```python
import numpy as np

kernel_name = "gpu_gemm_a2a_m8192_k8192_n8192_f32_silu"


ABSMAX_RTOL = 0.001

COMPUTE_DTYPE = "float32"
EPILOGUE = "silu"
IN_QUANT = None
OUT_QUANT = None
IN_DTYPES = {"x": "float32", "w_mat": "float32"}
WORLD_SIZE = 8

def setup_inputs():

    x = np.random.randn(8192, 8192).astype(np.float32)
    w_mat = (np.random.randn(8192, 8192) / np.sqrt(8192)).astype(np.float32)
    return {"x": x, "w_mat": w_mat}

def _epilogue(y):
    return (y / (1.0 + np.exp(-np.clip(y, -60.0, 60.0)))).astype(np.float32)

def reference(x, w_mat):
    y = (x @ w_mat).astype(np.float32)
    return _epilogue(y)
```

```cpp
#include <hip/hip_runtime.h>
#include <rccl/rccl.h>
#include <cstdio>
#include <cstdint>

#define HIP_CHECK(expr) do { hipError_t _e = (expr); if (_e != hipSuccess) { \
    fprintf(stderr, "HIP error %s at %s:%d\n", hipGetErrorString(_e), __FILE__, __LINE__); \
    return; } } while (0)

typedef float    f32x2 __attribute__((ext_vector_type(2)));
typedef float    f32x4 __attribute__((ext_vector_type(4)));
typedef _Float16 f16x2 __attribute__((ext_vector_type(2)));
typedef _Float16 f16x8 __attribute__((ext_vector_type(8)));
typedef unsigned u32x4 __attribute__((ext_vector_type(4)));
#define LAS __attribute__((address_space(3)))
#define DI __device__ __forceinline__

constexpr int NR = 8;
constexpr int ML = 1024;
constexpr int NN = 8192, KK = 8192;
constexpr int NL = NN / NR;
constexpr size_t XH_BYTES = (size_t)ML * KK * 2;
constexpr size_t WS_XH = 0;
constexpr size_t WS_FLAGS = (size_t)256 << 20;
constexpr size_t WS_NEED = WS_FLAGS + 65536;
constexpr int FL_CALL = 0, FL_TMO = 32, FL_DONE = 64;
constexpr unsigned long long DEADLINE_TICKS = 50000000ull;

constexpr int KT = 128;
constexpr int NKT = KK / KT;
#define ROT 1
#define ABL_NOA 0
#define ABL_NOW 0
#define ABL_NOMFMA 0
#define AUX_A 0
#define AUX_W 0
constexpr int STAGE_B = 32768;
constexpr int NSTAGE = 3;
constexpr int LDS_BYTES = NSTAGE * STAGE_B;

struct Args {
    const float* x; const float* w;
    void* const* out_all; void* const* ws_all;
    int rank; int phase;
};

DI unsigned cvtpk(float lo, float hi) { f32x2 v = {lo, hi}; return __builtin_bit_cast(unsigned, __builtin_convertvector(v, f16x2)); }
DI float silu(float y) {
    const float c = __builtin_amdgcn_fmed3f(y, -60.0f, 60.0f);
    const float e = __builtin_amdgcn_exp2f(c * -1.4426950408889634f);
    return y * __builtin_amdgcn_rcpf(1.0f + e);
}
#define SB() __builtin_amdgcn_sched_barrier(0)

__global__ __launch_bounds__(512, 2) void fused_kernel(Args a) {
    extern __shared__ __attribute__((aligned(16))) char smem[];
    const int tid = threadIdx.x, lane = tid & 63;
    const int wv = __builtin_amdgcn_readfirstlane(tid >> 6);
    char* ws = (char*)a.ws_all[a.rank];
    unsigned* fl = (unsigned*)(ws + WS_FLAGS);
    const int g = lane >> 4, c = lane & 15;

    if (a.phase == 0) {
        if (blockIdx.x == 0 && tid == 0) fl[FL_CALL] = fl[FL_CALL] + 1u;
        const int gw = blockIdx.x * 8 + wv;
        const int mf = gw >> 5, kg = gw & 31;
        const float* src = a.x + (size_t)(mf * 16 + c) * KK + kg * 256 + g * 8;
        f32x4 v[8][2];
#pragma unroll
        for (int s = 0; s < 8; ++s) { v[s][0] = *(const f32x4*)(src + s * 32); v[s][1] = *(const f32x4*)(src + s * 32 + 4); }
        u32x4* dst = (u32x4*)(ws + WS_XH) + ((size_t)(mf * 256 + kg * 8) * 64 + lane);
#pragma unroll
        for (int s = 0; s < 8; ++s) {
            u32x4 o;
            o[0] = cvtpk(v[s][0][0], v[s][0][1]); o[1] = cvtpk(v[s][0][2], v[s][0][3]);
            o[2] = cvtpk(v[s][1][0], v[s][1][1]); o[3] = cvtpk(v[s][1][2], v[s][1][3]);
            dst[s * 64] = o;
        }
        return;
    }

    const unsigned call = __builtin_amdgcn_readfirstlane(fl[FL_CALL]);
    const int xcd = blockIdx.x & 7, idx = blockIdx.x >> 3;
    const int mt = idx & 3, nt = idx >> 2;
    const int dest = (xcd + a.rank) & 7;
    const int n0 = dest * NL + nt * 128;
    const __amdgpu_buffer_rsrc_t rA = __builtin_amdgcn_make_buffer_rsrc((void*)(ws + WS_XH), (short)0, ABL_NOA ? 0 : (int)XH_BYTES, 0x00020000);
    const __amdgpu_buffer_rsrc_t rW = __builtin_amdgcn_make_buffer_rsrc((void*)a.w, (short)0, ABL_NOW ? 0 : (int)((size_t)KK * NN * 4), 0x00020000);
    const unsigned vA = lane * 16;
    const unsigned sA0 = (unsigned)(mt * 16 + 2 * wv) * 262144u;
    const unsigned vW = (unsigned)(g * 8 * NN + 4 * c) * 4u;
    const unsigned sW0 = (unsigned)(((wv >> 1) * 32) * NN + n0 + (wv & 1) * 64) * 4u;
    LAS char* lds = (LAS char*)smem;
    const unsigned lrd = lane * 16;
    const unsigned lwr = lane * 16 + wv * 4096;

    f32x4 acc[2][8];
#pragma unroll
    for (int i = 0; i < 2; ++i)
#pragma unroll
        for (int j = 0; j < 8; ++j) acc[i][j] = (f32x4){0.f, 0.f, 0.f, 0.f};
    u32x4 A[2][4][2];
    f32x4 Wr[2][8];
    u32x4 B[8];

#define WIN 4
#define XOFF 1
    const int rot = ROT ? (mt + nt) & (WIN - 1) : 0;
    const int woff = XOFF ? xcd & (WIN - 1) : 0;
#define TMAP(T) ((unsigned)(((((T) + woff) & ~(WIN - 1)) + (((T) + woff + rot) & (WIN - 1))) & (NKT - 1)))
#define LOAD_A(P, KS, MF, T) A[P][KS][MF] = __builtin_amdgcn_raw_buffer_load_b128(rA, vA + (KS) * 1024, sA0 + (MF) * 262144u + TMAP(T) * 4096u, AUX_A)
#define LOAD_W(P, J, T) Wr[P][J] = __builtin_bit_cast(f32x4, __builtin_amdgcn_raw_buffer_load_b128(rW, vW, sW0 + (TMAP(T) * KT + (J)) * (NN * 4u), AUX_W))
#define CVT_W(P, TT) (u32x4){cvtpk(Wr[P][0][TT], Wr[P][1][TT]), cvtpk(Wr[P][2][TT], Wr[P][3][TT]), cvtpk(Wr[P][4][TT], Wr[P][5][TT]), cvtpk(Wr[P][6][TT], Wr[P][7][TT])}
#define LDS_RD(OFF) (*(LAS const u32x4*)(lds + (OFF)))
#define LDS_WR(OFF, V) (*(LAS u32x4*)(lds + (OFF)) = (V))

#pragma unroll
    for (int j = 0; j < 8; ++j) LOAD_W(0, j, 0);
#pragma unroll
    for (int ks = 0; ks < 4; ++ks) { LOAD_A(0, ks, 0, 0); LOAD_A(0, ks, 1, 0); }
#pragma unroll
    for (int j = 0; j < 8; ++j) LOAD_W(1, j, 1);
#pragma unroll
    for (int ks = 0; ks < 4; ++ks) { LOAD_A(1, ks, 0, 1); LOAD_A(1, ks, 1, 1); }
#pragma unroll
    for (int tt = 0; tt < 4; ++tt) LDS_WR(lwr + tt * 1024, CVT_W(0, tt));
    SB();
#pragma unroll
    for (int j = 0; j < 8; ++j) LOAD_W(0, j, 2);
    asm volatile("s_waitcnt lgkmcnt(0)\n\ts_barrier" ::: "memory");
    SB();
#pragma unroll
    for (int nf = 0; nf < 8; ++nf) B[nf] = LDS_RD(lrd + nf * 1024);
    unsigned scur = 0, snxt = STAGE_B;

#define TILE(P, T) do { \
    const unsigned rdc = lrd + scur, rdn = lrd + snxt, wrn = lwr + snxt; \
    const int tw = (T) + 3 < NKT ? (T) + 3 : NKT - 1, ta = (T) + 2 < NKT ? (T) + 2 : NKT - 1; \
    _Pragma("unroll") for (int ks = 0; ks < 4; ++ks) { \
        if (ks == 3) { asm volatile("s_barrier" ::: "memory"); SB(); } \
        _Pragma("unroll") for (int nf = 0; nf < 8; ++nf) { \
            if (ABL_NOMFMA) { asm volatile("" :: "v"(A[P][ks][0]), "v"(A[P][ks][1]), "v"(B[nf])); } else { \
            acc[0][nf] = __builtin_amdgcn_mfma_f32_16x16x32_f16(__builtin_bit_cast(f16x8, A[P][ks][0]), __builtin_bit_cast(f16x8, B[nf]), acc[0][nf], 0, 0, 0); \
            acc[1][nf] = __builtin_amdgcn_mfma_f32_16x16x32_f16(__builtin_bit_cast(f16x8, A[P][ks][1]), __builtin_bit_cast(f16x8, B[nf]), acc[1][nf], 0, 0, 0); } \
            B[nf] = ks < 3 ? LDS_RD(rdc + ((ks + 1) * 8 + nf) * 1024) : LDS_RD(rdn + nf * 1024); \
            if (ks == 0 && (nf & 1)) LDS_WR(wrn + (nf >> 1) * 1024, CVT_W((P) ^ 1, nf >> 1)); \
            if (ks == 1) LOAD_W((P) ^ 1, nf, tw); \
            if (ks > 0 && nf == 0) LOAD_A(P, ks - 1, 0, ta); \
            if (ks > 0 && nf == 1) LOAD_A(P, ks - 1, 1, ta); \
            SB(); \
        } \
    } \
    LOAD_A(P, 3, 0, ta); LOAD_A(P, 3, 1, ta); SB(); \
    scur = snxt; snxt = snxt + STAGE_B >= (unsigned)LDS_BYTES ? 0u : snxt + STAGE_B; \
} while (0)

    for (int t = 0; t < NKT; t += 2) {
        TILE(0, t);
        TILE(1, t + 1);
    }

    {
        float* ob = (float*)a.out_all[dest] + (size_t)(a.rank * ML + mt * 256 + wv * 32 + 4 * g) * NL + nt * 128 + 4 * c;
#pragma unroll
        for (int mf = 0; mf < 2; ++mf)
#pragma unroll
            for (int r = 0; r < 4; ++r)
#pragma unroll
                for (int h = 0; h < 2; ++h) {
                    f32x4 o = {silu(acc[mf][4 * h + 0][r]), silu(acc[mf][4 * h + 1][r]), silu(acc[mf][4 * h + 2][r]), silu(acc[mf][4 * h + 3][r])};
                    float* p = ob + (size_t)(mf * 16 + r) * NL + h * 64;
                    asm volatile("global_store_dwordx4 %0, %1, off sc0 sc1\n\ts_nop 1" :: "v"(p), "v"(__builtin_bit_cast(u32x4, o)) : "memory");
                }
    }
    asm volatile("s_waitcnt vmcnt(0)" ::: "memory");
    __syncthreads();
    if (tid == 0 && dest != a.rank) {
        unsigned* pf = (unsigned*)((char*)a.ws_all[dest] + WS_FLAGS) + FL_DONE + 32 * a.rank;
        __hip_atomic_fetch_add(pf, 1u, __ATOMIC_RELAXED, __HIP_MEMORY_SCOPE_SYSTEM);
    }
    if (blockIdx.x == 0 && tid < 64) {
        bool bad = false;
        if (lane < NR && lane != a.rank) {
            const unsigned want = call * 32u;
            const unsigned long long t0 = __builtin_amdgcn_s_memrealtime();
            while (__hip_atomic_load(fl + FL_DONE + 32 * lane, __ATOMIC_RELAXED, __HIP_MEMORY_SCOPE_SYSTEM) < want) {
                if (__hip_atomic_load(fl + FL_TMO, __ATOMIC_RELAXED, __HIP_MEMORY_SCOPE_AGENT) == call
                    || __builtin_amdgcn_s_memrealtime() - t0 > DEADLINE_TICKS) {
                    __hip_atomic_store(fl + FL_TMO, call, __ATOMIC_RELAXED, __HIP_MEMORY_SCOPE_AGENT);
                    bad = true;
                    break;
                }
                __builtin_amdgcn_s_sleep(2);
            }
        }
        if (bad) ((float*)a.out_all[a.rank])[(size_t)(a.rank * ML) * NL] = __builtin_nanf("");
    }
}

extern "C" void kernel_launch(int rank, int nranks, ncclComm_t comm,
                              hipStream_t stream,
                              void* const* d_in, const int* in_sizes, int n_in,
                              void* d_out, int out_size,
                              const int* in_dtypes, int out_dtype,
                              void* const* d_out_all,
                              void* const* d_ws_all, size_t ws_bytes) {
    if (nranks != NR || (unsigned)rank >= (unsigned)NR || n_in != 2 || in_sizes[0] != ML * KK || in_sizes[1] != KK * NN
        || out_size != NN * NL || ws_bytes < WS_NEED) {
        fprintf(stderr, "kernel_launch: unexpected problem shape\n");
        return;
    }
    HIP_CHECK(hipFuncSetAttribute((const void*)fused_kernel, hipFuncAttributeMaxDynamicSharedMemorySize, LDS_BYTES));
    Args a;
    a.x = (const float*)d_in[0]; a.w = (const float*)d_in[1];
    a.out_all = d_out_all; a.ws_all = d_ws_all; a.rank = rank;
    a.phase = 0;
    hipLaunchKernelGGL(fused_kernel, dim3(256), dim3(512), 0, stream, a);
    a.phase = 1;
    hipLaunchKernelGGL(fused_kernel, dim3(256), dim3(512), LDS_BYTES, stream, a);
    HIP_CHECK(hipGetLastError());
}
```

```cpp
#include <hip/hip_runtime.h>
#include <rccl/rccl.h>
#include <cstdio>
#include <cstdint>
#define USE_ign
#define GEN_RD 0
#define GEN_OWN 1
#define GEMM_ASM \
    "s_mov_b32 s36, %[t0]\n\t" \
    "s_mov_b32 s43, %[sW0]\n\t" \
    "s_mov_b32 s44, %[sW1]\n\t" \
    "s_mov_b32 s45, %[sW2]\n\t" \
    "s_mov_b32 s46, %[sW3]\n\t" \
    "s_mov_b32 s47, 16384\n\t" \
    "s_mov_b32 s34, 0\n\t" \
    "s_mov_b32 s54, 0\n\t" \
    "s_mov_b32 s51, 0\n\t" \
    "s_mov_b32 s50, %[sExp0]\n\t" \
    "s_mov_b32 s59, 0x00020000\n\t" \
    "s_mov_b32 s56, %[wsLo]\n\t" \
    "s_mov_b32 s57, %[wsHi]\n\t" \
    "s_mov_b32 s58, 0\n\t" \
    "s_mov_b32 s48, 1\n\t" \
    "s_lshl_b32 s66, s36, 22\n\t" \
    "s_lshl_b32 s38, s36, 15\n\t" \
    "s_or_b32 s38, s38, %[sA0]\n\t" \
    "s_mov_b32 s68, %[sXs0]\n\t" \
    "s_mov_b32 s69, %[sXd0]\n\t" \
    "s_mov_b32 s70, 0\n\t" \
    "s_mov_b32 s71, %[sXc0]\n\t" \
    "s_add_u32 s63, s68, 131072\n\t" \
    "s_add_u32 s64, s68, 262144\n\t" \
    "s_add_u32 s65, s68, 393216\n\t" \
    "v_mov_b32 v19, 2\n\t" \
    "s_cmp_lg_u32 %[pop], 1\n\t" \
    "s_cbranch_scc1 EARLY0%=\n\t" \
    "buffer_load_dwordx4 v[20:23], %[vXs], %[rX], s68 offen offset:0 nt\n\t" \
    "buffer_load_dwordx4 v[24:27], %[vXs], %[rX], s63 offen offset:0 nt\n\t" \
    "buffer_load_dwordx4 v[28:31], %[vXs], %[rX], s64 offen offset:0 nt\n\t" \
    "buffer_load_dwordx4 v[32:35], %[vXs], %[rX], s65 offen offset:0 nt\n\t" \
    "buffer_load_dwordx4 v[36:39], %[vXs], %[rX], s68 offen offset:256 nt\n\t" \
    "buffer_load_dwordx4 v[40:43], %[vXs], %[rX], s63 offen offset:256 nt\n\t" \
    "buffer_load_dwordx4 v[44:47], %[vXs], %[rX], s64 offen offset:256 nt\n\t" \
    "buffer_load_dwordx4 v[48:51], %[vXs], %[rX], s65 offen offset:256 nt\n\t" \
    "buffer_load_dwordx4 v[52:55], %[vXs], %[rX], s68 offen offset:512 nt\n\t" \
    "buffer_load_dwordx4 v[56:59], %[vXs], %[rX], s63 offen offset:512 nt\n\t" \
    "buffer_load_dwordx4 v[60:63], %[vXs], %[rX], s64 offen offset:512 nt\n\t" \
    "buffer_load_dwordx4 v[64:67], %[vXs], %[rX], s65 offen offset:512 nt\n\t" \
    "buffer_load_dwordx4 v[68:71], %[vXs], %[rX], s68 offen offset:768 nt\n\t" \
    "buffer_load_dwordx4 v[72:75], %[vXs], %[rX], s63 offen offset:768 nt\n\t" \
    "buffer_load_dwordx4 v[76:79], %[vXs], %[rX], s64 offen offset:768 nt\n\t" \
    "buffer_load_dwordx4 v[80:83], %[vXs], %[rX], s65 offen offset:768 nt\n\t" \
    "s_waitcnt lgkmcnt(0)\n\t" \
    "s_waitcnt vmcnt(12)\n\t" \
    "v_cvt_pk_f16_f32 v20, v20, v21\n\t" \
    "v_cvt_pk_f16_f32 v21, v22, v23\n\t" \
    "ds_write_b64 %[vXw], v[20:21] offset:0\n\t" \
    "v_cvt_pk_f16_f32 v24, v24, v25\n\t" \
    "v_cvt_pk_f16_f32 v25, v26, v27\n\t" \
    "ds_write_b64 %[vXw], v[24:25] offset:16\n\t" \
    "v_cvt_pk_f16_f32 v28, v28, v29\n\t" \
    "v_cvt_pk_f16_f32 v29, v30, v31\n\t" \
    "ds_write_b64 %[vXw], v[28:29] offset:32\n\t" \
    "v_cvt_pk_f16_f32 v32, v32, v33\n\t" \
    "v_cvt_pk_f16_f32 v33, v34, v35\n\t" \
    "ds_write_b64 %[vXw], v[32:33] offset:48\n\t" \
    "ds_read_b128 v[28:31], %[vStg] offset:0\n\t" \
    "ds_read_b128 v[32:35], %[vStg] offset:1024\n\t" \
    "s_waitcnt vmcnt(8)\n\t" \
    "v_cvt_pk_f16_f32 v36, v36, v37\n\t" \
    "v_cvt_pk_f16_f32 v37, v38, v39\n\t" \
    "ds_write_b64 %[vXw], v[36:37] offset:2048\n\t" \
    "v_cvt_pk_f16_f32 v40, v40, v41\n\t" \
    "v_cvt_pk_f16_f32 v41, v42, v43\n\t" \
    "ds_write_b64 %[vXw], v[40:41] offset:2064\n\t" \
    "v_cvt_pk_f16_f32 v44, v44, v45\n\t" \
    "v_cvt_pk_f16_f32 v45, v46, v47\n\t" \
    "ds_write_b64 %[vXw], v[44:45] offset:2080\n\t" \
    "v_cvt_pk_f16_f32 v48, v48, v49\n\t" \
    "v_cvt_pk_f16_f32 v49, v50, v51\n\t" \
    "ds_write_b64 %[vXw], v[48:49] offset:2096\n\t" \
    "s_waitcnt lgkmcnt(4)\n\t" \
    "s_add_u32 s41, s69, 0\n\t" \
    "s_add_u32 s72, %[wsLo], s41\n\t" \
    "s_addc_u32 s73, %[wsHi], 0\n\t" \
    "global_store_dwordx4 %[vA0], v[28:31], s[72:73] sc0 sc1\n\t" \
    "s_add_u32 s41, s69, 8192\n\t" \
    "s_add_u32 s72, %[wsLo], s41\n\t" \
    "s_addc_u32 s73, %[wsHi], 0\n\t" \
    "global_store_dwordx4 %[vA0], v[32:35], s[72:73] sc0 sc1\n\t" \
    "ds_read_b128 v[44:47], %[vStg] offset:2048\n\t" \
    "ds_read_b128 v[48:51], %[vStg] offset:3072\n\t" \
    "s_waitcnt vmcnt(6)\n\t" \
    "v_cvt_pk_f16_f32 v52, v52, v53\n\t" \
    "v_cvt_pk_f16_f32 v53, v54, v55\n\t" \
    "ds_write_b64 %[vXw], v[52:53] offset:4096\n\t" \
    "v_cvt_pk_f16_f32 v56, v56, v57\n\t" \
    "v_cvt_pk_f16_f32 v57, v58, v59\n\t" \
    "ds_write_b64 %[vXw], v[56:57] offset:4112\n\t" \
    "v_cvt_pk_f16_f32 v60, v60, v61\n\t" \
    "v_cvt_pk_f16_f32 v61, v62, v63\n\t" \
    "ds_write_b64 %[vXw], v[60:61] offset:4128\n\t" \
    "v_cvt_pk_f16_f32 v64, v64, v65\n\t" \
    "v_cvt_pk_f16_f32 v65, v66, v67\n\t" \
    "ds_write_b64 %[vXw], v[64:65] offset:4144\n\t" \
    "s_waitcnt lgkmcnt(4)\n\t" \
    "s_add_u32 s41, s69, 16384\n\t" \
    "s_add_u32 s72, %[wsLo], s41\n\t" \
    "s_addc_u32 s73, %[wsHi], 0\n\t" \
    "global_store_dwordx4 %[vA0], v[44:47], s[72:73] sc0 sc1\n\t" \
    "s_add_u32 s41, s69, 24576\n\t" \
    "s_add_u32 s72, %[wsLo], s41\n\t" \
    "s_addc_u32 s73, %[wsHi], 0\n\t" \
    "global_store_dwordx4 %[vA0], v[48:51], s[72:73] sc0 sc1\n\t" \
    "ds_read_b128 v[60:63], %[vStg] offset:4096\n\t" \
    "ds_read_b128 v[64:67], %[vStg] offset:5120\n\t" \
    "s_waitcnt vmcnt(4)\n\t" \
    "v_cvt_pk_f16_f32 v68, v68, v69\n\t" \
    "v_cvt_pk_f16_f32 v69, v70, v71\n\t" \
    "ds_write_b64 %[vXw], v[68:69] offset:6144\n\t" \
    "v_cvt_pk_f16_f32 v72, v72, v73\n\t" \
    "v_cvt_pk_f16_f32 v73, v74, v75\n\t" \
    "ds_write_b64 %[vXw], v[72:73] offset:6160\n\t" \
    "v_cvt_pk_f16_f32 v76, v76, v77\n\t" \
    "v_cvt_pk_f16_f32 v77, v78, v79\n\t" \
    "ds_write_b64 %[vXw], v[76:77] offset:6176\n\t" \
    "v_cvt_pk_f16_f32 v80, v80, v81\n\t" \
    "v_cvt_pk_f16_f32 v81, v82, v83\n\t" \
    "ds_write_b64 %[vXw], v[80:81] offset:6192\n\t" \
    "s_waitcnt lgkmcnt(4)\n\t" \
    "s_add_u32 s41, s69, 32768\n\t" \
    "s_add_u32 s72, %[wsLo], s41\n\t" \
    "s_addc_u32 s73, %[wsHi], 0\n\t" \
    "global_store_dwordx4 %[vA0], v[60:63], s[72:73] sc0 sc1\n\t" \
    "s_add_u32 s41, s69, 40960\n\t" \
    "s_add_u32 s72, %[wsLo], s41\n\t" \
    "s_addc_u32 s73, %[wsHi], 0\n\t" \
    "global_store_dwordx4 %[vA0], v[64:67], s[72:73] sc0 sc1\n\t" \
    "ds_read_b128 v[76:79], %[vStg] offset:6144\n\t" \
    "ds_read_b128 v[80:83], %[vStg] offset:7168\n\t" \
    "s_waitcnt lgkmcnt(0)\n\t" \
    "s_add_u32 s41, s69, 49152\n\t" \
    "s_add_u32 s72, %[wsLo], s41\n\t" \
    "s_addc_u32 s73, %[wsHi], 0\n\t" \
    "global_store_dwordx4 %[vA0], v[76:79], s[72:73] sc0 sc1\n\t" \
    "s_add_u32 s41, s69, 57344\n\t" \
    "s_add_u32 s72, %[wsLo], s41\n\t" \
    "s_addc_u32 s73, %[wsHi], 0\n\t" \
    "global_store_dwordx4 %[vA0], v[80:83], s[72:73] sc0 sc1\n\t" \
    "buffer_load_dwordx4 v[84:87], %[vXs], %[rX], s68 offen offset:1024 nt\n\t" \
    "buffer_load_dwordx4 v[88:91], %[vXs], %[rX], s63 offen offset:1024 nt\n\t" \
    "buffer_load_dwordx4 v[92:95], %[vXs], %[rX], s64 offen offset:1024 nt\n\t" \
    "buffer_load_dwordx4 v[96:99], %[vXs], %[rX], s65 offen offset:1024 nt\n\t" \
    "buffer_load_dwordx4 v[100:103], %[vXs], %[rX], s68 offen offset:1280 nt\n\t" \
    "buffer_load_dwordx4 v[104:107], %[vXs], %[rX], s63 offen offset:1280 nt\n\t" \
    "buffer_load_dwordx4 v[108:111], %[vXs], %[rX], s64 offen offset:1280 nt\n\t" \
    "buffer_load_dwordx4 v[112:115], %[vXs], %[rX], s65 offen offset:1280 nt\n\t" \
    "buffer_load_dwordx4 v[116:119], %[vXs], %[rX], s68 offen offset:1536 nt\n\t" \
    "buffer_load_dwordx4 v[120:123], %[vXs], %[rX], s63 offen offset:1536 nt\n\t" \
    "buffer_load_dwordx4 v[124:127], %[vXs], %[rX], s64 offen offset:1536 nt\n\t" \
    "buffer_load_dwordx4 v[128:131], %[vXs], %[rX], s65 offen offset:1536 nt\n\t" \
    "buffer_load_dwordx4 v[132:135], %[vXs], %[rX], s68 offen offset:1792 nt\n\t" \
    "buffer_load_dwordx4 v[136:139], %[vXs], %[rX], s63 offen offset:1792 nt\n\t" \
    "buffer_load_dwordx4 v[140:143], %[vXs], %[rX], s64 offen offset:1792 nt\n\t" \
    "buffer_load_dwordx4 v[144:147], %[vXs], %[rX], s65 offen offset:1792 nt\n\t" \
    "s_waitcnt vmcnt(16)\n\t" \
    "s_mov_b64 exec, 1\n\tbuffer_atomic_add v19, off, %[rWS], s71 sc1\n\ts_mov_b64 exec, -1\n\t" \
    "s_add_u32 s71, s71, 8192\n\t" \
    "s_nop 0\n\t" \
    "v_mov_b32 v19, 2\n\t" \
    "s_waitcnt vmcnt(13)\n\t" \
    "v_cvt_pk_f16_f32 v84, v84, v85\n\t" \
    "v_cvt_pk_f16_f32 v85, v86, v87\n\t" \
    "ds_write_b64 %[vXw], v[84:85] offset:8192\n\t" \
    "v_cvt_pk_f16_f32 v88, v88, v89\n\t" \
    "v_cvt_pk_f16_f32 v89, v90, v91\n\t" \
    "ds_write_b64 %[vXw], v[88:89] offset:8208\n\t" \
    "v_cvt_pk_f16_f32 v92, v92, v93\n\t" \
    "v_cvt_pk_f16_f32 v93, v94, v95\n\t" \
    "ds_write_b64 %[vXw], v[92:93] offset:8224\n\t" \
    "v_cvt_pk_f16_f32 v96, v96, v97\n\t" \
    "v_cvt_pk_f16_f32 v97, v98, v99\n\t" \
    "ds_write_b64 %[vXw], v[96:97] offset:8240\n\t" \
    "ds_read_b128 v[92:95], %[vStg] offset:8192\n\t" \
    "ds_read_b128 v[96:99], %[vStg] offset:9216\n\t" \
    "s_waitcnt vmcnt(9)\n\t" \
    "v_cvt_pk_f16_f32 v100, v100, v101\n\t" \
    "v_cvt_pk_f16_f32 v101, v102, v103\n\t" \
    "ds_write_b64 %[vXw], v[100:101] offset:10240\n\t" \
    "v_cvt_pk_f16_f32 v104, v104, v105\n\t" \
    "v_cvt_pk_f16_f32 v105, v106, v107\n\t" \
    "ds_write_b64 %[vXw], v[104:105] offset:10256\n\t" \
    "v_cvt_pk_f16_f32 v108, v108, v109\n\t" \
    "v_cvt_pk_f16_f32 v109, v110, v111\n\t" \
    "ds_write_b64 %[vXw], v[108:109] offset:10272\n\t" \
    "v_cvt_pk_f16_f32 v112, v112, v113\n\t" \
    "v_cvt_pk_f16_f32 v113, v114, v115\n\t" \
    "ds_write_b64 %[vXw], v[112:113] offset:10288\n\t" \
    "s_waitcnt lgkmcnt(4)\n\t" \
    "s_add_u32 s41, s69, 65536\n\t" \
    "s_add_u32 s72, %[wsLo], s41\n\t" \
    "s_addc_u32 s73, %[wsHi], 0\n\t" \
    "global_store_dwordx4 %[vA0], v[92:95], s[72:73] sc0 sc1\n\t" \
    "s_add_u32 s41, s69, 73728\n\t" \
    "s_add_u32 s72, %[wsLo], s41\n\t" \
    "s_addc_u32 s73, %[wsHi], 0\n\t" \
    "global_store_dwordx4 %[vA0], v[96:99], s[72:73] sc0 sc1\n\t" \
    "ds_read_b128 v[108:111], %[vStg] offset:10240\n\t" \
    "ds_read_b128 v[112:115], %[vStg] offset:11264\n\t" \
    "s_waitcnt vmcnt(7)\n\t" \
    "v_cvt_pk_f16_f32 v116, v116, v117\n\t" \
    "v_cvt_pk_f16_f32 v117, v118, v119\n\t" \
    "ds_write_b64 %[vXw], v[116:117] offset:12288\n\t" \
    "v_cvt_pk_f16_f32 v120, v120, v121\n\t" \
    "v_cvt_pk_f16_f32 v121, v122, v123\n\t" \
    "ds_write_b64 %[vXw], v[120:121] offset:12304\n\t" \
    "v_cvt_pk_f16_f32 v124, v124, v125\n\t" \
    "v_cvt_pk_f16_f32 v125, v126, v127\n\t" \
    "ds_write_b64 %[vXw], v[124:125] offset:12320\n\t" \
    "v_cvt_pk_f16_f32 v128, v128, v129\n\t" \
    "v_cvt_pk_f16_f32 v129, v130, v131\n\t" \
    "ds_write_b64 %[vXw], v[128:129] offset:12336\n\t" \
    "s_waitcnt lgkmcnt(4)\n\t" \
    "s_add_u32 s41, s69, 81920\n\t" \
    "s_add_u32 s72, %[wsLo], s41\n\t" \
    "s_addc_u32 s73, %[wsHi], 0\n\t" \
    "global_store_dwordx4 %[vA0], v[108:111], s[72:73] sc0 sc1\n\t" \
    "s_add_u32 s41, s69, 90112\n\t" \
    "s_add_u32 s72, %[wsLo], s41\n\t" \
    "s_addc_u32 s73, %[wsHi], 0\n\t" \
    "global_store_dwordx4 %[vA0], v[112:115], s[72:73] sc0 sc1\n\t" \
    "ds_read_b128 v[124:127], %[vStg] offset:12288\n\t" \
    "ds_read_b128 v[128:131], %[vStg] offset:13312\n\t" \
    "s_waitcnt vmcnt(5)\n\t" \
    "v_cvt_pk_f16_f32 v132, v132, v133\n\t" \
    "v_cvt_pk_f16_f32 v133, v134, v135\n\t" \
    "ds_write_b64 %[vXw], v[132:133] offset:14336\n\t" \
    "v_cvt_pk_f16_f32 v136, v136, v137\n\t" \
    "v_cvt_pk_f16_f32 v137, v138, v139\n\t" \
    "ds_write_b64 %[vXw], v[136:137] offset:14352\n\t" \
    "v_cvt_pk_f16_f32 v140, v140, v141\n\t" \
    "v_cvt_pk_f16_f32 v141, v142, v143\n\t" \
    "ds_write_b64 %[vXw], v[140:141] offset:14368\n\t" \
    "v_cvt_pk_f16_f32 v144, v144, v145\n\t" \
    "v_cvt_pk_f16_f32 v145, v146, v147\n\t" \
    "ds_write_b64 %[vXw], v[144:145] offset:14384\n\t" \
    "s_waitcnt lgkmcnt(4)\n\t" \
    "s_add_u32 s41, s69, 98304\n\t" \
    "s_add_u32 s72, %[wsLo], s41\n\t" \
    "s_addc_u32 s73, %[wsHi], 0\n\t" \
    "global_store_dwordx4 %[vA0], v[124:127], s[72:73] sc0 sc1\n\t" \
    "s_add_u32 s41, s69, 106496\n\t" \
    "s_add_u32 s72, %[wsLo], s41\n\t" \
    "s_addc_u32 s73, %[wsHi], 0\n\t" \
    "global_store_dwordx4 %[vA0], v[128:131], s[72:73] sc0 sc1\n\t" \
    "ds_read_b128 v[140:143], %[vStg] offset:14336\n\t" \
    "ds_read_b128 v[144:147], %[vStg] offset:15360\n\t" \
    "s_waitcnt lgkmcnt(0)\n\t" \
    "s_add_u32 s41, s69, 114688\n\t" \
    "s_add_u32 s72, %[wsLo], s41\n\t" \
    "s_addc_u32 s73, %[wsHi], 0\n\t" \
    "global_store_dwordx4 %[vA0], v[140:143], s[72:73] sc0 sc1\n\t" \
    "s_add_u32 s41, s69, 122880\n\t" \
    "s_add_u32 s72, %[wsLo], s41\n\t" \
    "s_addc_u32 s73, %[wsHi], 0\n\t" \
    "global_store_dwordx4 %[vA0], v[144:147], s[72:73] sc0 sc1\n\t" \
    "buffer_load_dwordx4 v[20:23], %[vXs], %[rX], s68 offen offset:2048 nt\n\t" \
    "buffer_load_dwordx4 v[24:27], %[vXs], %[rX], s63 offen offset:2048 nt\n\t" \
    "buffer_load_dwordx4 v[28:31], %[vXs], %[rX], s64 offen offset:2048 nt\n\t" \
    "buffer_load_dwordx4 v[32:35], %[vXs], %[rX], s65 offen offset:2048 nt\n\t" \
    "buffer_load_dwordx4 v[36:39], %[vXs], %[rX], s68 offen offset:2304 nt\n\t" \
    "buffer_load_dwordx4 v[40:43], %[vXs], %[rX], s63 offen offset:2304 nt\n\t" \
    "buffer_load_dwordx4 v[44:47], %[vXs], %[rX], s64 offen offset:2304 nt\n\t" \
    "buffer_load_dwordx4 v[48:51], %[vXs], %[rX], s65 offen offset:2304 nt\n\t" \
    "buffer_load_dwordx4 v[52:55], %[vXs], %[rX], s68 offen offset:2560 nt\n\t" \
    "buffer_load_dwordx4 v[56:59], %[vXs], %[rX], s63 offen offset:2560 nt\n\t" \
    "buffer_load_dwordx4 v[60:63], %[vXs], %[rX], s64 offen offset:2560 nt\n\t" \
    "buffer_load_dwordx4 v[64:67], %[vXs], %[rX], s65 offen offset:2560 nt\n\t" \
    "buffer_load_dwordx4 v[68:71], %[vXs], %[rX], s68 offen offset:2816 nt\n\t" \
    "buffer_load_dwordx4 v[72:75], %[vXs], %[rX], s63 offen offset:2816 nt\n\t" \
    "buffer_load_dwordx4 v[76:79], %[vXs], %[rX], s64 offen offset:2816 nt\n\t" \
    "buffer_load_dwordx4 v[80:83], %[vXs], %[rX], s65 offen offset:2816 nt\n\t" \
    "s_waitcnt vmcnt(16)\n\t" \
    "s_mov_b64 exec, 1\n\tbuffer_atomic_add v19, off, %[rWS], s71 sc1\n\ts_mov_b64 exec, -1\n\t" \
    "s_add_u32 s71, s71, 8192\n\t" \
    "s_nop 0\n\t" \
    "v_mov_b32 v19, 2\n\t" \
    "s_waitcnt vmcnt(13)\n\t" \
    "v_cvt_pk_f16_f32 v20, v20, v21\n\t" \
    "v_cvt_pk_f16_f32 v21, v22, v23\n\t" \
    "ds_write_b64 %[vXw], v[20:21] offset:0\n\t" \
    "v_cvt_pk_f16_f32 v24, v24, v25\n\t" \
    "v_cvt_pk_f16_f32 v25, v26, v27\n\t" \
    "ds_write_b64 %[vXw], v[24:25] offset:16\n\t" \
    "v_cvt_pk_f16_f32 v28, v28, v29\n\t" \
    "v_cvt_pk_f16_f32 v29, v30, v31\n\t" \
    "ds_write_b64 %[vXw], v[28:29] offset:32\n\t" \
    "v_cvt_pk_f16_f32 v32, v32, v33\n\t" \
    "v_cvt_pk_f16_f32 v33, v34, v35\n\t" \
    "ds_write_b64 %[vXw], v[32:33] offset:48\n\t" \
    "ds_read_b128 v[28:31], %[vStg] offset:0\n\t" \
    "ds_read_b128 v[32:35], %[vStg] offset:1024\n\t" \
    "s_waitcnt vmcnt(9)\n\t" \
    "v_cvt_pk_f16_f32 v36, v36, v37\n\t" \
    "v_cvt_pk_f16_f32 v37, v38, v39\n\t" \
    "ds_write_b64 %[vXw], v[36:37] offset:2048\n\t" \
    "v_cvt_pk_f16_f32 v40, v40, v41\n\t" \
    "v_cvt_pk_f16_f32 v41, v42, v43\n\t" \
    "ds_write_b64 %[vXw], v[40:41] offset:2064\n\t" \
    "v_cvt_pk_f16_f32 v44, v44, v45\n\t" \
    "v_cvt_pk_f16_f32 v45, v46, v47\n\t" \
    "ds_write_b64 %[vXw], v[44:45] offset:2080\n\t" \
    "v_cvt_pk_f16_f32 v48, v48, v49\n\t" \
    "v_cvt_pk_f16_f32 v49, v50, v51\n\t" \
    "ds_write_b64 %[vXw], v[48:49] offset:2096\n\t" \
    "s_waitcnt lgkmcnt(4)\n\t" \
    "s_add_u32 s41, s69, 131072\n\t" \
    "s_add_u32 s72, %[wsLo], s41\n\t" \
    "s_addc_u32 s73, %[wsHi], 0\n\t" \
    "global_store_dwordx4 %[vA0], v[28:31], s[72:73] sc0 sc1\n\t" \
    "s_add_u32 s41, s69, 139264\n\t" \
    "s_add_u32 s72, %[wsLo], s41\n\t" \
    "s_addc_u32 s73, %[wsHi], 0\n\t" \
    "global_store_dwordx4 %[vA0], v[32:35], s[72:73] sc0 sc1\n\t" \
    "ds_read_b128 v[44:47], %[vStg] offset:2048\n\t" \
    "ds_read_b128 v[48:51], %[vStg] offset:3072\n\t" \
    "s_waitcnt vmcnt(7)\n\t" \
    "v_cvt_pk_f16_f32 v52, v52, v53\n\t" \
    "v_cvt_pk_f16_f32 v53, v54, v55\n\t" \
    "ds_write_b64 %[vXw], v[52:53] offset:4096\n\t" \
    "v_cvt_pk_f16_f32 v56, v56, v57\n\t" \
    "v_cvt_pk_f16_f32 v57, v58, v59\n\t" \
    "ds_write_b64 %[vXw], v[56:57] offset:4112\n\t" \
    "v_cvt_pk_f16_f32 v60, v60, v61\n\t" \
    "v_cvt_pk_f16_f32 v61, v62, v63\n\t" \
    "ds_write_b64 %[vXw], v[60:61] offset:4128\n\t" \
    "v_cvt_pk_f16_f32 v64, v64, v65\n\t" \
    "v_cvt_pk_f16_f32 v65, v66, v67\n\t" \
    "ds_write_b64 %[vXw], v[64:65] offset:4144\n\t" \
    "s_waitcnt lgkmcnt(4)\n\t" \
    "s_add_u32 s41, s69, 147456\n\t" \
    "s_add_u32 s72, %[wsLo], s41\n\t" \
    "s_addc_u32 s73, %[wsHi], 0\n\t" \
    "global_store_dwordx4 %[vA0], v[44:47], s[72:73] sc0 sc1\n\t" \
    "s_add_u32 s41, s69, 155648\n\t" \
    "s_add_u32 s72, %[wsLo], s41\n\t" \
    "s_addc_u32 s73, %[wsHi], 0\n\t" \
    "global_store_dwordx4 %[vA0], v[48:51], s[72:73] sc0 sc1\n\t" \
    "ds_read_b128 v[60:63], %[vStg] offset:4096\n\t" \
    "ds_read_b128 v[64:67], %[vStg] offset:5120\n\t" \
    "s_waitcnt vmcnt(5)\n\t" \
    "v_cvt_pk_f16_f32 v68, v68, v69\n\t" \
    "v_cvt_pk_f16_f32 v69, v70, v71\n\t" \
    "ds_write_b64 %[vXw], v[68:69] offset:6144\n\t" \
    "v_cvt_pk_f16_f32 v72, v72, v73\n\t" \
    "v_cvt_pk_f16_f32 v73, v74, v75\n\t" \
    "ds_write_b64 %[vXw], v[72:73] offset:6160\n\t" \
    "v_cvt_pk_f16_f32 v76, v76, v77\n\t" \
    "v_cvt_pk_f16_f32 v77, v78, v79\n\t" \
    "ds_write_b64 %[vXw], v[76:77] offset:6176\n\t" \
    "v_cvt_pk_f16_f32 v80, v80, v81\n\t" \
    "v_cvt_pk_f16_f32 v81, v82, v83\n\t" \
    "ds_write_b64 %[vXw], v[80:81] offset:6192\n\t" \
    "s_waitcnt lgkmcnt(4)\n\t" \
    "s_add_u32 s41, s69, 163840\n\t" \
    "s_add_u32 s72, %[wsLo], s41\n\t" \
    "s_addc_u32 s73, %[wsHi], 0\n\t" \
    "global_store_dwordx4 %[vA0], v[60:63], s[72:73] sc0 sc1\n\t" \
    "s_add_u32 s41, s69, 172032\n\t" \
    "s_add_u32 s72, %[wsLo], s41\n\t" \
    "s_addc_u32 s73, %[wsHi], 0\n\t" \
    "global_store_dwordx4 %[vA0], v[64:67], s[72:73] sc0 sc1\n\t" \
    "ds_read_b128 v[76:79], %[vStg] offset:6144\n\t" \
    "ds_read_b128 v[80:83], %[vStg] offset:7168\n\t" \
    "s_waitcnt lgkmcnt(0)\n\t" \
    "s_add_u32 s41, s69, 180224\n\t" \
    "s_add_u32 s72, %[wsLo], s41\n\t" \
    "s_addc_u32 s73, %[wsHi], 0\n\t" \
    "global_store_dwordx4 %[vA0], v[76:79], s[72:73] sc0 sc1\n\t" \
    "s_add_u32 s41, s69, 188416\n\t" \
    "s_add_u32 s72, %[wsLo], s41\n\t" \
    "s_addc_u32 s73, %[wsHi], 0\n\t" \
    "global_store_dwordx4 %[vA0], v[80:83], s[72:73] sc0 sc1\n\t" \
    "buffer_load_dwordx4 v[84:87], %[vXs], %[rX], s68 offen offset:3072 nt\n\t" \
    "buffer_load_dwordx4 v[88:91], %[vXs], %[rX], s63 offen offset:3072 nt\n\t" \
    "buffer_load_dwordx4 v[92:95], %[vXs], %[rX], s64 offen offset:3072 nt\n\t" \
    "buffer_load_dwordx4 v[96:99], %[vXs], %[rX], s65 offen offset:3072 nt\n\t" \
    "buffer_load_dwordx4 v[100:103], %[vXs], %[rX], s68 offen offset:3328 nt\n\t" \
    "buffer_load_dwordx4 v[104:107], %[vXs], %[rX], s63 offen offset:3328 nt\n\t" \
    "buffer_load_dwordx4 v[108:111], %[vXs], %[rX], s64 offen offset:3328 nt\n\t" \
    "buffer_load_dwordx4 v[112:115], %[vXs], %[rX], s65 offen offset:3328 nt\n\t" \
    "buffer_load_dwordx4 v[116:119], %[vXs], %[rX], s68 offen offset:3584 nt\n\t" \
    "buffer_load_dwordx4 v[120:123], %[vXs], %[rX], s63 offen offset:3584 nt\n\t" \
    "buffer_load_dwordx4 v[124:127], %[vXs], %[rX], s64 offen offset:3584 nt\n\t" \
    "buffer_load_dwordx4 v[128:131], %[vXs], %[rX], s65 offen offset:3584 nt\n\t" \
    "buffer_load_dwordx4 v[132:135], %[vXs], %[rX], s68 offen offset:3840 nt\n\t" \
    "buffer_load_dwordx4 v[136:139], %[vXs], %[rX], s63 offen offset:3840 nt\n\t" \
    "buffer_load_dwordx4 v[140:143], %[vXs], %[rX], s64 offen offset:3840 nt\n\t" \
    "buffer_load_dwordx4 v[144:147], %[vXs], %[rX], s65 offen offset:3840 nt\n\t" \
    "s_waitcnt vmcnt(16)\n\t" \
    "s_mov_b64 exec, 1\n\tbuffer_atomic_add v19, off, %[rWS], s71 sc1\n\ts_mov_b64 exec, -1\n\t" \
    "s_add_u32 s71, s71, 8192\n\t" \
    "s_nop 0\n\t" \
    "v_mov_b32 v19, 2\n\t" \
    "s_waitcnt vmcnt(13)\n\t" \
    "v_cvt_pk_f16_f32 v84, v84, v85\n\t" \
    "v_cvt_pk_f16_f32 v85, v86, v87\n\t" \
    "ds_write_b64 %[vXw], v[84:85] offset:8192\n\t" \
    "v_cvt_pk_f16_f32 v88, v88, v89\n\t" \
    "v_cvt_pk_f16_f32 v89, v90, v91\n\t" \
    "ds_write_b64 %[vXw], v[88:89] offset:8208\n\t" \
    "v_cvt_pk_f16_f32 v92, v92, v93\n\t" \
    "v_cvt_pk_f16_f32 v93, v94, v95\n\t" \
    "ds_write_b64 %[vXw], v[92:93] offset:8224\n\t" \
    "v_cvt_pk_f16_f32 v96, v96, v97\n\t" \
    "v_cvt_pk_f16_f32 v97, v98, v99\n\t" \
    "ds_write_b64 %[vXw], v[96:97] offset:8240\n\t" \
    "ds_read_b128 v[92:95], %[vStg] offset:8192\n\t" \
    "ds_read_b128 v[96:99], %[vStg] offset:9216\n\t" \
    "s_waitcnt vmcnt(9)\n\t" \
    "v_cvt_pk_f16_f32 v100, v100, v101\n\t" \
    "v_cvt_pk_f16_f32 v101, v102, v103\n\t" \
    "ds_write_b64 %[vXw], v[100:101] offset:10240\n\t" \
    "v_cvt_pk_f16_f32 v104, v104, v105\n\t" \
    "v_cvt_pk_f16_f32 v105, v106, v107\n\t" \
    "ds_write_b64 %[vXw], v[104:105] offset:10256\n\t" \
    "v_cvt_pk_f16_f32 v108, v108, v109\n\t" \
    "v_cvt_pk_f16_f32 v109, v110, v111\n\t" \
    "ds_write_b64 %[vXw], v[108:109] offset:10272\n\t" \
    "v_cvt_pk_f16_f32 v112, v112, v113\n\t" \
    "v_cvt_pk_f16_f32 v113, v114, v115\n\t" \
    "ds_write_b64 %[vXw], v[112:113] offset:10288\n\t" \
    "s_waitcnt lgkmcnt(4)\n\t" \
    "s_add_u32 s41, s69, 196608\n\t" \
    "s_add_u32 s72, %[wsLo], s41\n\t" \
    "s_addc_u32 s73, %[wsHi], 0\n\t" \
    "global_store_dwordx4 %[vA0], v[92:95], s[72:73] sc0 sc1\n\t" \
    "s_add_u32 s41, s69, 204800\n\t" \
    "s_add_u32 s72, %[wsLo], s41\n\t" \
    "s_addc_u32 s73, %[wsHi], 0\n\t" \
    "global_store_dwordx4 %[vA0], v[96:99], s[72:73] sc0 sc1\n\t" \
    "ds_read_b128 v[108:111], %[vStg] offset:10240\n\t" \
    "ds_read_b128 v[112:115], %[vStg] offset:11264\n\t" \
    "s_waitcnt vmcnt(7)\n\t" \
    "v_cvt_pk_f16_f32 v116, v116, v117\n\t" \
    "v_cvt_pk_f16_f32 v117, v118, v119\n\t" \
    "ds_write_b64 %[vXw], v[116:117] offset:12288\n\t" \
    "v_cvt_pk_f16_f32 v120, v120, v121\n\t" \
    "v_cvt_pk_f16_f32 v121, v122, v123\n\t" \
    "ds_write_b64 %[vXw], v[120:121] offset:12304\n\t" \
    "v_cvt_pk_f16_f32 v124, v124, v125\n\t" \
    "v_cvt_pk_f16_f32 v125, v126, v127\n\t" \
    "ds_write_b64 %[vXw], v[124:125] offset:12320\n\t" \
    "v_cvt_pk_f16_f32 v128, v128, v129\n\t" \
    "v_cvt_pk_f16_f32 v129, v130, v131\n\t" \
    "ds_write_b64 %[vXw], v[128:129] offset:12336\n\t" \
    "s_waitcnt lgkmcnt(4)\n\t" \
    "s_add_u32 s41, s69, 212992\n\t" \
    "s_add_u32 s72, %[wsLo], s41\n\t" \
    "s_addc_u32 s73, %[wsHi], 0\n\t" \
    "global_store_dwordx4 %[vA0], v[108:111], s[72:73] sc0 sc1\n\t" \
    "s_add_u32 s41, s69, 221184\n\t" \
    "s_add_u32 s72, %[wsLo], s41\n\t" \
    "s_addc_u32 s73, %[wsHi], 0\n\t" \
    "global_store_dwordx4 %[vA0], v[112:115], s[72:73] sc0 sc1\n\t" \
    "ds_read_b128 v[124:127], %[vStg] offset:12288\n\t" \
    "ds_read_b128 v[128:131], %[vStg] offset:13312\n\t" \
    "s_waitcnt vmcnt(5)\n\t" \
    "v_cvt_pk_f16_f32 v132, v132, v133\n\t" \
    "v_cvt_pk_f16_f32 v133, v134, v135\n\t" \
    "ds_write_b64 %[vXw], v[132:133] offset:14336\n\t" \
    "v_cvt_pk_f16_f32 v136, v136, v137\n\t" \
    "v_cvt_pk_f16_f32 v137, v138, v139\n\t" \
    "ds_write_b64 %[vXw], v[136:137] offset:14352\n\t" \
    "v_cvt_pk_f16_f32 v140, v140, v141\n\t" \
    "v_cvt_pk_f16_f32 v141, v142, v143\n\t" \
    "ds_write_b64 %[vXw], v[140:141] offset:14368\n\t" \
    "v_cvt_pk_f16_f32 v144, v144, v145\n\t" \
    "v_cvt_pk_f16_f32 v145, v146, v147\n\t" \
    "ds_write_b64 %[vXw], v[144:145] offset:14384\n\t" \
    "s_waitcnt lgkmcnt(4)\n\t" \
    "s_add_u32 s41, s69, 229376\n\t" \
    "s_add_u32 s72, %[wsLo], s41\n\t" \
    "s_addc_u32 s73, %[wsHi], 0\n\t" \
    "global_store_dwordx4 %[vA0], v[124:127], s[72:73] sc0 sc1\n\t" \
    "s_add_u32 s41, s69, 237568\n\t" \
    "s_add_u32 s72, %[wsLo], s41\n\t" \
    "s_addc_u32 s73, %[wsHi], 0\n\t" \
    "global_store_dwordx4 %[vA0], v[128:131], s[72:73] sc0 sc1\n\t" \
    "ds_read_b128 v[140:143], %[vStg] offset:14336\n\t" \
    "ds_read_b128 v[144:147], %[vStg] offset:15360\n\t" \
    "s_waitcnt lgkmcnt(0)\n\t" \
    "s_add_u32 s41, s69, 245760\n\t" \
    "s_add_u32 s72, %[wsLo], s41\n\t" \
    "s_addc_u32 s73, %[wsHi], 0\n\t" \
    "global_store_dwordx4 %[vA0], v[140:143], s[72:73] sc0 sc1\n\t" \
    "s_add_u32 s41, s69, 253952\n\t" \
    "s_add_u32 s72, %[wsLo], s41\n\t" \
    "s_addc_u32 s73, %[wsHi], 0\n\t" \
    "global_store_dwordx4 %[vA0], v[144:147], s[72:73] sc0 sc1\n\t" \
    "s_waitcnt vmcnt(0)\n\t" \
    "s_mov_b64 exec, 1\n\tbuffer_atomic_add v19, off, %[rWS], s71 sc1\n\ts_mov_b64 exec, -1\n\t" \
    "s_add_u32 s71, s71, 8192\n\t" \
    "s_nop 0\n\t" \
    "v_mov_b32 v19, 2\n\t" \
    "EARLY0%=:\n\t" \
    "EARLY%=:\n\t" \
    "v_mov_b32 v18, 1\n\t" \
    "s_mov_b64 exec, 1\n\t" \
    "buffer_atomic_add v18, off, %[rWS], %[sJunk] offset:4 sc0 sc1\n\t" \
    "s_mov_b64 exec, -1\n\t" \
    "s_add_u32 s39, s66, s43\n\t" \
    "s_add_u32 s39, s39, %[sWk0]\n\t" \
    "buffer_load_dwordx4 v[164:167], %[vW], %[rW], s39 offen\n\t" \
    "s_add_u32 s39, s39, 32768\n\t" \
    "buffer_load_dwordx4 v[168:171], %[vW], %[rW], s39 offen\n\t" \
    "s_add_u32 s39, s39, 32768\n\t" \
    "buffer_load_dwordx4 v[172:175], %[vW], %[rW], s39 offen\n\t" \
    "s_add_u32 s39, s39, 32768\n\t" \
    "buffer_load_dwordx4 v[176:179], %[vW], %[rW], s39 offen\n\t" \
    "s_add_u32 s39, s39, 32768\n\t" \
    "buffer_load_dwordx4 v[180:183], %[vW], %[rW], s39 offen\n\t" \
    "s_add_u32 s39, s39, 32768\n\t" \
    "buffer_load_dwordx4 v[184:187], %[vW], %[rW], s39 offen\n\t" \
    "s_add_u32 s39, s39, 32768\n\t" \
    "buffer_load_dwordx4 v[188:191], %[vW], %[rW], s39 offen\n\t" \
    "s_add_u32 s39, s39, 32768\n\t" \
    "buffer_load_dwordx4 v[192:195], %[vW], %[rW], s39 offen\n\t" \
    "s_add_u32 s40, s66, s43\n\t" \
    "s_add_u32 s40, s40, %[sWk1]\n\t" \
    "buffer_load_dwordx4 v[196:199], %[vW], %[rW], s40 offen nt\n\t" \
    "s_add_u32 s40, s40, 32768\n\t" \
    "buffer_load_dwordx4 v[200:203], %[vW], %[rW], s40 offen nt\n\t" \
    "s_add_u32 s40, s40, 32768\n\t" \
    "buffer_load_dwordx4 v[204:207], %[vW], %[rW], s40 offen nt\n\t" \
    "s_add_u32 s40, s40, 32768\n\t" \
    "buffer_load_dwordx4 v[208:211], %[vW], %[rW], s40 offen nt\n\t" \
    "s_add_u32 s40, s40, 32768\n\t" \
    "buffer_load_dwordx4 v[212:215], %[vW], %[rW], s40 offen nt\n\t" \
    "s_add_u32 s40, s40, 32768\n\t" \
    "buffer_load_dwordx4 v[216:219], %[vW], %[rW], s40 offen nt\n\t" \
    "s_add_u32 s40, s40, 32768\n\t" \
    "buffer_load_dwordx4 v[220:223], %[vW], %[rW], s40 offen nt\n\t" \
    "s_add_u32 s40, s40, 32768\n\t" \
    "buffer_load_dwordx4 v[224:227], %[vW], %[rW], s40 offen nt\n\t" \
    "v_accvgpr_write_b32 a0, 0\n\t" \
    "v_accvgpr_write_b32 a1, 0\n\t" \
    "v_accvgpr_write_b32 a2, 0\n\t" \
    "v_accvgpr_write_b32 a3, 0\n\t" \
    "v_accvgpr_write_b32 a4, 0\n\t" \
    "v_accvgpr_write_b32 a5, 0\n\t" \
    "v_accvgpr_write_b32 a6, 0\n\t" \
    "v_accvgpr_write_b32 a7, 0\n\t" \
    "v_accvgpr_write_b32 a8, 0\n\t" \
    "v_accvgpr_write_b32 a9, 0\n\t" \
    "v_accvgpr_write_b32 a10, 0\n\t" \
    "v_accvgpr_write_b32 a11, 0\n\t" \
    "v_accvgpr_write_b32 a12, 0\n\t" \
    "v_accvgpr_write_b32 a13, 0\n\t" \
    "v_accvgpr_write_b32 a14, 0\n\t" \
    "v_accvgpr_write_b32 a15, 0\n\t" \
    "v_accvgpr_write_b32 a16, 0\n\t" \
    "v_accvgpr_write_b32 a17, 0\n\t" \
    "v_accvgpr_write_b32 a18, 0\n\t" \
    "v_accvgpr_write_b32 a19, 0\n\t" \
    "v_accvgpr_write_b32 a20, 0\n\t" \
    "v_accvgpr_write_b32 a21, 0\n\t" \
    "v_accvgpr_write_b32 a22, 0\n\t" \
    "v_accvgpr_write_b32 a23, 0\n\t" \
    "v_accvgpr_write_b32 a24, 0\n\t" \
    "v_accvgpr_write_b32 a25, 0\n\t" \
    "v_accvgpr_write_b32 a26, 0\n\t" \
    "v_accvgpr_write_b32 a27, 0\n\t" \
    "v_accvgpr_write_b32 a28, 0\n\t" \
    "v_accvgpr_write_b32 a29, 0\n\t" \
    "v_accvgpr_write_b32 a30, 0\n\t" \
    "v_accvgpr_write_b32 a31, 0\n\t" \
    "v_accvgpr_write_b32 a32, 0\n\t" \
    "v_accvgpr_write_b32 a33, 0\n\t" \
    "v_accvgpr_write_b32 a34, 0\n\t" \
    "v_accvgpr_write_b32 a35, 0\n\t" \
    "v_accvgpr_write_b32 a36, 0\n\t" \
    "v_accvgpr_write_b32 a37, 0\n\t" \
    "v_accvgpr_write_b32 a38, 0\n\t" \
    "v_accvgpr_write_b32 a39, 0\n\t" \
    "v_accvgpr_write_b32 a40, 0\n\t" \
    "v_accvgpr_write_b32 a41, 0\n\t" \
    "v_accvgpr_write_b32 a42, 0\n\t" \
    "v_accvgpr_write_b32 a43, 0\n\t" \
    "v_accvgpr_write_b32 a44, 0\n\t" \
    "v_accvgpr_write_b32 a45, 0\n\t" \
    "v_accvgpr_write_b32 a46, 0\n\t" \
    "v_accvgpr_write_b32 a47, 0\n\t" \
    "v_accvgpr_write_b32 a48, 0\n\t" \
    "v_accvgpr_write_b32 a49, 0\n\t" \
    "v_accvgpr_write_b32 a50, 0\n\t" \
    "v_accvgpr_write_b32 a51, 0\n\t" \
    "v_accvgpr_write_b32 a52, 0\n\t" \
    "v_accvgpr_write_b32 a53, 0\n\t" \
    "v_accvgpr_write_b32 a54, 0\n\t" \
    "v_accvgpr_write_b32 a55, 0\n\t" \
    "v_accvgpr_write_b32 a56, 0\n\t" \
    "v_accvgpr_write_b32 a57, 0\n\t" \
    "v_accvgpr_write_b32 a58, 0\n\t" \
    "v_accvgpr_write_b32 a59, 0\n\t" \
    "v_accvgpr_write_b32 a60, 0\n\t" \
    "v_accvgpr_write_b32 a61, 0\n\t" \
    "v_accvgpr_write_b32 a62, 0\n\t" \
    "v_accvgpr_write_b32 a63, 0\n\t" \
    "v_accvgpr_write_b32 a64, 0\n\t" \
    "v_accvgpr_write_b32 a65, 0\n\t" \
    "v_accvgpr_write_b32 a66, 0\n\t" \
    "v_accvgpr_write_b32 a67, 0\n\t" \
    "v_accvgpr_write_b32 a68, 0\n\t" \
    "v_accvgpr_write_b32 a69, 0\n\t" \
    "v_accvgpr_write_b32 a70, 0\n\t" \
    "v_accvgpr_write_b32 a71, 0\n\t" \
    "v_accvgpr_write_b32 a72, 0\n\t" \
    "v_accvgpr_write_b32 a73, 0\n\t" \
    "v_accvgpr_write_b32 a74, 0\n\t" \
    "v_accvgpr_write_b32 a75, 0\n\t" \
    "v_accvgpr_write_b32 a76, 0\n\t" \
    "v_accvgpr_write_b32 a77, 0\n\t" \
    "v_accvgpr_write_b32 a78, 0\n\t" \
    "v_accvgpr_write_b32 a79, 0\n\t" \
    "v_accvgpr_write_b32 a80, 0\n\t" \
    "v_accvgpr_write_b32 a81, 0\n\t" \
    "v_accvgpr_write_b32 a82, 0\n\t" \
    "v_accvgpr_write_b32 a83, 0\n\t" \
    "v_accvgpr_write_b32 a84, 0\n\t" \
    "v_accvgpr_write_b32 a85, 0\n\t" \
    "v_accvgpr_write_b32 a86, 0\n\t" \
    "v_accvgpr_write_b32 a87, 0\n\t" \
    "v_accvgpr_write_b32 a88, 0\n\t" \
    "v_accvgpr_write_b32 a89, 0\n\t" \
    "v_accvgpr_write_b32 a90, 0\n\t" \
    "v_accvgpr_write_b32 a91, 0\n\t" \
    "v_accvgpr_write_b32 a92, 0\n\t" \
    "v_accvgpr_write_b32 a93, 0\n\t" \
    "v_accvgpr_write_b32 a94, 0\n\t" \
    "v_accvgpr_write_b32 a95, 0\n\t" \
    "v_accvgpr_write_b32 a96, 0\n\t" \
    "v_accvgpr_write_b32 a97, 0\n\t" \
    "v_accvgpr_write_b32 a98, 0\n\t" \
    "v_accvgpr_write_b32 a99, 0\n\t" \
    "v_accvgpr_write_b32 a100, 0\n\t" \
    "v_accvgpr_write_b32 a101, 0\n\t" \
    "v_accvgpr_write_b32 a102, 0\n\t" \
    "v_accvgpr_write_b32 a103, 0\n\t" \
    "v_accvgpr_write_b32 a104, 0\n\t" \
    "v_accvgpr_write_b32 a105, 0\n\t" \
    "v_accvgpr_write_b32 a106, 0\n\t" \
    "v_accvgpr_write_b32 a107, 0\n\t" \
    "v_accvgpr_write_b32 a108, 0\n\t" \
    "v_accvgpr_write_b32 a109, 0\n\t" \
    "v_accvgpr_write_b32 a110, 0\n\t" \
    "v_accvgpr_write_b32 a111, 0\n\t" \
    "v_accvgpr_write_b32 a112, 0\n\t" \
    "v_accvgpr_write_b32 a113, 0\n\t" \
    "v_accvgpr_write_b32 a114, 0\n\t" \
    "v_accvgpr_write_b32 a115, 0\n\t" \
    "v_accvgpr_write_b32 a116, 0\n\t" \
    "v_accvgpr_write_b32 a117, 0\n\t" \
    "v_accvgpr_write_b32 a118, 0\n\t" \
    "v_accvgpr_write_b32 a119, 0\n\t" \
    "v_accvgpr_write_b32 a120, 0\n\t" \
    "v_accvgpr_write_b32 a121, 0\n\t" \
    "v_accvgpr_write_b32 a122, 0\n\t" \
    "v_accvgpr_write_b32 a123, 0\n\t" \
    "v_accvgpr_write_b32 a124, 0\n\t" \
    "v_accvgpr_write_b32 a125, 0\n\t" \
    "v_accvgpr_write_b32 a126, 0\n\t" \
    "v_accvgpr_write_b32 a127, 0\n\t" \
    "v_accvgpr_write_b32 a128, 0\n\t" \
    "v_accvgpr_write_b32 a129, 0\n\t" \
    "v_accvgpr_write_b32 a130, 0\n\t" \
    "v_accvgpr_write_b32 a131, 0\n\t" \
    "v_accvgpr_write_b32 a132, 0\n\t" \
    "v_accvgpr_write_b32 a133, 0\n\t" \
    "v_accvgpr_write_b32 a134, 0\n\t" \
    "v_accvgpr_write_b32 a135, 0\n\t" \
    "v_accvgpr_write_b32 a136, 0\n\t" \
    "v_accvgpr_write_b32 a137, 0\n\t" \
    "v_accvgpr_write_b32 a138, 0\n\t" \
    "v_accvgpr_write_b32 a139, 0\n\t" \
    "v_accvgpr_write_b32 a140, 0\n\t" \
    "v_accvgpr_write_b32 a141, 0\n\t" \
    "v_accvgpr_write_b32 a142, 0\n\t" \
    "v_accvgpr_write_b32 a143, 0\n\t" \
    "v_accvgpr_write_b32 a144, 0\n\t" \
    "v_accvgpr_write_b32 a145, 0\n\t" \
    "v_accvgpr_write_b32 a146, 0\n\t" \
    "v_accvgpr_write_b32 a147, 0\n\t" \
    "v_accvgpr_write_b32 a148, 0\n\t" \
    "v_accvgpr_write_b32 a149, 0\n\t" \
    "v_accvgpr_write_b32 a150, 0\n\t" \
    "v_accvgpr_write_b32 a151, 0\n\t" \
    "v_accvgpr_write_b32 a152, 0\n\t" \
    "v_accvgpr_write_b32 a153, 0\n\t" \
    "v_accvgpr_write_b32 a154, 0\n\t" \
    "v_accvgpr_write_b32 a155, 0\n\t" \
    "v_accvgpr_write_b32 a156, 0\n\t" \
    "v_accvgpr_write_b32 a157, 0\n\t" \
    "v_accvgpr_write_b32 a158, 0\n\t" \
    "v_accvgpr_write_b32 a159, 0\n\t" \
    "v_accvgpr_write_b32 a160, 0\n\t" \
    "v_accvgpr_write_b32 a161, 0\n\t" \
    "v_accvgpr_write_b32 a162, 0\n\t" \
    "v_accvgpr_write_b32 a163, 0\n\t" \
    "v_accvgpr_write_b32 a164, 0\n\t" \
    "v_accvgpr_write_b32 a165, 0\n\t" \
    "v_accvgpr_write_b32 a166, 0\n\t" \
    "v_accvgpr_write_b32 a167, 0\n\t" \
    "v_accvgpr_write_b32 a168, 0\n\t" \
    "v_accvgpr_write_b32 a169, 0\n\t" \
    "v_accvgpr_write_b32 a170, 0\n\t" \
    "v_accvgpr_write_b32 a171, 0\n\t" \
    "v_accvgpr_write_b32 a172, 0\n\t" \
    "v_accvgpr_write_b32 a173, 0\n\t" \
    "v_accvgpr_write_b32 a174, 0\n\t" \
    "v_accvgpr_write_b32 a175, 0\n\t" \
    "v_accvgpr_write_b32 a176, 0\n\t" \
    "v_accvgpr_write_b32 a177, 0\n\t" \
    "v_accvgpr_write_b32 a178, 0\n\t" \
    "v_accvgpr_write_b32 a179, 0\n\t" \
    "v_accvgpr_write_b32 a180, 0\n\t" \
    "v_accvgpr_write_b32 a181, 0\n\t" \
    "v_accvgpr_write_b32 a182, 0\n\t" \
    "v_accvgpr_write_b32 a183, 0\n\t" \
    "v_accvgpr_write_b32 a184, 0\n\t" \
    "v_accvgpr_write_b32 a185, 0\n\t" \
    "v_accvgpr_write_b32 a186, 0\n\t" \
    "v_accvgpr_write_b32 a187, 0\n\t" \
    "v_accvgpr_write_b32 a188, 0\n\t" \
    "v_accvgpr_write_b32 a189, 0\n\t" \
    "v_accvgpr_write_b32 a190, 0\n\t" \
    "v_accvgpr_write_b32 a191, 0\n\t" \
    "v_accvgpr_write_b32 a192, 0\n\t" \
    "v_accvgpr_write_b32 a193, 0\n\t" \
    "v_accvgpr_write_b32 a194, 0\n\t" \
    "v_accvgpr_write_b32 a195, 0\n\t" \
    "v_accvgpr_write_b32 a196, 0\n\t" \
    "v_accvgpr_write_b32 a197, 0\n\t" \
    "v_accvgpr_write_b32 a198, 0\n\t" \
    "v_accvgpr_write_b32 a199, 0\n\t" \
    "v_accvgpr_write_b32 a200, 0\n\t" \
    "v_accvgpr_write_b32 a201, 0\n\t" \
    "v_accvgpr_write_b32 a202, 0\n\t" \
    "v_accvgpr_write_b32 a203, 0\n\t" \
    "v_accvgpr_write_b32 a204, 0\n\t" \
    "v_accvgpr_write_b32 a205, 0\n\t" \
    "v_accvgpr_write_b32 a206, 0\n\t" \
    "v_accvgpr_write_b32 a207, 0\n\t" \
    "v_accvgpr_write_b32 a208, 0\n\t" \
    "v_accvgpr_write_b32 a209, 0\n\t" \
    "v_accvgpr_write_b32 a210, 0\n\t" \
    "v_accvgpr_write_b32 a211, 0\n\t" \
    "v_accvgpr_write_b32 a212, 0\n\t" \
    "v_accvgpr_write_b32 a213, 0\n\t" \
    "v_accvgpr_write_b32 a214, 0\n\t" \
    "v_accvgpr_write_b32 a215, 0\n\t" \
    "v_accvgpr_write_b32 a216, 0\n\t" \
    "v_accvgpr_write_b32 a217, 0\n\t" \
    "v_accvgpr_write_b32 a218, 0\n\t" \
    "v_accvgpr_write_b32 a219, 0\n\t" \
    "v_accvgpr_write_b32 a220, 0\n\t" \
    "v_accvgpr_write_b32 a221, 0\n\t" \
    "v_accvgpr_write_b32 a222, 0\n\t" \
    "v_accvgpr_write_b32 a223, 0\n\t" \
    "v_accvgpr_write_b32 a224, 0\n\t" \
    "v_accvgpr_write_b32 a225, 0\n\t" \
    "v_accvgpr_write_b32 a226, 0\n\t" \
    "v_accvgpr_write_b32 a227, 0\n\t" \
    "v_accvgpr_write_b32 a228, 0\n\t" \
    "v_accvgpr_write_b32 a229, 0\n\t" \
    "v_accvgpr_write_b32 a230, 0\n\t" \
    "v_accvgpr_write_b32 a231, 0\n\t" \
    "v_accvgpr_write_b32 a232, 0\n\t" \
    "v_accvgpr_write_b32 a233, 0\n\t" \
    "v_accvgpr_write_b32 a234, 0\n\t" \
    "v_accvgpr_write_b32 a235, 0\n\t" \
    "v_accvgpr_write_b32 a236, 0\n\t" \
    "v_accvgpr_write_b32 a237, 0\n\t" \
    "v_accvgpr_write_b32 a238, 0\n\t" \
    "v_accvgpr_write_b32 a239, 0\n\t" \
    "v_accvgpr_write_b32 a240, 0\n\t" \
    "v_accvgpr_write_b32 a241, 0\n\t" \
    "v_accvgpr_write_b32 a242, 0\n\t" \
    "v_accvgpr_write_b32 a243, 0\n\t" \
    "v_accvgpr_write_b32 a244, 0\n\t" \
    "v_accvgpr_write_b32 a245, 0\n\t" \
    "v_accvgpr_write_b32 a246, 0\n\t" \
    "v_accvgpr_write_b32 a247, 0\n\t" \
    "v_accvgpr_write_b32 a248, 0\n\t" \
    "v_accvgpr_write_b32 a249, 0\n\t" \
    "v_accvgpr_write_b32 a250, 0\n\t" \
    "v_accvgpr_write_b32 a251, 0\n\t" \
    "v_accvgpr_write_b32 a252, 0\n\t" \
    "v_accvgpr_write_b32 a253, 0\n\t" \
    "v_accvgpr_write_b32 a254, 0\n\t" \
    "v_accvgpr_write_b32 a255, 0\n\t" \
    "v_mov_b32 v252, 0\n\t" \
    "v_mov_b32 v253, 0\n\t" \
    "v_mov_b32 v254, 0\n\t" \
    "v_mov_b32 v255, 0\n\t" \
    "s_waitcnt lgkmcnt(0)\n\t" \
    "s_waitcnt vmcnt(8)\n\t" \
    "v_readfirstlane_b32 s35, v18\n\t" \
    "v_mov_b32 v19, 0\n\t" \
    "v_cvt_pk_f16_f32 v228, v164, v168\n\t" \
    "v_cvt_pk_f16_f32 v229, v172, v176\n\t" \
    "ds_write_b64 %[vWr0], v[228:229] offset:0\n\t" \
    "v_cvt_pk_f16_f32 v230, v180, v184\n\t" \
    "v_cvt_pk_f16_f32 v231, v188, v192\n\t" \
    "ds_write_b64 %[vWr0], v[230:231] offset:8\n\t" \
    "v_cvt_pk_f16_f32 v228, v165, v169\n\t" \
    "v_cvt_pk_f16_f32 v229, v173, v177\n\t" \
    "ds_write_b64 %[vWr0], v[228:229] offset:1024\n\t" \
    "v_cvt_pk_f16_f32 v230, v181, v185\n\t" \
    "v_cvt_pk_f16_f32 v231, v189, v193\n\t" \
    "ds_write_b64 %[vWr0], v[230:231] offset:1032\n\t" \
    "v_cvt_pk_f16_f32 v228, v166, v170\n\t" \
    "v_cvt_pk_f16_f32 v229, v174, v178\n\t" \
    "ds_write_b64 %[vWr0], v[228:229] offset:2048\n\t" \
    "v_cvt_pk_f16_f32 v230, v182, v186\n\t" \
    "v_cvt_pk_f16_f32 v231, v190, v194\n\t" \
    "ds_write_b64 %[vWr0], v[230:231] offset:2056\n\t" \
    "v_cvt_pk_f16_f32 v228, v167, v171\n\t" \
    "v_cvt_pk_f16_f32 v229, v175, v179\n\t" \
    "ds_write_b64 %[vWr0], v[228:229] offset:3072\n\t" \
    "v_cvt_pk_f16_f32 v230, v183, v187\n\t" \
    "v_cvt_pk_f16_f32 v231, v191, v195\n\t" \
    "ds_write_b64 %[vWr0], v[230:231] offset:3080\n\t" \
    "s_add_u32 s66, s66, 0x400000\n\t" \
    "s_and_b32 s66, s66, 0xfffffff\n\t" \
    "s_add_u32 s39, s66, s43\n\t" \
    "s_add_u32 s39, s39, %[sWk0]\n\t" \
    "buffer_load_dwordx4 v[164:167], %[vW], %[rW], s39 offen\n\t" \
    "s_add_u32 s39, s39, 32768\n\t" \
    "buffer_load_dwordx4 v[168:171], %[vW], %[rW], s39 offen\n\t" \
    "s_add_u32 s39, s39, 32768\n\t" \
    "buffer_load_dwordx4 v[172:175], %[vW], %[rW], s39 offen\n\t" \
    "s_add_u32 s39, s39, 32768\n\t" \
    "buffer_load_dwordx4 v[176:179], %[vW], %[rW], s39 offen\n\t" \
    "s_add_u32 s39, s39, 32768\n\t" \
    "buffer_load_dwordx4 v[180:183], %[vW], %[rW], s39 offen\n\t" \
    "s_add_u32 s39, s39, 32768\n\t" \
    "buffer_load_dwordx4 v[184:187], %[vW], %[rW], s39 offen\n\t" \
    "s_add_u32 s39, s39, 32768\n\t" \
    "buffer_load_dwordx4 v[188:191], %[vW], %[rW], s39 offen\n\t" \
    "s_add_u32 s39, s39, 32768\n\t" \
    "buffer_load_dwordx4 v[192:195], %[vW], %[rW], s39 offen\n\t" \
    "s_add_u32 s35, s35, 1\n\t" \
    "s_lshl_b32 s49, s35, 4\n\t" \
    "s_lshl_b32 s70, s35, 4\n\t" \
    "s_mov_b32 s34, 8192\n\t" \
    "s_waitcnt vmcnt(8)\n\t" \
    "v_cvt_pk_f16_f32 v228, v196, v200\n\t" \
    "v_cvt_pk_f16_f32 v229, v204, v208\n\t" \
    "ds_write_b64 %[vWr1], v[228:229] offset:0\n\t" \
    "v_cvt_pk_f16_f32 v230, v212, v216\n\t" \
    "v_cvt_pk_f16_f32 v231, v220, v224\n\t" \
    "ds_write_b64 %[vWr1], v[230:231] offset:8\n\t" \
    "v_cvt_pk_f16_f32 v228, v197, v201\n\t" \
    "v_cvt_pk_f16_f32 v229, v205, v209\n\t" \
    "ds_write_b64 %[vWr1], v[228:229] offset:1024\n\t" \
    "v_cvt_pk_f16_f32 v230, v213, v217\n\t" \
    "v_cvt_pk_f16_f32 v231, v221, v225\n\t" \
    "ds_write_b64 %[vWr1], v[230:231] offset:1032\n\t" \
    "v_cvt_pk_f16_f32 v228, v198, v202\n\t" \
    "v_cvt_pk_f16_f32 v229, v206, v210\n\t" \
    "ds_write_b64 %[vWr1], v[228:229] offset:2048\n\t" \
    "v_cvt_pk_f16_f32 v230, v214, v218\n\t" \
    "v_cvt_pk_f16_f32 v231, v222, v226\n\t" \
    "ds_write_b64 %[vWr1], v[230:231] offset:2056\n\t" \
    "v_cvt_pk_f16_f32 v228, v199, v203\n\t" \
    "v_cvt_pk_f16_f32 v229, v207, v211\n\t" \
    "ds_write_b64 %[vWr1], v[228:229] offset:3072\n\t" \
    "v_cvt_pk_f16_f32 v230, v215, v219\n\t" \
    "v_cvt_pk_f16_f32 v231, v223, v227\n\t" \
    "ds_write_b64 %[vWr1], v[230:231] offset:3080\n\t" \
    "s_mov_b32 s55, %[sXp0]\n\t" \
    "v_cmp_le_u32 s[76:77], s70, v19\n\t" \
    "s_nop 1\n\t" \
    "s_cmp_eq_u64 s[76:77], exec\n\t" \
    "s_cbranch_scc0 SLOW_PX%=\n\t" \
    "BACK_PX%=:\n\t" \
    "s_add_u32 s55, s38, 0\n\t" \
    "buffer_load_dwordx4 v[20:23], %[vA0], %[rWS], s55 offen offset:0 sc0 sc1\n\t" \
    "buffer_load_dwordx4 v[24:27], %[vA0], %[rWS], s55 offen offset:1024 sc0 sc1\n\t" \
    "buffer_load_dwordx4 v[28:31], %[vA0], %[rWS], s55 offen offset:2048 sc0 sc1\n\t" \
    "buffer_load_dwordx4 v[32:35], %[vA0], %[rWS], s55 offen offset:3072 sc0 sc1\n\t" \
    "s_add_u32 s55, s38, 4096\n\t" \
    "buffer_load_dwordx4 v[36:39], %[vA0], %[rWS], s55 offen offset:0 sc0 sc1\n\t" \
    "buffer_load_dwordx4 v[40:43], %[vA0], %[rWS], s55 offen offset:1024 sc0 sc1\n\t" \
    "buffer_load_dwordx4 v[44:47], %[vA0], %[rWS], s55 offen offset:2048 sc0 sc1\n\t" \
    "buffer_load_dwordx4 v[48:51], %[vA0], %[rWS], s55 offen offset:3072 sc0 sc1\n\t" \
    "s_waitcnt lgkmcnt(0)\n\t" \
    "s_barrier\n\t" \
    "ds_read_b128 v[148:151], %[vA0] offset:0\n\t" \
    "ds_read_b128 v[152:155], %[vA0] offset:8192\n\t" \
    "s_add_u32 s55, s38, 8192\n\t" \
    "buffer_load_dwordx4 v[52:55], %[vA0], %[rWS], s55 offen offset:0 sc0 sc1\n\t" \
    "buffer_load_dwordx4 v[56:59], %[vA0], %[rWS], s55 offen offset:1024 sc0 sc1\n\t" \
    "buffer_load_dwordx4 v[60:63], %[vA0], %[rWS], s55 offen offset:2048 sc0 sc1\n\t" \
    "buffer_load_dwordx4 v[64:67], %[vA0], %[rWS], s55 offen offset:3072 sc0 sc1\n\t" \
    "s_add_u32 s55, s38, 12288\n\t" \
    "buffer_load_dwordx4 v[68:71], %[vA0], %[rWS], s55 offen offset:0 sc0 sc1\n\t" \
    "buffer_load_dwordx4 v[72:75], %[vA0], %[rWS], s55 offen offset:1024 sc0 sc1\n\t" \
    "buffer_load_dwordx4 v[76:79], %[vA0], %[rWS], s55 offen offset:2048 sc0 sc1\n\t" \
    "buffer_load_dwordx4 v[80:83], %[vA0], %[rWS], s55 offen offset:3072 sc0 sc1\n\t" \
    "s_add_u32 s40, s66, s43\n\t" \
    "s_add_u32 s40, s40, %[sWk1]\n\t" \
    "buffer_load_dwordx4 v[196:199], %[vW], %[rW], s40 offen nt\n\t" \
    "s_add_u32 s40, s40, 32768\n\t" \
    "buffer_load_dwordx4 v[200:203], %[vW], %[rW], s40 offen nt\n\t" \
    "s_add_u32 s40, s40, 32768\n\t" \
    "buffer_load_dwordx4 v[204:207], %[vW], %[rW], s40 offen nt\n\t" \
    "s_add_u32 s40, s40, 32768\n\t" \
    "buffer_load_dwordx4 v[208:211], %[vW], %[rW], s40 offen nt\n\t" \
    "s_add_u32 s40, s40, 32768\n\t" \
    "buffer_load_dwordx4 v[212:215], %[vW], %[rW], s40 offen nt\n\t" \
    "s_add_u32 s40, s40, 32768\n\t" \
    "buffer_load_dwordx4 v[216:219], %[vW], %[rW], s40 offen nt\n\t" \
    "s_add_u32 s40, s40, 32768\n\t" \
    "buffer_load_dwordx4 v[220:223], %[vW], %[rW], s40 offen nt\n\t" \
    "s_add_u32 s40, s40, 32768\n\t" \
    "buffer_load_dwordx4 v[224:227], %[vW], %[rW], s40 offen nt\n\t" \
    "s_cmp_eq_u32 %[pop], 1\n\t" \
    "s_cselect_b32 s67, 80, 0\n\t" \
    "s_bitcmp1_b32 s67, 0\n\t" \
    "s_cbranch_scc1 LOOP1%=\n\t" \
    "LOOP0%=:\n\t" \
    "ds_read_b128 v[156:159], %[vA0] offset:1024\n\t" \
    "s_waitcnt vmcnt(23) lgkmcnt(2)\n\t" \
    "v_mfma_f32_16x16x32_f16 a[0:3], v[20:23], v[148:151], a[0:3]\n\t" \
    "s_mov_b32 s37, s38\n\t" \
    "s_add_u32 s55, s37, 16384\n\t" \
    "s_waitcnt vmcnt(15) lgkmcnt(1)\n\t" \
    "v_mfma_f32_16x16x32_f16 a[0:3], v[52:55], v[152:155], a[0:3]\n\t" \
    "s_waitcnt vmcnt(14)\n\t" \
    "v_mfma_f32_16x16x32_f16 a[32:35], v[56:59], v[152:155], a[32:35]\n\t" \
    "buffer_load_dwordx4 v[84:87], %[vA0], %[rWS], s55 offen offset:0 sc0 sc1\n\t" \
    "v_mfma_f32_16x16x32_f16 a[32:35], v[24:27], v[148:151], a[32:35]\n\t" \
    "v_mfma_f32_16x16x32_f16 a[64:67], v[28:31], v[148:151], a[64:67]\n\t" \
    "s_waitcnt vmcnt(14)\n\t" \
    "v_mfma_f32_16x16x32_f16 a[64:67], v[60:63], v[152:155], a[64:67]\n\t" \
    "s_waitcnt vmcnt(13)\n\t" \
    "v_mfma_f32_16x16x32_f16 a[96:99], v[64:67], v[152:155], a[96:99]\n\t" \
    "buffer_load_dwordx4 v[88:91], %[vA0], %[rWS], s55 offen offset:1024 sc0 sc1\n\t" \
    "v_mfma_f32_16x16x32_f16 a[96:99], v[32:35], v[148:151], a[96:99]\n\t" \
    "s_nop 15\n\t" \
    "s_nop 3\n\t" \
    "ds_read_b128 v[160:163], %[vA0] offset:9216\n\t" \
    "v_mfma_f32_16x16x32_f16 a[128:131], v[36:39], v[148:151], a[128:131]\n\t" \
    "s_waitcnt vmcnt(13)\n\t" \
    "v_mfma_f32_16x16x32_f16 a[128:131], v[68:71], v[152:155], a[128:131]\n\t" \
    "s_waitcnt vmcnt(12)\n\t" \
    "v_mfma_f32_16x16x32_f16 a[160:163], v[72:75], v[152:155], a[160:163]\n\t" \
    "buffer_load_dwordx4 v[92:95], %[vA0], %[rWS], s55 offen offset:2048 sc0 sc1\n\t" \
    "v_mfma_f32_16x16x32_f16 a[160:163], v[40:43], v[148:151], a[160:163]\n\t" \
    "v_mfma_f32_16x16x32_f16 a[192:195], v[44:47], v[148:151], a[192:195]\n\t" \
    "s_waitcnt vmcnt(12)\n\t" \
    "v_mfma_f32_16x16x32_f16 a[192:195], v[76:79], v[152:155], a[192:195]\n\t" \
    "s_waitcnt vmcnt(11)\n\t" \
    "v_mfma_f32_16x16x32_f16 a[224:227], v[80:83], v[152:155], a[224:227]\n\t" \
    "buffer_load_dwordx4 v[96:99], %[vA0], %[rWS], s55 offen offset:3072 sc0 sc1\n\t" \
    "v_mfma_f32_16x16x32_f16 a[224:227], v[48:51], v[148:151], a[224:227]\n\t" \
    "s_nop 15\n\t" \
    "s_nop 3\n\t" \
    "ds_read_b128 v[148:151], %[vA0] offset:2048\n\t" \
    "s_waitcnt lgkmcnt(2)\n\t" \
    "v_mfma_f32_16x16x32_f16 a[228:231], v[48:51], v[156:159], a[228:231]\n\t" \
    "s_add_u32 s55, s37, 20480\n\t" \
    "s_waitcnt lgkmcnt(1)\n\t" \
    "v_mfma_f32_16x16x32_f16 a[228:231], v[80:83], v[160:163], a[228:231]\n\t" \
    "v_mfma_f32_16x16x32_f16 a[196:199], v[76:79], v[160:163], a[196:199]\n\t" \
    "buffer_load_dwordx4 v[100:103], %[vA0], %[rWS], s55 offen offset:0 sc0 sc1\n\t" \
    "v_mfma_f32_16x16x32_f16 a[196:199], v[44:47], v[156:159], a[196:199]\n\t" \
    "v_mfma_f32_16x16x32_f16 a[164:167], v[40:43], v[156:159], a[164:167]\n\t" \
    "v_mfma_f32_16x16x32_f16 a[164:167], v[72:75], v[160:163], a[164:167]\n\t" \
    "v_mfma_f32_16x16x32_f16 a[132:135], v[68:71], v[160:163], a[132:135]\n\t" \
    "buffer_load_dwordx4 v[104:107], %[vA0], %[rWS], s55 offen offset:1024 sc0 sc1\n\t" \
    "v_mfma_f32_16x16x32_f16 a[132:135], v[36:39], v[156:159], a[132:135]\n\t" \
    "s_nop 15\n\t" \
    "s_nop 3\n\t" \
    "ds_read_b128 v[152:155], %[vA0] offset:10240\n\t" \
    "v_mfma_f32_16x16x32_f16 a[100:103], v[32:35], v[156:159], a[100:103]\n\t" \
    "v_cvt_pk_f16_f32 v228, v164, v168\n\t" \
    "v_mfma_f32_16x16x32_f16 a[100:103], v[64:67], v[160:163], a[100:103]\n\t" \
    "v_mfma_f32_16x16x32_f16 a[68:71], v[60:63], v[160:163], a[68:71]\n\t" \
    "buffer_load_dwordx4 v[108:111], %[vA0], %[rWS], s55 offen offset:2048 sc0 sc1\n\t" \
    "v_mfma_f32_16x16x32_f16 a[68:71], v[28:31], v[156:159], a[68:71]\n\t" \
    "v_cvt_pk_f16_f32 v229, v172, v176\n\t" \
    "v_mfma_f32_16x16x32_f16 a[36:39], v[24:27], v[156:159], a[36:39]\n\t" \
    "ds_write_b64 %[vWr0], v[228:229] offset:32768\n\t" \
    "v_mfma_f32_16x16x32_f16 a[36:39], v[56:59], v[160:163], a[36:39]\n\t" \
    "v_cvt_pk_f16_f32 v230, v180, v184\n\t" \
    "v_mfma_f32_16x16x32_f16 a[4:7], v[52:55], v[160:163], a[4:7]\n\t" \
    "buffer_load_dwordx4 v[112:115], %[vA0], %[rWS], s55 offen offset:3072 sc0 sc1\n\t" \
    "v_mfma_f32_16x16x32_f16 a[4:7], v[20:23], v[156:159], a[4:7]\n\t" \
    "s_nop 15\n\t" \
    "s_nop 3\n\t" \
    "ds_read_b128 v[156:159], %[vA0] offset:3072\n\t" \
    "s_waitcnt lgkmcnt(3)\n\t" \
    "v_mfma_f32_16x16x32_f16 a[8:11], v[20:23], v[148:151], a[8:11]\n\t" \
    "v_cvt_pk_f16_f32 v231, v188, v192\n\t" \
    "s_waitcnt lgkmcnt(2)\n\t" \
    "v_mfma_f32_16x16x32_f16 a[8:11], v[52:55], v[152:155], a[8:11]\n\t" \
    "ds_write_b64 %[vWr0], v[230:231] offset:32776\n\t" \
    "v_mfma_f32_16x16x32_f16 a[40:43], v[56:59], v[152:155], a[40:43]\n\t" \
    "s_add_u32 s55, s37, 24576\n\t" \
    "v_cvt_pk_f16_f32 v228, v165, v169\n\t" \
    "v_mfma_f32_16x16x32_f16 a[40:43], v[24:27], v[148:151], a[40:43]\n\t" \
    "buffer_load_dwordx4 v[116:119], %[vA0], %[rWS], s55 offen offset:0 sc0 sc1\n\t" \
    "v_mfma_f32_16x16x32_f16 a[72:75], v[28:31], v[148:151], a[72:75]\n\t" \
    "v_cvt_pk_f16_f32 v229, v173, v177\n\t" \
    "s_add_u32 s38, s38, 32768\n\t" \
    "v_mfma_f32_16x16x32_f16 a[72:75], v[60:63], v[152:155], a[72:75]\n\t" \
    "s_and_b32 s38, s38, 0x1fffff\n\t" \
    "s_or_b32 s38, s38, %[sA0]\n\t" \
    "v_mfma_f32_16x16x32_f16 a[104:107], v[64:67], v[152:155], a[104:107]\n\t" \
    "ds_write_b64 %[vWr0], v[228:229] offset:33792\n\t" \
    "v_mfma_f32_16x16x32_f16 a[104:107], v[32:35], v[148:151], a[104:107]\n\t" \
    "s_nop 15\n\t" \
    "s_nop 3\n\t" \
    "ds_read_b128 v[160:163], %[vA0] offset:11264\n\t" \
    "v_mfma_f32_16x16x32_f16 a[136:139], v[36:39], v[148:151], a[136:139]\n\t" \
    "v_cvt_pk_f16_f32 v230, v181, v185\n\t" \
    "v_mfma_f32_16x16x32_f16 a[136:139], v[68:71], v[152:155], a[136:139]\n\t" \
    "buffer_load_dwordx4 v[120:123], %[vA0], %[rWS], s55 offen offset:1024 sc0 sc1\n\t" \
    "v_mfma_f32_16x16x32_f16 a[168:171], v[72:75], v[152:155], a[168:171]\n\t" \
    "v_cvt_pk_f16_f32 v231, v189, v193\n\t" \
    "v_mfma_f32_16x16x32_f16 a[168:171], v[40:43], v[148:151], a[168:171]\n\t" \
    "ds_write_b64 %[vWr0], v[230:231] offset:33800\n\t" \
    "v_mfma_f32_16x16x32_f16 a[200:203], v[44:47], v[148:151], a[200:203]\n\t" \
    "v_cvt_pk_f16_f32 v228, v166, v170\n\t" \
    "v_mfma_f32_16x16x32_f16 a[200:203], v[76:79], v[152:155], a[200:203]\n\t" \
    "buffer_load_dwordx4 v[124:127], %[vA0], %[rWS], s55 offen offset:2048 sc0 sc1\n\t" \
    "v_mfma_f32_16x16x32_f16 a[232:235], v[80:83], v[152:155], a[232:235]\n\t" \
    "v_cvt_pk_f16_f32 v229, v174, v178\n\t" \
    "v_mfma_f32_16x16x32_f16 a[232:235], v[48:51], v[148:151], a[232:235]\n\t" \
    "s_nop 15\n\t" \
    "s_nop 3\n\t" \
    "ds_read_b128 v[148:151], %[vA0] offset:4096\n\t" \
    "s_waitcnt lgkmcnt(5)\n\t" \
    "v_mfma_f32_16x16x32_f16 a[236:239], v[48:51], v[156:159], a[236:239]\n\t" \
    "ds_write_b64 %[vWr0], v[228:229] offset:34816\n\t" \
    "s_waitcnt lgkmcnt(3)\n\t" \
    "v_mfma_f32_16x16x32_f16 a[236:239], v[80:83], v[160:163], a[236:239]\n\t" \
    "v_cvt_pk_f16_f32 v230, v182, v186\n\t" \
    "v_mfma_f32_16x16x32_f16 a[204:207], v[76:79], v[160:163], a[204:207]\n\t" \
    "buffer_load_dwordx4 v[128:131], %[vA0], %[rWS], s55 offen offset:3072 sc0 sc1\n\t" \
    "v_mfma_f32_16x16x32_f16 a[204:207], v[44:47], v[156:159], a[204:207]\n\t" \
    "v_cvt_pk_f16_f32 v231, v190, v194\n\t" \
    "v_mfma_f32_16x16x32_f16 a[172:175], v[40:43], v[156:159], a[172:175]\n\t" \
    "ds_write_b64 %[vWr0], v[230:231] offset:34824\n\t" \
    "v_mfma_f32_16x16x32_f16 a[172:175], v[72:75], v[160:163], a[172:175]\n\t" \
    "s_add_u32 s55, s37, 28672\n\t" \
    "v_cvt_pk_f16_f32 v228, v167, v171\n\t" \
    "v_mfma_f32_16x16x32_f16 a[140:143], v[68:71], v[160:163], a[140:143]\n\t" \
    "buffer_load_dwordx4 v[132:135], %[vA0], %[rWS], s55 offen offset:0 sc0 sc1\n\t" \
    "v_mfma_f32_16x16x32_f16 a[140:143], v[36:39], v[156:159], a[140:143]\n\t" \
    "s_nop 15\n\t" \
    "s_nop 3\n\t" \
    "ds_read_b128 v[152:155], %[vA0] offset:12288\n\t" \
    "v_mfma_f32_16x16x32_f16 a[108:111], v[32:35], v[156:159], a[108:111]\n\t" \
    "v_cvt_pk_f16_f32 v229, v175, v179\n\t" \
    "v_mfma_f32_16x16x32_f16 a[108:111], v[64:67], v[160:163], a[108:111]\n\t" \
    "ds_write_b64 %[vWr0], v[228:229] offset:35840\n\t" \
    "v_mfma_f32_16x16x32_f16 a[76:79], v[60:63], v[160:163], a[76:79]\n\t" \
    "v_cvt_pk_f16_f32 v230, v183, v187\n\t" \
    "v_mfma_f32_16x16x32_f16 a[76:79], v[28:31], v[156:159], a[76:79]\n\t" \
    "buffer_load_dwordx4 v[136:139], %[vA0], %[rWS], s55 offen offset:1024 sc0 sc1\n\t" \
    "v_mfma_f32_16x16x32_f16 a[44:47], v[24:27], v[156:159], a[44:47]\n\t" \
    "s_add_u32 s66, s66, 0x400000\n\t" \
    "s_and_b32 s66, s66, 0xfffffff\n\t" \
    "v_mfma_f32_16x16x32_f16 a[44:47], v[56:59], v[160:163], a[44:47]\n\t" \
    "v_cvt_pk_f16_f32 v231, v191, v195\n\t" \
    "v_mfma_f32_16x16x32_f16 a[12:15], v[52:55], v[160:163], a[12:15]\n\t" \
    "ds_write_b64 %[vWr0], v[230:231] offset:35848\n\t" \
    "v_mfma_f32_16x16x32_f16 a[12:15], v[20:23], v[156:159], a[12:15]\n\t" \
    "s_nop 15\n\t" \
    "s_nop 3\n\t" \
    "ds_read_b128 v[156:159], %[vA0] offset:5120\n\t" \
    "s_waitcnt lgkmcnt(6)\n\t" \
    "v_mfma_f32_16x16x32_f16 a[16:19], v[20:23], v[148:151], a[16:19]\n\t" \
    "buffer_load_dwordx4 v[140:143], %[vA0], %[rWS], s55 offen offset:2048 sc0 sc1\n\t" \
    "s_waitcnt lgkmcnt(3)\n\t" \
    "v_mfma_f32_16x16x32_f16 a[16:19], v[52:55], v[152:155], a[16:19]\n\t" \
    "s_and_b32 s41, s36, 7\n\t" \
    "s_cmp_gt_u32 s41, 5\n\t" \
    "v_mfma_f32_16x16x32_f16 a[48:51], v[56:59], v[152:155], a[48:51]\n\t" \
    "s_cselect_b32 s41, s44, s43\n\t" \
    "s_add_u32 s39, s66, s41\n\t" \
    "v_mfma_f32_16x16x32_f16 a[48:51], v[24:27], v[148:151], a[48:51]\n\t" \
    "s_add_u32 s39, s39, %[sWk0]\n\t" \
    "v_mfma_f32_16x16x32_f16 a[80:83], v[28:31], v[148:151], a[80:83]\n\t" \
    "buffer_load_dwordx4 v[144:147], %[vA0], %[rWS], s55 offen offset:3072 sc0 sc1\n\t" \
    "v_mfma_f32_16x16x32_f16 a[80:83], v[60:63], v[152:155], a[80:83]\n\t" \
    "buffer_load_dwordx4 v[164:167], %[vW], %[rW], s39 offen\n\t" \
    "v_mfma_f32_16x16x32_f16 a[112:115], v[64:67], v[152:155], a[112:115]\n\t" \
    "s_add_u32 s39, s39, 32768\n\t" \
    "v_mfma_f32_16x16x32_f16 a[112:115], v[32:35], v[148:151], a[112:115]\n\t" \
    "s_nop 15\n\t" \
    "s_nop 3\n\t" \
    "ds_read_b128 v[160:163], %[vA0] offset:13312\n\t" \
    "v_mfma_f32_16x16x32_f16 a[144:147], v[36:39], v[148:151], a[144:147]\n\t" \
    "buffer_load_dwordx4 v[168:171], %[vW], %[rW], s39 offen\n\t" \
    "v_mfma_f32_16x16x32_f16 a[144:147], v[68:71], v[152:155], a[144:147]\n\t" \
    "s_add_u32 s39, s39, 32768\n\t" \
    "v_mfma_f32_16x16x32_f16 a[176:179], v[72:75], v[152:155], a[176:179]\n\t" \
    "buffer_load_dwordx4 v[172:175], %[vW], %[rW], s39 offen\n\t" \
    "v_mfma_f32_16x16x32_f16 a[176:179], v[40:43], v[148:151], a[176:179]\n\t" \
    "v_mfma_f32_16x16x32_f16 a[208:211], v[44:47], v[148:151], a[208:211]\n\t" \
    "s_add_u32 s39, s39, 32768\n\t" \
    "v_mfma_f32_16x16x32_f16 a[208:211], v[76:79], v[152:155], a[208:211]\n\t" \
    "v_mfma_f32_16x16x32_f16 a[240:243], v[80:83], v[152:155], a[240:243]\n\t" \
    "buffer_load_dwordx4 v[176:179], %[vW], %[rW], s39 offen\n\t" \
    "v_mfma_f32_16x16x32_f16 a[240:243], v[48:51], v[148:151], a[240:243]\n\t" \
    "s_nop 15\n\t" \
    "s_nop 3\n\t" \
    "ds_read_b128 v[148:151], %[vA0] offset:6144\n\t" \
    "s_waitcnt lgkmcnt(2)\n\t" \
    "v_mfma_f32_16x16x32_f16 a[244:247], v[48:51], v[156:159], a[244:247]\n\t" \
    "s_add_u32 s39, s39, 32768\n\t" \
    "s_waitcnt lgkmcnt(1)\n\t" \
    "v_mfma_f32_16x16x32_f16 a[244:247], v[80:83], v[160:163], a[244:247]\n\t" \
    "v_mfma_f32_16x16x32_f16 a[212:215], v[76:79], v[160:163], a[212:215]\n\t" \
    "buffer_load_dwordx4 v[180:183], %[vW], %[rW], s39 offen\n\t" \
    "v_mfma_f32_16x16x32_f16 a[212:215], v[44:47], v[156:159], a[212:215]\n\t" \
    "v_mfma_f32_16x16x32_f16 a[180:183], v[40:43], v[156:159], a[180:183]\n\t" \
    "s_add_u32 s39, s39, 32768\n\t" \
    "v_mfma_f32_16x16x32_f16 a[180:183], v[72:75], v[160:163], a[180:183]\n\t" \
    "v_mfma_f32_16x16x32_f16 a[148:151], v[68:71], v[160:163], a[148:151]\n\t" \
    "buffer_load_dwordx4 v[184:187], %[vW], %[rW], s39 offen\n\t" \
    "v_mfma_f32_16x16x32_f16 a[148:151], v[36:39], v[156:159], a[148:151]\n\t" \
    "s_nop 15\n\t" \
    "s_nop 3\n\t" \
    "ds_read_b128 v[152:155], %[vA0] offset:14336\n\t" \
    "v_mfma_f32_16x16x32_f16 a[116:119], v[32:35], v[156:159], a[116:119]\n\t" \
    "s_add_u32 s39, s39, 32768\n\t" \
    "v_mfma_f32_16x16x32_f16 a[116:119], v[64:67], v[160:163], a[116:119]\n\t" \
    "v_mfma_f32_16x16x32_f16 a[84:87], v[60:63], v[160:163], a[84:87]\n\t" \
    "buffer_load_dwordx4 v[188:191], %[vW], %[rW], s39 offen\n\t" \
    "v_mfma_f32_16x16x32_f16 a[84:87], v[28:31], v[156:159], a[84:87]\n\t" \
    "v_mfma_f32_16x16x32_f16 a[52:55], v[24:27], v[156:159], a[52:55]\n\t" \
    "s_add_u32 s39, s39, 32768\n\t" \
    "v_mfma_f32_16x16x32_f16 a[52:55], v[56:59], v[160:163], a[52:55]\n\t" \
    "v_mfma_f32_16x16x32_f16 a[20:23], v[52:55], v[160:163], a[20:23]\n\t" \
    "buffer_load_dwordx4 v[192:195], %[vW], %[rW], s39 offen\n\t" \
    "v_mfma_f32_16x16x32_f16 a[20:23], v[20:23], v[156:159], a[20:23]\n\t" \
    "s_nop 15\n\t" \
    "s_nop 3\n\t" \
    "ds_read_b128 v[156:159], %[vA0] offset:7168\n\t" \
    "s_waitcnt lgkmcnt(2)\n\t" \
    "v_mfma_f32_16x16x32_f16 a[24:27], v[20:23], v[148:151], a[24:27]\n\t" \
    "s_waitcnt vmcnt(30)\n\t" \
    "v_cvt_pk_f16_f32 v228, v196, v200\n\t" \
    "s_waitcnt lgkmcnt(1)\n\t" \
    "v_mfma_f32_16x16x32_f16 a[24:27], v[52:55], v[152:155], a[24:27]\n\t" \
    "v_mfma_f32_16x16x32_f16 a[56:59], v[56:59], v[152:155], a[56:59]\n\t" \
    "s_waitcnt vmcnt(28)\n\t" \
    "v_cvt_pk_f16_f32 v229, v204, v208\n\t" \
    "v_mfma_f32_16x16x32_f16 a[56:59], v[24:27], v[148:151], a[56:59]\n\t" \
    "ds_write_b64 %[vWr1], v[228:229] offset:32768\n\t" \
    "v_mfma_f32_16x16x32_f16 a[88:91], v[28:31], v[148:151], a[88:91]\n\t" \
    "s_waitcnt vmcnt(26)\n\t" \
    "v_cvt_pk_f16_f32 v230, v212, v216\n\t" \
    "v_mfma_f32_16x16x32_f16 a[88:91], v[60:63], v[152:155], a[88:91]\n\t" \
    "v_mfma_f32_16x16x32_f16 a[120:123], v[64:67], v[152:155], a[120:123]\n\t" \
    "s_waitcnt vmcnt(24)\n\t" \
    "v_cvt_pk_f16_f32 v231, v220, v224\n\t" \
    "v_mfma_f32_16x16x32_f16 a[120:123], v[32:35], v[148:151], a[120:123]\n\t" \
    "s_nop 15\n\t" \
    "s_nop 3\n\t" \
    "ds_read_b128 v[160:163], %[vA0] offset:15360\n\t" \
    "v_mfma_f32_16x16x32_f16 a[152:155], v[36:39], v[148:151], a[152:155]\n\t" \
    "ds_write_b64 %[vWr1], v[230:231] offset:32776\n\t" \
    "v_mfma_f32_16x16x32_f16 a[152:155], v[68:71], v[152:155], a[152:155]\n\t" \
    "v_cvt_pk_f16_f32 v228, v197, v201\n\t" \
    "v_mfma_f32_16x16x32_f16 a[184:187], v[72:75], v[152:155], a[184:187]\n\t" \
    "v_cvt_pk_f16_f32 v229, v205, v209\n\t" \
    "v_mfma_f32_16x16x32_f16 a[184:187], v[40:43], v[148:151], a[184:187]\n\t" \
    "ds_write_b64 %[vWr1], v[228:229] offset:33792\n\t" \
    "v_mfma_f32_16x16x32_f16 a[216:219], v[44:47], v[148:151], a[216:219]\n\t" \
    "v_cvt_pk_f16_f32 v230, v213, v217\n\t" \
    "v_mfma_f32_16x16x32_f16 a[216:219], v[76:79], v[152:155], a[216:219]\n\t" \
    "v_mfma_f32_16x16x32_f16 a[248:251], v[80:83], v[152:155], a[248:251]\n\t" \
    "v_cvt_pk_f16_f32 v231, v221, v225\n\t" \
    "v_mfma_f32_16x16x32_f16 a[248:251], v[48:51], v[148:151], a[248:251]\n\t" \
    "s_nop 15\n\t" \
    "s_nop 3\n\t" \
    "ds_read_b128 v[148:151], %[vA0] offset:16384\n\t" \
    "s_waitcnt lgkmcnt(5)\n\t" \
    "v_mfma_f32_16x16x32_f16 a[252:255], v[48:51], v[156:159], a[252:255]\n\t" \
    "ds_write_b64 %[vWr1], v[230:231] offset:33800\n\t" \
    "s_waitcnt lgkmcnt(4)\n\t" \
    "v_mfma_f32_16x16x32_f16 a[252:255], v[80:83], v[160:163], a[252:255]\n\t" \
    "v_cvt_pk_f16_f32 v228, v198, v202\n\t" \
    "v_mfma_f32_16x16x32_f16 a[220:223], v[76:79], v[160:163], a[220:223]\n\t" \
    "v_cvt_pk_f16_f32 v229, v206, v210\n\t" \
    "v_mfma_f32_16x16x32_f16 a[220:223], v[44:47], v[156:159], a[220:223]\n\t" \
    "ds_write_b64 %[vWr1], v[228:229] offset:34816\n\t" \
    "v_mfma_f32_16x16x32_f16 a[188:191], v[40:43], v[156:159], a[188:191]\n\t" \
    "v_cvt_pk_f16_f32 v230, v214, v218\n\t" \
    "v_mfma_f32_16x16x32_f16 a[188:191], v[72:75], v[160:163], a[188:191]\n\t" \
    "v_mfma_f32_16x16x32_f16 a[156:159], v[68:71], v[160:163], a[156:159]\n\t" \
    "v_cvt_pk_f16_f32 v231, v222, v226\n\t" \
    "v_mfma_f32_16x16x32_f16 a[156:159], v[36:39], v[156:159], a[156:159]\n\t" \
    "s_nop 15\n\t" \
    "s_nop 3\n\t" \
    "ds_read_b128 v[152:155], %[vA0] offset:24576\n\t" \
    "v_mfma_f32_16x16x32_f16 a[124:127], v[32:35], v[156:159], a[124:127]\n\t" \
    "ds_write_b64 %[vWr1], v[230:231] offset:34824\n\t" \
    "v_mfma_f32_16x16x32_f16 a[124:127], v[64:67], v[160:163], a[124:127]\n\t" \
    "v_cvt_pk_f16_f32 v228, v199, v203\n\t" \
    "v_mfma_f32_16x16x32_f16 a[92:95], v[60:63], v[160:163], a[92:95]\n\t" \
    "v_cvt_pk_f16_f32 v229, v207, v211\n\t" \
    "v_mfma_f32_16x16x32_f16 a[92:95], v[28:31], v[156:159], a[92:95]\n\t" \
    "ds_write_b64 %[vWr1], v[228:229] offset:35840\n\t" \
    "v_mfma_f32_16x16x32_f16 a[60:63], v[24:27], v[156:159], a[60:63]\n\t" \
    "v_cvt_pk_f16_f32 v230, v215, v219\n\t" \
    "v_mfma_f32_16x16x32_f16 a[60:63], v[56:59], v[160:163], a[60:63]\n\t" \
    "v_mfma_f32_16x16x32_f16 a[28:31], v[52:55], v[160:163], a[28:31]\n\t" \
    "v_cvt_pk_f16_f32 v231, v223, v227\n\t" \
    "v_mfma_f32_16x16x32_f16 a[28:31], v[20:23], v[156:159], a[28:31]\n\t" \
    "s_nop 15\n\t" \
    "s_nop 3\n\t" \
    "ds_read_b128 v[156:159], %[vA0] offset:17408\n\t" \
    "s_waitcnt vmcnt(23) lgkmcnt(6)\n\t" \
    "v_mfma_f32_16x16x32_f16 a[0:3], v[84:87], v[148:151], a[0:3]\n\t" \
    "ds_write_b64 %[vWr1], v[230:231] offset:35848\n\t" \
    "s_waitcnt vmcnt(15) lgkmcnt(4)\n\t" \
    "v_mfma_f32_16x16x32_f16 a[0:3], v[116:119], v[152:155], a[0:3]\n\t" \
    "s_add_u32 s55, s38, 0\n\t" \
    "s_waitcnt vmcnt(14)\n\t" \
    "v_mfma_f32_16x16x32_f16 a[32:35], v[120:123], v[152:155], a[32:35]\n\t" \
    "buffer_load_dwordx4 v[20:23], %[vA0], %[rWS], s55 offen offset:0 sc0 sc1\n\t" \
    "v_mfma_f32_16x16x32_f16 a[32:35], v[88:91], v[148:151], a[32:35]\n\t" \
    "v_mfma_f32_16x16x32_f16 a[64:67], v[92:95], v[148:151], a[64:67]\n\t" \
    "s_waitcnt vmcnt(14)\n\t" \
    "v_mfma_f32_16x16x32_f16 a[64:67], v[124:127], v[152:155], a[64:67]\n\t" \
    "s_waitcnt vmcnt(13)\n\t" \
    "v_mfma_f32_16x16x32_f16 a[96:99], v[128:131], v[152:155], a[96:99]\n\t" \
    "buffer_load_dwordx4 v[24:27], %[vA0], %[rWS], s55 offen offset:1024 sc0 sc1\n\t" \
    "v_mfma_f32_16x16x32_f16 a[96:99], v[96:99], v[148:151], a[96:99]\n\t" \
    "s_nop 15\n\t" \
    "s_nop 3\n\t" \
    "ds_read_b128 v[160:163], %[vA0] offset:25600\n\t" \
    "v_mfma_f32_16x16x32_f16 a[128:131], v[100:103], v[148:151], a[128:131]\n\t" \
    "s_waitcnt vmcnt(13)\n\t" \
    "v_mfma_f32_16x16x32_f16 a[128:131], v[132:135], v[152:155], a[128:131]\n\t" \
    "s_waitcnt vmcnt(12)\n\t" \
    "v_mfma_f32_16x16x32_f16 a[160:163], v[136:139], v[152:155], a[160:163]\n\t" \
    "buffer_load_dwordx4 v[28:31], %[vA0], %[rWS], s55 offen offset:2048 sc0 sc1\n\t" \
    "v_mfma_f32_16x16x32_f16 a[160:163], v[104:107], v[148:151], a[160:163]\n\t" \
    "v_mfma_f32_16x16x32_f16 a[192:195], v[108:111], v[148:151], a[192:195]\n\t" \
    "s_waitcnt vmcnt(12)\n\t" \
    "v_mfma_f32_16x16x32_f16 a[192:195], v[140:143], v[152:155], a[192:195]\n\t" \
    "s_waitcnt vmcnt(11)\n\t" \
    "v_mfma_f32_16x16x32_f16 a[224:227], v[144:147], v[152:155], a[224:227]\n\t" \
    "buffer_load_dwordx4 v[32:35], %[vA0], %[rWS], s55 offen offset:3072 sc0 sc1\n\t" \
    "v_mfma_f32_16x16x32_f16 a[224:227], v[112:115], v[148:151], a[224:227]\n\t" \
    "s_nop 15\n\t" \
    "s_nop 3\n\t" \
    "ds_read_b128 v[148:151], %[vA0] offset:18432\n\t" \
    "s_waitcnt lgkmcnt(3)\n\t" \
    "v_mfma_f32_16x16x32_f16 a[228:231], v[112:115], v[156:159], a[228:231]\n\t" \
    "s_waitcnt lgkmcnt(1)\n\t" \
    "v_mfma_f32_16x16x32_f16 a[228:231], v[144:147], v[160:163], a[228:231]\n\t" \
    "s_add_u32 s55, s38, 4096\n\t" \
    "v_mfma_f32_16x16x32_f16 a[196:199], v[140:143], v[160:163], a[196:199]\n\t" \
    "v_mfma_f32_16x16x32_f16 a[196:199], v[108:111], v[156:159], a[196:199]\n\t" \
    "buffer_load_dwordx4 v[36:39], %[vA0], %[rWS], s55 offen offset:0 sc0 sc1\n\t" \
    "v_mfma_f32_16x16x32_f16 a[164:167], v[104:107], v[156:159], a[164:167]\n\t" \
    "v_mfma_f32_16x16x32_f16 a[164:167], v[136:139], v[160:163], a[164:167]\n\t" \
    "v_mfma_f32_16x16x32_f16 a[132:135], v[132:135], v[160:163], a[132:135]\n\t" \
    "v_mfma_f32_16x16x32_f16 a[132:135], v[100:103], v[156:159], a[132:135]\n\t" \
    "s_nop 15\n\t" \
    "s_nop 3\n\t" \
    "ds_read_b128 v[152:155], %[vA0] offset:26624\n\t" \
    "v_mfma_f32_16x16x32_f16 a[100:103], v[96:99], v[156:159], a[100:103]\n\t" \
    "buffer_load_dwordx4 v[40:43], %[vA0], %[rWS], s55 offen offset:1024 sc0 sc1\n\t" \
    "v_mfma_f32_16x16x32_f16 a[100:103], v[128:131], v[160:163], a[100:103]\n\t" \
    "v_mfma_f32_16x16x32_f16 a[68:71], v[124:127], v[160:163], a[68:71]\n\t" \
    "v_mfma_f32_16x16x32_f16 a[68:71], v[92:95], v[156:159], a[68:71]\n\t" \
    "buffer_load_dwordx4 v[44:47], %[vA0], %[rWS], s55 offen offset:2048 sc0 sc1\n\t" \
    "v_mfma_f32_16x16x32_f16 a[36:39], v[88:91], v[156:159], a[36:39]\n\t" \
    "v_mfma_f32_16x16x32_f16 a[36:39], v[120:123], v[160:163], a[36:39]\n\t" \
    "v_mfma_f32_16x16x32_f16 a[4:7], v[116:119], v[160:163], a[4:7]\n\t" \
    "v_mfma_f32_16x16x32_f16 a[4:7], v[84:87], v[156:159], a[4:7]\n\t" \
    "s_nop 15\n\t" \
    "s_nop 3\n\t" \
    "ds_read_b128 v[156:159], %[vA0] offset:19456\n\t" \
    "s_waitcnt lgkmcnt(2)\n\t" \
    "v_mfma_f32_16x16x32_f16 a[8:11], v[84:87], v[148:151], a[8:11]\n\t" \
    "buffer_load_dwordx4 v[48:51], %[vA0], %[rWS], s55 offen offset:3072 sc0 sc1\n\t" \
    "s_waitcnt lgkmcnt(1)\n\t" \
    "v_mfma_f32_16x16x32_f16 a[8:11], v[116:119], v[152:155], a[8:11]\n\t" \
    "v_mfma_f32_16x16x32_f16 a[40:43], v[120:123], v[152:155], a[40:43]\n\t" \
    "s_add_u32 s55, s38, 8192\n\t" \
    "v_mfma_f32_16x16x32_f16 a[40:43], v[88:91], v[148:151], a[40:43]\n\t" \
    "v_mfma_f32_16x16x32_f16 a[72:75], v[92:95], v[148:151], a[72:75]\n\t" \
    "buffer_load_dwordx4 v[52:55], %[vA0], %[rWS], s55 offen offset:0 sc0 sc1\n\t" \
    "v_mfma_f32_16x16x32_f16 a[72:75], v[124:127], v[152:155], a[72:75]\n\t" \
    "v_mfma_f32_16x16x32_f16 a[104:107], v[128:131], v[152:155], a[104:107]\n\t" \
    "v_mfma_f32_16x16x32_f16 a[104:107], v[96:99], v[148:151], a[104:107]\n\t" \
    "s_nop 15\n\t" \
    "s_nop 3\n\t" \
    "ds_read_b128 v[160:163], %[vA0] offset:27648\n\t" \
    "v_mfma_f32_16x16x32_f16 a[136:139], v[100:103], v[148:151], a[136:139]\n\t" \
    "buffer_load_dwordx4 v[56:59], %[vA0], %[rWS], s55 offen offset:1024 sc0 sc1\n\t" \
    "v_mfma_f32_16x16x32_f16 a[136:139], v[132:135], v[152:155], a[136:139]\n\t" \
    "v_mfma_f32_16x16x32_f16 a[168:171], v[136:139], v[152:155], a[168:171]\n\t" \
    "v_mfma_f32_16x16x32_f16 a[168:171], v[104:107], v[148:151], a[168:171]\n\t" \
    "v_mfma_f32_16x16x32_f16 a[200:203], v[108:111], v[148:151], a[200:203]\n\t" \
    "buffer_load_dwordx4 v[60:63], %[vA0], %[rWS], s55 offen offset:2048 sc0 sc1\n\t" \
    "v_mfma_f32_16x16x32_f16 a[200:203], v[140:143], v[152:155], a[200:203]\n\t" \
    "v_mfma_f32_16x16x32_f16 a[232:235], v[144:147], v[152:155], a[232:235]\n\t" \
    "v_mfma_f32_16x16x32_f16 a[232:235], v[112:115], v[148:151], a[232:235]\n\t" \
    "s_nop 15\n\t" \
    "s_nop 3\n\t" \
    "ds_read_b128 v[148:151], %[vA0] offset:20480\n\t" \
    "s_waitcnt lgkmcnt(2)\n\t" \
    "v_mfma_f32_16x16x32_f16 a[236:239], v[112:115], v[156:159], a[236:239]\n\t" \
    "buffer_load_dwordx4 v[64:67], %[vA0], %[rWS], s55 offen offset:3072 sc0 sc1\n\t" \
    "s_waitcnt lgkmcnt(1)\n\t" \
    "v_mfma_f32_16x16x32_f16 a[236:239], v[144:147], v[160:163], a[236:239]\n\t" \
    "v_mfma_f32_16x16x32_f16 a[204:207], v[140:143], v[160:163], a[204:207]\n\t" \
    "v_mfma_f32_16x16x32_f16 a[204:207], v[108:111], v[156:159], a[204:207]\n\t" \
    "s_add_u32 s55, s38, 12288\n\t" \
    "v_mfma_f32_16x16x32_f16 a[172:175], v[104:107], v[156:159], a[172:175]\n\t" \
    "v_mfma_f32_16x16x32_f16 a[172:175], v[136:139], v[160:163], a[172:175]\n\t" \
    "buffer_load_dwordx4 v[68:71], %[vA0], %[rWS], s55 offen offset:0 sc0 sc1\n\t" \
    "v_mfma_f32_16x16x32_f16 a[140:143], v[132:135], v[160:163], a[140:143]\n\t" \
    "v_mfma_f32_16x16x32_f16 a[140:143], v[100:103], v[156:159], a[140:143]\n\t" \
    "s_nop 15\n\t" \
    "s_nop 3\n\t" \
    "ds_read_b128 v[152:155], %[vA0] offset:28672\n\t" \
    "v_mfma_f32_16x16x32_f16 a[108:111], v[96:99], v[156:159], a[108:111]\n\t" \
    "v_mfma_f32_16x16x32_f16 a[108:111], v[128:131], v[160:163], a[108:111]\n\t" \
    "buffer_load_dwordx4 v[72:75], %[vA0], %[rWS], s55 offen offset:1024 sc0 sc1\n\t" \
    "v_mfma_f32_16x16x32_f16 a[76:79], v[124:127], v[160:163], a[76:79]\n\t" \
    "v_mfma_f32_16x16x32_f16 a[76:79], v[92:95], v[156:159], a[76:79]\n\t" \
    "v_mfma_f32_16x16x32_f16 a[44:47], v[88:91], v[156:159], a[44:47]\n\t" \
    "v_mfma_f32_16x16x32_f16 a[44:47], v[120:123], v[160:163], a[44:47]\n\t" \
    "buffer_load_dwordx4 v[76:79], %[vA0], %[rWS], s55 offen offset:2048 sc0 sc1\n\t" \
    "v_mfma_f32_16x16x32_f16 a[12:15], v[116:119], v[160:163], a[12:15]\n\t" \
    "s_and_b32 s41, s36, 7\n\t" \
    "s_cmp_gt_u32 s41, 5\n\t" \
    "v_mfma_f32_16x16x32_f16 a[12:15], v[84:87], v[156:159], a[12:15]\n\t" \
    "s_nop 15\n\t" \
    "s_nop 3\n\t" \
    "ds_read_b128 v[156:159], %[vA0] offset:21504\n\t" \
    "s_waitcnt lgkmcnt(2)\n\t" \
    "v_mfma_f32_16x16x32_f16 a[16:19], v[84:87], v[148:151], a[16:19]\n\t" \
    "s_cselect_b32 s41, s44, s43\n\t" \
    "s_add_u32 s40, s66, s41\n\t" \
    "s_waitcnt lgkmcnt(1)\n\t" \
    "v_mfma_f32_16x16x32_f16 a[16:19], v[116:119], v[152:155], a[16:19]\n\t" \
    "s_add_u32 s40, s40, %[sWk1]\n\t" \
    "v_mfma_f32_16x16x32_f16 a[48:51], v[120:123], v[152:155], a[48:51]\n\t" \
    "buffer_load_dwordx4 v[80:83], %[vA0], %[rWS], s55 offen offset:3072 sc0 sc1\n\t" \
    "v_mfma_f32_16x16x32_f16 a[48:51], v[88:91], v[148:151], a[48:51]\n\t" \
    "v_mfma_f32_16x16x32_f16 a[80:83], v[92:95], v[148:151], a[80:83]\n\t" \
    "v_mfma_f32_16x16x32_f16 a[80:83], v[124:127], v[152:155], a[80:83]\n\t" \
    "v_mfma_f32_16x16x32_f16 a[112:115], v[128:131], v[152:155], a[112:115]\n\t" \
    "buffer_load_dwordx4 v[196:199], %[vW], %[rW], s40 offen nt\n\t" \
    "v_mfma_f32_16x16x32_f16 a[112:115], v[96:99], v[148:151], a[112:115]\n\t" \
    "s_nop 15\n\t" \
    "s_nop 3\n\t" \
    "ds_read_b128 v[160:163], %[vA0] offset:29696\n\t" \
    "v_mfma_f32_16x16x32_f16 a[144:147], v[100:103], v[148:151], a[144:147]\n\t" \
    "v_mfma_f32_16x16x32_f16 a[144:147], v[132:135], v[152:155], a[144:147]\n\t" \
    "v_mfma_f32_16x16x32_f16 a[176:179], v[136:139], v[152:155], a[176:179]\n\t" \
    "s_add_u32 s40, s40, 32768\n\t" \
    "v_mfma_f32_16x16x32_f16 a[176:179], v[104:107], v[148:151], a[176:179]\n\t" \
    "v_mfma_f32_16x16x32_f16 a[208:211], v[108:111], v[148:151], a[208:211]\n\t" \
    "buffer_load_dwordx4 v[200:203], %[vW], %[rW], s40 offen nt\n\t" \
    "v_mfma_f32_16x16x32_f16 a[208:211], v[140:143], v[152:155], a[208:211]\n\t" \
    "v_mfma_f32_16x16x32_f16 a[240:243], v[144:147], v[152:155], a[240:243]\n\t" \
    "v_mfma_f32_16x16x32_f16 a[240:243], v[112:115], v[148:151], a[240:243]\n\t" \
    "s_nop 15\n\t" \
    "s_nop 3\n\t" \
    "ds_read_b128 v[148:151], %[vA0] offset:22528\n\t" \
    "s_waitcnt lgkmcnt(2)\n\t" \
    "v_mfma_f32_16x16x32_f16 a[244:247], v[112:115], v[156:159], a[244:247]\n\t" \
    "s_add_u32 s40, s40, 32768\n\t" \
    "s_waitcnt lgkmcnt(1)\n\t" \
    "v_mfma_f32_16x16x32_f16 a[244:247], v[144:147], v[160:163], a[244:247]\n\t" \
    "v_mfma_f32_16x16x32_f16 a[212:215], v[140:143], v[160:163], a[212:215]\n\t" \
    "buffer_load_dwordx4 v[204:207], %[vW], %[rW], s40 offen nt\n\t" \
    "v_mfma_f32_16x16x32_f16 a[212:215], v[108:111], v[156:159], a[212:215]\n\t" \
    "v_mfma_f32_16x16x32_f16 a[180:183], v[104:107], v[156:159], a[180:183]\n\t" \
    "v_mfma_f32_16x16x32_f16 a[180:183], v[136:139], v[160:163], a[180:183]\n\t" \
    "v_mfma_f32_16x16x32_f16 a[148:151], v[132:135], v[160:163], a[148:151]\n\t" \
    "s_add_u32 s40, s40, 32768\n\t" \
    "v_mfma_f32_16x16x32_f16 a[148:151], v[100:103], v[156:159], a[148:151]\n\t" \
    "s_nop 15\n\t" \
    "s_nop 3\n\t" \
    "ds_read_b128 v[152:155], %[vA0] offset:30720\n\t" \
    "v_mfma_f32_16x16x32_f16 a[116:119], v[96:99], v[156:159], a[116:119]\n\t" \
    "buffer_load_dwordx4 v[208:211], %[vW], %[rW], s40 offen nt\n\t" \
    "v_mfma_f32_16x16x32_f16 a[116:119], v[128:131], v[160:163], a[116:119]\n\t" \
    "v_mfma_f32_16x16x32_f16 a[84:87], v[124:127], v[160:163], a[84:87]\n\t" \
    "v_mfma_f32_16x16x32_f16 a[84:87], v[92:95], v[156:159], a[84:87]\n\t" \
    "v_mfma_f32_16x16x32_f16 a[52:55], v[88:91], v[156:159], a[52:55]\n\t" \
    "s_add_u32 s40, s40, 32768\n\t" \
    "v_mfma_f32_16x16x32_f16 a[52:55], v[120:123], v[160:163], a[52:55]\n\t" \
    "v_mfma_f32_16x16x32_f16 a[20:23], v[116:119], v[160:163], a[20:23]\n\t" \
    "buffer_load_dwordx4 v[212:215], %[vW], %[rW], s40 offen nt\n\t" \
    "v_mfma_f32_16x16x32_f16 a[20:23], v[84:87], v[156:159], a[20:23]\n\t" \
    "s_nop 15\n\t" \
    "s_nop 3\n\t" \
    "ds_read_b128 v[156:159], %[vA0] offset:23552\n\t" \
    "s_waitcnt lgkmcnt(2)\n\t" \
    "v_mfma_f32_16x16x32_f16 a[24:27], v[84:87], v[148:151], a[24:27]\n\t" \
    "s_waitcnt lgkmcnt(1)\n\t" \
    "v_mfma_f32_16x16x32_f16 a[24:27], v[116:119], v[152:155], a[24:27]\n\t" \
    "v_mfma_f32_16x16x32_f16 a[56:59], v[120:123], v[152:155], a[56:59]\n\t" \
    "s_add_u32 s40, s40, 32768\n\t" \
    "v_mfma_f32_16x16x32_f16 a[56:59], v[88:91], v[148:151], a[56:59]\n\t" \
    "v_mfma_f32_16x16x32_f16 a[88:91], v[92:95], v[148:151], a[88:91]\n\t" \
    "buffer_load_dwordx4 v[216:219], %[vW], %[rW], s40 offen nt\n\t" \
    "v_mfma_f32_16x16x32_f16 a[88:91], v[124:127], v[152:155], a[88:91]\n\t" \
    "v_mfma_f32_16x16x32_f16 a[120:123], v[128:131], v[152:155], a[120:123]\n\t" \
    "v_mfma_f32_16x16x32_f16 a[120:123], v[96:99], v[148:151], a[120:123]\n\t" \
    "s_nop 15\n\t" \
    "s_nop 3\n\t" \
    "ds_read_b128 v[160:163], %[vA0] offset:31744\n\t" \
    "v_mfma_f32_16x16x32_f16 a[152:155], v[100:103], v[148:151], a[152:155]\n\t" \
    "s_add_u32 s40, s40, 32768\n\t" \
    "v_mfma_f32_16x16x32_f16 a[152:155], v[132:135], v[152:155], a[152:155]\n\t" \
    "v_mfma_f32_16x16x32_f16 a[184:187], v[136:139], v[152:155], a[184:187]\n\t" \
    "buffer_load_dwordx4 v[220:223], %[vW], %[rW], s40 offen nt\n\t" \
    "v_mfma_f32_16x16x32_f16 a[184:187], v[104:107], v[148:151], a[184:187]\n\t" \
    "v_mfma_f32_16x16x32_f16 a[216:219], v[108:111], v[148:151], a[216:219]\n\t" \
    "v_mfma_f32_16x16x32_f16 a[216:219], v[140:143], v[152:155], a[216:219]\n\t" \
    "v_mfma_f32_16x16x32_f16 a[248:251], v[144:147], v[152:155], a[248:251]\n\t" \
    "s_add_u32 s40, s40, 32768\n\t" \
    "v_mfma_f32_16x16x32_f16 a[248:251], v[112:115], v[148:151], a[248:251]\n\t" \
    "s_nop 15\n\t" \
    "s_nop 3\n\t" \
    "s_waitcnt lgkmcnt(0)\n\t" \
    "s_barrier\n\t" \
    "ds_read_b128 v[148:151], %[vA0] offset:32768\n\t" \
    "v_mfma_f32_16x16x32_f16 a[252:255], v[112:115], v[156:159], a[252:255]\n\t" \
    "buffer_load_dwordx4 v[224:227], %[vW], %[rW], s40 offen nt\n\t" \
    "v_mfma_f32_16x16x32_f16 a[252:255], v[144:147], v[160:163], a[252:255]\n\t" \
    "v_mfma_f32_16x16x32_f16 a[220:223], v[140:143], v[160:163], a[220:223]\n\t" \
    "v_mfma_f32_16x16x32_f16 a[220:223], v[108:111], v[156:159], a[220:223]\n\t" \
    "v_mfma_f32_16x16x32_f16 a[188:191], v[104:107], v[156:159], a[188:191]\n\t" \
    "v_mfma_f32_16x16x32_f16 a[188:191], v[136:139], v[160:163], a[188:191]\n\t" \
    "v_mfma_f32_16x16x32_f16 a[156:159], v[132:135], v[160:163], a[156:159]\n\t" \
    "v_mfma_f32_16x16x32_f16 a[156:159], v[100:103], v[156:159], a[156:159]\n\t" \
    "s_nop 15\n\t" \
    "s_nop 3\n\t" \
    "ds_read_b128 v[152:155], %[vA0] offset:40960\n\t" \
    "v_mfma_f32_16x16x32_f16 a[124:127], v[96:99], v[156:159], a[124:127]\n\t" \
    "v_mfma_f32_16x16x32_f16 a[124:127], v[128:131], v[160:163], a[124:127]\n\t" \
    "v_mfma_f32_16x16x32_f16 a[92:95], v[124:127], v[160:163], a[92:95]\n\t" \
    "v_mfma_f32_16x16x32_f16 a[92:95], v[92:95], v[156:159], a[92:95]\n\t" \
    "v_mfma_f32_16x16x32_f16 a[60:63], v[88:91], v[156:159], a[60:63]\n\t" \
    "v_mfma_f32_16x16x32_f16 a[60:63], v[120:123], v[160:163], a[60:63]\n\t" \
    "v_mfma_f32_16x16x32_f16 a[28:31], v[116:119], v[160:163], a[28:31]\n\t" \
    "v_mfma_f32_16x16x32_f16 a[28:31], v[84:87], v[156:159], a[28:31]\n\t" \
    "s_nop 15\n\t" \
    "s_nop 3\n\t" \
    "s_add_u32 s36, s36, 1\n\t" \
    "ds_read_b128 v[156:159], %[vA0] offset:33792\n\t" \
    "s_waitcnt vmcnt(23) lgkmcnt(2)\n\t" \
    "v_mfma_f32_16x16x32_f16 a[0:3], v[20:23], v[148:151], a[0:3]\n\t" \
    "s_mov_b32 s37, s38\n\t" \
    "s_add_u32 s55, s37, 16384\n\t" \
    "s_waitcnt vmcnt(15) lgkmcnt(1)\n\t" \
    "v_mfma_f32_16x16x32_f16 a[0:3], v[52:55], v[152:155], a[0:3]\n\t" \
    "s_waitcnt vmcnt(14)\n\t" \
    "v_mfma_f32_16x16x32_f16 a[32:35], v[56:59], v[152:155], a[32:35]\n\t" \
    "buffer_load_dwordx4 v[84:87], %[vA0], %[rWS], s55 offen offset:0 sc0 sc1\n\t" \
    "v_mfma_f32_16x16x32_f16 a[32:35], v[24:27], v[148:151], a[32:35]\n\t" \
    "v_mfma_f32_16x16x32_f16 a[64:67], v[28:31], v[148:151], a[64:67]\n\t" \
    "s_waitcnt vmcnt(14)\n\t" \
    "v_mfma_f32_16x16x32_f16 a[64:67], v[60:63], v[152:155], a[64:67]\n\t" \
    "s_waitcnt vmcnt(13)\n\t" \
    "v_mfma_f32_16x16x32_f16 a[96:99], v[64:67], v[152:155], a[96:99]\n\t" \
    "buffer_load_dwordx4 v[88:91], %[vA0], %[rWS], s55 offen offset:1024 sc0 sc1\n\t" \
    "v_mfma_f32_16x16x32_f16 a[96:99], v[32:35], v[148:151], a[96:99]\n\t" \
    "s_nop 15\n\t" \
    "s_nop 3\n\t" \
    "ds_read_b128 v[160:163], %[vA0] offset:41984\n\t" \
    "v_mfma_f32_16x16x32_f16 a[128:131], v[36:39], v[148:151], a[128:131]\n\t" \
    "s_waitcnt vmcnt(13)\n\t" \
    "v_mfma_f32_16x16x32_f16 a[128:131], v[68:71], v[152:155], a[128:131]\n\t" \
    "s_waitcnt vmcnt(12)\n\t" \
    "v_mfma_f32_16x16x32_f16 a[160:163], v[72:75], v[152:155], a[160:163]\n\t" \
    "buffer_load_dwordx4 v[92:95], %[vA0], %[rWS], s55 offen offset:2048 sc0 sc1\n\t" \
    "v_mfma_f32_16x16x32_f16 a[160:163], v[40:43], v[148:151], a[160:163]\n\t" \
    "v_mfma_f32_16x16x32_f16 a[192:195], v[44:47], v[148:151], a[192:195]\n\t" \
    "s_waitcnt vmcnt(12)\n\t" \
    "v_mfma_f32_16x16x32_f16 a[192:195], v[76:79], v[152:155], a[192:195]\n\t" \
    "s_waitcnt vmcnt(11)\n\t" \
    "v_mfma_f32_16x16x32_f16 a[224:227], v[80:83], v[152:155], a[224:227]\n\t" \
    "buffer_load_dwordx4 v[96:99], %[vA0], %[rWS], s55 offen offset:3072 sc0 sc1\n\t" \
    "v_mfma_f32_16x16x32_f16 a[224:227], v[48:51], v[148:151], a[224:227]\n\t" \
    "s_nop 15\n\t" \
    "s_nop 3\n\t" \
    "ds_read_b128 v[148:151], %[vA0] offset:34816\n\t" \
    "s_waitcnt lgkmcnt(2)\n\t" \
    "v_mfma_f32_16x16x32_f16 a[228:231], v[48:51], v[156:159], a[228:231]\n\t" \
    "s_add_u32 s55, s37, 20480\n\t" \
    "s_waitcnt lgkmcnt(1)\n\t" \
    "v_mfma_f32_16x16x32_f16 a[228:231], v[80:83], v[160:163], a[228:231]\n\t" \
    "v_mfma_f32_16x16x32_f16 a[196:199], v[76:79], v[160:163], a[196:199]\n\t" \
    "buffer_load_dwordx4 v[100:103], %[vA0], %[rWS], s55 offen offset:0 sc0 sc1\n\t" \
    "v_mfma_f32_16x16x32_f16 a[196:199], v[44:47], v[156:159], a[196:199]\n\t" \
    "v_mfma_f32_16x16x32_f16 a[164:167], v[40:43], v[156:159], a[164:167]\n\t" \
    "v_mfma_f32_16x16x32_f16 a[164:167], v[72:75], v[160:163], a[164:167]\n\t" \
    "v_mfma_f32_16x16x32_f16 a[132:135], v[68:71], v[160:163], a[132:135]\n\t" \
    "buffer_load_dwordx4 v[104:107], %[vA0], %[rWS], s55 offen offset:1024 sc0 sc1\n\t" \
    "v_mfma_f32_16x16x32_f16 a[132:135], v[36:39], v[156:159], a[132:135]\n\t" \
    "s_nop 15\n\t" \
    "s_nop 3\n\t" \
    "ds_read_b128 v[152:155], %[vA0] offset:43008\n\t" \
    "v_mfma_f32_16x16x32_f16 a[100:103], v[32:35], v[156:159], a[100:103]\n\t" \
    "v_cvt_pk_f16_f32 v228, v164, v168\n\t" \
    "v_mfma_f32_16x16x32_f16 a[100:103], v[64:67], v[160:163], a[100:103]\n\t" \
    "v_mfma_f32_16x16x32_f16 a[68:71], v[60:63], v[160:163], a[68:71]\n\t" \
    "buffer_load_dwordx4 v[108:111], %[vA0], %[rWS], s55 offen offset:2048 sc0 sc1\n\t" \
    "v_mfma_f32_16x16x32_f16 a[68:71], v[28:31], v[156:159], a[68:71]\n\t" \
    "v_cvt_pk_f16_f32 v229, v172, v176\n\t" \
    "v_mfma_f32_16x16x32_f16 a[36:39], v[24:27], v[156:159], a[36:39]\n\t" \
    "ds_write_b64 %[vWr0], v[228:229] offset:0\n\t" \
    "v_mfma_f32_16x16x32_f16 a[36:39], v[56:59], v[160:163], a[36:39]\n\t" \
    "v_cvt_pk_f16_f32 v230, v180, v184\n\t" \
    "v_mfma_f32_16x16x32_f16 a[4:7], v[52:55], v[160:163], a[4:7]\n\t" \
    "buffer_load_dwordx4 v[112:115], %[vA0], %[rWS], s55 offen offset:3072 sc0 sc1\n\t" \
    "v_mfma_f32_16x16x32_f16 a[4:7], v[20:23], v[156:159], a[4:7]\n\t" \
    "s_nop 15\n\t" \
    "s_nop 3\n\t" \
    "ds_read_b128 v[156:159], %[vA0] offset:35840\n\t" \
    "s_waitcnt lgkmcnt(3)\n\t" \
    "v_mfma_f32_16x16x32_f16 a[8:11], v[20:23], v[148:151], a[8:11]\n\t" \
    "v_cvt_pk_f16_f32 v231, v188, v192\n\t" \
    "s_waitcnt lgkmcnt(2)\n\t" \
    "v_mfma_f32_16x16x32_f16 a[8:11], v[52:55], v[152:155], a[8:11]\n\t" \
    "ds_write_b64 %[vWr0], v[230:231] offset:8\n\t" \
    "v_mfma_f32_16x16x32_f16 a[40:43], v[56:59], v[152:155], a[40:43]\n\t" \
    "s_add_u32 s55, s37, 24576\n\t" \
    "v_cvt_pk_f16_f32 v228, v165, v169\n\t" \
    "v_mfma_f32_16x16x32_f16 a[40:43], v[24:27], v[148:151], a[40:43]\n\t" \
    "buffer_load_dwordx4 v[116:119], %[vA0], %[rWS], s55 offen offset:0 sc0 sc1\n\t" \
    "v_mfma_f32_16x16x32_f16 a[72:75], v[28:31], v[148:151], a[72:75]\n\t" \
    "v_cvt_pk_f16_f32 v229, v173, v177\n\t" \
    "s_add_u32 s38, s38, 32768\n\t" \
    "v_mfma_f32_16x16x32_f16 a[72:75], v[60:63], v[152:155], a[72:75]\n\t" \
    "s_and_b32 s38, s38, 0x1fffff\n\t" \
    "s_or_b32 s38, s38, %[sA0]\n\t" \
    "v_mfma_f32_16x16x32_f16 a[104:107], v[64:67], v[152:155], a[104:107]\n\t" \
    "ds_write_b64 %[vWr0], v[228:229] offset:1024\n\t" \
    "v_mfma_f32_16x16x32_f16 a[104:107], v[32:35], v[148:151], a[104:107]\n\t" \
    "s_nop 15\n\t" \
    "s_nop 3\n\t" \
    "ds_read_b128 v[160:163], %[vA0] offset:44032\n\t" \
    "v_mfma_f32_16x16x32_f16 a[136:139], v[36:39], v[148:151], a[136:139]\n\t" \
    "v_cvt_pk_f16_f32 v230, v181, v185\n\t" \
    "v_mfma_f32_16x16x32_f16 a[136:139], v[68:71], v[152:155], a[136:139]\n\t" \
    "buffer_load_dwordx4 v[120:123], %[vA0], %[rWS], s55 offen offset:1024 sc0 sc1\n\t" \
    "v_mfma_f32_16x16x32_f16 a[168:171], v[72:75], v[152:155], a[168:171]\n\t" \
    "v_cvt_pk_f16_f32 v231, v189, v193\n\t" \
    "v_mfma_f32_16x16x32_f16 a[168:171], v[40:43], v[148:151], a[168:171]\n\t" \
    "ds_write_b64 %[vWr0], v[230:231] offset:1032\n\t" \
    "v_mfma_f32_16x16x32_f16 a[200:203], v[44:47], v[148:151], a[200:203]\n\t" \
    "v_cvt_pk_f16_f32 v228, v166, v170\n\t" \
    "v_mfma_f32_16x16x32_f16 a[200:203], v[76:79], v[152:155], a[200:203]\n\t" \
    "buffer_load_dwordx4 v[124:127], %[vA0], %[rWS], s55 offen offset:2048 sc0 sc1\n\t" \
    "v_mfma_f32_16x16x32_f16 a[232:235], v[80:83], v[152:155], a[232:235]\n\t" \
    "v_cvt_pk_f16_f32 v229, v174, v178\n\t" \
    "v_mfma_f32_16x16x32_f16 a[232:235], v[48:51], v[148:151], a[232:235]\n\t" \
    "s_nop 15\n\t" \
    "s_nop 3\n\t" \
    "ds_read_b128 v[148:151], %[vA0] offset:36864\n\t" \
    "s_waitcnt lgkmcnt(5)\n\t" \
    "v_mfma_f32_16x16x32_f16 a[236:239], v[48:51], v[156:159], a[236:239]\n\t" \
    "ds_write_b64 %[vWr0], v[228:229] offset:2048\n\t" \
    "s_waitcnt lgkmcnt(3)\n\t" \
    "v_mfma_f32_16x16x32_f16 a[236:239], v[80:83], v[160:163], a[236:239]\n\t" \
    "v_cvt_pk_f16_f32 v230, v182, v186\n\t" \
    "v_mfma_f32_16x16x32_f16 a[204:207], v[76:79], v[160:163], a[204:207]\n\t" \
    "buffer_load_dwordx4 v[128:131], %[vA0], %[rWS], s55 offen offset:3072 sc0 sc1\n\t" \
    "v_mfma_f32_16x16x32_f16 a[204:207], v[44:47], v[156:159], a[204:207]\n\t" \
    "v_cvt_pk_f16_f32 v231, v190, v194\n\t" \
    "v_mfma_f32_16x16x32_f16 a[172:175], v[40:43], v[156:159], a[172:175]\n\t" \
    "ds_write_b64 %[vWr0], v[230:231] offset:2056\n\t" \
    "v_mfma_f32_16x16x32_f16 a[172:175], v[72:75], v[160:163], a[172:175]\n\t" \
    "s_add_u32 s55, s37, 28672\n\t" \
    "v_cvt_pk_f16_f32 v228, v167, v171\n\t" \
    "v_mfma_f32_16x16x32_f16 a[140:143], v[68:71], v[160:163], a[140:143]\n\t" \
    "buffer_load_dwordx4 v[132:135], %[vA0], %[rWS], s55 offen offset:0 sc0 sc1\n\t" \
    "v_mfma_f32_16x16x32_f16 a[140:143], v[36:39], v[156:159], a[140:143]\n\t" \
    "s_nop 15\n\t" \
    "s_nop 3\n\t" \
    "ds_read_b128 v[152:155], %[vA0] offset:45056\n\t" \
    "v_mfma_f32_16x16x32_f16 a[108:111], v[32:35], v[156:159], a[108:111]\n\t" \
    "v_cvt_pk_f16_f32 v229, v175, v179\n\t" \
    "v_mfma_f32_16x16x32_f16 a[108:111], v[64:67], v[160:163], a[108:111]\n\t" \
    "ds_write_b64 %[vWr0], v[228:229] offset:3072\n\t" \
    "v_mfma_f32_16x16x32_f16 a[76:79], v[60:63], v[160:163], a[76:79]\n\t" \
    "v_cvt_pk_f16_f32 v230, v183, v187\n\t" \
    "v_mfma_f32_16x16x32_f16 a[76:79], v[28:31], v[156:159], a[76:79]\n\t" \
    "buffer_load_dwordx4 v[136:139], %[vA0], %[rWS], s55 offen offset:1024 sc0 sc1\n\t" \
    "v_mfma_f32_16x16x32_f16 a[44:47], v[24:27], v[156:159], a[44:47]\n\t" \
    "s_add_u32 s66, s66, 0x400000\n\t" \
    "s_and_b32 s66, s66, 0xfffffff\n\t" \
    "v_mfma_f32_16x16x32_f16 a[44:47], v[56:59], v[160:163], a[44:47]\n\t" \
    "v_cvt_pk_f16_f32 v231, v191, v195\n\t" \
    "v_mfma_f32_16x16x32_f16 a[12:15], v[52:55], v[160:163], a[12:15]\n\t" \
    "ds_write_b64 %[vWr0], v[230:231] offset:3080\n\t" \
    "v_mfma_f32_16x16x32_f16 a[12:15], v[20:23], v[156:159], a[12:15]\n\t" \
    "s_nop 15\n\t" \
    "s_nop 3\n\t" \
    "ds_read_b128 v[156:159], %[vA0] offset:37888\n\t" \
    "s_waitcnt lgkmcnt(6)\n\t" \
    "v_mfma_f32_16x16x32_f16 a[16:19], v[20:23], v[148:151], a[16:19]\n\t" \
    "buffer_load_dwordx4 v[140:143], %[vA0], %[rWS], s55 offen offset:2048 sc0 sc1\n\t" \
    "s_waitcnt lgkmcnt(3)\n\t" \
    "v_mfma_f32_16x16x32_f16 a[16:19], v[52:55], v[152:155], a[16:19]\n\t" \
    "s_and_b32 s41, s36, 7\n\t" \
    "s_cmp_gt_u32 s41, 5\n\t" \
    "v_mfma_f32_16x16x32_f16 a[48:51], v[56:59], v[152:155], a[48:51]\n\t" \
    "s_cselect_b32 s41, s44, s43\n\t" \
    "s_add_u32 s39, s66, s41\n\t" \
    "v_mfma_f32_16x16x32_f16 a[48:51], v[24:27], v[148:151], a[48:51]\n\t" \
    "s_add_u32 s39, s39, %[sWk0]\n\t" \
    "v_mfma_f32_16x16x32_f16 a[80:83], v[28:31], v[148:151], a[80:83]\n\t" \
    "buffer_load_dword v19, %[vXf], %[rWS], s34 offen sc0 sc1\n\t" \
    "v_mfma_f32_16x16x32_f16 a[80:83], v[60:63], v[152:155], a[80:83]\n\t" \
    "buffer_load_dwordx4 v[144:147], %[vA0], %[rWS], s55 offen offset:3072 sc0 sc1\n\t" \
    "v_mfma_f32_16x16x32_f16 a[112:115], v[64:67], v[152:155], a[112:115]\n\t" \
    "buffer_load_dwordx4 v[164:167], %[vW], %[rW], s39 offen\n\t" \
    "v_mfma_f32_16x16x32_f16 a[112:115], v[32:35], v[148:151], a[112:115]\n\t" \
    "s_nop 15\n\t" \
    "s_nop 3\n\t" \
    "ds_read_b128 v[160:163], %[vA0] offset:46080\n\t" \
    "v_mfma_f32_16x16x32_f16 a[144:147], v[36:39], v[148:151], a[144:147]\n\t" \
    "s_add_u32 s39, s39, 32768\n\t" \
    "v_mfma_f32_16x16x32_f16 a[144:147], v[68:71], v[152:155], a[144:147]\n\t" \
    "buffer_load_dwordx4 v[168:171], %[vW], %[rW], s39 offen\n\t" \
    "v_mfma_f32_16x16x32_f16 a[176:179], v[72:75], v[152:155], a[176:179]\n\t" \
    "s_add_u32 s39, s39, 32768\n\t" \
    "v_mfma_f32_16x16x32_f16 a[176:179], v[40:43], v[148:151], a[176:179]\n\t" \
    "buffer_load_dwordx4 v[172:175], %[vW], %[rW], s39 offen\n\t" \
    "v_mfma_f32_16x16x32_f16 a[208:211], v[44:47], v[148:151], a[208:211]\n\t" \
    "s_add_u32 s39, s39, 32768\n\t" \
    "v_mfma_f32_16x16x32_f16 a[208:211], v[76:79], v[152:155], a[208:211]\n\t" \
    "v_mfma_f32_16x16x32_f16 a[240:243], v[80:83], v[152:155], a[240:243]\n\t" \
    "buffer_load_dwordx4 v[176:179], %[vW], %[rW], s39 offen\n\t" \
    "v_mfma_f32_16x16x32_f16 a[240:243], v[48:51], v[148:151], a[240:243]\n\t" \
    "s_nop 15\n\t" \
    "s_nop 3\n\t" \
    "ds_read_b128 v[148:151], %[vA0] offset:38912\n\t" \
    "s_waitcnt lgkmcnt(2)\n\t" \
    "v_mfma_f32_16x16x32_f16 a[244:247], v[48:51], v[156:159], a[244:247]\n\t" \
    "s_add_u32 s39, s39, 32768\n\t" \
    "s_waitcnt lgkmcnt(1)\n\t" \
    "v_mfma_f32_16x16x32_f16 a[244:247], v[80:83], v[160:163], a[244:247]\n\t" \
    "v_mfma_f32_16x16x32_f16 a[212:215], v[76:79], v[160:163], a[212:215]\n\t" \
    "buffer_load_dwordx4 v[180:183], %[vW], %[rW], s39 offen\n\t" \
    "v_mfma_f32_16x16x32_f16 a[212:215], v[44:47], v[156:159], a[212:215]\n\t" \
    "v_mfma_f32_16x16x32_f16 a[180:183], v[40:43], v[156:159], a[180:183]\n\t" \
    "s_add_u32 s39, s39, 32768\n\t" \
    "v_mfma_f32_16x16x32_f16 a[180:183], v[72:75], v[160:163], a[180:183]\n\t" \
    "v_mfma_f32_16x16x32_f16 a[148:151], v[68:71], v[160:163], a[148:151]\n\t" \
    "buffer_load_dwordx4 v[184:187], %[vW], %[rW], s39 offen\n\t" \
    "v_mfma_f32_16x16x32_f16 a[148:151], v[36:39], v[156:159], a[148:151]\n\t" \
    "s_nop 15\n\t" \
    "s_nop 3\n\t" \
    "ds_read_b128 v[152:155], %[vA0] offset:47104\n\t" \
    "v_mfma_f32_16x16x32_f16 a[116:119], v[32:35], v[156:159], a[116:119]\n\t" \
    "s_add_u32 s39, s39, 32768\n\t" \
    "v_mfma_f32_16x16x32_f16 a[116:119], v[64:67], v[160:163], a[116:119]\n\t" \
    "v_mfma_f32_16x16x32_f16 a[84:87], v[60:63], v[160:163], a[84:87]\n\t" \
    "buffer_load_dwordx4 v[188:191], %[vW], %[rW], s39 offen\n\t" \
    "v_mfma_f32_16x16x32_f16 a[84:87], v[28:31], v[156:159], a[84:87]\n\t" \
    "v_mfma_f32_16x16x32_f16 a[52:55], v[24:27], v[156:159], a[52:55]\n\t" \
    "s_add_u32 s39, s39, 32768\n\t" \
    "v_mfma_f32_16x16x32_f16 a[52:55], v[56:59], v[160:163], a[52:55]\n\t" \
    "v_mfma_f32_16x16x32_f16 a[20:23], v[52:55], v[160:163], a[20:23]\n\t" \
    "buffer_load_dwordx4 v[192:195], %[vW], %[rW], s39 offen\n\t" \
    "v_mfma_f32_16x16x32_f16 a[20:23], v[20:23], v[156:159], a[20:23]\n\t" \
    "s_nop 15\n\t" \
    "s_nop 3\n\t" \
    "ds_read_b128 v[156:159], %[vA0] offset:39936\n\t" \
    "s_waitcnt lgkmcnt(2)\n\t" \
    "v_mfma_f32_16x16x32_f16 a[24:27], v[20:23], v[148:151], a[24:27]\n\t" \
    "s_waitcnt vmcnt(31)\n\t" \
    "v_cvt_pk_f16_f32 v228, v196, v200\n\t" \
    "s_waitcnt lgkmcnt(1)\n\t" \
    "v_mfma_f32_16x16x32_f16 a[24:27], v[52:55], v[152:155], a[24:27]\n\t" \
    "v_mfma_f32_16x16x32_f16 a[56:59], v[56:59], v[152:155], a[56:59]\n\t" \
    "s_waitcnt vmcnt(29)\n\t" \
    "v_cvt_pk_f16_f32 v229, v204, v208\n\t" \
    "v_mfma_f32_16x16x32_f16 a[56:59], v[24:27], v[148:151], a[56:59]\n\t" \
    "ds_write_b64 %[vWr1], v[228:229] offset:0\n\t" \
    "v_mfma_f32_16x16x32_f16 a[88:91], v[28:31], v[148:151], a[88:91]\n\t" \
    "s_waitcnt vmcnt(27)\n\t" \
    "v_cvt_pk_f16_f32 v230, v212, v216\n\t" \
    "v_mfma_f32_16x16x32_f16 a[88:91], v[60:63], v[152:155], a[88:91]\n\t" \
    "v_mfma_f32_16x16x32_f16 a[120:123], v[64:67], v[152:155], a[120:123]\n\t" \
    "s_waitcnt vmcnt(25)\n\t" \
    "v_cvt_pk_f16_f32 v231, v220, v224\n\t" \
    "v_mfma_f32_16x16x32_f16 a[120:123], v[32:35], v[148:151], a[120:123]\n\t" \
    "s_nop 15\n\t" \
    "s_nop 3\n\t" \
    "ds_read_b128 v[160:163], %[vA0] offset:48128\n\t" \
    "v_mfma_f32_16x16x32_f16 a[152:155], v[36:39], v[148:151], a[152:155]\n\t" \
    "ds_write_b64 %[vWr1], v[230:231] offset:8\n\t" \
    "v_mfma_f32_16x16x32_f16 a[152:155], v[68:71], v[152:155], a[152:155]\n\t" \
    "v_cvt_pk_f16_f32 v228, v197, v201\n\t" \
    "v_mfma_f32_16x16x32_f16 a[184:187], v[72:75], v[152:155], a[184:187]\n\t" \
    "v_cvt_pk_f16_f32 v229, v205, v209\n\t" \
    "v_mfma_f32_16x16x32_f16 a[184:187], v[40:43], v[148:151], a[184:187]\n\t" \
    "ds_write_b64 %[vWr1], v[228:229] offset:1024\n\t" \
    "v_mfma_f32_16x16x32_f16 a[216:219], v[44:47], v[148:151], a[216:219]\n\t" \
    "v_cvt_pk_f16_f32 v230, v213, v217\n\t" \
    "v_mfma_f32_16x16x32_f16 a[216:219], v[76:79], v[152:155], a[216:219]\n\t" \
    "v_mfma_f32_16x16x32_f16 a[248:251], v[80:83], v[152:155], a[248:251]\n\t" \
    "v_cvt_pk_f16_f32 v231, v221, v225\n\t" \
    "v_mfma_f32_16x16x32_f16 a[248:251], v[48:51], v[148:151], a[248:251]\n\t" \
    "s_nop 15\n\t" \
    "s_nop 3\n\t" \
    "ds_read_b128 v[148:151], %[vA0] offset:49152\n\t" \
    "s_waitcnt lgkmcnt(5)\n\t" \
    "v_mfma_f32_16x16x32_f16 a[252:255], v[48:51], v[156:159], a[252:255]\n\t" \
    "ds_write_b64 %[vWr1], v[230:231] offset:1032\n\t" \
    "s_waitcnt lgkmcnt(4)\n\t" \
    "v_mfma_f32_16x16x32_f16 a[252:255], v[80:83], v[160:163], a[252:255]\n\t" \
    "v_cvt_pk_f16_f32 v228, v198, v202\n\t" \
    "v_mfma_f32_16x16x32_f16 a[220:223], v[76:79], v[160:163], a[220:223]\n\t" \
    "v_cvt_pk_f16_f32 v229, v206, v210\n\t" \
    "v_mfma_f32_16x16x32_f16 a[220:223], v[44:47], v[156:159], a[220:223]\n\t" \
    "s_waitcnt vmcnt(9)\n\t" \
    "v_cmp_le_u32 s[76:77], s70, v19\n\t" \
    "v_mfma_f32_16x16x32_f16 a[188:191], v[40:43], v[156:159], a[188:191]\n\t" \
    "s_nop 1\n\t" \
    "s_cmp_eq_u64 s[76:77], exec\n\t" \
    "s_cbranch_scc0 SLOW_PQ0%=\n\t" \
    "BACK_PQ0%=:\n\t" \
    "v_mfma_f32_16x16x32_f16 a[188:191], v[72:75], v[160:163], a[188:191]\n\t" \
    "ds_write_b64 %[vWr1], v[228:229] offset:2048\n\t" \
    "v_mfma_f32_16x16x32_f16 a[156:159], v[68:71], v[160:163], a[156:159]\n\t" \
    "v_cvt_pk_f16_f32 v230, v214, v218\n\t" \
    "v_cvt_pk_f16_f32 v231, v222, v226\n\t" \
    "v_mfma_f32_16x16x32_f16 a[156:159], v[36:39], v[156:159], a[156:159]\n\t" \
    "s_nop 15\n\t" \
    "s_nop 3\n\t" \
    "ds_read_b128 v[152:155], %[vA0] offset:57344\n\t" \
    "v_mfma_f32_16x16x32_f16 a[124:127], v[32:35], v[156:159], a[124:127]\n\t" \
    "ds_write_b64 %[vWr1], v[230:231] offset:2056\n\t" \
    "v_mfma_f32_16x16x32_f16 a[124:127], v[64:67], v[160:163], a[124:127]\n\t" \
    "v_cvt_pk_f16_f32 v228, v199, v203\n\t" \
    "v_mfma_f32_16x16x32_f16 a[92:95], v[60:63], v[160:163], a[92:95]\n\t" \
    "v_cvt_pk_f16_f32 v229, v207, v211\n\t" \
    "v_mfma_f32_16x16x32_f16 a[92:95], v[28:31], v[156:159], a[92:95]\n\t" \
    "ds_write_b64 %[vWr1], v[228:229] offset:3072\n\t" \
    "v_mfma_f32_16x16x32_f16 a[60:63], v[24:27], v[156:159], a[60:63]\n\t" \
    "v_cvt_pk_f16_f32 v230, v215, v219\n\t" \
    "v_mfma_f32_16x16x32_f16 a[60:63], v[56:59], v[160:163], a[60:63]\n\t" \
    "v_mfma_f32_16x16x32_f16 a[28:31], v[52:55], v[160:163], a[28:31]\n\t" \
    "v_cvt_pk_f16_f32 v231, v223, v227\n\t" \
    "v_mfma_f32_16x16x32_f16 a[28:31], v[20:23], v[156:159], a[28:31]\n\t" \
    "s_nop 15\n\t" \
    "s_nop 3\n\t" \
    "ds_read_b128 v[156:159], %[vA0] offset:50176\n\t" \
    "s_waitcnt lgkmcnt(6)\n\t" \
    "v_mfma_f32_16x16x32_f16 a[0:3], v[84:87], v[148:151], a[0:3]\n\t" \
    "ds_write_b64 %[vWr1], v[230:231] offset:3080\n\t" \
    "s_waitcnt lgkmcnt(4)\n\t" \
    "v_mfma_f32_16x16x32_f16 a[0:3], v[116:119], v[152:155], a[0:3]\n\t" \
    "s_add_u32 s55, s38, 0\n\t" \
    "v_mfma_f32_16x16x32_f16 a[32:35], v[120:123], v[152:155], a[32:35]\n\t" \
    "buffer_load_dwordx4 v[20:23], %[vA0], %[rWS], s55 offen offset:0 sc0 sc1\n\t" \
    "v_mfma_f32_16x16x32_f16 a[32:35], v[88:91], v[148:151], a[32:35]\n\t" \
    "v_mfma_f32_16x16x32_f16 a[64:67], v[92:95], v[148:151], a[64:67]\n\t" \
    "v_mfma_f32_16x16x32_f16 a[64:67], v[124:127], v[152:155], a[64:67]\n\t" \
    "v_mfma_f32_16x16x32_f16 a[96:99], v[128:131], v[152:155], a[96:99]\n\t" \
    "buffer_load_dwordx4 v[24:27], %[vA0], %[rWS], s55 offen offset:1024 sc0 sc1\n\t" \
    "v_mfma_f32_16x16x32_f16 a[96:99], v[96:99], v[148:151], a[96:99]\n\t" \
    "s_nop 15\n\t" \
    "s_nop 3\n\t" \
    "ds_read_b128 v[160:163], %[vA0] offset:58368\n\t" \
    "v_mfma_f32_16x16x32_f16 a[128:131], v[100:103], v[148:151], a[128:131]\n\t" \
    "v_mfma_f32_16x16x32_f16 a[128:131], v[132:135], v[152:155], a[128:131]\n\t" \
    "v_mfma_f32_16x16x32_f16 a[160:163], v[136:139], v[152:155], a[160:163]\n\t" \
    "buffer_load_dwordx4 v[28:31], %[vA0], %[rWS], s55 offen offset:2048 sc0 sc1\n\t" \
    "v_mfma_f32_16x16x32_f16 a[160:163], v[104:107], v[148:151], a[160:163]\n\t" \
    "v_mfma_f32_16x16x32_f16 a[192:195], v[108:111], v[148:151], a[192:195]\n\t" \
    "v_mfma_f32_16x16x32_f16 a[192:195], v[140:143], v[152:155], a[192:195]\n\t" \
    "s_waitcnt vmcnt(11)\n\t" \
    "v_mfma_f32_16x16x32_f16 a[224:227], v[144:147], v[152:155], a[224:227]\n\t" \
    "buffer_load_dwordx4 v[32:35], %[vA0], %[rWS], s55 offen offset:3072 sc0 sc1\n\t" \
    "v_mfma_f32_16x16x32_f16 a[224:227], v[112:115], v[148:151], a[224:227]\n\t" \
    "s_nop 15\n\t" \
    "s_nop 3\n\t" \
    "ds_read_b128 v[148:151], %[vA0] offset:51200\n\t" \
    "s_waitcnt lgkmcnt(3)\n\t" \
    "v_mfma_f32_16x16x32_f16 a[228:231], v[112:115], v[156:159], a[228:231]\n\t" \
    "s_waitcnt lgkmcnt(1)\n\t" \
    "v_mfma_f32_16x16x32_f16 a[228:231], v[144:147], v[160:163], a[228:231]\n\t" \
    "s_add_u32 s55, s38, 4096\n\t" \
    "v_mfma_f32_16x16x32_f16 a[196:199], v[140:143], v[160:163], a[196:199]\n\t" \
    "v_mfma_f32_16x16x32_f16 a[196:199], v[108:111], v[156:159], a[196:199]\n\t" \
    "buffer_load_dwordx4 v[36:39], %[vA0], %[rWS], s55 offen offset:0 sc0 sc1\n\t" \
    "v_mfma_f32_16x16x32_f16 a[164:167], v[104:107], v[156:159], a[164:167]\n\t" \
    "v_mfma_f32_16x16x32_f16 a[164:167], v[136:139], v[160:163], a[164:167]\n\t" \
    "v_mfma_f32_16x16x32_f16 a[132:135], v[132:135], v[160:163], a[132:135]\n\t" \
    "v_mfma_f32_16x16x32_f16 a[132:135], v[100:103], v[156:159], a[132:135]\n\t" \
    "s_nop 15\n\t" \
    "s_nop 3\n\t" \
    "ds_read_b128 v[152:155], %[vA0] offset:59392\n\t" \
    "v_mfma_f32_16x16x32_f16 a[100:103], v[96:99], v[156:159], a[100:103]\n\t" \
    "buffer_load_dwordx4 v[40:43], %[vA0], %[rWS], s55 offen offset:1024 sc0 sc1\n\t" \
    "v_mfma_f32_16x16x32_f16 a[100:103], v[128:131], v[160:163], a[100:103]\n\t" \
    "v_mfma_f32_16x16x32_f16 a[68:71], v[124:127], v[160:163], a[68:71]\n\t" \
    "v_mfma_f32_16x16x32_f16 a[68:71], v[92:95], v[156:159], a[68:71]\n\t" \
    "buffer_load_dwordx4 v[44:47], %[vA0], %[rWS], s55 offen offset:2048 sc0 sc1\n\t" \
    "v_mfma_f32_16x16x32_f16 a[36:39], v[88:91], v[156:159], a[36:39]\n\t" \
    "v_mfma_f32_16x16x32_f16 a[36:39], v[120:123], v[160:163], a[36:39]\n\t" \
    "v_mfma_f32_16x16x32_f16 a[4:7], v[116:119], v[160:163], a[4:7]\n\t" \
    "v_mfma_f32_16x16x32_f16 a[4:7], v[84:87], v[156:159], a[4:7]\n\t" \
    "s_nop 15\n\t" \
    "s_nop 3\n\t" \
    "ds_read_b128 v[156:159], %[vA0] offset:52224\n\t" \
    "s_waitcnt lgkmcnt(2)\n\t" \
    "v_mfma_f32_16x16x32_f16 a[8:11], v[84:87], v[148:151], a[8:11]\n\t" \
    "buffer_load_dwordx4 v[48:51], %[vA0], %[rWS], s55 offen offset:3072 sc0 sc1\n\t" \
    "s_waitcnt lgkmcnt(1)\n\t" \
    "v_mfma_f32_16x16x32_f16 a[8:11], v[116:119], v[152:155], a[8:11]\n\t" \
    "v_mfma_f32_16x16x32_f16 a[40:43], v[120:123], v[152:155], a[40:43]\n\t" \
    "s_add_u32 s55, s38, 8192\n\t" \
    "v_mfma_f32_16x16x32_f16 a[40:43], v[88:91], v[148:151], a[40:43]\n\t" \
    "v_mfma_f32_16x16x32_f16 a[72:75], v[92:95], v[148:151], a[72:75]\n\t" \
    "buffer_load_dwordx4 v[52:55], %[vA0], %[rWS], s55 offen offset:0 sc0 sc1\n\t" \
    "v_mfma_f32_16x16x32_f16 a[72:75], v[124:127], v[152:155], a[72:75]\n\t" \
    "v_mfma_f32_16x16x32_f16 a[104:107], v[128:131], v[152:155], a[104:107]\n\t" \
    "v_mfma_f32_16x16x32_f16 a[104:107], v[96:99], v[148:151], a[104:107]\n\t" \
    "s_nop 15\n\t" \
    "s_nop 3\n\t" \
    "ds_read_b128 v[160:163], %[vA0] offset:60416\n\t" \
    "v_mfma_f32_16x16x32_f16 a[136:139], v[100:103], v[148:151], a[136:139]\n\t" \
    "buffer_load_dwordx4 v[56:59], %[vA0], %[rWS], s55 offen offset:1024 sc0 sc1\n\t" \
    "v_mfma_f32_16x16x32_f16 a[136:139], v[132:135], v[152:155], a[136:139]\n\t" \
    "v_mfma_f32_16x16x32_f16 a[168:171], v[136:139], v[152:155], a[168:171]\n\t" \
    "v_mfma_f32_16x16x32_f16 a[168:171], v[104:107], v[148:151], a[168:171]\n\t" \
    "v_mfma_f32_16x16x32_f16 a[200:203], v[108:111], v[148:151], a[200:203]\n\t" \
    "buffer_load_dwordx4 v[60:63], %[vA0], %[rWS], s55 offen offset:2048 sc0 sc1\n\t" \
    "v_mfma_f32_16x16x32_f16 a[200:203], v[140:143], v[152:155], a[200:203]\n\t" \
    "v_mfma_f32_16x16x32_f16 a[232:235], v[144:147], v[152:155], a[232:235]\n\t" \
    "v_mfma_f32_16x16x32_f16 a[232:235], v[112:115], v[148:151], a[232:235]\n\t" \
    "s_nop 15\n\t" \
    "s_nop 3\n\t" \
    "ds_read_b128 v[148:151], %[vA0] offset:53248\n\t" \
    "s_waitcnt lgkmcnt(2)\n\t" \
    "v_mfma_f32_16x16x32_f16 a[236:239], v[112:115], v[156:159], a[236:239]\n\t" \
    "buffer_load_dwordx4 v[64:67], %[vA0], %[rWS], s55 offen offset:3072 sc0 sc1\n\t" \
    "s_waitcnt lgkmcnt(1)\n\t" \
    "v_mfma_f32_16x16x32_f16 a[236:239], v[144:147], v[160:163], a[236:239]\n\t" \
    "v_mfma_f32_16x16x32_f16 a[204:207], v[140:143], v[160:163], a[204:207]\n\t" \
    "v_mfma_f32_16x16x32_f16 a[204:207], v[108:111], v[156:159], a[204:207]\n\t" \
    "s_add_u32 s55, s38, 12288\n\t" \
    "v_mfma_f32_16x16x32_f16 a[172:175], v[104:107], v[156:159], a[172:175]\n\t" \
    "v_mfma_f32_16x16x32_f16 a[172:175], v[136:139], v[160:163], a[172:175]\n\t" \
    "buffer_load_dwordx4 v[68:71], %[vA0], %[rWS], s55 offen offset:0 sc0 sc1\n\t" \
    "v_mfma_f32_16x16x32_f16 a[140:143], v[132:135], v[160:163], a[140:143]\n\t" \
    "v_mfma_f32_16x16x32_f16 a[140:143], v[100:103], v[156:159], a[140:143]\n\t" \
    "s_nop 15\n\t" \
    "s_nop 3\n\t" \
    "ds_read_b128 v[152:155], %[vA0] offset:61440\n\t" \
    "v_mfma_f32_16x16x32_f16 a[108:111], v[96:99], v[156:159], a[108:111]\n\t" \
    "v_mfma_f32_16x16x32_f16 a[108:111], v[128:131], v[160:163], a[108:111]\n\t" \
    "buffer_load_dwordx4 v[72:75], %[vA0], %[rWS], s55 offen offset:1024 sc0 sc1\n\t" \
    "v_mfma_f32_16x16x32_f16 a[76:79], v[124:127], v[160:163], a[76:79]\n\t" \
    "v_mfma_f32_16x16x32_f16 a[76:79], v[92:95], v[156:159], a[76:79]\n\t" \
    "v_mfma_f32_16x16x32_f16 a[44:47], v[88:91], v[156:159], a[44:47]\n\t" \
    "v_mfma_f32_16x16x32_f16 a[44:47], v[120:123], v[160:163], a[44:47]\n\t" \
    "buffer_load_dwordx4 v[76:79], %[vA0], %[rWS], s55 offen offset:2048 sc0 sc1\n\t" \
    "v_mfma_f32_16x16x32_f16 a[12:15], v[116:119], v[160:163], a[12:15]\n\t" \
    "s_and_b32 s41, s36, 7\n\t" \
    "s_cmp_gt_u32 s41, 5\n\t" \
    "v_mfma_f32_16x16x32_f16 a[12:15], v[84:87], v[156:159], a[12:15]\n\t" \
    "s_nop 15\n\t" \
    "s_nop 3\n\t" \
    "ds_read_b128 v[156:159], %[vA0] offset:54272\n\t" \
    "s_waitcnt lgkmcnt(2)\n\t" \
    "v_mfma_f32_16x16x32_f16 a[16:19], v[84:87], v[148:151], a[16:19]\n\t" \
    "s_cselect_b32 s41, s44, s43\n\t" \
    "s_add_u32 s40, s66, s41\n\t" \
    "s_waitcnt lgkmcnt(1)\n\t" \
    "v_mfma_f32_16x16x32_f16 a[16:19], v[116:119], v[152:155], a[16:19]\n\t" \
    "s_add_u32 s40, s40, %[sWk1]\n\t" \
    "v_mfma_f32_16x16x32_f16 a[48:51], v[120:123], v[152:155], a[48:51]\n\t" \
    "buffer_load_dwordx4 v[80:83], %[vA0], %[rWS], s55 offen offset:3072 sc0 sc1\n\t" \
    "v_mfma_f32_16x16x32_f16 a[48:51], v[88:91], v[148:151], a[48:51]\n\t" \
    "v_mfma_f32_16x16x32_f16 a[80:83], v[92:95], v[148:151], a[80:83]\n\t" \
    "v_mfma_f32_16x16x32_f16 a[80:83], v[124:127], v[152:155], a[80:83]\n\t" \
    "v_mfma_f32_16x16x32_f16 a[112:115], v[128:131], v[152:155], a[112:115]\n\t" \
    "buffer_load_dwordx4 v[196:199], %[vW], %[rW], s40 offen nt\n\t" \
    "v_mfma_f32_16x16x32_f16 a[112:115], v[96:99], v[148:151], a[112:115]\n\t" \
    "s_nop 15\n\t" \
    "s_nop 3\n\t" \
    "ds_read_b128 v[160:163], %[vA0] offset:62464\n\t" \
    "v_mfma_f32_16x16x32_f16 a[144:147], v[100:103], v[148:151], a[144:147]\n\t" \
    "v_mfma_f32_16x16x32_f16 a[144:147], v[132:135], v[152:155], a[144:147]\n\t" \
    "v_mfma_f32_16x16x32_f16 a[176:179], v[136:139], v[152:155], a[176:179]\n\t" \
    "s_add_u32 s40, s40, 32768\n\t" \
    "v_mfma_f32_16x16x32_f16 a[176:179], v[104:107], v[148:151], a[176:179]\n\t" \
    "v_mfma_f32_16x16x32_f16 a[208:211], v[108:111], v[148:151], a[208:211]\n\t" \
    "buffer_load_dwordx4 v[200:203], %[vW], %[rW], s40 offen nt\n\t" \
    "v_mfma_f32_16x16x32_f16 a[208:211], v[140:143], v[152:155], a[208:211]\n\t" \
    "v_mfma_f32_16x16x32_f16 a[240:243], v[144:147], v[152:155], a[240:243]\n\t" \
    "v_mfma_f32_16x16x32_f16 a[240:243], v[112:115], v[148:151], a[240:243]\n\t" \
    "s_nop 15\n\t" \
    "s_nop 3\n\t" \
    "ds_read_b128 v[148:151], %[vA0] offset:55296\n\t" \
    "s_waitcnt lgkmcnt(2)\n\t" \
    "v_mfma_f32_16x16x32_f16 a[244:247], v[112:115], v[156:159], a[244:247]\n\t" \
    "s_add_u32 s40, s40, 32768\n\t" \
    "s_waitcnt lgkmcnt(1)\n\t" \
    "v_mfma_f32_16x16x32_f16 a[244:247], v[144:147], v[160:163], a[244:247]\n\t" \
    "v_mfma_f32_16x16x32_f16 a[212:215], v[140:143], v[160:163], a[212:215]\n\t" \
    "buffer_load_dwordx4 v[204:207], %[vW], %[rW], s40 offen nt\n\t" \
    "v_mfma_f32_16x16x32_f16 a[212:215], v[108:111], v[156:159], a[212:215]\n\t" \
    "v_mfma_f32_16x16x32_f16 a[180:183], v[104:107], v[156:159], a[180:183]\n\t" \
    "v_mfma_f32_16x16x32_f16 a[180:183], v[136:139], v[160:163], a[180:183]\n\t" \
    "v_mfma_f32_16x16x32_f16 a[148:151], v[132:135], v[160:163], a[148:151]\n\t" \
    "s_add_u32 s40, s40, 32768\n\t" \
    "v_mfma_f32_16x16x32_f16 a[148:151], v[100:103], v[156:159], a[148:151]\n\t" \
    "s_nop 15\n\t" \
    "s_nop 3\n\t" \
    "ds_read_b128 v[152:155], %[vA0] offset:63488\n\t" \
    "v_mfma_f32_16x16x32_f16 a[116:119], v[96:99], v[156:159], a[116:119]\n\t" \
    "buffer_load_dwordx4 v[208:211], %[vW], %[rW], s40 offen nt\n\t" \
    "v_mfma_f32_16x16x32_f16 a[116:119], v[128:131], v[160:163], a[116:119]\n\t" \
    "v_mfma_f32_16x16x32_f16 a[84:87], v[124:127], v[160:163], a[84:87]\n\t" \
    "v_mfma_f32_16x16x32_f16 a[84:87], v[92:95], v[156:159], a[84:87]\n\t" \
    "v_mfma_f32_16x16x32_f16 a[52:55], v[88:91], v[156:159], a[52:55]\n\t" \
    "s_add_u32 s40, s40, 32768\n\t" \
    "v_mfma_f32_16x16x32_f16 a[52:55], v[120:123], v[160:163], a[52:55]\n\t" \
    "v_mfma_f32_16x16x32_f16 a[20:23], v[116:119], v[160:163], a[20:23]\n\t" \
    "buffer_load_dwordx4 v[212:215], %[vW], %[rW], s40 offen nt\n\t" \
    "v_mfma_f32_16x16x32_f16 a[20:23], v[84:87], v[156:159], a[20:23]\n\t" \
    "s_nop 15\n\t" \
    "s_nop 3\n\t" \
    "ds_read_b128 v[156:159], %[vA0] offset:56320\n\t" \
    "s_waitcnt lgkmcnt(2)\n\t" \
    "v_mfma_f32_16x16x32_f16 a[24:27], v[84:87], v[148:151], a[24:27]\n\t" \
    "s_waitcnt lgkmcnt(1)\n\t" \
    "v_mfma_f32_16x16x32_f16 a[24:27], v[116:119], v[152:155], a[24:27]\n\t" \
    "v_mfma_f32_16x16x32_f16 a[56:59], v[120:123], v[152:155], a[56:59]\n\t" \
    "s_add_u32 s40, s40, 32768\n\t" \
    "v_mfma_f32_16x16x32_f16 a[56:59], v[88:91], v[148:151], a[56:59]\n\t" \
    "v_mfma_f32_16x16x32_f16 a[88:91], v[92:95], v[148:151], a[88:91]\n\t" \
    "buffer_load_dwordx4 v[216:219], %[vW], %[rW], s40 offen nt\n\t" \
    "v_mfma_f32_16x16x32_f16 a[88:91], v[124:127], v[152:155], a[88:91]\n\t" \
    "v_mfma_f32_16x16x32_f16 a[120:123], v[128:131], v[152:155], a[120:123]\n\t" \
    "v_mfma_f32_16x16x32_f16 a[120:123], v[96:99], v[148:151], a[120:123]\n\t" \
    "s_nop 15\n\t" \
    "s_nop 3\n\t" \
    "ds_read_b128 v[160:163], %[vA0] offset:64512\n\t" \
    "v_mfma_f32_16x16x32_f16 a[152:155], v[100:103], v[148:151], a[152:155]\n\t" \
    "s_add_u32 s40, s40, 32768\n\t" \
    "v_mfma_f32_16x16x32_f16 a[152:155], v[132:135], v[152:155], a[152:155]\n\t" \
    "v_mfma_f32_16x16x32_f16 a[184:187], v[136:139], v[152:155], a[184:187]\n\t" \
    "buffer_load_dwordx4 v[220:223], %[vW], %[rW], s40 offen nt\n\t" \
    "v_mfma_f32_16x16x32_f16 a[184:187], v[104:107], v[148:151], a[184:187]\n\t" \
    "v_mfma_f32_16x16x32_f16 a[216:219], v[108:111], v[148:151], a[216:219]\n\t" \
    "v_mfma_f32_16x16x32_f16 a[216:219], v[140:143], v[152:155], a[216:219]\n\t" \
    "v_mfma_f32_16x16x32_f16 a[248:251], v[144:147], v[152:155], a[248:251]\n\t" \
    "s_add_u32 s40, s40, 32768\n\t" \
    "v_mfma_f32_16x16x32_f16 a[248:251], v[112:115], v[148:151], a[248:251]\n\t" \
    "s_nop 15\n\t" \
    "s_nop 3\n\t" \
    "s_waitcnt lgkmcnt(0)\n\t" \
    "s_barrier\n\t" \
    "ds_read_b128 v[148:151], %[vA0] offset:0\n\t" \
    "v_mfma_f32_16x16x32_f16 a[252:255], v[112:115], v[156:159], a[252:255]\n\t" \
    "buffer_load_dwordx4 v[224:227], %[vW], %[rW], s40 offen nt\n\t" \
    "v_mfma_f32_16x16x32_f16 a[252:255], v[144:147], v[160:163], a[252:255]\n\t" \
    "v_mfma_f32_16x16x32_f16 a[220:223], v[140:143], v[160:163], a[220:223]\n\t" \
    "v_mfma_f32_16x16x32_f16 a[220:223], v[108:111], v[156:159], a[220:223]\n\t" \
    "v_mfma_f32_16x16x32_f16 a[188:191], v[104:107], v[156:159], a[188:191]\n\t" \
    "v_mfma_f32_16x16x32_f16 a[188:191], v[136:139], v[160:163], a[188:191]\n\t" \
    "v_mfma_f32_16x16x32_f16 a[156:159], v[132:135], v[160:163], a[156:159]\n\t" \
    "v_mfma_f32_16x16x32_f16 a[156:159], v[100:103], v[156:159], a[156:159]\n\t" \
    "s_nop 15\n\t" \
    "s_nop 3\n\t" \
    "ds_read_b128 v[152:155], %[vA0] offset:8192\n\t" \
    "v_mfma_f32_16x16x32_f16 a[124:127], v[96:99], v[156:159], a[124:127]\n\t" \
    "v_mfma_f32_16x16x32_f16 a[124:127], v[128:131], v[160:163], a[124:127]\n\t" \
    "v_mfma_f32_16x16x32_f16 a[92:95], v[124:127], v[160:163], a[92:95]\n\t" \
    "v_mfma_f32_16x16x32_f16 a[92:95], v[92:95], v[156:159], a[92:95]\n\t" \
    "v_mfma_f32_16x16x32_f16 a[60:63], v[88:91], v[156:159], a[60:63]\n\t" \
    "v_mfma_f32_16x16x32_f16 a[60:63], v[120:123], v[160:163], a[60:63]\n\t" \
    "v_mfma_f32_16x16x32_f16 a[28:31], v[116:119], v[160:163], a[28:31]\n\t" \
    "v_mfma_f32_16x16x32_f16 a[28:31], v[84:87], v[156:159], a[28:31]\n\t" \
    "s_nop 15\n\t" \
    "s_nop 3\n\t" \
    "s_add_u32 s36, s36, 1\n\t" \
    "s_add_u32 s34, s34, 8192\n\t" \
    "s_min_u32 s34, s34, 24576\n\t" \
    "s_and_b32 s41, s36, 7\n\t" \
    "s_cbranch_scc0 Q0END%=\n\t" \
    "s_bitcmp1_b32 s67, s41\n\t" \
    "s_cbranch_scc0 LOOP0%=\n\t" \
    "LOOP1%=:\n\t" \
    "ds_read_b128 v[156:159], %[vA0] offset:1024\n\t" \
    "s_waitcnt vmcnt(23) lgkmcnt(2)\n\t" \
    "v_mfma_f32_16x16x32_f16 a[0:3], v[20:23], v[148:151], a[0:3]\n\t" \
    "s_mov_b32 s37, s38\n\t" \
    "s_add_u32 s55, s37, 16384\n\t" \
    "s_waitcnt vmcnt(15) lgkmcnt(1)\n\t" \
    "v_mfma_f32_16x16x32_f16 a[0:3], v[52:55], v[152:155], a[0:3]\n\t" \
    "s_waitcnt vmcnt(14)\n\t" \
    "v_mfma_f32_16x16x32_f16 a[32:35], v[56:59], v[152:155], a[32:35]\n\t" \
    "buffer_load_dwordx4 v[84:87], %[vA0], %[rWS], s55 offen offset:0 sc0 sc1\n\t" \
    "v_mfma_f32_16x16x32_f16 a[32:35], v[24:27], v[148:151], a[32:35]\n\t" \
    "v_mfma_f32_16x16x32_f16 a[64:67], v[28:31], v[148:151], a[64:67]\n\t" \
    "s_waitcnt vmcnt(14)\n\t" \
    "v_mfma_f32_16x16x32_f16 a[64:67], v[60:63], v[152:155], a[64:67]\n\t" \
    "s_waitcnt vmcnt(13)\n\t" \
    "v_mfma_f32_16x16x32_f16 a[96:99], v[64:67], v[152:155], a[96:99]\n\t" \
    "buffer_load_dwordx4 v[88:91], %[vA0], %[rWS], s55 offen offset:1024 sc0 sc1\n\t" \
    "v_mfma_f32_16x16x32_f16 a[96:99], v[32:35], v[148:151], a[96:99]\n\t" \
    "s_nop 9\n\t" \
    "ds_read_b128 v[160:163], %[vA0] offset:9216\n\t" \
    "v_mfma_f32_16x16x32_f16 a[128:131], v[36:39], v[148:151], a[128:131]\n\t" \
    "s_waitcnt vmcnt(13)\n\t" \
    "v_mfma_f32_16x16x32_f16 a[128:131], v[68:71], v[152:155], a[128:131]\n\t" \
    "s_waitcnt vmcnt(12)\n\t" \
    "v_mfma_f32_16x16x32_f16 a[160:163], v[72:75], v[152:155], a[160:163]\n\t" \
    "buffer_load_dwordx4 v[92:95], %[vA0], %[rWS], s55 offen offset:2048 sc0 sc1\n\t" \
    "v_mfma_f32_16x16x32_f16 a[160:163], v[40:43], v[148:151], a[160:163]\n\t" \
    "v_mfma_f32_16x16x32_f16 a[192:195], v[44:47], v[148:151], a[192:195]\n\t" \
    "s_waitcnt vmcnt(12)\n\t" \
    "v_mfma_f32_16x16x32_f16 a[192:195], v[76:79], v[152:155], a[192:195]\n\t" \
    "s_waitcnt vmcnt(11)\n\t" \
    "v_mfma_f32_16x16x32_f16 a[224:227], v[80:83], v[152:155], a[224:227]\n\t" \
    "buffer_load_dwordx4 v[96:99], %[vA0], %[rWS], s55 offen offset:3072 sc0 sc1\n\t" \
    "v_mfma_f32_16x16x32_f16 a[224:227], v[48:51], v[148:151], a[224:227]\n\t" \
    "s_nop 9\n\t" \
    "ds_read_b128 v[148:151], %[vA0] offset:2048\n\t" \
    "s_waitcnt lgkmcnt(2)\n\t" \
    "v_mfma_f32_16x16x32_f16 a[228:231], v[48:51], v[156:159], a[228:231]\n\t" \
    "s_add_u32 s55, s37, 20480\n\t" \
    "s_waitcnt lgkmcnt(1)\n\t" \
    "v_mfma_f32_16x16x32_f16 a[228:231], v[80:83], v[160:163], a[228:231]\n\t" \
    "v_mfma_f32_16x16x32_f16 a[196:199], v[76:79], v[160:163], a[196:199]\n\t" \
    "buffer_load_dwordx4 v[100:103], %[vA0], %[rWS], s55 offen offset:0 sc0 sc1\n\t" \
    "v_mfma_f32_16x16x32_f16 a[196:199], v[44:47], v[156:159], a[196:199]\n\t" \
    "v_mfma_f32_16x16x32_f16 a[164:167], v[40:43], v[156:159], a[164:167]\n\t" \
    "v_mfma_f32_16x16x32_f16 a[164:167], v[72:75], v[160:163], a[164:167]\n\t" \
    "v_mfma_f32_16x16x32_f16 a[132:135], v[68:71], v[160:163], a[132:135]\n\t" \
    "buffer_load_dwordx4 v[104:107], %[vA0], %[rWS], s55 offen offset:1024 sc0 sc1\n\t" \
    "v_mfma_f32_16x16x32_f16 a[132:135], v[36:39], v[156:159], a[132:135]\n\t" \
    "s_nop 9\n\t" \
    "ds_read_b128 v[152:155], %[vA0] offset:10240\n\t" \
    "v_mfma_f32_16x16x32_f16 a[100:103], v[32:35], v[156:159], a[100:103]\n\t" \
    "v_cvt_pk_f16_f32 v228, v164, v168\n\t" \
    "v_mfma_f32_16x16x32_f16 a[100:103], v[64:67], v[160:163], a[100:103]\n\t" \
    "v_mfma_f32_16x16x32_f16 a[68:71], v[60:63], v[160:163], a[68:71]\n\t" \
    "buffer_load_dwordx4 v[108:111], %[vA0], %[rWS], s55 offen offset:2048 sc0 sc1\n\t" \
    "v_mfma_f32_16x16x32_f16 a[68:71], v[28:31], v[156:159], a[68:71]\n\t" \
    "v_cvt_pk_f16_f32 v229, v172, v176\n\t" \
    "v_mfma_f32_16x16x32_f16 a[36:39], v[24:27], v[156:159], a[36:39]\n\t" \
    "ds_write_b64 %[vWr0], v[228:229] offset:32768\n\t" \
    "v_mfma_f32_16x16x32_f16 a[36:39], v[56:59], v[160:163], a[36:39]\n\t" \
    "v_cvt_pk_f16_f32 v230, v180, v184\n\t" \
    "v_mfma_f32_16x16x32_f16 a[4:7], v[52:55], v[160:163], a[4:7]\n\t" \
    "buffer_load_dwordx4 v[112:115], %[vA0], %[rWS], s55 offen offset:3072 sc0 sc1\n\t" \
    "v_mfma_f32_16x16x32_f16 a[4:7], v[20:23], v[156:159], a[4:7]\n\t" \
    "s_nop 9\n\t" \
    "ds_read_b128 v[156:159], %[vA0] offset:3072\n\t" \
    "s_waitcnt lgkmcnt(3)\n\t" \
    "v_mfma_f32_16x16x32_f16 a[8:11], v[20:23], v[148:151], a[8:11]\n\t" \
    "v_cvt_pk_f16_f32 v231, v188, v192\n\t" \
    "s_waitcnt lgkmcnt(2)\n\t" \
    "v_mfma_f32_16x16x32_f16 a[8:11], v[52:55], v[152:155], a[8:11]\n\t" \
    "ds_write_b64 %[vWr0], v[230:231] offset:32776\n\t" \
    "v_mfma_f32_16x16x32_f16 a[40:43], v[56:59], v[152:155], a[40:43]\n\t" \
    "s_add_u32 s55, s37, 24576\n\t" \
    "v_cvt_pk_f16_f32 v228, v165, v169\n\t" \
    "v_mfma_f32_16x16x32_f16 a[40:43], v[24:27], v[148:151], a[40:43]\n\t" \
    "buffer_load_dwordx4 v[116:119], %[vA0], %[rWS], s55 offen offset:0 sc0 sc1\n\t" \
    "v_mfma_f32_16x16x32_f16 a[72:75], v[28:31], v[148:151], a[72:75]\n\t" \
    "v_cvt_pk_f16_f32 v229, v173, v177\n\t" \
    "s_add_u32 s38, s38, 32768\n\t" \
    "v_mfma_f32_16x16x32_f16 a[72:75], v[60:63], v[152:155], a[72:75]\n\t" \
    "s_and_b32 s38, s38, 0x1fffff\n\t" \
    "s_or_b32 s38, s38, %[sA0]\n\t" \
    "v_mfma_f32_16x16x32_f16 a[104:107], v[64:67], v[152:155], a[104:107]\n\t" \
    "ds_write_b64 %[vWr0], v[228:229] offset:33792\n\t" \
    "v_mfma_f32_16x16x32_f16 a[104:107], v[32:35], v[148:151], a[104:107]\n\t" \
    "s_nop 9\n\t" \
    "ds_read_b128 v[160:163], %[vA0] offset:11264\n\t" \
    "v_mfma_f32_16x16x32_f16 a[136:139], v[36:39], v[148:151], a[136:139]\n\t" \
    "v_cvt_pk_f16_f32 v230, v181, v185\n\t" \
    "v_mfma_f32_16x16x32_f16 a[136:139], v[68:71], v[152:155], a[136:139]\n\t" \
    "buffer_load_dwordx4 v[120:123], %[vA0], %[rWS], s55 offen offset:1024 sc0 sc1\n\t" \
    "v_mfma_f32_16x16x32_f16 a[168:171], v[72:75], v[152:155], a[168:171]\n\t" \
    "v_cvt_pk_f16_f32 v231, v189, v193\n\t" \
    "v_mfma_f32_16x16x32_f16 a[168:171], v[40:43], v[148:151], a[168:171]\n\t" \
    "ds_write_b64 %[vWr0], v[230:231] offset:33800\n\t" \
    "v_mfma_f32_16x16x32_f16 a[200:203], v[44:47], v[148:151], a[200:203]\n\t" \
    "v_cvt_pk_f16_f32 v228, v166, v170\n\t" \
    "v_mfma_f32_16x16x32_f16 a[200:203], v[76:79], v[152:155], a[200:203]\n\t" \
    "buffer_load_dwordx4 v[124:127], %[vA0], %[rWS], s55 offen offset:2048 sc0 sc1\n\t" \
    "v_mfma_f32_16x16x32_f16 a[232:235], v[80:83], v[152:155], a[232:235]\n\t" \
    "v_cvt_pk_f16_f32 v229, v174, v178\n\t" \
    "v_mfma_f32_16x16x32_f16 a[232:235], v[48:51], v[148:151], a[232:235]\n\t" \
    "s_nop 9\n\t" \
    "ds_read_b128 v[148:151], %[vA0] offset:4096\n\t" \
    "s_waitcnt lgkmcnt(5)\n\t" \
    "v_mfma_f32_16x16x32_f16 a[236:239], v[48:51], v[156:159], a[236:239]\n\t" \
    "ds_write_b64 %[vWr0], v[228:229] offset:34816\n\t" \
    "s_waitcnt lgkmcnt(3)\n\t" \
    "v_mfma_f32_16x16x32_f16 a[236:239], v[80:83], v[160:163], a[236:239]\n\t" \
    "v_cvt_pk_f16_f32 v230, v182, v186\n\t" \
    "v_mfma_f32_16x16x32_f16 a[204:207], v[76:79], v[160:163], a[204:207]\n\t" \
    "buffer_load_dwordx4 v[128:131], %[vA0], %[rWS], s55 offen offset:3072 sc0 sc1\n\t" \
    "v_mfma_f32_16x16x32_f16 a[204:207], v[44:47], v[156:159], a[204:207]\n\t" \
    "v_cvt_pk_f16_f32 v231, v190, v194\n\t" \
    "v_mfma_f32_16x16x32_f16 a[172:175], v[40:43], v[156:159], a[172:175]\n\t" \
    "ds_write_b64 %[vWr0], v[230:231] offset:34824\n\t" \
    "v_mfma_f32_16x16x32_f16 a[172:175], v[72:75], v[160:163], a[172:175]\n\t" \
    "s_add_u32 s55, s37, 28672\n\t" \
    "v_cvt_pk_f16_f32 v228, v167, v171\n\t" \
    "v_mfma_f32_16x16x32_f16 a[140:143], v[68:71], v[160:163], a[140:143]\n\t" \
    "buffer_load_dwordx4 v[132:135], %[vA0], %[rWS], s55 offen offset:0 sc0 sc1\n\t" \
    "v_mfma_f32_16x16x32_f16 a[140:143], v[36:39], v[156:159], a[140:143]\n\t" \
    "s_nop 9\n\t" \
    "ds_read_b128 v[152:155], %[vA0] offset:12288\n\t" \
    "v_mfma_f32_16x16x32_f16 a[108:111], v[32:35], v[156:159], a[108:111]\n\t" \
    "v_cvt_pk_f16_f32 v229, v175, v179\n\t" \
    "v_mfma_f32_16x16x32_f16 a[108:111], v[64:67], v[160:163], a[108:111]\n\t" \
    "ds_write_b64 %[vWr0], v[228:229] offset:35840\n\t" \
    "v_mfma_f32_16x16x32_f16 a[76:79], v[60:63], v[160:163], a[76:79]\n\t" \
    "v_cvt_pk_f16_f32 v230, v183, v187\n\t" \
    "v_mfma_f32_16x16x32_f16 a[76:79], v[28:31], v[156:159], a[76:79]\n\t" \
    "buffer_load_dwordx4 v[136:139], %[vA0], %[rWS], s55 offen offset:1024 sc0 sc1\n\t" \
    "v_mfma_f32_16x16x32_f16 a[44:47], v[24:27], v[156:159], a[44:47]\n\t" \
    "s_add_u32 s66, s66, 0x400000\n\t" \
    "s_and_b32 s66, s66, 0xfffffff\n\t" \
    "v_mfma_f32_16x16x32_f16 a[44:47], v[56:59], v[160:163], a[44:47]\n\t" \
    "v_cvt_pk_f16_f32 v231, v191, v195\n\t" \
    "v_mfma_f32_16x16x32_f16 a[12:15], v[52:55], v[160:163], a[12:15]\n\t" \
    "ds_write_b64 %[vWr0], v[230:231] offset:35848\n\t" \
    "v_mfma_f32_16x16x32_f16 a[12:15], v[20:23], v[156:159], a[12:15]\n\t" \
    "s_nop 9\n\t" \
    "ds_read_b128 v[156:159], %[vA0] offset:5120\n\t" \
    "s_waitcnt lgkmcnt(6)\n\t" \
    "v_mfma_f32_16x16x32_f16 a[16:19], v[20:23], v[148:151], a[16:19]\n\t" \
    "buffer_load_dwordx4 v[140:143], %[vA0], %[rWS], s55 offen offset:2048 sc0 sc1\n\t" \
    "s_waitcnt lgkmcnt(3)\n\t" \
    "v_mfma_f32_16x16x32_f16 a[16:19], v[52:55], v[152:155], a[16:19]\n\t" \
    "s_and_b32 s41, s36, 7\n\t" \
    "s_cmp_gt_u32 s41, 5\n\t" \
    "v_mfma_f32_16x16x32_f16 a[48:51], v[56:59], v[152:155], a[48:51]\n\t" \
    "s_cselect_b32 s41, s44, s43\n\t" \
    "s_add_u32 s39, s66, s41\n\t" \
    "v_mfma_f32_16x16x32_f16 a[48:51], v[24:27], v[148:151], a[48:51]\n\t" \
    "s_add_u32 s39, s39, %[sWk0]\n\t" \
    "v_mfma_f32_16x16x32_f16 a[80:83], v[28:31], v[148:151], a[80:83]\n\t" \
    "buffer_load_dwordx4 v[144:147], %[vA0], %[rWS], s55 offen offset:3072 sc0 sc1\n\t" \
    "v_mfma_f32_16x16x32_f16 a[80:83], v[60:63], v[152:155], a[80:83]\n\t" \
    "buffer_load_dwordx4 v[164:167], %[vW], %[rW], s39 offen\n\t" \
    "v_mfma_f32_16x16x32_f16 a[112:115], v[64:67], v[152:155], a[112:115]\n\t" \
    "s_add_u32 s39, s39, 32768\n\t" \
    "v_mfma_f32_16x16x32_f16 a[112:115], v[32:35], v[148:151], a[112:115]\n\t" \
    "s_nop 9\n\t" \
    "ds_read_b128 v[160:163], %[vA0] offset:13312\n\t" \
    "v_mfma_f32_16x16x32_f16 a[144:147], v[36:39], v[148:151], a[144:147]\n\t" \
    "buffer_load_dwordx4 v[168:171], %[vW], %[rW], s39 offen\n\t" \
    "v_mfma_f32_16x16x32_f16 a[144:147], v[68:71], v[152:155], a[144:147]\n\t" \
    "s_add_u32 s39, s39, 32768\n\t" \
    "v_mfma_f32_16x16x32_f16 a[176:179], v[72:75], v[152:155], a[176:179]\n\t" \
    "buffer_load_dwordx4 v[172:175], %[vW], %[rW], s39 offen\n\t" \
    "v_mfma_f32_16x16x32_f16 a[176:179], v[40:43], v[148:151], a[176:179]\n\t" \
    "v_mfma_f32_16x16x32_f16 a[208:211], v[44:47], v[148:151], a[208:211]\n\t" \
    "s_add_u32 s39, s39, 32768\n\t" \
    "v_mfma_f32_16x16x32_f16 a[208:211], v[76:79], v[152:155], a[208:211]\n\t" \
    "v_mfma_f32_16x16x32_f16 a[240:243], v[80:83], v[152:155], a[240:243]\n\t" \
    "buffer_load_dwordx4 v[176:179], %[vW], %[rW], s39 offen\n\t" \
    "v_mfma_f32_16x16x32_f16 a[240:243], v[48:51], v[148:151], a[240:243]\n\t" \
    "s_nop 9\n\t" \
    "ds_read_b128 v[148:151], %[vA0] offset:6144\n\t" \
    "s_waitcnt lgkmcnt(2)\n\t" \
    "v_mfma_f32_16x16x32_f16 a[244:247], v[48:51], v[156:159], a[244:247]\n\t" \
    "s_add_u32 s39, s39, 32768\n\t" \
    "s_waitcnt lgkmcnt(1)\n\t" \
    "v_mfma_f32_16x16x32_f16 a[244:247], v[80:83], v[160:163], a[244:247]\n\t" \
    "v_mfma_f32_16x16x32_f16 a[212:215], v[76:79], v[160:163], a[212:215]\n\t" \
    "buffer_load_dwordx4 v[180:183], %[vW], %[rW], s39 offen\n\t" \
    "v_mfma_f32_16x16x32_f16 a[212:215], v[44:47], v[156:159], a[212:215]\n\t" \
    "v_mfma_f32_16x16x32_f16 a[180:183], v[40:43], v[156:159], a[180:183]\n\t" \
    "s_add_u32 s39, s39, 32768\n\t" \
    "v_mfma_f32_16x16x32_f16 a[180:183], v[72:75], v[160:163], a[180:183]\n\t" \
    "v_mfma_f32_16x16x32_f16 a[148:151], v[68:71], v[160:163], a[148:151]\n\t" \
    "buffer_load_dwordx4 v[184:187], %[vW], %[rW], s39 offen\n\t" \
    "v_mfma_f32_16x16x32_f16 a[148:151], v[36:39], v[156:159], a[148:151]\n\t" \
    "s_nop 9\n\t" \
    "ds_read_b128 v[152:155], %[vA0] offset:14336\n\t" \
    "v_mfma_f32_16x16x32_f16 a[116:119], v[32:35], v[156:159], a[116:119]\n\t" \
    "s_add_u32 s39, s39, 32768\n\t" \
    "v_mfma_f32_16x16x32_f16 a[116:119], v[64:67], v[160:163], a[116:119]\n\t" \
    "v_mfma_f32_16x16x32_f16 a[84:87], v[60:63], v[160:163], a[84:87]\n\t" \
    "buffer_load_dwordx4 v[188:191], %[vW], %[rW], s39 offen\n\t" \
    "v_mfma_f32_16x16x32_f16 a[84:87], v[28:31], v[156:159], a[84:87]\n\t" \
    "v_mfma_f32_16x16x32_f16 a[52:55], v[24:27], v[156:159], a[52:55]\n\t" \
    "s_add_u32 s39, s39, 32768\n\t" \
    "v_mfma_f32_16x16x32_f16 a[52:55], v[56:59], v[160:163], a[52:55]\n\t" \
    "v_mfma_f32_16x16x32_f16 a[20:23], v[52:55], v[160:163], a[20:23]\n\t" \
    "buffer_load_dwordx4 v[192:195], %[vW], %[rW], s39 offen\n\t" \
    "v_mfma_f32_16x16x32_f16 a[20:23], v[20:23], v[156:159], a[20:23]\n\t" \
    "s_nop 9\n\t" \
    "ds_read_b128 v[156:159], %[vA0] offset:7168\n\t" \
    "s_waitcnt lgkmcnt(2)\n\t" \
    "v_mfma_f32_16x16x32_f16 a[24:27], v[20:23], v[148:151], a[24:27]\n\t" \
    "s_waitcnt vmcnt(30)\n\t" \
    "v_cvt_pk_f16_f32 v228, v196, v200\n\t" \
    "s_waitcnt lgkmcnt(1)\n\t" \
    "v_mfma_f32_16x16x32_f16 a[24:27], v[52:55], v[152:155], a[24:27]\n\t" \
    "v_mfma_f32_16x16x32_f16 a[56:59], v[56:59], v[152:155], a[56:59]\n\t" \
    "s_waitcnt vmcnt(28)\n\t" \
    "v_cvt_pk_f16_f32 v229, v204, v208\n\t" \
    "v_mfma_f32_16x16x32_f16 a[56:59], v[24:27], v[148:151], a[56:59]\n\t" \
    "ds_write_b64 %[vWr1], v[228:229] offset:32768\n\t" \
    "v_mfma_f32_16x16x32_f16 a[88:91], v[28:31], v[148:151], a[88:91]\n\t" \
    "s_waitcnt vmcnt(26)\n\t" \
    "v_cvt_pk_f16_f32 v230, v212, v216\n\t" \
    "v_mfma_f32_16x16x32_f16 a[88:91], v[60:63], v[152:155], a[88:91]\n\t" \
    "v_mfma_f32_16x16x32_f16 a[120:123], v[64:67], v[152:155], a[120:123]\n\t" \
    "s_waitcnt vmcnt(24)\n\t" \
    "v_cvt_pk_f16_f32 v231, v220, v224\n\t" \
    "v_mfma_f32_16x16x32_f16 a[120:123], v[32:35], v[148:151], a[120:123]\n\t" \
    "s_nop 9\n\t" \
    "ds_read_b128 v[160:163], %[vA0] offset:15360\n\t" \
    "v_mfma_f32_16x16x32_f16 a[152:155], v[36:39], v[148:151], a[152:155]\n\t" \
    "ds_write_b64 %[vWr1], v[230:231] offset:32776\n\t" \
    "v_mfma_f32_16x16x32_f16 a[152:155], v[68:71], v[152:155], a[152:155]\n\t" \
    "v_cvt_pk_f16_f32 v228, v197, v201\n\t" \
    "v_mfma_f32_16x16x32_f16 a[184:187], v[72:75], v[152:155], a[184:187]\n\t" \
    "v_cvt_pk_f16_f32 v229, v205, v209\n\t" \
    "v_mfma_f32_16x16x32_f16 a[184:187], v[40:43], v[148:151], a[184:187]\n\t" \
    "ds_write_b64 %[vWr1], v[228:229] offset:33792\n\t" \
    "v_mfma_f32_16x16x32_f16 a[216:219], v[44:47], v[148:151], a[216:219]\n\t" \
    "v_cvt_pk_f16_f32 v230, v213, v217\n\t" \
    "v_mfma_f32_16x16x32_f16 a[216:219], v[76:79], v[152:155], a[216:219]\n\t" \
    "v_mfma_f32_16x16x32_f16 a[248:251], v[80:83], v[152:155], a[248:251]\n\t" \
    "v_cvt_pk_f16_f32 v231, v221, v225\n\t" \
    "v_mfma_f32_16x16x32_f16 a[248:251], v[48:51], v[148:151], a[248:251]\n\t" \
    "s_nop 9\n\t" \
    "ds_read_b128 v[148:151], %[vA0] offset:16384\n\t" \
    "s_waitcnt lgkmcnt(5)\n\t" \
    "v_mfma_f32_16x16x32_f16 a[252:255], v[48:51], v[156:159], a[252:255]\n\t" \
    "ds_write_b64 %[vWr1], v[230:231] offset:33800\n\t" \
    "s_waitcnt lgkmcnt(4)\n\t" \
    "v_mfma_f32_16x16x32_f16 a[252:255], v[80:83], v[160:163], a[252:255]\n\t" \
    "v_cvt_pk_f16_f32 v228, v198, v202\n\t" \
    "v_mfma_f32_16x16x32_f16 a[220:223], v[76:79], v[160:163], a[220:223]\n\t" \
    "v_cvt_pk_f16_f32 v229, v206, v210\n\t" \
    "v_mfma_f32_16x16x32_f16 a[220:223], v[44:47], v[156:159], a[220:223]\n\t" \
    "ds_write_b64 %[vWr1], v[228:229] offset:34816\n\t" \
    "v_mfma_f32_16x16x32_f16 a[188:191], v[40:43], v[156:159], a[188:191]\n\t" \
    "v_cvt_pk_f16_f32 v230, v214, v218\n\t" \
    "v_mfma_f32_16x16x32_f16 a[188:191], v[72:75], v[160:163], a[188:191]\n\t" \
    "v_mfma_f32_16x16x32_f16 a[156:159], v[68:71], v[160:163], a[156:159]\n\t" \
    "v_cvt_pk_f16_f32 v231, v222, v226\n\t" \
    "v_mfma_f32_16x16x32_f16 a[156:159], v[36:39], v[156:159], a[156:159]\n\t" \
    "s_nop 9\n\t" \
    "ds_read_b128 v[152:155], %[vA0] offset:24576\n\t" \
    "v_mfma_f32_16x16x32_f16 a[124:127], v[32:35], v[156:159], a[124:127]\n\t" \
    "ds_write_b64 %[vWr1], v[230:231] offset:34824\n\t" \
    "v_mfma_f32_16x16x32_f16 a[124:127], v[64:67], v[160:163], a[124:127]\n\t" \
    "v_cvt_pk_f16_f32 v228, v199, v203\n\t" \
    "v_mfma_f32_16x16x32_f16 a[92:95], v[60:63], v[160:163], a[92:95]\n\t" \
    "v_cvt_pk_f16_f32 v229, v207, v211\n\t" \
    "v_mfma_f32_16x16x32_f16 a[92:95], v[28:31], v[156:159], a[92:95]\n\t" \
    "ds_write_b64 %[vWr1], v[228:229] offset:35840\n\t" \
    "v_mfma_f32_16x16x32_f16 a[60:63], v[24:27], v[156:159], a[60:63]\n\t" \
    "v_cvt_pk_f16_f32 v230, v215, v219\n\t" \
    "v_mfma_f32_16x16x32_f16 a[60:63], v[56:59], v[160:163], a[60:63]\n\t" \
    "v_mfma_f32_16x16x32_f16 a[28:31], v[52:55], v[160:163], a[28:31]\n\t" \
    "v_cvt_pk_f16_f32 v231, v223, v227\n\t" \
    "v_mfma_f32_16x16x32_f16 a[28:31], v[20:23], v[156:159], a[28:31]\n\t" \
    "s_nop 9\n\t" \
    "ds_read_b128 v[156:159], %[vA0] offset:17408\n\t" \
    "s_waitcnt vmcnt(23) lgkmcnt(6)\n\t" \
    "v_mfma_f32_16x16x32_f16 a[0:3], v[84:87], v[148:151], a[0:3]\n\t" \
    "ds_write_b64 %[vWr1], v[230:231] offset:35848\n\t" \
    "s_waitcnt vmcnt(15) lgkmcnt(4)\n\t" \
    "v_mfma_f32_16x16x32_f16 a[0:3], v[116:119], v[152:155], a[0:3]\n\t" \
    "s_add_u32 s55, s38, 0\n\t" \
    "s_waitcnt vmcnt(14)\n\t" \
    "v_mfma_f32_16x16x32_f16 a[32:35], v[120:123], v[152:155], a[32:35]\n\t" \
    "buffer_load_dwordx4 v[20:23], %[vA0], %[rWS], s55 offen offset:0 sc0 sc1\n\t" \
    "v_mfma_f32_16x16x32_f16 a[32:35], v[88:91], v[148:151], a[32:35]\n\t" \
    "v_mfma_f32_16x16x32_f16 a[64:67], v[92:95], v[148:151], a[64:67]\n\t" \
    "s_waitcnt vmcnt(14)\n\t" \
    "v_mfma_f32_16x16x32_f16 a[64:67], v[124:127], v[152:155], a[64:67]\n\t" \
    "s_waitcnt vmcnt(13)\n\t" \
    "v_mfma_f32_16x16x32_f16 a[96:99], v[128:131], v[152:155], a[96:99]\n\t" \
    "buffer_load_dwordx4 v[24:27], %[vA0], %[rWS], s55 offen offset:1024 sc0 sc1\n\t" \
    "v_mfma_f32_16x16x32_f16 a[96:99], v[96:99], v[148:151], a[96:99]\n\t" \
    "s_nop 9\n\t" \
    "ds_read_b128 v[160:163], %[vA0] offset:25600\n\t" \
    "v_mfma_f32_16x16x32_f16 a[128:131], v[100:103], v[148:151], a[128:131]\n\t" \
    "s_waitcnt vmcnt(13)\n\t" \
    "v_mfma_f32_16x16x32_f16 a[128:131], v[132:135], v[152:155], a[128:131]\n\t" \
    "s_waitcnt vmcnt(12)\n\t" \
    "v_mfma_f32_16x16x32_f16 a[160:163], v[136:139], v[152:155], a[160:163]\n\t" \
    "buffer_load_dwordx4 v[28:31], %[vA0], %[rWS], s55 offen offset:2048 sc0 sc1\n\t" \
    "v_mfma_f32_16x16x32_f16 a[160:163], v[104:107], v[148:151], a[160:163]\n\t" \
    "v_mfma_f32_16x16x32_f16 a[192:195], v[108:111], v[148:151], a[192:195]\n\t" \
    "s_waitcnt vmcnt(12)\n\t" \
    "v_mfma_f32_16x16x32_f16 a[192:195], v[140:143], v[152:155], a[192:195]\n\t" \
    "s_waitcnt vmcnt(11)\n\t" \
    "v_mfma_f32_16x16x32_f16 a[224:227], v[144:147], v[152:155], a[224:227]\n\t" \
    "buffer_load_dwordx4 v[32:35], %[vA0], %[rWS], s55 offen offset:3072 sc0 sc1\n\t" \
    "v_mfma_f32_16x16x32_f16 a[224:227], v[112:115], v[148:151], a[224:227]\n\t" \
    "s_nop 9\n\t" \
    "ds_read_b128 v[148:151], %[vA0] offset:18432\n\t" \
    "s_waitcnt lgkmcnt(3)\n\t" \
    "v_mfma_f32_16x16x32_f16 a[228:231], v[112:115], v[156:159], a[228:231]\n\t" \
    "s_waitcnt lgkmcnt(1)\n\t" \
    "v_mfma_f32_16x16x32_f16 a[228:231], v[144:147], v[160:163], a[228:231]\n\t" \
    "s_add_u32 s55, s38, 4096\n\t" \
    "v_mfma_f32_16x16x32_f16 a[196:199], v[140:143], v[160:163], a[196:199]\n\t" \
    "v_mfma_f32_16x16x32_f16 a[196:199], v[108:111], v[156:159], a[196:199]\n\t" \
    "buffer_load_dwordx4 v[36:39], %[vA0], %[rWS], s55 offen offset:0 sc0 sc1\n\t" \
    "v_mfma_f32_16x16x32_f16 a[164:167], v[104:107], v[156:159], a[164:167]\n\t" \
    "v_mfma_f32_16x16x32_f16 a[164:167], v[136:139], v[160:163], a[164:167]\n\t" \
    "v_mfma_f32_16x16x32_f16 a[132:135], v[132:135], v[160:163], a[132:135]\n\t" \
    "v_mfma_f32_16x16x32_f16 a[132:135], v[100:103], v[156:159], a[132:135]\n\t" \
    "s_nop 9\n\t" \
    "ds_read_b128 v[152:155], %[vA0] offset:26624\n\t" \
    "v_mfma_f32_16x16x32_f16 a[100:103], v[96:99], v[156:159], a[100:103]\n\t" \
    "buffer_load_dwordx4 v[40:43], %[vA0], %[rWS], s55 offen offset:1024 sc0 sc1\n\t" \
    "v_mfma_f32_16x16x32_f16 a[100:103], v[128:131], v[160:163], a[100:103]\n\t" \
    "v_mfma_f32_16x16x32_f16 a[68:71], v[124:127], v[160:163], a[68:71]\n\t" \
    "v_mfma_f32_16x16x32_f16 a[68:71], v[92:95], v[156:159], a[68:71]\n\t" \
    "buffer_load_dwordx4 v[44:47], %[vA0], %[rWS], s55 offen offset:2048 sc0 sc1\n\t" \
    "v_mfma_f32_16x16x32_f16 a[36:39], v[88:91], v[156:159], a[36:39]\n\t" \
    "v_mfma_f32_16x16x32_f16 a[36:39], v[120:123], v[160:163], a[36:39]\n\t" \
    "v_mfma_f32_16x16x32_f16 a[4:7], v[116:119], v[160:163], a[4:7]\n\t" \
    "v_mfma_f32_16x16x32_f16 a[4:7], v[84:87], v[156:159], a[4:7]\n\t" \
    "s_nop 9\n\t" \
    "ds_read_b128 v[156:159], %[vA0] offset:19456\n\t" \
    "s_waitcnt lgkmcnt(2)\n\t" \
    "v_mfma_f32_16x16x32_f16 a[8:11], v[84:87], v[148:151], a[8:11]\n\t" \
    "buffer_load_dwordx4 v[48:51], %[vA0], %[rWS], s55 offen offset:3072 sc0 sc1\n\t" \
    "s_waitcnt lgkmcnt(1)\n\t" \
    "v_mfma_f32_16x16x32_f16 a[8:11], v[116:119], v[152:155], a[8:11]\n\t" \
    "v_mfma_f32_16x16x32_f16 a[40:43], v[120:123], v[152:155], a[40:43]\n\t" \
    "s_add_u32 s55, s38, 8192\n\t" \
    "v_mfma_f32_16x16x32_f16 a[40:43], v[88:91], v[148:151], a[40:43]\n\t" \
    "v_mfma_f32_16x16x32_f16 a[72:75], v[92:95], v[148:151], a[72:75]\n\t" \
    "buffer_load_dwordx4 v[52:55], %[vA0], %[rWS], s55 offen offset:0 sc0 sc1\n\t" \
    "v_mfma_f32_16x16x32_f16 a[72:75], v[124:127], v[152:155], a[72:75]\n\t" \
    "v_mfma_f32_16x16x32_f16 a[104:107], v[128:131], v[152:155], a[104:107]\n\t" \
    "v_mfma_f32_16x16x32_f16 a[104:107], v[96:99], v[148:151], a[104:107]\n\t" \
    "s_nop 9\n\t" \
    "ds_read_b128 v[160:163], %[vA0] offset:27648\n\t" \
    "v_mfma_f32_16x16x32_f16 a[136:139], v[100:103], v[148:151], a[136:139]\n\t" \
    "buffer_load_dwordx4 v[56:59], %[vA0], %[rWS], s55 offen offset:1024 sc0 sc1\n\t" \
    "v_mfma_f32_16x16x32_f16 a[136:139], v[132:135], v[152:155], a[136:139]\n\t" \
    "v_mfma_f32_16x16x32_f16 a[168:171], v[136:139], v[152:155], a[168:171]\n\t" \
    "v_mfma_f32_16x16x32_f16 a[168:171], v[104:107], v[148:151], a[168:171]\n\t" \
    "v_mfma_f32_16x16x32_f16 a[200:203], v[108:111], v[148:151], a[200:203]\n\t" \
    "buffer_load_dwordx4 v[60:63], %[vA0], %[rWS], s55 offen offset:2048 sc0 sc1\n\t" \
    "v_mfma_f32_16x16x32_f16 a[200:203], v[140:143], v[152:155], a[200:203]\n\t" \
    "v_mfma_f32_16x16x32_f16 a[232:235], v[144:147], v[152:155], a[232:235]\n\t" \
    "v_mfma_f32_16x16x32_f16 a[232:235], v[112:115], v[148:151], a[232:235]\n\t" \
    "s_nop 9\n\t" \
    "ds_read_b128 v[148:151], %[vA0] offset:20480\n\t" \
    "s_waitcnt lgkmcnt(2)\n\t" \
    "v_mfma_f32_16x16x32_f16 a[236:239], v[112:115], v[156:159], a[236:239]\n\t" \
    "buffer_load_dwordx4 v[64:67], %[vA0], %[rWS], s55 offen offset:3072 sc0 sc1\n\t" \
    "s_waitcnt lgkmcnt(1)\n\t" \
    "v_mfma_f32_16x16x32_f16 a[236:239], v[144:147], v[160:163], a[236:239]\n\t" \
    "v_mfma_f32_16x16x32_f16 a[204:207], v[140:143], v[160:163], a[204:207]\n\t" \
    "v_mfma_f32_16x16x32_f16 a[204:207], v[108:111], v[156:159], a[204:207]\n\t" \
    "s_add_u32 s55, s38, 12288\n\t" \
    "v_mfma_f32_16x16x32_f16 a[172:175], v[104:107], v[156:159], a[172:175]\n\t" \
    "v_mfma_f32_16x16x32_f16 a[172:175], v[136:139], v[160:163], a[172:175]\n\t" \
    "buffer_load_dwordx4 v[68:71], %[vA0], %[rWS], s55 offen offset:0 sc0 sc1\n\t" \
    "v_mfma_f32_16x16x32_f16 a[140:143], v[132:135], v[160:163], a[140:143]\n\t" \
    "v_mfma_f32_16x16x32_f16 a[140:143], v[100:103], v[156:159], a[140:143]\n\t" \
    "s_nop 9\n\t" \
    "ds_read_b128 v[152:155], %[vA0] offset:28672\n\t" \
    "v_mfma_f32_16x16x32_f16 a[108:111], v[96:99], v[156:159], a[108:111]\n\t" \
    "v_mfma_f32_16x16x32_f16 a[108:111], v[128:131], v[160:163], a[108:111]\n\t" \
    "buffer_load_dwordx4 v[72:75], %[vA0], %[rWS], s55 offen offset:1024 sc0 sc1\n\t" \
    "v_mfma_f32_16x16x32_f16 a[76:79], v[124:127], v[160:163], a[76:79]\n\t" \
    "v_mfma_f32_16x16x32_f16 a[76:79], v[92:95], v[156:159], a[76:79]\n\t" \
    "v_mfma_f32_16x16x32_f16 a[44:47], v[88:91], v[156:159], a[44:47]\n\t" \
    "v_mfma_f32_16x16x32_f16 a[44:47], v[120:123], v[160:163], a[44:47]\n\t" \
    "buffer_load_dwordx4 v[76:79], %[vA0], %[rWS], s55 offen offset:2048 sc0 sc1\n\t" \
    "v_mfma_f32_16x16x32_f16 a[12:15], v[116:119], v[160:163], a[12:15]\n\t" \
    "s_and_b32 s41, s36, 7\n\t" \
    "s_cmp_gt_u32 s41, 5\n\t" \
    "v_mfma_f32_16x16x32_f16 a[12:15], v[84:87], v[156:159], a[12:15]\n\t" \
    "s_nop 9\n\t" \
    "ds_read_b128 v[156:159], %[vA0] offset:21504\n\t" \
    "s_waitcnt lgkmcnt(2)\n\t" \
    "v_mfma_f32_16x16x32_f16 a[16:19], v[84:87], v[148:151], a[16:19]\n\t" \
    "s_cselect_b32 s41, s44, s43\n\t" \
    "s_add_u32 s40, s66, s41\n\t" \
    "s_waitcnt lgkmcnt(1)\n\t" \
    "v_mfma_f32_16x16x32_f16 a[16:19], v[116:119], v[152:155], a[16:19]\n\t" \
    "s_add_u32 s40, s40, %[sWk1]\n\t" \
    "v_mfma_f32_16x16x32_f16 a[48:51], v[120:123], v[152:155], a[48:51]\n\t" \
    "buffer_load_dwordx4 v[80:83], %[vA0], %[rWS], s55 offen offset:3072 sc0 sc1\n\t" \
    "v_mfma_f32_16x16x32_f16 a[48:51], v[88:91], v[148:151], a[48:51]\n\t" \
    "v_mfma_f32_16x16x32_f16 a[80:83], v[92:95], v[148:151], a[80:83]\n\t" \
    "v_mfma_f32_16x16x32_f16 a[80:83], v[124:127], v[152:155], a[80:83]\n\t" \
    "v_mfma_f32_16x16x32_f16 a[112:115], v[128:131], v[152:155], a[112:115]\n\t" \
    "buffer_load_dwordx4 v[196:199], %[vW], %[rW], s40 offen nt\n\t" \
    "v_mfma_f32_16x16x32_f16 a[112:115], v[96:99], v[148:151], a[112:115]\n\t" \
    "s_nop 9\n\t" \
    "ds_read_b128 v[160:163], %[vA0] offset:29696\n\t" \
    "v_mfma_f32_16x16x32_f16 a[144:147], v[100:103], v[148:151], a[144:147]\n\t" \
    "v_mfma_f32_16x16x32_f16 a[144:147], v[132:135], v[152:155], a[144:147]\n\t" \
    "v_mfma_f32_16x16x32_f16 a[176:179], v[136:139], v[152:155], a[176:179]\n\t" \
    "s_add_u32 s40, s40, 32768\n\t" \
    "v_mfma_f32_16x16x32_f16 a[176:179], v[104:107], v[148:151], a[176:179]\n\t" \
    "v_mfma_f32_16x16x32_f16 a[208:211], v[108:111], v[148:151], a[208:211]\n\t" \
    "buffer_load_dwordx4 v[200:203], %[vW], %[rW], s40 offen nt\n\t" \
    "v_mfma_f32_16x16x32_f16 a[208:211], v[140:143], v[152:155], a[208:211]\n\t" \
    "v_mfma_f32_16x16x32_f16 a[240:243], v[144:147], v[152:155], a[240:243]\n\t" \
    "v_mfma_f32_16x16x32_f16 a[240:243], v[112:115], v[148:151], a[240:243]\n\t" \
    "s_nop 9\n\t" \
    "ds_read_b128 v[148:151], %[vA0] offset:22528\n\t" \
    "s_waitcnt lgkmcnt(2)\n\t" \
    "v_mfma_f32_16x16x32_f16 a[244:247], v[112:115], v[156:159], a[244:247]\n\t" \
    "s_add_u32 s40, s40, 32768\n\t" \
    "s_waitcnt lgkmcnt(1)\n\t" \
    "v_mfma_f32_16x16x32_f16 a[244:247], v[144:147], v[160:163], a[244:247]\n\t" \
    "v_mfma_f32_16x16x32_f16 a[212:215], v[140:143], v[160:163], a[212:215]\n\t" \
    "buffer_load_dwordx4 v[204:207], %[vW], %[rW], s40 offen nt\n\t" \
    "v_mfma_f32_16x16x32_f16 a[212:215], v[108:111], v[156:159], a[212:215]\n\t" \
    "v_mfma_f32_16x16x32_f16 a[180:183], v[104:107], v[156:159], a[180:183]\n\t" \
    "v_mfma_f32_16x16x32_f16 a[180:183], v[136:139], v[160:163], a[180:183]\n\t" \
    "v_mfma_f32_16x16x32_f16 a[148:151], v[132:135], v[160:163], a[148:151]\n\t" \
    "s_add_u32 s40, s40, 32768\n\t" \
    "v_mfma_f32_16x16x32_f16 a[148:151], v[100:103], v[156:159], a[148:151]\n\t" \
    "s_nop 9\n\t" \
    "ds_read_b128 v[152:155], %[vA0] offset:30720\n\t" \
    "v_mfma_f32_16x16x32_f16 a[116:119], v[96:99], v[156:159], a[116:119]\n\t" \
    "buffer_load_dwordx4 v[208:211], %[vW], %[rW], s40 offen nt\n\t" \
    "v_mfma_f32_16x16x32_f16 a[116:119], v[128:131], v[160:163], a[116:119]\n\t" \
    "v_mfma_f32_16x16x32_f16 a[84:87], v[124:127], v[160:163], a[84:87]\n\t" \
    "v_mfma_f32_16x16x32_f16 a[84:87], v[92:95], v[156:159], a[84:87]\n\t" \
    "v_mfma_f32_16x16x32_f16 a[52:55], v[88:91], v[156:159], a[52:55]\n\t" \
    "s_add_u32 s40, s40, 32768\n\t" \
    "v_mfma_f32_16x16x32_f16 a[52:55], v[120:123], v[160:163], a[52:55]\n\t" \
    "v_mfma_f32_16x16x32_f16 a[20:23], v[116:119], v[160:163], a[20:23]\n\t" \
    "buffer_load_dwordx4 v[212:215], %[vW], %[rW], s40 offen nt\n\t" \
    "v_mfma_f32_16x16x32_f16 a[20:23], v[84:87], v[156:159], a[20:23]\n\t" \
    "s_nop 9\n\t" \
    "ds_read_b128 v[156:159], %[vA0] offset:23552\n\t" \
    "s_waitcnt lgkmcnt(2)\n\t" \
    "v_mfma_f32_16x16x32_f16 a[24:27], v[84:87], v[148:151], a[24:27]\n\t" \
    "s_waitcnt lgkmcnt(1)\n\t" \
    "v_mfma_f32_16x16x32_f16 a[24:27], v[116:119], v[152:155], a[24:27]\n\t" \
    "v_mfma_f32_16x16x32_f16 a[56:59], v[120:123], v[152:155], a[56:59]\n\t" \
    "s_add_u32 s40, s40, 32768\n\t" \
    "v_mfma_f32_16x16x32_f16 a[56:59], v[88:91], v[148:151], a[56:59]\n\t" \
    "v_mfma_f32_16x16x32_f16 a[88:91], v[92:95], v[148:151], a[88:91]\n\t" \
    "buffer_load_dwordx4 v[216:219], %[vW], %[rW], s40 offen nt\n\t" \
    "v_mfma_f32_16x16x32_f16 a[88:91], v[124:127], v[152:155], a[88:91]\n\t" \
    "v_mfma_f32_16x16x32_f16 a[120:123], v[128:131], v[152:155], a[120:123]\n\t" \
    "v_mfma_f32_16x16x32_f16 a[120:123], v[96:99], v[148:151], a[120:123]\n\t" \
    "s_nop 9\n\t" \
    "ds_read_b128 v[160:163], %[vA0] offset:31744\n\t" \
    "v_mfma_f32_16x16x32_f16 a[152:155], v[100:103], v[148:151], a[152:155]\n\t" \
    "s_add_u32 s40, s40, 32768\n\t" \
    "v_mfma_f32_16x16x32_f16 a[152:155], v[132:135], v[152:155], a[152:155]\n\t" \
    "v_mfma_f32_16x16x32_f16 a[184:187], v[136:139], v[152:155], a[184:187]\n\t" \
    "buffer_load_dwordx4 v[220:223], %[vW], %[rW], s40 offen nt\n\t" \
    "v_mfma_f32_16x16x32_f16 a[184:187], v[104:107], v[148:151], a[184:187]\n\t" \
    "v_mfma_f32_16x16x32_f16 a[216:219], v[108:111], v[148:151], a[216:219]\n\t" \
    "v_mfma_f32_16x16x32_f16 a[216:219], v[140:143], v[152:155], a[216:219]\n\t" \
    "v_mfma_f32_16x16x32_f16 a[248:251], v[144:147], v[152:155], a[248:251]\n\t" \
    "s_add_u32 s40, s40, 32768\n\t" \
    "v_mfma_f32_16x16x32_f16 a[248:251], v[112:115], v[148:151], a[248:251]\n\t" \
    "s_nop 9\n\t" \
    "s_waitcnt lgkmcnt(0)\n\t" \
    "s_barrier\n\t" \
    "ds_read_b128 v[148:151], %[vA0] offset:32768\n\t" \
    "v_mfma_f32_16x16x32_f16 a[252:255], v[112:115], v[156:159], a[252:255]\n\t" \
    "buffer_load_dwordx4 v[224:227], %[vW], %[rW], s40 offen nt\n\t" \
    "v_mfma_f32_16x16x32_f16 a[252:255], v[144:147], v[160:163], a[252:255]\n\t" \
    "v_mfma_f32_16x16x32_f16 a[220:223], v[140:143], v[160:163], a[220:223]\n\t" \
    "v_mfma_f32_16x16x32_f16 a[220:223], v[108:111], v[156:159], a[220:223]\n\t" \
    "v_mfma_f32_16x16x32_f16 a[188:191], v[104:107], v[156:159], a[188:191]\n\t" \
    "v_mfma_f32_16x16x32_f16 a[188:191], v[136:139], v[160:163], a[188:191]\n\t" \
    "v_mfma_f32_16x16x32_f16 a[156:159], v[132:135], v[160:163], a[156:159]\n\t" \
    "v_mfma_f32_16x16x32_f16 a[156:159], v[100:103], v[156:159], a[156:159]\n\t" \
    "s_nop 9\n\t" \
    "ds_read_b128 v[152:155], %[vA0] offset:40960\n\t" \
    "v_mfma_f32_16x16x32_f16 a[124:127], v[96:99], v[156:159], a[124:127]\n\t" \
    "v_mfma_f32_16x16x32_f16 a[124:127], v[128:131], v[160:163], a[124:127]\n\t" \
    "v_mfma_f32_16x16x32_f16 a[92:95], v[124:127], v[160:163], a[92:95]\n\t" \
    "v_mfma_f32_16x16x32_f16 a[92:95], v[92:95], v[156:159], a[92:95]\n\t" \
    "v_mfma_f32_16x16x32_f16 a[60:63], v[88:91], v[156:159], a[60:63]\n\t" \
    "v_mfma_f32_16x16x32_f16 a[60:63], v[120:123], v[160:163], a[60:63]\n\t" \
    "v_mfma_f32_16x16x32_f16 a[28:31], v[116:119], v[160:163], a[28:31]\n\t" \
    "v_mfma_f32_16x16x32_f16 a[28:31], v[84:87], v[156:159], a[28:31]\n\t" \
    "s_nop 9\n\t" \
    "s_add_u32 s36, s36, 1\n\t" \
    "ds_read_b128 v[156:159], %[vA0] offset:33792\n\t" \
    "s_waitcnt vmcnt(23) lgkmcnt(2)\n\t" \
    "v_mfma_f32_16x16x32_f16 a[0:3], v[20:23], v[148:151], a[0:3]\n\t" \
    "s_mov_b32 s37, s38\n\t" \
    "s_add_u32 s55, s37, 16384\n\t" \
    "s_waitcnt vmcnt(15) lgkmcnt(1)\n\t" \
    "v_mfma_f32_16x16x32_f16 a[0:3], v[52:55], v[152:155], a[0:3]\n\t" \
    "s_waitcnt vmcnt(14)\n\t" \
    "v_mfma_f32_16x16x32_f16 a[32:35], v[56:59], v[152:155], a[32:35]\n\t" \
    "buffer_load_dwordx4 v[84:87], %[vA0], %[rWS], s55 offen offset:0 sc0 sc1\n\t" \
    "v_mfma_f32_16x16x32_f16 a[32:35], v[24:27], v[148:151], a[32:35]\n\t" \
    "v_mfma_f32_16x16x32_f16 a[64:67], v[28:31], v[148:151], a[64:67]\n\t" \
    "s_waitcnt vmcnt(14)\n\t" \
    "v_mfma_f32_16x16x32_f16 a[64:67], v[60:63], v[152:155], a[64:67]\n\t" \
    "s_waitcnt vmcnt(13)\n\t" \
    "v_mfma_f32_16x16x32_f16 a[96:99], v[64:67], v[152:155], a[96:99]\n\t" \
    "buffer_load_dwordx4 v[88:91], %[vA0], %[rWS], s55 offen offset:1024 sc0 sc1\n\t" \
    "v_mfma_f32_16x16x32_f16 a[96:99], v[32:35], v[148:151], a[96:99]\n\t" \
    "s_nop 9\n\t" \
    "ds_read_b128 v[160:163], %[vA0] offset:41984\n\t" \
    "v_mfma_f32_16x16x32_f16 a[128:131], v[36:39], v[148:151], a[128:131]\n\t" \
    "s_waitcnt vmcnt(13)\n\t" \
    "v_mfma_f32_16x16x32_f16 a[128:131], v[68:71], v[152:155], a[128:131]\n\t" \
    "s_waitcnt vmcnt(12)\n\t" \
    "v_mfma_f32_16x16x32_f16 a[160:163], v[72:75], v[152:155], a[160:163]\n\t" \
    "buffer_load_dwordx4 v[92:95], %[vA0], %[rWS], s55 offen offset:2048 sc0 sc1\n\t" \
    "v_mfma_f32_16x16x32_f16 a[160:163], v[40:43], v[148:151], a[160:163]\n\t" \
    "v_mfma_f32_16x16x32_f16 a[192:195], v[44:47], v[148:151], a[192:195]\n\t" \
    "s_waitcnt vmcnt(12)\n\t" \
    "v_mfma_f32_16x16x32_f16 a[192:195], v[76:79], v[152:155], a[192:195]\n\t" \
    "s_waitcnt vmcnt(11)\n\t" \
    "v_mfma_f32_16x16x32_f16 a[224:227], v[80:83], v[152:155], a[224:227]\n\t" \
    "buffer_load_dwordx4 v[96:99], %[vA0], %[rWS], s55 offen offset:3072 sc0 sc1\n\t" \
    "v_mfma_f32_16x16x32_f16 a[224:227], v[48:51], v[148:151], a[224:227]\n\t" \
    "s_nop 9\n\t" \
    "ds_read_b128 v[148:151], %[vA0] offset:34816\n\t" \
    "s_waitcnt lgkmcnt(2)\n\t" \
    "v_mfma_f32_16x16x32_f16 a[228:231], v[48:51], v[156:159], a[228:231]\n\t" \
    "s_add_u32 s55, s37, 20480\n\t" \
    "s_waitcnt lgkmcnt(1)\n\t" \
    "v_mfma_f32_16x16x32_f16 a[228:231], v[80:83], v[160:163], a[228:231]\n\t" \
    "v_mfma_f32_16x16x32_f16 a[196:199], v[76:79], v[160:163], a[196:199]\n\t" \
    "buffer_load_dwordx4 v[100:103], %[vA0], %[rWS], s55 offen offset:0 sc0 sc1\n\t" \
    "v_mfma_f32_16x16x32_f16 a[196:199], v[44:47], v[156:159], a[196:199]\n\t" \
    "v_mfma_f32_16x16x32_f16 a[164:167], v[40:43], v[156:159], a[164:167]\n\t" \
    "v_mfma_f32_16x16x32_f16 a[164:167], v[72:75], v[160:163], a[164:167]\n\t" \
    "v_mfma_f32_16x16x32_f16 a[132:135], v[68:71], v[160:163], a[132:135]\n\t" \
    "buffer_load_dwordx4 v[104:107], %[vA0], %[rWS], s55 offen offset:1024 sc0 sc1\n\t" \
    "v_mfma_f32_16x16x32_f16 a[132:135], v[36:39], v[156:159], a[132:135]\n\t" \
    "s_nop 9\n\t" \
    "ds_read_b128 v[152:155], %[vA0] offset:43008\n\t" \
    "v_mfma_f32_16x16x32_f16 a[100:103], v[32:35], v[156:159], a[100:103]\n\t" \
    "v_cvt_pk_f16_f32 v228, v164, v168\n\t" \
    "v_mfma_f32_16x16x32_f16 a[100:103], v[64:67], v[160:163], a[100:103]\n\t" \
    "v_mfma_f32_16x16x32_f16 a[68:71], v[60:63], v[160:163], a[68:71]\n\t" \
    "buffer_load_dwordx4 v[108:111], %[vA0], %[rWS], s55 offen offset:2048 sc0 sc1\n\t" \
    "v_mfma_f32_16x16x32_f16 a[68:71], v[28:31], v[156:159], a[68:71]\n\t" \
    "v_cvt_pk_f16_f32 v229, v172, v176\n\t" \
    "v_mfma_f32_16x16x32_f16 a[36:39], v[24:27], v[156:159], a[36:39]\n\t" \
    "ds_write_b64 %[vWr0], v[228:229] offset:0\n\t" \
    "v_mfma_f32_16x16x32_f16 a[36:39], v[56:59], v[160:163], a[36:39]\n\t" \
    "v_cvt_pk_f16_f32 v230, v180, v184\n\t" \
    "v_mfma_f32_16x16x32_f16 a[4:7], v[52:55], v[160:163], a[4:7]\n\t" \
    "buffer_load_dwordx4 v[112:115], %[vA0], %[rWS], s55 offen offset:3072 sc0 sc1\n\t" \
    "v_mfma_f32_16x16x32_f16 a[4:7], v[20:23], v[156:159], a[4:7]\n\t" \
    "s_nop 9\n\t" \
    "ds_read_b128 v[156:159], %[vA0] offset:35840\n\t" \
    "s_waitcnt lgkmcnt(3)\n\t" \
    "v_mfma_f32_16x16x32_f16 a[8:11], v[20:23], v[148:151], a[8:11]\n\t" \
    "v_cvt_pk_f16_f32 v231, v188, v192\n\t" \
    "s_waitcnt lgkmcnt(2)\n\t" \
    "v_mfma_f32_16x16x32_f16 a[8:11], v[52:55], v[152:155], a[8:11]\n\t" \
    "ds_write_b64 %[vWr0], v[230:231] offset:8\n\t" \
    "v_mfma_f32_16x16x32_f16 a[40:43], v[56:59], v[152:155], a[40:43]\n\t" \
    "s_add_u32 s55, s37, 24576\n\t" \
    "v_cvt_pk_f16_f32 v228, v165, v169\n\t" \
    "v_mfma_f32_16x16x32_f16 a[40:43], v[24:27], v[148:151], a[40:43]\n\t" \
    "buffer_load_dwordx4 v[116:119], %[vA0], %[rWS], s55 offen offset:0 sc0 sc1\n\t" \
    "v_mfma_f32_16x16x32_f16 a[72:75], v[28:31], v[148:151], a[72:75]\n\t" \
    "v_cvt_pk_f16_f32 v229, v173, v177\n\t" \
    "s_add_u32 s38, s38, 32768\n\t" \
    "v_mfma_f32_16x16x32_f16 a[72:75], v[60:63], v[152:155], a[72:75]\n\t" \
    "s_and_b32 s38, s38, 0x1fffff\n\t" \
    "s_or_b32 s38, s38, %[sA0]\n\t" \
    "v_mfma_f32_16x16x32_f16 a[104:107], v[64:67], v[152:155], a[104:107]\n\t" \
    "ds_write_b64 %[vWr0], v[228:229] offset:1024\n\t" \
    "v_mfma_f32_16x16x32_f16 a[104:107], v[32:35], v[148:151], a[104:107]\n\t" \
    "s_nop 9\n\t" \
    "ds_read_b128 v[160:163], %[vA0] offset:44032\n\t" \
    "v_mfma_f32_16x16x32_f16 a[136:139], v[36:39], v[148:151], a[136:139]\n\t" \
    "v_cvt_pk_f16_f32 v230, v181, v185\n\t" \
    "v_mfma_f32_16x16x32_f16 a[136:139], v[68:71], v[152:155], a[136:139]\n\t" \
    "buffer_load_dwordx4 v[120:123], %[vA0], %[rWS], s55 offen offset:1024 sc0 sc1\n\t" \
    "v_mfma_f32_16x16x32_f16 a[168:171], v[72:75], v[152:155], a[168:171]\n\t" \
    "v_cvt_pk_f16_f32 v231, v189, v193\n\t" \
    "v_mfma_f32_16x16x32_f16 a[168:171], v[40:43], v[148:151], a[168:171]\n\t" \
    "ds_write_b64 %[vWr0], v[230:231] offset:1032\n\t" \
    "v_mfma_f32_16x16x32_f16 a[200:203], v[44:47], v[148:151], a[200:203]\n\t" \
    "v_cvt_pk_f16_f32 v228, v166, v170\n\t" \
    "v_mfma_f32_16x16x32_f16 a[200:203], v[76:79], v[152:155], a[200:203]\n\t" \
    "buffer_load_dwordx4 v[124:127], %[vA0], %[rWS], s55 offen offset:2048 sc0 sc1\n\t" \
    "v_mfma_f32_16x16x32_f16 a[232:235], v[80:83], v[152:155], a[232:235]\n\t" \
    "v_cvt_pk_f16_f32 v229, v174, v178\n\t" \
    "v_mfma_f32_16x16x32_f16 a[232:235], v[48:51], v[148:151], a[232:235]\n\t" \
    "s_nop 9\n\t" \
    "ds_read_b128 v[148:151], %[vA0] offset:36864\n\t" \
    "s_waitcnt lgkmcnt(5)\n\t" \
    "v_mfma_f32_16x16x32_f16 a[236:239], v[48:51], v[156:159], a[236:239]\n\t" \
    "ds_write_b64 %[vWr0], v[228:229] offset:2048\n\t" \
    "s_waitcnt lgkmcnt(3)\n\t" \
    "v_mfma_f32_16x16x32_f16 a[236:239], v[80:83], v[160:163], a[236:239]\n\t" \
    "v_cvt_pk_f16_f32 v230, v182, v186\n\t" \
    "v_mfma_f32_16x16x32_f16 a[204:207], v[76:79], v[160:163], a[204:207]\n\t" \
    "buffer_load_dwordx4 v[128:131], %[vA0], %[rWS], s55 offen offset:3072 sc0 sc1\n\t" \
    "v_mfma_f32_16x16x32_f16 a[204:207], v[44:47], v[156:159], a[204:207]\n\t" \
    "v_cvt_pk_f16_f32 v231, v190, v194\n\t" \
    "v_mfma_f32_16x16x32_f16 a[172:175], v[40:43], v[156:159], a[172:175]\n\t" \
    "ds_write_b64 %[vWr0], v[230:231] offset:2056\n\t" \
    "v_mfma_f32_16x16x32_f16 a[172:175], v[72:75], v[160:163], a[172:175]\n\t" \
    "s_add_u32 s55, s37, 28672\n\t" \
    "v_cvt_pk_f16_f32 v228, v167, v171\n\t" \
    "v_mfma_f32_16x16x32_f16 a[140:143], v[68:71], v[160:163], a[140:143]\n\t" \
    "buffer_load_dwordx4 v[132:135], %[vA0], %[rWS], s55 offen offset:0 sc0 sc1\n\t" \
    "v_mfma_f32_16x16x32_f16 a[140:143], v[36:39], v[156:159], a[140:143]\n\t" \
    "s_nop 9\n\t" \
    "ds_read_b128 v[152:155], %[vA0] offset:45056\n\t" \
    "v_mfma_f32_16x16x32_f16 a[108:111], v[32:35], v[156:159], a[108:111]\n\t" \
    "v_cvt_pk_f16_f32 v229, v175, v179\n\t" \
    "v_mfma_f32_16x16x32_f16 a[108:111], v[64:67], v[160:163], a[108:111]\n\t" \
    "ds_write_b64 %[vWr0], v[228:229] offset:3072\n\t" \
    "v_mfma_f32_16x16x32_f16 a[76:79], v[60:63], v[160:163], a[76:79]\n\t" \
    "v_cvt_pk_f16_f32 v230, v183, v187\n\t" \
    "v_mfma_f32_16x16x32_f16 a[76:79], v[28:31], v[156:159], a[76:79]\n\t" \
    "buffer_load_dwordx4 v[136:139], %[vA0], %[rWS], s55 offen offset:1024 sc0 sc1\n\t" \
    "v_mfma_f32_16x16x32_f16 a[44:47], v[24:27], v[156:159], a[44:47]\n\t" \
    "s_add_u32 s66, s66, 0x400000\n\t" \
    "s_and_b32 s66, s66, 0xfffffff\n\t" \
    "v_mfma_f32_16x16x32_f16 a[44:47], v[56:59], v[160:163], a[44:47]\n\t" \
    "v_cvt_pk_f16_f32 v231, v191, v195\n\t" \
    "v_mfma_f32_16x16x32_f16 a[12:15], v[52:55], v[160:163], a[12:15]\n\t" \
    "ds_write_b64 %[vWr0], v[230:231] offset:3080\n\t" \
    "v_mfma_f32_16x16x32_f16 a[12:15], v[20:23], v[156:159], a[12:15]\n\t" \
    "s_nop 9\n\t" \
    "ds_read_b128 v[156:159], %[vA0] offset:37888\n\t" \
    "s_waitcnt lgkmcnt(6)\n\t" \
    "v_mfma_f32_16x16x32_f16 a[16:19], v[20:23], v[148:151], a[16:19]\n\t" \
    "buffer_load_dwordx4 v[140:143], %[vA0], %[rWS], s55 offen offset:2048 sc0 sc1\n\t" \
    "s_waitcnt lgkmcnt(3)\n\t" \
    "v_mfma_f32_16x16x32_f16 a[16:19], v[52:55], v[152:155], a[16:19]\n\t" \
    "s_and_b32 s41, s36, 7\n\t" \
    "s_cmp_gt_u32 s41, 5\n\t" \
    "v_mfma_f32_16x16x32_f16 a[48:51], v[56:59], v[152:155], a[48:51]\n\t" \
    "s_cselect_b32 s41, s44, s43\n\t" \
    "s_add_u32 s39, s66, s41\n\t" \
    "v_mfma_f32_16x16x32_f16 a[48:51], v[24:27], v[148:151], a[48:51]\n\t" \
    "s_add_u32 s39, s39, %[sWk0]\n\t" \
    "v_mfma_f32_16x16x32_f16 a[80:83], v[28:31], v[148:151], a[80:83]\n\t" \
    "buffer_load_dword v19, %[vXf], %[rWS], s34 offen sc0 sc1\n\t" \
    "v_mfma_f32_16x16x32_f16 a[80:83], v[60:63], v[152:155], a[80:83]\n\t" \
    "buffer_load_dwordx4 v[144:147], %[vA0], %[rWS], s55 offen offset:3072 sc0 sc1\n\t" \
    "v_mfma_f32_16x16x32_f16 a[112:115], v[64:67], v[152:155], a[112:115]\n\t" \
    "buffer_load_dwordx4 v[164:167], %[vW], %[rW], s39 offen\n\t" \
    "v_mfma_f32_16x16x32_f16 a[112:115], v[32:35], v[148:151], a[112:115]\n\t" \
    "s_nop 9\n\t" \
    "ds_read_b128 v[160:163], %[vA0] offset:46080\n\t" \
    "v_mfma_f32_16x16x32_f16 a[144:147], v[36:39], v[148:151], a[144:147]\n\t" \
    "s_add_u32 s39, s39, 32768\n\t" \
    "v_mfma_f32_16x16x32_f16 a[144:147], v[68:71], v[152:155], a[144:147]\n\t" \
    "buffer_load_dwordx4 v[168:171], %[vW], %[rW], s39 offen\n\t" \
    "v_mfma_f32_16x16x32_f16 a[176:179], v[72:75], v[152:155], a[176:179]\n\t" \
    "s_add_u32 s39, s39, 32768\n\t" \
    "v_mfma_f32_16x16x32_f16 a[176:179], v[40:43], v[148:151], a[176:179]\n\t" \
    "buffer_load_dwordx4 v[172:175], %[vW], %[rW], s39 offen\n\t" \
    "v_mfma_f32_16x16x32_f16 a[208:211], v[44:47], v[148:151], a[208:211]\n\t" \
    "s_add_u32 s39, s39, 32768\n\t" \
    "v_mfma_f32_16x16x32_f16 a[208:211], v[76:79], v[152:155], a[208:211]\n\t" \
    "v_mfma_f32_16x16x32_f16 a[240:243], v[80:83], v[152:155], a[240:243]\n\t" \
    "buffer_load_dwordx4 v[176:179], %[vW], %[rW], s39 offen\n\t" \
    "v_mfma_f32_16x16x32_f16 a[240:243], v[48:51], v[148:151], a[240:243]\n\t" \
    "s_nop 9\n\t" \
    "ds_read_b128 v[148:151], %[vA0] offset:38912\n\t" \
    "s_waitcnt lgkmcnt(2)\n\t" \
    "v_mfma_f32_16x16x32_f16 a[244:247], v[48:51], v[156:159], a[244:247]\n\t" \
    "s_add_u32 s39, s39, 32768\n\t" \
    "s_waitcnt lgkmcnt(1)\n\t" \
    "v_mfma_f32_16x16x32_f16 a[244:247], v[80:83], v[160:163], a[244:247]\n\t" \
    "v_mfma_f32_16x16x32_f16 a[212:215], v[76:79], v[160:163], a[212:215]\n\t" \
    "buffer_load_dwordx4 v[180:183], %[vW], %[rW], s39 offen\n\t" \
    "v_mfma_f32_16x16x32_f16 a[212:215], v[44:47], v[156:159], a[212:215]\n\t" \
    "v_mfma_f32_16x16x32_f16 a[180:183], v[40:43], v[156:159], a[180:183]\n\t" \
    "s_add_u32 s39, s39, 32768\n\t" \
    "v_mfma_f32_16x16x32_f16 a[180:183], v[72:75], v[160:163], a[180:183]\n\t" \
    "v_mfma_f32_16x16x32_f16 a[148:151], v[68:71], v[160:163], a[148:151]\n\t" \
    "buffer_load_dwordx4 v[184:187], %[vW], %[rW], s39 offen\n\t" \
    "v_mfma_f32_16x16x32_f16 a[148:151], v[36:39], v[156:159], a[148:151]\n\t" \
    "s_nop 9\n\t" \
    "ds_read_b128 v[152:155], %[vA0] offset:47104\n\t" \
    "v_mfma_f32_16x16x32_f16 a[116:119], v[32:35], v[156:159], a[116:119]\n\t" \
    "s_add_u32 s39, s39, 32768\n\t" \
    "v_mfma_f32_16x16x32_f16 a[116:119], v[64:67], v[160:163], a[116:119]\n\t" \
    "v_mfma_f32_16x16x32_f16 a[84:87], v[60:63], v[160:163], a[84:87]\n\t" \
    "buffer_load_dwordx4 v[188:191], %[vW], %[rW], s39 offen\n\t" \
    "v_mfma_f32_16x16x32_f16 a[84:87], v[28:31], v[156:159], a[84:87]\n\t" \
    "v_mfma_f32_16x16x32_f16 a[52:55], v[24:27], v[156:159], a[52:55]\n\t" \
    "s_add_u32 s39, s39, 32768\n\t" \
    "v_mfma_f32_16x16x32_f16 a[52:55], v[56:59], v[160:163], a[52:55]\n\t" \
    "v_mfma_f32_16x16x32_f16 a[20:23], v[52:55], v[160:163], a[20:23]\n\t" \
    "buffer_load_dwordx4 v[192:195], %[vW], %[rW], s39 offen\n\t" \
    "v_mfma_f32_16x16x32_f16 a[20:23], v[20:23], v[156:159], a[20:23]\n\t" \
    "s_nop 9\n\t" \
    "ds_read_b128 v[156:159], %[vA0] offset:39936\n\t" \
    "s_waitcnt lgkmcnt(2)\n\t" \
    "v_mfma_f32_16x16x32_f16 a[24:27], v[20:23], v[148:151], a[24:27]\n\t" \
    "s_waitcnt vmcnt(31)\n\t" \
    "v_cvt_pk_f16_f32 v228, v196, v200\n\t" \
    "s_waitcnt lgkmcnt(1)\n\t" \
    "v_mfma_f32_16x16x32_f16 a[24:27], v[52:55], v[152:155], a[24:27]\n\t" \
    "v_mfma_f32_16x16x32_f16 a[56:59], v[56:59], v[152:155], a[56:59]\n\t" \
    "s_waitcnt vmcnt(29)\n\t" \
    "v_cvt_pk_f16_f32 v229, v204, v208\n\t" \
    "v_mfma_f32_16x16x32_f16 a[56:59], v[24:27], v[148:151], a[56:59]\n\t" \
    "ds_write_b64 %[vWr1], v[228:229] offset:0\n\t" \
    "v_mfma_f32_16x16x32_f16 a[88:91], v[28:31], v[148:151], a[88:91]\n\t" \
    "s_waitcnt vmcnt(27)\n\t" \
    "v_cvt_pk_f16_f32 v230, v212, v216\n\t" \
    "v_mfma_f32_16x16x32_f16 a[88:91], v[60:63], v[152:155], a[88:91]\n\t" \
    "v_mfma_f32_16x16x32_f16 a[120:123], v[64:67], v[152:155], a[120:123]\n\t" \
    "s_waitcnt vmcnt(25)\n\t" \
    "v_cvt_pk_f16_f32 v231, v220, v224\n\t" \
    "v_mfma_f32_16x16x32_f16 a[120:123], v[32:35], v[148:151], a[120:123]\n\t" \
    "s_nop 9\n\t" \
    "ds_read_b128 v[160:163], %[vA0] offset:48128\n\t" \
    "v_mfma_f32_16x16x32_f16 a[152:155], v[36:39], v[148:151], a[152:155]\n\t" \
    "ds_write_b64 %[vWr1], v[230:231] offset:8\n\t" \
    "v_mfma_f32_16x16x32_f16 a[152:155], v[68:71], v[152:155], a[152:155]\n\t" \
    "v_cvt_pk_f16_f32 v228, v197, v201\n\t" \
    "v_mfma_f32_16x16x32_f16 a[184:187], v[72:75], v[152:155], a[184:187]\n\t" \
    "v_cvt_pk_f16_f32 v229, v205, v209\n\t" \
    "v_mfma_f32_16x16x32_f16 a[184:187], v[40:43], v[148:151], a[184:187]\n\t" \
    "ds_write_b64 %[vWr1], v[228:229] offset:1024\n\t" \
    "v_mfma_f32_16x16x32_f16 a[216:219], v[44:47], v[148:151], a[216:219]\n\t" \
    "v_cvt_pk_f16_f32 v230, v213, v217\n\t" \
    "v_mfma_f32_16x16x32_f16 a[216:219], v[76:79], v[152:155], a[216:219]\n\t" \
    "v_mfma_f32_16x16x32_f16 a[248:251], v[80:83], v[152:155], a[248:251]\n\t" \
    "v_cvt_pk_f16_f32 v231, v221, v225\n\t" \
    "v_mfma_f32_16x16x32_f16 a[248:251], v[48:51], v[148:151], a[248:251]\n\t" \
    "s_nop 9\n\t" \
    "ds_read_b128 v[148:151], %[vA0] offset:49152\n\t" \
    "s_waitcnt lgkmcnt(5)\n\t" \
    "v_mfma_f32_16x16x32_f16 a[252:255], v[48:51], v[156:159], a[252:255]\n\t" \
    "ds_write_b64 %[vWr1], v[230:231] offset:1032\n\t" \
    "s_waitcnt lgkmcnt(4)\n\t" \
    "v_mfma_f32_16x16x32_f16 a[252:255], v[80:83], v[160:163], a[252:255]\n\t" \
    "v_cvt_pk_f16_f32 v228, v198, v202\n\t" \
    "v_mfma_f32_16x16x32_f16 a[220:223], v[76:79], v[160:163], a[220:223]\n\t" \
    "v_cvt_pk_f16_f32 v229, v206, v210\n\t" \
    "v_mfma_f32_16x16x32_f16 a[220:223], v[44:47], v[156:159], a[220:223]\n\t" \
    "s_waitcnt vmcnt(9)\n\t" \
    "v_cmp_le_u32 s[76:77], s70, v19\n\t" \
    "v_mfma_f32_16x16x32_f16 a[188:191], v[40:43], v[156:159], a[188:191]\n\t" \
    "s_nop 1\n\t" \
    "s_cmp_eq_u64 s[76:77], exec\n\t" \
    "s_cbranch_scc0 SLOW_PQ1%=\n\t" \
    "BACK_PQ1%=:\n\t" \
    "v_mfma_f32_16x16x32_f16 a[188:191], v[72:75], v[160:163], a[188:191]\n\t" \
    "ds_write_b64 %[vWr1], v[228:229] offset:2048\n\t" \
    "v_mfma_f32_16x16x32_f16 a[156:159], v[68:71], v[160:163], a[156:159]\n\t" \
    "v_cvt_pk_f16_f32 v230, v214, v218\n\t" \
    "v_cvt_pk_f16_f32 v231, v222, v226\n\t" \
    "v_mfma_f32_16x16x32_f16 a[156:159], v[36:39], v[156:159], a[156:159]\n\t" \
    "s_nop 9\n\t" \
    "ds_read_b128 v[152:155], %[vA0] offset:57344\n\t" \
    "v_mfma_f32_16x16x32_f16 a[124:127], v[32:35], v[156:159], a[124:127]\n\t" \
    "ds_write_b64 %[vWr1], v[230:231] offset:2056\n\t" \
    "v_mfma_f32_16x16x32_f16 a[124:127], v[64:67], v[160:163], a[124:127]\n\t" \
    "v_cvt_pk_f16_f32 v228, v199, v203\n\t" \
    "v_mfma_f32_16x16x32_f16 a[92:95], v[60:63], v[160:163], a[92:95]\n\t" \
    "v_cvt_pk_f16_f32 v229, v207, v211\n\t" \
    "v_mfma_f32_16x16x32_f16 a[92:95], v[28:31], v[156:159], a[92:95]\n\t" \
    "ds_write_b64 %[vWr1], v[228:229] offset:3072\n\t" \
    "v_mfma_f32_16x16x32_f16 a[60:63], v[24:27], v[156:159], a[60:63]\n\t" \
    "v_cvt_pk_f16_f32 v230, v215, v219\n\t" \
    "v_mfma_f32_16x16x32_f16 a[60:63], v[56:59], v[160:163], a[60:63]\n\t" \
    "v_mfma_f32_16x16x32_f16 a[28:31], v[52:55], v[160:163], a[28:31]\n\t" \
    "v_cvt_pk_f16_f32 v231, v223, v227\n\t" \
    "v_mfma_f32_16x16x32_f16 a[28:31], v[20:23], v[156:159], a[28:31]\n\t" \
    "s_nop 9\n\t" \
    "ds_read_b128 v[156:159], %[vA0] offset:50176\n\t" \
    "s_waitcnt lgkmcnt(6)\n\t" \
    "v_mfma_f32_16x16x32_f16 a[0:3], v[84:87], v[148:151], a[0:3]\n\t" \
    "ds_write_b64 %[vWr1], v[230:231] offset:3080\n\t" \
    "s_waitcnt lgkmcnt(4)\n\t" \
    "v_mfma_f32_16x16x32_f16 a[0:3], v[116:119], v[152:155], a[0:3]\n\t" \
    "s_add_u32 s55, s38, 0\n\t" \
    "v_mfma_f32_16x16x32_f16 a[32:35], v[120:123], v[152:155], a[32:35]\n\t" \
    "buffer_load_dwordx4 v[20:23], %[vA0], %[rWS], s55 offen offset:0 sc0 sc1\n\t" \
    "v_mfma_f32_16x16x32_f16 a[32:35], v[88:91], v[148:151], a[32:35]\n\t" \
    "v_mfma_f32_16x16x32_f16 a[64:67], v[92:95], v[148:151], a[64:67]\n\t" \
    "v_mfma_f32_16x16x32_f16 a[64:67], v[124:127], v[152:155], a[64:67]\n\t" \
    "v_mfma_f32_16x16x32_f16 a[96:99], v[128:131], v[152:155], a[96:99]\n\t" \
    "buffer_load_dwordx4 v[24:27], %[vA0], %[rWS], s55 offen offset:1024 sc0 sc1\n\t" \
    "v_mfma_f32_16x16x32_f16 a[96:99], v[96:99], v[148:151], a[96:99]\n\t" \
    "s_nop 9\n\t" \
    "ds_read_b128 v[160:163], %[vA0] offset:58368\n\t" \
    "v_mfma_f32_16x16x32_f16 a[128:131], v[100:103], v[148:151], a[128:131]\n\t" \
    "v_mfma_f32_16x16x32_f16 a[128:131], v[132:135], v[152:155], a[128:131]\n\t" \
    "v_mfma_f32_16x16x32_f16 a[160:163], v[136:139], v[152:155], a[160:163]\n\t" \
    "buffer_load_dwordx4 v[28:31], %[vA0], %[rWS], s55 offen offset:2048 sc0 sc1\n\t" \
    "v_mfma_f32_16x16x32_f16 a[160:163], v[104:107], v[148:151], a[160:163]\n\t" \
    "v_mfma_f32_16x16x32_f16 a[192:195], v[108:111], v[148:151], a[192:195]\n\t" \
    "v_mfma_f32_16x16x32_f16 a[192:195], v[140:143], v[152:155], a[192:195]\n\t" \
    "s_waitcnt vmcnt(11)\n\t" \
    "v_mfma_f32_16x16x32_f16 a[224:227], v[144:147], v[152:155], a[224:227]\n\t" \
    "buffer_load_dwordx4 v[32:35], %[vA0], %[rWS], s55 offen offset:3072 sc0 sc1\n\t" \
    "v_mfma_f32_16x16x32_f16 a[224:227], v[112:115], v[148:151], a[224:227]\n\t" \
    "s_nop 9\n\t" \
    "ds_read_b128 v[148:151], %[vA0] offset:51200\n\t" \
    "s_waitcnt lgkmcnt(3)\n\t" \
    "v_mfma_f32_16x16x32_f16 a[228:231], v[112:115], v[156:159], a[228:231]\n\t" \
    "s_waitcnt lgkmcnt(1)\n\t" \
    "v_mfma_f32_16x16x32_f16 a[228:231], v[144:147], v[160:163], a[228:231]\n\t" \
    "s_add_u32 s55, s38, 4096\n\t" \
    "v_mfma_f32_16x16x32_f16 a[196:199], v[140:143], v[160:163], a[196:199]\n\t" \
    "v_mfma_f32_16x16x32_f16 a[196:199], v[108:111], v[156:159], a[196:199]\n\t" \
    "buffer_load_dwordx4 v[36:39], %[vA0], %[rWS], s55 offen offset:0 sc0 sc1\n\t" \
    "v_mfma_f32_16x16x32_f16 a[164:167], v[104:107], v[156:159], a[164:167]\n\t" \
    "v_mfma_f32_16x16x32_f16 a[164:167], v[136:139], v[160:163], a[164:167]\n\t" \
    "v_mfma_f32_16x16x32_f16 a[132:135], v[132:135], v[160:163], a[132:135]\n\t" \
    "v_mfma_f32_16x16x32_f16 a[132:135], v[100:103], v[156:159], a[132:135]\n\t" \
    "s_nop 9\n\t" \
    "ds_read_b128 v[152:155], %[vA0] offset:59392\n\t" \
    "v_mfma_f32_16x16x32_f16 a[100:103], v[96:99], v[156:159], a[100:103]\n\t" \
    "buffer_load_dwordx4 v[40:43], %[vA0], %[rWS], s55 offen offset:1024 sc0 sc1\n\t" \
    "v_mfma_f32_16x16x32_f16 a[100:103], v[128:131], v[160:163], a[100:103]\n\t" \
    "v_mfma_f32_16x16x32_f16 a[68:71], v[124:127], v[160:163], a[68:71]\n\t" \
    "v_mfma_f32_16x16x32_f16 a[68:71], v[92:95], v[156:159], a[68:71]\n\t" \
    "buffer_load_dwordx4 v[44:47], %[vA0], %[rWS], s55 offen offset:2048 sc0 sc1\n\t" \
    "v_mfma_f32_16x16x32_f16 a[36:39], v[88:91], v[156:159], a[36:39]\n\t" \
    "v_mfma_f32_16x16x32_f16 a[36:39], v[120:123], v[160:163], a[36:39]\n\t" \
    "v_mfma_f32_16x16x32_f16 a[4:7], v[116:119], v[160:163], a[4:7]\n\t" \
    "v_mfma_f32_16x16x32_f16 a[4:7], v[84:87], v[156:159], a[4:7]\n\t" \
    "s_nop 9\n\t" \
    "ds_read_b128 v[156:159], %[vA0] offset:52224\n\t" \
    "s_waitcnt lgkmcnt(2)\n\t" \
    "v_mfma_f32_16x16x32_f16 a[8:11], v[84:87], v[148:151], a[8:11]\n\t" \
    "buffer_load_dwordx4 v[48:51], %[vA0], %[rWS], s55 offen offset:3072 sc0 sc1\n\t" \
    "s_waitcnt lgkmcnt(1)\n\t" \
    "v_mfma_f32_16x16x32_f16 a[8:11], v[116:119], v[152:155], a[8:11]\n\t" \
    "v_mfma_f32_16x16x32_f16 a[40:43], v[120:123], v[152:155], a[40:43]\n\t" \
    "s_add_u32 s55, s38, 8192\n\t" \
    "v_mfma_f32_16x16x32_f16 a[40:43], v[88:91], v[148:151], a[40:43]\n\t" \
    "v_mfma_f32_16x16x32_f16 a[72:75], v[92:95], v[148:151], a[72:75]\n\t" \
    "buffer_load_dwordx4 v[52:55], %[vA0], %[rWS], s55 offen offset:0 sc0 sc1\n\t" \
    "v_mfma_f32_16x16x32_f16 a[72:75], v[124:127], v[152:155], a[72:75]\n\t" \
    "v_mfma_f32_16x16x32_f16 a[104:107], v[128:131], v[152:155], a[104:107]\n\t" \
    "v_mfma_f32_16x16x32_f16 a[104:107], v[96:99], v[148:151], a[104:107]\n\t" \
    "s_nop 9\n\t" \
    "ds_read_b128 v[160:163], %[vA0] offset:60416\n\t" \
    "v_mfma_f32_16x16x32_f16 a[136:139], v[100:103], v[148:151], a[136:139]\n\t" \
    "buffer_load_dwordx4 v[56:59], %[vA0], %[rWS], s55 offen offset:1024 sc0 sc1\n\t" \
    "v_mfma_f32_16x16x32_f16 a[136:139], v[132:135], v[152:155], a[136:139]\n\t" \
    "v_mfma_f32_16x16x32_f16 a[168:171], v[136:139], v[152:155], a[168:171]\n\t" \
    "v_mfma_f32_16x16x32_f16 a[168:171], v[104:107], v[148:151], a[168:171]\n\t" \
    "v_mfma_f32_16x16x32_f16 a[200:203], v[108:111], v[148:151], a[200:203]\n\t" \
    "buffer_load_dwordx4 v[60:63], %[vA0], %[rWS], s55 offen offset:2048 sc0 sc1\n\t" \
    "v_mfma_f32_16x16x32_f16 a[200:203], v[140:143], v[152:155], a[200:203]\n\t" \
    "v_mfma_f32_16x16x32_f16 a[232:235], v[144:147], v[152:155], a[232:235]\n\t" \
    "v_mfma_f32_16x16x32_f16 a[232:235], v[112:115], v[148:151], a[232:235]\n\t" \
    "s_nop 9\n\t" \
    "ds_read_b128 v[148:151], %[vA0] offset:53248\n\t" \
    "s_waitcnt lgkmcnt(2)\n\t" \
    "v_mfma_f32_16x16x32_f16 a[236:239], v[112:115], v[156:159], a[236:239]\n\t" \
    "buffer_load_dwordx4 v[64:67], %[vA0], %[rWS], s55 offen offset:3072 sc0 sc1\n\t" \
    "s_waitcnt lgkmcnt(1)\n\t" \
    "v_mfma_f32_16x16x32_f16 a[236:239], v[144:147], v[160:163], a[236:239]\n\t" \
    "v_mfma_f32_16x16x32_f16 a[204:207], v[140:143], v[160:163], a[204:207]\n\t" \
    "v_mfma_f32_16x16x32_f16 a[204:207], v[108:111], v[156:159], a[204:207]\n\t" \
    "s_add_u32 s55, s38, 12288\n\t" \
    "v_mfma_f32_16x16x32_f16 a[172:175], v[104:107], v[156:159], a[172:175]\n\t" \
    "v_mfma_f32_16x16x32_f16 a[172:175], v[136:139], v[160:163], a[172:175]\n\t" \
    "buffer_load_dwordx4 v[68:71], %[vA0], %[rWS], s55 offen offset:0 sc0 sc1\n\t" \
    "v_mfma_f32_16x16x32_f16 a[140:143], v[132:135], v[160:163], a[140:143]\n\t" \
    "v_mfma_f32_16x16x32_f16 a[140:143], v[100:103], v[156:159], a[140:143]\n\t" \
    "s_nop 9\n\t" \
    "ds_read_b128 v[152:155], %[vA0] offset:61440\n\t" \
    "v_mfma_f32_16x16x32_f16 a[108:111], v[96:99], v[156:159], a[108:111]\n\t" \
    "v_mfma_f32_16x16x32_f16 a[108:111], v[128:131], v[160:163], a[108:111]\n\t" \
    "buffer_load_dwordx4 v[72:75], %[vA0], %[rWS], s55 offen offset:1024 sc0 sc1\n\t" \
    "v_mfma_f32_16x16x32_f16 a[76:79], v[124:127], v[160:163], a[76:79]\n\t" \
    "v_mfma_f32_16x16x32_f16 a[76:79], v[92:95], v[156:159], a[76:79]\n\t" \
    "v_mfma_f32_16x16x32_f16 a[44:47], v[88:91], v[156:159], a[44:47]\n\t" \
    "v_mfma_f32_16x16x32_f16 a[44:47], v[120:123], v[160:163], a[44:47]\n\t" \
    "buffer_load_dwordx4 v[76:79], %[vA0], %[rWS], s55 offen offset:2048 sc0 sc1\n\t" \
    "v_mfma_f32_16x16x32_f16 a[12:15], v[116:119], v[160:163], a[12:15]\n\t" \
    "s_and_b32 s41, s36, 7\n\t" \
    "s_cmp_gt_u32 s41, 5\n\t" \
    "v_mfma_f32_16x16x32_f16 a[12:15], v[84:87], v[156:159], a[12:15]\n\t" \
    "s_nop 9\n\t" \
    "ds_read_b128 v[156:159], %[vA0] offset:54272\n\t" \
    "s_waitcnt lgkmcnt(2)\n\t" \
    "v_mfma_f32_16x16x32_f16 a[16:19], v[84:87], v[148:151], a[16:19]\n\t" \
    "s_cselect_b32 s41, s44, s43\n\t" \
    "s_add_u32 s40, s66, s41\n\t" \
    "s_waitcnt lgkmcnt(1)\n\t" \
    "v_mfma_f32_16x16x32_f16 a[16:19], v[116:119], v[152:155], a[16:19]\n\t" \
    "s_add_u32 s40, s40, %[sWk1]\n\t" \
    "v_mfma_f32_16x16x32_f16 a[48:51], v[120:123], v[152:155], a[48:51]\n\t" \
    "buffer_load_dwordx4 v[80:83], %[vA0], %[rWS], s55 offen offset:3072 sc0 sc1\n\t" \
    "v_mfma_f32_16x16x32_f16 a[48:51], v[88:91], v[148:151], a[48:51]\n\t" \
    "v_mfma_f32_16x16x32_f16 a[80:83], v[92:95], v[148:151], a[80:83]\n\t" \
    "v_mfma_f32_16x16x32_f16 a[80:83], v[124:127], v[152:155], a[80:83]\n\t" \
    "v_mfma_f32_16x16x32_f16 a[112:115], v[128:131], v[152:155], a[112:115]\n\t" \
    "buffer_load_dwordx4 v[196:199], %[vW], %[rW], s40 offen nt\n\t" \
    "v_mfma_f32_16x16x32_f16 a[112:115], v[96:99], v[148:151], a[112:115]\n\t" \
    "s_nop 9\n\t" \
    "ds_read_b128 v[160:163], %[vA0] offset:62464\n\t" \
    "v_mfma_f32_16x16x32_f16 a[144:147], v[100:103], v[148:151], a[144:147]\n\t" \
    "v_mfma_f32_16x16x32_f16 a[144:147], v[132:135], v[152:155], a[144:147]\n\t" \
    "v_mfma_f32_16x16x32_f16 a[176:179], v[136:139], v[152:155], a[176:179]\n\t" \
    "s_add_u32 s40, s40, 32768\n\t" \
    "v_mfma_f32_16x16x32_f16 a[176:179], v[104:107], v[148:151], a[176:179]\n\t" \
    "v_mfma_f32_16x16x32_f16 a[208:211], v[108:111], v[148:151], a[208:211]\n\t" \
    "buffer_load_dwordx4 v[200:203], %[vW], %[rW], s40 offen nt\n\t" \
    "v_mfma_f32_16x16x32_f16 a[208:211], v[140:143], v[152:155], a[208:211]\n\t" \
    "v_mfma_f32_16x16x32_f16 a[240:243], v[144:147], v[152:155], a[240:243]\n\t" \
    "v_mfma_f32_16x16x32_f16 a[240:243], v[112:115], v[148:151], a[240:243]\n\t" \
    "s_nop 9\n\t" \
    "ds_read_b128 v[148:151], %[vA0] offset:55296\n\t" \
    "s_waitcnt lgkmcnt(2)\n\t" \
    "v_mfma_f32_16x16x32_f16 a[244:247], v[112:115], v[156:159], a[244:247]\n\t" \
    "s_add_u32 s40, s40, 32768\n\t" \
    "s_waitcnt lgkmcnt(1)\n\t" \
    "v_mfma_f32_16x16x32_f16 a[244:247], v[144:147], v[160:163], a[244:247]\n\t" \
    "v_mfma_f32_16x16x32_f16 a[212:215], v[140:143], v[160:163], a[212:215]\n\t" \
    "buffer_load_dwordx4 v[204:207], %[vW], %[rW], s40 offen nt\n\t" \
    "v_mfma_f32_16x16x32_f16 a[212:215], v[108:111], v[156:159], a[212:215]\n\t" \
    "v_mfma_f32_16x16x32_f16 a[180:183], v[104:107], v[156:159], a[180:183]\n\t" \
    "v_mfma_f32_16x16x32_f16 a[180:183], v[136:139], v[160:163], a[180:183]\n\t" \
    "v_mfma_f32_16x16x32_f16 a[148:151], v[132:135], v[160:163], a[148:151]\n\t" \
    "s_add_u32 s40, s40, 32768\n\t" \
    "v_mfma_f32_16x16x32_f16 a[148:151], v[100:103], v[156:159], a[148:151]\n\t" \
    "s_nop 9\n\t" \
    "ds_read_b128 v[152:155], %[vA0] offset:63488\n\t" \
    "v_mfma_f32_16x16x32_f16 a[116:119], v[96:99], v[156:159], a[116:119]\n\t" \
    "buffer_load_dwordx4 v[208:211], %[vW], %[rW], s40 offen nt\n\t" \
    "v_mfma_f32_16x16x32_f16 a[116:119], v[128:131], v[160:163], a[116:119]\n\t" \
    "v_mfma_f32_16x16x32_f16 a[84:87], v[124:127], v[160:163], a[84:87]\n\t" \
    "v_mfma_f32_16x16x32_f16 a[84:87], v[92:95], v[156:159], a[84:87]\n\t" \
    "v_mfma_f32_16x16x32_f16 a[52:55], v[88:91], v[156:159], a[52:55]\n\t" \
    "s_add_u32 s40, s40, 32768\n\t" \
    "v_mfma_f32_16x16x32_f16 a[52:55], v[120:123], v[160:163], a[52:55]\n\t" \
    "v_mfma_f32_16x16x32_f16 a[20:23], v[116:119], v[160:163], a[20:23]\n\t" \
    "buffer_load_dwordx4 v[212:215], %[vW], %[rW], s40 offen nt\n\t" \
    "v_mfma_f32_16x16x32_f16 a[20:23], v[84:87], v[156:159], a[20:23]\n\t" \
    "s_nop 9\n\t" \
    "ds_read_b128 v[156:159], %[vA0] offset:56320\n\t" \
    "s_waitcnt lgkmcnt(2)\n\t" \
    "v_mfma_f32_16x16x32_f16 a[24:27], v[84:87], v[148:151], a[24:27]\n\t" \
    "s_waitcnt lgkmcnt(1)\n\t" \
    "v_mfma_f32_16x16x32_f16 a[24:27], v[116:119], v[152:155], a[24:27]\n\t" \
    "v_mfma_f32_16x16x32_f16 a[56:59], v[120:123], v[152:155], a[56:59]\n\t" \
    "s_add_u32 s40, s40, 32768\n\t" \
    "v_mfma_f32_16x16x32_f16 a[56:59], v[88:91], v[148:151], a[56:59]\n\t" \
    "v_mfma_f32_16x16x32_f16 a[88:91], v[92:95], v[148:151], a[88:91]\n\t" \
    "buffer_load_dwordx4 v[216:219], %[vW], %[rW], s40 offen nt\n\t" \
    "v_mfma_f32_16x16x32_f16 a[88:91], v[124:127], v[152:155], a[88:91]\n\t" \
    "v_mfma_f32_16x16x32_f16 a[120:123], v[128:131], v[152:155], a[120:123]\n\t" \
    "v_mfma_f32_16x16x32_f16 a[120:123], v[96:99], v[148:151], a[120:123]\n\t" \
    "s_nop 9\n\t" \
    "ds_read_b128 v[160:163], %[vA0] offset:64512\n\t" \
    "v_mfma_f32_16x16x32_f16 a[152:155], v[100:103], v[148:151], a[152:155]\n\t" \
    "s_add_u32 s40, s40, 32768\n\t" \
    "v_mfma_f32_16x16x32_f16 a[152:155], v[132:135], v[152:155], a[152:155]\n\t" \
    "v_mfma_f32_16x16x32_f16 a[184:187], v[136:139], v[152:155], a[184:187]\n\t" \
    "buffer_load_dwordx4 v[220:223], %[vW], %[rW], s40 offen nt\n\t" \
    "v_mfma_f32_16x16x32_f16 a[184:187], v[104:107], v[148:151], a[184:187]\n\t" \
    "v_mfma_f32_16x16x32_f16 a[216:219], v[108:111], v[148:151], a[216:219]\n\t" \
    "v_mfma_f32_16x16x32_f16 a[216:219], v[140:143], v[152:155], a[216:219]\n\t" \
    "v_mfma_f32_16x16x32_f16 a[248:251], v[144:147], v[152:155], a[248:251]\n\t" \
    "s_add_u32 s40, s40, 32768\n\t" \
    "v_mfma_f32_16x16x32_f16 a[248:251], v[112:115], v[148:151], a[248:251]\n\t" \
    "s_nop 9\n\t" \
    "s_waitcnt lgkmcnt(0)\n\t" \
    "s_barrier\n\t" \
    "ds_read_b128 v[148:151], %[vA0] offset:0\n\t" \
    "v_mfma_f32_16x16x32_f16 a[252:255], v[112:115], v[156:159], a[252:255]\n\t" \
    "buffer_load_dwordx4 v[224:227], %[vW], %[rW], s40 offen nt\n\t" \
    "v_mfma_f32_16x16x32_f16 a[252:255], v[144:147], v[160:163], a[252:255]\n\t" \
    "v_mfma_f32_16x16x32_f16 a[220:223], v[140:143], v[160:163], a[220:223]\n\t" \
    "v_mfma_f32_16x16x32_f16 a[220:223], v[108:111], v[156:159], a[220:223]\n\t" \
    "v_mfma_f32_16x16x32_f16 a[188:191], v[104:107], v[156:159], a[188:191]\n\t" \
    "v_mfma_f32_16x16x32_f16 a[188:191], v[136:139], v[160:163], a[188:191]\n\t" \
    "v_mfma_f32_16x16x32_f16 a[156:159], v[132:135], v[160:163], a[156:159]\n\t" \
    "v_mfma_f32_16x16x32_f16 a[156:159], v[100:103], v[156:159], a[156:159]\n\t" \
    "s_nop 9\n\t" \
    "ds_read_b128 v[152:155], %[vA0] offset:8192\n\t" \
    "v_mfma_f32_16x16x32_f16 a[124:127], v[96:99], v[156:159], a[124:127]\n\t" \
    "v_mfma_f32_16x16x32_f16 a[124:127], v[128:131], v[160:163], a[124:127]\n\t" \
    "v_mfma_f32_16x16x32_f16 a[92:95], v[124:127], v[160:163], a[92:95]\n\t" \
    "v_mfma_f32_16x16x32_f16 a[92:95], v[92:95], v[156:159], a[92:95]\n\t" \
    "v_mfma_f32_16x16x32_f16 a[60:63], v[88:91], v[156:159], a[60:63]\n\t" \
    "v_mfma_f32_16x16x32_f16 a[60:63], v[120:123], v[160:163], a[60:63]\n\t" \
    "v_mfma_f32_16x16x32_f16 a[28:31], v[116:119], v[160:163], a[28:31]\n\t" \
    "v_mfma_f32_16x16x32_f16 a[28:31], v[84:87], v[156:159], a[28:31]\n\t" \
    "s_nop 9\n\t" \
    "s_add_u32 s36, s36, 1\n\t" \
    "s_add_u32 s34, s34, 8192\n\t" \
    "s_min_u32 s34, s34, 24576\n\t" \
    "s_and_b32 s41, s36, 7\n\t" \
    "s_cbranch_scc0 Q0END%=\n\t" \
    "s_bitcmp1_b32 s67, s41\n\t" \
    "s_cbranch_scc1 LOOP1%=\n\t" \
    "s_branch LOOP0%=\n\t" \
    "Q0END%=:\n\t" \
    "s_add_u32 s55, s38, 16384\n\t" \
    "buffer_load_dwordx4 v[84:87], %[vA0], %[rWS], s55 offen offset:0 sc0 sc1\n\t" \
    "buffer_load_dwordx4 v[88:91], %[vA0], %[rWS], s55 offen offset:1024 sc0 sc1\n\t" \
    "buffer_load_dwordx4 v[92:95], %[vA0], %[rWS], s55 offen offset:2048 sc0 sc1\n\t" \
    "buffer_load_dwordx4 v[96:99], %[vA0], %[rWS], s55 offen offset:3072 sc0 sc1\n\t" \
    "s_waitcnt vmcnt(4) lgkmcnt(0)\n\t" \
    "ds_read_b128 v[152:155], %[vA0] offset:1024\n\t" \
    "ds_read_b128 v[156:159], %[vA0] offset:2048\n\t" \
    "LOOPQ%=:\n\t" \
    "s_mov_b32 s43, s44\n\t" \
    "s_mov_b32 s44, s45\n\t" \
    "s_mov_b32 s45, s46\n\t" \
    "s_mov_b32 s46, 1073741824\n\t" \
    "s_cmp_eq_u32 s48, 3\n\t" \
    "s_cselect_b32 s47, 1073741824, 0\n\t" \
    "s_cmp_ge_u32 s48, 2\n\t" \
    "s_cselect_b32 s68, %[own], 9\n\t" \
    "s_cselect_b32 s70, %[ign], 0\n\t" \
    "s_mov_b32 s71, s70\n\t" \
    "s_cmp_eq_u32 s48, 2\n\t" \
    "s_cbranch_scc1 FLUSH0%=\n\t" \
    "s_nop 7\n\t" \
    "s_add_u32 s41, s50, 0\n\t" \
    "s_add_u32 s60, %[wsLo], s41\n\t" \
    "s_addc_u32 s61, %[wsHi], 0\n\t" \
    "s_cmp_eq_u32 s68, 0\n\t" \
    "s_cbranch_scc1 FO1_0%=\n\t" \
    "v_accvgpr_read_b32 v236, a16\n\t" \
    "v_accvgpr_read_b32 v237, a20\n\t" \
    "v_accvgpr_read_b32 v238, a24\n\t" \
    "v_accvgpr_read_b32 v239, a28\n\t" \
    "global_store_dwordx4 %[vA0], v[236:239], s[60:61] offset:0 sc0 sc1\n\t" \
    "ds_write_b128 %[vStg], a[48:51] offset:0\n\t" \
    "v_accvgpr_read_b32 v240, a17\n\t" \
    "v_accvgpr_read_b32 v241, a21\n\t" \
    "v_accvgpr_read_b32 v242, a25\n\t" \
    "v_accvgpr_read_b32 v243, a29\n\t" \
    "global_store_dwordx4 %[vA0], v[240:243], s[60:61] offset:1024 sc0 sc1\n\t" \
    "ds_write_b128 %[vStg], a[52:55] offset:1024\n\t" \
    "v_accvgpr_read_b32 v244, a18\n\t" \
    "v_accvgpr_read_b32 v245, a22\n\t" \
    "v_accvgpr_read_b32 v246, a26\n\t" \
    "v_accvgpr_read_b32 v247, a30\n\t" \
    "global_store_dwordx4 %[vA0], v[244:247], s[60:61] offset:2048 sc0 sc1\n\t" \
    "ds_write_b128 %[vStg], a[56:59] offset:2048\n\t" \
    "v_accvgpr_read_b32 v248, a19\n\t" \
    "v_accvgpr_read_b32 v249, a23\n\t" \
    "v_accvgpr_read_b32 v250, a27\n\t" \
    "v_accvgpr_read_b32 v251, a31\n\t" \
    "global_store_dwordx4 %[vA0], v[248:251], s[60:61] offset:3072 sc0 sc1\n\t" \
    "ds_write_b128 %[vStg], a[60:63] offset:3072\n\t" \
    "FB1_0%=:\n\t" \
    "s_add_u32 s41, s50, 8192\n\t" \
    "s_add_u32 s60, %[wsLo], s41\n\t" \
    "s_addc_u32 s61, %[wsHi], 0\n\t" \
    "s_cmp_eq_u32 s68, 1\n\t" \
    "s_cbranch_scc1 FO1_2%=\n\t" \
    "v_accvgpr_read_b32 v236, a80\n\t" \
    "v_accvgpr_read_b32 v237, a84\n\t" \
    "v_accvgpr_read_b32 v238, a88\n\t" \
    "v_accvgpr_read_b32 v239, a92\n\t" \
    "global_store_dwordx4 %[vA0], v[236:239], s[60:61] offset:0 sc0 sc1\n\t" \
    "ds_write_b128 %[vStg], a[112:115] offset:4096\n\t" \
    "v_accvgpr_read_b32 v240, a81\n\t" \
    "v_accvgpr_read_b32 v241, a85\n\t" \
    "v_accvgpr_read_b32 v242, a89\n\t" \
    "v_accvgpr_read_b32 v243, a93\n\t" \
    "global_store_dwordx4 %[vA0], v[240:243], s[60:61] offset:1024 sc0 sc1\n\t" \
    "ds_write_b128 %[vStg], a[116:119] offset:5120\n\t" \
    "v_accvgpr_read_b32 v244, a82\n\t" \
    "v_accvgpr_read_b32 v245, a86\n\t" \
    "v_accvgpr_read_b32 v246, a90\n\t" \
    "v_accvgpr_read_b32 v247, a94\n\t" \
    "global_store_dwordx4 %[vA0], v[244:247], s[60:61] offset:2048 sc0 sc1\n\t" \
    "ds_write_b128 %[vStg], a[120:123] offset:6144\n\t" \
    "v_accvgpr_read_b32 v248, a83\n\t" \
    "v_accvgpr_read_b32 v249, a87\n\t" \
    "v_accvgpr_read_b32 v250, a91\n\t" \
    "v_accvgpr_read_b32 v251, a95\n\t" \
    "global_store_dwordx4 %[vA0], v[248:251], s[60:61] offset:3072 sc0 sc1\n\t" \
    "ds_write_b128 %[vStg], a[124:127] offset:7168\n\t" \
    "FB1_2%=:\n\t" \
    "s_add_u32 s41, s50, 16384\n\t" \
    "s_add_u32 s60, %[wsLo], s41\n\t" \
    "s_addc_u32 s61, %[wsHi], 0\n\t" \
    "s_cmp_eq_u32 s68, 2\n\t" \
    "s_cbranch_scc1 FO1_4%=\n\t" \
    "v_accvgpr_read_b32 v236, a144\n\t" \
    "v_accvgpr_read_b32 v237, a148\n\t" \
    "v_accvgpr_read_b32 v238, a152\n\t" \
    "v_accvgpr_read_b32 v239, a156\n\t" \
    "global_store_dwordx4 %[vA0], v[236:239], s[60:61] offset:0 sc0 sc1\n\t" \
    "ds_write_b128 %[vStg], a[176:179] offset:8192\n\t" \
    "v_accvgpr_read_b32 v240, a145\n\t" \
    "v_accvgpr_read_b32 v241, a149\n\t" \
    "v_accvgpr_read_b32 v242, a153\n\t" \
    "v_accvgpr_read_b32 v243, a157\n\t" \
    "global_store_dwordx4 %[vA0], v[240:243], s[60:61] offset:1024 sc0 sc1\n\t" \
    "ds_write_b128 %[vStg], a[180:183] offset:9216\n\t" \
    "v_accvgpr_read_b32 v244, a146\n\t" \
    "v_accvgpr_read_b32 v245, a150\n\t" \
    "v_accvgpr_read_b32 v246, a154\n\t" \
    "v_accvgpr_read_b32 v247, a158\n\t" \
    "global_store_dwordx4 %[vA0], v[244:247], s[60:61] offset:2048 sc0 sc1\n\t" \
    "ds_write_b128 %[vStg], a[184:187] offset:10240\n\t" \
    "v_accvgpr_read_b32 v248, a147\n\t" \
    "v_accvgpr_read_b32 v249, a151\n\t" \
    "v_accvgpr_read_b32 v250, a155\n\t" \
    "v_accvgpr_read_b32 v251, a159\n\t" \
    "global_store_dwordx4 %[vA0], v[248:251], s[60:61] offset:3072 sc0 sc1\n\t" \
    "ds_write_b128 %[vStg], a[188:191] offset:11264\n\t" \
    "FB1_4%=:\n\t" \
    "s_add_u32 s41, s50, 24576\n\t" \
    "s_add_u32 s60, %[wsLo], s41\n\t" \
    "s_addc_u32 s61, %[wsHi], 0\n\t" \
    "s_cmp_eq_u32 s68, 3\n\t" \
    "s_cbranch_scc1 FO1_6%=\n\t" \
    "v_accvgpr_read_b32 v236, a208\n\t" \
    "v_accvgpr_read_b32 v237, a212\n\t" \
    "v_accvgpr_read_b32 v238, a216\n\t" \
    "v_accvgpr_read_b32 v239, a220\n\t" \
    "global_store_dwordx4 %[vA0], v[236:239], s[60:61] offset:0 sc0 sc1\n\t" \
    "ds_write_b128 %[vStg], a[240:243] offset:12288\n\t" \
    "v_accvgpr_read_b32 v240, a209\n\t" \
    "v_accvgpr_read_b32 v241, a213\n\t" \
    "v_accvgpr_read_b32 v242, a217\n\t" \
    "v_accvgpr_read_b32 v243, a221\n\t" \
    "global_store_dwordx4 %[vA0], v[240:243], s[60:61] offset:1024 sc0 sc1\n\t" \
    "ds_write_b128 %[vStg], a[244:247] offset:13312\n\t" \
    "v_accvgpr_read_b32 v244, a210\n\t" \
    "v_accvgpr_read_b32 v245, a214\n\t" \
    "v_accvgpr_read_b32 v246, a218\n\t" \
    "v_accvgpr_read_b32 v247, a222\n\t" \
    "global_store_dwordx4 %[vA0], v[244:247], s[60:61] offset:2048 sc0 sc1\n\t" \
    "ds_write_b128 %[vStg], a[248:251] offset:14336\n\t" \
    "v_accvgpr_read_b32 v248, a211\n\t" \
    "v_accvgpr_read_b32 v249, a215\n\t" \
    "v_accvgpr_read_b32 v250, a219\n\t" \
    "v_accvgpr_read_b32 v251, a223\n\t" \
    "global_store_dwordx4 %[vA0], v[248:251], s[60:61] offset:3072 sc0 sc1\n\t" \
    "ds_write_b128 %[vStg], a[252:255] offset:15360\n\t" \
    "FB1_6%=:\n\t" \
    "s_nop 1\n\t" \
    "s_nop 1\n\t" \
    "ds_read_b128 v[160:163], %[vA0] offset:3072\n\t" \
    "s_waitcnt vmcnt(32) lgkmcnt(3)\n\t" \
    "v_mfma_f32_16x16x32_f16 a[224:227], v[48:51], v[148:151], a[224:227]\n\t" \
    "s_mov_b32 s37, s38\n\t" \
    "v_cvt_pk_f16_f32 v228, v164, v168\n\t" \
    "v_mfma_f32_16x16x32_f16 a[192:195], v[44:47], v[148:151], a[192:195]\n\t" \
    "v_mfma_f32_16x16x32_f16 a[160:163], v[40:43], v[148:151], a[160:163]\n\t" \
    "v_cvt_pk_f16_f32 v229, v172, v176\n\t" \
    "v_mfma_f32_16x16x32_f16 a[128:131], v[36:39], v[148:151], a[128:131]\n\t" \
    "ds_write_b64 %[vWr0], v[228:229] offset:32768\n\t" \
    "v_mfma_f32_16x16x32_f16 a[96:99], v[32:35], v[148:151], a[96:99]\n\t" \
    "v_cvt_pk_f16_f32 v230, v180, v184\n\t" \
    "v_mfma_f32_16x16x32_f16 a[64:67], v[28:31], v[148:151], a[64:67]\n\t" \
    "v_mfma_f32_16x16x32_f16 a[32:35], v[24:27], v[148:151], a[32:35]\n\t" \
    "v_cvt_pk_f16_f32 v231, v188, v192\n\t" \
    "v_mfma_f32_16x16x32_f16 a[0:3], v[20:23], v[148:151], a[0:3]\n\t" \
    "ds_read_b128 v[148:151], %[vA0] offset:4096\n\t" \
    "s_waitcnt lgkmcnt(4)\n\t" \
    "v_mfma_f32_16x16x32_f16 a[4:7], v[20:23], v[152:155], a[4:7]\n\t" \
    "s_cmp_ge_u32 s48, 2\n\ts_cselect_b64 exec, -1, 0\n\ts_cselect_b32 s69, %[own], 9\n\tds_read_b128 v[252:255], %[vStg] offset:16384\n\ts_mov_b64 exec, -1\n\t" \
    "s_waitcnt lgkmcnt(4)\n\t" \
    "v_mfma_f32_16x16x32_f16 a[36:39], v[24:27], v[152:155], a[36:39]\n\t" \
    "ds_write_b64 %[vWr0], v[230:231] offset:32776\n\t" \
    "v_mfma_f32_16x16x32_f16 a[68:71], v[28:31], v[152:155], a[68:71]\n\t" \
    "v_cvt_pk_f16_f32 v228, v165, v169\n\t" \
    "s_add_u32 s55, s37, 20480\n\t" \
    "v_mfma_f32_16x16x32_f16 a[100:103], v[32:35], v[152:155], a[100:103]\n\t" \
    "v_cvt_pk_f16_f32 v229, v173, v177\n\t" \
    "v_mfma_f32_16x16x32_f16 a[132:135], v[36:39], v[152:155], a[132:135]\n\t" \
    "buffer_load_dwordx4 v[100:103], %[vA0], %[rWS], s55 offen offset:0 sc0 sc1\n\t" \
    "v_mfma_f32_16x16x32_f16 a[164:167], v[40:43], v[152:155], a[164:167]\n\t" \
    "ds_write_b64 %[vWr0], v[228:229] offset:33792\n\t" \
    "v_mfma_f32_16x16x32_f16 a[196:199], v[44:47], v[152:155], a[196:199]\n\t" \
    "v_cvt_pk_f16_f32 v230, v181, v185\n\t" \
    "v_cvt_pk_f16_f32 v231, v189, v193\n\t" \
    "v_mfma_f32_16x16x32_f16 a[228:231], v[48:51], v[152:155], a[228:231]\n\t" \
    "ds_read_b128 v[152:155], %[vA0] offset:5120\n\t" \
    "s_waitcnt lgkmcnt(6)\n\t" \
    "v_mfma_f32_16x16x32_f16 a[232:235], v[48:51], v[156:159], a[232:235]\n\t" \
    "ds_write_b64 %[vWr0], v[230:231] offset:33800\n\t" \
    "v_mfma_f32_16x16x32_f16 a[200:203], v[44:47], v[156:159], a[200:203]\n\t" \
    "buffer_load_dwordx4 v[104:107], %[vA0], %[rWS], s55 offen offset:1024 sc0 sc1\n\t" \
    "v_mfma_f32_16x16x32_f16 a[168:171], v[40:43], v[156:159], a[168:171]\n\t" \
    "v_cvt_pk_f16_f32 v228, v166, v170\n\t" \
    "v_cvt_pk_f16_f32 v229, v174, v178\n\t" \
    "v_mfma_f32_16x16x32_f16 a[136:139], v[36:39], v[156:159], a[136:139]\n\t" \
    "ds_write_b64 %[vWr0], v[228:229] offset:34816\n\t" \
    "v_mfma_f32_16x16x32_f16 a[104:107], v[32:35], v[156:159], a[104:107]\n\t" \
    "v_cvt_pk_f16_f32 v230, v182, v186\n\t" \
    "v_mfma_f32_16x16x32_f16 a[72:75], v[28:31], v[156:159], a[72:75]\n\t" \
    "buffer_load_dwordx4 v[108:111], %[vA0], %[rWS], s55 offen offset:2048 sc0 sc1\n\t" \
    "v_mfma_f32_16x16x32_f16 a[40:43], v[24:27], v[156:159], a[40:43]\n\t" \
    "s_add_u32 s66, s66, 0x400000\n\t" \
    "s_and_b32 s66, s66, 0xfffffff\n\t" \
    "v_mfma_f32_16x16x32_f16 a[8:11], v[20:23], v[156:159], a[8:11]\n\t" \
    "ds_read_b128 v[156:159], %[vA0] offset:6144\n\t" \
    "s_waitcnt lgkmcnt(8)\n\t" \
    "v_mfma_f32_16x16x32_f16 a[12:15], v[20:23], v[160:163], a[12:15]\n\t" \
    "v_cvt_pk_f16_f32 v231, v190, v194\n\t" \
    "v_mfma_f32_16x16x32_f16 a[44:47], v[24:27], v[160:163], a[44:47]\n\t" \
    "ds_write_b64 %[vWr0], v[230:231] offset:34824\n\t" \
    "v_mfma_f32_16x16x32_f16 a[76:79], v[28:31], v[160:163], a[76:79]\n\t" \
    "v_cvt_pk_f16_f32 v228, v167, v171\n\t" \
    "s_add_u32 s39, s66, s43\n\t" \
    "v_mfma_f32_16x16x32_f16 a[108:111], v[32:35], v[160:163], a[108:111]\n\t" \
    "s_add_u32 s39, s39, %[sWk0]\n\t" \
    "v_mfma_f32_16x16x32_f16 a[140:143], v[36:39], v[160:163], a[140:143]\n\t" \
    "buffer_load_dwordx4 v[112:115], %[vA0], %[rWS], s55 offen offset:3072 sc0 sc1\n\t" \
    "v_mfma_f32_16x16x32_f16 a[172:175], v[40:43], v[160:163], a[172:175]\n\t" \
    "v_cvt_pk_f16_f32 v229, v175, v179\n\t" \
    "v_mfma_f32_16x16x32_f16 a[204:207], v[44:47], v[160:163], a[204:207]\n\t" \
    "ds_write_b64 %[vWr0], v[228:229] offset:35840\n\t" \
    "v_mfma_f32_16x16x32_f16 a[236:239], v[48:51], v[160:163], a[236:239]\n\t" \
    "ds_read_b128 v[160:163], %[vA0] offset:7168\n\t" \
    "s_waitcnt lgkmcnt(9)\n\t" \
    "v_mfma_f32_16x16x32_f16 a[240:243], v[48:51], v[148:151], 0\n\t" \
    "v_cvt_pk_f16_f32 v230, v183, v187\n\t" \
    "v_cvt_pk_f16_f32 v231, v191, v195\n\t" \
    "v_mfma_f32_16x16x32_f16 a[208:211], v[44:47], v[148:151], 0\n\t" \
    "ds_write_b64 %[vWr0], v[230:231] offset:35848\n\t" \
    "v_mfma_f32_16x16x32_f16 a[176:179], v[40:43], v[148:151], 0\n\t" \
    "buffer_load_dwordx4 v[164:167], %[vW], %[rW], s39 offen\n\t" \
    "v_mfma_f32_16x16x32_f16 a[144:147], v[36:39], v[148:151], 0\n\t" \
    "s_add_u32 s38, s38, 32768\n\t" \
    "s_and_b32 s38, s38, 0x1fffff\n\t" \
    "v_mfma_f32_16x16x32_f16 a[112:115], v[32:35], v[148:151], 0\n\t" \
    "s_or_b32 s38, s38, %[sA0]\n\t" \
    "s_add_u32 s39, s39, 32768\n\t" \
    "v_mfma_f32_16x16x32_f16 a[80:83], v[28:31], v[148:151], 0\n\t" \
    "s_add_u32 s41, %[own], 0\n\t" \
    "s_and_b32 s41, s41, 3\n\t" \
    "v_mfma_f32_16x16x32_f16 a[48:51], v[24:27], v[148:151], 0\n\t" \
    "s_lshl_b32 s41, s41, 12\n\t" \
    "v_add_u32 v18, s41, %[vStg]\n\t" \
    "v_mfma_f32_16x16x32_f16 a[16:19], v[20:23], v[148:151], 0\n\t" \
    "ds_read_b128 v[148:151], %[vA0] offset:8192\n\t" \
    "s_waitcnt lgkmcnt(8)\n\t" \
    "v_mfma_f32_16x16x32_f16 a[20:23], v[20:23], v[152:155], 0\n\t" \
    "s_lshl_b32 s41, s41, 1\n\t" \
    "s_add_u32 s41, s41, s50\n\t" \
    "v_mfma_f32_16x16x32_f16 a[52:55], v[24:27], v[152:155], 0\n\t" \
    "s_add_u32 s41, s41, 4096\n\t" \
    "s_add_u32 s60, %[wsLo], s41\n\t" \
    "v_mfma_f32_16x16x32_f16 a[84:87], v[28:31], v[152:155], 0\n\t" \
    "s_addc_u32 s61, %[wsHi], 0\n\t" \
    "v_mfma_f32_16x16x32_f16 a[116:119], v[32:35], v[152:155], 0\n\t" \
    "buffer_load_dwordx4 v[168:171], %[vW], %[rW], s39 offen\n\t" \
    "v_mfma_f32_16x16x32_f16 a[148:151], v[36:39], v[152:155], 0\n\t" \
    "s_add_u32 s39, s39, 32768\n\t" \
    "s_add_u32 s65, s49, 1\n\t" \
    "v_mfma_f32_16x16x32_f16 a[180:183], v[40:43], v[152:155], 0\n\t" \
    "v_mov_b32 v19, s65\n\t" \
    "v_mfma_f32_16x16x32_f16 a[212:215], v[44:47], v[152:155], 0\n\t" \
    "buffer_load_dwordx4 v[172:175], %[vW], %[rW], s39 offen\n\t" \
    "v_mfma_f32_16x16x32_f16 a[244:247], v[48:51], v[152:155], 0\n\t" \
    "ds_read_b128 v[152:155], %[vA0] offset:9216\n\t" \
    "s_waitcnt lgkmcnt(6)\n\t" \
    "v_mfma_f32_16x16x32_f16 a[248:251], v[48:51], v[156:159], 0\n\t" \
    "ds_read_b128 v[236:239], v18 offset:0\n\t" \
    "v_mfma_f32_16x16x32_f16 a[216:219], v[44:47], v[156:159], 0\n\t" \
    "ds_read_b128 v[240:243], v18 offset:1024\n\t" \
    "v_mfma_f32_16x16x32_f16 a[184:187], v[40:43], v[156:159], 0\n\t" \
    "s_add_u32 s39, s39, 32768\n\t" \
    "v_mfma_f32_16x16x32_f16 a[152:155], v[36:39], v[156:159], 0\n\t" \
    "ds_read_b128 v[244:247], v18 offset:2048\n\t" \
    "v_mfma_f32_16x16x32_f16 a[120:123], v[32:35], v[156:159], 0\n\t" \
    "s_waitcnt vmcnt(7)\n\t" \
    "buffer_store_dword v19, %[vFs], %[rWS], 0 offen sc0 sc1\n\t" \
    "v_mfma_f32_16x16x32_f16 a[88:91], v[28:31], v[156:159], 0\n\t" \
    "ds_read_b128 v[248:251], v18 offset:3072\n\t" \
    "v_mfma_f32_16x16x32_f16 a[56:59], v[24:27], v[156:159], 0\n\t" \
    "buffer_load_dwordx4 v[176:179], %[vW], %[rW], s39 offen\n\t" \
    "v_mfma_f32_16x16x32_f16 a[24:27], v[20:23], v[156:159], 0\n\t" \
    "ds_read_b128 v[156:159], %[vA0] offset:10240\n\t" \
    "s_waitcnt lgkmcnt(8)\n\t" \
    "v_mfma_f32_16x16x32_f16 a[28:31], v[20:23], v[160:163], 0\n\t" \
    "s_cmp_ge_u32 s48, 2\n\t" \
    "s_cselect_b64 s[74:75], -1, 0\n\t" \
    "v_mfma_f32_16x16x32_f16 a[60:63], v[24:27], v[160:163], 0\n\t" \
    "s_cselect_b64 s[72:73], 0, -1\n\t" \
    "s_add_u32 s39, s39, 32768\n\t" \
    "v_mfma_f32_16x16x32_f16 a[92:95], v[28:31], v[160:163], 0\n\t" \
    "buffer_store_dwordx4 v[232:235], %[vO], s[56:59], 0 offen sc0 sc1 nt\n\t" \
    "v_mfma_f32_16x16x32_f16 a[124:127], v[32:35], v[160:163], 0\n\t" \
    "buffer_load_dwordx4 v[180:183], %[vW], %[rW], s39 offen\n\t" \
    "v_mfma_f32_16x16x32_f16 a[156:159], v[36:39], v[160:163], 0\n\t" \
    "s_add_u32 s39, s39, 32768\n\t" \
    "v_mfma_f32_16x16x32_f16 a[188:191], v[40:43], v[160:163], 0\n\t" \
    "buffer_load_dwordx4 v[184:187], %[vW], %[rW], s39 offen\n\t" \
    "v_mfma_f32_16x16x32_f16 a[220:223], v[44:47], v[160:163], 0\n\t" \
    "s_waitcnt lgkmcnt(3)\n\t" \
    "v_swap_b32 v237, v240\n\t" \
    "v_mfma_f32_16x16x32_f16 a[252:255], v[48:51], v[160:163], 0\n\t" \
    "s_branch FLUSHED%=\n\t" \
    "FLUSH0%=:\n\t" \
    "s_nop 7\n\t" \
    "s_add_u32 s41, s50, 0\n\t" \
    "s_add_u32 s60, %[wsLo], s41\n\t" \
    "s_addc_u32 s61, %[wsHi], 0\n\t" \
    "s_cmp_eq_u32 s68, 0\n\t" \
    "s_cbranch_scc1 FO0_0%=\n\t" \
    "v_accvgpr_read_b32 v236, a0\n\t" \
    "v_accvgpr_read_b32 v237, a4\n\t" \
    "v_accvgpr_read_b32 v238, a8\n\t" \
    "v_accvgpr_read_b32 v239, a12\n\t" \
    "global_store_dwordx4 %[vA0], v[236:239], s[60:61] offset:0 sc0 sc1\n\t" \
    "ds_write_b128 %[vStg], a[32:35] offset:0\n\t" \
    "v_accvgpr_read_b32 v240, a1\n\t" \
    "v_accvgpr_read_b32 v241, a5\n\t" \
    "v_accvgpr_read_b32 v242, a9\n\t" \
    "v_accvgpr_read_b32 v243, a13\n\t" \
    "global_store_dwordx4 %[vA0], v[240:243], s[60:61] offset:1024 sc0 sc1\n\t" \
    "ds_write_b128 %[vStg], a[36:39] offset:1024\n\t" \
    "v_accvgpr_read_b32 v244, a2\n\t" \
    "v_accvgpr_read_b32 v245, a6\n\t" \
    "v_accvgpr_read_b32 v246, a10\n\t" \
    "v_accvgpr_read_b32 v247, a14\n\t" \
    "global_store_dwordx4 %[vA0], v[244:247], s[60:61] offset:2048 sc0 sc1\n\t" \
    "ds_write_b128 %[vStg], a[40:43] offset:2048\n\t" \
    "v_accvgpr_read_b32 v248, a3\n\t" \
    "v_accvgpr_read_b32 v249, a7\n\t" \
    "v_accvgpr_read_b32 v250, a11\n\t" \
    "v_accvgpr_read_b32 v251, a15\n\t" \
    "global_store_dwordx4 %[vA0], v[248:251], s[60:61] offset:3072 sc0 sc1\n\t" \
    "ds_write_b128 %[vStg], a[44:47] offset:3072\n\t" \
    "FB0_0%=:\n\t" \
    "s_add_u32 s41, s50, 8192\n\t" \
    "s_add_u32 s60, %[wsLo], s41\n\t" \
    "s_addc_u32 s61, %[wsHi], 0\n\t" \
    "s_cmp_eq_u32 s68, 1\n\t" \
    "s_cbranch_scc1 FO0_2%=\n\t" \
    "v_accvgpr_read_b32 v236, a64\n\t" \
    "v_accvgpr_read_b32 v237, a68\n\t" \
    "v_accvgpr_read_b32 v238, a72\n\t" \
    "v_accvgpr_read_b32 v239, a76\n\t" \
    "global_store_dwordx4 %[vA0], v[236:239], s[60:61] offset:0 sc0 sc1\n\t" \
    "ds_write_b128 %[vStg], a[96:99] offset:4096\n\t" \
    "v_accvgpr_read_b32 v240, a65\n\t" \
    "v_accvgpr_read_b32 v241, a69\n\t" \
    "v_accvgpr_read_b32 v242, a73\n\t" \
    "v_accvgpr_read_b32 v243, a77\n\t" \
    "global_store_dwordx4 %[vA0], v[240:243], s[60:61] offset:1024 sc0 sc1\n\t" \
    "ds_write_b128 %[vStg], a[100:103] offset:5120\n\t" \
    "v_accvgpr_read_b32 v244, a66\n\t" \
    "v_accvgpr_read_b32 v245, a70\n\t" \
    "v_accvgpr_read_b32 v246, a74\n\t" \
    "v_accvgpr_read_b32 v247, a78\n\t" \
    "global_store_dwordx4 %[vA0], v[244:247], s[60:61] offset:2048 sc0 sc1\n\t" \
    "ds_write_b128 %[vStg], a[104:107] offset:6144\n\t" \
    "v_accvgpr_read_b32 v248, a67\n\t" \
    "v_accvgpr_read_b32 v249, a71\n\t" \
    "v_accvgpr_read_b32 v250, a75\n\t" \
    "v_accvgpr_read_b32 v251, a79\n\t" \
    "global_store_dwordx4 %[vA0], v[248:251], s[60:61] offset:3072 sc0 sc1\n\t" \
    "ds_write_b128 %[vStg], a[108:111] offset:7168\n\t" \
    "FB0_2%=:\n\t" \
    "s_add_u32 s41, s50, 16384\n\t" \
    "s_add_u32 s60, %[wsLo], s41\n\t" \
    "s_addc_u32 s61, %[wsHi], 0\n\t" \
    "s_cmp_eq_u32 s68, 2\n\t" \
    "s_cbranch_scc1 FO0_4%=\n\t" \
    "v_accvgpr_read_b32 v236, a128\n\t" \
    "v_accvgpr_read_b32 v237, a132\n\t" \
    "v_accvgpr_read_b32 v238, a136\n\t" \
    "v_accvgpr_read_b32 v239, a140\n\t" \
    "global_store_dwordx4 %[vA0], v[236:239], s[60:61] offset:0 sc0 sc1\n\t" \
    "ds_write_b128 %[vStg], a[160:163] offset:8192\n\t" \
    "v_accvgpr_read_b32 v240, a129\n\t" \
    "v_accvgpr_read_b32 v241, a133\n\t" \
    "v_accvgpr_read_b32 v242, a137\n\t" \
    "v_accvgpr_read_b32 v243, a141\n\t" \
    "global_store_dwordx4 %[vA0], v[240:243], s[60:61] offset:1024 sc0 sc1\n\t" \
    "ds_write_b128 %[vStg], a[164:167] offset:9216\n\t" \
    "v_accvgpr_read_b32 v244, a130\n\t" \
    "v_accvgpr_read_b32 v245, a134\n\t" \
    "v_accvgpr_read_b32 v246, a138\n\t" \
    "v_accvgpr_read_b32 v247, a142\n\t" \
    "global_store_dwordx4 %[vA0], v[244:247], s[60:61] offset:2048 sc0 sc1\n\t" \
    "ds_write_b128 %[vStg], a[168:171] offset:10240\n\t" \
    "v_accvgpr_read_b32 v248, a131\n\t" \
    "v_accvgpr_read_b32 v249, a135\n\t" \
    "v_accvgpr_read_b32 v250, a139\n\t" \
    "v_accvgpr_read_b32 v251, a143\n\t" \
    "global_store_dwordx4 %[vA0], v[248:251], s[60:61] offset:3072 sc0 sc1\n\t" \
    "ds_write_b128 %[vStg], a[172:175] offset:11264\n\t" \
    "FB0_4%=:\n\t" \
    "s_add_u32 s41, s50, 24576\n\t" \
    "s_add_u32 s60, %[wsLo], s41\n\t" \
    "s_addc_u32 s61, %[wsHi], 0\n\t" \
    "s_cmp_eq_u32 s68, 3\n\t" \
    "s_cbranch_scc1 FO0_6%=\n\t" \
    "v_accvgpr_read_b32 v236, a192\n\t" \
    "v_accvgpr_read_b32 v237, a196\n\t" \
    "v_accvgpr_read_b32 v238, a200\n\t" \
    "v_accvgpr_read_b32 v239, a204\n\t" \
    "global_store_dwordx4 %[vA0], v[236:239], s[60:61] offset:0 sc0 sc1\n\t" \
    "ds_write_b128 %[vStg], a[224:227] offset:12288\n\t" \
    "v_accvgpr_read_b32 v240, a193\n\t" \
    "v_accvgpr_read_b32 v241, a197\n\t" \
    "v_accvgpr_read_b32 v242, a201\n\t" \
    "v_accvgpr_read_b32 v243, a205\n\t" \
    "global_store_dwordx4 %[vA0], v[240:243], s[60:61] offset:1024 sc0 sc1\n\t" \
    "ds_write_b128 %[vStg], a[228:231] offset:13312\n\t" \
    "v_accvgpr_read_b32 v244, a194\n\t" \
    "v_accvgpr_read_b32 v245, a198\n\t" \
    "v_accvgpr_read_b32 v246, a202\n\t" \
    "v_accvgpr_read_b32 v247, a206\n\t" \
    "global_store_dwordx4 %[vA0], v[244:247], s[60:61] offset:2048 sc0 sc1\n\t" \
    "ds_write_b128 %[vStg], a[232:235] offset:14336\n\t" \
    "v_accvgpr_read_b32 v248, a195\n\t" \
    "v_accvgpr_read_b32 v249, a199\n\t" \
    "v_accvgpr_read_b32 v250, a203\n\t" \
    "v_accvgpr_read_b32 v251, a207\n\t" \
    "global_store_dwordx4 %[vA0], v[248:251], s[60:61] offset:3072 sc0 sc1\n\t" \
    "ds_write_b128 %[vStg], a[236:239] offset:15360\n\t" \
    "FB0_6%=:\n\t" \
    "s_nop 1\n\t" \
    "s_nop 1\n\t" \
    "ds_read_b128 v[160:163], %[vA0] offset:3072\n\t" \
    "s_waitcnt vmcnt(32) lgkmcnt(3)\n\t" \
    "v_mfma_f32_16x16x32_f16 a[224:227], v[48:51], v[148:151], 0\n\t" \
    "s_mov_b32 s37, s38\n\t" \
    "v_cvt_pk_f16_f32 v228, v164, v168\n\t" \
    "v_mfma_f32_16x16x32_f16 a[192:195], v[44:47], v[148:151], 0\n\t" \
    "v_mfma_f32_16x16x32_f16 a[160:163], v[40:43], v[148:151], 0\n\t" \
    "v_cvt_pk_f16_f32 v229, v172, v176\n\t" \
    "v_mfma_f32_16x16x32_f16 a[128:131], v[36:39], v[148:151], 0\n\t" \
    "ds_write_b64 %[vWr0], v[228:229] offset:32768\n\t" \
    "v_mfma_f32_16x16x32_f16 a[96:99], v[32:35], v[148:151], 0\n\t" \
    "v_cvt_pk_f16_f32 v230, v180, v184\n\t" \
    "v_mfma_f32_16x16x32_f16 a[64:67], v[28:31], v[148:151], 0\n\t" \
    "v_mfma_f32_16x16x32_f16 a[32:35], v[24:27], v[148:151], 0\n\t" \
    "v_cvt_pk_f16_f32 v231, v188, v192\n\t" \
    "v_mfma_f32_16x16x32_f16 a[0:3], v[20:23], v[148:151], 0\n\t" \
    "ds_read_b128 v[148:151], %[vA0] offset:4096\n\t" \
    "s_waitcnt lgkmcnt(4)\n\t" \
    "v_mfma_f32_16x16x32_f16 a[4:7], v[20:23], v[152:155], 0\n\t" \
    "s_cmp_ge_u32 s48, 2\n\ts_cselect_b64 exec, -1, 0\n\ts_cselect_b32 s69, %[own], 9\n\tds_read_b128 v[252:255], %[vStg] offset:16384\n\ts_mov_b64 exec, -1\n\t" \
    "s_waitcnt lgkmcnt(4)\n\t" \
    "v_mfma_f32_16x16x32_f16 a[36:39], v[24:27], v[152:155], 0\n\t" \
    "ds_write_b64 %[vWr0], v[230:231] offset:32776\n\t" \
    "v_mfma_f32_16x16x32_f16 a[68:71], v[28:31], v[152:155], 0\n\t" \
    "v_cvt_pk_f16_f32 v228, v165, v169\n\t" \
    "s_add_u32 s55, s37, 20480\n\t" \
    "v_mfma_f32_16x16x32_f16 a[100:103], v[32:35], v[152:155], 0\n\t" \
    "v_cvt_pk_f16_f32 v229, v173, v177\n\t" \
    "v_mfma_f32_16x16x32_f16 a[132:135], v[36:39], v[152:155], 0\n\t" \
    "buffer_load_dwordx4 v[100:103], %[vA0], %[rWS], s55 offen offset:0 sc0 sc1\n\t" \
    "v_mfma_f32_16x16x32_f16 a[164:167], v[40:43], v[152:155], 0\n\t" \
    "ds_write_b64 %[vWr0], v[228:229] offset:33792\n\t" \
    "v_mfma_f32_16x16x32_f16 a[196:199], v[44:47], v[152:155], 0\n\t" \
    "v_cvt_pk_f16_f32 v230, v181, v185\n\t" \
    "v_cvt_pk_f16_f32 v231, v189, v193\n\t" \
    "v_mfma_f32_16x16x32_f16 a[228:231], v[48:51], v[152:155], 0\n\t" \
    "ds_read_b128 v[152:155], %[vA0] offset:5120\n\t" \
    "s_waitcnt lgkmcnt(6)\n\t" \
    "v_mfma_f32_16x16x32_f16 a[232:235], v[48:51], v[156:159], 0\n\t" \
    "ds_write_b64 %[vWr0], v[230:231] offset:33800\n\t" \
    "v_mfma_f32_16x16x32_f16 a[200:203], v[44:47], v[156:159], 0\n\t" \
    "buffer_load_dwordx4 v[104:107], %[vA0], %[rWS], s55 offen offset:1024 sc0 sc1\n\t" \
    "v_mfma_f32_16x16x32_f16 a[168:171], v[40:43], v[156:159], 0\n\t" \
    "v_cvt_pk_f16_f32 v228, v166, v170\n\t" \
    "v_cvt_pk_f16_f32 v229, v174, v178\n\t" \
    "v_mfma_f32_16x16x32_f16 a[136:139], v[36:39], v[156:159], 0\n\t" \
    "ds_write_b64 %[vWr0], v[228:229] offset:34816\n\t" \
    "v_mfma_f32_16x16x32_f16 a[104:107], v[32:35], v[156:159], 0\n\t" \
    "v_cvt_pk_f16_f32 v230, v182, v186\n\t" \
    "v_mfma_f32_16x16x32_f16 a[72:75], v[28:31], v[156:159], 0\n\t" \
    "buffer_load_dwordx4 v[108:111], %[vA0], %[rWS], s55 offen offset:2048 sc0 sc1\n\t" \
    "v_mfma_f32_16x16x32_f16 a[40:43], v[24:27], v[156:159], 0\n\t" \
    "s_add_u32 s66, s66, 0x400000\n\t" \
    "s_and_b32 s66, s66, 0xfffffff\n\t" \
    "v_mfma_f32_16x16x32_f16 a[8:11], v[20:23], v[156:159], 0\n\t" \
    "ds_read_b128 v[156:159], %[vA0] offset:6144\n\t" \
    "s_waitcnt lgkmcnt(8)\n\t" \
    "v_mfma_f32_16x16x32_f16 a[12:15], v[20:23], v[160:163], 0\n\t" \
    "v_cvt_pk_f16_f32 v231, v190, v194\n\t" \
    "v_mfma_f32_16x16x32_f16 a[44:47], v[24:27], v[160:163], 0\n\t" \
    "ds_write_b64 %[vWr0], v[230:231] offset:34824\n\t" \
    "v_mfma_f32_16x16x32_f16 a[76:79], v[28:31], v[160:163], 0\n\t" \
    "v_cvt_pk_f16_f32 v228, v167, v171\n\t" \
    "s_add_u32 s39, s66, s43\n\t" \
    "v_mfma_f32_16x16x32_f16 a[108:111], v[32:35], v[160:163], 0\n\t" \
    "s_add_u32 s39, s39, %[sWk0]\n\t" \
    "v_mfma_f32_16x16x32_f16 a[140:143], v[36:39], v[160:163], 0\n\t" \
    "buffer_load_dwordx4 v[112:115], %[vA0], %[rWS], s55 offen offset:3072 sc0 sc1\n\t" \
    "v_mfma_f32_16x16x32_f16 a[172:175], v[40:43], v[160:163], 0\n\t" \
    "v_cvt_pk_f16_f32 v229, v175, v179\n\t" \
    "v_mfma_f32_16x16x32_f16 a[204:207], v[44:47], v[160:163], 0\n\t" \
    "ds_write_b64 %[vWr0], v[228:229] offset:35840\n\t" \
    "v_mfma_f32_16x16x32_f16 a[236:239], v[48:51], v[160:163], 0\n\t" \
    "ds_read_b128 v[160:163], %[vA0] offset:7168\n\t" \
    "s_waitcnt lgkmcnt(9)\n\t" \
    "v_mfma_f32_16x16x32_f16 a[240:243], v[48:51], v[148:151], a[240:243]\n\t" \
    "v_cvt_pk_f16_f32 v230, v183, v187\n\t" \
    "v_cvt_pk_f16_f32 v231, v191, v195\n\t" \
    "v_mfma_f32_16x16x32_f16 a[208:211], v[44:47], v[148:151], a[208:211]\n\t" \
    "ds_write_b64 %[vWr0], v[230:231] offset:35848\n\t" \
    "v_mfma_f32_16x16x32_f16 a[176:179], v[40:43], v[148:151], a[176:179]\n\t" \
    "buffer_load_dwordx4 v[164:167], %[vW], %[rW], s39 offen\n\t" \
    "v_mfma_f32_16x16x32_f16 a[144:147], v[36:39], v[148:151], a[144:147]\n\t" \
    "s_add_u32 s38, s38, 32768\n\t" \
    "s_and_b32 s38, s38, 0x1fffff\n\t" \
    "v_mfma_f32_16x16x32_f16 a[112:115], v[32:35], v[148:151], a[112:115]\n\t" \
    "s_or_b32 s38, s38, %[sA0]\n\t" \
    "s_add_u32 s39, s39, 32768\n\t" \
    "v_mfma_f32_16x16x32_f16 a[80:83], v[28:31], v[148:151], a[80:83]\n\t" \
    "s_add_u32 s41, %[own], 0\n\t" \
    "s_and_b32 s41, s41, 3\n\t" \
    "v_mfma_f32_16x16x32_f16 a[48:51], v[24:27], v[148:151], a[48:51]\n\t" \
    "s_lshl_b32 s41, s41, 12\n\t" \
    "v_add_u32 v18, s41, %[vStg]\n\t" \
    "v_mfma_f32_16x16x32_f16 a[16:19], v[20:23], v[148:151], a[16:19]\n\t" \
    "ds_read_b128 v[148:151], %[vA0] offset:8192\n\t" \
    "s_waitcnt lgkmcnt(8)\n\t" \
    "v_mfma_f32_16x16x32_f16 a[20:23], v[20:23], v[152:155], a[20:23]\n\t" \
    "s_lshl_b32 s41, s41, 1\n\t" \
    "s_add_u32 s41, s41, s50\n\t" \
    "v_mfma_f32_16x16x32_f16 a[52:55], v[24:27], v[152:155], a[52:55]\n\t" \
    "s_add_u32 s41, s41, 4096\n\t" \
    "s_add_u32 s60, %[wsLo], s41\n\t" \
    "v_mfma_f32_16x16x32_f16 a[84:87], v[28:31], v[152:155], a[84:87]\n\t" \
    "s_addc_u32 s61, %[wsHi], 0\n\t" \
    "v_mfma_f32_16x16x32_f16 a[116:119], v[32:35], v[152:155], a[116:119]\n\t" \
    "buffer_load_dwordx4 v[168:171], %[vW], %[rW], s39 offen\n\t" \
    "v_mfma_f32_16x16x32_f16 a[148:151], v[36:39], v[152:155], a[148:151]\n\t" \
    "s_add_u32 s39, s39, 32768\n\t" \
    "s_add_u32 s65, s49, 1\n\t" \
    "v_mfma_f32_16x16x32_f16 a[180:183], v[40:43], v[152:155], a[180:183]\n\t" \
    "v_mov_b32 v19, s65\n\t" \
    "v_mfma_f32_16x16x32_f16 a[212:215], v[44:47], v[152:155], a[212:215]\n\t" \
    "buffer_load_dwordx4 v[172:175], %[vW], %[rW], s39 offen\n\t" \
    "v_mfma_f32_16x16x32_f16 a[244:247], v[48:51], v[152:155], a[244:247]\n\t" \
    "ds_read_b128 v[152:155], %[vA0] offset:9216\n\t" \
    "s_waitcnt lgkmcnt(6)\n\t" \
    "v_mfma_f32_16x16x32_f16 a[248:251], v[48:51], v[156:159], a[248:251]\n\t" \
    "ds_read_b128 v[236:239], v18 offset:0\n\t" \
    "v_mfma_f32_16x16x32_f16 a[216:219], v[44:47], v[156:159], a[216:219]\n\t" \
    "ds_read_b128 v[240:243], v18 offset:1024\n\t" \
    "v_mfma_f32_16x16x32_f16 a[184:187], v[40:43], v[156:159], a[184:187]\n\t" \
    "s_add_u32 s39, s39, 32768\n\t" \
    "v_mfma_f32_16x16x32_f16 a[152:155], v[36:39], v[156:159], a[152:155]\n\t" \
    "ds_read_b128 v[244:247], v18 offset:2048\n\t" \
    "v_mfma_f32_16x16x32_f16 a[120:123], v[32:35], v[156:159], a[120:123]\n\t" \
    "s_waitcnt vmcnt(7)\n\t" \
    "buffer_store_dword v19, %[vFs], %[rWS], 0 offen sc0 sc1\n\t" \
    "v_mfma_f32_16x16x32_f16 a[88:91], v[28:31], v[156:159], a[88:91]\n\t" \
    "ds_read_b128 v[248:251], v18 offset:3072\n\t" \
    "v_mfma_f32_16x16x32_f16 a[56:59], v[24:27], v[156:159], a[56:59]\n\t" \
    "buffer_load_dwordx4 v[176:179], %[vW], %[rW], s39 offen\n\t" \
    "v_mfma_f32_16x16x32_f16 a[24:27], v[20:23], v[156:159], a[24:27]\n\t" \
    "ds_read_b128 v[156:159], %[vA0] offset:10240\n\t" \
    "s_waitcnt lgkmcnt(8)\n\t" \
    "v_mfma_f32_16x16x32_f16 a[28:31], v[20:23], v[160:163], a[28:31]\n\t" \
    "s_cmp_ge_u32 s48, 2\n\t" \
    "s_cselect_b64 s[74:75], -1, 0\n\t" \
    "v_mfma_f32_16x16x32_f16 a[60:63], v[24:27], v[160:163], a[60:63]\n\t" \
    "s_cselect_b64 s[72:73], 0, -1\n\t" \
    "s_add_u32 s39, s39, 32768\n\t" \
    "v_mfma_f32_16x16x32_f16 a[92:95], v[28:31], v[160:163], a[92:95]\n\t" \
    "buffer_store_dwordx4 v[232:235], %[vO], s[56:59], 0 offen sc0 sc1 nt\n\t" \
    "v_mfma_f32_16x16x32_f16 a[124:127], v[32:35], v[160:163], a[124:127]\n\t" \
    "buffer_load_dwordx4 v[180:183], %[vW], %[rW], s39 offen\n\t" \
    "v_mfma_f32_16x16x32_f16 a[156:159], v[36:39], v[160:163], a[156:159]\n\t" \
    "s_add_u32 s39, s39, 32768\n\t" \
    "v_mfma_f32_16x16x32_f16 a[188:191], v[40:43], v[160:163], a[188:191]\n\t" \
    "buffer_load_dwordx4 v[184:187], %[vW], %[rW], s39 offen\n\t" \
    "v_mfma_f32_16x16x32_f16 a[220:223], v[44:47], v[160:163], a[220:223]\n\t" \
    "s_waitcnt lgkmcnt(3)\n\t" \
    "v_swap_b32 v237, v240\n\t" \
    "v_mfma_f32_16x16x32_f16 a[252:255], v[48:51], v[160:163], a[252:255]\n\t" \
    "FLUSHED%=:\n\t" \
    "KS1%=:\n\t" \
    "ds_read_b128 v[160:163], %[vA0] offset:11264\n\t" \
    "v_mfma_f32_16x16x32_f16 a[224:227], v[80:83], v[148:151], a[224:227]\n\t" \
    "s_waitcnt lgkmcnt(3)\n\t" \
    "v_swap_b32 v238, v244\n\t" \
    "v_mfma_f32_16x16x32_f16 a[192:195], v[76:79], v[148:151], a[192:195]\n\t" \
    "s_add_u32 s39, s39, 32768\n\t" \
    "v_mfma_f32_16x16x32_f16 a[160:163], v[72:75], v[148:151], a[160:163]\n\t" \
    "s_waitcnt lgkmcnt(2)\n\t" \
    "v_swap_b32 v239, v248\n\t" \
    "v_mfma_f32_16x16x32_f16 a[128:131], v[68:71], v[148:151], a[128:131]\n\t" \
    "v_swap_b32 v242, v245\n\t" \
    "v_mfma_f32_16x16x32_f16 a[96:99], v[64:67], v[148:151], a[96:99]\n\t" \
    "buffer_load_dwordx4 v[188:191], %[vW], %[rW], s39 offen\n\t" \
    "v_mfma_f32_16x16x32_f16 a[64:67], v[60:63], v[148:151], a[64:67]\n\t" \
    "v_swap_b32 v243, v249\n\t" \
    "v_mfma_f32_16x16x32_f16 a[32:35], v[56:59], v[148:151], a[32:35]\n\t" \
    "v_swap_b32 v247, v250\n\t" \
    "v_mfma_f32_16x16x32_f16 a[0:3], v[52:55], v[148:151], a[0:3]\n\t" \
    "ds_read_b128 v[148:151], %[vA0] offset:12288\n\t" \
    "v_mfma_f32_16x16x32_f16 a[4:7], v[52:55], v[152:155], a[4:7]\n\t" \
    "s_add_u32 s39, s39, 32768\n\t" \
    "v_mfma_f32_16x16x32_f16 a[36:39], v[56:59], v[152:155], a[36:39]\n\t" \
    "s_mov_b64 exec, s[72:73]\n\tglobal_store_dwordx4 %[vA0], v[236:239], s[60:61] offset:0 sc0 sc1\n\ts_mov_b64 exec, -1\n\t" \
    "v_mfma_f32_16x16x32_f16 a[68:71], v[60:63], v[152:155], a[68:71]\n\t" \
    "s_mov_b64 exec, s[74:75]\n\tds_write_b128 %[vStg], v[236:239] offset:20480\n\ts_mov_b64 exec, -1\n\t" \
    "v_mfma_f32_16x16x32_f16 a[100:103], v[64:67], v[152:155], a[100:103]\n\t" \
    "buffer_load_dwordx4 v[192:195], %[vW], %[rW], s39 offen\n\t" \
    "v_mfma_f32_16x16x32_f16 a[132:135], v[68:71], v[152:155], a[132:135]\n\t" \
    "v_cvt_pk_f16_f32 v228, v196, v200\n\t" \
    "v_mfma_f32_16x16x32_f16 a[164:167], v[72:75], v[152:155], a[164:167]\n\t" \
    "s_mov_b64 exec, s[72:73]\n\tglobal_store_dwordx4 %[vA0], v[240:243], s[60:61] offset:1024 sc0 sc1\n\ts_mov_b64 exec, -1\n\t" \
    "v_mfma_f32_16x16x32_f16 a[196:199], v[76:79], v[152:155], a[196:199]\n\t" \
    "s_mov_b64 exec, s[74:75]\n\tds_write_b128 %[vStg], v[240:243] offset:21504\n\ts_mov_b64 exec, -1\n\t" \
    "v_mfma_f32_16x16x32_f16 a[228:231], v[80:83], v[152:155], a[228:231]\n\t" \
    "ds_read_b128 v[152:155], %[vA0] offset:13312\n\t" \
    "s_waitcnt lgkmcnt(3)\n\t" \
    "v_mfma_f32_16x16x32_f16 a[232:235], v[80:83], v[156:159], a[232:235]\n\t" \
    "v_cvt_pk_f16_f32 v229, v204, v208\n\t" \
    "v_mfma_f32_16x16x32_f16 a[200:203], v[76:79], v[156:159], a[200:203]\n\t" \
    "ds_write_b64 %[vWr1], v[228:229] offset:32768\n\t" \
    "v_mfma_f32_16x16x32_f16 a[168:171], v[72:75], v[156:159], a[168:171]\n\t" \
    "v_cvt_pk_f16_f32 v230, v212, v216\n\t" \
    "v_mfma_f32_16x16x32_f16 a[136:139], v[68:71], v[156:159], a[136:139]\n\t" \
    "s_mov_b64 exec, s[72:73]\n\tglobal_store_dwordx4 %[vA0], v[244:247], s[60:61] offset:2048 sc0 sc1\n\ts_mov_b64 exec, -1\n\t" \
    "v_mfma_f32_16x16x32_f16 a[104:107], v[64:67], v[156:159], a[104:107]\n\t" \
    "s_mov_b64 exec, s[74:75]\n\tds_write_b128 %[vStg], v[244:247] offset:22528\n\ts_mov_b64 exec, -1\n\t" \
    "s_waitcnt lgkmcnt(4)\n\t" \
    "v_mfma_f32_16x16x32_f16 a[72:75], v[60:63], v[156:159], a[72:75]\n\t" \
    "v_cvt_pk_f16_f32 v231, v220, v224\n\t" \
    "v_mfma_f32_16x16x32_f16 a[40:43], v[56:59], v[156:159], a[40:43]\n\t" \
    "ds_write_b64 %[vWr1], v[230:231] offset:32776\n\t" \
    "v_mfma_f32_16x16x32_f16 a[8:11], v[52:55], v[156:159], a[8:11]\n\t" \
    "ds_read_b128 v[156:159], %[vA0] offset:14336\n\t" \
    "s_waitcnt lgkmcnt(5)\n\t" \
    "v_mfma_f32_16x16x32_f16 a[12:15], v[52:55], v[160:163], a[12:15]\n\t" \
    "v_cvt_pk_f16_f32 v228, v197, v201\n\t" \
    "v_mfma_f32_16x16x32_f16 a[44:47], v[56:59], v[160:163], a[44:47]\n\t" \
    "s_mov_b64 exec, s[72:73]\n\tglobal_store_dwordx4 %[vA0], v[248:251], s[60:61] offset:3072 sc0 sc1\n\ts_mov_b64 exec, -1\n\t" \
    "v_mfma_f32_16x16x32_f16 a[76:79], v[60:63], v[160:163], a[76:79]\n\t" \
    "s_mov_b64 exec, s[74:75]\n\tds_write_b128 %[vStg], v[248:251] offset:23552\n\ts_mov_b64 exec, -1\n\t" \
    "s_waitcnt lgkmcnt(5)\n\t" \
    "v_mfma_f32_16x16x32_f16 a[108:111], v[64:67], v[160:163], a[108:111]\n\t" \
    "v_cvt_pk_f16_f32 v229, v205, v209\n\t" \
    "v_mfma_f32_16x16x32_f16 a[140:143], v[68:71], v[160:163], a[140:143]\n\t" \
    "ds_write_b64 %[vWr1], v[228:229] offset:33792\n\t" \
    "v_mfma_f32_16x16x32_f16 a[172:175], v[72:75], v[160:163], a[172:175]\n\t" \
    "v_cvt_pk_f16_f32 v230, v213, v217\n\t" \
    "v_cvt_pk_f16_f32 v231, v221, v225\n\t" \
    "v_mfma_f32_16x16x32_f16 a[204:207], v[76:79], v[160:163], a[204:207]\n\t" \
    "s_add_u32 s55, s37, 24576\n\t" \
    "v_mfma_f32_16x16x32_f16 a[236:239], v[80:83], v[160:163], a[236:239]\n\t" \
    "ds_read_b128 v[160:163], %[vA0] offset:15360\n\t" \
    "s_waitcnt lgkmcnt(6)\n\t" \
    "v_mfma_f32_16x16x32_f16 a[240:243], v[80:83], v[148:151], a[240:243]\n\t" \
    "ds_write_b64 %[vWr1], v[230:231] offset:33800\n\t" \
    "v_mfma_f32_16x16x32_f16 a[208:211], v[76:79], v[148:151], a[208:211]\n\t" \
    "v_cvt_pk_f16_f32 v228, v198, v202\n\t" \
    "v_mfma_f32_16x16x32_f16 a[176:179], v[72:75], v[148:151], a[176:179]\n\t" \
    "buffer_load_dwordx4 v[116:119], %[vA0], %[rWS], s55 offen offset:0 sc0 sc1\n\t" \
    "v_mfma_f32_16x16x32_f16 a[144:147], v[68:71], v[148:151], a[144:147]\n\t" \
    "v_cvt_pk_f16_f32 v229, v206, v210\n\t" \
    "v_mfma_f32_16x16x32_f16 a[112:115], v[64:67], v[148:151], a[112:115]\n\t" \
    "ds_write_b64 %[vWr1], v[228:229] offset:34816\n\t" \
    "v_mfma_f32_16x16x32_f16 a[80:83], v[60:63], v[148:151], a[80:83]\n\t" \
    "v_cvt_pk_f16_f32 v230, v214, v218\n\t" \
    "v_mfma_f32_16x16x32_f16 a[48:51], v[56:59], v[148:151], a[48:51]\n\t" \
    "buffer_load_dwordx4 v[120:123], %[vA0], %[rWS], s55 offen offset:1024 sc0 sc1\n\t" \
    "v_mfma_f32_16x16x32_f16 a[16:19], v[52:55], v[148:151], a[16:19]\n\t" \
    "ds_read_b128 v[148:151], %[vA0] offset:16384\n\t" \
    "s_waitcnt lgkmcnt(8)\n\t" \
    "v_mfma_f32_16x16x32_f16 a[20:23], v[52:55], v[152:155], a[20:23]\n\t" \
    "v_cvt_pk_f16_f32 v231, v222, v226\n\t" \
    "v_mfma_f32_16x16x32_f16 a[52:55], v[56:59], v[152:155], a[52:55]\n\t" \
    "ds_write_b64 %[vWr1], v[230:231] offset:34824\n\t" \
    "v_mfma_f32_16x16x32_f16 a[84:87], v[60:63], v[152:155], a[84:87]\n\t" \
    "v_cvt_pk_f16_f32 v228, v199, v203\n\t" \
    "v_cvt_pk_f16_f32 v229, v207, v211\n\t" \
    "v_mfma_f32_16x16x32_f16 a[116:119], v[64:67], v[152:155], a[116:119]\n\t" \
    "buffer_load_dwordx4 v[124:127], %[vA0], %[rWS], s55 offen offset:2048 sc0 sc1\n\t" \
    "v_mfma_f32_16x16x32_f16 a[148:151], v[68:71], v[152:155], a[148:151]\n\t" \
    "ds_write_b64 %[vWr1], v[228:229] offset:35840\n\t" \
    "v_mfma_f32_16x16x32_f16 a[180:183], v[72:75], v[152:155], a[180:183]\n\t" \
    "v_cvt_pk_f16_f32 v230, v215, v219\n\t" \
    "v_cvt_pk_f16_f32 v231, v223, v227\n\t" \
    "v_mfma_f32_16x16x32_f16 a[212:215], v[76:79], v[152:155], a[212:215]\n\t" \
    "ds_write_b64 %[vWr1], v[230:231] offset:35848\n\t" \
    "v_mfma_f32_16x16x32_f16 a[244:247], v[80:83], v[152:155], a[244:247]\n\t" \
    "ds_read_b128 v[152:155], %[vA0] offset:17408\n\t" \
    "s_waitcnt lgkmcnt(9)\n\t" \
    "v_mfma_f32_16x16x32_f16 a[248:251], v[80:83], v[156:159], a[248:251]\n\t" \
    "buffer_load_dwordx4 v[128:131], %[vA0], %[rWS], s55 offen offset:3072 sc0 sc1\n\t" \
    "v_mfma_f32_16x16x32_f16 a[216:219], v[76:79], v[156:159], a[216:219]\n\t" \
    "s_add_u32 s55, s37, 28672\n\t" \
    "v_mfma_f32_16x16x32_f16 a[184:187], v[72:75], v[156:159], a[184:187]\n\t" \
    "buffer_load_dwordx4 v[132:135], %[vA0], %[rWS], s55 offen offset:0 sc0 sc1\n\t" \
    "v_mfma_f32_16x16x32_f16 a[152:155], v[68:71], v[156:159], a[152:155]\n\t" \
    "buffer_load_dwordx4 v[136:139], %[vA0], %[rWS], s55 offen offset:1024 sc0 sc1\n\t" \
    "v_mfma_f32_16x16x32_f16 a[120:123], v[64:67], v[156:159], a[120:123]\n\t" \
    "buffer_load_dwordx4 v[140:143], %[vA0], %[rWS], s55 offen offset:2048 sc0 sc1\n\t" \
    "v_mfma_f32_16x16x32_f16 a[88:91], v[60:63], v[156:159], a[88:91]\n\t" \
    "buffer_load_dwordx4 v[144:147], %[vA0], %[rWS], s55 offen offset:3072 sc0 sc1\n\t" \
    "v_mfma_f32_16x16x32_f16 a[56:59], v[56:59], v[156:159], a[56:59]\n\t" \
    "v_mfma_f32_16x16x32_f16 a[24:27], v[52:55], v[156:159], a[24:27]\n\t" \
    "ds_read_b128 v[156:159], %[vA0] offset:18432\n\t" \
    "s_waitcnt lgkmcnt(8)\n\t" \
    "v_mfma_f32_16x16x32_f16 a[28:31], v[52:55], v[160:163], a[28:31]\n\t" \
    "s_add_u32 s55, s38, 0\n\t" \
    "v_mfma_f32_16x16x32_f16 a[60:63], v[56:59], v[160:163], a[60:63]\n\t" \
    "v_mfma_f32_16x16x32_f16 a[92:95], v[60:63], v[160:163], a[92:95]\n\t" \
    "buffer_load_dwordx4 v[20:23], %[vA0], %[rWS], s55 offen offset:0 sc0 sc1\n\t" \
    "v_mfma_f32_16x16x32_f16 a[124:127], v[64:67], v[160:163], a[124:127]\n\t" \
    "v_mfma_f32_16x16x32_f16 a[156:159], v[68:71], v[160:163], a[156:159]\n\t" \
    "v_mfma_f32_16x16x32_f16 a[188:191], v[72:75], v[160:163], a[188:191]\n\t" \
    "v_mfma_f32_16x16x32_f16 a[220:223], v[76:79], v[160:163], a[220:223]\n\t" \
    "v_mfma_f32_16x16x32_f16 a[252:255], v[80:83], v[160:163], a[252:255]\n\t" \
    "ds_read_b128 v[160:163], %[vA0] offset:19456\n\t" \
    "s_waitcnt vmcnt(19) lgkmcnt(6)\n\t" \
    "v_mfma_f32_16x16x32_f16 a[224:227], v[112:115], v[148:151], a[224:227]\n\t" \
    "buffer_load_dwordx4 v[24:27], %[vA0], %[rWS], s55 offen offset:1024 sc0 sc1\n\t" \
    "v_mfma_f32_16x16x32_f16 a[192:195], v[108:111], v[148:151], a[192:195]\n\t" \
    "v_mfma_f32_16x16x32_f16 a[160:163], v[104:107], v[148:151], a[160:163]\n\t" \
    "v_mfma_f32_16x16x32_f16 a[128:131], v[100:103], v[148:151], a[128:131]\n\t" \
    "v_mfma_f32_16x16x32_f16 a[96:99], v[96:99], v[148:151], a[96:99]\n\t" \
    "buffer_load_dwordx4 v[28:31], %[vA0], %[rWS], s55 offen offset:2048 sc0 sc1\n\t" \
    "v_mfma_f32_16x16x32_f16 a[64:67], v[92:95], v[148:151], a[64:67]\n\t" \
    "s_add_u32 s49, s49, 1\n\t" \
    "v_mfma_f32_16x16x32_f16 a[32:35], v[88:91], v[148:151], a[32:35]\n\t" \
    "buffer_load_dword v19, %[vFl], %[rWS], 0 offen sc0 sc1\n\t" \
    "v_mfma_f32_16x16x32_f16 a[0:3], v[84:87], v[148:151], a[0:3]\n\t" \
    "ds_read_b128 v[148:151], %[vA0] offset:20480\n\t" \
    "s_waitcnt lgkmcnt(3)\n\t" \
    "v_mfma_f32_16x16x32_f16 a[4:7], v[84:87], v[152:155], a[4:7]\n\t" \
    "v_mfma_f32_16x16x32_f16 a[36:39], v[88:91], v[152:155], a[36:39]\n\t" \
    "buffer_load_dwordx4 v[32:35], %[vA0], %[rWS], s55 offen offset:3072 sc0 sc1\n\t" \
    "v_mfma_f32_16x16x32_f16 a[68:71], v[92:95], v[152:155], a[68:71]\n\t" \
    "v_mfma_f32_16x16x32_f16 a[100:103], v[96:99], v[152:155], a[100:103]\n\t" \
    "v_mfma_f32_16x16x32_f16 a[132:135], v[100:103], v[152:155], a[132:135]\n\t" \
    "v_mfma_f32_16x16x32_f16 a[164:167], v[104:107], v[152:155], a[164:167]\n\t" \
    "s_add_u32 s55, s38, 4096\n\t" \
    "v_mfma_f32_16x16x32_f16 a[196:199], v[108:111], v[152:155], a[196:199]\n\t" \
    "v_mfma_f32_16x16x32_f16 a[228:231], v[112:115], v[152:155], a[228:231]\n\t" \
    "ds_read_b128 v[152:155], %[vA0] offset:21504\n\t" \
    "s_waitcnt lgkmcnt(3)\n\t" \
    "v_mfma_f32_16x16x32_f16 a[232:235], v[112:115], v[156:159], a[232:235]\n\t" \
    "buffer_load_dwordx4 v[36:39], %[vA0], %[rWS], s55 offen offset:0 sc0 sc1\n\t" \
    "v_mfma_f32_16x16x32_f16 a[200:203], v[108:111], v[156:159], a[200:203]\n\t" \
    "v_mfma_f32_16x16x32_f16 a[168:171], v[104:107], v[156:159], a[168:171]\n\t" \
    "v_mfma_f32_16x16x32_f16 a[136:139], v[100:103], v[156:159], a[136:139]\n\t" \
    "v_mfma_f32_16x16x32_f16 a[104:107], v[96:99], v[156:159], a[104:107]\n\t" \
    "buffer_load_dwordx4 v[40:43], %[vA0], %[rWS], s55 offen offset:1024 sc0 sc1\n\t" \
    "v_mfma_f32_16x16x32_f16 a[72:75], v[92:95], v[156:159], a[72:75]\n\t" \
    "v_readlane_b32 s52, %[vTab0], s51\n\t" \
    "v_mfma_f32_16x16x32_f16 a[40:43], v[88:91], v[156:159], a[40:43]\n\t" \
    "s_add_u32 s42, s51, 32\n\t" \
    "v_mfma_f32_16x16x32_f16 a[8:11], v[84:87], v[156:159], a[8:11]\n\t" \
    "ds_read_b128 v[156:159], %[vA0] offset:22528\n\t" \
    "s_waitcnt lgkmcnt(3)\n\t" \
    "v_mfma_f32_16x16x32_f16 a[12:15], v[84:87], v[160:163], a[12:15]\n\t" \
    "v_readlane_b32 s53, %[vTab1], s51\n\t" \
    "v_mfma_f32_16x16x32_f16 a[44:47], v[88:91], v[160:163], a[44:47]\n\t" \
    "v_readlane_b32 s42, %[vTab0], s42\n\t" \
    "v_mfma_f32_16x16x32_f16 a[76:79], v[92:95], v[160:163], a[76:79]\n\t" \
    "s_add_u32 s51, s51, 1\n\t" \
    "v_mfma_f32_16x16x32_f16 a[108:111], v[96:99], v[160:163], a[108:111]\n\t" \
    "buffer_load_dwordx4 v[44:47], %[vA0], %[rWS], s55 offen offset:2048 sc0 sc1\n\t" \
    "v_mfma_f32_16x16x32_f16 a[140:143], v[100:103], v[160:163], a[140:143]\n\t" \
    "s_add_u32 s40, s66, s43\n\t" \
    "s_add_u32 s40, s40, %[sWk1]\n\t" \
    "v_mfma_f32_16x16x32_f16 a[172:175], v[104:107], v[160:163], a[172:175]\n\t" \
    "s_mov_b32 s56, s42\n\t" \
    "s_and_b32 s57, s53, 0xffff\n\t" \
    "v_mfma_f32_16x16x32_f16 a[204:207], v[108:111], v[160:163], a[204:207]\n\t" \
    "s_and_b32 s58, s53, 0x10000\n\t" \
    "s_bfe_i32 s53, s53, 0x10011\n\t" \
    "v_mfma_f32_16x16x32_f16 a[236:239], v[112:115], v[160:163], a[236:239]\n\t" \
    "ds_read_b128 v[160:163], %[vA0] offset:23552\n\t" \
    "s_waitcnt lgkmcnt(3)\n\t" \
    "v_mfma_f32_16x16x32_f16 a[240:243], v[112:115], v[148:151], a[240:243]\n\t" \
    "buffer_load_dwordx4 v[48:51], %[vA0], %[rWS], s55 offen offset:3072 sc0 sc1\n\t" \
    "v_mfma_f32_16x16x32_f16 a[208:211], v[108:111], v[148:151], a[208:211]\n\t" \
    "s_waitcnt vmcnt(5)\n\t" \
    "v_cmp_le_u32 s[76:77], s49, v19\n\t" \
    "v_mfma_f32_16x16x32_f16 a[176:179], v[104:107], v[148:151], a[176:179]\n\t" \
    "s_nop 1\n\t" \
    "s_or_b64 s[76:77], s[76:77], s[70:71]\n\t" \
    "v_mfma_f32_16x16x32_f16 a[144:147], v[100:103], v[148:151], a[144:147]\n\t" \
    "s_cmp_eq_u64 s[76:77], exec\n\t" \
    "s_cbranch_scc0 SLOW_P0%=\n\t" \
    "BACK_P0%=:\n\t" \
    "v_mfma_f32_16x16x32_f16 a[112:115], v[96:99], v[148:151], a[112:115]\n\t" \
    "buffer_load_dwordx4 v[196:199], %[vW], %[rW], s40 offen nt\n\t" \
    "v_mfma_f32_16x16x32_f16 a[80:83], v[92:95], v[148:151], a[80:83]\n\t" \
    "v_mfma_f32_16x16x32_f16 a[48:51], v[88:91], v[148:151], a[48:51]\n\t" \
    "s_add_u32 s40, s40, 32768\n\t" \
    "v_mfma_f32_16x16x32_f16 a[16:19], v[84:87], v[148:151], a[16:19]\n\t" \
    "ds_read_b128 v[148:151], %[vA0] offset:24576\n\t" \
    "s_waitcnt lgkmcnt(3)\n\t" \
    "v_mfma_f32_16x16x32_f16 a[20:23], v[84:87], v[152:155], a[20:23]\n\t" \
    "s_cmp_eq_u32 s69, 0\n\t" \
    "s_cselect_b32 s42, 1073741824, s52\n\t" \
    "v_mfma_f32_16x16x32_f16 a[52:55], v[88:91], v[152:155], a[52:55]\n\t" \
    "buffer_load_dwordx4 v[200:203], %[vW], %[rW], s40 offen nt\n\t" \
    "v_mfma_f32_16x16x32_f16 a[84:87], v[92:95], v[152:155], a[84:87]\n\t" \
    "buffer_load_dwordx4 v[236:239], %[vA0], %[rWS], s42 offen sc0 sc1\n\t" \
    "v_mfma_f32_16x16x32_f16 a[116:119], v[96:99], v[152:155], a[116:119]\n\t" \
    "s_add_u32 s52, s52, 262144\n\t" \
    "s_cmp_eq_u32 s69, 1\n\t" \
    "v_mfma_f32_16x16x32_f16 a[148:151], v[100:103], v[152:155], a[148:151]\n\t" \
    "s_cselect_b32 s42, 1073741824, s52\n\t" \
    "v_mfma_f32_16x16x32_f16 a[180:183], v[104:107], v[152:155], a[180:183]\n\t" \
    "buffer_load_dwordx4 v[240:243], %[vA0], %[rWS], s42 offen sc0 sc1\n\t" \
    "v_mfma_f32_16x16x32_f16 a[212:215], v[108:111], v[152:155], a[212:215]\n\t" \
    "s_add_u32 s40, s40, 32768\n\t" \
    "s_add_u32 s52, s52, 262144\n\t" \
    "v_mfma_f32_16x16x32_f16 a[244:247], v[112:115], v[152:155], a[244:247]\n\t" \
    "ds_read_b128 v[152:155], %[vA0] offset:25600\n\t" \
    "s_waitcnt lgkmcnt(3)\n\t" \
    "v_mfma_f32_16x16x32_f16 a[248:251], v[112:115], v[156:159], a[248:251]\n\t" \
    "s_cmp_eq_u32 s69, 2\n\t" \
    "s_cselect_b32 s42, 1073741824, s52\n\t" \
    "v_mfma_f32_16x16x32_f16 a[216:219], v[108:111], v[156:159], a[216:219]\n\t" \
    "buffer_load_dwordx4 v[204:207], %[vW], %[rW], s40 offen nt\n\t" \
    "v_mfma_f32_16x16x32_f16 a[184:187], v[104:107], v[156:159], a[184:187]\n\t" \
    "buffer_load_dwordx4 v[244:247], %[vA0], %[rWS], s42 offen sc0 sc1\n\t" \
    "v_mfma_f32_16x16x32_f16 a[152:155], v[100:103], v[156:159], a[152:155]\n\t" \
    "s_add_u32 s52, s52, 262144\n\t" \
    "s_cmp_eq_u32 s69, 3\n\t" \
    "v_mfma_f32_16x16x32_f16 a[120:123], v[96:99], v[156:159], a[120:123]\n\t" \
    "s_cselect_b32 s42, 1073741824, s52\n\t" \
    "s_add_u32 s40, s40, 32768\n\t" \
    "v_mfma_f32_16x16x32_f16 a[88:91], v[92:95], v[156:159], a[88:91]\n\t" \
    "buffer_load_dwordx4 v[248:251], %[vA0], %[rWS], s42 offen sc0 sc1\n\t" \
    "v_mfma_f32_16x16x32_f16 a[56:59], v[88:91], v[156:159], a[56:59]\n\t" \
    "buffer_load_dwordx4 v[208:211], %[vW], %[rW], s40 offen nt\n\t" \
    "v_mfma_f32_16x16x32_f16 a[24:27], v[84:87], v[156:159], a[24:27]\n\t" \
    "ds_read_b128 v[156:159], %[vA0] offset:26624\n\t" \
    "s_waitcnt lgkmcnt(3)\n\t" \
    "v_mfma_f32_16x16x32_f16 a[28:31], v[84:87], v[160:163], a[28:31]\n\t" \
    "s_add_u32 s40, s40, 32768\n\t" \
    "v_mfma_f32_16x16x32_f16 a[60:63], v[88:91], v[160:163], a[60:63]\n\t" \
    "buffer_load_dwordx4 v[212:215], %[vW], %[rW], s40 offen nt\n\t" \
    "v_mfma_f32_16x16x32_f16 a[92:95], v[92:95], v[160:163], a[92:95]\n\t" \
    "v_mfma_f32_16x16x32_f16 a[124:127], v[96:99], v[160:163], a[124:127]\n\t" \
    "s_add_u32 s40, s40, 32768\n\t" \
    "v_mfma_f32_16x16x32_f16 a[156:159], v[100:103], v[160:163], a[156:159]\n\t" \
    "v_mfma_f32_16x16x32_f16 a[188:191], v[104:107], v[160:163], a[188:191]\n\t" \
    "buffer_load_dwordx4 v[216:219], %[vW], %[rW], s40 offen nt\n\t" \
    "v_mfma_f32_16x16x32_f16 a[220:223], v[108:111], v[160:163], a[220:223]\n\t" \
    "v_mfma_f32_16x16x32_f16 a[252:255], v[112:115], v[160:163], a[252:255]\n\t" \
    "ds_read_b128 v[160:163], %[vA0] offset:27648\n\t" \
    "s_waitcnt lgkmcnt(3)\n\t" \
    "v_mfma_f32_16x16x32_f16 a[224:227], v[144:147], v[148:151], a[224:227]\n\t" \
    "s_add_u32 s40, s40, 32768\n\t" \
    "v_mfma_f32_16x16x32_f16 a[192:195], v[140:143], v[148:151], a[192:195]\n\t" \
    "v_mfma_f32_16x16x32_f16 a[160:163], v[136:139], v[148:151], a[160:163]\n\t" \
    "buffer_load_dwordx4 v[220:223], %[vW], %[rW], s40 offen nt\n\t" \
    "v_mfma_f32_16x16x32_f16 a[128:131], v[132:135], v[148:151], a[128:131]\n\t" \
    "v_mfma_f32_16x16x32_f16 a[96:99], v[128:131], v[148:151], a[96:99]\n\t" \
    "v_mfma_f32_16x16x32_f16 a[64:67], v[124:127], v[148:151], a[64:67]\n\t" \
    "v_mfma_f32_16x16x32_f16 a[32:35], v[120:123], v[148:151], a[32:35]\n\t" \
    "s_add_u32 s40, s40, 32768\n\t" \
    "v_mfma_f32_16x16x32_f16 a[0:3], v[116:119], v[148:151], a[0:3]\n\t" \
    "ds_read_b128 v[148:151], %[vA0] offset:28672\n\t" \
    "s_waitcnt lgkmcnt(3)\n\t" \
    "v_mfma_f32_16x16x32_f16 a[4:7], v[116:119], v[152:155], a[4:7]\n\t" \
    "buffer_load_dwordx4 v[224:227], %[vW], %[rW], s40 offen nt\n\t" \
    "v_mfma_f32_16x16x32_f16 a[36:39], v[120:123], v[152:155], a[36:39]\n\t" \
    "v_mfma_f32_16x16x32_f16 a[68:71], v[124:127], v[152:155], a[68:71]\n\t" \
    "v_mfma_f32_16x16x32_f16 a[100:103], v[128:131], v[152:155], a[100:103]\n\t" \
    "s_add_u32 s55, s38, 8192\n\t" \
    "v_mfma_f32_16x16x32_f16 a[132:135], v[132:135], v[152:155], a[132:135]\n\t" \
    "v_and_b32 v252, s53, v252\n\t" \
    "v_mfma_f32_16x16x32_f16 a[164:167], v[136:139], v[152:155], a[164:167]\n\t" \
    "buffer_load_dwordx4 v[52:55], %[vA0], %[rWS], s55 offen offset:0 sc0 sc1\n\t" \
    "v_mfma_f32_16x16x32_f16 a[196:199], v[140:143], v[152:155], a[196:199]\n\t" \
    "v_and_b32 v253, s53, v253\n\t" \
    "v_and_b32 v254, s53, v254\n\t" \
    "v_mfma_f32_16x16x32_f16 a[228:231], v[144:147], v[152:155], a[228:231]\n\t" \
    "ds_read_b128 v[152:155], %[vA0] offset:29696\n\t" \
    "s_waitcnt lgkmcnt(3)\n\t" \
    "v_mfma_f32_16x16x32_f16 a[232:235], v[144:147], v[156:159], a[232:235]\n\t" \
    "v_and_b32 v255, s53, v255\n\t" \
    "s_waitcnt vmcnt(9)\n\t" \
    "v_add_f32 v236, v236, v240\n\t" \
    "v_mfma_f32_16x16x32_f16 a[200:203], v[140:143], v[156:159], a[200:203]\n\t" \
    "v_add_f32 v237, v237, v241\n\t" \
    "v_mfma_f32_16x16x32_f16 a[168:171], v[136:139], v[156:159], a[168:171]\n\t" \
    "buffer_load_dwordx4 v[56:59], %[vA0], %[rWS], s55 offen offset:1024 sc0 sc1\n\t" \
    "v_mfma_f32_16x16x32_f16 a[136:139], v[132:135], v[156:159], a[136:139]\n\t" \
    "v_add_f32 v238, v238, v242\n\t" \
    "v_add_f32 v239, v239, v243\n\t" \
    "v_mfma_f32_16x16x32_f16 a[104:107], v[128:131], v[156:159], a[104:107]\n\t" \
    "v_add_f32 v252, v252, v236\n\t" \
    "v_mfma_f32_16x16x32_f16 a[72:75], v[124:127], v[156:159], a[72:75]\n\t" \
    "v_add_f32 v253, v253, v237\n\t" \
    "v_mfma_f32_16x16x32_f16 a[40:43], v[120:123], v[156:159], a[40:43]\n\t" \
    "v_add_f32 v254, v254, v238\n\t" \
    "v_mfma_f32_16x16x32_f16 a[8:11], v[116:119], v[156:159], a[8:11]\n\t" \
    "ds_read_b128 v[156:159], %[vA0] offset:30720\n\t" \
    "s_waitcnt lgkmcnt(3)\n\t" \
    "v_mfma_f32_16x16x32_f16 a[12:15], v[116:119], v[160:163], a[12:15]\n\t" \
    "buffer_load_dwordx4 v[60:63], %[vA0], %[rWS], s55 offen offset:2048 sc0 sc1\n\t" \
    "v_mfma_f32_16x16x32_f16 a[44:47], v[120:123], v[160:163], a[44:47]\n\t" \
    "v_add_f32 v255, v255, v239\n\t" \
    "s_waitcnt vmcnt(8)\n\t" \
    "v_add_f32 v244, v244, v248\n\t" \
    "v_mfma_f32_16x16x32_f16 a[76:79], v[124:127], v[160:163], a[76:79]\n\t" \
    "v_add_f32 v245, v245, v249\n\t" \
    "v_add_f32 v246, v246, v250\n\t" \
    "v_mfma_f32_16x16x32_f16 a[108:111], v[128:131], v[160:163], a[108:111]\n\t" \
    "v_add_f32 v247, v247, v251\n\t" \
    "v_mfma_f32_16x16x32_f16 a[140:143], v[132:135], v[160:163], a[140:143]\n\t" \
    "buffer_load_dwordx4 v[64:67], %[vA0], %[rWS], s55 offen offset:3072 sc0 sc1\n\t" \
    "v_mfma_f32_16x16x32_f16 a[172:175], v[136:139], v[160:163], a[172:175]\n\t" \
    "v_add_f32 v252, v252, v244\n\t" \
    "v_add_f32 v253, v253, v245\n\t" \
    "v_mfma_f32_16x16x32_f16 a[204:207], v[140:143], v[160:163], a[204:207]\n\t" \
    "v_add_f32 v254, v254, v246\n\t" \
    "v_mfma_f32_16x16x32_f16 a[236:239], v[144:147], v[160:163], a[236:239]\n\t" \
    "ds_read_b128 v[160:163], %[vA0] offset:31744\n\t" \
    "s_waitcnt lgkmcnt(3)\n\t" \
    "v_mfma_f32_16x16x32_f16 a[240:243], v[144:147], v[148:151], a[240:243]\n\t" \
    "s_add_u32 s55, s38, 12288\n\t" \
    "v_add_f32 v255, v255, v247\n\t" \
    "v_mfma_f32_16x16x32_f16 a[208:211], v[140:143], v[148:151], a[208:211]\n\t" \
    "v_med3_f32 v236, v252, %[m60], %[p60]\n\t" \
    "v_mfma_f32_16x16x32_f16 a[176:179], v[136:139], v[148:151], a[176:179]\n\t" \
    "buffer_load_dwordx4 v[68:71], %[vA0], %[rWS], s55 offen offset:0 sc0 sc1\n\t" \
    "v_mfma_f32_16x16x32_f16 a[144:147], v[132:135], v[148:151], a[144:147]\n\t" \
    "v_med3_f32 v237, v253, %[m60], %[p60]\n\t" \
    "v_med3_f32 v238, v254, %[m60], %[p60]\n\t" \
    "v_mfma_f32_16x16x32_f16 a[112:115], v[128:131], v[148:151], a[112:115]\n\t" \
    "v_med3_f32 v239, v255, %[m60], %[p60]\n\t" \
    "v_mul_f32 v236, 0xbfb8aa3b, v236\n\t" \
    "v_mfma_f32_16x16x32_f16 a[80:83], v[124:127], v[148:151], a[80:83]\n\t" \
    "v_mul_f32 v237, 0xbfb8aa3b, v237\n\t" \
    "v_mfma_f32_16x16x32_f16 a[48:51], v[120:123], v[148:151], a[48:51]\n\t" \
    "v_mul_f32 v238, 0xbfb8aa3b, v238\n\t" \
    "v_mfma_f32_16x16x32_f16 a[16:19], v[116:119], v[148:151], a[16:19]\n\t" \
    "s_waitcnt lgkmcnt(0)\n\t" \
    "s_barrier\n\t" \
    "ds_read_b128 v[148:151], %[vA0] offset:32768\n\t" \
    "v_mfma_f32_16x16x32_f16 a[20:23], v[116:119], v[152:155], a[20:23]\n\t" \
    "buffer_load_dwordx4 v[72:75], %[vA0], %[rWS], s55 offen offset:1024 sc0 sc1\n\t" \
    "v_mfma_f32_16x16x32_f16 a[52:55], v[120:123], v[152:155], a[52:55]\n\t" \
    "v_mul_f32 v239, 0xbfb8aa3b, v239\n\t" \
    "v_mfma_f32_16x16x32_f16 a[84:87], v[124:127], v[152:155], a[84:87]\n\t" \
    "v_exp_f32 v236, v236\n\t" \
    "v_mfma_f32_16x16x32_f16 a[116:119], v[128:131], v[152:155], a[116:119]\n\t" \
    "v_exp_f32 v237, v237\n\t" \
    "v_mfma_f32_16x16x32_f16 a[148:151], v[132:135], v[152:155], a[148:151]\n\t" \
    "v_exp_f32 v238, v238\n\t" \
    "v_mfma_f32_16x16x32_f16 a[180:183], v[136:139], v[152:155], a[180:183]\n\t" \
    "v_exp_f32 v239, v239\n\t" \
    "v_mfma_f32_16x16x32_f16 a[212:215], v[140:143], v[152:155], a[212:215]\n\t" \
    "buffer_load_dwordx4 v[76:79], %[vA0], %[rWS], s55 offen offset:2048 sc0 sc1\n\t" \
    "v_mfma_f32_16x16x32_f16 a[244:247], v[144:147], v[152:155], a[244:247]\n\t" \
    "ds_read_b128 v[152:155], %[vA0] offset:33792\n\t" \
    "v_mfma_f32_16x16x32_f16 a[248:251], v[144:147], v[156:159], a[248:251]\n\t" \
    "v_add_f32 v236, 1.0, v236\n\t" \
    "v_add_f32 v237, 1.0, v237\n\t" \
    "v_mfma_f32_16x16x32_f16 a[216:219], v[140:143], v[156:159], a[216:219]\n\t" \
    "v_add_f32 v238, 1.0, v238\n\t" \
    "v_add_f32 v239, 1.0, v239\n\t" \
    "v_mfma_f32_16x16x32_f16 a[184:187], v[136:139], v[156:159], a[184:187]\n\t" \
    "v_rcp_f32 v236, v236\n\t" \
    "v_mfma_f32_16x16x32_f16 a[152:155], v[132:135], v[156:159], a[152:155]\n\t" \
    "buffer_load_dwordx4 v[80:83], %[vA0], %[rWS], s55 offen offset:3072 sc0 sc1\n\t" \
    "v_mfma_f32_16x16x32_f16 a[120:123], v[128:131], v[156:159], a[120:123]\n\t" \
    "v_rcp_f32 v237, v237\n\t" \
    "v_mfma_f32_16x16x32_f16 a[88:91], v[124:127], v[156:159], a[88:91]\n\t" \
    "s_add_u32 s55, s38, 16384\n\t" \
    "v_mfma_f32_16x16x32_f16 a[56:59], v[120:123], v[156:159], a[56:59]\n\t" \
    "v_rcp_f32 v238, v238\n\t" \
    "v_mfma_f32_16x16x32_f16 a[24:27], v[116:119], v[156:159], a[24:27]\n\t" \
    "ds_read_b128 v[156:159], %[vA0] offset:34816\n\t" \
    "v_mfma_f32_16x16x32_f16 a[28:31], v[116:119], v[160:163], a[28:31]\n\t" \
    "v_rcp_f32 v239, v239\n\t" \
    "v_mfma_f32_16x16x32_f16 a[60:63], v[120:123], v[160:163], a[60:63]\n\t" \
    "buffer_load_dwordx4 v[84:87], %[vA0], %[rWS], s55 offen offset:0 sc0 sc1\n\t" \
    "v_mfma_f32_16x16x32_f16 a[92:95], v[124:127], v[160:163], a[92:95]\n\t" \
    "v_mul_f32 v232, v252, v236\n\t" \
    "v_mul_f32 v233, v253, v237\n\t" \
    "v_mfma_f32_16x16x32_f16 a[124:127], v[128:131], v[160:163], a[124:127]\n\t" \
    "v_mul_f32 v234, v254, v238\n\t" \
    "v_mfma_f32_16x16x32_f16 a[156:159], v[132:135], v[160:163], a[156:159]\n\t" \
    "buffer_load_dwordx4 v[88:91], %[vA0], %[rWS], s55 offen offset:1024 sc0 sc1\n\t" \
    "v_mfma_f32_16x16x32_f16 a[188:191], v[136:139], v[160:163], a[188:191]\n\t" \
    "v_mul_f32 v235, v255, v239\n\t" \
    "v_mfma_f32_16x16x32_f16 a[220:223], v[140:143], v[160:163], a[220:223]\n\t" \
    "buffer_load_dwordx4 v[92:95], %[vA0], %[rWS], s55 offen offset:2048 sc0 sc1\n\t" \
    "v_mfma_f32_16x16x32_f16 a[252:255], v[144:147], v[160:163], a[252:255]\n\t" \
    "buffer_load_dwordx4 v[96:99], %[vA0], %[rWS], s55 offen offset:3072 sc0 sc1\n\t" \
    "s_add_u32 s36, s36, 1\n\t" \
    "ds_read_b128 v[160:163], %[vA0] offset:35840\n\t" \
    "s_waitcnt lgkmcnt(3)\n\t" \
    "v_mfma_f32_16x16x32_f16 a[224:227], v[48:51], v[148:151], a[224:227]\n\t" \
    "s_mov_b32 s37, s38\n\t" \
    "v_cvt_pk_f16_f32 v228, v164, v168\n\t" \
    "v_mfma_f32_16x16x32_f16 a[192:195], v[44:47], v[148:151], a[192:195]\n\t" \
    "v_mfma_f32_16x16x32_f16 a[160:163], v[40:43], v[148:151], a[160:163]\n\t" \
    "v_cvt_pk_f16_f32 v229, v172, v176\n\t" \
    "v_mfma_f32_16x16x32_f16 a[128:131], v[36:39], v[148:151], a[128:131]\n\t" \
    "ds_write_b64 %[vWr0], v[228:229] offset:0\n\t" \
    "v_mfma_f32_16x16x32_f16 a[96:99], v[32:35], v[148:151], a[96:99]\n\t" \
    "v_cvt_pk_f16_f32 v230, v180, v184\n\t" \
    "v_mfma_f32_16x16x32_f16 a[64:67], v[28:31], v[148:151], a[64:67]\n\t" \
    "v_mfma_f32_16x16x32_f16 a[32:35], v[24:27], v[148:151], a[32:35]\n\t" \
    "v_cvt_pk_f16_f32 v231, v188, v192\n\t" \
    "v_mfma_f32_16x16x32_f16 a[0:3], v[20:23], v[148:151], a[0:3]\n\t" \
    "ds_read_b128 v[148:151], %[vA0] offset:36864\n\t" \
    "s_waitcnt lgkmcnt(4)\n\t" \
    "v_mfma_f32_16x16x32_f16 a[4:7], v[20:23], v[152:155], a[4:7]\n\t" \
    "s_cmp_ge_u32 s48, 2\n\ts_cselect_b64 exec, -1, 0\n\ts_cselect_b32 s69, %[own], 9\n\tds_read_b128 v[252:255], %[vStg] offset:17408\n\ts_mov_b64 exec, -1\n\t" \
    "s_waitcnt lgkmcnt(4)\n\t" \
    "v_mfma_f32_16x16x32_f16 a[36:39], v[24:27], v[152:155], a[36:39]\n\t" \
    "ds_write_b64 %[vWr0], v[230:231] offset:8\n\t" \
    "v_mfma_f32_16x16x32_f16 a[68:71], v[28:31], v[152:155], a[68:71]\n\t" \
    "v_cvt_pk_f16_f32 v228, v165, v169\n\t" \
    "s_add_u32 s55, s37, 20480\n\t" \
    "v_mfma_f32_16x16x32_f16 a[100:103], v[32:35], v[152:155], a[100:103]\n\t" \
    "v_cvt_pk_f16_f32 v229, v173, v177\n\t" \
    "v_mfma_f32_16x16x32_f16 a[132:135], v[36:39], v[152:155], a[132:135]\n\t" \
    "buffer_load_dwordx4 v[100:103], %[vA0], %[rWS], s55 offen offset:0 sc0 sc1\n\t" \
    "v_mfma_f32_16x16x32_f16 a[164:167], v[40:43], v[152:155], a[164:167]\n\t" \
    "ds_write_b64 %[vWr0], v[228:229] offset:1024\n\t" \
    "v_mfma_f32_16x16x32_f16 a[196:199], v[44:47], v[152:155], a[196:199]\n\t" \
    "v_cvt_pk_f16_f32 v230, v181, v185\n\t" \
    "v_cvt_pk_f16_f32 v231, v189, v193\n\t" \
    "v_mfma_f32_16x16x32_f16 a[228:231], v[48:51], v[152:155], a[228:231]\n\t" \
    "ds_read_b128 v[152:155], %[vA0] offset:37888\n\t" \
    "s_waitcnt lgkmcnt(6)\n\t" \
    "v_mfma_f32_16x16x32_f16 a[232:235], v[48:51], v[156:159], a[232:235]\n\t" \
    "ds_write_b64 %[vWr0], v[230:231] offset:1032\n\t" \
    "v_mfma_f32_16x16x32_f16 a[200:203], v[44:47], v[156:159], a[200:203]\n\t" \
    "buffer_load_dwordx4 v[104:107], %[vA0], %[rWS], s55 offen offset:1024 sc0 sc1\n\t" \
    "v_mfma_f32_16x16x32_f16 a[168:171], v[40:43], v[156:159], a[168:171]\n\t" \
    "v_cvt_pk_f16_f32 v228, v166, v170\n\t" \
    "v_cvt_pk_f16_f32 v229, v174, v178\n\t" \
    "v_mfma_f32_16x16x32_f16 a[136:139], v[36:39], v[156:159], a[136:139]\n\t" \
    "ds_write_b64 %[vWr0], v[228:229] offset:2048\n\t" \
    "v_mfma_f32_16x16x32_f16 a[104:107], v[32:35], v[156:159], a[104:107]\n\t" \
    "v_cvt_pk_f16_f32 v230, v182, v186\n\t" \
    "v_mfma_f32_16x16x32_f16 a[72:75], v[28:31], v[156:159], a[72:75]\n\t" \
    "buffer_load_dwordx4 v[108:111], %[vA0], %[rWS], s55 offen offset:2048 sc0 sc1\n\t" \
    "v_mfma_f32_16x16x32_f16 a[40:43], v[24:27], v[156:159], a[40:43]\n\t" \
    "s_add_u32 s66, s66, 0x400000\n\t" \
    "s_and_b32 s66, s66, 0xfffffff\n\t" \
    "v_mfma_f32_16x16x32_f16 a[8:11], v[20:23], v[156:159], a[8:11]\n\t" \
    "ds_read_b128 v[156:159], %[vA0] offset:38912\n\t" \
    "s_waitcnt lgkmcnt(8)\n\t" \
    "v_mfma_f32_16x16x32_f16 a[12:15], v[20:23], v[160:163], a[12:15]\n\t" \
    "v_cvt_pk_f16_f32 v231, v190, v194\n\t" \
    "v_mfma_f32_16x16x32_f16 a[44:47], v[24:27], v[160:163], a[44:47]\n\t" \
    "ds_write_b64 %[vWr0], v[230:231] offset:2056\n\t" \
    "v_mfma_f32_16x16x32_f16 a[76:79], v[28:31], v[160:163], a[76:79]\n\t" \
    "v_cvt_pk_f16_f32 v228, v167, v171\n\t" \
    "s_add_u32 s39, s66, s43\n\t" \
    "v_mfma_f32_16x16x32_f16 a[108:111], v[32:35], v[160:163], a[108:111]\n\t" \
    "s_add_u32 s39, s39, %[sWk0]\n\t" \
    "v_mfma_f32_16x16x32_f16 a[140:143], v[36:39], v[160:163], a[140:143]\n\t" \
    "buffer_load_dwordx4 v[112:115], %[vA0], %[rWS], s55 offen offset:3072 sc0 sc1\n\t" \
    "v_mfma_f32_16x16x32_f16 a[172:175], v[40:43], v[160:163], a[172:175]\n\t" \
    "v_cvt_pk_f16_f32 v229, v175, v179\n\t" \
    "v_mfma_f32_16x16x32_f16 a[204:207], v[44:47], v[160:163], a[204:207]\n\t" \
    "ds_write_b64 %[vWr0], v[228:229] offset:3072\n\t" \
    "v_mfma_f32_16x16x32_f16 a[236:239], v[48:51], v[160:163], a[236:239]\n\t" \
    "ds_read_b128 v[160:163], %[vA0] offset:39936\n\t" \
    "s_waitcnt lgkmcnt(9)\n\t" \
    "v_mfma_f32_16x16x32_f16 a[240:243], v[48:51], v[148:151], a[240:243]\n\t" \
    "v_cvt_pk_f16_f32 v230, v183, v187\n\t" \
    "v_cvt_pk_f16_f32 v231, v191, v195\n\t" \
    "v_mfma_f32_16x16x32_f16 a[208:211], v[44:47], v[148:151], a[208:211]\n\t" \
    "ds_write_b64 %[vWr0], v[230:231] offset:3080\n\t" \
    "v_mfma_f32_16x16x32_f16 a[176:179], v[40:43], v[148:151], a[176:179]\n\t" \
    "buffer_load_dwordx4 v[164:167], %[vW], %[rW], s39 offen\n\t" \
    "v_mfma_f32_16x16x32_f16 a[144:147], v[36:39], v[148:151], a[144:147]\n\t" \
    "s_add_u32 s38, s38, 32768\n\t" \
    "s_and_b32 s38, s38, 0x1fffff\n\t" \
    "v_mfma_f32_16x16x32_f16 a[112:115], v[32:35], v[148:151], a[112:115]\n\t" \
    "s_or_b32 s38, s38, %[sA0]\n\t" \
    "s_add_u32 s39, s39, 32768\n\t" \
    "v_mfma_f32_16x16x32_f16 a[80:83], v[28:31], v[148:151], a[80:83]\n\t" \
    "s_add_u32 s41, %[own], 1\n\t" \
    "s_and_b32 s41, s41, 3\n\t" \
    "v_mfma_f32_16x16x32_f16 a[48:51], v[24:27], v[148:151], a[48:51]\n\t" \
    "s_lshl_b32 s41, s41, 12\n\t" \
    "v_add_u32 v18, s41, %[vStg]\n\t" \
    "v_mfma_f32_16x16x32_f16 a[16:19], v[20:23], v[148:151], a[16:19]\n\t" \
    "ds_read_b128 v[148:151], %[vA0] offset:40960\n\t" \
    "s_waitcnt lgkmcnt(8)\n\t" \
    "v_mfma_f32_16x16x32_f16 a[20:23], v[20:23], v[152:155], a[20:23]\n\t" \
    "s_lshl_b32 s41, s41, 1\n\t" \
    "s_add_u32 s41, s41, s50\n\t" \
    "v_mfma_f32_16x16x32_f16 a[52:55], v[24:27], v[152:155], a[52:55]\n\t" \
    "s_add_u32 s41, s41, 4096\n\t" \
    "s_add_u32 s60, %[wsLo], s41\n\t" \
    "v_mfma_f32_16x16x32_f16 a[84:87], v[28:31], v[152:155], a[84:87]\n\t" \
    "s_addc_u32 s61, %[wsHi], 0\n\t" \
    "v_mfma_f32_16x16x32_f16 a[116:119], v[32:35], v[152:155], a[116:119]\n\t" \
    "buffer_load_dwordx4 v[168:171], %[vW], %[rW], s39 offen\n\t" \
    "v_mfma_f32_16x16x32_f16 a[148:151], v[36:39], v[152:155], a[148:151]\n\t" \
    "s_add_u32 s39, s39, 32768\n\t" \
    "v_mfma_f32_16x16x32_f16 a[180:183], v[40:43], v[152:155], a[180:183]\n\t" \
    "buffer_load_dwordx4 v[172:175], %[vW], %[rW], s39 offen\n\t" \
    "v_mfma_f32_16x16x32_f16 a[212:215], v[44:47], v[152:155], a[212:215]\n\t" \
    "ds_read_b128 v[236:239], v18 offset:0\n\t" \
    "v_mfma_f32_16x16x32_f16 a[244:247], v[48:51], v[152:155], a[244:247]\n\t" \
    "ds_read_b128 v[152:155], %[vA0] offset:41984\n\t" \
    "s_waitcnt lgkmcnt(7)\n\t" \
    "v_mfma_f32_16x16x32_f16 a[248:251], v[48:51], v[156:159], a[248:251]\n\t" \
    "ds_read_b128 v[240:243], v18 offset:1024\n\t" \
    "v_mfma_f32_16x16x32_f16 a[216:219], v[44:47], v[156:159], a[216:219]\n\t" \
    "s_add_u32 s39, s39, 32768\n\t" \
    "v_mfma_f32_16x16x32_f16 a[184:187], v[40:43], v[156:159], a[184:187]\n\t" \
    "ds_read_b128 v[244:247], v18 offset:2048\n\t" \
    "v_mfma_f32_16x16x32_f16 a[152:155], v[36:39], v[156:159], a[152:155]\n\t" \
    "ds_read_b128 v[248:251], v18 offset:3072\n\t" \
    "v_mfma_f32_16x16x32_f16 a[120:123], v[32:35], v[156:159], a[120:123]\n\t" \
    "buffer_load_dwordx4 v[176:179], %[vW], %[rW], s39 offen\n\t" \
    "v_mfma_f32_16x16x32_f16 a[88:91], v[28:31], v[156:159], a[88:91]\n\t" \
    "s_add_u32 s39, s39, 32768\n\t" \
    "v_mfma_f32_16x16x32_f16 a[56:59], v[24:27], v[156:159], a[56:59]\n\t" \
    "buffer_store_dwordx4 v[232:235], %[vO], s[56:59], 0 offen sc0 sc1 nt\n\t" \
    "v_mfma_f32_16x16x32_f16 a[24:27], v[20:23], v[156:159], a[24:27]\n\t" \
    "ds_read_b128 v[156:159], %[vA0] offset:43008\n\t" \
    "s_waitcnt lgkmcnt(8)\n\t" \
    "v_mfma_f32_16x16x32_f16 a[28:31], v[20:23], v[160:163], a[28:31]\n\t" \
    "buffer_load_dwordx4 v[180:183], %[vW], %[rW], s39 offen\n\t" \
    "v_mfma_f32_16x16x32_f16 a[60:63], v[24:27], v[160:163], a[60:63]\n\t" \
    "s_add_u32 s39, s39, 32768\n\t" \
    "v_mfma_f32_16x16x32_f16 a[92:95], v[28:31], v[160:163], a[92:95]\n\t" \
    "buffer_load_dwordx4 v[184:187], %[vW], %[rW], s39 offen\n\t" \
    "v_mfma_f32_16x16x32_f16 a[124:127], v[32:35], v[160:163], a[124:127]\n\t" \
    "s_waitcnt lgkmcnt(3)\n\t" \
    "v_swap_b32 v237, v240\n\t" \
    "v_mfma_f32_16x16x32_f16 a[156:159], v[36:39], v[160:163], a[156:159]\n\t" \
    "s_waitcnt lgkmcnt(2)\n\t" \
    "v_swap_b32 v238, v244\n\t" \
    "v_mfma_f32_16x16x32_f16 a[188:191], v[40:43], v[160:163], a[188:191]\n\t" \
    "s_add_u32 s39, s39, 32768\n\t" \
    "v_mfma_f32_16x16x32_f16 a[220:223], v[44:47], v[160:163], a[220:223]\n\t" \
    "s_waitcnt lgkmcnt(1)\n\t" \
    "v_swap_b32 v239, v248\n\t" \
    "v_mfma_f32_16x16x32_f16 a[252:255], v[48:51], v[160:163], a[252:255]\n\t" \
    "ds_read_b128 v[160:163], %[vA0] offset:44032\n\t" \
    "s_waitcnt vmcnt(15)\n\t" \
    "v_mfma_f32_16x16x32_f16 a[224:227], v[80:83], v[148:151], a[224:227]\n\t" \
    "v_swap_b32 v242, v245\n\t" \
    "v_mfma_f32_16x16x32_f16 a[192:195], v[76:79], v[148:151], a[192:195]\n\t" \
    "buffer_load_dwordx4 v[188:191], %[vW], %[rW], s39 offen\n\t" \
    "v_mfma_f32_16x16x32_f16 a[160:163], v[72:75], v[148:151], a[160:163]\n\t" \
    "v_swap_b32 v243, v249\n\t" \
    "v_mfma_f32_16x16x32_f16 a[128:131], v[68:71], v[148:151], a[128:131]\n\t" \
    "v_swap_b32 v247, v250\n\t" \
    "v_mfma_f32_16x16x32_f16 a[96:99], v[64:67], v[148:151], a[96:99]\n\t" \
    "s_add_u32 s39, s39, 32768\n\t" \
    "v_mfma_f32_16x16x32_f16 a[64:67], v[60:63], v[148:151], a[64:67]\n\t" \
    "global_store_dwordx4 %[vA0], v[236:239], s[60:61] offset:0 sc0 sc1\n\t" \
    "v_mfma_f32_16x16x32_f16 a[32:35], v[56:59], v[148:151], a[32:35]\n\t" \
    "buffer_load_dwordx4 v[192:195], %[vW], %[rW], s39 offen\n\t" \
    "v_mfma_f32_16x16x32_f16 a[0:3], v[52:55], v[148:151], a[0:3]\n\t" \
    "ds_read_b128 v[148:151], %[vA0] offset:45056\n\t" \
    "v_mfma_f32_16x16x32_f16 a[4:7], v[52:55], v[152:155], a[4:7]\n\t" \
    "v_cvt_pk_f16_f32 v228, v196, v200\n\t" \
    "v_mfma_f32_16x16x32_f16 a[36:39], v[56:59], v[152:155], a[36:39]\n\t" \
    "global_store_dwordx4 %[vA0], v[240:243], s[60:61] offset:1024 sc0 sc1\n\t" \
    "v_mfma_f32_16x16x32_f16 a[68:71], v[60:63], v[152:155], a[68:71]\n\t" \
    "v_cvt_pk_f16_f32 v229, v204, v208\n\t" \
    "v_mfma_f32_16x16x32_f16 a[100:103], v[64:67], v[152:155], a[100:103]\n\t" \
    "ds_write_b64 %[vWr1], v[228:229] offset:0\n\t" \
    "v_mfma_f32_16x16x32_f16 a[132:135], v[68:71], v[152:155], a[132:135]\n\t" \
    "v_cvt_pk_f16_f32 v230, v212, v216\n\t" \
    "v_mfma_f32_16x16x32_f16 a[164:167], v[72:75], v[152:155], a[164:167]\n\t" \
    "global_store_dwordx4 %[vA0], v[244:247], s[60:61] offset:2048 sc0 sc1\n\t" \
    "v_mfma_f32_16x16x32_f16 a[196:199], v[76:79], v[152:155], a[196:199]\n\t" \
    "v_cvt_pk_f16_f32 v231, v220, v224\n\t" \
    "v_mfma_f32_16x16x32_f16 a[228:231], v[80:83], v[152:155], a[228:231]\n\t" \
    "ds_read_b128 v[152:155], %[vA0] offset:46080\n\t" \
    "s_waitcnt lgkmcnt(4)\n\t" \
    "v_mfma_f32_16x16x32_f16 a[232:235], v[80:83], v[156:159], a[232:235]\n\t" \
    "ds_write_b64 %[vWr1], v[230:231] offset:8\n\t" \
    "v_mfma_f32_16x16x32_f16 a[200:203], v[76:79], v[156:159], a[200:203]\n\t" \
    "v_cvt_pk_f16_f32 v228, v197, v201\n\t" \
    "v_mfma_f32_16x16x32_f16 a[168:171], v[72:75], v[156:159], a[168:171]\n\t" \
    "global_store_dwordx4 %[vA0], v[248:251], s[60:61] offset:3072 sc0 sc1\n\t" \
    "v_mfma_f32_16x16x32_f16 a[136:139], v[68:71], v[156:159], a[136:139]\n\t" \
    "v_cvt_pk_f16_f32 v229, v205, v209\n\t" \
    "v_mfma_f32_16x16x32_f16 a[104:107], v[64:67], v[156:159], a[104:107]\n\t" \
    "ds_write_b64 %[vWr1], v[228:229] offset:1024\n\t" \
    "v_mfma_f32_16x16x32_f16 a[72:75], v[60:63], v[156:159], a[72:75]\n\t" \
    "v_cvt_pk_f16_f32 v230, v213, v217\n\t" \
    "v_cvt_pk_f16_f32 v231, v221, v225\n\t" \
    "v_mfma_f32_16x16x32_f16 a[40:43], v[56:59], v[156:159], a[40:43]\n\t" \
    "s_add_u32 s55, s37, 24576\n\t" \
    "v_mfma_f32_16x16x32_f16 a[8:11], v[52:55], v[156:159], a[8:11]\n\t" \
    "ds_read_b128 v[156:159], %[vA0] offset:47104\n\t" \
    "s_waitcnt lgkmcnt(6)\n\t" \
    "v_mfma_f32_16x16x32_f16 a[12:15], v[52:55], v[160:163], a[12:15]\n\t" \
    "ds_write_b64 %[vWr1], v[230:231] offset:1032\n\t" \
    "v_mfma_f32_16x16x32_f16 a[44:47], v[56:59], v[160:163], a[44:47]\n\t" \
    "v_cvt_pk_f16_f32 v228, v198, v202\n\t" \
    "v_mfma_f32_16x16x32_f16 a[76:79], v[60:63], v[160:163], a[76:79]\n\t" \
    "buffer_load_dwordx4 v[116:119], %[vA0], %[rWS], s55 offen offset:0 sc0 sc1\n\t" \
    "v_mfma_f32_16x16x32_f16 a[108:111], v[64:67], v[160:163], a[108:111]\n\t" \
    "v_cvt_pk_f16_f32 v229, v206, v210\n\t" \
    "v_mfma_f32_16x16x32_f16 a[140:143], v[68:71], v[160:163], a[140:143]\n\t" \
    "ds_write_b64 %[vWr1], v[228:229] offset:2048\n\t" \
    "v_mfma_f32_16x16x32_f16 a[172:175], v[72:75], v[160:163], a[172:175]\n\t" \
    "v_cvt_pk_f16_f32 v230, v214, v218\n\t" \
    "v_mfma_f32_16x16x32_f16 a[204:207], v[76:79], v[160:163], a[204:207]\n\t" \
    "buffer_load_dwordx4 v[120:123], %[vA0], %[rWS], s55 offen offset:1024 sc0 sc1\n\t" \
    "v_mfma_f32_16x16x32_f16 a[236:239], v[80:83], v[160:163], a[236:239]\n\t" \
    "ds_read_b128 v[160:163], %[vA0] offset:48128\n\t" \
    "s_waitcnt lgkmcnt(8)\n\t" \
    "v_mfma_f32_16x16x32_f16 a[240:243], v[80:83], v[148:151], a[240:243]\n\t" \
    "v_cvt_pk_f16_f32 v231, v222, v226\n\t" \
    "v_mfma_f32_16x16x32_f16 a[208:211], v[76:79], v[148:151], a[208:211]\n\t" \
    "ds_write_b64 %[vWr1], v[230:231] offset:2056\n\t" \
    "v_mfma_f32_16x16x32_f16 a[176:179], v[72:75], v[148:151], a[176:179]\n\t" \
    "v_cvt_pk_f16_f32 v228, v199, v203\n\t" \
    "v_cvt_pk_f16_f32 v229, v207, v211\n\t" \
    "v_mfma_f32_16x16x32_f16 a[144:147], v[68:71], v[148:151], a[144:147]\n\t" \
    "buffer_load_dwordx4 v[124:127], %[vA0], %[rWS], s55 offen offset:2048 sc0 sc1\n\t" \
    "v_mfma_f32_16x16x32_f16 a[112:115], v[64:67], v[148:151], a[112:115]\n\t" \
    "ds_write_b64 %[vWr1], v[228:229] offset:3072\n\t" \
    "v_mfma_f32_16x16x32_f16 a[80:83], v[60:63], v[148:151], a[80:83]\n\t" \
    "v_cvt_pk_f16_f32 v230, v215, v219\n\t" \
    "v_cvt_pk_f16_f32 v231, v223, v227\n\t" \
    "v_mfma_f32_16x16x32_f16 a[48:51], v[56:59], v[148:151], a[48:51]\n\t" \
    "ds_write_b64 %[vWr1], v[230:231] offset:3080\n\t" \
    "v_mfma_f32_16x16x32_f16 a[16:19], v[52:55], v[148:151], a[16:19]\n\t" \
    "ds_read_b128 v[148:151], %[vA0] offset:49152\n\t" \
    "s_waitcnt lgkmcnt(10)\n\t" \
    "v_mfma_f32_16x16x32_f16 a[20:23], v[52:55], v[152:155], a[20:23]\n\t" \
    "buffer_load_dwordx4 v[128:131], %[vA0], %[rWS], s55 offen offset:3072 sc0 sc1\n\t" \
    "v_mfma_f32_16x16x32_f16 a[52:55], v[56:59], v[152:155], a[52:55]\n\t" \
    "s_add_u32 s55, s37, 28672\n\t" \
    "v_mfma_f32_16x16x32_f16 a[84:87], v[60:63], v[152:155], a[84:87]\n\t" \
    "buffer_load_dwordx4 v[132:135], %[vA0], %[rWS], s55 offen offset:0 sc0 sc1\n\t" \
    "v_mfma_f32_16x16x32_f16 a[116:119], v[64:67], v[152:155], a[116:119]\n\t" \
    "buffer_load_dwordx4 v[136:139], %[vA0], %[rWS], s55 offen offset:1024 sc0 sc1\n\t" \
    "v_mfma_f32_16x16x32_f16 a[148:151], v[68:71], v[152:155], a[148:151]\n\t" \
    "v_mfma_f32_16x16x32_f16 a[180:183], v[72:75], v[152:155], a[180:183]\n\t" \
    "v_mfma_f32_16x16x32_f16 a[212:215], v[76:79], v[152:155], a[212:215]\n\t" \
    "v_mfma_f32_16x16x32_f16 a[244:247], v[80:83], v[152:155], a[244:247]\n\t" \
    "ds_read_b128 v[152:155], %[vA0] offset:50176\n\t" \
    "s_waitcnt lgkmcnt(8)\n\t" \
    "v_mfma_f32_16x16x32_f16 a[248:251], v[80:83], v[156:159], a[248:251]\n\t" \
    "buffer_load_dwordx4 v[140:143], %[vA0], %[rWS], s55 offen offset:2048 sc0 sc1\n\t" \
    "v_mfma_f32_16x16x32_f16 a[216:219], v[76:79], v[156:159], a[216:219]\n\t" \
    "v_mfma_f32_16x16x32_f16 a[184:187], v[72:75], v[156:159], a[184:187]\n\t" \
    "v_mfma_f32_16x16x32_f16 a[152:155], v[68:71], v[156:159], a[152:155]\n\t" \
    "v_mfma_f32_16x16x32_f16 a[120:123], v[64:67], v[156:159], a[120:123]\n\t" \
    "v_mfma_f32_16x16x32_f16 a[88:91], v[60:63], v[156:159], a[88:91]\n\t" \
    "buffer_load_dwordx4 v[144:147], %[vA0], %[rWS], s55 offen offset:3072 sc0 sc1\n\t" \
    "v_mfma_f32_16x16x32_f16 a[56:59], v[56:59], v[156:159], a[56:59]\n\t" \
    "v_mfma_f32_16x16x32_f16 a[24:27], v[52:55], v[156:159], a[24:27]\n\t" \
    "ds_read_b128 v[156:159], %[vA0] offset:51200\n\t" \
    "s_waitcnt lgkmcnt(6)\n\t" \
    "v_mfma_f32_16x16x32_f16 a[28:31], v[52:55], v[160:163], a[28:31]\n\t" \
    "s_add_u32 s55, s38, 0\n\t" \
    "v_mfma_f32_16x16x32_f16 a[60:63], v[56:59], v[160:163], a[60:63]\n\t" \
    "v_mfma_f32_16x16x32_f16 a[92:95], v[60:63], v[160:163], a[92:95]\n\t" \
    "buffer_load_dwordx4 v[20:23], %[vA0], %[rWS], s55 offen offset:0 sc0 sc1\n\t" \
    "v_mfma_f32_16x16x32_f16 a[124:127], v[64:67], v[160:163], a[124:127]\n\t" \
    "v_mfma_f32_16x16x32_f16 a[156:159], v[68:71], v[160:163], a[156:159]\n\t" \
    "v_mfma_f32_16x16x32_f16 a[188:191], v[72:75], v[160:163], a[188:191]\n\t" \
    "v_mfma_f32_16x16x32_f16 a[220:223], v[76:79], v[160:163], a[220:223]\n\t" \
    "v_mfma_f32_16x16x32_f16 a[252:255], v[80:83], v[160:163], a[252:255]\n\t" \
    "ds_read_b128 v[160:163], %[vA0] offset:52224\n\t" \
    "s_waitcnt vmcnt(22) lgkmcnt(3)\n\t" \
    "v_mfma_f32_16x16x32_f16 a[224:227], v[112:115], v[148:151], a[224:227]\n\t" \
    "buffer_load_dwordx4 v[24:27], %[vA0], %[rWS], s55 offen offset:1024 sc0 sc1\n\t" \
    "v_mfma_f32_16x16x32_f16 a[192:195], v[108:111], v[148:151], a[192:195]\n\t" \
    "v_mfma_f32_16x16x32_f16 a[160:163], v[104:107], v[148:151], a[160:163]\n\t" \
    "v_mfma_f32_16x16x32_f16 a[128:131], v[100:103], v[148:151], a[128:131]\n\t" \
    "v_mfma_f32_16x16x32_f16 a[96:99], v[96:99], v[148:151], a[96:99]\n\t" \
    "buffer_load_dwordx4 v[28:31], %[vA0], %[rWS], s55 offen offset:2048 sc0 sc1\n\t" \
    "v_mfma_f32_16x16x32_f16 a[64:67], v[92:95], v[148:151], a[64:67]\n\t" \
    "v_mfma_f32_16x16x32_f16 a[32:35], v[88:91], v[148:151], a[32:35]\n\t" \
    "v_mfma_f32_16x16x32_f16 a[0:3], v[84:87], v[148:151], a[0:3]\n\t" \
    "ds_read_b128 v[148:151], %[vA0] offset:53248\n\t" \
    "s_waitcnt lgkmcnt(3)\n\t" \
    "v_mfma_f32_16x16x32_f16 a[4:7], v[84:87], v[152:155], a[4:7]\n\t" \
    "v_mfma_f32_16x16x32_f16 a[36:39], v[88:91], v[152:155], a[36:39]\n\t" \
    "buffer_load_dwordx4 v[32:35], %[vA0], %[rWS], s55 offen offset:3072 sc0 sc1\n\t" \
    "v_mfma_f32_16x16x32_f16 a[68:71], v[92:95], v[152:155], a[68:71]\n\t" \
    "v_mfma_f32_16x16x32_f16 a[100:103], v[96:99], v[152:155], a[100:103]\n\t" \
    "v_mfma_f32_16x16x32_f16 a[132:135], v[100:103], v[152:155], a[132:135]\n\t" \
    "v_mfma_f32_16x16x32_f16 a[164:167], v[104:107], v[152:155], a[164:167]\n\t" \
    "s_add_u32 s55, s38, 4096\n\t" \
    "v_mfma_f32_16x16x32_f16 a[196:199], v[108:111], v[152:155], a[196:199]\n\t" \
    "v_mfma_f32_16x16x32_f16 a[228:231], v[112:115], v[152:155], a[228:231]\n\t" \
    "ds_read_b128 v[152:155], %[vA0] offset:54272\n\t" \
    "s_waitcnt lgkmcnt(3)\n\t" \
    "v_mfma_f32_16x16x32_f16 a[232:235], v[112:115], v[156:159], a[232:235]\n\t" \
    "buffer_load_dwordx4 v[36:39], %[vA0], %[rWS], s55 offen offset:0 sc0 sc1\n\t" \
    "v_mfma_f32_16x16x32_f16 a[200:203], v[108:111], v[156:159], a[200:203]\n\t" \
    "v_mfma_f32_16x16x32_f16 a[168:171], v[104:107], v[156:159], a[168:171]\n\t" \
    "v_mfma_f32_16x16x32_f16 a[136:139], v[100:103], v[156:159], a[136:139]\n\t" \
    "v_mfma_f32_16x16x32_f16 a[104:107], v[96:99], v[156:159], a[104:107]\n\t" \
    "buffer_load_dwordx4 v[40:43], %[vA0], %[rWS], s55 offen offset:1024 sc0 sc1\n\t" \
    "v_mfma_f32_16x16x32_f16 a[72:75], v[92:95], v[156:159], a[72:75]\n\t" \
    "v_readlane_b32 s52, %[vTab0], s51\n\t" \
    "v_mfma_f32_16x16x32_f16 a[40:43], v[88:91], v[156:159], a[40:43]\n\t" \
    "s_add_u32 s42, s51, 32\n\t" \
    "v_mfma_f32_16x16x32_f16 a[8:11], v[84:87], v[156:159], a[8:11]\n\t" \
    "ds_read_b128 v[156:159], %[vA0] offset:55296\n\t" \
    "s_waitcnt lgkmcnt(3)\n\t" \
    "v_mfma_f32_16x16x32_f16 a[12:15], v[84:87], v[160:163], a[12:15]\n\t" \
    "v_readlane_b32 s53, %[vTab1], s51\n\t" \
    "v_mfma_f32_16x16x32_f16 a[44:47], v[88:91], v[160:163], a[44:47]\n\t" \
    "v_readlane_b32 s42, %[vTab0], s42\n\t" \
    "v_mfma_f32_16x16x32_f16 a[76:79], v[92:95], v[160:163], a[76:79]\n\t" \
    "s_add_u32 s51, s51, 1\n\t" \
    "v_mfma_f32_16x16x32_f16 a[108:111], v[96:99], v[160:163], a[108:111]\n\t" \
    "buffer_load_dwordx4 v[44:47], %[vA0], %[rWS], s55 offen offset:2048 sc0 sc1\n\t" \
    "v_mfma_f32_16x16x32_f16 a[140:143], v[100:103], v[160:163], a[140:143]\n\t" \
    "s_add_u32 s40, s66, s43\n\t" \
    "s_add_u32 s40, s40, %[sWk1]\n\t" \
    "v_mfma_f32_16x16x32_f16 a[172:175], v[104:107], v[160:163], a[172:175]\n\t" \
    "s_mov_b32 s56, s42\n\t" \
    "s_and_b32 s57, s53, 0xffff\n\t" \
    "v_mfma_f32_16x16x32_f16 a[204:207], v[108:111], v[160:163], a[204:207]\n\t" \
    "s_and_b32 s58, s53, 0x10000\n\t" \
    "s_bfe_i32 s53, s53, 0x10011\n\t" \
    "v_mfma_f32_16x16x32_f16 a[236:239], v[112:115], v[160:163], a[236:239]\n\t" \
    "ds_read_b128 v[160:163], %[vA0] offset:56320\n\t" \
    "s_waitcnt lgkmcnt(3)\n\t" \
    "v_mfma_f32_16x16x32_f16 a[240:243], v[112:115], v[148:151], a[240:243]\n\t" \
    "buffer_load_dwordx4 v[48:51], %[vA0], %[rWS], s55 offen offset:3072 sc0 sc1\n\t" \
    "v_mfma_f32_16x16x32_f16 a[208:211], v[108:111], v[148:151], a[208:211]\n\t" \
    "v_mfma_f32_16x16x32_f16 a[176:179], v[104:107], v[148:151], a[176:179]\n\t" \
    "v_mfma_f32_16x16x32_f16 a[144:147], v[100:103], v[148:151], a[144:147]\n\t" \
    "buffer_load_dwordx4 v[196:199], %[vW], %[rW], s40 offen nt\n\t" \
    "v_mfma_f32_16x16x32_f16 a[112:115], v[96:99], v[148:151], a[112:115]\n\t" \
    "v_mfma_f32_16x16x32_f16 a[80:83], v[92:95], v[148:151], a[80:83]\n\t" \
    "v_mfma_f32_16x16x32_f16 a[48:51], v[88:91], v[148:151], a[48:51]\n\t" \
    "s_add_u32 s40, s40, 32768\n\t" \
    "v_mfma_f32_16x16x32_f16 a[16:19], v[84:87], v[148:151], a[16:19]\n\t" \
    "ds_read_b128 v[148:151], %[vA0] offset:57344\n\t" \
    "s_waitcnt lgkmcnt(3)\n\t" \
    "v_mfma_f32_16x16x32_f16 a[20:23], v[84:87], v[152:155], a[20:23]\n\t" \
    "s_cmp_eq_u32 s69, 0\n\t" \
    "s_cselect_b32 s42, 1073741824, s52\n\t" \
    "v_mfma_f32_16x16x32_f16 a[52:55], v[88:91], v[152:155], a[52:55]\n\t" \
    "buffer_load_dwordx4 v[200:203], %[vW], %[rW], s40 offen nt\n\t" \
    "v_mfma_f32_16x16x32_f16 a[84:87], v[92:95], v[152:155], a[84:87]\n\t" \
    "buffer_load_dwordx4 v[236:239], %[vA0], %[rWS], s42 offen sc0 sc1\n\t" \
    "v_mfma_f32_16x16x32_f16 a[116:119], v[96:99], v[152:155], a[116:119]\n\t" \
    "s_add_u32 s52, s52, 262144\n\t" \
    "s_cmp_eq_u32 s69, 1\n\t" \
    "v_mfma_f32_16x16x32_f16 a[148:151], v[100:103], v[152:155], a[148:151]\n\t" \
    "s_cselect_b32 s42, 1073741824, s52\n\t" \
    "v_mfma_f32_16x16x32_f16 a[180:183], v[104:107], v[152:155], a[180:183]\n\t" \
    "buffer_load_dwordx4 v[240:243], %[vA0], %[rWS], s42 offen sc0 sc1\n\t" \
    "v_mfma_f32_16x16x32_f16 a[212:215], v[108:111], v[152:155], a[212:215]\n\t" \
    "s_add_u32 s40, s40, 32768\n\t" \
    "s_add_u32 s52, s52, 262144\n\t" \
    "v_mfma_f32_16x16x32_f16 a[244:247], v[112:115], v[152:155], a[244:247]\n\t" \
    "ds_read_b128 v[152:155], %[vA0] offset:58368\n\t" \
    "s_waitcnt lgkmcnt(3)\n\t" \
    "v_mfma_f32_16x16x32_f16 a[248:251], v[112:115], v[156:159], a[248:251]\n\t" \
    "s_cmp_eq_u32 s69, 2\n\t" \
    "s_cselect_b32 s42, 1073741824, s52\n\t" \
    "v_mfma_f32_16x16x32_f16 a[216:219], v[108:111], v[156:159], a[216:219]\n\t" \
    "buffer_load_dwordx4 v[204:207], %[vW], %[rW], s40 offen nt\n\t" \
    "v_mfma_f32_16x16x32_f16 a[184:187], v[104:107], v[156:159], a[184:187]\n\t" \
    "buffer_load_dwordx4 v[244:247], %[vA0], %[rWS], s42 offen sc0 sc1\n\t" \
    "v_mfma_f32_16x16x32_f16 a[152:155], v[100:103], v[156:159], a[152:155]\n\t" \
    "s_add_u32 s52, s52, 262144\n\t" \
    "s_cmp_eq_u32 s69, 3\n\t" \
    "v_mfma_f32_16x16x32_f16 a[120:123], v[96:99], v[156:159], a[120:123]\n\t" \
    "s_cselect_b32 s42, 1073741824, s52\n\t" \
    "s_add_u32 s40, s40, 32768\n\t" \
    "v_mfma_f32_16x16x32_f16 a[88:91], v[92:95], v[156:159], a[88:91]\n\t" \
    "buffer_load_dwordx4 v[248:251], %[vA0], %[rWS], s42 offen sc0 sc1\n\t" \
    "v_mfma_f32_16x16x32_f16 a[56:59], v[88:91], v[156:159], a[56:59]\n\t" \
    "buffer_load_dwordx4 v[208:211], %[vW], %[rW], s40 offen nt\n\t" \
    "v_mfma_f32_16x16x32_f16 a[24:27], v[84:87], v[156:159], a[24:27]\n\t" \
    "ds_read_b128 v[156:159], %[vA0] offset:59392\n\t" \
    "s_waitcnt lgkmcnt(3)\n\t" \
    "v_mfma_f32_16x16x32_f16 a[28:31], v[84:87], v[160:163], a[28:31]\n\t" \
    "s_add_u32 s40, s40, 32768\n\t" \
    "v_mfma_f32_16x16x32_f16 a[60:63], v[88:91], v[160:163], a[60:63]\n\t" \
    "buffer_load_dwordx4 v[212:215], %[vW], %[rW], s40 offen nt\n\t" \
    "v_mfma_f32_16x16x32_f16 a[92:95], v[92:95], v[160:163], a[92:95]\n\t" \
    "v_mfma_f32_16x16x32_f16 a[124:127], v[96:99], v[160:163], a[124:127]\n\t" \
    "s_add_u32 s40, s40, 32768\n\t" \
    "v_mfma_f32_16x16x32_f16 a[156:159], v[100:103], v[160:163], a[156:159]\n\t" \
    "v_mfma_f32_16x16x32_f16 a[188:191], v[104:107], v[160:163], a[188:191]\n\t" \
    "buffer_load_dwordx4 v[216:219], %[vW], %[rW], s40 offen nt\n\t" \
    "v_mfma_f32_16x16x32_f16 a[220:223], v[108:111], v[160:163], a[220:223]\n\t" \
    "v_mfma_f32_16x16x32_f16 a[252:255], v[112:115], v[160:163], a[252:255]\n\t" \
    "ds_read_b128 v[160:163], %[vA0] offset:60416\n\t" \
    "s_waitcnt vmcnt(18) lgkmcnt(3)\n\t" \
    "v_mfma_f32_16x16x32_f16 a[224:227], v[144:147], v[148:151], a[224:227]\n\t" \
    "s_add_u32 s40, s40, 32768\n\t" \
    "v_mfma_f32_16x16x32_f16 a[192:195], v[140:143], v[148:151], a[192:195]\n\t" \
    "v_mfma_f32_16x16x32_f16 a[160:163], v[136:139], v[148:151], a[160:163]\n\t" \
    "buffer_load_dwordx4 v[220:223], %[vW], %[rW], s40 offen nt\n\t" \
    "v_mfma_f32_16x16x32_f16 a[128:131], v[132:135], v[148:151], a[128:131]\n\t" \
    "v_mfma_f32_16x16x32_f16 a[96:99], v[128:131], v[148:151], a[96:99]\n\t" \
    "v_mfma_f32_16x16x32_f16 a[64:67], v[124:127], v[148:151], a[64:67]\n\t" \
    "v_mfma_f32_16x16x32_f16 a[32:35], v[120:123], v[148:151], a[32:35]\n\t" \
    "s_add_u32 s40, s40, 32768\n\t" \
    "v_mfma_f32_16x16x32_f16 a[0:3], v[116:119], v[148:151], a[0:3]\n\t" \
    "ds_read_b128 v[148:151], %[vA0] offset:61440\n\t" \
    "s_waitcnt lgkmcnt(3)\n\t" \
    "v_mfma_f32_16x16x32_f16 a[4:7], v[116:119], v[152:155], a[4:7]\n\t" \
    "buffer_load_dwordx4 v[224:227], %[vW], %[rW], s40 offen nt\n\t" \
    "v_mfma_f32_16x16x32_f16 a[36:39], v[120:123], v[152:155], a[36:39]\n\t" \
    "v_mfma_f32_16x16x32_f16 a[68:71], v[124:127], v[152:155], a[68:71]\n\t" \
    "v_mfma_f32_16x16x32_f16 a[100:103], v[128:131], v[152:155], a[100:103]\n\t" \
    "s_add_u32 s55, s38, 8192\n\t" \
    "v_mfma_f32_16x16x32_f16 a[132:135], v[132:135], v[152:155], a[132:135]\n\t" \
    "v_and_b32 v252, s53, v252\n\t" \
    "v_mfma_f32_16x16x32_f16 a[164:167], v[136:139], v[152:155], a[164:167]\n\t" \
    "buffer_load_dwordx4 v[52:55], %[vA0], %[rWS], s55 offen offset:0 sc0 sc1\n\t" \
    "v_mfma_f32_16x16x32_f16 a[196:199], v[140:143], v[152:155], a[196:199]\n\t" \
    "v_and_b32 v253, s53, v253\n\t" \
    "v_and_b32 v254, s53, v254\n\t" \
    "v_mfma_f32_16x16x32_f16 a[228:231], v[144:147], v[152:155], a[228:231]\n\t" \
    "ds_read_b128 v[152:155], %[vA0] offset:62464\n\t" \
    "s_waitcnt lgkmcnt(3)\n\t" \
    "v_mfma_f32_16x16x32_f16 a[232:235], v[144:147], v[156:159], a[232:235]\n\t" \
    "v_and_b32 v255, s53, v255\n\t" \
    "s_waitcnt vmcnt(9)\n\t" \
    "v_add_f32 v236, v236, v240\n\t" \
    "v_mfma_f32_16x16x32_f16 a[200:203], v[140:143], v[156:159], a[200:203]\n\t" \
    "v_add_f32 v237, v237, v241\n\t" \
    "v_mfma_f32_16x16x32_f16 a[168:171], v[136:139], v[156:159], a[168:171]\n\t" \
    "buffer_load_dwordx4 v[56:59], %[vA0], %[rWS], s55 offen offset:1024 sc0 sc1\n\t" \
    "v_mfma_f32_16x16x32_f16 a[136:139], v[132:135], v[156:159], a[136:139]\n\t" \
    "v_add_f32 v238, v238, v242\n\t" \
    "v_add_f32 v239, v239, v243\n\t" \
    "v_mfma_f32_16x16x32_f16 a[104:107], v[128:131], v[156:159], a[104:107]\n\t" \
    "v_add_f32 v252, v252, v236\n\t" \
    "v_mfma_f32_16x16x32_f16 a[72:75], v[124:127], v[156:159], a[72:75]\n\t" \
    "v_add_f32 v253, v253, v237\n\t" \
    "v_mfma_f32_16x16x32_f16 a[40:43], v[120:123], v[156:159], a[40:43]\n\t" \
    "v_add_f32 v254, v254, v238\n\t" \
    "v_mfma_f32_16x16x32_f16 a[8:11], v[116:119], v[156:159], a[8:11]\n\t" \
    "ds_read_b128 v[156:159], %[vA0] offset:63488\n\t" \
    "s_waitcnt lgkmcnt(3)\n\t" \
    "v_mfma_f32_16x16x32_f16 a[12:15], v[116:119], v[160:163], a[12:15]\n\t" \
    "buffer_load_dwordx4 v[60:63], %[vA0], %[rWS], s55 offen offset:2048 sc0 sc1\n\t" \
    "v_mfma_f32_16x16x32_f16 a[44:47], v[120:123], v[160:163], a[44:47]\n\t" \
    "v_add_f32 v255, v255, v239\n\t" \
    "s_waitcnt vmcnt(8)\n\t" \
    "v_add_f32 v244, v244, v248\n\t" \
    "v_mfma_f32_16x16x32_f16 a[76:79], v[124:127], v[160:163], a[76:79]\n\t" \
    "v_add_f32 v245, v245, v249\n\t" \
    "v_add_f32 v246, v246, v250\n\t" \
    "v_mfma_f32_16x16x32_f16 a[108:111], v[128:131], v[160:163], a[108:111]\n\t" \
    "v_add_f32 v247, v247, v251\n\t" \
    "v_mfma_f32_16x16x32_f16 a[140:143], v[132:135], v[160:163], a[140:143]\n\t" \
    "buffer_load_dwordx4 v[64:67], %[vA0], %[rWS], s55 offen offset:3072 sc0 sc1\n\t" \
    "v_mfma_f32_16x16x32_f16 a[172:175], v[136:139], v[160:163], a[172:175]\n\t" \
    "v_add_f32 v252, v252, v244\n\t" \
    "v_add_f32 v253, v253, v245\n\t" \
    "v_mfma_f32_16x16x32_f16 a[204:207], v[140:143], v[160:163], a[204:207]\n\t" \
    "v_add_f32 v254, v254, v246\n\t" \
    "v_mfma_f32_16x16x32_f16 a[236:239], v[144:147], v[160:163], a[236:239]\n\t" \
    "ds_read_b128 v[160:163], %[vA0] offset:64512\n\t" \
    "s_waitcnt lgkmcnt(3)\n\t" \
    "v_mfma_f32_16x16x32_f16 a[240:243], v[144:147], v[148:151], a[240:243]\n\t" \
    "s_add_u32 s55, s38, 12288\n\t" \
    "v_add_f32 v255, v255, v247\n\t" \
    "v_mfma_f32_16x16x32_f16 a[208:211], v[140:143], v[148:151], a[208:211]\n\t" \
    "v_med3_f32 v236, v252, %[m60], %[p60]\n\t" \
    "v_mfma_f32_16x16x32_f16 a[176:179], v[136:139], v[148:151], a[176:179]\n\t" \
    "buffer_load_dwordx4 v[68:71], %[vA0], %[rWS], s55 offen offset:0 sc0 sc1\n\t" \
    "v_mfma_f32_16x16x32_f16 a[144:147], v[132:135], v[148:151], a[144:147]\n\t" \
    "v_med3_f32 v237, v253, %[m60], %[p60]\n\t" \
    "v_med3_f32 v238, v254, %[m60], %[p60]\n\t" \
    "v_mfma_f32_16x16x32_f16 a[112:115], v[128:131], v[148:151], a[112:115]\n\t" \
    "v_med3_f32 v239, v255, %[m60], %[p60]\n\t" \
    "v_mul_f32 v236, 0xbfb8aa3b, v236\n\t" \
    "v_mfma_f32_16x16x32_f16 a[80:83], v[124:127], v[148:151], a[80:83]\n\t" \
    "v_mul_f32 v237, 0xbfb8aa3b, v237\n\t" \
    "v_mfma_f32_16x16x32_f16 a[48:51], v[120:123], v[148:151], a[48:51]\n\t" \
    "v_mul_f32 v238, 0xbfb8aa3b, v238\n\t" \
    "v_mfma_f32_16x16x32_f16 a[16:19], v[116:119], v[148:151], a[16:19]\n\t" \
    "s_waitcnt lgkmcnt(0)\n\t" \
    "s_barrier\n\t" \
    "ds_read_b128 v[148:151], %[vA0] offset:0\n\t" \
    "v_mfma_f32_16x16x32_f16 a[20:23], v[116:119], v[152:155], a[20:23]\n\t" \
    "buffer_load_dwordx4 v[72:75], %[vA0], %[rWS], s55 offen offset:1024 sc0 sc1\n\t" \
    "v_mfma_f32_16x16x32_f16 a[52:55], v[120:123], v[152:155], a[52:55]\n\t" \
    "v_mul_f32 v239, 0xbfb8aa3b, v239\n\t" \
    "v_mfma_f32_16x16x32_f16 a[84:87], v[124:127], v[152:155], a[84:87]\n\t" \
    "v_exp_f32 v236, v236\n\t" \
    "v_mfma_f32_16x16x32_f16 a[116:119], v[128:131], v[152:155], a[116:119]\n\t" \
    "v_exp_f32 v237, v237\n\t" \
    "v_mfma_f32_16x16x32_f16 a[148:151], v[132:135], v[152:155], a[148:151]\n\t" \
    "v_exp_f32 v238, v238\n\t" \
    "v_mfma_f32_16x16x32_f16 a[180:183], v[136:139], v[152:155], a[180:183]\n\t" \
    "v_exp_f32 v239, v239\n\t" \
    "v_mfma_f32_16x16x32_f16 a[212:215], v[140:143], v[152:155], a[212:215]\n\t" \
    "buffer_load_dwordx4 v[76:79], %[vA0], %[rWS], s55 offen offset:2048 sc0 sc1\n\t" \
    "v_mfma_f32_16x16x32_f16 a[244:247], v[144:147], v[152:155], a[244:247]\n\t" \
    "ds_read_b128 v[152:155], %[vA0] offset:1024\n\t" \
    "v_mfma_f32_16x16x32_f16 a[248:251], v[144:147], v[156:159], a[248:251]\n\t" \
    "v_add_f32 v236, 1.0, v236\n\t" \
    "v_add_f32 v237, 1.0, v237\n\t" \
    "v_mfma_f32_16x16x32_f16 a[216:219], v[140:143], v[156:159], a[216:219]\n\t" \
    "v_add_f32 v238, 1.0, v238\n\t" \
    "v_add_f32 v239, 1.0, v239\n\t" \
    "v_mfma_f32_16x16x32_f16 a[184:187], v[136:139], v[156:159], a[184:187]\n\t" \
    "v_rcp_f32 v236, v236\n\t" \
    "v_mfma_f32_16x16x32_f16 a[152:155], v[132:135], v[156:159], a[152:155]\n\t" \
    "buffer_load_dwordx4 v[80:83], %[vA0], %[rWS], s55 offen offset:3072 sc0 sc1\n\t" \
    "v_mfma_f32_16x16x32_f16 a[120:123], v[128:131], v[156:159], a[120:123]\n\t" \
    "v_rcp_f32 v237, v237\n\t" \
    "v_mfma_f32_16x16x32_f16 a[88:91], v[124:127], v[156:159], a[88:91]\n\t" \
    "s_add_u32 s55, s38, 16384\n\t" \
    "v_mfma_f32_16x16x32_f16 a[56:59], v[120:123], v[156:159], a[56:59]\n\t" \
    "v_rcp_f32 v238, v238\n\t" \
    "v_mfma_f32_16x16x32_f16 a[24:27], v[116:119], v[156:159], a[24:27]\n\t" \
    "ds_read_b128 v[156:159], %[vA0] offset:2048\n\t" \
    "v_mfma_f32_16x16x32_f16 a[28:31], v[116:119], v[160:163], a[28:31]\n\t" \
    "v_rcp_f32 v239, v239\n\t" \
    "v_mfma_f32_16x16x32_f16 a[60:63], v[120:123], v[160:163], a[60:63]\n\t" \
    "buffer_load_dwordx4 v[84:87], %[vA0], %[rWS], s55 offen offset:0 sc0 sc1\n\t" \
    "v_mfma_f32_16x16x32_f16 a[92:95], v[124:127], v[160:163], a[92:95]\n\t" \
    "v_mul_f32 v232, v252, v236\n\t" \
    "v_mul_f32 v233, v253, v237\n\t" \
    "v_mfma_f32_16x16x32_f16 a[124:127], v[128:131], v[160:163], a[124:127]\n\t" \
    "v_mul_f32 v234, v254, v238\n\t" \
    "v_mfma_f32_16x16x32_f16 a[156:159], v[132:135], v[160:163], a[156:159]\n\t" \
    "buffer_load_dwordx4 v[88:91], %[vA0], %[rWS], s55 offen offset:1024 sc0 sc1\n\t" \
    "v_mfma_f32_16x16x32_f16 a[188:191], v[136:139], v[160:163], a[188:191]\n\t" \
    "v_mul_f32 v235, v255, v239\n\t" \
    "v_mfma_f32_16x16x32_f16 a[220:223], v[140:143], v[160:163], a[220:223]\n\t" \
    "buffer_load_dwordx4 v[92:95], %[vA0], %[rWS], s55 offen offset:2048 sc0 sc1\n\t" \
    "v_mfma_f32_16x16x32_f16 a[252:255], v[144:147], v[160:163], a[252:255]\n\t" \
    "buffer_load_dwordx4 v[96:99], %[vA0], %[rWS], s55 offen offset:3072 sc0 sc1\n\t" \
    "s_add_u32 s36, s36, 1\n\t" \
    "ds_read_b128 v[160:163], %[vA0] offset:3072\n\t" \
    "s_waitcnt lgkmcnt(3)\n\t" \
    "v_mfma_f32_16x16x32_f16 a[224:227], v[48:51], v[148:151], a[224:227]\n\t" \
    "s_mov_b32 s37, s38\n\t" \
    "v_cvt_pk_f16_f32 v228, v164, v168\n\t" \
    "v_mfma_f32_16x16x32_f16 a[192:195], v[44:47], v[148:151], a[192:195]\n\t" \
    "v_mfma_f32_16x16x32_f16 a[160:163], v[40:43], v[148:151], a[160:163]\n\t" \
    "v_cvt_pk_f16_f32 v229, v172, v176\n\t" \
    "v_mfma_f32_16x16x32_f16 a[128:131], v[36:39], v[148:151], a[128:131]\n\t" \
    "ds_write_b64 %[vWr0], v[228:229] offset:32768\n\t" \
    "v_mfma_f32_16x16x32_f16 a[96:99], v[32:35], v[148:151], a[96:99]\n\t" \
    "v_cvt_pk_f16_f32 v230, v180, v184\n\t" \
    "v_mfma_f32_16x16x32_f16 a[64:67], v[28:31], v[148:151], a[64:67]\n\t" \
    "v_mfma_f32_16x16x32_f16 a[32:35], v[24:27], v[148:151], a[32:35]\n\t" \
    "v_cvt_pk_f16_f32 v231, v188, v192\n\t" \
    "v_mfma_f32_16x16x32_f16 a[0:3], v[20:23], v[148:151], a[0:3]\n\t" \
    "ds_read_b128 v[148:151], %[vA0] offset:4096\n\t" \
    "s_waitcnt lgkmcnt(4)\n\t" \
    "v_mfma_f32_16x16x32_f16 a[4:7], v[20:23], v[152:155], a[4:7]\n\t" \
    "s_cmp_ge_u32 s48, 2\n\ts_cselect_b64 exec, -1, 0\n\ts_cselect_b32 s69, %[own], 9\n\tds_read_b128 v[252:255], %[vStg] offset:18432\n\ts_mov_b64 exec, -1\n\t" \
    "s_waitcnt lgkmcnt(4)\n\t" \
    "v_mfma_f32_16x16x32_f16 a[36:39], v[24:27], v[152:155], a[36:39]\n\t" \
    "ds_write_b64 %[vWr0], v[230:231] offset:32776\n\t" \
    "v_mfma_f32_16x16x32_f16 a[68:71], v[28:31], v[152:155], a[68:71]\n\t" \
    "v_cvt_pk_f16_f32 v228, v165, v169\n\t" \
    "s_add_u32 s55, s37, 20480\n\t" \
    "v_mfma_f32_16x16x32_f16 a[100:103], v[32:35], v[152:155], a[100:103]\n\t" \
    "v_cvt_pk_f16_f32 v229, v173, v177\n\t" \
    "v_mfma_f32_16x16x32_f16 a[132:135], v[36:39], v[152:155], a[132:135]\n\t" \
    "buffer_load_dwordx4 v[100:103], %[vA0], %[rWS], s55 offen offset:0 sc0 sc1\n\t" \
    "v_mfma_f32_16x16x32_f16 a[164:167], v[40:43], v[152:155], a[164:167]\n\t" \
    "ds_write_b64 %[vWr0], v[228:229] offset:33792\n\t" \
    "v_mfma_f32_16x16x32_f16 a[196:199], v[44:47], v[152:155], a[196:199]\n\t" \
    "v_cvt_pk_f16_f32 v230, v181, v185\n\t" \
    "v_cvt_pk_f16_f32 v231, v189, v193\n\t" \
    "v_mfma_f32_16x16x32_f16 a[228:231], v[48:51], v[152:155], a[228:231]\n\t" \
    "ds_read_b128 v[152:155], %[vA0] offset:5120\n\t" \
    "s_waitcnt lgkmcnt(6)\n\t" \
    "v_mfma_f32_16x16x32_f16 a[232:235], v[48:51], v[156:159], a[232:235]\n\t" \
    "ds_write_b64 %[vWr0], v[230:231] offset:33800\n\t" \
    "v_mfma_f32_16x16x32_f16 a[200:203], v[44:47], v[156:159], a[200:203]\n\t" \
    "buffer_load_dwordx4 v[104:107], %[vA0], %[rWS], s55 offen offset:1024 sc0 sc1\n\t" \
    "v_mfma_f32_16x16x32_f16 a[168:171], v[40:43], v[156:159], a[168:171]\n\t" \
    "v_cvt_pk_f16_f32 v228, v166, v170\n\t" \
    "v_cvt_pk_f16_f32 v229, v174, v178\n\t" \
    "v_mfma_f32_16x16x32_f16 a[136:139], v[36:39], v[156:159], a[136:139]\n\t" \
    "ds_write_b64 %[vWr0], v[228:229] offset:34816\n\t" \
    "v_mfma_f32_16x16x32_f16 a[104:107], v[32:35], v[156:159], a[104:107]\n\t" \
    "v_cvt_pk_f16_f32 v230, v182, v186\n\t" \
    "v_mfma_f32_16x16x32_f16 a[72:75], v[28:31], v[156:159], a[72:75]\n\t" \
    "buffer_load_dwordx4 v[108:111], %[vA0], %[rWS], s55 offen offset:2048 sc0 sc1\n\t" \
    "v_mfma_f32_16x16x32_f16 a[40:43], v[24:27], v[156:159], a[40:43]\n\t" \
    "s_add_u32 s66, s66, 0x400000\n\t" \
    "s_and_b32 s66, s66, 0xfffffff\n\t" \
    "v_mfma_f32_16x16x32_f16 a[8:11], v[20:23], v[156:159], a[8:11]\n\t" \
    "ds_read_b128 v[156:159], %[vA0] offset:6144\n\t" \
    "s_waitcnt lgkmcnt(8)\n\t" \
    "v_mfma_f32_16x16x32_f16 a[12:15], v[20:23], v[160:163], a[12:15]\n\t" \
    "v_cvt_pk_f16_f32 v231, v190, v194\n\t" \
    "v_mfma_f32_16x16x32_f16 a[44:47], v[24:27], v[160:163], a[44:47]\n\t" \
    "ds_write_b64 %[vWr0], v[230:231] offset:34824\n\t" \
    "v_mfma_f32_16x16x32_f16 a[76:79], v[28:31], v[160:163], a[76:79]\n\t" \
    "v_cvt_pk_f16_f32 v228, v167, v171\n\t" \
    "s_add_u32 s39, s66, s43\n\t" \
    "v_mfma_f32_16x16x32_f16 a[108:111], v[32:35], v[160:163], a[108:111]\n\t" \
    "s_add_u32 s39, s39, %[sWk0]\n\t" \
    "v_mfma_f32_16x16x32_f16 a[140:143], v[36:39], v[160:163], a[140:143]\n\t" \
    "buffer_load_dwordx4 v[112:115], %[vA0], %[rWS], s55 offen offset:3072 sc0 sc1\n\t" \
    "v_mfma_f32_16x16x32_f16 a[172:175], v[40:43], v[160:163], a[172:175]\n\t" \
    "v_cvt_pk_f16_f32 v229, v175, v179\n\t" \
    "v_mfma_f32_16x16x32_f16 a[204:207], v[44:47], v[160:163], a[204:207]\n\t" \
    "ds_write_b64 %[vWr0], v[228:229] offset:35840\n\t" \
    "v_mfma_f32_16x16x32_f16 a[236:239], v[48:51], v[160:163], a[236:239]\n\t" \
    "ds_read_b128 v[160:163], %[vA0] offset:7168\n\t" \
    "s_waitcnt lgkmcnt(9)\n\t" \
    "v_mfma_f32_16x16x32_f16 a[240:243], v[48:51], v[148:151], a[240:243]\n\t" \
    "v_cvt_pk_f16_f32 v230, v183, v187\n\t" \
    "v_cvt_pk_f16_f32 v231, v191, v195\n\t" \
    "v_mfma_f32_16x16x32_f16 a[208:211], v[44:47], v[148:151], a[208:211]\n\t" \
    "ds_write_b64 %[vWr0], v[230:231] offset:35848\n\t" \
    "v_mfma_f32_16x16x32_f16 a[176:179], v[40:43], v[148:151], a[176:179]\n\t" \
    "buffer_load_dwordx4 v[164:167], %[vW], %[rW], s39 offen\n\t" \
    "v_mfma_f32_16x16x32_f16 a[144:147], v[36:39], v[148:151], a[144:147]\n\t" \
    "s_add_u32 s38, s38, 32768\n\t" \
    "s_and_b32 s38, s38, 0x1fffff\n\t" \
    "v_mfma_f32_16x16x32_f16 a[112:115], v[32:35], v[148:151], a[112:115]\n\t" \
    "s_or_b32 s38, s38, %[sA0]\n\t" \
    "s_add_u32 s39, s39, 32768\n\t" \
    "v_mfma_f32_16x16x32_f16 a[80:83], v[28:31], v[148:151], a[80:83]\n\t" \
    "s_add_u32 s41, %[own], 2\n\t" \
    "s_and_b32 s41, s41, 3\n\t" \
    "v_mfma_f32_16x16x32_f16 a[48:51], v[24:27], v[148:151], a[48:51]\n\t" \
    "s_lshl_b32 s41, s41, 12\n\t" \
    "v_add_u32 v18, s41, %[vStg]\n\t" \
    "v_mfma_f32_16x16x32_f16 a[16:19], v[20:23], v[148:151], a[16:19]\n\t" \
    "ds_read_b128 v[148:151], %[vA0] offset:8192\n\t" \
    "s_waitcnt lgkmcnt(8)\n\t" \
    "v_mfma_f32_16x16x32_f16 a[20:23], v[20:23], v[152:155], a[20:23]\n\t" \
    "s_lshl_b32 s41, s41, 1\n\t" \
    "s_add_u32 s41, s41, s50\n\t" \
    "v_mfma_f32_16x16x32_f16 a[52:55], v[24:27], v[152:155], a[52:55]\n\t" \
    "s_add_u32 s41, s41, 4096\n\t" \
    "s_add_u32 s60, %[wsLo], s41\n\t" \
    "v_mfma_f32_16x16x32_f16 a[84:87], v[28:31], v[152:155], a[84:87]\n\t" \
    "s_addc_u32 s61, %[wsHi], 0\n\t" \
    "v_mfma_f32_16x16x32_f16 a[116:119], v[32:35], v[152:155], a[116:119]\n\t" \
    "buffer_load_dwordx4 v[168:171], %[vW], %[rW], s39 offen\n\t" \
    "v_mfma_f32_16x16x32_f16 a[148:151], v[36:39], v[152:155], a[148:151]\n\t" \
    "s_add_u32 s39, s39, 32768\n\t" \
    "v_mfma_f32_16x16x32_f16 a[180:183], v[40:43], v[152:155], a[180:183]\n\t" \
    "buffer_load_dwordx4 v[172:175], %[vW], %[rW], s39 offen\n\t" \
    "v_mfma_f32_16x16x32_f16 a[212:215], v[44:47], v[152:155], a[212:215]\n\t" \
    "ds_read_b128 v[236:239], v18 offset:0\n\t" \
    "v_mfma_f32_16x16x32_f16 a[244:247], v[48:51], v[152:155], a[244:247]\n\t" \
    "ds_read_b128 v[152:155], %[vA0] offset:9216\n\t" \
    "s_waitcnt lgkmcnt(7)\n\t" \
    "v_mfma_f32_16x16x32_f16 a[248:251], v[48:51], v[156:159], a[248:251]\n\t" \
    "ds_read_b128 v[240:243], v18 offset:1024\n\t" \
    "v_mfma_f32_16x16x32_f16 a[216:219], v[44:47], v[156:159], a[216:219]\n\t" \
    "s_add_u32 s39, s39, 32768\n\t" \
    "v_mfma_f32_16x16x32_f16 a[184:187], v[40:43], v[156:159], a[184:187]\n\t" \
    "ds_read_b128 v[244:247], v18 offset:2048\n\t" \
    "v_mfma_f32_16x16x32_f16 a[152:155], v[36:39], v[156:159], a[152:155]\n\t" \
    "ds_read_b128 v[248:251], v18 offset:3072\n\t" \
    "v_mfma_f32_16x16x32_f16 a[120:123], v[32:35], v[156:159], a[120:123]\n\t" \
    "buffer_load_dwordx4 v[176:179], %[vW], %[rW], s39 offen\n\t" \
    "v_mfma_f32_16x16x32_f16 a[88:91], v[28:31], v[156:159], a[88:91]\n\t" \
    "s_add_u32 s39, s39, 32768\n\t" \
    "v_mfma_f32_16x16x32_f16 a[56:59], v[24:27], v[156:159], a[56:59]\n\t" \
    "buffer_store_dwordx4 v[232:235], %[vO], s[56:59], 0 offen sc0 sc1 nt\n\t" \
    "v_mfma_f32_16x16x32_f16 a[24:27], v[20:23], v[156:159], a[24:27]\n\t" \
    "ds_read_b128 v[156:159], %[vA0] offset:10240\n\t" \
    "s_waitcnt lgkmcnt(8)\n\t" \
    "v_mfma_f32_16x16x32_f16 a[28:31], v[20:23], v[160:163], a[28:31]\n\t" \
    "buffer_load_dwordx4 v[180:183], %[vW], %[rW], s39 offen\n\t" \
    "v_mfma_f32_16x16x32_f16 a[60:63], v[24:27], v[160:163], a[60:63]\n\t" \
    "s_add_u32 s39, s39, 32768\n\t" \
    "v_mfma_f32_16x16x32_f16 a[92:95], v[28:31], v[160:163], a[92:95]\n\t" \
    "buffer_load_dwordx4 v[184:187], %[vW], %[rW], s39 offen\n\t" \
    "v_mfma_f32_16x16x32_f16 a[124:127], v[32:35], v[160:163], a[124:127]\n\t" \
    "s_waitcnt lgkmcnt(3)\n\t" \
    "v_swap_b32 v237, v240\n\t" \
    "v_mfma_f32_16x16x32_f16 a[156:159], v[36:39], v[160:163], a[156:159]\n\t" \
    "s_waitcnt lgkmcnt(2)\n\t" \
    "v_swap_b32 v238, v244\n\t" \
    "v_mfma_f32_16x16x32_f16 a[188:191], v[40:43], v[160:163], a[188:191]\n\t" \
    "s_add_u32 s39, s39, 32768\n\t" \
    "v_mfma_f32_16x16x32_f16 a[220:223], v[44:47], v[160:163], a[220:223]\n\t" \
    "s_waitcnt lgkmcnt(1)\n\t" \
    "v_swap_b32 v239, v248\n\t" \
    "v_mfma_f32_16x16x32_f16 a[252:255], v[48:51], v[160:163], a[252:255]\n\t" \
    "ds_read_b128 v[160:163], %[vA0] offset:11264\n\t" \
    "s_waitcnt vmcnt(15)\n\t" \
    "v_mfma_f32_16x16x32_f16 a[224:227], v[80:83], v[148:151], a[224:227]\n\t" \
    "v_swap_b32 v242, v245\n\t" \
    "v_mfma_f32_16x16x32_f16 a[192:195], v[76:79], v[148:151], a[192:195]\n\t" \
    "buffer_load_dwordx4 v[188:191], %[vW], %[rW], s39 offen\n\t" \
    "v_mfma_f32_16x16x32_f16 a[160:163], v[72:75], v[148:151], a[160:163]\n\t" \
    "v_swap_b32 v243, v249\n\t" \
    "v_mfma_f32_16x16x32_f16 a[128:131], v[68:71], v[148:151], a[128:131]\n\t" \
    "v_swap_b32 v247, v250\n\t" \
    "v_mfma_f32_16x16x32_f16 a[96:99], v[64:67], v[148:151], a[96:99]\n\t" \
    "s_add_u32 s39, s39, 32768\n\t" \
    "v_mfma_f32_16x16x32_f16 a[64:67], v[60:63], v[148:151], a[64:67]\n\t" \
    "global_store_dwordx4 %[vA0], v[236:239], s[60:61] offset:0 sc0 sc1\n\t" \
    "v_mfma_f32_16x16x32_f16 a[32:35], v[56:59], v[148:151], a[32:35]\n\t" \
    "buffer_load_dwordx4 v[192:195], %[vW], %[rW], s39 offen\n\t" \
    "v_mfma_f32_16x16x32_f16 a[0:3], v[52:55], v[148:151], a[0:3]\n\t" \
    "ds_read_b128 v[148:151], %[vA0] offset:12288\n\t" \
    "v_mfma_f32_16x16x32_f16 a[4:7], v[52:55], v[152:155], a[4:7]\n\t" \
    "v_cvt_pk_f16_f32 v228, v196, v200\n\t" \
    "v_mfma_f32_16x16x32_f16 a[36:39], v[56:59], v[152:155], a[36:39]\n\t" \
    "global_store_dwordx4 %[vA0], v[240:243], s[60:61] offset:1024 sc0 sc1\n\t" \
    "v_mfma_f32_16x16x32_f16 a[68:71], v[60:63], v[152:155], a[68:71]\n\t" \
    "v_cvt_pk_f16_f32 v229, v204, v208\n\t" \
    "v_mfma_f32_16x16x32_f16 a[100:103], v[64:67], v[152:155], a[100:103]\n\t" \
    "ds_write_b64 %[vWr1], v[228:229] offset:32768\n\t" \
    "v_mfma_f32_16x16x32_f16 a[132:135], v[68:71], v[152:155], a[132:135]\n\t" \
    "v_cvt_pk_f16_f32 v230, v212, v216\n\t" \
    "v_mfma_f32_16x16x32_f16 a[164:167], v[72:75], v[152:155], a[164:167]\n\t" \
    "global_store_dwordx4 %[vA0], v[244:247], s[60:61] offset:2048 sc0 sc1\n\t" \
    "v_mfma_f32_16x16x32_f16 a[196:199], v[76:79], v[152:155], a[196:199]\n\t" \
    "v_cvt_pk_f16_f32 v231, v220, v224\n\t" \
    "v_mfma_f32_16x16x32_f16 a[228:231], v[80:83], v[152:155], a[228:231]\n\t" \
    "ds_read_b128 v[152:155], %[vA0] offset:13312\n\t" \
    "s_waitcnt lgkmcnt(4)\n\t" \
    "v_mfma_f32_16x16x32_f16 a[232:235], v[80:83], v[156:159], a[232:235]\n\t" \
    "ds_write_b64 %[vWr1], v[230:231] offset:32776\n\t" \
    "v_mfma_f32_16x16x32_f16 a[200:203], v[76:79], v[156:159], a[200:203]\n\t" \
    "v_cvt_pk_f16_f32 v228, v197, v201\n\t" \
    "v_mfma_f32_16x16x32_f16 a[168:171], v[72:75], v[156:159], a[168:171]\n\t" \
    "global_store_dwordx4 %[vA0], v[248:251], s[60:61] offset:3072 sc0 sc1\n\t" \
    "v_mfma_f32_16x16x32_f16 a[136:139], v[68:71], v[156:159], a[136:139]\n\t" \
    "v_cvt_pk_f16_f32 v229, v205, v209\n\t" \
    "v_mfma_f32_16x16x32_f16 a[104:107], v[64:67], v[156:159], a[104:107]\n\t" \
    "ds_write_b64 %[vWr1], v[228:229] offset:33792\n\t" \
    "v_mfma_f32_16x16x32_f16 a[72:75], v[60:63], v[156:159], a[72:75]\n\t" \
    "v_cvt_pk_f16_f32 v230, v213, v217\n\t" \
    "v_cvt_pk_f16_f32 v231, v221, v225\n\t" \
    "v_mfma_f32_16x16x32_f16 a[40:43], v[56:59], v[156:159], a[40:43]\n\t" \
    "s_add_u32 s55, s37, 24576\n\t" \
    "v_mfma_f32_16x16x32_f16 a[8:11], v[52:55], v[156:159], a[8:11]\n\t" \
    "ds_read_b128 v[156:159], %[vA0] offset:14336\n\t" \
    "s_waitcnt lgkmcnt(6)\n\t" \
    "v_mfma_f32_16x16x32_f16 a[12:15], v[52:55], v[160:163], a[12:15]\n\t" \
    "ds_write_b64 %[vWr1], v[230:231] offset:33800\n\t" \
    "v_mfma_f32_16x16x32_f16 a[44:47], v[56:59], v[160:163], a[44:47]\n\t" \
    "v_cvt_pk_f16_f32 v228, v198, v202\n\t" \
    "v_mfma_f32_16x16x32_f16 a[76:79], v[60:63], v[160:163], a[76:79]\n\t" \
    "buffer_load_dwordx4 v[116:119], %[vA0], %[rWS], s55 offen offset:0 sc0 sc1\n\t" \
    "v_mfma_f32_16x16x32_f16 a[108:111], v[64:67], v[160:163], a[108:111]\n\t" \
    "v_cvt_pk_f16_f32 v229, v206, v210\n\t" \
    "v_mfma_f32_16x16x32_f16 a[140:143], v[68:71], v[160:163], a[140:143]\n\t" \
    "ds_write_b64 %[vWr1], v[228:229] offset:34816\n\t" \
    "v_mfma_f32_16x16x32_f16 a[172:175], v[72:75], v[160:163], a[172:175]\n\t" \
    "v_cvt_pk_f16_f32 v230, v214, v218\n\t" \
    "v_mfma_f32_16x16x32_f16 a[204:207], v[76:79], v[160:163], a[204:207]\n\t" \
    "buffer_load_dwordx4 v[120:123], %[vA0], %[rWS], s55 offen offset:1024 sc0 sc1\n\t" \
    "v_mfma_f32_16x16x32_f16 a[236:239], v[80:83], v[160:163], a[236:239]\n\t" \
    "ds_read_b128 v[160:163], %[vA0] offset:15360\n\t" \
    "s_waitcnt lgkmcnt(8)\n\t" \
    "v_mfma_f32_16x16x32_f16 a[240:243], v[80:83], v[148:151], a[240:243]\n\t" \
    "v_cvt_pk_f16_f32 v231, v222, v226\n\t" \
    "v_mfma_f32_16x16x32_f16 a[208:211], v[76:79], v[148:151], a[208:211]\n\t" \
    "ds_write_b64 %[vWr1], v[230:231] offset:34824\n\t" \
    "v_mfma_f32_16x16x32_f16 a[176:179], v[72:75], v[148:151], a[176:179]\n\t" \
    "v_cvt_pk_f16_f32 v228, v199, v203\n\t" \
    "v_cvt_pk_f16_f32 v229, v207, v211\n\t" \
    "v_mfma_f32_16x16x32_f16 a[144:147], v[68:71], v[148:151], a[144:147]\n\t" \
    "buffer_load_dwordx4 v[124:127], %[vA0], %[rWS], s55 offen offset:2048 sc0 sc1\n\t" \
    "v_mfma_f32_16x16x32_f16 a[112:115], v[64:67], v[148:151], a[112:115]\n\t" \
    "ds_write_b64 %[vWr1], v[228:229] offset:35840\n\t" \
    "v_mfma_f32_16x16x32_f16 a[80:83], v[60:63], v[148:151], a[80:83]\n\t" \
    "v_cvt_pk_f16_f32 v230, v215, v219\n\t" \
    "v_cvt_pk_f16_f32 v231, v223, v227\n\t" \
    "v_mfma_f32_16x16x32_f16 a[48:51], v[56:59], v[148:151], a[48:51]\n\t" \
    "ds_write_b64 %[vWr1], v[230:231] offset:35848\n\t" \
    "v_mfma_f32_16x16x32_f16 a[16:19], v[52:55], v[148:151], a[16:19]\n\t" \
    "ds_read_b128 v[148:151], %[vA0] offset:16384\n\t" \
    "s_waitcnt lgkmcnt(10)\n\t" \
    "v_mfma_f32_16x16x32_f16 a[20:23], v[52:55], v[152:155], a[20:23]\n\t" \
    "buffer_load_dwordx4 v[128:131], %[vA0], %[rWS], s55 offen offset:3072 sc0 sc1\n\t" \
    "v_mfma_f32_16x16x32_f16 a[52:55], v[56:59], v[152:155], a[52:55]\n\t" \
    "s_add_u32 s55, s37, 28672\n\t" \
    "v_mfma_f32_16x16x32_f16 a[84:87], v[60:63], v[152:155], a[84:87]\n\t" \
    "buffer_load_dwordx4 v[132:135], %[vA0], %[rWS], s55 offen offset:0 sc0 sc1\n\t" \
    "v_mfma_f32_16x16x32_f16 a[116:119], v[64:67], v[152:155], a[116:119]\n\t" \
    "buffer_load_dwordx4 v[136:139], %[vA0], %[rWS], s55 offen offset:1024 sc0 sc1\n\t" \
    "v_mfma_f32_16x16x32_f16 a[148:151], v[68:71], v[152:155], a[148:151]\n\t" \
    "v_mfma_f32_16x16x32_f16 a[180:183], v[72:75], v[152:155], a[180:183]\n\t" \
    "v_mfma_f32_16x16x32_f16 a[212:215], v[76:79], v[152:155], a[212:215]\n\t" \
    "v_mfma_f32_16x16x32_f16 a[244:247], v[80:83], v[152:155], a[244:247]\n\t" \
    "ds_read_b128 v[152:155], %[vA0] offset:17408\n\t" \
    "s_waitcnt lgkmcnt(8)\n\t" \
    "v_mfma_f32_16x16x32_f16 a[248:251], v[80:83], v[156:159], a[248:251]\n\t" \
    "buffer_load_dwordx4 v[140:143], %[vA0], %[rWS], s55 offen offset:2048 sc0 sc1\n\t" \
    "v_mfma_f32_16x16x32_f16 a[216:219], v[76:79], v[156:159], a[216:219]\n\t" \
    "v_mfma_f32_16x16x32_f16 a[184:187], v[72:75], v[156:159], a[184:187]\n\t" \
    "v_mfma_f32_16x16x32_f16 a[152:155], v[68:71], v[156:159], a[152:155]\n\t" \
    "v_mfma_f32_16x16x32_f16 a[120:123], v[64:67], v[156:159], a[120:123]\n\t" \
    "v_mfma_f32_16x16x32_f16 a[88:91], v[60:63], v[156:159], a[88:91]\n\t" \
    "buffer_load_dwordx4 v[144:147], %[vA0], %[rWS], s55 offen offset:3072 sc0 sc1\n\t" \
    "v_mfma_f32_16x16x32_f16 a[56:59], v[56:59], v[156:159], a[56:59]\n\t" \
    "v_mfma_f32_16x16x32_f16 a[24:27], v[52:55], v[156:159], a[24:27]\n\t" \
    "ds_read_b128 v[156:159], %[vA0] offset:18432\n\t" \
    "s_waitcnt lgkmcnt(6)\n\t" \
    "v_mfma_f32_16x16x32_f16 a[28:31], v[52:55], v[160:163], a[28:31]\n\t" \
    "s_add_u32 s55, s38, 0\n\t" \
    "v_mfma_f32_16x16x32_f16 a[60:63], v[56:59], v[160:163], a[60:63]\n\t" \
    "v_mfma_f32_16x16x32_f16 a[92:95], v[60:63], v[160:163], a[92:95]\n\t" \
    "buffer_load_dwordx4 v[20:23], %[vA0], %[rWS], s55 offen offset:0 sc0 sc1\n\t" \
    "v_mfma_f32_16x16x32_f16 a[124:127], v[64:67], v[160:163], a[124:127]\n\t" \
    "v_mfma_f32_16x16x32_f16 a[156:159], v[68:71], v[160:163], a[156:159]\n\t" \
    "v_mfma_f32_16x16x32_f16 a[188:191], v[72:75], v[160:163], a[188:191]\n\t" \
    "v_mfma_f32_16x16x32_f16 a[220:223], v[76:79], v[160:163], a[220:223]\n\t" \
    "v_mfma_f32_16x16x32_f16 a[252:255], v[80:83], v[160:163], a[252:255]\n\t" \
    "ds_read_b128 v[160:163], %[vA0] offset:19456\n\t" \
    "s_waitcnt vmcnt(22) lgkmcnt(3)\n\t" \
    "v_mfma_f32_16x16x32_f16 a[224:227], v[112:115], v[148:151], a[224:227]\n\t" \
    "buffer_load_dwordx4 v[24:27], %[vA0], %[rWS], s55 offen offset:1024 sc0 sc1\n\t" \
    "v_mfma_f32_16x16x32_f16 a[192:195], v[108:111], v[148:151], a[192:195]\n\t" \
    "v_mfma_f32_16x16x32_f16 a[160:163], v[104:107], v[148:151], a[160:163]\n\t" \
    "v_mfma_f32_16x16x32_f16 a[128:131], v[100:103], v[148:151], a[128:131]\n\t" \
    "v_mfma_f32_16x16x32_f16 a[96:99], v[96:99], v[148:151], a[96:99]\n\t" \
    "buffer_load_dwordx4 v[28:31], %[vA0], %[rWS], s55 offen offset:2048 sc0 sc1\n\t" \
    "v_mfma_f32_16x16x32_f16 a[64:67], v[92:95], v[148:151], a[64:67]\n\t" \
    "v_mfma_f32_16x16x32_f16 a[32:35], v[88:91], v[148:151], a[32:35]\n\t" \
    "v_mfma_f32_16x16x32_f16 a[0:3], v[84:87], v[148:151], a[0:3]\n\t" \
    "ds_read_b128 v[148:151], %[vA0] offset:20480\n\t" \
    "s_waitcnt lgkmcnt(3)\n\t" \
    "v_mfma_f32_16x16x32_f16 a[4:7], v[84:87], v[152:155], a[4:7]\n\t" \
    "v_mfma_f32_16x16x32_f16 a[36:39], v[88:91], v[152:155], a[36:39]\n\t" \
    "buffer_load_dwordx4 v[32:35], %[vA0], %[rWS], s55 offen offset:3072 sc0 sc1\n\t" \
    "v_mfma_f32_16x16x32_f16 a[68:71], v[92:95], v[152:155], a[68:71]\n\t" \
    "v_mfma_f32_16x16x32_f16 a[100:103], v[96:99], v[152:155], a[100:103]\n\t" \
    "v_mfma_f32_16x16x32_f16 a[132:135], v[100:103], v[152:155], a[132:135]\n\t" \
    "v_mfma_f32_16x16x32_f16 a[164:167], v[104:107], v[152:155], a[164:167]\n\t" \
    "s_add_u32 s55, s38, 4096\n\t" \
    "v_mfma_f32_16x16x32_f16 a[196:199], v[108:111], v[152:155], a[196:199]\n\t" \
    "v_mfma_f32_16x16x32_f16 a[228:231], v[112:115], v[152:155], a[228:231]\n\t" \
    "ds_read_b128 v[152:155], %[vA0] offset:21504\n\t" \
    "s_waitcnt lgkmcnt(3)\n\t" \
    "v_mfma_f32_16x16x32_f16 a[232:235], v[112:115], v[156:159], a[232:235]\n\t" \
    "buffer_load_dwordx4 v[36:39], %[vA0], %[rWS], s55 offen offset:0 sc0 sc1\n\t" \
    "v_mfma_f32_16x16x32_f16 a[200:203], v[108:111], v[156:159], a[200:203]\n\t" \
    "v_mfma_f32_16x16x32_f16 a[168:171], v[104:107], v[156:159], a[168:171]\n\t" \
    "v_mfma_f32_16x16x32_f16 a[136:139], v[100:103], v[156:159], a[136:139]\n\t" \
    "v_mfma_f32_16x16x32_f16 a[104:107], v[96:99], v[156:159], a[104:107]\n\t" \
    "buffer_load_dwordx4 v[40:43], %[vA0], %[rWS], s55 offen offset:1024 sc0 sc1\n\t" \
    "v_mfma_f32_16x16x32_f16 a[72:75], v[92:95], v[156:159], a[72:75]\n\t" \
    "v_readlane_b32 s52, %[vTab0], s51\n\t" \
    "v_mfma_f32_16x16x32_f16 a[40:43], v[88:91], v[156:159], a[40:43]\n\t" \
    "s_add_u32 s42, s51, 32\n\t" \
    "v_mfma_f32_16x16x32_f16 a[8:11], v[84:87], v[156:159], a[8:11]\n\t" \
    "ds_read_b128 v[156:159], %[vA0] offset:22528\n\t" \
    "s_waitcnt lgkmcnt(3)\n\t" \
    "v_mfma_f32_16x16x32_f16 a[12:15], v[84:87], v[160:163], a[12:15]\n\t" \
    "v_readlane_b32 s53, %[vTab1], s51\n\t" \
    "v_mfma_f32_16x16x32_f16 a[44:47], v[88:91], v[160:163], a[44:47]\n\t" \
    "v_readlane_b32 s42, %[vTab0], s42\n\t" \
    "v_mfma_f32_16x16x32_f16 a[76:79], v[92:95], v[160:163], a[76:79]\n\t" \
    "s_add_u32 s51, s51, 1\n\t" \
    "v_mfma_f32_16x16x32_f16 a[108:111], v[96:99], v[160:163], a[108:111]\n\t" \
    "buffer_load_dwordx4 v[44:47], %[vA0], %[rWS], s55 offen offset:2048 sc0 sc1\n\t" \
    "v_mfma_f32_16x16x32_f16 a[140:143], v[100:103], v[160:163], a[140:143]\n\t" \
    "s_add_u32 s40, s66, s43\n\t" \
    "s_add_u32 s40, s40, %[sWk1]\n\t" \
    "v_mfma_f32_16x16x32_f16 a[172:175], v[104:107], v[160:163], a[172:175]\n\t" \
    "s_mov_b32 s56, s42\n\t" \
    "s_and_b32 s57, s53, 0xffff\n\t" \
    "v_mfma_f32_16x16x32_f16 a[204:207], v[108:111], v[160:163], a[204:207]\n\t" \
    "s_and_b32 s58, s53, 0x10000\n\t" \
    "s_bfe_i32 s53, s53, 0x10011\n\t" \
    "v_mfma_f32_16x16x32_f16 a[236:239], v[112:115], v[160:163], a[236:239]\n\t" \
    "ds_read_b128 v[160:163], %[vA0] offset:23552\n\t" \
    "s_waitcnt lgkmcnt(3)\n\t" \
    "v_mfma_f32_16x16x32_f16 a[240:243], v[112:115], v[148:151], a[240:243]\n\t" \
    "buffer_load_dwordx4 v[48:51], %[vA0], %[rWS], s55 offen offset:3072 sc0 sc1\n\t" \
    "v_mfma_f32_16x16x32_f16 a[208:211], v[108:111], v[148:151], a[208:211]\n\t" \
    "v_mfma_f32_16x16x32_f16 a[176:179], v[104:107], v[148:151], a[176:179]\n\t" \
    "v_mfma_f32_16x16x32_f16 a[144:147], v[100:103], v[148:151], a[144:147]\n\t" \
    "buffer_load_dwordx4 v[196:199], %[vW], %[rW], s40 offen nt\n\t" \
    "v_mfma_f32_16x16x32_f16 a[112:115], v[96:99], v[148:151], a[112:115]\n\t" \
    "v_mfma_f32_16x16x32_f16 a[80:83], v[92:95], v[148:151], a[80:83]\n\t" \
    "v_mfma_f32_16x16x32_f16 a[48:51], v[88:91], v[148:151], a[48:51]\n\t" \
    "s_add_u32 s40, s40, 32768\n\t" \
    "v_mfma_f32_16x16x32_f16 a[16:19], v[84:87], v[148:151], a[16:19]\n\t" \
    "ds_read_b128 v[148:151], %[vA0] offset:24576\n\t" \
    "s_waitcnt lgkmcnt(3)\n\t" \
    "v_mfma_f32_16x16x32_f16 a[20:23], v[84:87], v[152:155], a[20:23]\n\t" \
    "s_cmp_eq_u32 s69, 0\n\t" \
    "s_cselect_b32 s42, 1073741824, s52\n\t" \
    "v_mfma_f32_16x16x32_f16 a[52:55], v[88:91], v[152:155], a[52:55]\n\t" \
    "buffer_load_dwordx4 v[200:203], %[vW], %[rW], s40 offen nt\n\t" \
    "v_mfma_f32_16x16x32_f16 a[84:87], v[92:95], v[152:155], a[84:87]\n\t" \
    "buffer_load_dwordx4 v[236:239], %[vA0], %[rWS], s42 offen sc0 sc1\n\t" \
    "v_mfma_f32_16x16x32_f16 a[116:119], v[96:99], v[152:155], a[116:119]\n\t" \
    "s_add_u32 s52, s52, 262144\n\t" \
    "s_cmp_eq_u32 s69, 1\n\t" \
    "v_mfma_f32_16x16x32_f16 a[148:151], v[100:103], v[152:155], a[148:151]\n\t" \
    "s_cselect_b32 s42, 1073741824, s52\n\t" \
    "v_mfma_f32_16x16x32_f16 a[180:183], v[104:107], v[152:155], a[180:183]\n\t" \
    "buffer_load_dwordx4 v[240:243], %[vA0], %[rWS], s42 offen sc0 sc1\n\t" \
    "v_mfma_f32_16x16x32_f16 a[212:215], v[108:111], v[152:155], a[212:215]\n\t" \
    "s_add_u32 s40, s40, 32768\n\t" \
    "s_add_u32 s52, s52, 262144\n\t" \
    "v_mfma_f32_16x16x32_f16 a[244:247], v[112:115], v[152:155], a[244:247]\n\t" \
    "ds_read_b128 v[152:155], %[vA0] offset:25600\n\t" \
    "s_waitcnt lgkmcnt(3)\n\t" \
    "v_mfma_f32_16x16x32_f16 a[248:251], v[112:115], v[156:159], a[248:251]\n\t" \
    "s_cmp_eq_u32 s69, 2\n\t" \
    "s_cselect_b32 s42, 1073741824, s52\n\t" \
    "v_mfma_f32_16x16x32_f16 a[216:219], v[108:111], v[156:159], a[216:219]\n\t" \
    "buffer_load_dwordx4 v[204:207], %[vW], %[rW], s40 offen nt\n\t" \
    "v_mfma_f32_16x16x32_f16 a[184:187], v[104:107], v[156:159], a[184:187]\n\t" \
    "buffer_load_dwordx4 v[244:247], %[vA0], %[rWS], s42 offen sc0 sc1\n\t" \
    "v_mfma_f32_16x16x32_f16 a[152:155], v[100:103], v[156:159], a[152:155]\n\t" \
    "s_add_u32 s52, s52, 262144\n\t" \
    "s_cmp_eq_u32 s69, 3\n\t" \
    "v_mfma_f32_16x16x32_f16 a[120:123], v[96:99], v[156:159], a[120:123]\n\t" \
    "s_cselect_b32 s42, 1073741824, s52\n\t" \
    "s_add_u32 s40, s40, 32768\n\t" \
    "v_mfma_f32_16x16x32_f16 a[88:91], v[92:95], v[156:159], a[88:91]\n\t" \
    "buffer_load_dwordx4 v[248:251], %[vA0], %[rWS], s42 offen sc0 sc1\n\t" \
    "v_mfma_f32_16x16x32_f16 a[56:59], v[88:91], v[156:159], a[56:59]\n\t" \
    "buffer_load_dwordx4 v[208:211], %[vW], %[rW], s40 offen nt\n\t" \
    "v_mfma_f32_16x16x32_f16 a[24:27], v[84:87], v[156:159], a[24:27]\n\t" \
    "ds_read_b128 v[156:159], %[vA0] offset:26624\n\t" \
    "s_waitcnt lgkmcnt(3)\n\t" \
    "v_mfma_f32_16x16x32_f16 a[28:31], v[84:87], v[160:163], a[28:31]\n\t" \
    "s_add_u32 s40, s40, 32768\n\t" \
    "v_mfma_f32_16x16x32_f16 a[60:63], v[88:91], v[160:163], a[60:63]\n\t" \
    "buffer_load_dwordx4 v[212:215], %[vW], %[rW], s40 offen nt\n\t" \
    "v_mfma_f32_16x16x32_f16 a[92:95], v[92:95], v[160:163], a[92:95]\n\t" \
    "v_mfma_f32_16x16x32_f16 a[124:127], v[96:99], v[160:163], a[124:127]\n\t" \
    "s_add_u32 s40, s40, 32768\n\t" \
    "v_mfma_f32_16x16x32_f16 a[156:159], v[100:103], v[160:163], a[156:159]\n\t" \
    "v_mfma_f32_16x16x32_f16 a[188:191], v[104:107], v[160:163], a[188:191]\n\t" \
    "buffer_load_dwordx4 v[216:219], %[vW], %[rW], s40 offen nt\n\t" \
    "v_mfma_f32_16x16x32_f16 a[220:223], v[108:111], v[160:163], a[220:223]\n\t" \
    "v_mfma_f32_16x16x32_f16 a[252:255], v[112:115], v[160:163], a[252:255]\n\t" \
    "ds_read_b128 v[160:163], %[vA0] offset:27648\n\t" \
    "s_waitcnt vmcnt(18) lgkmcnt(3)\n\t" \
    "v_mfma_f32_16x16x32_f16 a[224:227], v[144:147], v[148:151], a[224:227]\n\t" \
    "s_add_u32 s40, s40, 32768\n\t" \
    "v_mfma_f32_16x16x32_f16 a[192:195], v[140:143], v[148:151], a[192:195]\n\t" \
    "v_mfma_f32_16x16x32_f16 a[160:163], v[136:139], v[148:151], a[160:163]\n\t" \
    "buffer_load_dwordx4 v[220:223], %[vW], %[rW], s40 offen nt\n\t" \
    "v_mfma_f32_16x16x32_f16 a[128:131], v[132:135], v[148:151], a[128:131]\n\t" \
    "v_mfma_f32_16x16x32_f16 a[96:99], v[128:131], v[148:151], a[96:99]\n\t" \
    "v_mfma_f32_16x16x32_f16 a[64:67], v[124:127], v[148:151], a[64:67]\n\t" \
    "v_mfma_f32_16x16x32_f16 a[32:35], v[120:123], v[148:151], a[32:35]\n\t" \
    "s_add_u32 s40, s40, 32768\n\t" \
    "v_mfma_f32_16x16x32_f16 a[0:3], v[116:119], v[148:151], a[0:3]\n\t" \
    "ds_read_b128 v[148:151], %[vA0] offset:28672\n\t" \
    "s_waitcnt lgkmcnt(3)\n\t" \
    "v_mfma_f32_16x16x32_f16 a[4:7], v[116:119], v[152:155], a[4:7]\n\t" \
    "buffer_load_dwordx4 v[224:227], %[vW], %[rW], s40 offen nt\n\t" \
    "v_mfma_f32_16x16x32_f16 a[36:39], v[120:123], v[152:155], a[36:39]\n\t" \
    "v_mfma_f32_16x16x32_f16 a[68:71], v[124:127], v[152:155], a[68:71]\n\t" \
    "v_mfma_f32_16x16x32_f16 a[100:103], v[128:131], v[152:155], a[100:103]\n\t" \
    "s_add_u32 s55, s38, 8192\n\t" \
    "v_mfma_f32_16x16x32_f16 a[132:135], v[132:135], v[152:155], a[132:135]\n\t" \
    "v_and_b32 v252, s53, v252\n\t" \
    "v_mfma_f32_16x16x32_f16 a[164:167], v[136:139], v[152:155], a[164:167]\n\t" \
    "buffer_load_dwordx4 v[52:55], %[vA0], %[rWS], s55 offen offset:0 sc0 sc1\n\t" \
    "v_mfma_f32_16x16x32_f16 a[196:199], v[140:143], v[152:155], a[196:199]\n\t" \
    "v_and_b32 v253, s53, v253\n\t" \
    "v_and_b32 v254, s53, v254\n\t" \
    "v_mfma_f32_16x16x32_f16 a[228:231], v[144:147], v[152:155], a[228:231]\n\t" \
    "ds_read_b128 v[152:155], %[vA0] offset:29696\n\t" \
    "s_waitcnt lgkmcnt(3)\n\t" \
    "v_mfma_f32_16x16x32_f16 a[232:235], v[144:147], v[156:159], a[232:235]\n\t" \
    "v_and_b32 v255, s53, v255\n\t" \
    "s_waitcnt vmcnt(9)\n\t" \
    "v_add_f32 v236, v236, v240\n\t" \
    "v_mfma_f32_16x16x32_f16 a[200:203], v[140:143], v[156:159], a[200:203]\n\t" \
    "v_add_f32 v237, v237, v241\n\t" \
    "v_mfma_f32_16x16x32_f16 a[168:171], v[136:139], v[156:159], a[168:171]\n\t" \
    "buffer_load_dwordx4 v[56:59], %[vA0], %[rWS], s55 offen offset:1024 sc0 sc1\n\t" \
    "v_mfma_f32_16x16x32_f16 a[136:139], v[132:135], v[156:159], a[136:139]\n\t" \
    "v_add_f32 v238, v238, v242\n\t" \
    "v_add_f32 v239, v239, v243\n\t" \
    "v_mfma_f32_16x16x32_f16 a[104:107], v[128:131], v[156:159], a[104:107]\n\t" \
    "v_add_f32 v252, v252, v236\n\t" \
    "v_mfma_f32_16x16x32_f16 a[72:75], v[124:127], v[156:159], a[72:75]\n\t" \
    "v_add_f32 v253, v253, v237\n\t" \
    "v_mfma_f32_16x16x32_f16 a[40:43], v[120:123], v[156:159], a[40:43]\n\t" \
    "v_add_f32 v254, v254, v238\n\t" \
    "v_mfma_f32_16x16x32_f16 a[8:11], v[116:119], v[156:159], a[8:11]\n\t" \
    "ds_read_b128 v[156:159], %[vA0] offset:30720\n\t" \
    "s_waitcnt lgkmcnt(3)\n\t" \
    "v_mfma_f32_16x16x32_f16 a[12:15], v[116:119], v[160:163], a[12:15]\n\t" \
    "buffer_load_dwordx4 v[60:63], %[vA0], %[rWS], s55 offen offset:2048 sc0 sc1\n\t" \
    "v_mfma_f32_16x16x32_f16 a[44:47], v[120:123], v[160:163], a[44:47]\n\t" \
    "v_add_f32 v255, v255, v239\n\t" \
    "s_waitcnt vmcnt(8)\n\t" \
    "v_add_f32 v244, v244, v248\n\t" \
    "v_mfma_f32_16x16x32_f16 a[76:79], v[124:127], v[160:163], a[76:79]\n\t" \
    "v_add_f32 v245, v245, v249\n\t" \
    "v_add_f32 v246, v246, v250\n\t" \
    "v_mfma_f32_16x16x32_f16 a[108:111], v[128:131], v[160:163], a[108:111]\n\t" \
    "v_add_f32 v247, v247, v251\n\t" \
    "v_mfma_f32_16x16x32_f16 a[140:143], v[132:135], v[160:163], a[140:143]\n\t" \
    "buffer_load_dwordx4 v[64:67], %[vA0], %[rWS], s55 offen offset:3072 sc0 sc1\n\t" \
    "v_mfma_f32_16x16x32_f16 a[172:175], v[136:139], v[160:163], a[172:175]\n\t" \
    "v_add_f32 v252, v252, v244\n\t" \
    "v_add_f32 v253, v253, v245\n\t" \
    "v_mfma_f32_16x16x32_f16 a[204:207], v[140:143], v[160:163], a[204:207]\n\t" \
    "v_add_f32 v254, v254, v246\n\t" \
    "v_mfma_f32_16x16x32_f16 a[236:239], v[144:147], v[160:163], a[236:239]\n\t" \
    "ds_read_b128 v[160:163], %[vA0] offset:31744\n\t" \
    "s_waitcnt lgkmcnt(3)\n\t" \
    "v_mfma_f32_16x16x32_f16 a[240:243], v[144:147], v[148:151], a[240:243]\n\t" \
    "s_add_u32 s55, s38, 12288\n\t" \
    "v_add_f32 v255, v255, v247\n\t" \
    "v_mfma_f32_16x16x32_f16 a[208:211], v[140:143], v[148:151], a[208:211]\n\t" \
    "v_med3_f32 v236, v252, %[m60], %[p60]\n\t" \
    "v_mfma_f32_16x16x32_f16 a[176:179], v[136:139], v[148:151], a[176:179]\n\t" \
    "buffer_load_dwordx4 v[68:71], %[vA0], %[rWS], s55 offen offset:0 sc0 sc1\n\t" \
    "v_mfma_f32_16x16x32_f16 a[144:147], v[132:135], v[148:151], a[144:147]\n\t" \
    "v_med3_f32 v237, v253, %[m60], %[p60]\n\t" \
    "v_med3_f32 v238, v254, %[m60], %[p60]\n\t" \
    "v_mfma_f32_16x16x32_f16 a[112:115], v[128:131], v[148:151], a[112:115]\n\t" \
    "v_med3_f32 v239, v255, %[m60], %[p60]\n\t" \
    "v_mul_f32 v236, 0xbfb8aa3b, v236\n\t" \
    "v_mfma_f32_16x16x32_f16 a[80:83], v[124:127], v[148:151], a[80:83]\n\t" \
    "v_mul_f32 v237, 0xbfb8aa3b, v237\n\t" \
    "v_mfma_f32_16x16x32_f16 a[48:51], v[120:123], v[148:151], a[48:51]\n\t" \
    "v_mul_f32 v238, 0xbfb8aa3b, v238\n\t" \
    "v_mfma_f32_16x16x32_f16 a[16:19], v[116:119], v[148:151], a[16:19]\n\t" \
    "s_waitcnt lgkmcnt(0)\n\t" \
    "s_barrier\n\t" \
    "ds_read_b128 v[148:151], %[vA0] offset:32768\n\t" \
    "v_mfma_f32_16x16x32_f16 a[20:23], v[116:119], v[152:155], a[20:23]\n\t" \
    "buffer_load_dwordx4 v[72:75], %[vA0], %[rWS], s55 offen offset:1024 sc0 sc1\n\t" \
    "v_mfma_f32_16x16x32_f16 a[52:55], v[120:123], v[152:155], a[52:55]\n\t" \
    "v_mul_f32 v239, 0xbfb8aa3b, v239\n\t" \
    "v_mfma_f32_16x16x32_f16 a[84:87], v[124:127], v[152:155], a[84:87]\n\t" \
    "v_exp_f32 v236, v236\n\t" \
    "v_mfma_f32_16x16x32_f16 a[116:119], v[128:131], v[152:155], a[116:119]\n\t" \
    "v_exp_f32 v237, v237\n\t" \
    "v_mfma_f32_16x16x32_f16 a[148:151], v[132:135], v[152:155], a[148:151]\n\t" \
    "v_exp_f32 v238, v238\n\t" \
    "v_mfma_f32_16x16x32_f16 a[180:183], v[136:139], v[152:155], a[180:183]\n\t" \
    "v_exp_f32 v239, v239\n\t" \
    "v_mfma_f32_16x16x32_f16 a[212:215], v[140:143], v[152:155], a[212:215]\n\t" \
    "buffer_load_dwordx4 v[76:79], %[vA0], %[rWS], s55 offen offset:2048 sc0 sc1\n\t" \
    "v_mfma_f32_16x16x32_f16 a[244:247], v[144:147], v[152:155], a[244:247]\n\t" \
    "ds_read_b128 v[152:155], %[vA0] offset:33792\n\t" \
    "v_mfma_f32_16x16x32_f16 a[248:251], v[144:147], v[156:159], a[248:251]\n\t" \
    "v_add_f32 v236, 1.0, v236\n\t" \
    "v_add_f32 v237, 1.0, v237\n\t" \
    "v_mfma_f32_16x16x32_f16 a[216:219], v[140:143], v[156:159], a[216:219]\n\t" \
    "v_add_f32 v238, 1.0, v238\n\t" \
    "v_add_f32 v239, 1.0, v239\n\t" \
    "v_mfma_f32_16x16x32_f16 a[184:187], v[136:139], v[156:159], a[184:187]\n\t" \
    "v_rcp_f32 v236, v236\n\t" \
    "v_mfma_f32_16x16x32_f16 a[152:155], v[132:135], v[156:159], a[152:155]\n\t" \
    "buffer_load_dwordx4 v[80:83], %[vA0], %[rWS], s55 offen offset:3072 sc0 sc1\n\t" \
    "v_mfma_f32_16x16x32_f16 a[120:123], v[128:131], v[156:159], a[120:123]\n\t" \
    "v_rcp_f32 v237, v237\n\t" \
    "v_mfma_f32_16x16x32_f16 a[88:91], v[124:127], v[156:159], a[88:91]\n\t" \
    "s_add_u32 s55, s38, 16384\n\t" \
    "v_mfma_f32_16x16x32_f16 a[56:59], v[120:123], v[156:159], a[56:59]\n\t" \
    "v_rcp_f32 v238, v238\n\t" \
    "v_mfma_f32_16x16x32_f16 a[24:27], v[116:119], v[156:159], a[24:27]\n\t" \
    "ds_read_b128 v[156:159], %[vA0] offset:34816\n\t" \
    "v_mfma_f32_16x16x32_f16 a[28:31], v[116:119], v[160:163], a[28:31]\n\t" \
    "v_rcp_f32 v239, v239\n\t" \
    "v_mfma_f32_16x16x32_f16 a[60:63], v[120:123], v[160:163], a[60:63]\n\t" \
    "buffer_load_dwordx4 v[84:87], %[vA0], %[rWS], s55 offen offset:0 sc0 sc1\n\t" \
    "v_mfma_f32_16x16x32_f16 a[92:95], v[124:127], v[160:163], a[92:95]\n\t" \
    "v_mul_f32 v232, v252, v236\n\t" \
    "v_mul_f32 v233, v253, v237\n\t" \
    "v_mfma_f32_16x16x32_f16 a[124:127], v[128:131], v[160:163], a[124:127]\n\t" \
    "v_mul_f32 v234, v254, v238\n\t" \
    "v_mfma_f32_16x16x32_f16 a[156:159], v[132:135], v[160:163], a[156:159]\n\t" \
    "buffer_load_dwordx4 v[88:91], %[vA0], %[rWS], s55 offen offset:1024 sc0 sc1\n\t" \
    "v_mfma_f32_16x16x32_f16 a[188:191], v[136:139], v[160:163], a[188:191]\n\t" \
    "v_mul_f32 v235, v255, v239\n\t" \
    "v_mfma_f32_16x16x32_f16 a[220:223], v[140:143], v[160:163], a[220:223]\n\t" \
    "buffer_load_dwordx4 v[92:95], %[vA0], %[rWS], s55 offen offset:2048 sc0 sc1\n\t" \
    "v_mfma_f32_16x16x32_f16 a[252:255], v[144:147], v[160:163], a[252:255]\n\t" \
    "buffer_load_dwordx4 v[96:99], %[vA0], %[rWS], s55 offen offset:3072 sc0 sc1\n\t" \
    "s_add_u32 s36, s36, 1\n\t" \
    "ds_read_b128 v[160:163], %[vA0] offset:35840\n\t" \
    "s_waitcnt lgkmcnt(3)\n\t" \
    "v_mfma_f32_16x16x32_f16 a[224:227], v[48:51], v[148:151], a[224:227]\n\t" \
    "s_mov_b32 s37, s38\n\t" \
    "v_cvt_pk_f16_f32 v228, v164, v168\n\t" \
    "v_mfma_f32_16x16x32_f16 a[192:195], v[44:47], v[148:151], a[192:195]\n\t" \
    "v_mfma_f32_16x16x32_f16 a[160:163], v[40:43], v[148:151], a[160:163]\n\t" \
    "v_cvt_pk_f16_f32 v229, v172, v176\n\t" \
    "v_mfma_f32_16x16x32_f16 a[128:131], v[36:39], v[148:151], a[128:131]\n\t" \
    "ds_write_b64 %[vWr0], v[228:229] offset:0\n\t" \
    "v_mfma_f32_16x16x32_f16 a[96:99], v[32:35], v[148:151], a[96:99]\n\t" \
    "v_cvt_pk_f16_f32 v230, v180, v184\n\t" \
    "v_mfma_f32_16x16x32_f16 a[64:67], v[28:31], v[148:151], a[64:67]\n\t" \
    "v_mfma_f32_16x16x32_f16 a[32:35], v[24:27], v[148:151], a[32:35]\n\t" \
    "v_cvt_pk_f16_f32 v231, v188, v192\n\t" \
    "v_mfma_f32_16x16x32_f16 a[0:3], v[20:23], v[148:151], a[0:3]\n\t" \
    "ds_read_b128 v[148:151], %[vA0] offset:36864\n\t" \
    "s_waitcnt lgkmcnt(4)\n\t" \
    "v_mfma_f32_16x16x32_f16 a[4:7], v[20:23], v[152:155], a[4:7]\n\t" \
    "s_cmp_ge_u32 s48, 2\n\ts_cselect_b64 exec, -1, 0\n\ts_cselect_b32 s69, %[own], 9\n\tds_read_b128 v[252:255], %[vStg] offset:19456\n\ts_mov_b64 exec, -1\n\t" \
    "s_waitcnt lgkmcnt(4)\n\t" \
    "v_mfma_f32_16x16x32_f16 a[36:39], v[24:27], v[152:155], a[36:39]\n\t" \
    "ds_write_b64 %[vWr0], v[230:231] offset:8\n\t" \
    "v_mfma_f32_16x16x32_f16 a[68:71], v[28:31], v[152:155], a[68:71]\n\t" \
    "v_cvt_pk_f16_f32 v228, v165, v169\n\t" \
    "s_add_u32 s55, s37, 20480\n\t" \
    "v_mfma_f32_16x16x32_f16 a[100:103], v[32:35], v[152:155], a[100:103]\n\t" \
    "v_cvt_pk_f16_f32 v229, v173, v177\n\t" \
    "v_mfma_f32_16x16x32_f16 a[132:135], v[36:39], v[152:155], a[132:135]\n\t" \
    "buffer_load_dwordx4 v[100:103], %[vA0], %[rWS], s55 offen offset:0 sc0 sc1\n\t" \
    "v_mfma_f32_16x16x32_f16 a[164:167], v[40:43], v[152:155], a[164:167]\n\t" \
    "ds_write_b64 %[vWr0], v[228:229] offset:1024\n\t" \
    "v_mfma_f32_16x16x32_f16 a[196:199], v[44:47], v[152:155], a[196:199]\n\t" \
    "v_cvt_pk_f16_f32 v230, v181, v185\n\t" \
    "v_cvt_pk_f16_f32 v231, v189, v193\n\t" \
    "v_mfma_f32_16x16x32_f16 a[228:231], v[48:51], v[152:155], a[228:231]\n\t" \
    "ds_read_b128 v[152:155], %[vA0] offset:37888\n\t" \
    "s_waitcnt lgkmcnt(6)\n\t" \
    "v_mfma_f32_16x16x32_f16 a[232:235], v[48:51], v[156:159], a[232:235]\n\t" \
    "ds_write_b64 %[vWr0], v[230:231] offset:1032\n\t" \
    "v_mfma_f32_16x16x32_f16 a[200:203], v[44:47], v[156:159], a[200:203]\n\t" \
    "buffer_load_dwordx4 v[104:107], %[vA0], %[rWS], s55 offen offset:1024 sc0 sc1\n\t" \
    "v_mfma_f32_16x16x32_f16 a[168:171], v[40:43], v[156:159], a[168:171]\n\t" \
    "v_cvt_pk_f16_f32 v228, v166, v170\n\t" \
    "v_cvt_pk_f16_f32 v229, v174, v178\n\t" \
    "v_mfma_f32_16x16x32_f16 a[136:139], v[36:39], v[156:159], a[136:139]\n\t" \
    "ds_write_b64 %[vWr0], v[228:229] offset:2048\n\t" \
    "v_mfma_f32_16x16x32_f16 a[104:107], v[32:35], v[156:159], a[104:107]\n\t" \
    "v_cvt_pk_f16_f32 v230, v182, v186\n\t" \
    "v_mfma_f32_16x16x32_f16 a[72:75], v[28:31], v[156:159], a[72:75]\n\t" \
    "buffer_load_dwordx4 v[108:111], %[vA0], %[rWS], s55 offen offset:2048 sc0 sc1\n\t" \
    "v_mfma_f32_16x16x32_f16 a[40:43], v[24:27], v[156:159], a[40:43]\n\t" \
    "s_add_u32 s66, s66, 0x400000\n\t" \
    "s_and_b32 s66, s66, 0xfffffff\n\t" \
    "v_mfma_f32_16x16x32_f16 a[8:11], v[20:23], v[156:159], a[8:11]\n\t" \
    "ds_read_b128 v[156:159], %[vA0] offset:38912\n\t" \
    "s_waitcnt lgkmcnt(8)\n\t" \
    "v_mfma_f32_16x16x32_f16 a[12:15], v[20:23], v[160:163], a[12:15]\n\t" \
    "v_cvt_pk_f16_f32 v231, v190, v194\n\t" \
    "v_mfma_f32_16x16x32_f16 a[44:47], v[24:27], v[160:163], a[44:47]\n\t" \
    "ds_write_b64 %[vWr0], v[230:231] offset:2056\n\t" \
    "v_mfma_f32_16x16x32_f16 a[76:79], v[28:31], v[160:163], a[76:79]\n\t" \
    "v_cvt_pk_f16_f32 v228, v167, v171\n\t" \
    "s_add_u32 s39, s66, s43\n\t" \
    "v_mfma_f32_16x16x32_f16 a[108:111], v[32:35], v[160:163], a[108:111]\n\t" \
    "s_add_u32 s39, s39, %[sWk0]\n\t" \
    "v_mfma_f32_16x16x32_f16 a[140:143], v[36:39], v[160:163], a[140:143]\n\t" \
    "buffer_load_dwordx4 v[112:115], %[vA0], %[rWS], s55 offen offset:3072 sc0 sc1\n\t" \
    "v_mfma_f32_16x16x32_f16 a[172:175], v[40:43], v[160:163], a[172:175]\n\t" \
    "v_cvt_pk_f16_f32 v229, v175, v179\n\t" \
    "v_mfma_f32_16x16x32_f16 a[204:207], v[44:47], v[160:163], a[204:207]\n\t" \
    "ds_write_b64 %[vWr0], v[228:229] offset:3072\n\t" \
    "v_mfma_f32_16x16x32_f16 a[236:239], v[48:51], v[160:163], a[236:239]\n\t" \
    "ds_read_b128 v[160:163], %[vA0] offset:39936\n\t" \
    "s_waitcnt lgkmcnt(9)\n\t" \
    "v_mfma_f32_16x16x32_f16 a[240:243], v[48:51], v[148:151], a[240:243]\n\t" \
    "v_cvt_pk_f16_f32 v230, v183, v187\n\t" \
    "v_cvt_pk_f16_f32 v231, v191, v195\n\t" \
    "v_mfma_f32_16x16x32_f16 a[208:211], v[44:47], v[148:151], a[208:211]\n\t" \
    "ds_write_b64 %[vWr0], v[230:231] offset:3080\n\t" \
    "v_mfma_f32_16x16x32_f16 a[176:179], v[40:43], v[148:151], a[176:179]\n\t" \
    "buffer_load_dwordx4 v[164:167], %[vW], %[rW], s39 offen\n\t" \
    "v_mfma_f32_16x16x32_f16 a[144:147], v[36:39], v[148:151], a[144:147]\n\t" \
    "s_add_u32 s38, s38, 32768\n\t" \
    "s_and_b32 s38, s38, 0x1fffff\n\t" \
    "v_mfma_f32_16x16x32_f16 a[112:115], v[32:35], v[148:151], a[112:115]\n\t" \
    "s_or_b32 s38, s38, %[sA0]\n\t" \
    "s_add_u32 s39, s39, 32768\n\t" \
    "v_mfma_f32_16x16x32_f16 a[80:83], v[28:31], v[148:151], a[80:83]\n\t" \
    "s_add_u32 s41, %[own], 3\n\t" \
    "s_and_b32 s41, s41, 3\n\t" \
    "v_mfma_f32_16x16x32_f16 a[48:51], v[24:27], v[148:151], a[48:51]\n\t" \
    "s_lshl_b32 s41, s41, 12\n\t" \
    "v_add_u32 v18, s41, %[vStg]\n\t" \
    "v_mfma_f32_16x16x32_f16 a[16:19], v[20:23], v[148:151], a[16:19]\n\t" \
    "ds_read_b128 v[148:151], %[vA0] offset:40960\n\t" \
    "s_waitcnt lgkmcnt(8)\n\t" \
    "v_mfma_f32_16x16x32_f16 a[20:23], v[20:23], v[152:155], a[20:23]\n\t" \
    "s_lshl_b32 s41, s41, 1\n\t" \
    "s_add_u32 s41, s41, s50\n\t" \
    "v_mfma_f32_16x16x32_f16 a[52:55], v[24:27], v[152:155], a[52:55]\n\t" \
    "s_add_u32 s41, s41, 4096\n\t" \
    "s_add_u32 s60, %[wsLo], s41\n\t" \
    "v_mfma_f32_16x16x32_f16 a[84:87], v[28:31], v[152:155], a[84:87]\n\t" \
    "s_addc_u32 s61, %[wsHi], 0\n\t" \
    "v_mfma_f32_16x16x32_f16 a[116:119], v[32:35], v[152:155], a[116:119]\n\t" \
    "buffer_load_dwordx4 v[168:171], %[vW], %[rW], s39 offen\n\t" \
    "v_mfma_f32_16x16x32_f16 a[148:151], v[36:39], v[152:155], a[148:151]\n\t" \
    "s_add_u32 s39, s39, 32768\n\t" \
    "v_mfma_f32_16x16x32_f16 a[180:183], v[40:43], v[152:155], a[180:183]\n\t" \
    "buffer_load_dwordx4 v[172:175], %[vW], %[rW], s39 offen\n\t" \
    "v_mfma_f32_16x16x32_f16 a[212:215], v[44:47], v[152:155], a[212:215]\n\t" \
    "ds_read_b128 v[236:239], v18 offset:0\n\t" \
    "v_mfma_f32_16x16x32_f16 a[244:247], v[48:51], v[152:155], a[244:247]\n\t" \
    "ds_read_b128 v[152:155], %[vA0] offset:41984\n\t" \
    "s_waitcnt lgkmcnt(7)\n\t" \
    "v_mfma_f32_16x16x32_f16 a[248:251], v[48:51], v[156:159], a[248:251]\n\t" \
    "ds_read_b128 v[240:243], v18 offset:1024\n\t" \
    "v_mfma_f32_16x16x32_f16 a[216:219], v[44:47], v[156:159], a[216:219]\n\t" \
    "s_add_u32 s39, s39, 32768\n\t" \
    "v_mfma_f32_16x16x32_f16 a[184:187], v[40:43], v[156:159], a[184:187]\n\t" \
    "ds_read_b128 v[244:247], v18 offset:2048\n\t" \
    "v_mfma_f32_16x16x32_f16 a[152:155], v[36:39], v[156:159], a[152:155]\n\t" \
    "ds_read_b128 v[248:251], v18 offset:3072\n\t" \
    "v_mfma_f32_16x16x32_f16 a[120:123], v[32:35], v[156:159], a[120:123]\n\t" \
    "buffer_load_dwordx4 v[176:179], %[vW], %[rW], s39 offen\n\t" \
    "v_mfma_f32_16x16x32_f16 a[88:91], v[28:31], v[156:159], a[88:91]\n\t" \
    "s_add_u32 s39, s39, 32768\n\t" \
    "v_mfma_f32_16x16x32_f16 a[56:59], v[24:27], v[156:159], a[56:59]\n\t" \
    "buffer_store_dwordx4 v[232:235], %[vO], s[56:59], 0 offen sc0 sc1 nt\n\t" \
    "v_mfma_f32_16x16x32_f16 a[24:27], v[20:23], v[156:159], a[24:27]\n\t" \
    "ds_read_b128 v[156:159], %[vA0] offset:43008\n\t" \
    "s_waitcnt lgkmcnt(8)\n\t" \
    "v_mfma_f32_16x16x32_f16 a[28:31], v[20:23], v[160:163], a[28:31]\n\t" \
    "buffer_load_dwordx4 v[180:183], %[vW], %[rW], s39 offen\n\t" \
    "v_mfma_f32_16x16x32_f16 a[60:63], v[24:27], v[160:163], a[60:63]\n\t" \
    "s_add_u32 s39, s39, 32768\n\t" \
    "v_mfma_f32_16x16x32_f16 a[92:95], v[28:31], v[160:163], a[92:95]\n\t" \
    "buffer_load_dwordx4 v[184:187], %[vW], %[rW], s39 offen\n\t" \
    "v_mfma_f32_16x16x32_f16 a[124:127], v[32:35], v[160:163], a[124:127]\n\t" \
    "s_waitcnt lgkmcnt(3)\n\t" \
    "v_swap_b32 v237, v240\n\t" \
    "v_mfma_f32_16x16x32_f16 a[156:159], v[36:39], v[160:163], a[156:159]\n\t" \
    "s_waitcnt lgkmcnt(2)\n\t" \
    "v_swap_b32 v238, v244\n\t" \
    "v_mfma_f32_16x16x32_f16 a[188:191], v[40:43], v[160:163], a[188:191]\n\t" \
    "s_add_u32 s39, s39, 32768\n\t" \
    "v_mfma_f32_16x16x32_f16 a[220:223], v[44:47], v[160:163], a[220:223]\n\t" \
    "s_waitcnt lgkmcnt(1)\n\t" \
    "v_swap_b32 v239, v248\n\t" \
    "v_mfma_f32_16x16x32_f16 a[252:255], v[48:51], v[160:163], a[252:255]\n\t" \
    "ds_read_b128 v[160:163], %[vA0] offset:44032\n\t" \
    "s_waitcnt vmcnt(15)\n\t" \
    "v_mfma_f32_16x16x32_f16 a[224:227], v[80:83], v[148:151], a[224:227]\n\t" \
    "v_swap_b32 v242, v245\n\t" \
    "v_mfma_f32_16x16x32_f16 a[192:195], v[76:79], v[148:151], a[192:195]\n\t" \
    "buffer_load_dwordx4 v[188:191], %[vW], %[rW], s39 offen\n\t" \
    "v_mfma_f32_16x16x32_f16 a[160:163], v[72:75], v[148:151], a[160:163]\n\t" \
    "v_swap_b32 v243, v249\n\t" \
    "v_mfma_f32_16x16x32_f16 a[128:131], v[68:71], v[148:151], a[128:131]\n\t" \
    "v_swap_b32 v247, v250\n\t" \
    "v_mfma_f32_16x16x32_f16 a[96:99], v[64:67], v[148:151], a[96:99]\n\t" \
    "s_add_u32 s39, s39, 32768\n\t" \
    "v_mfma_f32_16x16x32_f16 a[64:67], v[60:63], v[148:151], a[64:67]\n\t" \
    "global_store_dwordx4 %[vA0], v[236:239], s[60:61] offset:0 sc0 sc1\n\t" \
    "v_mfma_f32_16x16x32_f16 a[32:35], v[56:59], v[148:151], a[32:35]\n\t" \
    "buffer_load_dwordx4 v[192:195], %[vW], %[rW], s39 offen\n\t" \
    "v_mfma_f32_16x16x32_f16 a[0:3], v[52:55], v[148:151], a[0:3]\n\t" \
    "ds_read_b128 v[148:151], %[vA0] offset:45056\n\t" \
    "v_mfma_f32_16x16x32_f16 a[4:7], v[52:55], v[152:155], a[4:7]\n\t" \
    "v_cvt_pk_f16_f32 v228, v196, v200\n\t" \
    "v_mfma_f32_16x16x32_f16 a[36:39], v[56:59], v[152:155], a[36:39]\n\t" \
    "global_store_dwordx4 %[vA0], v[240:243], s[60:61] offset:1024 sc0 sc1\n\t" \
    "v_mfma_f32_16x16x32_f16 a[68:71], v[60:63], v[152:155], a[68:71]\n\t" \
    "v_cvt_pk_f16_f32 v229, v204, v208\n\t" \
    "v_mfma_f32_16x16x32_f16 a[100:103], v[64:67], v[152:155], a[100:103]\n\t" \
    "ds_write_b64 %[vWr1], v[228:229] offset:0\n\t" \
    "v_mfma_f32_16x16x32_f16 a[132:135], v[68:71], v[152:155], a[132:135]\n\t" \
    "v_cvt_pk_f16_f32 v230, v212, v216\n\t" \
    "v_mfma_f32_16x16x32_f16 a[164:167], v[72:75], v[152:155], a[164:167]\n\t" \
    "global_store_dwordx4 %[vA0], v[244:247], s[60:61] offset:2048 sc0 sc1\n\t" \
    "v_mfma_f32_16x16x32_f16 a[196:199], v[76:79], v[152:155], a[196:199]\n\t" \
    "v_cvt_pk_f16_f32 v231, v220, v224\n\t" \
    "v_mfma_f32_16x16x32_f16 a[228:231], v[80:83], v[152:155], a[228:231]\n\t" \
    "ds_read_b128 v[152:155], %[vA0] offset:46080\n\t" \
    "s_waitcnt lgkmcnt(4)\n\t" \
    "v_mfma_f32_16x16x32_f16 a[232:235], v[80:83], v[156:159], a[232:235]\n\t" \
    "ds_write_b64 %[vWr1], v[230:231] offset:8\n\t" \
    "v_mfma_f32_16x16x32_f16 a[200:203], v[76:79], v[156:159], a[200:203]\n\t" \
    "v_cvt_pk_f16_f32 v228, v197, v201\n\t" \
    "v_mfma_f32_16x16x32_f16 a[168:171], v[72:75], v[156:159], a[168:171]\n\t" \
    "global_store_dwordx4 %[vA0], v[248:251], s[60:61] offset:3072 sc0 sc1\n\t" \
    "v_mfma_f32_16x16x32_f16 a[136:139], v[68:71], v[156:159], a[136:139]\n\t" \
    "v_cvt_pk_f16_f32 v229, v205, v209\n\t" \
    "v_mfma_f32_16x16x32_f16 a[104:107], v[64:67], v[156:159], a[104:107]\n\t" \
    "ds_write_b64 %[vWr1], v[228:229] offset:1024\n\t" \
    "v_mfma_f32_16x16x32_f16 a[72:75], v[60:63], v[156:159], a[72:75]\n\t" \
    "v_cvt_pk_f16_f32 v230, v213, v217\n\t" \
    "v_cvt_pk_f16_f32 v231, v221, v225\n\t" \
    "v_mfma_f32_16x16x32_f16 a[40:43], v[56:59], v[156:159], a[40:43]\n\t" \
    "s_add_u32 s55, s37, 24576\n\t" \
    "v_mfma_f32_16x16x32_f16 a[8:11], v[52:55], v[156:159], a[8:11]\n\t" \
    "ds_read_b128 v[156:159], %[vA0] offset:47104\n\t" \
    "s_waitcnt lgkmcnt(6)\n\t" \
    "v_mfma_f32_16x16x32_f16 a[12:15], v[52:55], v[160:163], a[12:15]\n\t" \
    "ds_write_b64 %[vWr1], v[230:231] offset:1032\n\t" \
    "v_mfma_f32_16x16x32_f16 a[44:47], v[56:59], v[160:163], a[44:47]\n\t" \
    "v_cvt_pk_f16_f32 v228, v198, v202\n\t" \
    "v_mfma_f32_16x16x32_f16 a[76:79], v[60:63], v[160:163], a[76:79]\n\t" \
    "buffer_load_dwordx4 v[116:119], %[vA0], %[rWS], s55 offen offset:0 sc0 sc1\n\t" \
    "v_mfma_f32_16x16x32_f16 a[108:111], v[64:67], v[160:163], a[108:111]\n\t" \
    "v_cvt_pk_f16_f32 v229, v206, v210\n\t" \
    "v_mfma_f32_16x16x32_f16 a[140:143], v[68:71], v[160:163], a[140:143]\n\t" \
    "ds_write_b64 %[vWr1], v[228:229] offset:2048\n\t" \
    "v_mfma_f32_16x16x32_f16 a[172:175], v[72:75], v[160:163], a[172:175]\n\t" \
    "v_cvt_pk_f16_f32 v230, v214, v218\n\t" \
    "v_mfma_f32_16x16x32_f16 a[204:207], v[76:79], v[160:163], a[204:207]\n\t" \
    "buffer_load_dwordx4 v[120:123], %[vA0], %[rWS], s55 offen offset:1024 sc0 sc1\n\t" \
    "v_mfma_f32_16x16x32_f16 a[236:239], v[80:83], v[160:163], a[236:239]\n\t" \
    "ds_read_b128 v[160:163], %[vA0] offset:48128\n\t" \
    "s_waitcnt lgkmcnt(8)\n\t" \
    "v_mfma_f32_16x16x32_f16 a[240:243], v[80:83], v[148:151], a[240:243]\n\t" \
    "v_cvt_pk_f16_f32 v231, v222, v226\n\t" \
    "v_mfma_f32_16x16x32_f16 a[208:211], v[76:79], v[148:151], a[208:211]\n\t" \
    "ds_write_b64 %[vWr1], v[230:231] offset:2056\n\t" \
    "v_mfma_f32_16x16x32_f16 a[176:179], v[72:75], v[148:151], a[176:179]\n\t" \
    "v_cvt_pk_f16_f32 v228, v199, v203\n\t" \
    "v_cvt_pk_f16_f32 v229, v207, v211\n\t" \
    "v_mfma_f32_16x16x32_f16 a[144:147], v[68:71], v[148:151], a[144:147]\n\t" \
    "buffer_load_dwordx4 v[124:127], %[vA0], %[rWS], s55 offen offset:2048 sc0 sc1\n\t" \
    "v_mfma_f32_16x16x32_f16 a[112:115], v[64:67], v[148:151], a[112:115]\n\t" \
    "ds_write_b64 %[vWr1], v[228:229] offset:3072\n\t" \
    "v_mfma_f32_16x16x32_f16 a[80:83], v[60:63], v[148:151], a[80:83]\n\t" \
    "v_cvt_pk_f16_f32 v230, v215, v219\n\t" \
    "v_cvt_pk_f16_f32 v231, v223, v227\n\t" \
    "v_mfma_f32_16x16x32_f16 a[48:51], v[56:59], v[148:151], a[48:51]\n\t" \
    "ds_write_b64 %[vWr1], v[230:231] offset:3080\n\t" \
    "v_mfma_f32_16x16x32_f16 a[16:19], v[52:55], v[148:151], a[16:19]\n\t" \
    "ds_read_b128 v[148:151], %[vA0] offset:49152\n\t" \
    "s_waitcnt lgkmcnt(10)\n\t" \
    "v_mfma_f32_16x16x32_f16 a[20:23], v[52:55], v[152:155], a[20:23]\n\t" \
    "buffer_load_dwordx4 v[128:131], %[vA0], %[rWS], s55 offen offset:3072 sc0 sc1\n\t" \
    "v_mfma_f32_16x16x32_f16 a[52:55], v[56:59], v[152:155], a[52:55]\n\t" \
    "s_add_u32 s55, s37, 28672\n\t" \
    "v_mfma_f32_16x16x32_f16 a[84:87], v[60:63], v[152:155], a[84:87]\n\t" \
    "buffer_load_dwordx4 v[132:135], %[vA0], %[rWS], s55 offen offset:0 sc0 sc1\n\t" \
    "v_mfma_f32_16x16x32_f16 a[116:119], v[64:67], v[152:155], a[116:119]\n\t" \
    "buffer_load_dwordx4 v[136:139], %[vA0], %[rWS], s55 offen offset:1024 sc0 sc1\n\t" \
    "v_mfma_f32_16x16x32_f16 a[148:151], v[68:71], v[152:155], a[148:151]\n\t" \
    "v_mfma_f32_16x16x32_f16 a[180:183], v[72:75], v[152:155], a[180:183]\n\t" \
    "v_mfma_f32_16x16x32_f16 a[212:215], v[76:79], v[152:155], a[212:215]\n\t" \
    "v_mfma_f32_16x16x32_f16 a[244:247], v[80:83], v[152:155], a[244:247]\n\t" \
    "ds_read_b128 v[152:155], %[vA0] offset:50176\n\t" \
    "s_waitcnt lgkmcnt(8)\n\t" \
    "v_mfma_f32_16x16x32_f16 a[248:251], v[80:83], v[156:159], a[248:251]\n\t" \
    "buffer_load_dwordx4 v[140:143], %[vA0], %[rWS], s55 offen offset:2048 sc0 sc1\n\t" \
    "v_mfma_f32_16x16x32_f16 a[216:219], v[76:79], v[156:159], a[216:219]\n\t" \
    "v_mfma_f32_16x16x32_f16 a[184:187], v[72:75], v[156:159], a[184:187]\n\t" \
    "v_mfma_f32_16x16x32_f16 a[152:155], v[68:71], v[156:159], a[152:155]\n\t" \
    "v_mfma_f32_16x16x32_f16 a[120:123], v[64:67], v[156:159], a[120:123]\n\t" \
    "v_mfma_f32_16x16x32_f16 a[88:91], v[60:63], v[156:159], a[88:91]\n\t" \
    "buffer_load_dwordx4 v[144:147], %[vA0], %[rWS], s55 offen offset:3072 sc0 sc1\n\t" \
    "v_mfma_f32_16x16x32_f16 a[56:59], v[56:59], v[156:159], a[56:59]\n\t" \
    "v_mfma_f32_16x16x32_f16 a[24:27], v[52:55], v[156:159], a[24:27]\n\t" \
    "ds_read_b128 v[156:159], %[vA0] offset:51200\n\t" \
    "s_waitcnt lgkmcnt(6)\n\t" \
    "v_mfma_f32_16x16x32_f16 a[28:31], v[52:55], v[160:163], a[28:31]\n\t" \
    "s_add_u32 s55, s38, 0\n\t" \
    "v_mfma_f32_16x16x32_f16 a[60:63], v[56:59], v[160:163], a[60:63]\n\t" \
    "v_mfma_f32_16x16x32_f16 a[92:95], v[60:63], v[160:163], a[92:95]\n\t" \
    "buffer_load_dwordx4 v[20:23], %[vA0], %[rWS], s55 offen offset:0 sc0 sc1\n\t" \
    "v_mfma_f32_16x16x32_f16 a[124:127], v[64:67], v[160:163], a[124:127]\n\t" \
    "v_mfma_f32_16x16x32_f16 a[156:159], v[68:71], v[160:163], a[156:159]\n\t" \
    "v_mfma_f32_16x16x32_f16 a[188:191], v[72:75], v[160:163], a[188:191]\n\t" \
    "v_mfma_f32_16x16x32_f16 a[220:223], v[76:79], v[160:163], a[220:223]\n\t" \
    "v_mfma_f32_16x16x32_f16 a[252:255], v[80:83], v[160:163], a[252:255]\n\t" \
    "ds_read_b128 v[160:163], %[vA0] offset:52224\n\t" \
    "s_waitcnt vmcnt(22) lgkmcnt(3)\n\t" \
    "v_mfma_f32_16x16x32_f16 a[224:227], v[112:115], v[148:151], a[224:227]\n\t" \
    "buffer_load_dwordx4 v[24:27], %[vA0], %[rWS], s55 offen offset:1024 sc0 sc1\n\t" \
    "v_mfma_f32_16x16x32_f16 a[192:195], v[108:111], v[148:151], a[192:195]\n\t" \
    "v_mfma_f32_16x16x32_f16 a[160:163], v[104:107], v[148:151], a[160:163]\n\t" \
    "v_mfma_f32_16x16x32_f16 a[128:131], v[100:103], v[148:151], a[128:131]\n\t" \
    "v_mfma_f32_16x16x32_f16 a[96:99], v[96:99], v[148:151], a[96:99]\n\t" \
    "buffer_load_dwordx4 v[28:31], %[vA0], %[rWS], s55 offen offset:2048 sc0 sc1\n\t" \
    "v_mfma_f32_16x16x32_f16 a[64:67], v[92:95], v[148:151], a[64:67]\n\t" \
    "v_mfma_f32_16x16x32_f16 a[32:35], v[88:91], v[148:151], a[32:35]\n\t" \
    "v_mfma_f32_16x16x32_f16 a[0:3], v[84:87], v[148:151], a[0:3]\n\t" \
    "ds_read_b128 v[148:151], %[vA0] offset:53248\n\t" \
    "s_waitcnt lgkmcnt(3)\n\t" \
    "v_mfma_f32_16x16x32_f16 a[4:7], v[84:87], v[152:155], a[4:7]\n\t" \
    "v_mfma_f32_16x16x32_f16 a[36:39], v[88:91], v[152:155], a[36:39]\n\t" \
    "buffer_load_dwordx4 v[32:35], %[vA0], %[rWS], s55 offen offset:3072 sc0 sc1\n\t" \
    "v_mfma_f32_16x16x32_f16 a[68:71], v[92:95], v[152:155], a[68:71]\n\t" \
    "v_mfma_f32_16x16x32_f16 a[100:103], v[96:99], v[152:155], a[100:103]\n\t" \
    "v_mfma_f32_16x16x32_f16 a[132:135], v[100:103], v[152:155], a[132:135]\n\t" \
    "v_mfma_f32_16x16x32_f16 a[164:167], v[104:107], v[152:155], a[164:167]\n\t" \
    "s_add_u32 s55, s38, 4096\n\t" \
    "v_mfma_f32_16x16x32_f16 a[196:199], v[108:111], v[152:155], a[196:199]\n\t" \
    "v_mfma_f32_16x16x32_f16 a[228:231], v[112:115], v[152:155], a[228:231]\n\t" \
    "ds_read_b128 v[152:155], %[vA0] offset:54272\n\t" \
    "s_waitcnt lgkmcnt(3)\n\t" \
    "v_mfma_f32_16x16x32_f16 a[232:235], v[112:115], v[156:159], a[232:235]\n\t" \
    "buffer_load_dwordx4 v[36:39], %[vA0], %[rWS], s55 offen offset:0 sc0 sc1\n\t" \
    "v_mfma_f32_16x16x32_f16 a[200:203], v[108:111], v[156:159], a[200:203]\n\t" \
    "v_mfma_f32_16x16x32_f16 a[168:171], v[104:107], v[156:159], a[168:171]\n\t" \
    "v_mfma_f32_16x16x32_f16 a[136:139], v[100:103], v[156:159], a[136:139]\n\t" \
    "v_mfma_f32_16x16x32_f16 a[104:107], v[96:99], v[156:159], a[104:107]\n\t" \
    "buffer_load_dwordx4 v[40:43], %[vA0], %[rWS], s55 offen offset:1024 sc0 sc1\n\t" \
    "v_mfma_f32_16x16x32_f16 a[72:75], v[92:95], v[156:159], a[72:75]\n\t" \
    "v_readlane_b32 s52, %[vTab0], s51\n\t" \
    "v_mfma_f32_16x16x32_f16 a[40:43], v[88:91], v[156:159], a[40:43]\n\t" \
    "s_add_u32 s42, s51, 32\n\t" \
    "v_mfma_f32_16x16x32_f16 a[8:11], v[84:87], v[156:159], a[8:11]\n\t" \
    "ds_read_b128 v[156:159], %[vA0] offset:55296\n\t" \
    "s_waitcnt lgkmcnt(3)\n\t" \
    "v_mfma_f32_16x16x32_f16 a[12:15], v[84:87], v[160:163], a[12:15]\n\t" \
    "v_readlane_b32 s53, %[vTab1], s51\n\t" \
    "v_mfma_f32_16x16x32_f16 a[44:47], v[88:91], v[160:163], a[44:47]\n\t" \
    "v_readlane_b32 s42, %[vTab0], s42\n\t" \
    "v_mfma_f32_16x16x32_f16 a[76:79], v[92:95], v[160:163], a[76:79]\n\t" \
    "s_add_u32 s51, s51, 1\n\t" \
    "v_mfma_f32_16x16x32_f16 a[108:111], v[96:99], v[160:163], a[108:111]\n\t" \
    "buffer_load_dwordx4 v[44:47], %[vA0], %[rWS], s55 offen offset:2048 sc0 sc1\n\t" \
    "v_mfma_f32_16x16x32_f16 a[140:143], v[100:103], v[160:163], a[140:143]\n\t" \
    "s_add_u32 s40, s66, s43\n\t" \
    "s_add_u32 s40, s40, %[sWk1]\n\t" \
    "v_mfma_f32_16x16x32_f16 a[172:175], v[104:107], v[160:163], a[172:175]\n\t" \
    "s_mov_b32 s56, s42\n\t" \
    "s_and_b32 s57, s53, 0xffff\n\t" \
    "v_mfma_f32_16x16x32_f16 a[204:207], v[108:111], v[160:163], a[204:207]\n\t" \
    "s_and_b32 s58, s53, 0x10000\n\t" \
    "s_bfe_i32 s53, s53, 0x10011\n\t" \
    "v_mfma_f32_16x16x32_f16 a[236:239], v[112:115], v[160:163], a[236:239]\n\t" \
    "ds_read_b128 v[160:163], %[vA0] offset:56320\n\t" \
    "s_waitcnt lgkmcnt(3)\n\t" \
    "v_mfma_f32_16x16x32_f16 a[240:243], v[112:115], v[148:151], a[240:243]\n\t" \
    "buffer_load_dwordx4 v[48:51], %[vA0], %[rWS], s55 offen offset:3072 sc0 sc1\n\t" \
    "v_mfma_f32_16x16x32_f16 a[208:211], v[108:111], v[148:151], a[208:211]\n\t" \
    "v_mfma_f32_16x16x32_f16 a[176:179], v[104:107], v[148:151], a[176:179]\n\t" \
    "v_mfma_f32_16x16x32_f16 a[144:147], v[100:103], v[148:151], a[144:147]\n\t" \
    "buffer_load_dwordx4 v[196:199], %[vW], %[rW], s40 offen nt\n\t" \
    "v_mfma_f32_16x16x32_f16 a[112:115], v[96:99], v[148:151], a[112:115]\n\t" \
    "v_mfma_f32_16x16x32_f16 a[80:83], v[92:95], v[148:151], a[80:83]\n\t" \
    "v_mfma_f32_16x16x32_f16 a[48:51], v[88:91], v[148:151], a[48:51]\n\t" \
    "s_add_u32 s40, s40, 32768\n\t" \
    "v_mfma_f32_16x16x32_f16 a[16:19], v[84:87], v[148:151], a[16:19]\n\t" \
    "ds_read_b128 v[148:151], %[vA0] offset:57344\n\t" \
    "s_waitcnt lgkmcnt(3)\n\t" \
    "v_mfma_f32_16x16x32_f16 a[20:23], v[84:87], v[152:155], a[20:23]\n\t" \
    "s_cmp_eq_u32 s69, 0\n\t" \
    "s_cselect_b32 s42, 1073741824, s52\n\t" \
    "v_mfma_f32_16x16x32_f16 a[52:55], v[88:91], v[152:155], a[52:55]\n\t" \
    "buffer_load_dwordx4 v[200:203], %[vW], %[rW], s40 offen nt\n\t" \
    "v_mfma_f32_16x16x32_f16 a[84:87], v[92:95], v[152:155], a[84:87]\n\t" \
    "buffer_load_dwordx4 v[236:239], %[vA0], %[rWS], s42 offen sc0 sc1\n\t" \
    "v_mfma_f32_16x16x32_f16 a[116:119], v[96:99], v[152:155], a[116:119]\n\t" \
    "s_add_u32 s52, s52, 262144\n\t" \
    "s_cmp_eq_u32 s69, 1\n\t" \
    "v_mfma_f32_16x16x32_f16 a[148:151], v[100:103], v[152:155], a[148:151]\n\t" \
    "s_cselect_b32 s42, 1073741824, s52\n\t" \
    "v_mfma_f32_16x16x32_f16 a[180:183], v[104:107], v[152:155], a[180:183]\n\t" \
    "buffer_load_dwordx4 v[240:243], %[vA0], %[rWS], s42 offen sc0 sc1\n\t" \
    "v_mfma_f32_16x16x32_f16 a[212:215], v[108:111], v[152:155], a[212:215]\n\t" \
    "s_add_u32 s40, s40, 32768\n\t" \
    "s_add_u32 s52, s52, 262144\n\t" \
    "v_mfma_f32_16x16x32_f16 a[244:247], v[112:115], v[152:155], a[244:247]\n\t" \
    "ds_read_b128 v[152:155], %[vA0] offset:58368\n\t" \
    "s_waitcnt lgkmcnt(3)\n\t" \
    "v_mfma_f32_16x16x32_f16 a[248:251], v[112:115], v[156:159], a[248:251]\n\t" \
    "s_cmp_eq_u32 s69, 2\n\t" \
    "s_cselect_b32 s42, 1073741824, s52\n\t" \
    "v_mfma_f32_16x16x32_f16 a[216:219], v[108:111], v[156:159], a[216:219]\n\t" \
    "buffer_load_dwordx4 v[204:207], %[vW], %[rW], s40 offen nt\n\t" \
    "v_mfma_f32_16x16x32_f16 a[184:187], v[104:107], v[156:159], a[184:187]\n\t" \
    "buffer_load_dwordx4 v[244:247], %[vA0], %[rWS], s42 offen sc0 sc1\n\t" \
    "v_mfma_f32_16x16x32_f16 a[152:155], v[100:103], v[156:159], a[152:155]\n\t" \
    "s_add_u32 s52, s52, 262144\n\t" \
    "s_add_u32 s65, s49, 1\n\t" \
    "v_mfma_f32_16x16x32_f16 a[120:123], v[96:99], v[156:159], a[120:123]\n\t" \
    "v_mov_b32 v19, s65\n\t" \
    "s_cmp_eq_u32 s69, 3\n\t" \
    "v_mfma_f32_16x16x32_f16 a[88:91], v[92:95], v[156:159], a[88:91]\n\t" \
    "s_cselect_b32 s42, 1073741824, s52\n\t" \
    "s_add_u32 s40, s40, 32768\n\t" \
    "v_mfma_f32_16x16x32_f16 a[56:59], v[88:91], v[156:159], a[56:59]\n\t" \
    "buffer_load_dwordx4 v[248:251], %[vA0], %[rWS], s42 offen sc0 sc1\n\t" \
    "v_mfma_f32_16x16x32_f16 a[24:27], v[84:87], v[156:159], a[24:27]\n\t" \
    "ds_read_b128 v[156:159], %[vA0] offset:59392\n\t" \
    "s_waitcnt lgkmcnt(3)\n\t" \
    "v_mfma_f32_16x16x32_f16 a[28:31], v[84:87], v[160:163], a[28:31]\n\t" \
    "buffer_load_dwordx4 v[208:211], %[vW], %[rW], s40 offen nt\n\t" \
    "v_mfma_f32_16x16x32_f16 a[60:63], v[88:91], v[160:163], a[60:63]\n\t" \
    "s_waitcnt vmcnt(24)\n\t" \
    "buffer_store_dword v19, %[vFs], %[rWS], 0 offen sc0 sc1\n\t" \
    "v_mfma_f32_16x16x32_f16 a[92:95], v[92:95], v[160:163], a[92:95]\n\t" \
    "s_add_u32 s40, s40, 32768\n\t" \
    "v_mfma_f32_16x16x32_f16 a[124:127], v[96:99], v[160:163], a[124:127]\n\t" \
    "buffer_load_dwordx4 v[212:215], %[vW], %[rW], s40 offen nt\n\t" \
    "v_mfma_f32_16x16x32_f16 a[156:159], v[100:103], v[160:163], a[156:159]\n\t" \
    "s_add_u32 s40, s40, 32768\n\t" \
    "v_mfma_f32_16x16x32_f16 a[188:191], v[104:107], v[160:163], a[188:191]\n\t" \
    "buffer_load_dwordx4 v[216:219], %[vW], %[rW], s40 offen nt\n\t" \
    "v_mfma_f32_16x16x32_f16 a[220:223], v[108:111], v[160:163], a[220:223]\n\t" \
    "v_mfma_f32_16x16x32_f16 a[252:255], v[112:115], v[160:163], a[252:255]\n\t" \
    "ds_read_b128 v[160:163], %[vA0] offset:60416\n\t" \
    "s_waitcnt vmcnt(19) lgkmcnt(3)\n\t" \
    "v_mfma_f32_16x16x32_f16 a[224:227], v[144:147], v[148:151], a[224:227]\n\t" \
    "s_add_u32 s40, s40, 32768\n\t" \
    "v_mfma_f32_16x16x32_f16 a[192:195], v[140:143], v[148:151], a[192:195]\n\t" \
    "v_mfma_f32_16x16x32_f16 a[160:163], v[136:139], v[148:151], a[160:163]\n\t" \
    "buffer_load_dwordx4 v[220:223], %[vW], %[rW], s40 offen nt\n\t" \
    "v_mfma_f32_16x16x32_f16 a[128:131], v[132:135], v[148:151], a[128:131]\n\t" \
    "v_mfma_f32_16x16x32_f16 a[96:99], v[128:131], v[148:151], a[96:99]\n\t" \
    "v_mfma_f32_16x16x32_f16 a[64:67], v[124:127], v[148:151], a[64:67]\n\t" \
    "v_mfma_f32_16x16x32_f16 a[32:35], v[120:123], v[148:151], a[32:35]\n\t" \
    "s_add_u32 s40, s40, 32768\n\t" \
    "v_mfma_f32_16x16x32_f16 a[0:3], v[116:119], v[148:151], a[0:3]\n\t" \
    "ds_read_b128 v[148:151], %[vA0] offset:61440\n\t" \
    "s_waitcnt lgkmcnt(3)\n\t" \
    "v_mfma_f32_16x16x32_f16 a[4:7], v[116:119], v[152:155], a[4:7]\n\t" \
    "buffer_load_dwordx4 v[224:227], %[vW], %[rW], s40 offen nt\n\t" \
    "v_mfma_f32_16x16x32_f16 a[36:39], v[120:123], v[152:155], a[36:39]\n\t" \
    "v_mfma_f32_16x16x32_f16 a[68:71], v[124:127], v[152:155], a[68:71]\n\t" \
    "v_mfma_f32_16x16x32_f16 a[100:103], v[128:131], v[152:155], a[100:103]\n\t" \
    "s_add_u32 s55, s38, 8192\n\t" \
    "v_mfma_f32_16x16x32_f16 a[132:135], v[132:135], v[152:155], a[132:135]\n\t" \
    "v_and_b32 v252, s53, v252\n\t" \
    "v_mfma_f32_16x16x32_f16 a[164:167], v[136:139], v[152:155], a[164:167]\n\t" \
    "buffer_load_dwordx4 v[52:55], %[vA0], %[rWS], s55 offen offset:0 sc0 sc1\n\t" \
    "v_mfma_f32_16x16x32_f16 a[196:199], v[140:143], v[152:155], a[196:199]\n\t" \
    "v_and_b32 v253, s53, v253\n\t" \
    "v_and_b32 v254, s53, v254\n\t" \
    "v_mfma_f32_16x16x32_f16 a[228:231], v[144:147], v[152:155], a[228:231]\n\t" \
    "ds_read_b128 v[152:155], %[vA0] offset:62464\n\t" \
    "s_waitcnt lgkmcnt(3)\n\t" \
    "v_mfma_f32_16x16x32_f16 a[232:235], v[144:147], v[156:159], a[232:235]\n\t" \
    "v_and_b32 v255, s53, v255\n\t" \
    "s_waitcnt vmcnt(10)\n\t" \
    "v_add_f32 v236, v236, v240\n\t" \
    "v_mfma_f32_16x16x32_f16 a[200:203], v[140:143], v[156:159], a[200:203]\n\t" \
    "v_add_f32 v237, v237, v241\n\t" \
    "v_mfma_f32_16x16x32_f16 a[168:171], v[136:139], v[156:159], a[168:171]\n\t" \
    "buffer_load_dwordx4 v[56:59], %[vA0], %[rWS], s55 offen offset:1024 sc0 sc1\n\t" \
    "v_mfma_f32_16x16x32_f16 a[136:139], v[132:135], v[156:159], a[136:139]\n\t" \
    "v_add_f32 v238, v238, v242\n\t" \
    "v_add_f32 v239, v239, v243\n\t" \
    "v_mfma_f32_16x16x32_f16 a[104:107], v[128:131], v[156:159], a[104:107]\n\t" \
    "v_add_f32 v252, v252, v236\n\t" \
    "v_mfma_f32_16x16x32_f16 a[72:75], v[124:127], v[156:159], a[72:75]\n\t" \
    "v_add_f32 v253, v253, v237\n\t" \
    "v_mfma_f32_16x16x32_f16 a[40:43], v[120:123], v[156:159], a[40:43]\n\t" \
    "v_add_f32 v254, v254, v238\n\t" \
    "v_mfma_f32_16x16x32_f16 a[8:11], v[116:119], v[156:159], a[8:11]\n\t" \
    "ds_read_b128 v[156:159], %[vA0] offset:63488\n\t" \
    "s_waitcnt lgkmcnt(3)\n\t" \
    "v_mfma_f32_16x16x32_f16 a[12:15], v[116:119], v[160:163], a[12:15]\n\t" \
    "buffer_load_dwordx4 v[60:63], %[vA0], %[rWS], s55 offen offset:2048 sc0 sc1\n\t" \
    "v_mfma_f32_16x16x32_f16 a[44:47], v[120:123], v[160:163], a[44:47]\n\t" \
    "v_add_f32 v255, v255, v239\n\t" \
    "s_waitcnt vmcnt(9)\n\t" \
    "v_add_f32 v244, v244, v248\n\t" \
    "v_mfma_f32_16x16x32_f16 a[76:79], v[124:127], v[160:163], a[76:79]\n\t" \
    "v_add_f32 v245, v245, v249\n\t" \
    "v_add_f32 v246, v246, v250\n\t" \
    "v_mfma_f32_16x16x32_f16 a[108:111], v[128:131], v[160:163], a[108:111]\n\t" \
    "v_add_f32 v247, v247, v251\n\t" \
    "v_mfma_f32_16x16x32_f16 a[140:143], v[132:135], v[160:163], a[140:143]\n\t" \
    "buffer_load_dwordx4 v[64:67], %[vA0], %[rWS], s55 offen offset:3072 sc0 sc1\n\t" \
    "v_mfma_f32_16x16x32_f16 a[172:175], v[136:139], v[160:163], a[172:175]\n\t" \
    "v_add_f32 v252, v252, v244\n\t" \
    "v_add_f32 v253, v253, v245\n\t" \
    "v_mfma_f32_16x16x32_f16 a[204:207], v[140:143], v[160:163], a[204:207]\n\t" \
    "v_add_f32 v254, v254, v246\n\t" \
    "v_mfma_f32_16x16x32_f16 a[236:239], v[144:147], v[160:163], a[236:239]\n\t" \
    "ds_read_b128 v[160:163], %[vA0] offset:64512\n\t" \
    "s_waitcnt lgkmcnt(3)\n\t" \
    "v_mfma_f32_16x16x32_f16 a[240:243], v[144:147], v[148:151], a[240:243]\n\t" \
    "s_add_u32 s55, s38, 12288\n\t" \
    "v_add_f32 v255, v255, v247\n\t" \
    "v_mfma_f32_16x16x32_f16 a[208:211], v[140:143], v[148:151], a[208:211]\n\t" \
    "v_med3_f32 v236, v252, %[m60], %[p60]\n\t" \
    "v_mfma_f32_16x16x32_f16 a[176:179], v[136:139], v[148:151], a[176:179]\n\t" \
    "buffer_load_dwordx4 v[68:71], %[vA0], %[rWS], s55 offen offset:0 sc0 sc1\n\t" \
    "v_mfma_f32_16x16x32_f16 a[144:147], v[132:135], v[148:151], a[144:147]\n\t" \
    "v_med3_f32 v237, v253, %[m60], %[p60]\n\t" \
    "v_med3_f32 v238, v254, %[m60], %[p60]\n\t" \
    "v_mfma_f32_16x16x32_f16 a[112:115], v[128:131], v[148:151], a[112:115]\n\t" \
    "v_med3_f32 v239, v255, %[m60], %[p60]\n\t" \
    "v_mul_f32 v236, 0xbfb8aa3b, v236\n\t" \
    "v_mfma_f32_16x16x32_f16 a[80:83], v[124:127], v[148:151], a[80:83]\n\t" \
    "v_mul_f32 v237, 0xbfb8aa3b, v237\n\t" \
    "v_mfma_f32_16x16x32_f16 a[48:51], v[120:123], v[148:151], a[48:51]\n\t" \
    "v_mul_f32 v238, 0xbfb8aa3b, v238\n\t" \
    "v_mfma_f32_16x16x32_f16 a[16:19], v[116:119], v[148:151], a[16:19]\n\t" \
    "s_waitcnt lgkmcnt(0)\n\t" \
    "s_barrier\n\t" \
    "ds_read_b128 v[148:151], %[vA0] offset:0\n\t" \
    "v_mfma_f32_16x16x32_f16 a[20:23], v[116:119], v[152:155], a[20:23]\n\t" \
    "buffer_load_dwordx4 v[72:75], %[vA0], %[rWS], s55 offen offset:1024 sc0 sc1\n\t" \
    "v_mfma_f32_16x16x32_f16 a[52:55], v[120:123], v[152:155], a[52:55]\n\t" \
    "v_mul_f32 v239, 0xbfb8aa3b, v239\n\t" \
    "v_mfma_f32_16x16x32_f16 a[84:87], v[124:127], v[152:155], a[84:87]\n\t" \
    "v_exp_f32 v236, v236\n\t" \
    "v_mfma_f32_16x16x32_f16 a[116:119], v[128:131], v[152:155], a[116:119]\n\t" \
    "v_exp_f32 v237, v237\n\t" \
    "v_mfma_f32_16x16x32_f16 a[148:151], v[132:135], v[152:155], a[148:151]\n\t" \
    "v_exp_f32 v238, v238\n\t" \
    "v_mfma_f32_16x16x32_f16 a[180:183], v[136:139], v[152:155], a[180:183]\n\t" \
    "v_exp_f32 v239, v239\n\t" \
    "v_mfma_f32_16x16x32_f16 a[212:215], v[140:143], v[152:155], a[212:215]\n\t" \
    "buffer_load_dwordx4 v[76:79], %[vA0], %[rWS], s55 offen offset:2048 sc0 sc1\n\t" \
    "v_mfma_f32_16x16x32_f16 a[244:247], v[144:147], v[152:155], a[244:247]\n\t" \
    "ds_read_b128 v[152:155], %[vA0] offset:1024\n\t" \
    "v_mfma_f32_16x16x32_f16 a[248:251], v[144:147], v[156:159], a[248:251]\n\t" \
    "v_add_f32 v236, 1.0, v236\n\t" \
    "v_add_f32 v237, 1.0, v237\n\t" \
    "v_mfma_f32_16x16x32_f16 a[216:219], v[140:143], v[156:159], a[216:219]\n\t" \
    "v_add_f32 v238, 1.0, v238\n\t" \
    "v_add_f32 v239, 1.0, v239\n\t" \
    "v_mfma_f32_16x16x32_f16 a[184:187], v[136:139], v[156:159], a[184:187]\n\t" \
    "v_rcp_f32 v236, v236\n\t" \
    "v_mfma_f32_16x16x32_f16 a[152:155], v[132:135], v[156:159], a[152:155]\n\t" \
    "buffer_load_dwordx4 v[80:83], %[vA0], %[rWS], s55 offen offset:3072 sc0 sc1\n\t" \
    "v_mfma_f32_16x16x32_f16 a[120:123], v[128:131], v[156:159], a[120:123]\n\t" \
    "v_rcp_f32 v237, v237\n\t" \
    "v_mfma_f32_16x16x32_f16 a[88:91], v[124:127], v[156:159], a[88:91]\n\t" \
    "s_add_u32 s55, s38, 16384\n\t" \
    "v_mfma_f32_16x16x32_f16 a[56:59], v[120:123], v[156:159], a[56:59]\n\t" \
    "v_rcp_f32 v238, v238\n\t" \
    "v_mfma_f32_16x16x32_f16 a[24:27], v[116:119], v[156:159], a[24:27]\n\t" \
    "ds_read_b128 v[156:159], %[vA0] offset:2048\n\t" \
    "v_mfma_f32_16x16x32_f16 a[28:31], v[116:119], v[160:163], a[28:31]\n\t" \
    "v_rcp_f32 v239, v239\n\t" \
    "v_mfma_f32_16x16x32_f16 a[60:63], v[120:123], v[160:163], a[60:63]\n\t" \
    "buffer_load_dwordx4 v[84:87], %[vA0], %[rWS], s55 offen offset:0 sc0 sc1\n\t" \
    "v_mfma_f32_16x16x32_f16 a[92:95], v[124:127], v[160:163], a[92:95]\n\t" \
    "v_mul_f32 v232, v252, v236\n\t" \
    "v_mul_f32 v233, v253, v237\n\t" \
    "v_mfma_f32_16x16x32_f16 a[124:127], v[128:131], v[160:163], a[124:127]\n\t" \
    "v_mul_f32 v234, v254, v238\n\t" \
    "v_mfma_f32_16x16x32_f16 a[156:159], v[132:135], v[160:163], a[156:159]\n\t" \
    "buffer_load_dwordx4 v[88:91], %[vA0], %[rWS], s55 offen offset:1024 sc0 sc1\n\t" \
    "v_mfma_f32_16x16x32_f16 a[188:191], v[136:139], v[160:163], a[188:191]\n\t" \
    "v_mul_f32 v235, v255, v239\n\t" \
    "v_mfma_f32_16x16x32_f16 a[220:223], v[140:143], v[160:163], a[220:223]\n\t" \
    "buffer_load_dwordx4 v[92:95], %[vA0], %[rWS], s55 offen offset:2048 sc0 sc1\n\t" \
    "v_mfma_f32_16x16x32_f16 a[252:255], v[144:147], v[160:163], a[252:255]\n\t" \
    "buffer_load_dwordx4 v[96:99], %[vA0], %[rWS], s55 offen offset:3072 sc0 sc1\n\t" \
    "s_add_u32 s36, s36, 1\n\t" \
    "ds_read_b128 v[160:163], %[vA0] offset:3072\n\t" \
    "s_waitcnt lgkmcnt(3)\n\t" \
    "v_mfma_f32_16x16x32_f16 a[224:227], v[48:51], v[148:151], a[224:227]\n\t" \
    "s_mov_b32 s37, s38\n\t" \
    "v_cvt_pk_f16_f32 v228, v164, v168\n\t" \
    "v_mfma_f32_16x16x32_f16 a[192:195], v[44:47], v[148:151], a[192:195]\n\t" \
    "v_mfma_f32_16x16x32_f16 a[160:163], v[40:43], v[148:151], a[160:163]\n\t" \
    "v_cvt_pk_f16_f32 v229, v172, v176\n\t" \
    "v_mfma_f32_16x16x32_f16 a[128:131], v[36:39], v[148:151], a[128:131]\n\t" \
    "ds_write_b64 %[vWr0], v[228:229] offset:32768\n\t" \
    "v_mfma_f32_16x16x32_f16 a[96:99], v[32:35], v[148:151], a[96:99]\n\t" \
    "v_cvt_pk_f16_f32 v230, v180, v184\n\t" \
    "v_mfma_f32_16x16x32_f16 a[64:67], v[28:31], v[148:151], a[64:67]\n\t" \
    "v_mfma_f32_16x16x32_f16 a[32:35], v[24:27], v[148:151], a[32:35]\n\t" \
    "v_cvt_pk_f16_f32 v231, v188, v192\n\t" \
    "v_mfma_f32_16x16x32_f16 a[0:3], v[20:23], v[148:151], a[0:3]\n\t" \
    "ds_read_b128 v[148:151], %[vA0] offset:4096\n\t" \
    "s_waitcnt lgkmcnt(4)\n\t" \
    "v_mfma_f32_16x16x32_f16 a[4:7], v[20:23], v[152:155], a[4:7]\n\t" \
    "s_cmp_ge_u32 s48, 2\n\ts_cselect_b64 exec, -1, 0\n\ts_cselect_b32 s69, %[own], 9\n\tds_read_b128 v[252:255], %[vStg] offset:20480\n\ts_mov_b64 exec, -1\n\t" \
    "s_waitcnt lgkmcnt(4)\n\t" \
    "v_mfma_f32_16x16x32_f16 a[36:39], v[24:27], v[152:155], a[36:39]\n\t" \
    "ds_write_b64 %[vWr0], v[230:231] offset:32776\n\t" \
    "v_mfma_f32_16x16x32_f16 a[68:71], v[28:31], v[152:155], a[68:71]\n\t" \
    "v_cvt_pk_f16_f32 v228, v165, v169\n\t" \
    "s_add_u32 s55, s37, 20480\n\t" \
    "v_mfma_f32_16x16x32_f16 a[100:103], v[32:35], v[152:155], a[100:103]\n\t" \
    "v_cvt_pk_f16_f32 v229, v173, v177\n\t" \
    "v_mfma_f32_16x16x32_f16 a[132:135], v[36:39], v[152:155], a[132:135]\n\t" \
    "buffer_load_dwordx4 v[100:103], %[vA0], %[rWS], s55 offen offset:0 sc0 sc1\n\t" \
    "v_mfma_f32_16x16x32_f16 a[164:167], v[40:43], v[152:155], a[164:167]\n\t" \
    "ds_write_b64 %[vWr0], v[228:229] offset:33792\n\t" \
    "v_mfma_f32_16x16x32_f16 a[196:199], v[44:47], v[152:155], a[196:199]\n\t" \
    "v_cvt_pk_f16_f32 v230, v181, v185\n\t" \
    "v_cvt_pk_f16_f32 v231, v189, v193\n\t" \
    "v_mfma_f32_16x16x32_f16 a[228:231], v[48:51], v[152:155], a[228:231]\n\t" \
    "ds_read_b128 v[152:155], %[vA0] offset:5120\n\t" \
    "s_waitcnt lgkmcnt(6)\n\t" \
    "v_mfma_f32_16x16x32_f16 a[232:235], v[48:51], v[156:159], a[232:235]\n\t" \
    "ds_write_b64 %[vWr0], v[230:231] offset:33800\n\t" \
    "v_mfma_f32_16x16x32_f16 a[200:203], v[44:47], v[156:159], a[200:203]\n\t" \
    "buffer_load_dwordx4 v[104:107], %[vA0], %[rWS], s55 offen offset:1024 sc0 sc1\n\t" \
    "v_mfma_f32_16x16x32_f16 a[168:171], v[40:43], v[156:159], a[168:171]\n\t" \
    "v_cvt_pk_f16_f32 v228, v166, v170\n\t" \
    "v_cvt_pk_f16_f32 v229, v174, v178\n\t" \
    "v_mfma_f32_16x16x32_f16 a[136:139], v[36:39], v[156:159], a[136:139]\n\t" \
    "ds_write_b64 %[vWr0], v[228:229] offset:34816\n\t" \
    "v_mfma_f32_16x16x32_f16 a[104:107], v[32:35], v[156:159], a[104:107]\n\t" \
    "v_cvt_pk_f16_f32 v230, v182, v186\n\t" \
    "v_mfma_f32_16x16x32_f16 a[72:75], v[28:31], v[156:159], a[72:75]\n\t" \
    "buffer_load_dwordx4 v[108:111], %[vA0], %[rWS], s55 offen offset:2048 sc0 sc1\n\t" \
    "v_mfma_f32_16x16x32_f16 a[40:43], v[24:27], v[156:159], a[40:43]\n\t" \
    "s_add_u32 s66, s66, 0x400000\n\t" \
    "s_and_b32 s66, s66, 0xfffffff\n\t" \
    "v_mfma_f32_16x16x32_f16 a[8:11], v[20:23], v[156:159], a[8:11]\n\t" \
    "ds_read_b128 v[156:159], %[vA0] offset:6144\n\t" \
    "s_waitcnt lgkmcnt(8)\n\t" \
    "v_mfma_f32_16x16x32_f16 a[12:15], v[20:23], v[160:163], a[12:15]\n\t" \
    "v_cvt_pk_f16_f32 v231, v190, v194\n\t" \
    "v_mfma_f32_16x16x32_f16 a[44:47], v[24:27], v[160:163], a[44:47]\n\t" \
    "ds_write_b64 %[vWr0], v[230:231] offset:34824\n\t" \
    "v_mfma_f32_16x16x32_f16 a[76:79], v[28:31], v[160:163], a[76:79]\n\t" \
    "v_cvt_pk_f16_f32 v228, v167, v171\n\t" \
    "s_add_u32 s39, s66, s43\n\t" \
    "v_mfma_f32_16x16x32_f16 a[108:111], v[32:35], v[160:163], a[108:111]\n\t" \
    "s_add_u32 s39, s39, %[sWk0]\n\t" \
    "v_mfma_f32_16x16x32_f16 a[140:143], v[36:39], v[160:163], a[140:143]\n\t" \
    "buffer_load_dwordx4 v[112:115], %[vA0], %[rWS], s55 offen offset:3072 sc0 sc1\n\t" \
    "v_mfma_f32_16x16x32_f16 a[172:175], v[40:43], v[160:163], a[172:175]\n\t" \
    "v_cvt_pk_f16_f32 v229, v175, v179\n\t" \
    "v_mfma_f32_16x16x32_f16 a[204:207], v[44:47], v[160:163], a[204:207]\n\t" \
    "ds_write_b64 %[vWr0], v[228:229] offset:35840\n\t" \
    "v_mfma_f32_16x16x32_f16 a[236:239], v[48:51], v[160:163], a[236:239]\n\t" \
    "ds_read_b128 v[160:163], %[vA0] offset:7168\n\t" \
    "s_waitcnt lgkmcnt(9)\n\t" \
    "v_mfma_f32_16x16x32_f16 a[240:243], v[48:51], v[148:151], a[240:243]\n\t" \
    "v_cvt_pk_f16_f32 v230, v183, v187\n\t" \
    "v_cvt_pk_f16_f32 v231, v191, v195\n\t" \
    "v_mfma_f32_16x16x32_f16 a[208:211], v[44:47], v[148:151], a[208:211]\n\t" \
    "ds_write_b64 %[vWr0], v[230:231] offset:35848\n\t" \
    "v_mfma_f32_16x16x32_f16 a[176:179], v[40:43], v[148:151], a[176:179]\n\t" \
    "buffer_load_dwordx4 v[164:167], %[vW], %[rW], s39 offen\n\t" \
    "v_mfma_f32_16x16x32_f16 a[144:147], v[36:39], v[148:151], a[144:147]\n\t" \
    "s_add_u32 s38, s38, 32768\n\t" \
    "s_and_b32 s38, s38, 0x1fffff\n\t" \
    "v_mfma_f32_16x16x32_f16 a[112:115], v[32:35], v[148:151], a[112:115]\n\t" \
    "s_or_b32 s38, s38, %[sA0]\n\t" \
    "s_add_u32 s39, s39, 32768\n\t" \
    "v_mfma_f32_16x16x32_f16 a[80:83], v[28:31], v[148:151], a[80:83]\n\t" \
    "buffer_load_dwordx4 v[168:171], %[vW], %[rW], s39 offen\n\t" \
    "v_mfma_f32_16x16x32_f16 a[48:51], v[24:27], v[148:151], a[48:51]\n\t" \
    "v_mfma_f32_16x16x32_f16 a[16:19], v[20:23], v[148:151], a[16:19]\n\t" \
    "ds_read_b128 v[148:151], %[vA0] offset:8192\n\t" \
    "s_waitcnt lgkmcnt(8)\n\t" \
    "v_mfma_f32_16x16x32_f16 a[20:23], v[20:23], v[152:155], a[20:23]\n\t" \
    "s_add_u32 s39, s39, 32768\n\t" \
    "v_mfma_f32_16x16x32_f16 a[52:55], v[24:27], v[152:155], a[52:55]\n\t" \
    "buffer_load_dwordx4 v[172:175], %[vW], %[rW], s39 offen\n\t" \
    "v_mfma_f32_16x16x32_f16 a[84:87], v[28:31], v[152:155], a[84:87]\n\t" \
    "v_mfma_f32_16x16x32_f16 a[116:119], v[32:35], v[152:155], a[116:119]\n\t" \
    "s_add_u32 s39, s39, 32768\n\t" \
    "v_mfma_f32_16x16x32_f16 a[148:151], v[36:39], v[152:155], a[148:151]\n\t" \
    "v_mfma_f32_16x16x32_f16 a[180:183], v[40:43], v[152:155], a[180:183]\n\t" \
    "buffer_load_dwordx4 v[176:179], %[vW], %[rW], s39 offen\n\t" \
    "v_mfma_f32_16x16x32_f16 a[212:215], v[44:47], v[152:155], a[212:215]\n\t" \
    "v_mfma_f32_16x16x32_f16 a[244:247], v[48:51], v[152:155], a[244:247]\n\t" \
    "ds_read_b128 v[152:155], %[vA0] offset:9216\n\t" \
    "s_waitcnt lgkmcnt(6)\n\t" \
    "v_mfma_f32_16x16x32_f16 a[248:251], v[48:51], v[156:159], a[248:251]\n\t" \
    "s_add_u32 s39, s39, 32768\n\t" \
    "v_mfma_f32_16x16x32_f16 a[216:219], v[44:47], v[156:159], a[216:219]\n\t" \
    "buffer_store_dwordx4 v[232:235], %[vO], s[56:59], 0 offen sc0 sc1 nt\n\t" \
    "v_mfma_f32_16x16x32_f16 a[184:187], v[40:43], v[156:159], a[184:187]\n\t" \
    "buffer_load_dwordx4 v[180:183], %[vW], %[rW], s39 offen\n\t" \
    "v_mfma_f32_16x16x32_f16 a[152:155], v[36:39], v[156:159], a[152:155]\n\t" \
    "s_add_u32 s39, s39, 32768\n\t" \
    "v_mfma_f32_16x16x32_f16 a[120:123], v[32:35], v[156:159], a[120:123]\n\t" \
    "v_mfma_f32_16x16x32_f16 a[88:91], v[28:31], v[156:159], a[88:91]\n\t" \
    "buffer_load_dwordx4 v[184:187], %[vW], %[rW], s39 offen\n\t" \
    "v_mfma_f32_16x16x32_f16 a[56:59], v[24:27], v[156:159], a[56:59]\n\t" \
    "v_mfma_f32_16x16x32_f16 a[24:27], v[20:23], v[156:159], a[24:27]\n\t" \
    "ds_read_b128 v[156:159], %[vA0] offset:10240\n\t" \
    "s_waitcnt lgkmcnt(4)\n\t" \
    "v_mfma_f32_16x16x32_f16 a[28:31], v[20:23], v[160:163], a[28:31]\n\t" \
    "s_add_u32 s39, s39, 32768\n\t" \
    "v_mfma_f32_16x16x32_f16 a[60:63], v[24:27], v[160:163], a[60:63]\n\t" \
    "buffer_load_dwordx4 v[188:191], %[vW], %[rW], s39 offen\n\t" \
    "v_mfma_f32_16x16x32_f16 a[92:95], v[28:31], v[160:163], a[92:95]\n\t" \
    "v_mfma_f32_16x16x32_f16 a[124:127], v[32:35], v[160:163], a[124:127]\n\t" \
    "s_add_u32 s39, s39, 32768\n\t" \
    "v_mfma_f32_16x16x32_f16 a[156:159], v[36:39], v[160:163], a[156:159]\n\t" \
    "v_mfma_f32_16x16x32_f16 a[188:191], v[40:43], v[160:163], a[188:191]\n\t" \
    "buffer_load_dwordx4 v[192:195], %[vW], %[rW], s39 offen\n\t" \
    "v_mfma_f32_16x16x32_f16 a[220:223], v[44:47], v[160:163], a[220:223]\n\t" \
    "v_mfma_f32_16x16x32_f16 a[252:255], v[48:51], v[160:163], a[252:255]\n\t" \
    "ds_read_b128 v[160:163], %[vA0] offset:11264\n\t" \
    "s_waitcnt vmcnt(17) lgkmcnt(3)\n\t" \
    "v_mfma_f32_16x16x32_f16 a[224:227], v[80:83], v[148:151], a[224:227]\n\t" \
    "v_cvt_pk_f16_f32 v228, v196, v200\n\t" \
    "v_mfma_f32_16x16x32_f16 a[192:195], v[76:79], v[148:151], a[192:195]\n\t" \
    "v_mfma_f32_16x16x32_f16 a[160:163], v[72:75], v[148:151], a[160:163]\n\t" \
    "v_cvt_pk_f16_f32 v229, v204, v208\n\t" \
    "v_mfma_f32_16x16x32_f16 a[128:131], v[68:71], v[148:151], a[128:131]\n\t" \
    "ds_write_b64 %[vWr1], v[228:229] offset:32768\n\t" \
    "v_mfma_f32_16x16x32_f16 a[96:99], v[64:67], v[148:151], a[96:99]\n\t" \
    "v_cvt_pk_f16_f32 v230, v212, v216\n\t" \
    "v_mfma_f32_16x16x32_f16 a[64:67], v[60:63], v[148:151], a[64:67]\n\t" \
    "v_mfma_f32_16x16x32_f16 a[32:35], v[56:59], v[148:151], a[32:35]\n\t" \
    "v_cvt_pk_f16_f32 v231, v220, v224\n\t" \
    "v_mfma_f32_16x16x32_f16 a[0:3], v[52:55], v[148:151], a[0:3]\n\t" \
    "ds_read_b128 v[148:151], %[vA0] offset:12288\n\t" \
    "s_waitcnt lgkmcnt(4)\n\t" \
    "v_mfma_f32_16x16x32_f16 a[4:7], v[52:55], v[152:155], a[4:7]\n\t" \
    "ds_write_b64 %[vWr1], v[230:231] offset:32776\n\t" \
    "v_mfma_f32_16x16x32_f16 a[36:39], v[56:59], v[152:155], a[36:39]\n\t" \
    "v_cvt_pk_f16_f32 v228, v197, v201\n\t" \
    "v_mfma_f32_16x16x32_f16 a[68:71], v[60:63], v[152:155], a[68:71]\n\t" \
    "v_cvt_pk_f16_f32 v229, v205, v209\n\t" \
    "v_mfma_f32_16x16x32_f16 a[100:103], v[64:67], v[152:155], a[100:103]\n\t" \
    "ds_write_b64 %[vWr1], v[228:229] offset:33792\n\t" \
    "v_mfma_f32_16x16x32_f16 a[132:135], v[68:71], v[152:155], a[132:135]\n\t" \
    "v_cvt_pk_f16_f32 v230, v213, v217\n\t" \
    "v_mfma_f32_16x16x32_f16 a[164:167], v[72:75], v[152:155], a[164:167]\n\t" \
    "v_mfma_f32_16x16x32_f16 a[196:199], v[76:79], v[152:155], a[196:199]\n\t" \
    "v_cvt_pk_f16_f32 v231, v221, v225\n\t" \
    "v_mfma_f32_16x16x32_f16 a[228:231], v[80:83], v[152:155], a[228:231]\n\t" \
    "ds_read_b128 v[152:155], %[vA0] offset:13312\n\t" \
    "s_waitcnt lgkmcnt(6)\n\t" \
    "v_mfma_f32_16x16x32_f16 a[232:235], v[80:83], v[156:159], a[232:235]\n\t" \
    "s_add_u32 s55, s37, 24576\n\t" \
    "v_mfma_f32_16x16x32_f16 a[200:203], v[76:79], v[156:159], a[200:203]\n\t" \
    "ds_write_b64 %[vWr1], v[230:231] offset:33800\n\t" \
    "v_mfma_f32_16x16x32_f16 a[168:171], v[72:75], v[156:159], a[168:171]\n\t" \
    "v_cvt_pk_f16_f32 v228, v198, v202\n\t" \
    "v_mfma_f32_16x16x32_f16 a[136:139], v[68:71], v[156:159], a[136:139]\n\t" \
    "buffer_load_dwordx4 v[116:119], %[vA0], %[rWS], s55 offen offset:0 sc0 sc1\n\t" \
    "v_mfma_f32_16x16x32_f16 a[104:107], v[64:67], v[156:159], a[104:107]\n\t" \
    "v_cvt_pk_f16_f32 v229, v206, v210\n\t" \
    "v_mfma_f32_16x16x32_f16 a[72:75], v[60:63], v[156:159], a[72:75]\n\t" \
    "ds_write_b64 %[vWr1], v[228:229] offset:34816\n\t" \
    "v_mfma_f32_16x16x32_f16 a[40:43], v[56:59], v[156:159], a[40:43]\n\t" \
    "v_cvt_pk_f16_f32 v230, v214, v218\n\t" \
    "v_mfma_f32_16x16x32_f16 a[8:11], v[52:55], v[156:159], a[8:11]\n\t" \
    "ds_read_b128 v[156:159], %[vA0] offset:14336\n\t" \
    "s_waitcnt lgkmcnt(8)\n\t" \
    "v_mfma_f32_16x16x32_f16 a[12:15], v[52:55], v[160:163], a[12:15]\n\t" \
    "buffer_load_dwordx4 v[120:123], %[vA0], %[rWS], s55 offen offset:1024 sc0 sc1\n\t" \
    "v_mfma_f32_16x16x32_f16 a[44:47], v[56:59], v[160:163], a[44:47]\n\t" \
    "v_cvt_pk_f16_f32 v231, v222, v226\n\t" \
    "v_mfma_f32_16x16x32_f16 a[76:79], v[60:63], v[160:163], a[76:79]\n\t" \
    "ds_write_b64 %[vWr1], v[230:231] offset:34824\n\t" \
    "v_mfma_f32_16x16x32_f16 a[108:111], v[64:67], v[160:163], a[108:111]\n\t" \
    "v_cvt_pk_f16_f32 v228, v199, v203\n\t" \
    "v_cvt_pk_f16_f32 v229, v207, v211\n\t" \
    "v_mfma_f32_16x16x32_f16 a[140:143], v[68:71], v[160:163], a[140:143]\n\t" \
    "buffer_load_dwordx4 v[124:127], %[vA0], %[rWS], s55 offen offset:2048 sc0 sc1\n\t" \
    "v_mfma_f32_16x16x32_f16 a[172:175], v[72:75], v[160:163], a[172:175]\n\t" \
    "ds_write_b64 %[vWr1], v[228:229] offset:35840\n\t" \
    "v_mfma_f32_16x16x32_f16 a[204:207], v[76:79], v[160:163], a[204:207]\n\t" \
    "v_cvt_pk_f16_f32 v230, v215, v219\n\t" \
    "v_cvt_pk_f16_f32 v231, v223, v227\n\t" \
    "v_mfma_f32_16x16x32_f16 a[236:239], v[80:83], v[160:163], a[236:239]\n\t" \
    "ds_read_b128 v[160:163], %[vA0] offset:15360\n\t" \
    "s_waitcnt lgkmcnt(9)\n\t" \
    "v_mfma_f32_16x16x32_f16 a[240:243], v[80:83], v[148:151], a[240:243]\n\t" \
    "ds_write_b64 %[vWr1], v[230:231] offset:35848\n\t" \
    "v_mfma_f32_16x16x32_f16 a[208:211], v[76:79], v[148:151], a[208:211]\n\t" \
    "buffer_load_dwordx4 v[128:131], %[vA0], %[rWS], s55 offen offset:3072 sc0 sc1\n\t" \
    "v_mfma_f32_16x16x32_f16 a[176:179], v[72:75], v[148:151], a[176:179]\n\t" \
    "v_mfma_f32_16x16x32_f16 a[144:147], v[68:71], v[148:151], a[144:147]\n\t" \
    "s_add_u32 s55, s37, 28672\n\t" \
    "v_mfma_f32_16x16x32_f16 a[112:115], v[64:67], v[148:151], a[112:115]\n\t" \
    "v_mfma_f32_16x16x32_f16 a[80:83], v[60:63], v[148:151], a[80:83]\n\t" \
    "buffer_load_dwordx4 v[132:135], %[vA0], %[rWS], s55 offen offset:0 sc0 sc1\n\t" \
    "v_mfma_f32_16x16x32_f16 a[48:51], v[56:59], v[148:151], a[48:51]\n\t" \
    "v_mfma_f32_16x16x32_f16 a[16:19], v[52:55], v[148:151], a[16:19]\n\t" \
    "ds_read_b128 v[148:151], %[vA0] offset:16384\n\t" \
    "s_waitcnt lgkmcnt(8)\n\t" \
    "v_mfma_f32_16x16x32_f16 a[20:23], v[52:55], v[152:155], a[20:23]\n\t" \
    "v_mfma_f32_16x16x32_f16 a[52:55], v[56:59], v[152:155], a[52:55]\n\t" \
    "v_mfma_f32_16x16x32_f16 a[84:87], v[60:63], v[152:155], a[84:87]\n\t" \
    "buffer_load_dwordx4 v[136:139], %[vA0], %[rWS], s55 offen offset:1024 sc0 sc1\n\t" \
    "v_mfma_f32_16x16x32_f16 a[116:119], v[64:67], v[152:155], a[116:119]\n\t" \
    "v_mfma_f32_16x16x32_f16 a[148:151], v[68:71], v[152:155], a[148:151]\n\t" \
    "v_mfma_f32_16x16x32_f16 a[180:183], v[72:75], v[152:155], a[180:183]\n\t" \
    "v_mfma_f32_16x16x32_f16 a[212:215], v[76:79], v[152:155], a[212:215]\n\t" \
    "v_mfma_f32_16x16x32_f16 a[244:247], v[80:83], v[152:155], a[244:247]\n\t" \
    "ds_read_b128 v[152:155], %[vA0] offset:17408\n\t" \
    "s_waitcnt lgkmcnt(6)\n\t" \
    "v_mfma_f32_16x16x32_f16 a[248:251], v[80:83], v[156:159], a[248:251]\n\t" \
    "buffer_load_dwordx4 v[140:143], %[vA0], %[rWS], s55 offen offset:2048 sc0 sc1\n\t" \
    "v_mfma_f32_16x16x32_f16 a[216:219], v[76:79], v[156:159], a[216:219]\n\t" \
    "v_mfma_f32_16x16x32_f16 a[184:187], v[72:75], v[156:159], a[184:187]\n\t" \
    "v_mfma_f32_16x16x32_f16 a[152:155], v[68:71], v[156:159], a[152:155]\n\t" \
    "v_mfma_f32_16x16x32_f16 a[120:123], v[64:67], v[156:159], a[120:123]\n\t" \
    "v_mfma_f32_16x16x32_f16 a[88:91], v[60:63], v[156:159], a[88:91]\n\t" \
    "buffer_load_dwordx4 v[144:147], %[vA0], %[rWS], s55 offen offset:3072 sc0 sc1\n\t" \
    "v_mfma_f32_16x16x32_f16 a[56:59], v[56:59], v[156:159], a[56:59]\n\t" \
    "v_mfma_f32_16x16x32_f16 a[24:27], v[52:55], v[156:159], a[24:27]\n\t" \
    "ds_read_b128 v[156:159], %[vA0] offset:18432\n\t" \
    "s_waitcnt lgkmcnt(4)\n\t" \
    "v_mfma_f32_16x16x32_f16 a[28:31], v[52:55], v[160:163], a[28:31]\n\t" \
    "s_add_u32 s55, s38, 0\n\t" \
    "v_mfma_f32_16x16x32_f16 a[60:63], v[56:59], v[160:163], a[60:63]\n\t" \
    "v_mfma_f32_16x16x32_f16 a[92:95], v[60:63], v[160:163], a[92:95]\n\t" \
    "buffer_load_dwordx4 v[20:23], %[vA0], %[rWS], s55 offen offset:0 sc0 sc1\n\t" \
    "v_mfma_f32_16x16x32_f16 a[124:127], v[64:67], v[160:163], a[124:127]\n\t" \
    "v_mfma_f32_16x16x32_f16 a[156:159], v[68:71], v[160:163], a[156:159]\n\t" \
    "v_mfma_f32_16x16x32_f16 a[188:191], v[72:75], v[160:163], a[188:191]\n\t" \
    "v_mfma_f32_16x16x32_f16 a[220:223], v[76:79], v[160:163], a[220:223]\n\t" \
    "v_mfma_f32_16x16x32_f16 a[252:255], v[80:83], v[160:163], a[252:255]\n\t" \
    "ds_read_b128 v[160:163], %[vA0] offset:19456\n\t" \
    "s_waitcnt vmcnt(18) lgkmcnt(3)\n\t" \
    "v_mfma_f32_16x16x32_f16 a[224:227], v[112:115], v[148:151], a[224:227]\n\t" \
    "buffer_load_dwordx4 v[24:27], %[vA0], %[rWS], s55 offen offset:1024 sc0 sc1\n\t" \
    "v_mfma_f32_16x16x32_f16 a[192:195], v[108:111], v[148:151], a[192:195]\n\t" \
    "v_mfma_f32_16x16x32_f16 a[160:163], v[104:107], v[148:151], a[160:163]\n\t" \
    "v_mfma_f32_16x16x32_f16 a[128:131], v[100:103], v[148:151], a[128:131]\n\t" \
    "v_mfma_f32_16x16x32_f16 a[96:99], v[96:99], v[148:151], a[96:99]\n\t" \
    "buffer_load_dwordx4 v[28:31], %[vA0], %[rWS], s55 offen offset:2048 sc0 sc1\n\t" \
    "v_mfma_f32_16x16x32_f16 a[64:67], v[92:95], v[148:151], a[64:67]\n\t" \
    "s_add_u32 s49, s49, 1\n\t" \
    "v_mfma_f32_16x16x32_f16 a[32:35], v[88:91], v[148:151], a[32:35]\n\t" \
    "buffer_load_dword v19, %[vFl], %[rWS], 0 offen sc0 sc1\n\t" \
    "v_mfma_f32_16x16x32_f16 a[0:3], v[84:87], v[148:151], a[0:3]\n\t" \
    "ds_read_b128 v[148:151], %[vA0] offset:20480\n\t" \
    "s_waitcnt lgkmcnt(3)\n\t" \
    "v_mfma_f32_16x16x32_f16 a[4:7], v[84:87], v[152:155], a[4:7]\n\t" \
    "v_mfma_f32_16x16x32_f16 a[36:39], v[88:91], v[152:155], a[36:39]\n\t" \
    "buffer_load_dwordx4 v[32:35], %[vA0], %[rWS], s55 offen offset:3072 sc0 sc1\n\t" \
    "v_mfma_f32_16x16x32_f16 a[68:71], v[92:95], v[152:155], a[68:71]\n\t" \
    "v_mfma_f32_16x16x32_f16 a[100:103], v[96:99], v[152:155], a[100:103]\n\t" \
    "v_mfma_f32_16x16x32_f16 a[132:135], v[100:103], v[152:155], a[132:135]\n\t" \
    "v_mfma_f32_16x16x32_f16 a[164:167], v[104:107], v[152:155], a[164:167]\n\t" \
    "s_add_u32 s55, s38, 4096\n\t" \
    "v_mfma_f32_16x16x32_f16 a[196:199], v[108:111], v[152:155], a[196:199]\n\t" \
    "v_mfma_f32_16x16x32_f16 a[228:231], v[112:115], v[152:155], a[228:231]\n\t" \
    "ds_read_b128 v[152:155], %[vA0] offset:21504\n\t" \
    "s_waitcnt lgkmcnt(3)\n\t" \
    "v_mfma_f32_16x16x32_f16 a[232:235], v[112:115], v[156:159], a[232:235]\n\t" \
    "buffer_load_dwordx4 v[36:39], %[vA0], %[rWS], s55 offen offset:0 sc0 sc1\n\t" \
    "v_mfma_f32_16x16x32_f16 a[200:203], v[108:111], v[156:159], a[200:203]\n\t" \
    "v_mfma_f32_16x16x32_f16 a[168:171], v[104:107], v[156:159], a[168:171]\n\t" \
    "v_mfma_f32_16x16x32_f16 a[136:139], v[100:103], v[156:159], a[136:139]\n\t" \
    "v_mfma_f32_16x16x32_f16 a[104:107], v[96:99], v[156:159], a[104:107]\n\t" \
    "buffer_load_dwordx4 v[40:43], %[vA0], %[rWS], s55 offen offset:1024 sc0 sc1\n\t" \
    "v_mfma_f32_16x16x32_f16 a[72:75], v[92:95], v[156:159], a[72:75]\n\t" \
    "v_readlane_b32 s52, %[vTab0], s51\n\t" \
    "v_mfma_f32_16x16x32_f16 a[40:43], v[88:91], v[156:159], a[40:43]\n\t" \
    "s_add_u32 s42, s51, 32\n\t" \
    "v_mfma_f32_16x16x32_f16 a[8:11], v[84:87], v[156:159], a[8:11]\n\t" \
    "ds_read_b128 v[156:159], %[vA0] offset:22528\n\t" \
    "s_waitcnt lgkmcnt(3)\n\t" \
    "v_mfma_f32_16x16x32_f16 a[12:15], v[84:87], v[160:163], a[12:15]\n\t" \
    "v_readlane_b32 s53, %[vTab1], s51\n\t" \
    "v_mfma_f32_16x16x32_f16 a[44:47], v[88:91], v[160:163], a[44:47]\n\t" \
    "v_readlane_b32 s42, %[vTab0], s42\n\t" \
    "v_mfma_f32_16x16x32_f16 a[76:79], v[92:95], v[160:163], a[76:79]\n\t" \
    "s_add_u32 s51, s51, 1\n\t" \
    "v_mfma_f32_16x16x32_f16 a[108:111], v[96:99], v[160:163], a[108:111]\n\t" \
    "buffer_load_dwordx4 v[44:47], %[vA0], %[rWS], s55 offen offset:2048 sc0 sc1\n\t" \
    "v_mfma_f32_16x16x32_f16 a[140:143], v[100:103], v[160:163], a[140:143]\n\t" \
    "s_add_u32 s40, s66, s43\n\t" \
    "s_add_u32 s40, s40, %[sWk1]\n\t" \
    "v_mfma_f32_16x16x32_f16 a[172:175], v[104:107], v[160:163], a[172:175]\n\t" \
    "s_mov_b32 s56, s42\n\t" \
    "s_and_b32 s57, s53, 0xffff\n\t" \
    "v_mfma_f32_16x16x32_f16 a[204:207], v[108:111], v[160:163], a[204:207]\n\t" \
    "s_and_b32 s58, s53, 0x10000\n\t" \
    "s_bfe_i32 s53, s53, 0x10011\n\t" \
    "v_mfma_f32_16x16x32_f16 a[236:239], v[112:115], v[160:163], a[236:239]\n\t" \
    "ds_read_b128 v[160:163], %[vA0] offset:23552\n\t" \
    "s_waitcnt lgkmcnt(3)\n\t" \
    "v_mfma_f32_16x16x32_f16 a[240:243], v[112:115], v[148:151], a[240:243]\n\t" \
    "buffer_load_dwordx4 v[48:51], %[vA0], %[rWS], s55 offen offset:3072 sc0 sc1\n\t" \
    "v_mfma_f32_16x16x32_f16 a[208:211], v[108:111], v[148:151], a[208:211]\n\t" \
    "s_waitcnt vmcnt(5)\n\t" \
    "v_cmp_le_u32 s[76:77], s49, v19\n\t" \
    "v_mfma_f32_16x16x32_f16 a[176:179], v[104:107], v[148:151], a[176:179]\n\t" \
    "s_nop 1\n\t" \
    "s_or_b64 s[76:77], s[76:77], s[70:71]\n\t" \
    "v_mfma_f32_16x16x32_f16 a[144:147], v[100:103], v[148:151], a[144:147]\n\t" \
    "s_cmp_eq_u64 s[76:77], exec\n\t" \
    "s_cbranch_scc0 SLOW_P4%=\n\t" \
    "BACK_P4%=:\n\t" \
    "v_mfma_f32_16x16x32_f16 a[112:115], v[96:99], v[148:151], a[112:115]\n\t" \
    "buffer_load_dwordx4 v[196:199], %[vW], %[rW], s40 offen nt\n\t" \
    "v_mfma_f32_16x16x32_f16 a[80:83], v[92:95], v[148:151], a[80:83]\n\t" \
    "v_mfma_f32_16x16x32_f16 a[48:51], v[88:91], v[148:151], a[48:51]\n\t" \
    "s_add_u32 s40, s40, 32768\n\t" \
    "v_mfma_f32_16x16x32_f16 a[16:19], v[84:87], v[148:151], a[16:19]\n\t" \
    "ds_read_b128 v[148:151], %[vA0] offset:24576\n\t" \
    "s_waitcnt lgkmcnt(3)\n\t" \
    "v_mfma_f32_16x16x32_f16 a[20:23], v[84:87], v[152:155], a[20:23]\n\t" \
    "s_cmp_eq_u32 s69, 0\n\t" \
    "s_cselect_b32 s42, 1073741824, s52\n\t" \
    "v_mfma_f32_16x16x32_f16 a[52:55], v[88:91], v[152:155], a[52:55]\n\t" \
    "buffer_load_dwordx4 v[200:203], %[vW], %[rW], s40 offen nt\n\t" \
    "v_mfma_f32_16x16x32_f16 a[84:87], v[92:95], v[152:155], a[84:87]\n\t" \
    "buffer_load_dwordx4 v[236:239], %[vA0], %[rWS], s42 offen sc0 sc1\n\t" \
    "v_mfma_f32_16x16x32_f16 a[116:119], v[96:99], v[152:155], a[116:119]\n\t" \
    "s_add_u32 s52, s52, 262144\n\t" \
    "s_cmp_eq_u32 s69, 1\n\t" \
    "v_mfma_f32_16x16x32_f16 a[148:151], v[100:103], v[152:155], a[148:151]\n\t" \
    "s_cselect_b32 s42, 1073741824, s52\n\t" \
    "v_mfma_f32_16x16x32_f16 a[180:183], v[104:107], v[152:155], a[180:183]\n\t" \
    "buffer_load_dwordx4 v[240:243], %[vA0], %[rWS], s42 offen sc0 sc1\n\t" \
    "v_mfma_f32_16x16x32_f16 a[212:215], v[108:111], v[152:155], a[212:215]\n\t" \
    "s_add_u32 s40, s40, 32768\n\t" \
    "s_add_u32 s52, s52, 262144\n\t" \
    "v_mfma_f32_16x16x32_f16 a[244:247], v[112:115], v[152:155], a[244:247]\n\t" \
    "ds_read_b128 v[152:155], %[vA0] offset:25600\n\t" \
    "s_waitcnt lgkmcnt(3)\n\t" \
    "v_mfma_f32_16x16x32_f16 a[248:251], v[112:115], v[156:159], a[248:251]\n\t" \
    "s_cmp_eq_u32 s69, 2\n\t" \
    "s_cselect_b32 s42, 1073741824, s52\n\t" \
    "v_mfma_f32_16x16x32_f16 a[216:219], v[108:111], v[156:159], a[216:219]\n\t" \
    "buffer_load_dwordx4 v[204:207], %[vW], %[rW], s40 offen nt\n\t" \
    "v_mfma_f32_16x16x32_f16 a[184:187], v[104:107], v[156:159], a[184:187]\n\t" \
    "buffer_load_dwordx4 v[244:247], %[vA0], %[rWS], s42 offen sc0 sc1\n\t" \
    "v_mfma_f32_16x16x32_f16 a[152:155], v[100:103], v[156:159], a[152:155]\n\t" \
    "s_add_u32 s52, s52, 262144\n\t" \
    "s_cmp_eq_u32 s69, 3\n\t" \
    "v_mfma_f32_16x16x32_f16 a[120:123], v[96:99], v[156:159], a[120:123]\n\t" \
    "s_cselect_b32 s42, 1073741824, s52\n\t" \
    "s_add_u32 s40, s40, 32768\n\t" \
    "v_mfma_f32_16x16x32_f16 a[88:91], v[92:95], v[156:159], a[88:91]\n\t" \
    "buffer_load_dwordx4 v[248:251], %[vA0], %[rWS], s42 offen sc0 sc1\n\t" \
    "v_mfma_f32_16x16x32_f16 a[56:59], v[88:91], v[156:159], a[56:59]\n\t" \
    "buffer_load_dwordx4 v[208:211], %[vW], %[rW], s40 offen nt\n\t" \
    "v_mfma_f32_16x16x32_f16 a[24:27], v[84:87], v[156:159], a[24:27]\n\t" \
    "ds_read_b128 v[156:159], %[vA0] offset:26624\n\t" \
    "s_waitcnt lgkmcnt(3)\n\t" \
    "v_mfma_f32_16x16x32_f16 a[28:31], v[84:87], v[160:163], a[28:31]\n\t" \
    "s_add_u32 s40, s40, 32768\n\t" \
    "v_mfma_f32_16x16x32_f16 a[60:63], v[88:91], v[160:163], a[60:63]\n\t" \
    "buffer_load_dwordx4 v[212:215], %[vW], %[rW], s40 offen nt\n\t" \
    "v_mfma_f32_16x16x32_f16 a[92:95], v[92:95], v[160:163], a[92:95]\n\t" \
    "v_mfma_f32_16x16x32_f16 a[124:127], v[96:99], v[160:163], a[124:127]\n\t" \
    "s_add_u32 s40, s40, 32768\n\t" \
    "v_mfma_f32_16x16x32_f16 a[156:159], v[100:103], v[160:163], a[156:159]\n\t" \
    "v_mfma_f32_16x16x32_f16 a[188:191], v[104:107], v[160:163], a[188:191]\n\t" \
    "buffer_load_dwordx4 v[216:219], %[vW], %[rW], s40 offen nt\n\t" \
    "v_mfma_f32_16x16x32_f16 a[220:223], v[108:111], v[160:163], a[220:223]\n\t" \
    "v_mfma_f32_16x16x32_f16 a[252:255], v[112:115], v[160:163], a[252:255]\n\t" \
    "ds_read_b128 v[160:163], %[vA0] offset:27648\n\t" \
    "s_waitcnt lgkmcnt(3)\n\t" \
    "v_mfma_f32_16x16x32_f16 a[224:227], v[144:147], v[148:151], a[224:227]\n\t" \
    "s_add_u32 s40, s40, 32768\n\t" \
    "v_mfma_f32_16x16x32_f16 a[192:195], v[140:143], v[148:151], a[192:195]\n\t" \
    "v_mfma_f32_16x16x32_f16 a[160:163], v[136:139], v[148:151], a[160:163]\n\t" \
    "buffer_load_dwordx4 v[220:223], %[vW], %[rW], s40 offen nt\n\t" \
    "v_mfma_f32_16x16x32_f16 a[128:131], v[132:135], v[148:151], a[128:131]\n\t" \
    "v_mfma_f32_16x16x32_f16 a[96:99], v[128:131], v[148:151], a[96:99]\n\t" \
    "v_mfma_f32_16x16x32_f16 a[64:67], v[124:127], v[148:151], a[64:67]\n\t" \
    "v_mfma_f32_16x16x32_f16 a[32:35], v[120:123], v[148:151], a[32:35]\n\t" \
    "s_add_u32 s40, s40, 32768\n\t" \
    "v_mfma_f32_16x16x32_f16 a[0:3], v[116:119], v[148:151], a[0:3]\n\t" \
    "ds_read_b128 v[148:151], %[vA0] offset:28672\n\t" \
    "s_waitcnt lgkmcnt(3)\n\t" \
    "v_mfma_f32_16x16x32_f16 a[4:7], v[116:119], v[152:155], a[4:7]\n\t" \
    "buffer_load_dwordx4 v[224:227], %[vW], %[rW], s40 offen nt\n\t" \
    "v_mfma_f32_16x16x32_f16 a[36:39], v[120:123], v[152:155], a[36:39]\n\t" \
    "v_mfma_f32_16x16x32_f16 a[68:71], v[124:127], v[152:155], a[68:71]\n\t" \
    "v_mfma_f32_16x16x32_f16 a[100:103], v[128:131], v[152:155], a[100:103]\n\t" \
    "s_add_u32 s55, s38, 8192\n\t" \
    "v_mfma_f32_16x16x32_f16 a[132:135], v[132:135], v[152:155], a[132:135]\n\t" \
    "v_and_b32 v252, s53, v252\n\t" \
    "v_mfma_f32_16x16x32_f16 a[164:167], v[136:139], v[152:155], a[164:167]\n\t" \
    "buffer_load_dwordx4 v[52:55], %[vA0], %[rWS], s55 offen offset:0 sc0 sc1\n\t" \
    "v_mfma_f32_16x16x32_f16 a[196:199], v[140:143], v[152:155], a[196:199]\n\t" \
    "v_and_b32 v253, s53, v253\n\t" \
    "v_and_b32 v254, s53, v254\n\t" \
    "v_mfma_f32_16x16x32_f16 a[228:231], v[144:147], v[152:155], a[228:231]\n\t" \
    "ds_read_b128 v[152:155], %[vA0] offset:29696\n\t" \
    "s_waitcnt lgkmcnt(3)\n\t" \
    "v_mfma_f32_16x16x32_f16 a[232:235], v[144:147], v[156:159], a[232:235]\n\t" \
    "v_and_b32 v255, s53, v255\n\t" \
    "s_waitcnt vmcnt(9)\n\t" \
    "v_add_f32 v236, v236, v240\n\t" \
    "v_mfma_f32_16x16x32_f16 a[200:203], v[140:143], v[156:159], a[200:203]\n\t" \
    "v_add_f32 v237, v237, v241\n\t" \
    "v_mfma_f32_16x16x32_f16 a[168:171], v[136:139], v[156:159], a[168:171]\n\t" \
    "buffer_load_dwordx4 v[56:59], %[vA0], %[rWS], s55 offen offset:1024 sc0 sc1\n\t" \
    "v_mfma_f32_16x16x32_f16 a[136:139], v[132:135], v[156:159], a[136:139]\n\t" \
    "v_add_f32 v238, v238, v242\n\t" \
    "v_add_f32 v239, v239, v243\n\t" \
    "v_mfma_f32_16x16x32_f16 a[104:107], v[128:131], v[156:159], a[104:107]\n\t" \
    "v_add_f32 v252, v252, v236\n\t" \
    "v_mfma_f32_16x16x32_f16 a[72:75], v[124:127], v[156:159], a[72:75]\n\t" \
    "v_add_f32 v253, v253, v237\n\t" \
    "v_mfma_f32_16x16x32_f16 a[40:43], v[120:123], v[156:159], a[40:43]\n\t" \
    "v_add_f32 v254, v254, v238\n\t" \
    "v_mfma_f32_16x16x32_f16 a[8:11], v[116:119], v[156:159], a[8:11]\n\t" \
    "ds_read_b128 v[156:159], %[vA0] offset:30720\n\t" \
    "s_waitcnt lgkmcnt(3)\n\t" \
    "v_mfma_f32_16x16x32_f16 a[12:15], v[116:119], v[160:163], a[12:15]\n\t" \
    "buffer_load_dwordx4 v[60:63], %[vA0], %[rWS], s55 offen offset:2048 sc0 sc1\n\t" \
    "v_mfma_f32_16x16x32_f16 a[44:47], v[120:123], v[160:163], a[44:47]\n\t" \
    "v_add_f32 v255, v255, v239\n\t" \
    "s_waitcnt vmcnt(8)\n\t" \
    "v_add_f32 v244, v244, v248\n\t" \
    "v_mfma_f32_16x16x32_f16 a[76:79], v[124:127], v[160:163], a[76:79]\n\t" \
    "v_add_f32 v245, v245, v249\n\t" \
    "v_add_f32 v246, v246, v250\n\t" \
    "v_mfma_f32_16x16x32_f16 a[108:111], v[128:131], v[160:163], a[108:111]\n\t" \
    "v_add_f32 v247, v247, v251\n\t" \
    "v_mfma_f32_16x16x32_f16 a[140:143], v[132:135], v[160:163], a[140:143]\n\t" \
    "buffer_load_dwordx4 v[64:67], %[vA0], %[rWS], s55 offen offset:3072 sc0 sc1\n\t" \
    "v_mfma_f32_16x16x32_f16 a[172:175], v[136:139], v[160:163], a[172:175]\n\t" \
    "v_add_f32 v252, v252, v244\n\t" \
    "v_add_f32 v253, v253, v245\n\t" \
    "v_mfma_f32_16x16x32_f16 a[204:207], v[140:143], v[160:163], a[204:207]\n\t" \
    "v_add_f32 v254, v254, v246\n\t" \
    "v_mfma_f32_16x16x32_f16 a[236:239], v[144:147], v[160:163], a[236:239]\n\t" \
    "ds_read_b128 v[160:163], %[vA0] offset:31744\n\t" \
    "s_waitcnt lgkmcnt(3)\n\t" \
    "v_mfma_f32_16x16x32_f16 a[240:243], v[144:147], v[148:151], a[240:243]\n\t" \
    "s_add_u32 s55, s38, 12288\n\t" \
    "v_add_f32 v255, v255, v247\n\t" \
    "v_mfma_f32_16x16x32_f16 a[208:211], v[140:143], v[148:151], a[208:211]\n\t" \
    "v_med3_f32 v236, v252, %[m60], %[p60]\n\t" \
    "v_mfma_f32_16x16x32_f16 a[176:179], v[136:139], v[148:151], a[176:179]\n\t" \
    "buffer_load_dwordx4 v[68:71], %[vA0], %[rWS], s55 offen offset:0 sc0 sc1\n\t" \
    "v_mfma_f32_16x16x32_f16 a[144:147], v[132:135], v[148:151], a[144:147]\n\t" \
    "v_med3_f32 v237, v253, %[m60], %[p60]\n\t" \
    "v_med3_f32 v238, v254, %[m60], %[p60]\n\t" \
    "v_mfma_f32_16x16x32_f16 a[112:115], v[128:131], v[148:151], a[112:115]\n\t" \
    "v_med3_f32 v239, v255, %[m60], %[p60]\n\t" \
    "v_mul_f32 v236, 0xbfb8aa3b, v236\n\t" \
    "v_mfma_f32_16x16x32_f16 a[80:83], v[124:127], v[148:151], a[80:83]\n\t" \
    "v_mul_f32 v237, 0xbfb8aa3b, v237\n\t" \
    "v_mfma_f32_16x16x32_f16 a[48:51], v[120:123], v[148:151], a[48:51]\n\t" \
    "v_mul_f32 v238, 0xbfb8aa3b, v238\n\t" \
    "v_mfma_f32_16x16x32_f16 a[16:19], v[116:119], v[148:151], a[16:19]\n\t" \
    "s_waitcnt lgkmcnt(0)\n\t" \
    "s_barrier\n\t" \
    "ds_read_b128 v[148:151], %[vA0] offset:32768\n\t" \
    "v_mfma_f32_16x16x32_f16 a[20:23], v[116:119], v[152:155], a[20:23]\n\t" \
    "buffer_load_dwordx4 v[72:75], %[vA0], %[rWS], s55 offen offset:1024 sc0 sc1\n\t" \
    "v_mfma_f32_16x16x32_f16 a[52:55], v[120:123], v[152:155], a[52:55]\n\t" \
    "v_mul_f32 v239, 0xbfb8aa3b, v239\n\t" \
    "v_mfma_f32_16x16x32_f16 a[84:87], v[124:127], v[152:155], a[84:87]\n\t" \
    "v_exp_f32 v236, v236\n\t" \
    "v_mfma_f32_16x16x32_f16 a[116:119], v[128:131], v[152:155], a[116:119]\n\t" \
    "v_exp_f32 v237, v237\n\t" \
    "v_mfma_f32_16x16x32_f16 a[148:151], v[132:135], v[152:155], a[148:151]\n\t" \
    "v_exp_f32 v238, v238\n\t" \
    "v_mfma_f32_16x16x32_f16 a[180:183], v[136:139], v[152:155], a[180:183]\n\t" \
    "v_exp_f32 v239, v239\n\t" \
    "v_mfma_f32_16x16x32_f16 a[212:215], v[140:143], v[152:155], a[212:215]\n\t" \
    "buffer_load_dwordx4 v[76:79], %[vA0], %[rWS], s55 offen offset:2048 sc0 sc1\n\t" \
    "v_mfma_f32_16x16x32_f16 a[244:247], v[144:147], v[152:155], a[244:247]\n\t" \
    "ds_read_b128 v[152:155], %[vA0] offset:33792\n\t" \
    "v_mfma_f32_16x16x32_f16 a[248:251], v[144:147], v[156:159], a[248:251]\n\t" \
    "v_add_f32 v236, 1.0, v236\n\t" \
    "v_add_f32 v237, 1.0, v237\n\t" \
    "v_mfma_f32_16x16x32_f16 a[216:219], v[140:143], v[156:159], a[216:219]\n\t" \
    "v_add_f32 v238, 1.0, v238\n\t" \
    "v_add_f32 v239, 1.0, v239\n\t" \
    "v_mfma_f32_16x16x32_f16 a[184:187], v[136:139], v[156:159], a[184:187]\n\t" \
    "v_rcp_f32 v236, v236\n\t" \
    "v_mfma_f32_16x16x32_f16 a[152:155], v[132:135], v[156:159], a[152:155]\n\t" \
    "buffer_load_dwordx4 v[80:83], %[vA0], %[rWS], s55 offen offset:3072 sc0 sc1\n\t" \
    "v_mfma_f32_16x16x32_f16 a[120:123], v[128:131], v[156:159], a[120:123]\n\t" \
    "v_rcp_f32 v237, v237\n\t" \
    "v_mfma_f32_16x16x32_f16 a[88:91], v[124:127], v[156:159], a[88:91]\n\t" \
    "s_add_u32 s55, s38, 16384\n\t" \
    "v_mfma_f32_16x16x32_f16 a[56:59], v[120:123], v[156:159], a[56:59]\n\t" \
    "v_rcp_f32 v238, v238\n\t" \
    "v_mfma_f32_16x16x32_f16 a[24:27], v[116:119], v[156:159], a[24:27]\n\t" \
    "ds_read_b128 v[156:159], %[vA0] offset:34816\n\t" \
    "v_mfma_f32_16x16x32_f16 a[28:31], v[116:119], v[160:163], a[28:31]\n\t" \
    "v_rcp_f32 v239, v239\n\t" \
    "v_mfma_f32_16x16x32_f16 a[60:63], v[120:123], v[160:163], a[60:63]\n\t" \
    "buffer_load_dwordx4 v[84:87], %[vA0], %[rWS], s55 offen offset:0 sc0 sc1\n\t" \
    "v_mfma_f32_16x16x32_f16 a[92:95], v[124:127], v[160:163], a[92:95]\n\t" \
    "v_mul_f32 v232, v252, v236\n\t" \
    "v_mul_f32 v233, v253, v237\n\t" \
    "v_mfma_f32_16x16x32_f16 a[124:127], v[128:131], v[160:163], a[124:127]\n\t" \
    "v_mul_f32 v234, v254, v238\n\t" \
    "v_mfma_f32_16x16x32_f16 a[156:159], v[132:135], v[160:163], a[156:159]\n\t" \
    "buffer_load_dwordx4 v[88:91], %[vA0], %[rWS], s55 offen offset:1024 sc0 sc1\n\t" \
    "v_mfma_f32_16x16x32_f16 a[188:191], v[136:139], v[160:163], a[188:191]\n\t" \
    "v_mul_f32 v235, v255, v239\n\t" \
    "v_mfma_f32_16x16x32_f16 a[220:223], v[140:143], v[160:163], a[220:223]\n\t" \
    "buffer_load_dwordx4 v[92:95], %[vA0], %[rWS], s55 offen offset:2048 sc0 sc1\n\t" \
    "v_mfma_f32_16x16x32_f16 a[252:255], v[144:147], v[160:163], a[252:255]\n\t" \
    "buffer_load_dwordx4 v[96:99], %[vA0], %[rWS], s55 offen offset:3072 sc0 sc1\n\t" \
    "s_add_u32 s36, s36, 1\n\t" \
    "ds_read_b128 v[160:163], %[vA0] offset:35840\n\t" \
    "s_waitcnt lgkmcnt(3)\n\t" \
    "v_mfma_f32_16x16x32_f16 a[224:227], v[48:51], v[148:151], a[224:227]\n\t" \
    "s_mov_b32 s37, s38\n\t" \
    "v_cvt_pk_f16_f32 v228, v164, v168\n\t" \
    "v_mfma_f32_16x16x32_f16 a[192:195], v[44:47], v[148:151], a[192:195]\n\t" \
    "v_mfma_f32_16x16x32_f16 a[160:163], v[40:43], v[148:151], a[160:163]\n\t" \
    "v_cvt_pk_f16_f32 v229, v172, v176\n\t" \
    "v_mfma_f32_16x16x32_f16 a[128:131], v[36:39], v[148:151], a[128:131]\n\t" \
    "ds_write_b64 %[vWr0], v[228:229] offset:0\n\t" \
    "v_mfma_f32_16x16x32_f16 a[96:99], v[32:35], v[148:151], a[96:99]\n\t" \
    "v_cvt_pk_f16_f32 v230, v180, v184\n\t" \
    "v_mfma_f32_16x16x32_f16 a[64:67], v[28:31], v[148:151], a[64:67]\n\t" \
    "v_mfma_f32_16x16x32_f16 a[32:35], v[24:27], v[148:151], a[32:35]\n\t" \
    "v_cvt_pk_f16_f32 v231, v188, v192\n\t" \
    "v_mfma_f32_16x16x32_f16 a[0:3], v[20:23], v[148:151], a[0:3]\n\t" \
    "ds_read_b128 v[148:151], %[vA0] offset:36864\n\t" \
    "s_waitcnt lgkmcnt(4)\n\t" \
    "v_mfma_f32_16x16x32_f16 a[4:7], v[20:23], v[152:155], a[4:7]\n\t" \
    "s_cmp_ge_u32 s48, 2\n\ts_cselect_b64 exec, -1, 0\n\ts_cselect_b32 s69, %[own], 9\n\tds_read_b128 v[252:255], %[vStg] offset:21504\n\ts_mov_b64 exec, -1\n\t" \
    "s_waitcnt lgkmcnt(4)\n\t" \
    "v_mfma_f32_16x16x32_f16 a[36:39], v[24:27], v[152:155], a[36:39]\n\t" \
    "ds_write_b64 %[vWr0], v[230:231] offset:8\n\t" \
    "v_mfma_f32_16x16x32_f16 a[68:71], v[28:31], v[152:155], a[68:71]\n\t" \
    "v_cvt_pk_f16_f32 v228, v165, v169\n\t" \
    "s_add_u32 s55, s37, 20480\n\t" \
    "v_mfma_f32_16x16x32_f16 a[100:103], v[32:35], v[152:155], a[100:103]\n\t" \
    "v_cvt_pk_f16_f32 v229, v173, v177\n\t" \
    "v_mfma_f32_16x16x32_f16 a[132:135], v[36:39], v[152:155], a[132:135]\n\t" \
    "buffer_load_dwordx4 v[100:103], %[vA0], %[rWS], s55 offen offset:0 sc0 sc1\n\t" \
    "v_mfma_f32_16x16x32_f16 a[164:167], v[40:43], v[152:155], a[164:167]\n\t" \
    "ds_write_b64 %[vWr0], v[228:229] offset:1024\n\t" \
    "v_mfma_f32_16x16x32_f16 a[196:199], v[44:47], v[152:155], a[196:199]\n\t" \
    "v_cvt_pk_f16_f32 v230, v181, v185\n\t" \
    "v_cvt_pk_f16_f32 v231, v189, v193\n\t" \
    "v_mfma_f32_16x16x32_f16 a[228:231], v[48:51], v[152:155], a[228:231]\n\t" \
    "ds_read_b128 v[152:155], %[vA0] offset:37888\n\t" \
    "s_waitcnt lgkmcnt(6)\n\t" \
    "v_mfma_f32_16x16x32_f16 a[232:235], v[48:51], v[156:159], a[232:235]\n\t" \
    "ds_write_b64 %[vWr0], v[230:231] offset:1032\n\t" \
    "v_mfma_f32_16x16x32_f16 a[200:203], v[44:47], v[156:159], a[200:203]\n\t" \
    "buffer_load_dwordx4 v[104:107], %[vA0], %[rWS], s55 offen offset:1024 sc0 sc1\n\t" \
    "v_mfma_f32_16x16x32_f16 a[168:171], v[40:43], v[156:159], a[168:171]\n\t" \
    "v_cvt_pk_f16_f32 v228, v166, v170\n\t" \
    "v_cvt_pk_f16_f32 v229, v174, v178\n\t" \
    "v_mfma_f32_16x16x32_f16 a[136:139], v[36:39], v[156:159], a[136:139]\n\t" \
    "ds_write_b64 %[vWr0], v[228:229] offset:2048\n\t" \
    "v_mfma_f32_16x16x32_f16 a[104:107], v[32:35], v[156:159], a[104:107]\n\t" \
    "v_cvt_pk_f16_f32 v230, v182, v186\n\t" \
    "v_mfma_f32_16x16x32_f16 a[72:75], v[28:31], v[156:159], a[72:75]\n\t" \
    "buffer_load_dwordx4 v[108:111], %[vA0], %[rWS], s55 offen offset:2048 sc0 sc1\n\t" \
    "v_mfma_f32_16x16x32_f16 a[40:43], v[24:27], v[156:159], a[40:43]\n\t" \
    "s_add_u32 s66, s66, 0x400000\n\t" \
    "s_and_b32 s66, s66, 0xfffffff\n\t" \
    "v_mfma_f32_16x16x32_f16 a[8:11], v[20:23], v[156:159], a[8:11]\n\t" \
    "ds_read_b128 v[156:159], %[vA0] offset:38912\n\t" \
    "s_waitcnt lgkmcnt(8)\n\t" \
    "v_mfma_f32_16x16x32_f16 a[12:15], v[20:23], v[160:163], a[12:15]\n\t" \
    "v_cvt_pk_f16_f32 v231, v190, v194\n\t" \
    "v_mfma_f32_16x16x32_f16 a[44:47], v[24:27], v[160:163], a[44:47]\n\t" \
    "ds_write_b64 %[vWr0], v[230:231] offset:2056\n\t" \
    "v_mfma_f32_16x16x32_f16 a[76:79], v[28:31], v[160:163], a[76:79]\n\t" \
    "v_cvt_pk_f16_f32 v228, v167, v171\n\t" \
    "s_add_u32 s39, s66, s43\n\t" \
    "v_mfma_f32_16x16x32_f16 a[108:111], v[32:35], v[160:163], a[108:111]\n\t" \
    "s_add_u32 s39, s39, %[sWk0]\n\t" \
    "v_mfma_f32_16x16x32_f16 a[140:143], v[36:39], v[160:163], a[140:143]\n\t" \
    "buffer_load_dwordx4 v[112:115], %[vA0], %[rWS], s55 offen offset:3072 sc0 sc1\n\t" \
    "v_mfma_f32_16x16x32_f16 a[172:175], v[40:43], v[160:163], a[172:175]\n\t" \
    "v_cvt_pk_f16_f32 v229, v175, v179\n\t" \
    "v_mfma_f32_16x16x32_f16 a[204:207], v[44:47], v[160:163], a[204:207]\n\t" \
    "ds_write_b64 %[vWr0], v[228:229] offset:3072\n\t" \
    "v_mfma_f32_16x16x32_f16 a[236:239], v[48:51], v[160:163], a[236:239]\n\t" \
    "ds_read_b128 v[160:163], %[vA0] offset:39936\n\t" \
    "s_waitcnt lgkmcnt(9)\n\t" \
    "v_mfma_f32_16x16x32_f16 a[240:243], v[48:51], v[148:151], a[240:243]\n\t" \
    "v_cvt_pk_f16_f32 v230, v183, v187\n\t" \
    "v_cvt_pk_f16_f32 v231, v191, v195\n\t" \
    "v_mfma_f32_16x16x32_f16 a[208:211], v[44:47], v[148:151], a[208:211]\n\t" \
    "ds_write_b64 %[vWr0], v[230:231] offset:3080\n\t" \
    "v_mfma_f32_16x16x32_f16 a[176:179], v[40:43], v[148:151], a[176:179]\n\t" \
    "buffer_load_dwordx4 v[164:167], %[vW], %[rW], s39 offen\n\t" \
    "v_mfma_f32_16x16x32_f16 a[144:147], v[36:39], v[148:151], a[144:147]\n\t" \
    "s_add_u32 s38, s38, 32768\n\t" \
    "s_and_b32 s38, s38, 0x1fffff\n\t" \
    "v_mfma_f32_16x16x32_f16 a[112:115], v[32:35], v[148:151], a[112:115]\n\t" \
    "s_or_b32 s38, s38, %[sA0]\n\t" \
    "s_add_u32 s39, s39, 32768\n\t" \
    "v_mfma_f32_16x16x32_f16 a[80:83], v[28:31], v[148:151], a[80:83]\n\t" \
    "buffer_load_dwordx4 v[168:171], %[vW], %[rW], s39 offen\n\t" \
    "v_mfma_f32_16x16x32_f16 a[48:51], v[24:27], v[148:151], a[48:51]\n\t" \
    "v_mfma_f32_16x16x32_f16 a[16:19], v[20:23], v[148:151], a[16:19]\n\t" \
    "ds_read_b128 v[148:151], %[vA0] offset:40960\n\t" \
    "s_waitcnt lgkmcnt(8)\n\t" \
    "v_mfma_f32_16x16x32_f16 a[20:23], v[20:23], v[152:155], a[20:23]\n\t" \
    "s_add_u32 s39, s39, 32768\n\t" \
    "v_mfma_f32_16x16x32_f16 a[52:55], v[24:27], v[152:155], a[52:55]\n\t" \
    "buffer_load_dwordx4 v[172:175], %[vW], %[rW], s39 offen\n\t" \
    "v_mfma_f32_16x16x32_f16 a[84:87], v[28:31], v[152:155], a[84:87]\n\t" \
    "v_mfma_f32_16x16x32_f16 a[116:119], v[32:35], v[152:155], a[116:119]\n\t" \
    "s_add_u32 s39, s39, 32768\n\t" \
    "v_mfma_f32_16x16x32_f16 a[148:151], v[36:39], v[152:155], a[148:151]\n\t" \
    "v_mfma_f32_16x16x32_f16 a[180:183], v[40:43], v[152:155], a[180:183]\n\t" \
    "buffer_load_dwordx4 v[176:179], %[vW], %[rW], s39 offen\n\t" \
    "v_mfma_f32_16x16x32_f16 a[212:215], v[44:47], v[152:155], a[212:215]\n\t" \
    "v_mfma_f32_16x16x32_f16 a[244:247], v[48:51], v[152:155], a[244:247]\n\t" \
    "ds_read_b128 v[152:155], %[vA0] offset:41984\n\t" \
    "s_waitcnt lgkmcnt(6)\n\t" \
    "v_mfma_f32_16x16x32_f16 a[248:251], v[48:51], v[156:159], a[248:251]\n\t" \
    "s_add_u32 s39, s39, 32768\n\t" \
    "v_mfma_f32_16x16x32_f16 a[216:219], v[44:47], v[156:159], a[216:219]\n\t" \
    "buffer_store_dwordx4 v[232:235], %[vO], s[56:59], 0 offen sc0 sc1 nt\n\t" \
    "v_mfma_f32_16x16x32_f16 a[184:187], v[40:43], v[156:159], a[184:187]\n\t" \
    "buffer_load_dwordx4 v[180:183], %[vW], %[rW], s39 offen\n\t" \
    "v_mfma_f32_16x16x32_f16 a[152:155], v[36:39], v[156:159], a[152:155]\n\t" \
    "s_add_u32 s39, s39, 32768\n\t" \
    "v_mfma_f32_16x16x32_f16 a[120:123], v[32:35], v[156:159], a[120:123]\n\t" \
    "v_mfma_f32_16x16x32_f16 a[88:91], v[28:31], v[156:159], a[88:91]\n\t" \
    "buffer_load_dwordx4 v[184:187], %[vW], %[rW], s39 offen\n\t" \
    "v_mfma_f32_16x16x32_f16 a[56:59], v[24:27], v[156:159], a[56:59]\n\t" \
    "v_mfma_f32_16x16x32_f16 a[24:27], v[20:23], v[156:159], a[24:27]\n\t" \
    "ds_read_b128 v[156:159], %[vA0] offset:43008\n\t" \
    "s_waitcnt lgkmcnt(4)\n\t" \
    "v_mfma_f32_16x16x32_f16 a[28:31], v[20:23], v[160:163], a[28:31]\n\t" \
    "s_add_u32 s39, s39, 32768\n\t" \
    "v_mfma_f32_16x16x32_f16 a[60:63], v[24:27], v[160:163], a[60:63]\n\t" \
    "buffer_load_dwordx4 v[188:191], %[vW], %[rW], s39 offen\n\t" \
    "v_mfma_f32_16x16x32_f16 a[92:95], v[28:31], v[160:163], a[92:95]\n\t" \
    "v_mfma_f32_16x16x32_f16 a[124:127], v[32:35], v[160:163], a[124:127]\n\t" \
    "s_add_u32 s39, s39, 32768\n\t" \
    "v_mfma_f32_16x16x32_f16 a[156:159], v[36:39], v[160:163], a[156:159]\n\t" \
    "v_mfma_f32_16x16x32_f16 a[188:191], v[40:43], v[160:163], a[188:191]\n\t" \
    "buffer_load_dwordx4 v[192:195], %[vW], %[rW], s39 offen\n\t" \
    "v_mfma_f32_16x16x32_f16 a[220:223], v[44:47], v[160:163], a[220:223]\n\t" \
    "v_mfma_f32_16x16x32_f16 a[252:255], v[48:51], v[160:163], a[252:255]\n\t" \
    "ds_read_b128 v[160:163], %[vA0] offset:44032\n\t" \
    "s_waitcnt vmcnt(17) lgkmcnt(3)\n\t" \
    "v_mfma_f32_16x16x32_f16 a[224:227], v[80:83], v[148:151], a[224:227]\n\t" \
    "v_cvt_pk_f16_f32 v228, v196, v200\n\t" \
    "v_mfma_f32_16x16x32_f16 a[192:195], v[76:79], v[148:151], a[192:195]\n\t" \
    "v_mfma_f32_16x16x32_f16 a[160:163], v[72:75], v[148:151], a[160:163]\n\t" \
    "v_cvt_pk_f16_f32 v229, v204, v208\n\t" \
    "v_mfma_f32_16x16x32_f16 a[128:131], v[68:71], v[148:151], a[128:131]\n\t" \
    "ds_write_b64 %[vWr1], v[228:229] offset:0\n\t" \
    "v_mfma_f32_16x16x32_f16 a[96:99], v[64:67], v[148:151], a[96:99]\n\t" \
    "v_cvt_pk_f16_f32 v230, v212, v216\n\t" \
    "v_mfma_f32_16x16x32_f16 a[64:67], v[60:63], v[148:151], a[64:67]\n\t" \
    "v_mfma_f32_16x16x32_f16 a[32:35], v[56:59], v[148:151], a[32:35]\n\t" \
    "v_cvt_pk_f16_f32 v231, v220, v224\n\t" \
    "v_mfma_f32_16x16x32_f16 a[0:3], v[52:55], v[148:151], a[0:3]\n\t" \
    "ds_read_b128 v[148:151], %[vA0] offset:45056\n\t" \
    "s_waitcnt lgkmcnt(4)\n\t" \
    "v_mfma_f32_16x16x32_f16 a[4:7], v[52:55], v[152:155], a[4:7]\n\t" \
    "ds_write_b64 %[vWr1], v[230:231] offset:8\n\t" \
    "v_mfma_f32_16x16x32_f16 a[36:39], v[56:59], v[152:155], a[36:39]\n\t" \
    "v_cvt_pk_f16_f32 v228, v197, v201\n\t" \
    "v_mfma_f32_16x16x32_f16 a[68:71], v[60:63], v[152:155], a[68:71]\n\t" \
    "v_cvt_pk_f16_f32 v229, v205, v209\n\t" \
    "v_mfma_f32_16x16x32_f16 a[100:103], v[64:67], v[152:155], a[100:103]\n\t" \
    "ds_write_b64 %[vWr1], v[228:229] offset:1024\n\t" \
    "v_mfma_f32_16x16x32_f16 a[132:135], v[68:71], v[152:155], a[132:135]\n\t" \
    "v_cvt_pk_f16_f32 v230, v213, v217\n\t" \
    "v_mfma_f32_16x16x32_f16 a[164:167], v[72:75], v[152:155], a[164:167]\n\t" \
    "v_mfma_f32_16x16x32_f16 a[196:199], v[76:79], v[152:155], a[196:199]\n\t" \
    "v_cvt_pk_f16_f32 v231, v221, v225\n\t" \
    "v_mfma_f32_16x16x32_f16 a[228:231], v[80:83], v[152:155], a[228:231]\n\t" \
    "ds_read_b128 v[152:155], %[vA0] offset:46080\n\t" \
    "s_waitcnt lgkmcnt(6)\n\t" \
    "v_mfma_f32_16x16x32_f16 a[232:235], v[80:83], v[156:159], a[232:235]\n\t" \
    "s_add_u32 s55, s37, 24576\n\t" \
    "v_mfma_f32_16x16x32_f16 a[200:203], v[76:79], v[156:159], a[200:203]\n\t" \
    "ds_write_b64 %[vWr1], v[230:231] offset:1032\n\t" \
    "v_mfma_f32_16x16x32_f16 a[168:171], v[72:75], v[156:159], a[168:171]\n\t" \
    "v_cvt_pk_f16_f32 v228, v198, v202\n\t" \
    "v_mfma_f32_16x16x32_f16 a[136:139], v[68:71], v[156:159], a[136:139]\n\t" \
    "buffer_load_dwordx4 v[116:119], %[vA0], %[rWS], s55 offen offset:0 sc0 sc1\n\t" \
    "v_mfma_f32_16x16x32_f16 a[104:107], v[64:67], v[156:159], a[104:107]\n\t" \
    "v_cvt_pk_f16_f32 v229, v206, v210\n\t" \
    "v_mfma_f32_16x16x32_f16 a[72:75], v[60:63], v[156:159], a[72:75]\n\t" \
    "ds_write_b64 %[vWr1], v[228:229] offset:2048\n\t" \
    "v_mfma_f32_16x16x32_f16 a[40:43], v[56:59], v[156:159], a[40:43]\n\t" \
    "v_cvt_pk_f16_f32 v230, v214, v218\n\t" \
    "v_mfma_f32_16x16x32_f16 a[8:11], v[52:55], v[156:159], a[8:11]\n\t" \
    "ds_read_b128 v[156:159], %[vA0] offset:47104\n\t" \
    "s_waitcnt lgkmcnt(8)\n\t" \
    "v_mfma_f32_16x16x32_f16 a[12:15], v[52:55], v[160:163], a[12:15]\n\t" \
    "buffer_load_dwordx4 v[120:123], %[vA0], %[rWS], s55 offen offset:1024 sc0 sc1\n\t" \
    "v_mfma_f32_16x16x32_f16 a[44:47], v[56:59], v[160:163], a[44:47]\n\t" \
    "v_cvt_pk_f16_f32 v231, v222, v226\n\t" \
    "v_mfma_f32_16x16x32_f16 a[76:79], v[60:63], v[160:163], a[76:79]\n\t" \
    "ds_write_b64 %[vWr1], v[230:231] offset:2056\n\t" \
    "v_mfma_f32_16x16x32_f16 a[108:111], v[64:67], v[160:163], a[108:111]\n\t" \
    "v_cvt_pk_f16_f32 v228, v199, v203\n\t" \
    "v_cvt_pk_f16_f32 v229, v207, v211\n\t" \
    "v_mfma_f32_16x16x32_f16 a[140:143], v[68:71], v[160:163], a[140:143]\n\t" \
    "buffer_load_dwordx4 v[124:127], %[vA0], %[rWS], s55 offen offset:2048 sc0 sc1\n\t" \
    "v_mfma_f32_16x16x32_f16 a[172:175], v[72:75], v[160:163], a[172:175]\n\t" \
    "ds_write_b64 %[vWr1], v[228:229] offset:3072\n\t" \
    "v_mfma_f32_16x16x32_f16 a[204:207], v[76:79], v[160:163], a[204:207]\n\t" \
    "v_cvt_pk_f16_f32 v230, v215, v219\n\t" \
    "v_cvt_pk_f16_f32 v231, v223, v227\n\t" \
    "v_mfma_f32_16x16x32_f16 a[236:239], v[80:83], v[160:163], a[236:239]\n\t" \
    "ds_read_b128 v[160:163], %[vA0] offset:48128\n\t" \
    "s_waitcnt lgkmcnt(9)\n\t" \
    "v_mfma_f32_16x16x32_f16 a[240:243], v[80:83], v[148:151], a[240:243]\n\t" \
    "ds_write_b64 %[vWr1], v[230:231] offset:3080\n\t" \
    "v_mfma_f32_16x16x32_f16 a[208:211], v[76:79], v[148:151], a[208:211]\n\t" \
    "buffer_load_dwordx4 v[128:131], %[vA0], %[rWS], s55 offen offset:3072 sc0 sc1\n\t" \
    "v_mfma_f32_16x16x32_f16 a[176:179], v[72:75], v[148:151], a[176:179]\n\t" \
    "v_mfma_f32_16x16x32_f16 a[144:147], v[68:71], v[148:151], a[144:147]\n\t" \
    "s_add_u32 s55, s37, 28672\n\t" \
    "v_mfma_f32_16x16x32_f16 a[112:115], v[64:67], v[148:151], a[112:115]\n\t" \
    "v_mfma_f32_16x16x32_f16 a[80:83], v[60:63], v[148:151], a[80:83]\n\t" \
    "buffer_load_dwordx4 v[132:135], %[vA0], %[rWS], s55 offen offset:0 sc0 sc1\n\t" \
    "v_mfma_f32_16x16x32_f16 a[48:51], v[56:59], v[148:151], a[48:51]\n\t" \
    "v_mfma_f32_16x16x32_f16 a[16:19], v[52:55], v[148:151], a[16:19]\n\t" \
    "ds_read_b128 v[148:151], %[vA0] offset:49152\n\t" \
    "s_waitcnt lgkmcnt(8)\n\t" \
    "v_mfma_f32_16x16x32_f16 a[20:23], v[52:55], v[152:155], a[20:23]\n\t" \
    "v_mfma_f32_16x16x32_f16 a[52:55], v[56:59], v[152:155], a[52:55]\n\t" \
    "v_mfma_f32_16x16x32_f16 a[84:87], v[60:63], v[152:155], a[84:87]\n\t" \
    "buffer_load_dwordx4 v[136:139], %[vA0], %[rWS], s55 offen offset:1024 sc0 sc1\n\t" \
    "v_mfma_f32_16x16x32_f16 a[116:119], v[64:67], v[152:155], a[116:119]\n\t" \
    "v_mfma_f32_16x16x32_f16 a[148:151], v[68:71], v[152:155], a[148:151]\n\t" \
    "v_mfma_f32_16x16x32_f16 a[180:183], v[72:75], v[152:155], a[180:183]\n\t" \
    "v_mfma_f32_16x16x32_f16 a[212:215], v[76:79], v[152:155], a[212:215]\n\t" \
    "v_mfma_f32_16x16x32_f16 a[244:247], v[80:83], v[152:155], a[244:247]\n\t" \
    "ds_read_b128 v[152:155], %[vA0] offset:50176\n\t" \
    "s_waitcnt lgkmcnt(6)\n\t" \
    "v_mfma_f32_16x16x32_f16 a[248:251], v[80:83], v[156:159], a[248:251]\n\t" \
    "buffer_load_dwordx4 v[140:143], %[vA0], %[rWS], s55 offen offset:2048 sc0 sc1\n\t" \
    "v_mfma_f32_16x16x32_f16 a[216:219], v[76:79], v[156:159], a[216:219]\n\t" \
    "v_mfma_f32_16x16x32_f16 a[184:187], v[72:75], v[156:159], a[184:187]\n\t" \
    "v_mfma_f32_16x16x32_f16 a[152:155], v[68:71], v[156:159], a[152:155]\n\t" \
    "v_mfma_f32_16x16x32_f16 a[120:123], v[64:67], v[156:159], a[120:123]\n\t" \
    "v_mfma_f32_16x16x32_f16 a[88:91], v[60:63], v[156:159], a[88:91]\n\t" \
    "buffer_load_dwordx4 v[144:147], %[vA0], %[rWS], s55 offen offset:3072 sc0 sc1\n\t" \
    "v_mfma_f32_16x16x32_f16 a[56:59], v[56:59], v[156:159], a[56:59]\n\t" \
    "v_mfma_f32_16x16x32_f16 a[24:27], v[52:55], v[156:159], a[24:27]\n\t" \
    "ds_read_b128 v[156:159], %[vA0] offset:51200\n\t" \
    "s_waitcnt lgkmcnt(4)\n\t" \
    "v_mfma_f32_16x16x32_f16 a[28:31], v[52:55], v[160:163], a[28:31]\n\t" \
    "s_add_u32 s55, s38, 0\n\t" \
    "v_mfma_f32_16x16x32_f16 a[60:63], v[56:59], v[160:163], a[60:63]\n\t" \
    "v_mfma_f32_16x16x32_f16 a[92:95], v[60:63], v[160:163], a[92:95]\n\t" \
    "buffer_load_dwordx4 v[20:23], %[vA0], %[rWS], s55 offen offset:0 sc0 sc1\n\t" \
    "v_mfma_f32_16x16x32_f16 a[124:127], v[64:67], v[160:163], a[124:127]\n\t" \
    "v_mfma_f32_16x16x32_f16 a[156:159], v[68:71], v[160:163], a[156:159]\n\t" \
    "v_mfma_f32_16x16x32_f16 a[188:191], v[72:75], v[160:163], a[188:191]\n\t" \
    "v_mfma_f32_16x16x32_f16 a[220:223], v[76:79], v[160:163], a[220:223]\n\t" \
    "v_mfma_f32_16x16x32_f16 a[252:255], v[80:83], v[160:163], a[252:255]\n\t" \
    "ds_read_b128 v[160:163], %[vA0] offset:52224\n\t" \
    "s_waitcnt vmcnt(18) lgkmcnt(3)\n\t" \
    "v_mfma_f32_16x16x32_f16 a[224:227], v[112:115], v[148:151], a[224:227]\n\t" \
    "buffer_load_dwordx4 v[24:27], %[vA0], %[rWS], s55 offen offset:1024 sc0 sc1\n\t" \
    "v_mfma_f32_16x16x32_f16 a[192:195], v[108:111], v[148:151], a[192:195]\n\t" \
    "v_mfma_f32_16x16x32_f16 a[160:163], v[104:107], v[148:151], a[160:163]\n\t" \
    "v_mfma_f32_16x16x32_f16 a[128:131], v[100:103], v[148:151], a[128:131]\n\t" \
    "v_mfma_f32_16x16x32_f16 a[96:99], v[96:99], v[148:151], a[96:99]\n\t" \
    "buffer_load_dwordx4 v[28:31], %[vA0], %[rWS], s55 offen offset:2048 sc0 sc1\n\t" \
    "v_mfma_f32_16x16x32_f16 a[64:67], v[92:95], v[148:151], a[64:67]\n\t" \
    "v_mfma_f32_16x16x32_f16 a[32:35], v[88:91], v[148:151], a[32:35]\n\t" \
    "v_mfma_f32_16x16x32_f16 a[0:3], v[84:87], v[148:151], a[0:3]\n\t" \
    "ds_read_b128 v[148:151], %[vA0] offset:53248\n\t" \
    "s_waitcnt lgkmcnt(3)\n\t" \
    "v_mfma_f32_16x16x32_f16 a[4:7], v[84:87], v[152:155], a[4:7]\n\t" \
    "v_mfma_f32_16x16x32_f16 a[36:39], v[88:91], v[152:155], a[36:39]\n\t" \
    "buffer_load_dwordx4 v[32:35], %[vA0], %[rWS], s55 offen offset:3072 sc0 sc1\n\t" \
    "v_mfma_f32_16x16x32_f16 a[68:71], v[92:95], v[152:155], a[68:71]\n\t" \
    "v_mfma_f32_16x16x32_f16 a[100:103], v[96:99], v[152:155], a[100:103]\n\t" \
    "v_mfma_f32_16x16x32_f16 a[132:135], v[100:103], v[152:155], a[132:135]\n\t" \
    "v_mfma_f32_16x16x32_f16 a[164:167], v[104:107], v[152:155], a[164:167]\n\t" \
    "s_add_u32 s55, s38, 4096\n\t" \
    "v_mfma_f32_16x16x32_f16 a[196:199], v[108:111], v[152:155], a[196:199]\n\t" \
    "v_mfma_f32_16x16x32_f16 a[228:231], v[112:115], v[152:155], a[228:231]\n\t" \
    "ds_read_b128 v[152:155], %[vA0] offset:54272\n\t" \
    "s_waitcnt lgkmcnt(3)\n\t" \
    "v_mfma_f32_16x16x32_f16 a[232:235], v[112:115], v[156:159], a[232:235]\n\t" \
    "buffer_load_dwordx4 v[36:39], %[vA0], %[rWS], s55 offen offset:0 sc0 sc1\n\t" \
    "v_mfma_f32_16x16x32_f16 a[200:203], v[108:111], v[156:159], a[200:203]\n\t" \
    "v_mfma_f32_16x16x32_f16 a[168:171], v[104:107], v[156:159], a[168:171]\n\t" \
    "v_mfma_f32_16x16x32_f16 a[136:139], v[100:103], v[156:159], a[136:139]\n\t" \
    "v_mfma_f32_16x16x32_f16 a[104:107], v[96:99], v[156:159], a[104:107]\n\t" \
    "buffer_load_dwordx4 v[40:43], %[vA0], %[rWS], s55 offen offset:1024 sc0 sc1\n\t" \
    "v_mfma_f32_16x16x32_f16 a[72:75], v[92:95], v[156:159], a[72:75]\n\t" \
    "v_readlane_b32 s52, %[vTab0], s51\n\t" \
    "v_mfma_f32_16x16x32_f16 a[40:43], v[88:91], v[156:159], a[40:43]\n\t" \
    "s_add_u32 s42, s51, 32\n\t" \
    "v_mfma_f32_16x16x32_f16 a[8:11], v[84:87], v[156:159], a[8:11]\n\t" \
    "ds_read_b128 v[156:159], %[vA0] offset:55296\n\t" \
    "s_waitcnt lgkmcnt(3)\n\t" \
    "v_mfma_f32_16x16x32_f16 a[12:15], v[84:87], v[160:163], a[12:15]\n\t" \
    "v_readlane_b32 s53, %[vTab1], s51\n\t" \
    "v_mfma_f32_16x16x32_f16 a[44:47], v[88:91], v[160:163], a[44:47]\n\t" \
    "v_readlane_b32 s42, %[vTab0], s42\n\t" \
    "v_mfma_f32_16x16x32_f16 a[76:79], v[92:95], v[160:163], a[76:79]\n\t" \
    "s_add_u32 s51, s51, 1\n\t" \
    "v_mfma_f32_16x16x32_f16 a[108:111], v[96:99], v[160:163], a[108:111]\n\t" \
    "buffer_load_dwordx4 v[44:47], %[vA0], %[rWS], s55 offen offset:2048 sc0 sc1\n\t" \
    "v_mfma_f32_16x16x32_f16 a[140:143], v[100:103], v[160:163], a[140:143]\n\t" \
    "s_add_u32 s40, s66, s43\n\t" \
    "s_add_u32 s40, s40, %[sWk1]\n\t" \
    "v_mfma_f32_16x16x32_f16 a[172:175], v[104:107], v[160:163], a[172:175]\n\t" \
    "s_mov_b32 s56, s42\n\t" \
    "s_and_b32 s57, s53, 0xffff\n\t" \
    "v_mfma_f32_16x16x32_f16 a[204:207], v[108:111], v[160:163], a[204:207]\n\t" \
    "s_and_b32 s58, s53, 0x10000\n\t" \
    "s_bfe_i32 s53, s53, 0x10011\n\t" \
    "v_mfma_f32_16x16x32_f16 a[236:239], v[112:115], v[160:163], a[236:239]\n\t" \
    "ds_read_b128 v[160:163], %[vA0] offset:56320\n\t" \
    "s_waitcnt lgkmcnt(3)\n\t" \
    "v_mfma_f32_16x16x32_f16 a[240:243], v[112:115], v[148:151], a[240:243]\n\t" \
    "buffer_load_dwordx4 v[48:51], %[vA0], %[rWS], s55 offen offset:3072 sc0 sc1\n\t" \
    "v_mfma_f32_16x16x32_f16 a[208:211], v[108:111], v[148:151], a[208:211]\n\t" \
    "v_mfma_f32_16x16x32_f16 a[176:179], v[104:107], v[148:151], a[176:179]\n\t" \
    "v_mfma_f32_16x16x32_f16 a[144:147], v[100:103], v[148:151], a[144:147]\n\t" \
    "buffer_load_dwordx4 v[196:199], %[vW], %[rW], s40 offen nt\n\t" \
    "v_mfma_f32_16x16x32_f16 a[112:115], v[96:99], v[148:151], a[112:115]\n\t" \
    "v_mfma_f32_16x16x32_f16 a[80:83], v[92:95], v[148:151], a[80:83]\n\t" \
    "v_mfma_f32_16x16x32_f16 a[48:51], v[88:91], v[148:151], a[48:51]\n\t" \
    "s_add_u32 s40, s40, 32768\n\t" \
    "v_mfma_f32_16x16x32_f16 a[16:19], v[84:87], v[148:151], a[16:19]\n\t" \
    "ds_read_b128 v[148:151], %[vA0] offset:57344\n\t" \
    "s_waitcnt lgkmcnt(3)\n\t" \
    "v_mfma_f32_16x16x32_f16 a[20:23], v[84:87], v[152:155], a[20:23]\n\t" \
    "s_cmp_eq_u32 s69, 0\n\t" \
    "s_cselect_b32 s42, 1073741824, s52\n\t" \
    "v_mfma_f32_16x16x32_f16 a[52:55], v[88:91], v[152:155], a[52:55]\n\t" \
    "buffer_load_dwordx4 v[200:203], %[vW], %[rW], s40 offen nt\n\t" \
    "v_mfma_f32_16x16x32_f16 a[84:87], v[92:95], v[152:155], a[84:87]\n\t" \
    "buffer_load_dwordx4 v[236:239], %[vA0], %[rWS], s42 offen sc0 sc1\n\t" \
    "v_mfma_f32_16x16x32_f16 a[116:119], v[96:99], v[152:155], a[116:119]\n\t" \
    "s_add_u32 s52, s52, 262144\n\t" \
    "s_cmp_eq_u32 s69, 1\n\t" \
    "v_mfma_f32_16x16x32_f16 a[148:151], v[100:103], v[152:155], a[148:151]\n\t" \
    "s_cselect_b32 s42, 1073741824, s52\n\t" \
    "v_mfma_f32_16x16x32_f16 a[180:183], v[104:107], v[152:155], a[180:183]\n\t" \
    "buffer_load_dwordx4 v[240:243], %[vA0], %[rWS], s42 offen sc0 sc1\n\t" \
    "v_mfma_f32_16x16x32_f16 a[212:215], v[108:111], v[152:155], a[212:215]\n\t" \
    "s_add_u32 s40, s40, 32768\n\t" \
    "s_add_u32 s52, s52, 262144\n\t" \
    "v_mfma_f32_16x16x32_f16 a[244:247], v[112:115], v[152:155], a[244:247]\n\t" \
    "ds_read_b128 v[152:155], %[vA0] offset:58368\n\t" \
    "s_waitcnt lgkmcnt(3)\n\t" \
    "v_mfma_f32_16x16x32_f16 a[248:251], v[112:115], v[156:159], a[248:251]\n\t" \
    "s_cmp_eq_u32 s69, 2\n\t" \
    "s_cselect_b32 s42, 1073741824, s52\n\t" \
    "v_mfma_f32_16x16x32_f16 a[216:219], v[108:111], v[156:159], a[216:219]\n\t" \
    "buffer_load_dwordx4 v[204:207], %[vW], %[rW], s40 offen nt\n\t" \
    "v_mfma_f32_16x16x32_f16 a[184:187], v[104:107], v[156:159], a[184:187]\n\t" \
    "buffer_load_dwordx4 v[244:247], %[vA0], %[rWS], s42 offen sc0 sc1\n\t" \
    "v_mfma_f32_16x16x32_f16 a[152:155], v[100:103], v[156:159], a[152:155]\n\t" \
    "s_add_u32 s52, s52, 262144\n\t" \
    "s_cmp_eq_u32 s69, 3\n\t" \
    "v_mfma_f32_16x16x32_f16 a[120:123], v[96:99], v[156:159], a[120:123]\n\t" \
    "s_cselect_b32 s42, 1073741824, s52\n\t" \
    "s_add_u32 s40, s40, 32768\n\t" \
    "v_mfma_f32_16x16x32_f16 a[88:91], v[92:95], v[156:159], a[88:91]\n\t" \
    "buffer_load_dwordx4 v[248:251], %[vA0], %[rWS], s42 offen sc0 sc1\n\t" \
    "v_mfma_f32_16x16x32_f16 a[56:59], v[88:91], v[156:159], a[56:59]\n\t" \
    "buffer_load_dwordx4 v[208:211], %[vW], %[rW], s40 offen nt\n\t" \
    "v_mfma_f32_16x16x32_f16 a[24:27], v[84:87], v[156:159], a[24:27]\n\t" \
    "ds_read_b128 v[156:159], %[vA0] offset:59392\n\t" \
    "s_waitcnt lgkmcnt(3)\n\t" \
    "v_mfma_f32_16x16x32_f16 a[28:31], v[84:87], v[160:163], a[28:31]\n\t" \
    "s_add_u32 s40, s40, 32768\n\t" \
    "v_mfma_f32_16x16x32_f16 a[60:63], v[88:91], v[160:163], a[60:63]\n\t" \
    "buffer_load_dwordx4 v[212:215], %[vW], %[rW], s40 offen nt\n\t" \
    "v_mfma_f32_16x16x32_f16 a[92:95], v[92:95], v[160:163], a[92:95]\n\t" \
    "v_mfma_f32_16x16x32_f16 a[124:127], v[96:99], v[160:163], a[124:127]\n\t" \
    "s_add_u32 s40, s40, 32768\n\t" \
    "v_mfma_f32_16x16x32_f16 a[156:159], v[100:103], v[160:163], a[156:159]\n\t" \
    "v_mfma_f32_16x16x32_f16 a[188:191], v[104:107], v[160:163], a[188:191]\n\t" \
    "buffer_load_dwordx4 v[216:219], %[vW], %[rW], s40 offen nt\n\t" \
    "v_mfma_f32_16x16x32_f16 a[220:223], v[108:111], v[160:163], a[220:223]\n\t" \
    "v_mfma_f32_16x16x32_f16 a[252:255], v[112:115], v[160:163], a[252:255]\n\t" \
    "ds_read_b128 v[160:163], %[vA0] offset:60416\n\t" \
    "s_waitcnt vmcnt(18) lgkmcnt(3)\n\t" \
    "v_mfma_f32_16x16x32_f16 a[224:227], v[144:147], v[148:151], a[224:227]\n\t" \
    "s_add_u32 s40, s40, 32768\n\t" \
    "v_mfma_f32_16x16x32_f16 a[192:195], v[140:143], v[148:151], a[192:195]\n\t" \
    "v_mfma_f32_16x16x32_f16 a[160:163], v[136:139], v[148:151], a[160:163]\n\t" \
    "buffer_load_dwordx4 v[220:223], %[vW], %[rW], s40 offen nt\n\t" \
    "v_mfma_f32_16x16x32_f16 a[128:131], v[132:135], v[148:151], a[128:131]\n\t" \
    "v_mfma_f32_16x16x32_f16 a[96:99], v[128:131], v[148:151], a[96:99]\n\t" \
    "v_mfma_f32_16x16x32_f16 a[64:67], v[124:127], v[148:151], a[64:67]\n\t" \
    "v_mfma_f32_16x16x32_f16 a[32:35], v[120:123], v[148:151], a[32:35]\n\t" \
    "s_add_u32 s40, s40, 32768\n\t" \
    "v_mfma_f32_16x16x32_f16 a[0:3], v[116:119], v[148:151], a[0:3]\n\t" \
    "ds_read_b128 v[148:151], %[vA0] offset:61440\n\t" \
    "s_waitcnt lgkmcnt(3)\n\t" \
    "v_mfma_f32_16x16x32_f16 a[4:7], v[116:119], v[152:155], a[4:7]\n\t" \
    "buffer_load_dwordx4 v[224:227], %[vW], %[rW], s40 offen nt\n\t" \
    "v_mfma_f32_16x16x32_f16 a[36:39], v[120:123], v[152:155], a[36:39]\n\t" \
    "v_mfma_f32_16x16x32_f16 a[68:71], v[124:127], v[152:155], a[68:71]\n\t" \
    "v_mfma_f32_16x16x32_f16 a[100:103], v[128:131], v[152:155], a[100:103]\n\t" \
    "s_add_u32 s55, s38, 8192\n\t" \
    "v_mfma_f32_16x16x32_f16 a[132:135], v[132:135], v[152:155], a[132:135]\n\t" \
    "v_and_b32 v252, s53, v252\n\t" \
    "v_mfma_f32_16x16x32_f16 a[164:167], v[136:139], v[152:155], a[164:167]\n\t" \
    "buffer_load_dwordx4 v[52:55], %[vA0], %[rWS], s55 offen offset:0 sc0 sc1\n\t" \
    "v_mfma_f32_16x16x32_f16 a[196:199], v[140:143], v[152:155], a[196:199]\n\t" \
    "v_and_b32 v253, s53, v253\n\t" \
    "v_and_b32 v254, s53, v254\n\t" \
    "v_mfma_f32_16x16x32_f16 a[228:231], v[144:147], v[152:155], a[228:231]\n\t" \
    "ds_read_b128 v[152:155], %[vA0] offset:62464\n\t" \
    "s_waitcnt lgkmcnt(3)\n\t" \
    "v_mfma_f32_16x16x32_f16 a[232:235], v[144:147], v[156:159], a[232:235]\n\t" \
    "v_and_b32 v255, s53, v255\n\t" \
    "s_waitcnt vmcnt(9)\n\t" \
    "v_add_f32 v236, v236, v240\n\t" \
    "v_mfma_f32_16x16x32_f16 a[200:203], v[140:143], v[156:159], a[200:203]\n\t" \
    "v_add_f32 v237, v237, v241\n\t" \
    "v_mfma_f32_16x16x32_f16 a[168:171], v[136:139], v[156:159], a[168:171]\n\t" \
    "buffer_load_dwordx4 v[56:59], %[vA0], %[rWS], s55 offen offset:1024 sc0 sc1\n\t" \
    "v_mfma_f32_16x16x32_f16 a[136:139], v[132:135], v[156:159], a[136:139]\n\t" \
    "v_add_f32 v238, v238, v242\n\t" \
    "v_add_f32 v239, v239, v243\n\t" \
    "v_mfma_f32_16x16x32_f16 a[104:107], v[128:131], v[156:159], a[104:107]\n\t" \
    "v_add_f32 v252, v252, v236\n\t" \
    "v_mfma_f32_16x16x32_f16 a[72:75], v[124:127], v[156:159], a[72:75]\n\t" \
    "v_add_f32 v253, v253, v237\n\t" \
    "v_mfma_f32_16x16x32_f16 a[40:43], v[120:123], v[156:159], a[40:43]\n\t" \
    "v_add_f32 v254, v254, v238\n\t" \
    "v_mfma_f32_16x16x32_f16 a[8:11], v[116:119], v[156:159], a[8:11]\n\t" \
    "ds_read_b128 v[156:159], %[vA0] offset:63488\n\t" \
    "s_waitcnt lgkmcnt(3)\n\t" \
    "v_mfma_f32_16x16x32_f16 a[12:15], v[116:119], v[160:163], a[12:15]\n\t" \
    "buffer_load_dwordx4 v[60:63], %[vA0], %[rWS], s55 offen offset:2048 sc0 sc1\n\t" \
    "v_mfma_f32_16x16x32_f16 a[44:47], v[120:123], v[160:163], a[44:47]\n\t" \
    "v_add_f32 v255, v255, v239\n\t" \
    "s_waitcnt vmcnt(8)\n\t" \
    "v_add_f32 v244, v244, v248\n\t" \
    "v_mfma_f32_16x16x32_f16 a[76:79], v[124:127], v[160:163], a[76:79]\n\t" \
    "v_add_f32 v245, v245, v249\n\t" \
    "v_add_f32 v246, v246, v250\n\t" \
    "v_mfma_f32_16x16x32_f16 a[108:111], v[128:131], v[160:163], a[108:111]\n\t" \
    "v_add_f32 v247, v247, v251\n\t" \
    "v_mfma_f32_16x16x32_f16 a[140:143], v[132:135], v[160:163], a[140:143]\n\t" \
    "buffer_load_dwordx4 v[64:67], %[vA0], %[rWS], s55 offen offset:3072 sc0 sc1\n\t" \
    "v_mfma_f32_16x16x32_f16 a[172:175], v[136:139], v[160:163], a[172:175]\n\t" \
    "v_add_f32 v252, v252, v244\n\t" \
    "v_add_f32 v253, v253, v245\n\t" \
    "v_mfma_f32_16x16x32_f16 a[204:207], v[140:143], v[160:163], a[204:207]\n\t" \
    "v_add_f32 v254, v254, v246\n\t" \
    "v_mfma_f32_16x16x32_f16 a[236:239], v[144:147], v[160:163], a[236:239]\n\t" \
    "ds_read_b128 v[160:163], %[vA0] offset:64512\n\t" \
    "s_waitcnt lgkmcnt(3)\n\t" \
    "v_mfma_f32_16x16x32_f16 a[240:243], v[144:147], v[148:151], a[240:243]\n\t" \
    "s_add_u32 s55, s38, 12288\n\t" \
    "v_add_f32 v255, v255, v247\n\t" \
    "v_mfma_f32_16x16x32_f16 a[208:211], v[140:143], v[148:151], a[208:211]\n\t" \
    "v_med3_f32 v236, v252, %[m60], %[p60]\n\t" \
    "v_mfma_f32_16x16x32_f16 a[176:179], v[136:139], v[148:151], a[176:179]\n\t" \
    "buffer_load_dwordx4 v[68:71], %[vA0], %[rWS], s55 offen offset:0 sc0 sc1\n\t" \
    "v_mfma_f32_16x16x32_f16 a[144:147], v[132:135], v[148:151], a[144:147]\n\t" \
    "v_med3_f32 v237, v253, %[m60], %[p60]\n\t" \
    "v_med3_f32 v238, v254, %[m60], %[p60]\n\t" \
    "v_mfma_f32_16x16x32_f16 a[112:115], v[128:131], v[148:151], a[112:115]\n\t" \
    "v_med3_f32 v239, v255, %[m60], %[p60]\n\t" \
    "v_mul_f32 v236, 0xbfb8aa3b, v236\n\t" \
    "v_mfma_f32_16x16x32_f16 a[80:83], v[124:127], v[148:151], a[80:83]\n\t" \
    "v_mul_f32 v237, 0xbfb8aa3b, v237\n\t" \
    "v_mfma_f32_16x16x32_f16 a[48:51], v[120:123], v[148:151], a[48:51]\n\t" \
    "v_mul_f32 v238, 0xbfb8aa3b, v238\n\t" \
    "v_mfma_f32_16x16x32_f16 a[16:19], v[116:119], v[148:151], a[16:19]\n\t" \
    "s_waitcnt lgkmcnt(0)\n\t" \
    "s_barrier\n\t" \
    "ds_read_b128 v[148:151], %[vA0] offset:0\n\t" \
    "v_mfma_f32_16x16x32_f16 a[20:23], v[116:119], v[152:155], a[20:23]\n\t" \
    "buffer_load_dwordx4 v[72:75], %[vA0], %[rWS], s55 offen offset:1024 sc0 sc1\n\t" \
    "v_mfma_f32_16x16x32_f16 a[52:55], v[120:123], v[152:155], a[52:55]\n\t" \
    "v_mul_f32 v239, 0xbfb8aa3b, v239\n\t" \
    "v_mfma_f32_16x16x32_f16 a[84:87], v[124:127], v[152:155], a[84:87]\n\t" \
    "v_exp_f32 v236, v236\n\t" \
    "v_mfma_f32_16x16x32_f16 a[116:119], v[128:131], v[152:155], a[116:119]\n\t" \
    "v_exp_f32 v237, v237\n\t" \
    "v_mfma_f32_16x16x32_f16 a[148:151], v[132:135], v[152:155], a[148:151]\n\t" \
    "v_exp_f32 v238, v238\n\t" \
    "v_mfma_f32_16x16x32_f16 a[180:183], v[136:139], v[152:155], a[180:183]\n\t" \
    "v_exp_f32 v239, v239\n\t" \
    "v_mfma_f32_16x16x32_f16 a[212:215], v[140:143], v[152:155], a[212:215]\n\t" \
    "buffer_load_dwordx4 v[76:79], %[vA0], %[rWS], s55 offen offset:2048 sc0 sc1\n\t" \
    "v_mfma_f32_16x16x32_f16 a[244:247], v[144:147], v[152:155], a[244:247]\n\t" \
    "ds_read_b128 v[152:155], %[vA0] offset:1024\n\t" \
    "v_mfma_f32_16x16x32_f16 a[248:251], v[144:147], v[156:159], a[248:251]\n\t" \
    "v_add_f32 v236, 1.0, v236\n\t" \
    "v_add_f32 v237, 1.0, v237\n\t" \
    "v_mfma_f32_16x16x32_f16 a[216:219], v[140:143], v[156:159], a[216:219]\n\t" \
    "v_add_f32 v238, 1.0, v238\n\t" \
    "v_add_f32 v239, 1.0, v239\n\t" \
    "v_mfma_f32_16x16x32_f16 a[184:187], v[136:139], v[156:159], a[184:187]\n\t" \
    "v_rcp_f32 v236, v236\n\t" \
    "v_mfma_f32_16x16x32_f16 a[152:155], v[132:135], v[156:159], a[152:155]\n\t" \
    "buffer_load_dwordx4 v[80:83], %[vA0], %[rWS], s55 offen offset:3072 sc0 sc1\n\t" \
    "v_mfma_f32_16x16x32_f16 a[120:123], v[128:131], v[156:159], a[120:123]\n\t" \
    "v_rcp_f32 v237, v237\n\t" \
    "v_mfma_f32_16x16x32_f16 a[88:91], v[124:127], v[156:159], a[88:91]\n\t" \
    "s_add_u32 s55, s38, 16384\n\t" \
    "v_mfma_f32_16x16x32_f16 a[56:59], v[120:123], v[156:159], a[56:59]\n\t" \
    "v_rcp_f32 v238, v238\n\t" \
    "v_mfma_f32_16x16x32_f16 a[24:27], v[116:119], v[156:159], a[24:27]\n\t" \
    "ds_read_b128 v[156:159], %[vA0] offset:2048\n\t" \
    "v_mfma_f32_16x16x32_f16 a[28:31], v[116:119], v[160:163], a[28:31]\n\t" \
    "v_rcp_f32 v239, v239\n\t" \
    "v_mfma_f32_16x16x32_f16 a[60:63], v[120:123], v[160:163], a[60:63]\n\t" \
    "buffer_load_dwordx4 v[84:87], %[vA0], %[rWS], s55 offen offset:0 sc0 sc1\n\t" \
    "v_mfma_f32_16x16x32_f16 a[92:95], v[124:127], v[160:163], a[92:95]\n\t" \
    "v_mul_f32 v232, v252, v236\n\t" \
    "v_mul_f32 v233, v253, v237\n\t" \
    "v_mfma_f32_16x16x32_f16 a[124:127], v[128:131], v[160:163], a[124:127]\n\t" \
    "v_mul_f32 v234, v254, v238\n\t" \
    "v_mfma_f32_16x16x32_f16 a[156:159], v[132:135], v[160:163], a[156:159]\n\t" \
    "buffer_load_dwordx4 v[88:91], %[vA0], %[rWS], s55 offen offset:1024 sc0 sc1\n\t" \
    "v_mfma_f32_16x16x32_f16 a[188:191], v[136:139], v[160:163], a[188:191]\n\t" \
    "v_mul_f32 v235, v255, v239\n\t" \
    "v_mfma_f32_16x16x32_f16 a[220:223], v[140:143], v[160:163], a[220:223]\n\t" \
    "buffer_load_dwordx4 v[92:95], %[vA0], %[rWS], s55 offen offset:2048 sc0 sc1\n\t" \
    "v_mfma_f32_16x16x32_f16 a[252:255], v[144:147], v[160:163], a[252:255]\n\t" \
    "buffer_load_dwordx4 v[96:99], %[vA0], %[rWS], s55 offen offset:3072 sc0 sc1\n\t" \
    "s_add_u32 s36, s36, 1\n\t" \
    "ds_read_b128 v[160:163], %[vA0] offset:3072\n\t" \
    "s_waitcnt lgkmcnt(3)\n\t" \
    "v_mfma_f32_16x16x32_f16 a[224:227], v[48:51], v[148:151], a[224:227]\n\t" \
    "s_mov_b32 s37, s38\n\t" \
    "v_cvt_pk_f16_f32 v228, v164, v168\n\t" \
    "v_mfma_f32_16x16x32_f16 a[192:195], v[44:47], v[148:151], a[192:195]\n\t" \
    "v_mfma_f32_16x16x32_f16 a[160:163], v[40:43], v[148:151], a[160:163]\n\t" \
    "v_cvt_pk_f16_f32 v229, v172, v176\n\t" \
    "v_mfma_f32_16x16x32_f16 a[128:131], v[36:39], v[148:151], a[128:131]\n\t" \
    "ds_write_b64 %[vWr0], v[228:229] offset:32768\n\t" \
    "v_mfma_f32_16x16x32_f16 a[96:99], v[32:35], v[148:151], a[96:99]\n\t" \
    "v_cvt_pk_f16_f32 v230, v180, v184\n\t" \
    "v_mfma_f32_16x16x32_f16 a[64:67], v[28:31], v[148:151], a[64:67]\n\t" \
    "v_mfma_f32_16x16x32_f16 a[32:35], v[24:27], v[148:151], a[32:35]\n\t" \
    "v_cvt_pk_f16_f32 v231, v188, v192\n\t" \
    "v_mfma_f32_16x16x32_f16 a[0:3], v[20:23], v[148:151], a[0:3]\n\t" \
    "ds_read_b128 v[148:151], %[vA0] offset:4096\n\t" \
    "s_waitcnt lgkmcnt(4)\n\t" \
    "v_mfma_f32_16x16x32_f16 a[4:7], v[20:23], v[152:155], a[4:7]\n\t" \
    "s_cmp_ge_u32 s48, 2\n\ts_cselect_b64 exec, -1, 0\n\ts_cselect_b32 s69, %[own], 9\n\tds_read_b128 v[252:255], %[vStg] offset:22528\n\ts_mov_b64 exec, -1\n\t" \
    "s_waitcnt lgkmcnt(4)\n\t" \
    "v_mfma_f32_16x16x32_f16 a[36:39], v[24:27], v[152:155], a[36:39]\n\t" \
    "ds_write_b64 %[vWr0], v[230:231] offset:32776\n\t" \
    "v_mfma_f32_16x16x32_f16 a[68:71], v[28:31], v[152:155], a[68:71]\n\t" \
    "v_cvt_pk_f16_f32 v228, v165, v169\n\t" \
    "s_add_u32 s55, s37, 20480\n\t" \
    "v_mfma_f32_16x16x32_f16 a[100:103], v[32:35], v[152:155], a[100:103]\n\t" \
    "v_cvt_pk_f16_f32 v229, v173, v177\n\t" \
    "v_mfma_f32_16x16x32_f16 a[132:135], v[36:39], v[152:155], a[132:135]\n\t" \
    "buffer_load_dwordx4 v[100:103], %[vA0], %[rWS], s55 offen offset:0 sc0 sc1\n\t" \
    "v_mfma_f32_16x16x32_f16 a[164:167], v[40:43], v[152:155], a[164:167]\n\t" \
    "ds_write_b64 %[vWr0], v[228:229] offset:33792\n\t" \
    "v_mfma_f32_16x16x32_f16 a[196:199], v[44:47], v[152:155], a[196:199]\n\t" \
    "v_cvt_pk_f16_f32 v230, v181, v185\n\t" \
    "v_cvt_pk_f16_f32 v231, v189, v193\n\t" \
    "v_mfma_f32_16x16x32_f16 a[228:231], v[48:51], v[152:155], a[228:231]\n\t" \
    "ds_read_b128 v[152:155], %[vA0] offset:5120\n\t" \
    "s_waitcnt lgkmcnt(6)\n\t" \
    "v_mfma_f32_16x16x32_f16 a[232:235], v[48:51], v[156:159], a[232:235]\n\t" \
    "ds_write_b64 %[vWr0], v[230:231] offset:33800\n\t" \
    "v_mfma_f32_16x16x32_f16 a[200:203], v[44:47], v[156:159], a[200:203]\n\t" \
    "buffer_load_dwordx4 v[104:107], %[vA0], %[rWS], s55 offen offset:1024 sc0 sc1\n\t" \
    "v_mfma_f32_16x16x32_f16 a[168:171], v[40:43], v[156:159], a[168:171]\n\t" \
    "v_cvt_pk_f16_f32 v228, v166, v170\n\t" \
    "v_cvt_pk_f16_f32 v229, v174, v178\n\t" \
    "v_mfma_f32_16x16x32_f16 a[136:139], v[36:39], v[156:159], a[136:139]\n\t" \
    "ds_write_b64 %[vWr0], v[228:229] offset:34816\n\t" \
    "v_mfma_f32_16x16x32_f16 a[104:107], v[32:35], v[156:159], a[104:107]\n\t" \
    "v_cvt_pk_f16_f32 v230, v182, v186\n\t" \
    "v_mfma_f32_16x16x32_f16 a[72:75], v[28:31], v[156:159], a[72:75]\n\t" \
    "buffer_load_dwordx4 v[108:111], %[vA0], %[rWS], s55 offen offset:2048 sc0 sc1\n\t" \
    "v_mfma_f32_16x16x32_f16 a[40:43], v[24:27], v[156:159], a[40:43]\n\t" \
    "s_add_u32 s66, s66, 0x400000\n\t" \
    "s_and_b32 s66, s66, 0xfffffff\n\t" \
    "v_mfma_f32_16x16x32_f16 a[8:11], v[20:23], v[156:159], a[8:11]\n\t" \
    "ds_read_b128 v[156:159], %[vA0] offset:6144\n\t" \
    "s_waitcnt lgkmcnt(8)\n\t" \
    "v_mfma_f32_16x16x32_f16 a[12:15], v[20:23], v[160:163], a[12:15]\n\t" \
    "v_cvt_pk_f16_f32 v231, v190, v194\n\t" \
    "v_mfma_f32_16x16x32_f16 a[44:47], v[24:27], v[160:163], a[44:47]\n\t" \
    "ds_write_b64 %[vWr0], v[230:231] offset:34824\n\t" \
    "v_mfma_f32_16x16x32_f16 a[76:79], v[28:31], v[160:163], a[76:79]\n\t" \
    "v_cvt_pk_f16_f32 v228, v167, v171\n\t" \
    "s_add_u32 s39, s66, s44\n\t" \
    "v_mfma_f32_16x16x32_f16 a[108:111], v[32:35], v[160:163], a[108:111]\n\t" \
    "s_add_u32 s39, s39, %[sWk0]\n\t" \
    "v_mfma_f32_16x16x32_f16 a[140:143], v[36:39], v[160:163], a[140:143]\n\t" \
    "buffer_load_dwordx4 v[112:115], %[vA0], %[rWS], s55 offen offset:3072 sc0 sc1\n\t" \
    "v_mfma_f32_16x16x32_f16 a[172:175], v[40:43], v[160:163], a[172:175]\n\t" \
    "v_cvt_pk_f16_f32 v229, v175, v179\n\t" \
    "v_mfma_f32_16x16x32_f16 a[204:207], v[44:47], v[160:163], a[204:207]\n\t" \
    "ds_write_b64 %[vWr0], v[228:229] offset:35840\n\t" \
    "v_mfma_f32_16x16x32_f16 a[236:239], v[48:51], v[160:163], a[236:239]\n\t" \
    "ds_read_b128 v[160:163], %[vA0] offset:7168\n\t" \
    "s_waitcnt lgkmcnt(9)\n\t" \
    "v_mfma_f32_16x16x32_f16 a[240:243], v[48:51], v[148:151], a[240:243]\n\t" \
    "v_cvt_pk_f16_f32 v230, v183, v187\n\t" \
    "v_cvt_pk_f16_f32 v231, v191, v195\n\t" \
    "v_mfma_f32_16x16x32_f16 a[208:211], v[44:47], v[148:151], a[208:211]\n\t" \
    "ds_write_b64 %[vWr0], v[230:231] offset:35848\n\t" \
    "v_mfma_f32_16x16x32_f16 a[176:179], v[40:43], v[148:151], a[176:179]\n\t" \
    "buffer_load_dwordx4 v[164:167], %[vW], %[rW], s39 offen\n\t" \
    "v_mfma_f32_16x16x32_f16 a[144:147], v[36:39], v[148:151], a[144:147]\n\t" \
    "s_add_u32 s38, s38, 32768\n\t" \
    "s_and_b32 s38, s38, 0x1fffff\n\t" \
    "v_mfma_f32_16x16x32_f16 a[112:115], v[32:35], v[148:151], a[112:115]\n\t" \
    "s_or_b32 s38, s38, %[sA0]\n\t" \
    "s_add_u32 s39, s39, 32768\n\t" \
    "v_mfma_f32_16x16x32_f16 a[80:83], v[28:31], v[148:151], a[80:83]\n\t" \
    "buffer_load_dwordx4 v[168:171], %[vW], %[rW], s39 offen\n\t" \
    "v_mfma_f32_16x16x32_f16 a[48:51], v[24:27], v[148:151], a[48:51]\n\t" \
    "v_mfma_f32_16x16x32_f16 a[16:19], v[20:23], v[148:151], a[16:19]\n\t" \
    "ds_read_b128 v[148:151], %[vA0] offset:8192\n\t" \
    "s_waitcnt lgkmcnt(8)\n\t" \
    "v_mfma_f32_16x16x32_f16 a[20:23], v[20:23], v[152:155], a[20:23]\n\t" \
    "s_add_u32 s39, s39, 32768\n\t" \
    "v_mfma_f32_16x16x32_f16 a[52:55], v[24:27], v[152:155], a[52:55]\n\t" \
    "buffer_load_dwordx4 v[172:175], %[vW], %[rW], s39 offen\n\t" \
    "v_mfma_f32_16x16x32_f16 a[84:87], v[28:31], v[152:155], a[84:87]\n\t" \
    "v_mfma_f32_16x16x32_f16 a[116:119], v[32:35], v[152:155], a[116:119]\n\t" \
    "s_add_u32 s39, s39, 32768\n\t" \
    "v_mfma_f32_16x16x32_f16 a[148:151], v[36:39], v[152:155], a[148:151]\n\t" \
    "v_mfma_f32_16x16x32_f16 a[180:183], v[40:43], v[152:155], a[180:183]\n\t" \
    "buffer_load_dwordx4 v[176:179], %[vW], %[rW], s39 offen\n\t" \
    "v_mfma_f32_16x16x32_f16 a[212:215], v[44:47], v[152:155], a[212:215]\n\t" \
    "v_mfma_f32_16x16x32_f16 a[244:247], v[48:51], v[152:155], a[244:247]\n\t" \
    "ds_read_b128 v[152:155], %[vA0] offset:9216\n\t" \
    "s_waitcnt lgkmcnt(6)\n\t" \
    "v_mfma_f32_16x16x32_f16 a[248:251], v[48:51], v[156:159], a[248:251]\n\t" \
    "s_add_u32 s39, s39, 32768\n\t" \
    "v_mfma_f32_16x16x32_f16 a[216:219], v[44:47], v[156:159], a[216:219]\n\t" \
    "buffer_store_dwordx4 v[232:235], %[vO], s[56:59], 0 offen sc0 sc1 nt\n\t" \
    "v_mfma_f32_16x16x32_f16 a[184:187], v[40:43], v[156:159], a[184:187]\n\t" \
    "buffer_load_dwordx4 v[180:183], %[vW], %[rW], s39 offen\n\t" \
    "v_mfma_f32_16x16x32_f16 a[152:155], v[36:39], v[156:159], a[152:155]\n\t" \
    "s_add_u32 s39, s39, 32768\n\t" \
    "v_mfma_f32_16x16x32_f16 a[120:123], v[32:35], v[156:159], a[120:123]\n\t" \
    "v_mfma_f32_16x16x32_f16 a[88:91], v[28:31], v[156:159], a[88:91]\n\t" \
    "buffer_load_dwordx4 v[184:187], %[vW], %[rW], s39 offen\n\t" \
    "v_mfma_f32_16x16x32_f16 a[56:59], v[24:27], v[156:159], a[56:59]\n\t" \
    "v_mfma_f32_16x16x32_f16 a[24:27], v[20:23], v[156:159], a[24:27]\n\t" \
    "ds_read_b128 v[156:159], %[vA0] offset:10240\n\t" \
    "s_waitcnt lgkmcnt(4)\n\t" \
    "v_mfma_f32_16x16x32_f16 a[28:31], v[20:23], v[160:163], a[28:31]\n\t" \
    "s_add_u32 s39, s39, 32768\n\t" \
    "v_mfma_f32_16x16x32_f16 a[60:63], v[24:27], v[160:163], a[60:63]\n\t" \
    "buffer_load_dwordx4 v[188:191], %[vW], %[rW], s39 offen\n\t" \
    "v_mfma_f32_16x16x32_f16 a[92:95], v[28:31], v[160:163], a[92:95]\n\t" \
    "v_mfma_f32_16x16x32_f16 a[124:127], v[32:35], v[160:163], a[124:127]\n\t" \
    "s_add_u32 s39, s39, 32768\n\t" \
    "v_mfma_f32_16x16x32_f16 a[156:159], v[36:39], v[160:163], a[156:159]\n\t" \
    "v_mfma_f32_16x16x32_f16 a[188:191], v[40:43], v[160:163], a[188:191]\n\t" \
    "buffer_load_dwordx4 v[192:195], %[vW], %[rW], s39 offen\n\t" \
    "v_mfma_f32_16x16x32_f16 a[220:223], v[44:47], v[160:163], a[220:223]\n\t" \
    "v_mfma_f32_16x16x32_f16 a[252:255], v[48:51], v[160:163], a[252:255]\n\t" \
    "ds_read_b128 v[160:163], %[vA0] offset:11264\n\t" \
    "s_waitcnt vmcnt(17) lgkmcnt(3)\n\t" \
    "v_mfma_f32_16x16x32_f16 a[224:227], v[80:83], v[148:151], a[224:227]\n\t" \
    "v_cvt_pk_f16_f32 v228, v196, v200\n\t" \
    "v_mfma_f32_16x16x32_f16 a[192:195], v[76:79], v[148:151], a[192:195]\n\t" \
    "v_mfma_f32_16x16x32_f16 a[160:163], v[72:75], v[148:151], a[160:163]\n\t" \
    "v_cvt_pk_f16_f32 v229, v204, v208\n\t" \
    "v_mfma_f32_16x16x32_f16 a[128:131], v[68:71], v[148:151], a[128:131]\n\t" \
    "ds_write_b64 %[vWr1], v[228:229] offset:32768\n\t" \
    "v_mfma_f32_16x16x32_f16 a[96:99], v[64:67], v[148:151], a[96:99]\n\t" \
    "v_cvt_pk_f16_f32 v230, v212, v216\n\t" \
    "v_mfma_f32_16x16x32_f16 a[64:67], v[60:63], v[148:151], a[64:67]\n\t" \
    "v_mfma_f32_16x16x32_f16 a[32:35], v[56:59], v[148:151], a[32:35]\n\t" \
    "v_cvt_pk_f16_f32 v231, v220, v224\n\t" \
    "v_mfma_f32_16x16x32_f16 a[0:3], v[52:55], v[148:151], a[0:3]\n\t" \
    "ds_read_b128 v[148:151], %[vA0] offset:12288\n\t" \
    "s_waitcnt lgkmcnt(4)\n\t" \
    "v_mfma_f32_16x16x32_f16 a[4:7], v[52:55], v[152:155], a[4:7]\n\t" \
    "ds_write_b64 %[vWr1], v[230:231] offset:32776\n\t" \
    "v_mfma_f32_16x16x32_f16 a[36:39], v[56:59], v[152:155], a[36:39]\n\t" \
    "v_cvt_pk_f16_f32 v228, v197, v201\n\t" \
    "v_mfma_f32_16x16x32_f16 a[68:71], v[60:63], v[152:155], a[68:71]\n\t" \
    "v_cvt_pk_f16_f32 v229, v205, v209\n\t" \
    "v_mfma_f32_16x16x32_f16 a[100:103], v[64:67], v[152:155], a[100:103]\n\t" \
    "ds_write_b64 %[vWr1], v[228:229] offset:33792\n\t" \
    "v_mfma_f32_16x16x32_f16 a[132:135], v[68:71], v[152:155], a[132:135]\n\t" \
    "v_cvt_pk_f16_f32 v230, v213, v217\n\t" \
    "v_mfma_f32_16x16x32_f16 a[164:167], v[72:75], v[152:155], a[164:167]\n\t" \
    "v_mfma_f32_16x16x32_f16 a[196:199], v[76:79], v[152:155], a[196:199]\n\t" \
    "v_cvt_pk_f16_f32 v231, v221, v225\n\t" \
    "v_mfma_f32_16x16x32_f16 a[228:231], v[80:83], v[152:155], a[228:231]\n\t" \
    "ds_read_b128 v[152:155], %[vA0] offset:13312\n\t" \
    "s_waitcnt lgkmcnt(6)\n\t" \
    "v_mfma_f32_16x16x32_f16 a[232:235], v[80:83], v[156:159], a[232:235]\n\t" \
    "s_add_u32 s55, s37, 24576\n\t" \
    "v_mfma_f32_16x16x32_f16 a[200:203], v[76:79], v[156:159], a[200:203]\n\t" \
    "ds_write_b64 %[vWr1], v[230:231] offset:33800\n\t" \
    "v_mfma_f32_16x16x32_f16 a[168:171], v[72:75], v[156:159], a[168:171]\n\t" \
    "v_cvt_pk_f16_f32 v228, v198, v202\n\t" \
    "v_mfma_f32_16x16x32_f16 a[136:139], v[68:71], v[156:159], a[136:139]\n\t" \
    "buffer_load_dwordx4 v[116:119], %[vA0], %[rWS], s55 offen offset:0 sc0 sc1\n\t" \
    "v_mfma_f32_16x16x32_f16 a[104:107], v[64:67], v[156:159], a[104:107]\n\t" \
    "v_cvt_pk_f16_f32 v229, v206, v210\n\t" \
    "v_mfma_f32_16x16x32_f16 a[72:75], v[60:63], v[156:159], a[72:75]\n\t" \
    "ds_write_b64 %[vWr1], v[228:229] offset:34816\n\t" \
    "v_mfma_f32_16x16x32_f16 a[40:43], v[56:59], v[156:159], a[40:43]\n\t" \
    "v_cvt_pk_f16_f32 v230, v214, v218\n\t" \
    "v_mfma_f32_16x16x32_f16 a[8:11], v[52:55], v[156:159], a[8:11]\n\t" \
    "ds_read_b128 v[156:159], %[vA0] offset:14336\n\t" \
    "s_waitcnt lgkmcnt(8)\n\t" \
    "v_mfma_f32_16x16x32_f16 a[12:15], v[52:55], v[160:163], a[12:15]\n\t" \
    "buffer_load_dwordx4 v[120:123], %[vA0], %[rWS], s55 offen offset:1024 sc0 sc1\n\t" \
    "v_mfma_f32_16x16x32_f16 a[44:47], v[56:59], v[160:163], a[44:47]\n\t" \
    "v_cvt_pk_f16_f32 v231, v222, v226\n\t" \
    "v_mfma_f32_16x16x32_f16 a[76:79], v[60:63], v[160:163], a[76:79]\n\t" \
    "ds_write_b64 %[vWr1], v[230:231] offset:34824\n\t" \
    "v_mfma_f32_16x16x32_f16 a[108:111], v[64:67], v[160:163], a[108:111]\n\t" \
    "v_cvt_pk_f16_f32 v228, v199, v203\n\t" \
    "v_cvt_pk_f16_f32 v229, v207, v211\n\t" \
    "v_mfma_f32_16x16x32_f16 a[140:143], v[68:71], v[160:163], a[140:143]\n\t" \
    "buffer_load_dwordx4 v[124:127], %[vA0], %[rWS], s55 offen offset:2048 sc0 sc1\n\t" \
    "v_mfma_f32_16x16x32_f16 a[172:175], v[72:75], v[160:163], a[172:175]\n\t" \
    "ds_write_b64 %[vWr1], v[228:229] offset:35840\n\t" \
    "v_mfma_f32_16x16x32_f16 a[204:207], v[76:79], v[160:163], a[204:207]\n\t" \
    "v_cvt_pk_f16_f32 v230, v215, v219\n\t" \
    "v_cvt_pk_f16_f32 v231, v223, v227\n\t" \
    "v_mfma_f32_16x16x32_f16 a[236:239], v[80:83], v[160:163], a[236:239]\n\t" \
    "ds_read_b128 v[160:163], %[vA0] offset:15360\n\t" \
    "s_waitcnt lgkmcnt(9)\n\t" \
    "v_mfma_f32_16x16x32_f16 a[240:243], v[80:83], v[148:151], a[240:243]\n\t" \
    "ds_write_b64 %[vWr1], v[230:231] offset:35848\n\t" \
    "v_mfma_f32_16x16x32_f16 a[208:211], v[76:79], v[148:151], a[208:211]\n\t" \
    "buffer_load_dwordx4 v[128:131], %[vA0], %[rWS], s55 offen offset:3072 sc0 sc1\n\t" \
    "v_mfma_f32_16x16x32_f16 a[176:179], v[72:75], v[148:151], a[176:179]\n\t" \
    "v_mfma_f32_16x16x32_f16 a[144:147], v[68:71], v[148:151], a[144:147]\n\t" \
    "s_add_u32 s55, s37, 28672\n\t" \
    "v_mfma_f32_16x16x32_f16 a[112:115], v[64:67], v[148:151], a[112:115]\n\t" \
    "v_mfma_f32_16x16x32_f16 a[80:83], v[60:63], v[148:151], a[80:83]\n\t" \
    "buffer_load_dwordx4 v[132:135], %[vA0], %[rWS], s55 offen offset:0 sc0 sc1\n\t" \
    "v_mfma_f32_16x16x32_f16 a[48:51], v[56:59], v[148:151], a[48:51]\n\t" \
    "v_mfma_f32_16x16x32_f16 a[16:19], v[52:55], v[148:151], a[16:19]\n\t" \
    "ds_read_b128 v[148:151], %[vA0] offset:16384\n\t" \
    "s_waitcnt lgkmcnt(8)\n\t" \
    "v_mfma_f32_16x16x32_f16 a[20:23], v[52:55], v[152:155], a[20:23]\n\t" \
    "v_mfma_f32_16x16x32_f16 a[52:55], v[56:59], v[152:155], a[52:55]\n\t" \
    "v_mfma_f32_16x16x32_f16 a[84:87], v[60:63], v[152:155], a[84:87]\n\t" \
    "buffer_load_dwordx4 v[136:139], %[vA0], %[rWS], s55 offen offset:1024 sc0 sc1\n\t" \
    "v_mfma_f32_16x16x32_f16 a[116:119], v[64:67], v[152:155], a[116:119]\n\t" \
    "v_mfma_f32_16x16x32_f16 a[148:151], v[68:71], v[152:155], a[148:151]\n\t" \
    "v_mfma_f32_16x16x32_f16 a[180:183], v[72:75], v[152:155], a[180:183]\n\t" \
    "v_mfma_f32_16x16x32_f16 a[212:215], v[76:79], v[152:155], a[212:215]\n\t" \
    "v_mfma_f32_16x16x32_f16 a[244:247], v[80:83], v[152:155], a[244:247]\n\t" \
    "ds_read_b128 v[152:155], %[vA0] offset:17408\n\t" \
    "s_waitcnt lgkmcnt(6)\n\t" \
    "v_mfma_f32_16x16x32_f16 a[248:251], v[80:83], v[156:159], a[248:251]\n\t" \
    "buffer_load_dwordx4 v[140:143], %[vA0], %[rWS], s55 offen offset:2048 sc0 sc1\n\t" \
    "v_mfma_f32_16x16x32_f16 a[216:219], v[76:79], v[156:159], a[216:219]\n\t" \
    "v_mfma_f32_16x16x32_f16 a[184:187], v[72:75], v[156:159], a[184:187]\n\t" \
    "v_mfma_f32_16x16x32_f16 a[152:155], v[68:71], v[156:159], a[152:155]\n\t" \
    "v_mfma_f32_16x16x32_f16 a[120:123], v[64:67], v[156:159], a[120:123]\n\t" \
    "v_mfma_f32_16x16x32_f16 a[88:91], v[60:63], v[156:159], a[88:91]\n\t" \
    "buffer_load_dwordx4 v[144:147], %[vA0], %[rWS], s55 offen offset:3072 sc0 sc1\n\t" \
    "v_mfma_f32_16x16x32_f16 a[56:59], v[56:59], v[156:159], a[56:59]\n\t" \
    "v_mfma_f32_16x16x32_f16 a[24:27], v[52:55], v[156:159], a[24:27]\n\t" \
    "ds_read_b128 v[156:159], %[vA0] offset:18432\n\t" \
    "s_waitcnt lgkmcnt(4)\n\t" \
    "v_mfma_f32_16x16x32_f16 a[28:31], v[52:55], v[160:163], a[28:31]\n\t" \
    "s_add_u32 s55, s38, 0\n\t" \
    "v_mfma_f32_16x16x32_f16 a[60:63], v[56:59], v[160:163], a[60:63]\n\t" \
    "v_mfma_f32_16x16x32_f16 a[92:95], v[60:63], v[160:163], a[92:95]\n\t" \
    "buffer_load_dwordx4 v[20:23], %[vA0], %[rWS], s55 offen offset:0 sc0 sc1\n\t" \
    "v_mfma_f32_16x16x32_f16 a[124:127], v[64:67], v[160:163], a[124:127]\n\t" \
    "v_mfma_f32_16x16x32_f16 a[156:159], v[68:71], v[160:163], a[156:159]\n\t" \
    "v_mfma_f32_16x16x32_f16 a[188:191], v[72:75], v[160:163], a[188:191]\n\t" \
    "v_mfma_f32_16x16x32_f16 a[220:223], v[76:79], v[160:163], a[220:223]\n\t" \
    "v_mfma_f32_16x16x32_f16 a[252:255], v[80:83], v[160:163], a[252:255]\n\t" \
    "ds_read_b128 v[160:163], %[vA0] offset:19456\n\t" \
    "s_waitcnt vmcnt(18) lgkmcnt(3)\n\t" \
    "v_mfma_f32_16x16x32_f16 a[224:227], v[112:115], v[148:151], a[224:227]\n\t" \
    "buffer_load_dwordx4 v[24:27], %[vA0], %[rWS], s55 offen offset:1024 sc0 sc1\n\t" \
    "v_mfma_f32_16x16x32_f16 a[192:195], v[108:111], v[148:151], a[192:195]\n\t" \
    "v_mfma_f32_16x16x32_f16 a[160:163], v[104:107], v[148:151], a[160:163]\n\t" \
    "v_mfma_f32_16x16x32_f16 a[128:131], v[100:103], v[148:151], a[128:131]\n\t" \
    "v_mfma_f32_16x16x32_f16 a[96:99], v[96:99], v[148:151], a[96:99]\n\t" \
    "buffer_load_dwordx4 v[28:31], %[vA0], %[rWS], s55 offen offset:2048 sc0 sc1\n\t" \
    "v_mfma_f32_16x16x32_f16 a[64:67], v[92:95], v[148:151], a[64:67]\n\t" \
    "v_mfma_f32_16x16x32_f16 a[32:35], v[88:91], v[148:151], a[32:35]\n\t" \
    "v_mfma_f32_16x16x32_f16 a[0:3], v[84:87], v[148:151], a[0:3]\n\t" \
    "ds_read_b128 v[148:151], %[vA0] offset:20480\n\t" \
    "s_waitcnt lgkmcnt(3)\n\t" \
    "v_mfma_f32_16x16x32_f16 a[4:7], v[84:87], v[152:155], a[4:7]\n\t" \
    "v_mfma_f32_16x16x32_f16 a[36:39], v[88:91], v[152:155], a[36:39]\n\t" \
    "buffer_load_dwordx4 v[32:35], %[vA0], %[rWS], s55 offen offset:3072 sc0 sc1\n\t" \
    "v_mfma_f32_16x16x32_f16 a[68:71], v[92:95], v[152:155], a[68:71]\n\t" \
    "v_mfma_f32_16x16x32_f16 a[100:103], v[96:99], v[152:155], a[100:103]\n\t" \
    "v_mfma_f32_16x16x32_f16 a[132:135], v[100:103], v[152:155], a[132:135]\n\t" \
    "v_mfma_f32_16x16x32_f16 a[164:167], v[104:107], v[152:155], a[164:167]\n\t" \
    "s_add_u32 s55, s38, 4096\n\t" \
    "v_mfma_f32_16x16x32_f16 a[196:199], v[108:111], v[152:155], a[196:199]\n\t" \
    "v_mfma_f32_16x16x32_f16 a[228:231], v[112:115], v[152:155], a[228:231]\n\t" \
    "ds_read_b128 v[152:155], %[vA0] offset:21504\n\t" \
    "s_waitcnt lgkmcnt(3)\n\t" \
    "v_mfma_f32_16x16x32_f16 a[232:235], v[112:115], v[156:159], a[232:235]\n\t" \
    "buffer_load_dwordx4 v[36:39], %[vA0], %[rWS], s55 offen offset:0 sc0 sc1\n\t" \
    "v_mfma_f32_16x16x32_f16 a[200:203], v[108:111], v[156:159], a[200:203]\n\t" \
    "v_mfma_f32_16x16x32_f16 a[168:171], v[104:107], v[156:159], a[168:171]\n\t" \
    "v_mfma_f32_16x16x32_f16 a[136:139], v[100:103], v[156:159], a[136:139]\n\t" \
    "v_mfma_f32_16x16x32_f16 a[104:107], v[96:99], v[156:159], a[104:107]\n\t" \
    "buffer_load_dwordx4 v[40:43], %[vA0], %[rWS], s55 offen offset:1024 sc0 sc1\n\t" \
    "v_mfma_f32_16x16x32_f16 a[72:75], v[92:95], v[156:159], a[72:75]\n\t" \
    "v_readlane_b32 s52, %[vTab0], s51\n\t" \
    "v_mfma_f32_16x16x32_f16 a[40:43], v[88:91], v[156:159], a[40:43]\n\t" \
    "s_add_u32 s42, s51, 32\n\t" \
    "v_mfma_f32_16x16x32_f16 a[8:11], v[84:87], v[156:159], a[8:11]\n\t" \
    "ds_read_b128 v[156:159], %[vA0] offset:22528\n\t" \
    "s_waitcnt lgkmcnt(3)\n\t" \
    "v_mfma_f32_16x16x32_f16 a[12:15], v[84:87], v[160:163], a[12:15]\n\t" \
    "v_readlane_b32 s53, %[vTab1], s51\n\t" \
    "v_mfma_f32_16x16x32_f16 a[44:47], v[88:91], v[160:163], a[44:47]\n\t" \
    "v_readlane_b32 s42, %[vTab0], s42\n\t" \
    "v_mfma_f32_16x16x32_f16 a[76:79], v[92:95], v[160:163], a[76:79]\n\t" \
    "s_add_u32 s51, s51, 1\n\t" \
    "v_mfma_f32_16x16x32_f16 a[108:111], v[96:99], v[160:163], a[108:111]\n\t" \
    "buffer_load_dwordx4 v[44:47], %[vA0], %[rWS], s55 offen offset:2048 sc0 sc1\n\t" \
    "v_mfma_f32_16x16x32_f16 a[140:143], v[100:103], v[160:163], a[140:143]\n\t" \
    "s_add_u32 s40, s66, s44\n\t" \
    "s_add_u32 s40, s40, %[sWk1]\n\t" \
    "v_mfma_f32_16x16x32_f16 a[172:175], v[104:107], v[160:163], a[172:175]\n\t" \
    "s_mov_b32 s56, s42\n\t" \
    "s_and_b32 s57, s53, 0xffff\n\t" \
    "v_mfma_f32_16x16x32_f16 a[204:207], v[108:111], v[160:163], a[204:207]\n\t" \
    "s_and_b32 s58, s53, 0x10000\n\t" \
    "s_bfe_i32 s53, s53, 0x10011\n\t" \
    "v_mfma_f32_16x16x32_f16 a[236:239], v[112:115], v[160:163], a[236:239]\n\t" \
    "ds_read_b128 v[160:163], %[vA0] offset:23552\n\t" \
    "s_waitcnt lgkmcnt(3)\n\t" \
    "v_mfma_f32_16x16x32_f16 a[240:243], v[112:115], v[148:151], a[240:243]\n\t" \
    "buffer_load_dwordx4 v[48:51], %[vA0], %[rWS], s55 offen offset:3072 sc0 sc1\n\t" \
    "v_mfma_f32_16x16x32_f16 a[208:211], v[108:111], v[148:151], a[208:211]\n\t" \
    "v_mfma_f32_16x16x32_f16 a[176:179], v[104:107], v[148:151], a[176:179]\n\t" \
    "v_mfma_f32_16x16x32_f16 a[144:147], v[100:103], v[148:151], a[144:147]\n\t" \
    "buffer_load_dwordx4 v[196:199], %[vW], %[rW], s40 offen nt\n\t" \
    "v_mfma_f32_16x16x32_f16 a[112:115], v[96:99], v[148:151], a[112:115]\n\t" \
    "v_mfma_f32_16x16x32_f16 a[80:83], v[92:95], v[148:151], a[80:83]\n\t" \
    "v_mfma_f32_16x16x32_f16 a[48:51], v[88:91], v[148:151], a[48:51]\n\t" \
    "s_add_u32 s40, s40, 32768\n\t" \
    "v_mfma_f32_16x16x32_f16 a[16:19], v[84:87], v[148:151], a[16:19]\n\t" \
    "ds_read_b128 v[148:151], %[vA0] offset:24576\n\t" \
    "s_waitcnt lgkmcnt(3)\n\t" \
    "v_mfma_f32_16x16x32_f16 a[20:23], v[84:87], v[152:155], a[20:23]\n\t" \
    "s_cmp_eq_u32 s69, 0\n\t" \
    "s_cselect_b32 s42, 1073741824, s52\n\t" \
    "v_mfma_f32_16x16x32_f16 a[52:55], v[88:91], v[152:155], a[52:55]\n\t" \
    "buffer_load_dwordx4 v[200:203], %[vW], %[rW], s40 offen nt\n\t" \
    "v_mfma_f32_16x16x32_f16 a[84:87], v[92:95], v[152:155], a[84:87]\n\t" \
    "buffer_load_dwordx4 v[236:239], %[vA0], %[rWS], s42 offen sc0 sc1\n\t" \
    "v_mfma_f32_16x16x32_f16 a[116:119], v[96:99], v[152:155], a[116:119]\n\t" \
    "s_add_u32 s52, s52, 262144\n\t" \
    "s_cmp_eq_u32 s69, 1\n\t" \
    "v_mfma_f32_16x16x32_f16 a[148:151], v[100:103], v[152:155], a[148:151]\n\t" \
    "s_cselect_b32 s42, 1073741824, s52\n\t" \
    "v_mfma_f32_16x16x32_f16 a[180:183], v[104:107], v[152:155], a[180:183]\n\t" \
    "buffer_load_dwordx4 v[240:243], %[vA0], %[rWS], s42 offen sc0 sc1\n\t" \
    "v_mfma_f32_16x16x32_f16 a[212:215], v[108:111], v[152:155], a[212:215]\n\t" \
    "s_add_u32 s40, s40, 32768\n\t" \
    "s_add_u32 s52, s52, 262144\n\t" \
    "v_mfma_f32_16x16x32_f16 a[244:247], v[112:115], v[152:155], a[244:247]\n\t" \
    "ds_read_b128 v[152:155], %[vA0] offset:25600\n\t" \
    "s_waitcnt lgkmcnt(3)\n\t" \
    "v_mfma_f32_16x16x32_f16 a[248:251], v[112:115], v[156:159], a[248:251]\n\t" \
    "s_cmp_eq_u32 s69, 2\n\t" \
    "s_cselect_b32 s42, 1073741824, s52\n\t" \
    "v_mfma_f32_16x16x32_f16 a[216:219], v[108:111], v[156:159], a[216:219]\n\t" \
    "buffer_load_dwordx4 v[204:207], %[vW], %[rW], s40 offen nt\n\t" \
    "v_mfma_f32_16x16x32_f16 a[184:187], v[104:107], v[156:159], a[184:187]\n\t" \
    "buffer_load_dwordx4 v[244:247], %[vA0], %[rWS], s42 offen sc0 sc1\n\t" \
    "v_mfma_f32_16x16x32_f16 a[152:155], v[100:103], v[156:159], a[152:155]\n\t" \
    "s_add_u32 s52, s52, 262144\n\t" \
    "s_cmp_eq_u32 s69, 3\n\t" \
    "v_mfma_f32_16x16x32_f16 a[120:123], v[96:99], v[156:159], a[120:123]\n\t" \
    "s_cselect_b32 s42, 1073741824, s52\n\t" \
    "s_add_u32 s40, s40, 32768\n\t" \
    "v_mfma_f32_16x16x32_f16 a[88:91], v[92:95], v[156:159], a[88:91]\n\t" \
    "buffer_load_dwordx4 v[248:251], %[vA0], %[rWS], s42 offen sc0 sc1\n\t" \
    "v_mfma_f32_16x16x32_f16 a[56:59], v[88:91], v[156:159], a[56:59]\n\t" \
    "buffer_load_dwordx4 v[208:211], %[vW], %[rW], s40 offen nt\n\t" \
    "v_mfma_f32_16x16x32_f16 a[24:27], v[84:87], v[156:159], a[24:27]\n\t" \
    "ds_read_b128 v[156:159], %[vA0] offset:26624\n\t" \
    "s_waitcnt lgkmcnt(3)\n\t" \
    "v_mfma_f32_16x16x32_f16 a[28:31], v[84:87], v[160:163], a[28:31]\n\t" \
    "s_add_u32 s40, s40, 32768\n\t" \
    "v_mfma_f32_16x16x32_f16 a[60:63], v[88:91], v[160:163], a[60:63]\n\t" \
    "buffer_load_dwordx4 v[212:215], %[vW], %[rW], s40 offen nt\n\t" \
    "v_mfma_f32_16x16x32_f16 a[92:95], v[92:95], v[160:163], a[92:95]\n\t" \
    "v_mfma_f32_16x16x32_f16 a[124:127], v[96:99], v[160:163], a[124:127]\n\t" \
    "s_add_u32 s40, s40, 32768\n\t" \
    "v_mfma_f32_16x16x32_f16 a[156:159], v[100:103], v[160:163], a[156:159]\n\t" \
    "v_mfma_f32_16x16x32_f16 a[188:191], v[104:107], v[160:163], a[188:191]\n\t" \
    "buffer_load_dwordx4 v[216:219], %[vW], %[rW], s40 offen nt\n\t" \
    "v_mfma_f32_16x16x32_f16 a[220:223], v[108:111], v[160:163], a[220:223]\n\t" \
    "v_mfma_f32_16x16x32_f16 a[252:255], v[112:115], v[160:163], a[252:255]\n\t" \
    "ds_read_b128 v[160:163], %[vA0] offset:27648\n\t" \
    "s_waitcnt vmcnt(18) lgkmcnt(3)\n\t" \
    "v_mfma_f32_16x16x32_f16 a[224:227], v[144:147], v[148:151], a[224:227]\n\t" \
    "s_add_u32 s40, s40, 32768\n\t" \
    "v_mfma_f32_16x16x32_f16 a[192:195], v[140:143], v[148:151], a[192:195]\n\t" \
    "v_mfma_f32_16x16x32_f16 a[160:163], v[136:139], v[148:151], a[160:163]\n\t" \
    "buffer_load_dwordx4 v[220:223], %[vW], %[rW], s40 offen nt\n\t" \
    "v_mfma_f32_16x16x32_f16 a[128:131], v[132:135], v[148:151], a[128:131]\n\t" \
    "v_mfma_f32_16x16x32_f16 a[96:99], v[128:131], v[148:151], a[96:99]\n\t" \
    "v_mfma_f32_16x16x32_f16 a[64:67], v[124:127], v[148:151], a[64:67]\n\t" \
    "v_mfma_f32_16x16x32_f16 a[32:35], v[120:123], v[148:151], a[32:35]\n\t" \
    "s_add_u32 s40, s40, 32768\n\t" \
    "v_mfma_f32_16x16x32_f16 a[0:3], v[116:119], v[148:151], a[0:3]\n\t" \
    "ds_read_b128 v[148:151], %[vA0] offset:28672\n\t" \
    "s_waitcnt lgkmcnt(3)\n\t" \
    "v_mfma_f32_16x16x32_f16 a[4:7], v[116:119], v[152:155], a[4:7]\n\t" \
    "buffer_load_dwordx4 v[224:227], %[vW], %[rW], s40 offen nt\n\t" \
    "v_mfma_f32_16x16x32_f16 a[36:39], v[120:123], v[152:155], a[36:39]\n\t" \
    "v_mfma_f32_16x16x32_f16 a[68:71], v[124:127], v[152:155], a[68:71]\n\t" \
    "v_mfma_f32_16x16x32_f16 a[100:103], v[128:131], v[152:155], a[100:103]\n\t" \
    "s_add_u32 s55, s38, 8192\n\t" \
    "v_mfma_f32_16x16x32_f16 a[132:135], v[132:135], v[152:155], a[132:135]\n\t" \
    "v_and_b32 v252, s53, v252\n\t" \
    "v_mfma_f32_16x16x32_f16 a[164:167], v[136:139], v[152:155], a[164:167]\n\t" \
    "buffer_load_dwordx4 v[52:55], %[vA0], %[rWS], s55 offen offset:0 sc0 sc1\n\t" \
    "v_mfma_f32_16x16x32_f16 a[196:199], v[140:143], v[152:155], a[196:199]\n\t" \
    "v_and_b32 v253, s53, v253\n\t" \
    "v_and_b32 v254, s53, v254\n\t" \
    "v_mfma_f32_16x16x32_f16 a[228:231], v[144:147], v[152:155], a[228:231]\n\t" \
    "ds_read_b128 v[152:155], %[vA0] offset:29696\n\t" \
    "s_waitcnt lgkmcnt(3)\n\t" \
    "v_mfma_f32_16x16x32_f16 a[232:235], v[144:147], v[156:159], a[232:235]\n\t" \
    "v_and_b32 v255, s53, v255\n\t" \
    "s_waitcnt vmcnt(9)\n\t" \
    "v_add_f32 v236, v236, v240\n\t" \
    "v_mfma_f32_16x16x32_f16 a[200:203], v[140:143], v[156:159], a[200:203]\n\t" \
    "v_add_f32 v237, v237, v241\n\t" \
    "v_mfma_f32_16x16x32_f16 a[168:171], v[136:139], v[156:159], a[168:171]\n\t" \
    "buffer_load_dwordx4 v[56:59], %[vA0], %[rWS], s55 offen offset:1024 sc0 sc1\n\t" \
    "v_mfma_f32_16x16x32_f16 a[136:139], v[132:135], v[156:159], a[136:139]\n\t" \
    "v_add_f32 v238, v238, v242\n\t" \
    "v_add_f32 v239, v239, v243\n\t" \
    "v_mfma_f32_16x16x32_f16 a[104:107], v[128:131], v[156:159], a[104:107]\n\t" \
    "v_add_f32 v252, v252, v236\n\t" \
    "v_mfma_f32_16x16x32_f16 a[72:75], v[124:127], v[156:159], a[72:75]\n\t" \
    "v_add_f32 v253, v253, v237\n\t" \
    "v_mfma_f32_16x16x32_f16 a[40:43], v[120:123], v[156:159], a[40:43]\n\t" \
    "v_add_f32 v254, v254, v238\n\t" \
    "v_mfma_f32_16x16x32_f16 a[8:11], v[116:119], v[156:159], a[8:11]\n\t" \
    "ds_read_b128 v[156:159], %[vA0] offset:30720\n\t" \
    "s_waitcnt lgkmcnt(3)\n\t" \
    "v_mfma_f32_16x16x32_f16 a[12:15], v[116:119], v[160:163], a[12:15]\n\t" \
    "buffer_load_dwordx4 v[60:63], %[vA0], %[rWS], s55 offen offset:2048 sc0 sc1\n\t" \
    "v_mfma_f32_16x16x32_f16 a[44:47], v[120:123], v[160:163], a[44:47]\n\t" \
    "v_add_f32 v255, v255, v239\n\t" \
    "s_waitcnt vmcnt(8)\n\t" \
    "v_add_f32 v244, v244, v248\n\t" \
    "v_mfma_f32_16x16x32_f16 a[76:79], v[124:127], v[160:163], a[76:79]\n\t" \
    "v_add_f32 v245, v245, v249\n\t" \
    "v_add_f32 v246, v246, v250\n\t" \
    "v_mfma_f32_16x16x32_f16 a[108:111], v[128:131], v[160:163], a[108:111]\n\t" \
    "v_add_f32 v247, v247, v251\n\t" \
    "v_mfma_f32_16x16x32_f16 a[140:143], v[132:135], v[160:163], a[140:143]\n\t" \
    "buffer_load_dwordx4 v[64:67], %[vA0], %[rWS], s55 offen offset:3072 sc0 sc1\n\t" \
    "v_mfma_f32_16x16x32_f16 a[172:175], v[136:139], v[160:163], a[172:175]\n\t" \
    "v_add_f32 v252, v252, v244\n\t" \
    "v_add_f32 v253, v253, v245\n\t" \
    "v_mfma_f32_16x16x32_f16 a[204:207], v[140:143], v[160:163], a[204:207]\n\t" \
    "v_add_f32 v254, v254, v246\n\t" \
    "v_mfma_f32_16x16x32_f16 a[236:239], v[144:147], v[160:163], a[236:239]\n\t" \
    "ds_read_b128 v[160:163], %[vA0] offset:31744\n\t" \
    "s_waitcnt lgkmcnt(3)\n\t" \
    "v_mfma_f32_16x16x32_f16 a[240:243], v[144:147], v[148:151], a[240:243]\n\t" \
    "s_add_u32 s55, s38, 12288\n\t" \
    "v_add_f32 v255, v255, v247\n\t" \
    "v_mfma_f32_16x16x32_f16 a[208:211], v[140:143], v[148:151], a[208:211]\n\t" \
    "v_med3_f32 v236, v252, %[m60], %[p60]\n\t" \
    "v_mfma_f32_16x16x32_f16 a[176:179], v[136:139], v[148:151], a[176:179]\n\t" \
    "buffer_load_dwordx4 v[68:71], %[vA0], %[rWS], s55 offen offset:0 sc0 sc1\n\t" \
    "v_mfma_f32_16x16x32_f16 a[144:147], v[132:135], v[148:151], a[144:147]\n\t" \
    "v_med3_f32 v237, v253, %[m60], %[p60]\n\t" \
    "v_med3_f32 v238, v254, %[m60], %[p60]\n\t" \
    "v_mfma_f32_16x16x32_f16 a[112:115], v[128:131], v[148:151], a[112:115]\n\t" \
    "v_med3_f32 v239, v255, %[m60], %[p60]\n\t" \
    "v_mul_f32 v236, 0xbfb8aa3b, v236\n\t" \
    "v_mfma_f32_16x16x32_f16 a[80:83], v[124:127], v[148:151], a[80:83]\n\t" \
    "v_mul_f32 v237, 0xbfb8aa3b, v237\n\t" \
    "v_mfma_f32_16x16x32_f16 a[48:51], v[120:123], v[148:151], a[48:51]\n\t" \
    "v_mul_f32 v238, 0xbfb8aa3b, v238\n\t" \
    "v_mfma_f32_16x16x32_f16 a[16:19], v[116:119], v[148:151], a[16:19]\n\t" \
    "s_waitcnt lgkmcnt(0)\n\t" \
    "s_barrier\n\t" \
    "ds_read_b128 v[148:151], %[vA0] offset:32768\n\t" \
    "v_mfma_f32_16x16x32_f16 a[20:23], v[116:119], v[152:155], a[20:23]\n\t" \
    "buffer_load_dwordx4 v[72:75], %[vA0], %[rWS], s55 offen offset:1024 sc0 sc1\n\t" \
    "v_mfma_f32_16x16x32_f16 a[52:55], v[120:123], v[152:155], a[52:55]\n\t" \
    "v_mul_f32 v239, 0xbfb8aa3b, v239\n\t" \
    "v_mfma_f32_16x16x32_f16 a[84:87], v[124:127], v[152:155], a[84:87]\n\t" \
    "v_exp_f32 v236, v236\n\t" \
    "v_mfma_f32_16x16x32_f16 a[116:119], v[128:131], v[152:155], a[116:119]\n\t" \
    "v_exp_f32 v237, v237\n\t" \
    "v_mfma_f32_16x16x32_f16 a[148:151], v[132:135], v[152:155], a[148:151]\n\t" \
    "v_exp_f32 v238, v238\n\t" \
    "v_mfma_f32_16x16x32_f16 a[180:183], v[136:139], v[152:155], a[180:183]\n\t" \
    "v_exp_f32 v239, v239\n\t" \
    "v_mfma_f32_16x16x32_f16 a[212:215], v[140:143], v[152:155], a[212:215]\n\t" \
    "buffer_load_dwordx4 v[76:79], %[vA0], %[rWS], s55 offen offset:2048 sc0 sc1\n\t" \
    "v_mfma_f32_16x16x32_f16 a[244:247], v[144:147], v[152:155], a[244:247]\n\t" \
    "ds_read_b128 v[152:155], %[vA0] offset:33792\n\t" \
    "v_mfma_f32_16x16x32_f16 a[248:251], v[144:147], v[156:159], a[248:251]\n\t" \
    "v_add_f32 v236, 1.0, v236\n\t" \
    "v_add_f32 v237, 1.0, v237\n\t" \
    "v_mfma_f32_16x16x32_f16 a[216:219], v[140:143], v[156:159], a[216:219]\n\t" \
    "v_add_f32 v238, 1.0, v238\n\t" \
    "v_add_f32 v239, 1.0, v239\n\t" \
    "v_mfma_f32_16x16x32_f16 a[184:187], v[136:139], v[156:159], a[184:187]\n\t" \
    "v_rcp_f32 v236, v236\n\t" \
    "v_mfma_f32_16x16x32_f16 a[152:155], v[132:135], v[156:159], a[152:155]\n\t" \
    "buffer_load_dwordx4 v[80:83], %[vA0], %[rWS], s55 offen offset:3072 sc0 sc1\n\t" \
    "v_mfma_f32_16x16x32_f16 a[120:123], v[128:131], v[156:159], a[120:123]\n\t" \
    "v_rcp_f32 v237, v237\n\t" \
    "v_mfma_f32_16x16x32_f16 a[88:91], v[124:127], v[156:159], a[88:91]\n\t" \
    "s_add_u32 s55, s38, 16384\n\t" \
    "v_mfma_f32_16x16x32_f16 a[56:59], v[120:123], v[156:159], a[56:59]\n\t" \
    "v_rcp_f32 v238, v238\n\t" \
    "v_mfma_f32_16x16x32_f16 a[24:27], v[116:119], v[156:159], a[24:27]\n\t" \
    "ds_read_b128 v[156:159], %[vA0] offset:34816\n\t" \
    "v_mfma_f32_16x16x32_f16 a[28:31], v[116:119], v[160:163], a[28:31]\n\t" \
    "v_rcp_f32 v239, v239\n\t" \
    "v_mfma_f32_16x16x32_f16 a[60:63], v[120:123], v[160:163], a[60:63]\n\t" \
    "buffer_load_dwordx4 v[84:87], %[vA0], %[rWS], s55 offen offset:0 sc0 sc1\n\t" \
    "v_mfma_f32_16x16x32_f16 a[92:95], v[124:127], v[160:163], a[92:95]\n\t" \
    "v_mul_f32 v232, v252, v236\n\t" \
    "v_mul_f32 v233, v253, v237\n\t" \
    "v_mfma_f32_16x16x32_f16 a[124:127], v[128:131], v[160:163], a[124:127]\n\t" \
    "v_mul_f32 v234, v254, v238\n\t" \
    "v_mfma_f32_16x16x32_f16 a[156:159], v[132:135], v[160:163], a[156:159]\n\t" \
    "buffer_load_dwordx4 v[88:91], %[vA0], %[rWS], s55 offen offset:1024 sc0 sc1\n\t" \
    "v_mfma_f32_16x16x32_f16 a[188:191], v[136:139], v[160:163], a[188:191]\n\t" \
    "v_mul_f32 v235, v255, v239\n\t" \
    "v_mfma_f32_16x16x32_f16 a[220:223], v[140:143], v[160:163], a[220:223]\n\t" \
    "buffer_load_dwordx4 v[92:95], %[vA0], %[rWS], s55 offen offset:2048 sc0 sc1\n\t" \
    "v_mfma_f32_16x16x32_f16 a[252:255], v[144:147], v[160:163], a[252:255]\n\t" \
    "buffer_load_dwordx4 v[96:99], %[vA0], %[rWS], s55 offen offset:3072 sc0 sc1\n\t" \
    "s_add_u32 s36, s36, 1\n\t" \
    "s_cmp_eq_u32 s48, 3\n\t" \
    "s_cbranch_scc1 LASTT%=\n\t" \
    "ds_read_b128 v[160:163], %[vA0] offset:35840\n\t" \
    "s_waitcnt lgkmcnt(3)\n\t" \
    "v_mfma_f32_16x16x32_f16 a[224:227], v[48:51], v[148:151], a[224:227]\n\t" \
    "s_mov_b32 s37, s38\n\t" \
    "v_cvt_pk_f16_f32 v228, v164, v168\n\t" \
    "v_mfma_f32_16x16x32_f16 a[192:195], v[44:47], v[148:151], a[192:195]\n\t" \
    "v_mfma_f32_16x16x32_f16 a[160:163], v[40:43], v[148:151], a[160:163]\n\t" \
    "v_cvt_pk_f16_f32 v229, v172, v176\n\t" \
    "v_mfma_f32_16x16x32_f16 a[128:131], v[36:39], v[148:151], a[128:131]\n\t" \
    "ds_write_b64 %[vWr0], v[228:229] offset:0\n\t" \
    "v_mfma_f32_16x16x32_f16 a[96:99], v[32:35], v[148:151], a[96:99]\n\t" \
    "v_cvt_pk_f16_f32 v230, v180, v184\n\t" \
    "v_mfma_f32_16x16x32_f16 a[64:67], v[28:31], v[148:151], a[64:67]\n\t" \
    "v_mfma_f32_16x16x32_f16 a[32:35], v[24:27], v[148:151], a[32:35]\n\t" \
    "v_cvt_pk_f16_f32 v231, v188, v192\n\t" \
    "v_mfma_f32_16x16x32_f16 a[0:3], v[20:23], v[148:151], a[0:3]\n\t" \
    "ds_read_b128 v[148:151], %[vA0] offset:36864\n\t" \
    "s_waitcnt lgkmcnt(4)\n\t" \
    "v_mfma_f32_16x16x32_f16 a[4:7], v[20:23], v[152:155], a[4:7]\n\t" \
    "s_cmp_ge_u32 s48, 2\n\ts_cselect_b64 exec, -1, 0\n\ts_cselect_b32 s69, %[own], 9\n\tds_read_b128 v[252:255], %[vStg] offset:23552\n\ts_mov_b64 exec, -1\n\t" \
    "s_waitcnt lgkmcnt(4)\n\t" \
    "v_mfma_f32_16x16x32_f16 a[36:39], v[24:27], v[152:155], a[36:39]\n\t" \
    "ds_write_b64 %[vWr0], v[230:231] offset:8\n\t" \
    "v_mfma_f32_16x16x32_f16 a[68:71], v[28:31], v[152:155], a[68:71]\n\t" \
    "v_cvt_pk_f16_f32 v228, v165, v169\n\t" \
    "s_add_u32 s55, s37, 20480\n\t" \
    "v_mfma_f32_16x16x32_f16 a[100:103], v[32:35], v[152:155], a[100:103]\n\t" \
    "v_cvt_pk_f16_f32 v229, v173, v177\n\t" \
    "v_mfma_f32_16x16x32_f16 a[132:135], v[36:39], v[152:155], a[132:135]\n\t" \
    "buffer_load_dwordx4 v[100:103], %[vA0], %[rWS], s55 offen offset:0 sc0 sc1\n\t" \
    "v_mfma_f32_16x16x32_f16 a[164:167], v[40:43], v[152:155], a[164:167]\n\t" \
    "ds_write_b64 %[vWr0], v[228:229] offset:1024\n\t" \
    "v_mfma_f32_16x16x32_f16 a[196:199], v[44:47], v[152:155], a[196:199]\n\t" \
    "v_cvt_pk_f16_f32 v230, v181, v185\n\t" \
    "v_cvt_pk_f16_f32 v231, v189, v193\n\t" \
    "v_mfma_f32_16x16x32_f16 a[228:231], v[48:51], v[152:155], a[228:231]\n\t" \
    "ds_read_b128 v[152:155], %[vA0] offset:37888\n\t" \
    "s_waitcnt lgkmcnt(6)\n\t" \
    "v_mfma_f32_16x16x32_f16 a[232:235], v[48:51], v[156:159], a[232:235]\n\t" \
    "ds_write_b64 %[vWr0], v[230:231] offset:1032\n\t" \
    "v_mfma_f32_16x16x32_f16 a[200:203], v[44:47], v[156:159], a[200:203]\n\t" \
    "buffer_load_dwordx4 v[104:107], %[vA0], %[rWS], s55 offen offset:1024 sc0 sc1\n\t" \
    "v_mfma_f32_16x16x32_f16 a[168:171], v[40:43], v[156:159], a[168:171]\n\t" \
    "v_cvt_pk_f16_f32 v228, v166, v170\n\t" \
    "v_cvt_pk_f16_f32 v229, v174, v178\n\t" \
    "v_mfma_f32_16x16x32_f16 a[136:139], v[36:39], v[156:159], a[136:139]\n\t" \
    "ds_write_b64 %[vWr0], v[228:229] offset:2048\n\t" \
    "v_mfma_f32_16x16x32_f16 a[104:107], v[32:35], v[156:159], a[104:107]\n\t" \
    "v_cvt_pk_f16_f32 v230, v182, v186\n\t" \
    "v_mfma_f32_16x16x32_f16 a[72:75], v[28:31], v[156:159], a[72:75]\n\t" \
    "buffer_load_dwordx4 v[108:111], %[vA0], %[rWS], s55 offen offset:2048 sc0 sc1\n\t" \
    "v_mfma_f32_16x16x32_f16 a[40:43], v[24:27], v[156:159], a[40:43]\n\t" \
    "s_add_u32 s66, s66, 0x400000\n\t" \
    "s_and_b32 s66, s66, 0xfffffff\n\t" \
    "v_mfma_f32_16x16x32_f16 a[8:11], v[20:23], v[156:159], a[8:11]\n\t" \
    "ds_read_b128 v[156:159], %[vA0] offset:38912\n\t" \
    "s_waitcnt lgkmcnt(8)\n\t" \
    "v_mfma_f32_16x16x32_f16 a[12:15], v[20:23], v[160:163], a[12:15]\n\t" \
    "v_cvt_pk_f16_f32 v231, v190, v194\n\t" \
    "v_mfma_f32_16x16x32_f16 a[44:47], v[24:27], v[160:163], a[44:47]\n\t" \
    "ds_write_b64 %[vWr0], v[230:231] offset:2056\n\t" \
    "v_mfma_f32_16x16x32_f16 a[76:79], v[28:31], v[160:163], a[76:79]\n\t" \
    "v_cvt_pk_f16_f32 v228, v167, v171\n\t" \
    "s_add_u32 s39, s66, s44\n\t" \
    "v_mfma_f32_16x16x32_f16 a[108:111], v[32:35], v[160:163], a[108:111]\n\t" \
    "s_add_u32 s39, s39, %[sWk0]\n\t" \
    "v_mfma_f32_16x16x32_f16 a[140:143], v[36:39], v[160:163], a[140:143]\n\t" \
    "buffer_load_dwordx4 v[112:115], %[vA0], %[rWS], s55 offen offset:3072 sc0 sc1\n\t" \
    "v_mfma_f32_16x16x32_f16 a[172:175], v[40:43], v[160:163], a[172:175]\n\t" \
    "v_cvt_pk_f16_f32 v229, v175, v179\n\t" \
    "v_mfma_f32_16x16x32_f16 a[204:207], v[44:47], v[160:163], a[204:207]\n\t" \
    "ds_write_b64 %[vWr0], v[228:229] offset:3072\n\t" \
    "v_mfma_f32_16x16x32_f16 a[236:239], v[48:51], v[160:163], a[236:239]\n\t" \
    "ds_read_b128 v[160:163], %[vA0] offset:39936\n\t" \
    "s_waitcnt lgkmcnt(9)\n\t" \
    "v_mfma_f32_16x16x32_f16 a[240:243], v[48:51], v[148:151], a[240:243]\n\t" \
    "v_cvt_pk_f16_f32 v230, v183, v187\n\t" \
    "v_cvt_pk_f16_f32 v231, v191, v195\n\t" \
    "v_mfma_f32_16x16x32_f16 a[208:211], v[44:47], v[148:151], a[208:211]\n\t" \
    "ds_write_b64 %[vWr0], v[230:231] offset:3080\n\t" \
    "v_mfma_f32_16x16x32_f16 a[176:179], v[40:43], v[148:151], a[176:179]\n\t" \
    "buffer_load_dwordx4 v[164:167], %[vW], %[rW], s39 offen\n\t" \
    "v_mfma_f32_16x16x32_f16 a[144:147], v[36:39], v[148:151], a[144:147]\n\t" \
    "s_add_u32 s38, s38, 32768\n\t" \
    "s_and_b32 s38, s38, 0x1fffff\n\t" \
    "v_mfma_f32_16x16x32_f16 a[112:115], v[32:35], v[148:151], a[112:115]\n\t" \
    "s_or_b32 s38, s38, %[sA0]\n\t" \
    "s_add_u32 s38, s38, s47\n\t" \
    "v_mfma_f32_16x16x32_f16 a[80:83], v[28:31], v[148:151], a[80:83]\n\t" \
    "s_add_u32 s39, s39, 32768\n\t" \
    "v_mfma_f32_16x16x32_f16 a[48:51], v[24:27], v[148:151], a[48:51]\n\t" \
    "buffer_load_dwordx4 v[168:171], %[vW], %[rW], s39 offen\n\t" \
    "v_mfma_f32_16x16x32_f16 a[16:19], v[20:23], v[148:151], a[16:19]\n\t" \
    "ds_read_b128 v[148:151], %[vA0] offset:40960\n\t" \
    "s_waitcnt lgkmcnt(8)\n\t" \
    "v_mfma_f32_16x16x32_f16 a[20:23], v[20:23], v[152:155], a[20:23]\n\t" \
    "s_add_u32 s39, s39, 32768\n\t" \
    "v_mfma_f32_16x16x32_f16 a[52:55], v[24:27], v[152:155], a[52:55]\n\t" \
    "buffer_load_dwordx4 v[172:175], %[vW], %[rW], s39 offen\n\t" \
    "v_mfma_f32_16x16x32_f16 a[84:87], v[28:31], v[152:155], a[84:87]\n\t" \
    "v_mfma_f32_16x16x32_f16 a[116:119], v[32:35], v[152:155], a[116:119]\n\t" \
    "s_add_u32 s39, s39, 32768\n\t" \
    "v_mfma_f32_16x16x32_f16 a[148:151], v[36:39], v[152:155], a[148:151]\n\t" \
    "v_mfma_f32_16x16x32_f16 a[180:183], v[40:43], v[152:155], a[180:183]\n\t" \
    "buffer_load_dwordx4 v[176:179], %[vW], %[rW], s39 offen\n\t" \
    "v_mfma_f32_16x16x32_f16 a[212:215], v[44:47], v[152:155], a[212:215]\n\t" \
    "v_mfma_f32_16x16x32_f16 a[244:247], v[48:51], v[152:155], a[244:247]\n\t" \
    "ds_read_b128 v[152:155], %[vA0] offset:41984\n\t" \
    "s_waitcnt lgkmcnt(6)\n\t" \
    "v_mfma_f32_16x16x32_f16 a[248:251], v[48:51], v[156:159], a[248:251]\n\t" \
    "s_add_u32 s39, s39, 32768\n\t" \
    "v_mfma_f32_16x16x32_f16 a[216:219], v[44:47], v[156:159], a[216:219]\n\t" \
    "buffer_store_dwordx4 v[232:235], %[vO], s[56:59], 0 offen sc0 sc1 nt\n\t" \
    "v_mfma_f32_16x16x32_f16 a[184:187], v[40:43], v[156:159], a[184:187]\n\t" \
    "buffer_load_dwordx4 v[180:183], %[vW], %[rW], s39 offen\n\t" \
    "v_mfma_f32_16x16x32_f16 a[152:155], v[36:39], v[156:159], a[152:155]\n\t" \
    "s_add_u32 s39, s39, 32768\n\t" \
    "v_mfma_f32_16x16x32_f16 a[120:123], v[32:35], v[156:159], a[120:123]\n\t" \
    "v_mfma_f32_16x16x32_f16 a[88:91], v[28:31], v[156:159], a[88:91]\n\t" \
    "buffer_load_dwordx4 v[184:187], %[vW], %[rW], s39 offen\n\t" \
    "v_mfma_f32_16x16x32_f16 a[56:59], v[24:27], v[156:159], a[56:59]\n\t" \
    "v_mfma_f32_16x16x32_f16 a[24:27], v[20:23], v[156:159], a[24:27]\n\t" \
    "ds_read_b128 v[156:159], %[vA0] offset:43008\n\t" \
    "s_waitcnt lgkmcnt(4)\n\t" \
    "v_mfma_f32_16x16x32_f16 a[28:31], v[20:23], v[160:163], a[28:31]\n\t" \
    "s_add_u32 s39, s39, 32768\n\t" \
    "v_mfma_f32_16x16x32_f16 a[60:63], v[24:27], v[160:163], a[60:63]\n\t" \
    "buffer_load_dwordx4 v[188:191], %[vW], %[rW], s39 offen\n\t" \
    "v_mfma_f32_16x16x32_f16 a[92:95], v[28:31], v[160:163], a[92:95]\n\t" \
    "v_mfma_f32_16x16x32_f16 a[124:127], v[32:35], v[160:163], a[124:127]\n\t" \
    "s_add_u32 s39, s39, 32768\n\t" \
    "v_mfma_f32_16x16x32_f16 a[156:159], v[36:39], v[160:163], a[156:159]\n\t" \
    "v_mfma_f32_16x16x32_f16 a[188:191], v[40:43], v[160:163], a[188:191]\n\t" \
    "buffer_load_dwordx4 v[192:195], %[vW], %[rW], s39 offen\n\t" \
    "v_mfma_f32_16x16x32_f16 a[220:223], v[44:47], v[160:163], a[220:223]\n\t" \
    "v_mfma_f32_16x16x32_f16 a[252:255], v[48:51], v[160:163], a[252:255]\n\t" \
    "ds_read_b128 v[160:163], %[vA0] offset:44032\n\t" \
    "s_waitcnt vmcnt(17) lgkmcnt(3)\n\t" \
    "v_mfma_f32_16x16x32_f16 a[224:227], v[80:83], v[148:151], a[224:227]\n\t" \
    "v_cvt_pk_f16_f32 v228, v196, v200\n\t" \
    "v_mfma_f32_16x16x32_f16 a[192:195], v[76:79], v[148:151], a[192:195]\n\t" \
    "v_mfma_f32_16x16x32_f16 a[160:163], v[72:75], v[148:151], a[160:163]\n\t" \
    "v_cvt_pk_f16_f32 v229, v204, v208\n\t" \
    "v_mfma_f32_16x16x32_f16 a[128:131], v[68:71], v[148:151], a[128:131]\n\t" \
    "ds_write_b64 %[vWr1], v[228:229] offset:0\n\t" \
    "v_mfma_f32_16x16x32_f16 a[96:99], v[64:67], v[148:151], a[96:99]\n\t" \
    "v_cvt_pk_f16_f32 v230, v212, v216\n\t" \
    "v_mfma_f32_16x16x32_f16 a[64:67], v[60:63], v[148:151], a[64:67]\n\t" \
    "v_mfma_f32_16x16x32_f16 a[32:35], v[56:59], v[148:151], a[32:35]\n\t" \
    "v_cvt_pk_f16_f32 v231, v220, v224\n\t" \
    "v_mfma_f32_16x16x32_f16 a[0:3], v[52:55], v[148:151], a[0:3]\n\t" \
    "ds_read_b128 v[148:151], %[vA0] offset:45056\n\t" \
    "s_waitcnt lgkmcnt(4)\n\t" \
    "v_mfma_f32_16x16x32_f16 a[4:7], v[52:55], v[152:155], a[4:7]\n\t" \
    "ds_write_b64 %[vWr1], v[230:231] offset:8\n\t" \
    "v_mfma_f32_16x16x32_f16 a[36:39], v[56:59], v[152:155], a[36:39]\n\t" \
    "v_cvt_pk_f16_f32 v228, v197, v201\n\t" \
    "v_mfma_f32_16x16x32_f16 a[68:71], v[60:63], v[152:155], a[68:71]\n\t" \
    "v_cvt_pk_f16_f32 v229, v205, v209\n\t" \
    "v_mfma_f32_16x16x32_f16 a[100:103], v[64:67], v[152:155], a[100:103]\n\t" \
    "ds_write_b64 %[vWr1], v[228:229] offset:1024\n\t" \
    "v_mfma_f32_16x16x32_f16 a[132:135], v[68:71], v[152:155], a[132:135]\n\t" \
    "v_cvt_pk_f16_f32 v230, v213, v217\n\t" \
    "v_mfma_f32_16x16x32_f16 a[164:167], v[72:75], v[152:155], a[164:167]\n\t" \
    "v_mfma_f32_16x16x32_f16 a[196:199], v[76:79], v[152:155], a[196:199]\n\t" \
    "v_cvt_pk_f16_f32 v231, v221, v225\n\t" \
    "v_mfma_f32_16x16x32_f16 a[228:231], v[80:83], v[152:155], a[228:231]\n\t" \
    "ds_read_b128 v[152:155], %[vA0] offset:46080\n\t" \
    "s_waitcnt lgkmcnt(6)\n\t" \
    "v_mfma_f32_16x16x32_f16 a[232:235], v[80:83], v[156:159], a[232:235]\n\t" \
    "s_add_u32 s55, s37, 24576\n\t" \
    "v_mfma_f32_16x16x32_f16 a[200:203], v[76:79], v[156:159], a[200:203]\n\t" \
    "ds_write_b64 %[vWr1], v[230:231] offset:1032\n\t" \
    "v_mfma_f32_16x16x32_f16 a[168:171], v[72:75], v[156:159], a[168:171]\n\t" \
    "v_cvt_pk_f16_f32 v228, v198, v202\n\t" \
    "v_mfma_f32_16x16x32_f16 a[136:139], v[68:71], v[156:159], a[136:139]\n\t" \
    "buffer_load_dwordx4 v[116:119], %[vA0], %[rWS], s55 offen offset:0 sc0 sc1\n\t" \
    "v_mfma_f32_16x16x32_f16 a[104:107], v[64:67], v[156:159], a[104:107]\n\t" \
    "v_cvt_pk_f16_f32 v229, v206, v210\n\t" \
    "v_mfma_f32_16x16x32_f16 a[72:75], v[60:63], v[156:159], a[72:75]\n\t" \
    "ds_write_b64 %[vWr1], v[228:229] offset:2048\n\t" \
    "v_mfma_f32_16x16x32_f16 a[40:43], v[56:59], v[156:159], a[40:43]\n\t" \
    "v_cvt_pk_f16_f32 v230, v214, v218\n\t" \
    "v_mfma_f32_16x16x32_f16 a[8:11], v[52:55], v[156:159], a[8:11]\n\t" \
    "ds_read_b128 v[156:159], %[vA0] offset:47104\n\t" \
    "s_waitcnt lgkmcnt(8)\n\t" \
    "v_mfma_f32_16x16x32_f16 a[12:15], v[52:55], v[160:163], a[12:15]\n\t" \
    "buffer_load_dwordx4 v[120:123], %[vA0], %[rWS], s55 offen offset:1024 sc0 sc1\n\t" \
    "v_mfma_f32_16x16x32_f16 a[44:47], v[56:59], v[160:163], a[44:47]\n\t" \
    "v_cvt_pk_f16_f32 v231, v222, v226\n\t" \
    "v_mfma_f32_16x16x32_f16 a[76:79], v[60:63], v[160:163], a[76:79]\n\t" \
    "ds_write_b64 %[vWr1], v[230:231] offset:2056\n\t" \
    "v_mfma_f32_16x16x32_f16 a[108:111], v[64:67], v[160:163], a[108:111]\n\t" \
    "v_cvt_pk_f16_f32 v228, v199, v203\n\t" \
    "v_cvt_pk_f16_f32 v229, v207, v211\n\t" \
    "v_mfma_f32_16x16x32_f16 a[140:143], v[68:71], v[160:163], a[140:143]\n\t" \
    "buffer_load_dwordx4 v[124:127], %[vA0], %[rWS], s55 offen offset:2048 sc0 sc1\n\t" \
    "v_mfma_f32_16x16x32_f16 a[172:175], v[72:75], v[160:163], a[172:175]\n\t" \
    "ds_write_b64 %[vWr1], v[228:229] offset:3072\n\t" \
    "v_mfma_f32_16x16x32_f16 a[204:207], v[76:79], v[160:163], a[204:207]\n\t" \
    "v_cvt_pk_f16_f32 v230, v215, v219\n\t" \
    "v_cvt_pk_f16_f32 v231, v223, v227\n\t" \
    "v_mfma_f32_16x16x32_f16 a[236:239], v[80:83], v[160:163], a[236:239]\n\t" \
    "ds_read_b128 v[160:163], %[vA0] offset:48128\n\t" \
    "s_waitcnt lgkmcnt(9)\n\t" \
    "v_mfma_f32_16x16x32_f16 a[240:243], v[80:83], v[148:151], a[240:243]\n\t" \
    "ds_write_b64 %[vWr1], v[230:231] offset:3080\n\t" \
    "v_mfma_f32_16x16x32_f16 a[208:211], v[76:79], v[148:151], a[208:211]\n\t" \
    "buffer_load_dwordx4 v[128:131], %[vA0], %[rWS], s55 offen offset:3072 sc0 sc1\n\t" \
    "v_mfma_f32_16x16x32_f16 a[176:179], v[72:75], v[148:151], a[176:179]\n\t" \
    "v_mfma_f32_16x16x32_f16 a[144:147], v[68:71], v[148:151], a[144:147]\n\t" \
    "s_add_u32 s55, s37, 28672\n\t" \
    "v_mfma_f32_16x16x32_f16 a[112:115], v[64:67], v[148:151], a[112:115]\n\t" \
    "v_mfma_f32_16x16x32_f16 a[80:83], v[60:63], v[148:151], a[80:83]\n\t" \
    "buffer_load_dwordx4 v[132:135], %[vA0], %[rWS], s55 offen offset:0 sc0 sc1\n\t" \
    "v_mfma_f32_16x16x32_f16 a[48:51], v[56:59], v[148:151], a[48:51]\n\t" \
    "v_mfma_f32_16x16x32_f16 a[16:19], v[52:55], v[148:151], a[16:19]\n\t" \
    "ds_read_b128 v[148:151], %[vA0] offset:49152\n\t" \
    "s_waitcnt lgkmcnt(8)\n\t" \
    "v_mfma_f32_16x16x32_f16 a[20:23], v[52:55], v[152:155], a[20:23]\n\t" \
    "v_mfma_f32_16x16x32_f16 a[52:55], v[56:59], v[152:155], a[52:55]\n\t" \
    "v_mfma_f32_16x16x32_f16 a[84:87], v[60:63], v[152:155], a[84:87]\n\t" \
    "buffer_load_dwordx4 v[136:139], %[vA0], %[rWS], s55 offen offset:1024 sc0 sc1\n\t" \
    "v_mfma_f32_16x16x32_f16 a[116:119], v[64:67], v[152:155], a[116:119]\n\t" \
    "v_mfma_f32_16x16x32_f16 a[148:151], v[68:71], v[152:155], a[148:151]\n\t" \
    "v_mfma_f32_16x16x32_f16 a[180:183], v[72:75], v[152:155], a[180:183]\n\t" \
    "v_mfma_f32_16x16x32_f16 a[212:215], v[76:79], v[152:155], a[212:215]\n\t" \
    "v_mfma_f32_16x16x32_f16 a[244:247], v[80:83], v[152:155], a[244:247]\n\t" \
    "ds_read_b128 v[152:155], %[vA0] offset:50176\n\t" \
    "s_waitcnt lgkmcnt(6)\n\t" \
    "v_mfma_f32_16x16x32_f16 a[248:251], v[80:83], v[156:159], a[248:251]\n\t" \
    "buffer_load_dwordx4 v[140:143], %[vA0], %[rWS], s55 offen offset:2048 sc0 sc1\n\t" \
    "v_mfma_f32_16x16x32_f16 a[216:219], v[76:79], v[156:159], a[216:219]\n\t" \
    "v_mfma_f32_16x16x32_f16 a[184:187], v[72:75], v[156:159], a[184:187]\n\t" \
    "v_mfma_f32_16x16x32_f16 a[152:155], v[68:71], v[156:159], a[152:155]\n\t" \
    "v_mfma_f32_16x16x32_f16 a[120:123], v[64:67], v[156:159], a[120:123]\n\t" \
    "v_mfma_f32_16x16x32_f16 a[88:91], v[60:63], v[156:159], a[88:91]\n\t" \
    "buffer_load_dwordx4 v[144:147], %[vA0], %[rWS], s55 offen offset:3072 sc0 sc1\n\t" \
    "v_mfma_f32_16x16x32_f16 a[56:59], v[56:59], v[156:159], a[56:59]\n\t" \
    "v_mfma_f32_16x16x32_f16 a[24:27], v[52:55], v[156:159], a[24:27]\n\t" \
    "ds_read_b128 v[156:159], %[vA0] offset:51200\n\t" \
    "s_waitcnt lgkmcnt(4)\n\t" \
    "v_mfma_f32_16x16x32_f16 a[28:31], v[52:55], v[160:163], a[28:31]\n\t" \
    "s_add_u32 s55, s38, 0\n\t" \
    "v_mfma_f32_16x16x32_f16 a[60:63], v[56:59], v[160:163], a[60:63]\n\t" \
    "v_mfma_f32_16x16x32_f16 a[92:95], v[60:63], v[160:163], a[92:95]\n\t" \
    "buffer_load_dwordx4 v[20:23], %[vA0], %[rWS], s55 offen offset:0 sc0 sc1\n\t" \
    "v_mfma_f32_16x16x32_f16 a[124:127], v[64:67], v[160:163], a[124:127]\n\t" \
    "v_mfma_f32_16x16x32_f16 a[156:159], v[68:71], v[160:163], a[156:159]\n\t" \
    "v_mfma_f32_16x16x32_f16 a[188:191], v[72:75], v[160:163], a[188:191]\n\t" \
    "v_mfma_f32_16x16x32_f16 a[220:223], v[76:79], v[160:163], a[220:223]\n\t" \
    "v_mfma_f32_16x16x32_f16 a[252:255], v[80:83], v[160:163], a[252:255]\n\t" \
    "ds_read_b128 v[160:163], %[vA0] offset:52224\n\t" \
    "s_waitcnt vmcnt(18) lgkmcnt(3)\n\t" \
    "v_mfma_f32_16x16x32_f16 a[224:227], v[112:115], v[148:151], a[224:227]\n\t" \
    "buffer_load_dwordx4 v[24:27], %[vA0], %[rWS], s55 offen offset:1024 sc0 sc1\n\t" \
    "v_mfma_f32_16x16x32_f16 a[192:195], v[108:111], v[148:151], a[192:195]\n\t" \
    "v_mfma_f32_16x16x32_f16 a[160:163], v[104:107], v[148:151], a[160:163]\n\t" \
    "v_mfma_f32_16x16x32_f16 a[128:131], v[100:103], v[148:151], a[128:131]\n\t" \
    "v_mfma_f32_16x16x32_f16 a[96:99], v[96:99], v[148:151], a[96:99]\n\t" \
    "buffer_load_dwordx4 v[28:31], %[vA0], %[rWS], s55 offen offset:2048 sc0 sc1\n\t" \
    "v_mfma_f32_16x16x32_f16 a[64:67], v[92:95], v[148:151], a[64:67]\n\t" \
    "v_mfma_f32_16x16x32_f16 a[32:35], v[88:91], v[148:151], a[32:35]\n\t" \
    "v_mfma_f32_16x16x32_f16 a[0:3], v[84:87], v[148:151], a[0:3]\n\t" \
    "ds_read_b128 v[148:151], %[vA0] offset:53248\n\t" \
    "s_waitcnt lgkmcnt(3)\n\t" \
    "v_mfma_f32_16x16x32_f16 a[4:7], v[84:87], v[152:155], a[4:7]\n\t" \
    "v_mfma_f32_16x16x32_f16 a[36:39], v[88:91], v[152:155], a[36:39]\n\t" \
    "buffer_load_dwordx4 v[32:35], %[vA0], %[rWS], s55 offen offset:3072 sc0 sc1\n\t" \
    "v_mfma_f32_16x16x32_f16 a[68:71], v[92:95], v[152:155], a[68:71]\n\t" \
    "v_mfma_f32_16x16x32_f16 a[100:103], v[96:99], v[152:155], a[100:103]\n\t" \
    "v_mfma_f32_16x16x32_f16 a[132:135], v[100:103], v[152:155], a[132:135]\n\t" \
    "v_mfma_f32_16x16x32_f16 a[164:167], v[104:107], v[152:155], a[164:167]\n\t" \
    "s_add_u32 s55, s38, 4096\n\t" \
    "v_mfma_f32_16x16x32_f16 a[196:199], v[108:111], v[152:155], a[196:199]\n\t" \
    "v_mfma_f32_16x16x32_f16 a[228:231], v[112:115], v[152:155], a[228:231]\n\t" \
    "ds_read_b128 v[152:155], %[vA0] offset:54272\n\t" \
    "s_waitcnt lgkmcnt(3)\n\t" \
    "v_mfma_f32_16x16x32_f16 a[232:235], v[112:115], v[156:159], a[232:235]\n\t" \
    "buffer_load_dwordx4 v[36:39], %[vA0], %[rWS], s55 offen offset:0 sc0 sc1\n\t" \
    "v_mfma_f32_16x16x32_f16 a[200:203], v[108:111], v[156:159], a[200:203]\n\t" \
    "v_mfma_f32_16x16x32_f16 a[168:171], v[104:107], v[156:159], a[168:171]\n\t" \
    "v_mfma_f32_16x16x32_f16 a[136:139], v[100:103], v[156:159], a[136:139]\n\t" \
    "v_mfma_f32_16x16x32_f16 a[104:107], v[96:99], v[156:159], a[104:107]\n\t" \
    "buffer_load_dwordx4 v[40:43], %[vA0], %[rWS], s55 offen offset:1024 sc0 sc1\n\t" \
    "v_mfma_f32_16x16x32_f16 a[72:75], v[92:95], v[156:159], a[72:75]\n\t" \
    "v_readlane_b32 s52, %[vTab0], s51\n\t" \
    "v_mfma_f32_16x16x32_f16 a[40:43], v[88:91], v[156:159], a[40:43]\n\t" \
    "s_add_u32 s42, s51, 32\n\t" \
    "v_mfma_f32_16x16x32_f16 a[8:11], v[84:87], v[156:159], a[8:11]\n\t" \
    "ds_read_b128 v[156:159], %[vA0] offset:55296\n\t" \
    "s_waitcnt lgkmcnt(3)\n\t" \
    "v_mfma_f32_16x16x32_f16 a[12:15], v[84:87], v[160:163], a[12:15]\n\t" \
    "v_readlane_b32 s53, %[vTab1], s51\n\t" \
    "v_mfma_f32_16x16x32_f16 a[44:47], v[88:91], v[160:163], a[44:47]\n\t" \
    "v_readlane_b32 s42, %[vTab0], s42\n\t" \
    "v_mfma_f32_16x16x32_f16 a[76:79], v[92:95], v[160:163], a[76:79]\n\t" \
    "s_add_u32 s51, s51, 1\n\t" \
    "v_mfma_f32_16x16x32_f16 a[108:111], v[96:99], v[160:163], a[108:111]\n\t" \
    "buffer_load_dwordx4 v[44:47], %[vA0], %[rWS], s55 offen offset:2048 sc0 sc1\n\t" \
    "v_mfma_f32_16x16x32_f16 a[140:143], v[100:103], v[160:163], a[140:143]\n\t" \
    "s_add_u32 s40, s66, s44\n\t" \
    "s_add_u32 s40, s40, %[sWk1]\n\t" \
    "v_mfma_f32_16x16x32_f16 a[172:175], v[104:107], v[160:163], a[172:175]\n\t" \
    "s_mov_b32 s56, s42\n\t" \
    "s_and_b32 s57, s53, 0xffff\n\t" \
    "v_mfma_f32_16x16x32_f16 a[204:207], v[108:111], v[160:163], a[204:207]\n\t" \
    "s_and_b32 s58, s53, 0x10000\n\t" \
    "s_bfe_i32 s53, s53, 0x10011\n\t" \
    "v_mfma_f32_16x16x32_f16 a[236:239], v[112:115], v[160:163], a[236:239]\n\t" \
    "ds_read_b128 v[160:163], %[vA0] offset:56320\n\t" \
    "s_waitcnt lgkmcnt(3)\n\t" \
    "v_mfma_f32_16x16x32_f16 a[240:243], v[112:115], v[148:151], a[240:243]\n\t" \
    "buffer_load_dwordx4 v[48:51], %[vA0], %[rWS], s55 offen offset:3072 sc0 sc1\n\t" \
    "v_mfma_f32_16x16x32_f16 a[208:211], v[108:111], v[148:151], a[208:211]\n\t" \
    "v_mfma_f32_16x16x32_f16 a[176:179], v[104:107], v[148:151], a[176:179]\n\t" \
    "v_mfma_f32_16x16x32_f16 a[144:147], v[100:103], v[148:151], a[144:147]\n\t" \
    "buffer_load_dwordx4 v[196:199], %[vW], %[rW], s40 offen nt\n\t" \
    "v_mfma_f32_16x16x32_f16 a[112:115], v[96:99], v[148:151], a[112:115]\n\t" \
    "v_mfma_f32_16x16x32_f16 a[80:83], v[92:95], v[148:151], a[80:83]\n\t" \
    "v_mfma_f32_16x16x32_f16 a[48:51], v[88:91], v[148:151], a[48:51]\n\t" \
    "s_add_u32 s40, s40, 32768\n\t" \
    "v_mfma_f32_16x16x32_f16 a[16:19], v[84:87], v[148:151], a[16:19]\n\t" \
    "ds_read_b128 v[148:151], %[vA0] offset:57344\n\t" \
    "s_waitcnt lgkmcnt(3)\n\t" \
    "v_mfma_f32_16x16x32_f16 a[20:23], v[84:87], v[152:155], a[20:23]\n\t" \
    "s_cmp_eq_u32 s69, 0\n\t" \
    "s_cselect_b32 s42, 1073741824, s52\n\t" \
    "v_mfma_f32_16x16x32_f16 a[52:55], v[88:91], v[152:155], a[52:55]\n\t" \
    "buffer_load_dwordx4 v[200:203], %[vW], %[rW], s40 offen nt\n\t" \
    "v_mfma_f32_16x16x32_f16 a[84:87], v[92:95], v[152:155], a[84:87]\n\t" \
    "buffer_load_dwordx4 v[236:239], %[vA0], %[rWS], s42 offen sc0 sc1\n\t" \
    "v_mfma_f32_16x16x32_f16 a[116:119], v[96:99], v[152:155], a[116:119]\n\t" \
    "s_add_u32 s52, s52, 262144\n\t" \
    "s_cmp_eq_u32 s69, 1\n\t" \
    "v_mfma_f32_16x16x32_f16 a[148:151], v[100:103], v[152:155], a[148:151]\n\t" \
    "s_cselect_b32 s42, 1073741824, s52\n\t" \
    "v_mfma_f32_16x16x32_f16 a[180:183], v[104:107], v[152:155], a[180:183]\n\t" \
    "buffer_load_dwordx4 v[240:243], %[vA0], %[rWS], s42 offen sc0 sc1\n\t" \
    "v_mfma_f32_16x16x32_f16 a[212:215], v[108:111], v[152:155], a[212:215]\n\t" \
    "s_add_u32 s40, s40, 32768\n\t" \
    "s_add_u32 s52, s52, 262144\n\t" \
    "v_mfma_f32_16x16x32_f16 a[244:247], v[112:115], v[152:155], a[244:247]\n\t" \
    "ds_read_b128 v[152:155], %[vA0] offset:58368\n\t" \
    "s_waitcnt lgkmcnt(3)\n\t" \
    "v_mfma_f32_16x16x32_f16 a[248:251], v[112:115], v[156:159], a[248:251]\n\t" \
    "s_cmp_eq_u32 s69, 2\n\t" \
    "s_cselect_b32 s42, 1073741824, s52\n\t" \
    "v_mfma_f32_16x16x32_f16 a[216:219], v[108:111], v[156:159], a[216:219]\n\t" \
    "buffer_load_dwordx4 v[204:207], %[vW], %[rW], s40 offen nt\n\t" \
    "v_mfma_f32_16x16x32_f16 a[184:187], v[104:107], v[156:159], a[184:187]\n\t" \
    "buffer_load_dwordx4 v[244:247], %[vA0], %[rWS], s42 offen sc0 sc1\n\t" \
    "v_mfma_f32_16x16x32_f16 a[152:155], v[100:103], v[156:159], a[152:155]\n\t" \
    "s_add_u32 s52, s52, 262144\n\t" \
    "s_cmp_eq_u32 s69, 3\n\t" \
    "v_mfma_f32_16x16x32_f16 a[120:123], v[96:99], v[156:159], a[120:123]\n\t" \
    "s_cselect_b32 s42, 1073741824, s52\n\t" \
    "s_add_u32 s40, s40, 32768\n\t" \
    "v_mfma_f32_16x16x32_f16 a[88:91], v[92:95], v[156:159], a[88:91]\n\t" \
    "buffer_load_dwordx4 v[248:251], %[vA0], %[rWS], s42 offen sc0 sc1\n\t" \
    "v_mfma_f32_16x16x32_f16 a[56:59], v[88:91], v[156:159], a[56:59]\n\t" \
    "buffer_load_dwordx4 v[208:211], %[vW], %[rW], s40 offen nt\n\t" \
    "v_mfma_f32_16x16x32_f16 a[24:27], v[84:87], v[156:159], a[24:27]\n\t" \
    "ds_read_b128 v[156:159], %[vA0] offset:59392\n\t" \
    "s_waitcnt lgkmcnt(3)\n\t" \
    "v_mfma_f32_16x16x32_f16 a[28:31], v[84:87], v[160:163], a[28:31]\n\t" \
    "s_add_u32 s40, s40, 32768\n\t" \
    "v_mfma_f32_16x16x32_f16 a[60:63], v[88:91], v[160:163], a[60:63]\n\t" \
    "buffer_load_dwordx4 v[212:215], %[vW], %[rW], s40 offen nt\n\t" \
    "v_mfma_f32_16x16x32_f16 a[92:95], v[92:95], v[160:163], a[92:95]\n\t" \
    "v_mfma_f32_16x16x32_f16 a[124:127], v[96:99], v[160:163], a[124:127]\n\t" \
    "s_add_u32 s40, s40, 32768\n\t" \
    "v_mfma_f32_16x16x32_f16 a[156:159], v[100:103], v[160:163], a[156:159]\n\t" \
    "v_mfma_f32_16x16x32_f16 a[188:191], v[104:107], v[160:163], a[188:191]\n\t" \
    "buffer_load_dwordx4 v[216:219], %[vW], %[rW], s40 offen nt\n\t" \
    "v_mfma_f32_16x16x32_f16 a[220:223], v[108:111], v[160:163], a[220:223]\n\t" \
    "v_mfma_f32_16x16x32_f16 a[252:255], v[112:115], v[160:163], a[252:255]\n\t" \
    "ds_read_b128 v[160:163], %[vA0] offset:60416\n\t" \
    "s_waitcnt vmcnt(18) lgkmcnt(3)\n\t" \
    "v_mfma_f32_16x16x32_f16 a[224:227], v[144:147], v[148:151], a[224:227]\n\t" \
    "s_add_u32 s40, s40, 32768\n\t" \
    "v_mfma_f32_16x16x32_f16 a[192:195], v[140:143], v[148:151], a[192:195]\n\t" \
    "v_mfma_f32_16x16x32_f16 a[160:163], v[136:139], v[148:151], a[160:163]\n\t" \
    "buffer_load_dwordx4 v[220:223], %[vW], %[rW], s40 offen nt\n\t" \
    "v_mfma_f32_16x16x32_f16 a[128:131], v[132:135], v[148:151], a[128:131]\n\t" \
    "v_mfma_f32_16x16x32_f16 a[96:99], v[128:131], v[148:151], a[96:99]\n\t" \
    "v_mfma_f32_16x16x32_f16 a[64:67], v[124:127], v[148:151], a[64:67]\n\t" \
    "v_mfma_f32_16x16x32_f16 a[32:35], v[120:123], v[148:151], a[32:35]\n\t" \
    "s_add_u32 s40, s40, 32768\n\t" \
    "v_mfma_f32_16x16x32_f16 a[0:3], v[116:119], v[148:151], a[0:3]\n\t" \
    "ds_read_b128 v[148:151], %[vA0] offset:61440\n\t" \
    "s_waitcnt lgkmcnt(3)\n\t" \
    "v_mfma_f32_16x16x32_f16 a[4:7], v[116:119], v[152:155], a[4:7]\n\t" \
    "buffer_load_dwordx4 v[224:227], %[vW], %[rW], s40 offen nt\n\t" \
    "v_mfma_f32_16x16x32_f16 a[36:39], v[120:123], v[152:155], a[36:39]\n\t" \
    "v_mfma_f32_16x16x32_f16 a[68:71], v[124:127], v[152:155], a[68:71]\n\t" \
    "v_mfma_f32_16x16x32_f16 a[100:103], v[128:131], v[152:155], a[100:103]\n\t" \
    "s_add_u32 s55, s38, 8192\n\t" \
    "v_mfma_f32_16x16x32_f16 a[132:135], v[132:135], v[152:155], a[132:135]\n\t" \
    "v_and_b32 v252, s53, v252\n\t" \
    "v_mfma_f32_16x16x32_f16 a[164:167], v[136:139], v[152:155], a[164:167]\n\t" \
    "buffer_load_dwordx4 v[52:55], %[vA0], %[rWS], s55 offen offset:0 sc0 sc1\n\t" \
    "v_mfma_f32_16x16x32_f16 a[196:199], v[140:143], v[152:155], a[196:199]\n\t" \
    "v_and_b32 v253, s53, v253\n\t" \
    "v_and_b32 v254, s53, v254\n\t" \
    "v_mfma_f32_16x16x32_f16 a[228:231], v[144:147], v[152:155], a[228:231]\n\t" \
    "ds_read_b128 v[152:155], %[vA0] offset:62464\n\t" \
    "s_waitcnt lgkmcnt(3)\n\t" \
    "v_mfma_f32_16x16x32_f16 a[232:235], v[144:147], v[156:159], a[232:235]\n\t" \
    "v_and_b32 v255, s53, v255\n\t" \
    "s_waitcnt vmcnt(9)\n\t" \
    "v_add_f32 v236, v236, v240\n\t" \
    "v_mfma_f32_16x16x32_f16 a[200:203], v[140:143], v[156:159], a[200:203]\n\t" \
    "v_add_f32 v237, v237, v241\n\t" \
    "v_mfma_f32_16x16x32_f16 a[168:171], v[136:139], v[156:159], a[168:171]\n\t" \
    "buffer_load_dwordx4 v[56:59], %[vA0], %[rWS], s55 offen offset:1024 sc0 sc1\n\t" \
    "v_mfma_f32_16x16x32_f16 a[136:139], v[132:135], v[156:159], a[136:139]\n\t" \
    "v_add_f32 v238, v238, v242\n\t" \
    "v_add_f32 v239, v239, v243\n\t" \
    "v_mfma_f32_16x16x32_f16 a[104:107], v[128:131], v[156:159], a[104:107]\n\t" \
    "v_add_f32 v252, v252, v236\n\t" \
    "v_mfma_f32_16x16x32_f16 a[72:75], v[124:127], v[156:159], a[72:75]\n\t" \
    "v_add_f32 v253, v253, v237\n\t" \
    "v_mfma_f32_16x16x32_f16 a[40:43], v[120:123], v[156:159], a[40:43]\n\t" \
    "v_add_f32 v254, v254, v238\n\t" \
    "v_mfma_f32_16x16x32_f16 a[8:11], v[116:119], v[156:159], a[8:11]\n\t" \
    "ds_read_b128 v[156:159], %[vA0] offset:63488\n\t" \
    "s_waitcnt lgkmcnt(3)\n\t" \
    "v_mfma_f32_16x16x32_f16 a[12:15], v[116:119], v[160:163], a[12:15]\n\t" \
    "buffer_load_dwordx4 v[60:63], %[vA0], %[rWS], s55 offen offset:2048 sc0 sc1\n\t" \
    "v_mfma_f32_16x16x32_f16 a[44:47], v[120:123], v[160:163], a[44:47]\n\t" \
    "v_add_f32 v255, v255, v239\n\t" \
    "s_waitcnt vmcnt(8)\n\t" \
    "v_add_f32 v244, v244, v248\n\t" \
    "v_mfma_f32_16x16x32_f16 a[76:79], v[124:127], v[160:163], a[76:79]\n\t" \
    "v_add_f32 v245, v245, v249\n\t" \
    "v_add_f32 v246, v246, v250\n\t" \
    "v_mfma_f32_16x16x32_f16 a[108:111], v[128:131], v[160:163], a[108:111]\n\t" \
    "v_add_f32 v247, v247, v251\n\t" \
    "v_mfma_f32_16x16x32_f16 a[140:143], v[132:135], v[160:163], a[140:143]\n\t" \
    "buffer_load_dwordx4 v[64:67], %[vA0], %[rWS], s55 offen offset:3072 sc0 sc1\n\t" \
    "v_mfma_f32_16x16x32_f16 a[172:175], v[136:139], v[160:163], a[172:175]\n\t" \
    "v_add_f32 v252, v252, v244\n\t" \
    "v_add_f32 v253, v253, v245\n\t" \
    "v_mfma_f32_16x16x32_f16 a[204:207], v[140:143], v[160:163], a[204:207]\n\t" \
    "v_add_f32 v254, v254, v246\n\t" \
    "v_mfma_f32_16x16x32_f16 a[236:239], v[144:147], v[160:163], a[236:239]\n\t" \
    "ds_read_b128 v[160:163], %[vA0] offset:64512\n\t" \
    "s_waitcnt lgkmcnt(3)\n\t" \
    "v_mfma_f32_16x16x32_f16 a[240:243], v[144:147], v[148:151], a[240:243]\n\t" \
    "s_add_u32 s55, s38, 12288\n\t" \
    "v_add_f32 v255, v255, v247\n\t" \
    "v_mfma_f32_16x16x32_f16 a[208:211], v[140:143], v[148:151], a[208:211]\n\t" \
    "v_med3_f32 v236, v252, %[m60], %[p60]\n\t" \
    "v_mfma_f32_16x16x32_f16 a[176:179], v[136:139], v[148:151], a[176:179]\n\t" \
    "buffer_load_dwordx4 v[68:71], %[vA0], %[rWS], s55 offen offset:0 sc0 sc1\n\t" \
    "v_mfma_f32_16x16x32_f16 a[144:147], v[132:135], v[148:151], a[144:147]\n\t" \
    "v_med3_f32 v237, v253, %[m60], %[p60]\n\t" \
    "v_med3_f32 v238, v254, %[m60], %[p60]\n\t" \
    "v_mfma_f32_16x16x32_f16 a[112:115], v[128:131], v[148:151], a[112:115]\n\t" \
    "v_med3_f32 v239, v255, %[m60], %[p60]\n\t" \
    "v_mul_f32 v236, 0xbfb8aa3b, v236\n\t" \
    "v_mfma_f32_16x16x32_f16 a[80:83], v[124:127], v[148:151], a[80:83]\n\t" \
    "v_mul_f32 v237, 0xbfb8aa3b, v237\n\t" \
    "v_mfma_f32_16x16x32_f16 a[48:51], v[120:123], v[148:151], a[48:51]\n\t" \
    "v_mul_f32 v238, 0xbfb8aa3b, v238\n\t" \
    "v_mfma_f32_16x16x32_f16 a[16:19], v[116:119], v[148:151], a[16:19]\n\t" \
    "s_waitcnt lgkmcnt(0)\n\t" \
    "s_barrier\n\t" \
    "ds_read_b128 v[148:151], %[vA0] offset:0\n\t" \
    "v_mfma_f32_16x16x32_f16 a[20:23], v[116:119], v[152:155], a[20:23]\n\t" \
    "buffer_load_dwordx4 v[72:75], %[vA0], %[rWS], s55 offen offset:1024 sc0 sc1\n\t" \
    "v_mfma_f32_16x16x32_f16 a[52:55], v[120:123], v[152:155], a[52:55]\n\t" \
    "v_mul_f32 v239, 0xbfb8aa3b, v239\n\t" \
    "v_mfma_f32_16x16x32_f16 a[84:87], v[124:127], v[152:155], a[84:87]\n\t" \
    "v_exp_f32 v236, v236\n\t" \
    "v_mfma_f32_16x16x32_f16 a[116:119], v[128:131], v[152:155], a[116:119]\n\t" \
    "v_exp_f32 v237, v237\n\t" \
    "v_mfma_f32_16x16x32_f16 a[148:151], v[132:135], v[152:155], a[148:151]\n\t" \
    "v_exp_f32 v238, v238\n\t" \
    "v_mfma_f32_16x16x32_f16 a[180:183], v[136:139], v[152:155], a[180:183]\n\t" \
    "v_exp_f32 v239, v239\n\t" \
    "v_mfma_f32_16x16x32_f16 a[212:215], v[140:143], v[152:155], a[212:215]\n\t" \
    "buffer_load_dwordx4 v[76:79], %[vA0], %[rWS], s55 offen offset:2048 sc0 sc1\n\t" \
    "v_mfma_f32_16x16x32_f16 a[244:247], v[144:147], v[152:155], a[244:247]\n\t" \
    "ds_read_b128 v[152:155], %[vA0] offset:1024\n\t" \
    "v_mfma_f32_16x16x32_f16 a[248:251], v[144:147], v[156:159], a[248:251]\n\t" \
    "v_add_f32 v236, 1.0, v236\n\t" \
    "v_add_f32 v237, 1.0, v237\n\t" \
    "v_mfma_f32_16x16x32_f16 a[216:219], v[140:143], v[156:159], a[216:219]\n\t" \
    "v_add_f32 v238, 1.0, v238\n\t" \
    "v_add_f32 v239, 1.0, v239\n\t" \
    "v_mfma_f32_16x16x32_f16 a[184:187], v[136:139], v[156:159], a[184:187]\n\t" \
    "v_rcp_f32 v236, v236\n\t" \
    "v_mfma_f32_16x16x32_f16 a[152:155], v[132:135], v[156:159], a[152:155]\n\t" \
    "buffer_load_dwordx4 v[80:83], %[vA0], %[rWS], s55 offen offset:3072 sc0 sc1\n\t" \
    "v_mfma_f32_16x16x32_f16 a[120:123], v[128:131], v[156:159], a[120:123]\n\t" \
    "v_rcp_f32 v237, v237\n\t" \
    "v_mfma_f32_16x16x32_f16 a[88:91], v[124:127], v[156:159], a[88:91]\n\t" \
    "s_add_u32 s55, s38, 16384\n\t" \
    "v_mfma_f32_16x16x32_f16 a[56:59], v[120:123], v[156:159], a[56:59]\n\t" \
    "v_rcp_f32 v238, v238\n\t" \
    "v_mfma_f32_16x16x32_f16 a[24:27], v[116:119], v[156:159], a[24:27]\n\t" \
    "ds_read_b128 v[156:159], %[vA0] offset:2048\n\t" \
    "v_mfma_f32_16x16x32_f16 a[28:31], v[116:119], v[160:163], a[28:31]\n\t" \
    "v_rcp_f32 v239, v239\n\t" \
    "v_mfma_f32_16x16x32_f16 a[60:63], v[120:123], v[160:163], a[60:63]\n\t" \
    "buffer_load_dwordx4 v[84:87], %[vA0], %[rWS], s55 offen offset:0 sc0 sc1\n\t" \
    "v_mfma_f32_16x16x32_f16 a[92:95], v[124:127], v[160:163], a[92:95]\n\t" \
    "v_mul_f32 v232, v252, v236\n\t" \
    "v_mul_f32 v233, v253, v237\n\t" \
    "v_mfma_f32_16x16x32_f16 a[124:127], v[128:131], v[160:163], a[124:127]\n\t" \
    "v_mul_f32 v234, v254, v238\n\t" \
    "v_mfma_f32_16x16x32_f16 a[156:159], v[132:135], v[160:163], a[156:159]\n\t" \
    "buffer_load_dwordx4 v[88:91], %[vA0], %[rWS], s55 offen offset:1024 sc0 sc1\n\t" \
    "v_mfma_f32_16x16x32_f16 a[188:191], v[136:139], v[160:163], a[188:191]\n\t" \
    "v_mul_f32 v235, v255, v239\n\t" \
    "v_mfma_f32_16x16x32_f16 a[220:223], v[140:143], v[160:163], a[220:223]\n\t" \
    "buffer_load_dwordx4 v[92:95], %[vA0], %[rWS], s55 offen offset:2048 sc0 sc1\n\t" \
    "v_mfma_f32_16x16x32_f16 a[252:255], v[144:147], v[160:163], a[252:255]\n\t" \
    "buffer_load_dwordx4 v[96:99], %[vA0], %[rWS], s55 offen offset:3072 sc0 sc1\n\t" \
    "s_add_u32 s36, s36, 1\n\t" \
    "s_add_u32 s50, s50, 33554432\n\t" \
    "s_add_u32 s48, s48, 1\n\t" \
    "s_cmp_lt_u32 s48, 4\n\t" \
    "s_cbranch_scc1 LOOPQ%=\n\t" \
    "LASTT%=:\n\t" \
    "ds_read_b128 v[160:163], %[vA0] offset:35840\n\t" \
    "s_waitcnt lgkmcnt(3)\n\t" \
    "v_mfma_f32_16x16x32_f16 a[224:227], v[48:51], v[148:151], a[224:227]\n\t" \
    "s_mov_b32 s37, s38\n\t" \
    "v_mfma_f32_16x16x32_f16 a[192:195], v[44:47], v[148:151], a[192:195]\n\t" \
    "s_add_u32 s55, s37, 20480\n\t" \
    "v_mfma_f32_16x16x32_f16 a[160:163], v[40:43], v[148:151], a[160:163]\n\t" \
    "buffer_load_dwordx4 v[100:103], %[vA0], %[rWS], s55 offen offset:0 sc0 sc1\n\t" \
    "v_mfma_f32_16x16x32_f16 a[128:131], v[36:39], v[148:151], a[128:131]\n\t" \
    "v_mfma_f32_16x16x32_f16 a[96:99], v[32:35], v[148:151], a[96:99]\n\t" \
    "buffer_load_dwordx4 v[104:107], %[vA0], %[rWS], s55 offen offset:1024 sc0 sc1\n\t" \
    "v_mfma_f32_16x16x32_f16 a[64:67], v[28:31], v[148:151], a[64:67]\n\t" \
    "v_mfma_f32_16x16x32_f16 a[32:35], v[24:27], v[148:151], a[32:35]\n\t" \
    "s_cmp_ge_u32 s48, 2\n\ts_cselect_b64 exec, -1, 0\n\ts_cselect_b32 s69, %[own], 9\n\tds_read_b128 v[252:255], %[vStg] offset:23552\n\ts_mov_b64 exec, -1\n\t" \
    "s_waitcnt lgkmcnt(3)\n\t" \
    "v_mfma_f32_16x16x32_f16 a[0:3], v[20:23], v[148:151], a[0:3]\n\t" \
    "ds_read_b128 v[148:151], %[vA0] offset:40960\n\t" \
    "s_waitcnt lgkmcnt(3)\n\t" \
    "v_mfma_f32_16x16x32_f16 a[4:7], v[20:23], v[152:155], a[4:7]\n\t" \
    "buffer_load_dwordx4 v[108:111], %[vA0], %[rWS], s55 offen offset:2048 sc0 sc1\n\t" \
    "v_mfma_f32_16x16x32_f16 a[36:39], v[24:27], v[152:155], a[36:39]\n\t" \
    "v_mfma_f32_16x16x32_f16 a[68:71], v[28:31], v[152:155], a[68:71]\n\t" \
    "buffer_load_dwordx4 v[112:115], %[vA0], %[rWS], s55 offen offset:3072 sc0 sc1\n\t" \
    "v_mfma_f32_16x16x32_f16 a[100:103], v[32:35], v[152:155], a[100:103]\n\t" \
    "v_mfma_f32_16x16x32_f16 a[132:135], v[36:39], v[152:155], a[132:135]\n\t" \
    "v_mfma_f32_16x16x32_f16 a[164:167], v[40:43], v[152:155], a[164:167]\n\t" \
    "s_add_u32 s55, s37, 24576\n\t" \
    "v_mfma_f32_16x16x32_f16 a[196:199], v[44:47], v[152:155], a[196:199]\n\t" \
    "buffer_load_dwordx4 v[116:119], %[vA0], %[rWS], s55 offen offset:0 sc0 sc1\n\t" \
    "v_mfma_f32_16x16x32_f16 a[228:231], v[48:51], v[152:155], a[228:231]\n\t" \
    "ds_read_b128 v[152:155], %[vA0] offset:41984\n\t" \
    "s_waitcnt lgkmcnt(3)\n\t" \
    "v_mfma_f32_16x16x32_f16 a[232:235], v[48:51], v[156:159], a[232:235]\n\t" \
    "buffer_load_dwordx4 v[120:123], %[vA0], %[rWS], s55 offen offset:1024 sc0 sc1\n\t" \
    "v_mfma_f32_16x16x32_f16 a[200:203], v[44:47], v[156:159], a[200:203]\n\t" \
    "v_mfma_f32_16x16x32_f16 a[168:171], v[40:43], v[156:159], a[168:171]\n\t" \
    "v_mfma_f32_16x16x32_f16 a[136:139], v[36:39], v[156:159], a[136:139]\n\t" \
    "buffer_load_dwordx4 v[124:127], %[vA0], %[rWS], s55 offen offset:2048 sc0 sc1\n\t" \
    "v_mfma_f32_16x16x32_f16 a[104:107], v[32:35], v[156:159], a[104:107]\n\t" \
    "v_mfma_f32_16x16x32_f16 a[72:75], v[28:31], v[156:159], a[72:75]\n\t" \
    "v_mfma_f32_16x16x32_f16 a[40:43], v[24:27], v[156:159], a[40:43]\n\t" \
    "buffer_load_dwordx4 v[128:131], %[vA0], %[rWS], s55 offen offset:3072 sc0 sc1\n\t" \
    "v_mfma_f32_16x16x32_f16 a[8:11], v[20:23], v[156:159], a[8:11]\n\t" \
    "ds_read_b128 v[156:159], %[vA0] offset:43008\n\t" \
    "s_waitcnt lgkmcnt(3)\n\t" \
    "v_mfma_f32_16x16x32_f16 a[12:15], v[20:23], v[160:163], a[12:15]\n\t" \
    "v_mfma_f32_16x16x32_f16 a[44:47], v[24:27], v[160:163], a[44:47]\n\t" \
    "s_add_u32 s55, s37, 28672\n\t" \
    "v_mfma_f32_16x16x32_f16 a[76:79], v[28:31], v[160:163], a[76:79]\n\t" \
    "buffer_load_dwordx4 v[132:135], %[vA0], %[rWS], s55 offen offset:0 sc0 sc1\n\t" \
    "v_mfma_f32_16x16x32_f16 a[108:111], v[32:35], v[160:163], a[108:111]\n\t" \
    "v_mfma_f32_16x16x32_f16 a[140:143], v[36:39], v[160:163], a[140:143]\n\t" \
    "buffer_load_dwordx4 v[136:139], %[vA0], %[rWS], s55 offen offset:1024 sc0 sc1\n\t" \
    "v_mfma_f32_16x16x32_f16 a[172:175], v[40:43], v[160:163], a[172:175]\n\t" \
    "v_mfma_f32_16x16x32_f16 a[204:207], v[44:47], v[160:163], a[204:207]\n\t" \
    "v_mfma_f32_16x16x32_f16 a[236:239], v[48:51], v[160:163], a[236:239]\n\t" \
    "ds_read_b128 v[160:163], %[vA0] offset:44032\n\t" \
    "s_waitcnt vmcnt(14) lgkmcnt(3)\n\t" \
    "v_mfma_f32_16x16x32_f16 a[224:227], v[80:83], v[148:151], a[224:227]\n\t" \
    "buffer_load_dwordx4 v[140:143], %[vA0], %[rWS], s55 offen offset:2048 sc0 sc1\n\t" \
    "v_mfma_f32_16x16x32_f16 a[192:195], v[76:79], v[148:151], a[192:195]\n\t" \
    "v_mfma_f32_16x16x32_f16 a[160:163], v[72:75], v[148:151], a[160:163]\n\t" \
    "buffer_load_dwordx4 v[144:147], %[vA0], %[rWS], s55 offen offset:3072 sc0 sc1\n\t" \
    "v_mfma_f32_16x16x32_f16 a[128:131], v[68:71], v[148:151], a[128:131]\n\t" \
    "v_mfma_f32_16x16x32_f16 a[96:99], v[64:67], v[148:151], a[96:99]\n\t" \
    "v_mfma_f32_16x16x32_f16 a[64:67], v[60:63], v[148:151], a[64:67]\n\t" \
    "v_mfma_f32_16x16x32_f16 a[32:35], v[56:59], v[148:151], a[32:35]\n\t" \
    "v_mfma_f32_16x16x32_f16 a[0:3], v[52:55], v[148:151], a[0:3]\n\t" \
    "ds_read_b128 v[148:151], %[vA0] offset:49152\n\t" \
    "s_waitcnt lgkmcnt(3)\n\t" \
    "v_mfma_f32_16x16x32_f16 a[4:7], v[52:55], v[152:155], a[4:7]\n\t" \
    "v_mfma_f32_16x16x32_f16 a[36:39], v[56:59], v[152:155], a[36:39]\n\t" \
    "v_mfma_f32_16x16x32_f16 a[68:71], v[60:63], v[152:155], a[68:71]\n\t" \
    "buffer_store_dwordx4 v[232:235], %[vO], s[56:59], 0 offen sc0 sc1 nt\n\t" \
    "v_mfma_f32_16x16x32_f16 a[100:103], v[64:67], v[152:155], a[100:103]\n\t" \
    "v_mfma_f32_16x16x32_f16 a[132:135], v[68:71], v[152:155], a[132:135]\n\t" \
    "v_readlane_b32 s52, %[vTab0], s51\n\t" \
    "v_mfma_f32_16x16x32_f16 a[164:167], v[72:75], v[152:155], a[164:167]\n\t" \
    "s_add_u32 s42, s51, 32\n\t" \
    "v_mfma_f32_16x16x32_f16 a[196:199], v[76:79], v[152:155], a[196:199]\n\t" \
    "v_readlane_b32 s53, %[vTab1], s51\n\t" \
    "v_mfma_f32_16x16x32_f16 a[228:231], v[80:83], v[152:155], a[228:231]\n\t" \
    "ds_read_b128 v[152:155], %[vA0] offset:50176\n\t" \
    "s_waitcnt lgkmcnt(3)\n\t" \
    "v_mfma_f32_16x16x32_f16 a[232:235], v[80:83], v[156:159], a[232:235]\n\t" \
    "v_readlane_b32 s42, %[vTab0], s42\n\t" \
    "v_mfma_f32_16x16x32_f16 a[200:203], v[76:79], v[156:159], a[200:203]\n\t" \
    "s_add_u32 s51, s51, 1\n\t" \
    "v_mfma_f32_16x16x32_f16 a[168:171], v[72:75], v[156:159], a[168:171]\n\t" \
    "v_mfma_f32_16x16x32_f16 a[136:139], v[68:71], v[156:159], a[136:139]\n\t" \
    "v_mfma_f32_16x16x32_f16 a[104:107], v[64:67], v[156:159], a[104:107]\n\t" \
    "s_mov_b32 s56, s42\n\t" \
    "s_and_b32 s57, s53, 0xffff\n\t" \
    "v_mfma_f32_16x16x32_f16 a[72:75], v[60:63], v[156:159], a[72:75]\n\t" \
    "s_and_b32 s58, s53, 0x10000\n\t" \
    "s_bfe_i32 s53, s53, 0x10011\n\t" \
    "v_mfma_f32_16x16x32_f16 a[40:43], v[56:59], v[156:159], a[40:43]\n\t" \
    "v_mfma_f32_16x16x32_f16 a[8:11], v[52:55], v[156:159], a[8:11]\n\t" \
    "ds_read_b128 v[156:159], %[vA0] offset:51200\n\t" \
    "s_waitcnt lgkmcnt(3)\n\t" \
    "v_mfma_f32_16x16x32_f16 a[12:15], v[52:55], v[160:163], a[12:15]\n\t" \
    "v_mfma_f32_16x16x32_f16 a[44:47], v[56:59], v[160:163], a[44:47]\n\t" \
    "v_mfma_f32_16x16x32_f16 a[76:79], v[60:63], v[160:163], a[76:79]\n\t" \
    "v_mfma_f32_16x16x32_f16 a[108:111], v[64:67], v[160:163], a[108:111]\n\t" \
    "v_mfma_f32_16x16x32_f16 a[140:143], v[68:71], v[160:163], a[140:143]\n\t" \
    "v_mfma_f32_16x16x32_f16 a[172:175], v[72:75], v[160:163], a[172:175]\n\t" \
    "v_mfma_f32_16x16x32_f16 a[204:207], v[76:79], v[160:163], a[204:207]\n\t" \
    "v_mfma_f32_16x16x32_f16 a[236:239], v[80:83], v[160:163], a[236:239]\n\t" \
    "ds_read_b128 v[160:163], %[vA0] offset:52224\n\t" \
    "s_waitcnt vmcnt(9) lgkmcnt(3)\n\t" \
    "v_mfma_f32_16x16x32_f16 a[224:227], v[112:115], v[148:151], a[224:227]\n\t" \
    "s_cmp_eq_u32 s69, 0\n\t" \
    "s_cselect_b32 s42, 1073741824, s52\n\t" \
    "v_mfma_f32_16x16x32_f16 a[192:195], v[108:111], v[148:151], a[192:195]\n\t" \
    "buffer_load_dwordx4 v[236:239], %[vA0], %[rWS], s42 offen sc0 sc1\n\t" \
    "v_mfma_f32_16x16x32_f16 a[160:163], v[104:107], v[148:151], a[160:163]\n\t" \
    "s_add_u32 s52, s52, 262144\n\t" \
    "s_cmp_eq_u32 s69, 1\n\t" \
    "v_mfma_f32_16x16x32_f16 a[128:131], v[100:103], v[148:151], a[128:131]\n\t" \
    "s_cselect_b32 s42, 1073741824, s52\n\t" \
    "v_mfma_f32_16x16x32_f16 a[96:99], v[96:99], v[148:151], a[96:99]\n\t" \
    "buffer_load_dwordx4 v[240:243], %[vA0], %[rWS], s42 offen sc0 sc1\n\t" \
    "v_mfma_f32_16x16x32_f16 a[64:67], v[92:95], v[148:151], a[64:67]\n\t" \
    "s_add_u32 s52, s52, 262144\n\t" \
    "s_cmp_eq_u32 s69, 2\n\t" \
    "v_mfma_f32_16x16x32_f16 a[32:35], v[88:91], v[148:151], a[32:35]\n\t" \
    "s_cselect_b32 s42, 1073741824, s52\n\t" \
    "v_mfma_f32_16x16x32_f16 a[0:3], v[84:87], v[148:151], a[0:3]\n\t" \
    "ds_read_b128 v[148:151], %[vA0] offset:57344\n\t" \
    "s_waitcnt lgkmcnt(3)\n\t" \
    "v_mfma_f32_16x16x32_f16 a[4:7], v[84:87], v[152:155], a[4:7]\n\t" \
    "buffer_load_dwordx4 v[244:247], %[vA0], %[rWS], s42 offen sc0 sc1\n\t" \
    "v_mfma_f32_16x16x32_f16 a[36:39], v[88:91], v[152:155], a[36:39]\n\t" \
    "s_add_u32 s52, s52, 262144\n\t" \
    "s_cmp_eq_u32 s69, 3\n\t" \
    "v_mfma_f32_16x16x32_f16 a[68:71], v[92:95], v[152:155], a[68:71]\n\t" \
    "s_cselect_b32 s42, 1073741824, s52\n\t" \
    "v_mfma_f32_16x16x32_f16 a[100:103], v[96:99], v[152:155], a[100:103]\n\t" \
    "buffer_load_dwordx4 v[248:251], %[vA0], %[rWS], s42 offen sc0 sc1\n\t" \
    "v_mfma_f32_16x16x32_f16 a[132:135], v[100:103], v[152:155], a[132:135]\n\t" \
    "v_mfma_f32_16x16x32_f16 a[164:167], v[104:107], v[152:155], a[164:167]\n\t" \
    "v_mfma_f32_16x16x32_f16 a[196:199], v[108:111], v[152:155], a[196:199]\n\t" \
    "v_mfma_f32_16x16x32_f16 a[228:231], v[112:115], v[152:155], a[228:231]\n\t" \
    "ds_read_b128 v[152:155], %[vA0] offset:58368\n\t" \
    "s_waitcnt lgkmcnt(3)\n\t" \
    "v_mfma_f32_16x16x32_f16 a[232:235], v[112:115], v[156:159], a[232:235]\n\t" \
    "v_mfma_f32_16x16x32_f16 a[200:203], v[108:111], v[156:159], a[200:203]\n\t" \
    "v_mfma_f32_16x16x32_f16 a[168:171], v[104:107], v[156:159], a[168:171]\n\t" \
    "v_mfma_f32_16x16x32_f16 a[136:139], v[100:103], v[156:159], a[136:139]\n\t" \
    "v_mfma_f32_16x16x32_f16 a[104:107], v[96:99], v[156:159], a[104:107]\n\t" \
    "v_mfma_f32_16x16x32_f16 a[72:75], v[92:95], v[156:159], a[72:75]\n\t" \
    "v_mfma_f32_16x16x32_f16 a[40:43], v[88:91], v[156:159], a[40:43]\n\t" \
    "v_mfma_f32_16x16x32_f16 a[8:11], v[84:87], v[156:159], a[8:11]\n\t" \
    "ds_read_b128 v[156:159], %[vA0] offset:59392\n\t" \
    "s_waitcnt lgkmcnt(3)\n\t" \
    "v_mfma_f32_16x16x32_f16 a[12:15], v[84:87], v[160:163], a[12:15]\n\t" \
    "v_mfma_f32_16x16x32_f16 a[44:47], v[88:91], v[160:163], a[44:47]\n\t" \
    "v_mfma_f32_16x16x32_f16 a[76:79], v[92:95], v[160:163], a[76:79]\n\t" \
    "v_mfma_f32_16x16x32_f16 a[108:111], v[96:99], v[160:163], a[108:111]\n\t" \
    "v_mfma_f32_16x16x32_f16 a[140:143], v[100:103], v[160:163], a[140:143]\n\t" \
    "v_mfma_f32_16x16x32_f16 a[172:175], v[104:107], v[160:163], a[172:175]\n\t" \
    "v_mfma_f32_16x16x32_f16 a[204:207], v[108:111], v[160:163], a[204:207]\n\t" \
    "v_mfma_f32_16x16x32_f16 a[236:239], v[112:115], v[160:163], a[236:239]\n\t" \
    "ds_read_b128 v[160:163], %[vA0] offset:60416\n\t" \
    "s_waitcnt vmcnt(5) lgkmcnt(3)\n\t" \
    "v_mfma_f32_16x16x32_f16 a[224:227], v[144:147], v[148:151], a[224:227]\n\t" \
    "v_mfma_f32_16x16x32_f16 a[192:195], v[140:143], v[148:151], a[192:195]\n\t" \
    "v_mfma_f32_16x16x32_f16 a[160:163], v[136:139], v[148:151], a[160:163]\n\t" \
    "v_mfma_f32_16x16x32_f16 a[128:131], v[132:135], v[148:151], a[128:131]\n\t" \
    "v_mfma_f32_16x16x32_f16 a[96:99], v[128:131], v[148:151], a[96:99]\n\t" \
    "v_mfma_f32_16x16x32_f16 a[64:67], v[124:127], v[148:151], a[64:67]\n\t" \
    "v_mfma_f32_16x16x32_f16 a[32:35], v[120:123], v[148:151], a[32:35]\n\t" \
    "v_mfma_f32_16x16x32_f16 a[0:3], v[116:119], v[148:151], a[0:3]\n\t" \
    "ds_read_b128 v[148:151], %[vA0] offset:36864\n\t" \
    "s_waitcnt lgkmcnt(3)\n\t" \
    "v_mfma_f32_16x16x32_f16 a[4:7], v[116:119], v[152:155], a[4:7]\n\t" \
    "v_mfma_f32_16x16x32_f16 a[36:39], v[120:123], v[152:155], a[36:39]\n\t" \
    "v_mfma_f32_16x16x32_f16 a[68:71], v[124:127], v[152:155], a[68:71]\n\t" \
    "v_mfma_f32_16x16x32_f16 a[100:103], v[128:131], v[152:155], a[100:103]\n\t" \
    "v_mfma_f32_16x16x32_f16 a[132:135], v[132:135], v[152:155], a[132:135]\n\t" \
    "v_and_b32 v252, s53, v252\n\t" \
    "v_mfma_f32_16x16x32_f16 a[164:167], v[136:139], v[152:155], a[164:167]\n\t" \
    "v_and_b32 v253, s53, v253\n\t" \
    "v_mfma_f32_16x16x32_f16 a[196:199], v[140:143], v[152:155], a[196:199]\n\t" \
    "v_and_b32 v254, s53, v254\n\t" \
    "v_mfma_f32_16x16x32_f16 a[228:231], v[144:147], v[152:155], a[228:231]\n\t" \
    "ds_read_b128 v[152:155], %[vA0] offset:37888\n\t" \
    "s_waitcnt lgkmcnt(3)\n\t" \
    "v_mfma_f32_16x16x32_f16 a[232:235], v[144:147], v[156:159], a[232:235]\n\t" \
    "v_and_b32 v255, s53, v255\n\t" \
    "s_waitcnt vmcnt(2)\n\t" \
    "v_add_f32 v236, v236, v240\n\t" \
    "v_mfma_f32_16x16x32_f16 a[200:203], v[140:143], v[156:159], a[200:203]\n\t" \
    "v_add_f32 v237, v237, v241\n\t" \
    "v_mfma_f32_16x16x32_f16 a[168:171], v[136:139], v[156:159], a[168:171]\n\t" \
    "v_add_f32 v238, v238, v242\n\t" \
    "v_mfma_f32_16x16x32_f16 a[136:139], v[132:135], v[156:159], a[136:139]\n\t" \
    "v_add_f32 v239, v239, v243\n\t" \
    "v_mfma_f32_16x16x32_f16 a[104:107], v[128:131], v[156:159], a[104:107]\n\t" \
    "v_add_f32 v252, v252, v236\n\t" \
    "v_mfma_f32_16x16x32_f16 a[72:75], v[124:127], v[156:159], a[72:75]\n\t" \
    "v_add_f32 v253, v253, v237\n\t" \
    "v_mfma_f32_16x16x32_f16 a[40:43], v[120:123], v[156:159], a[40:43]\n\t" \
    "v_add_f32 v254, v254, v238\n\t" \
    "v_mfma_f32_16x16x32_f16 a[8:11], v[116:119], v[156:159], a[8:11]\n\t" \
    "ds_read_b128 v[156:159], %[vA0] offset:38912\n\t" \
    "s_waitcnt lgkmcnt(3)\n\t" \
    "v_mfma_f32_16x16x32_f16 a[12:15], v[116:119], v[160:163], a[12:15]\n\t" \
    "v_add_f32 v255, v255, v239\n\t" \
    "s_waitcnt vmcnt(0)\n\t" \
    "v_add_f32 v244, v244, v248\n\t" \
    "v_mfma_f32_16x16x32_f16 a[44:47], v[120:123], v[160:163], a[44:47]\n\t" \
    "v_add_f32 v245, v245, v249\n\t" \
    "v_mfma_f32_16x16x32_f16 a[76:79], v[124:127], v[160:163], a[76:79]\n\t" \
    "v_add_f32 v246, v246, v250\n\t" \
    "v_mfma_f32_16x16x32_f16 a[108:111], v[128:131], v[160:163], a[108:111]\n\t" \
    "v_add_f32 v247, v247, v251\n\t" \
    "v_mfma_f32_16x16x32_f16 a[140:143], v[132:135], v[160:163], a[140:143]\n\t" \
    "v_add_f32 v252, v252, v244\n\t" \
    "v_mfma_f32_16x16x32_f16 a[172:175], v[136:139], v[160:163], a[172:175]\n\t" \
    "v_add_f32 v253, v253, v245\n\t" \
    "v_mfma_f32_16x16x32_f16 a[204:207], v[140:143], v[160:163], a[204:207]\n\t" \
    "v_add_f32 v254, v254, v246\n\t" \
    "v_mfma_f32_16x16x32_f16 a[236:239], v[144:147], v[160:163], a[236:239]\n\t" \
    "ds_read_b128 v[160:163], %[vA0] offset:39936\n\t" \
    "s_waitcnt lgkmcnt(3)\n\t" \
    "v_mfma_f32_16x16x32_f16 a[240:243], v[48:51], v[148:151], a[240:243]\n\t" \
    "v_add_f32 v255, v255, v247\n\t" \
    "v_med3_f32 v236, v252, %[m60], %[p60]\n\t" \
    "v_mfma_f32_16x16x32_f16 a[208:211], v[44:47], v[148:151], a[208:211]\n\t" \
    "v_med3_f32 v237, v253, %[m60], %[p60]\n\t" \
    "v_mfma_f32_16x16x32_f16 a[176:179], v[40:43], v[148:151], a[176:179]\n\t" \
    "v_med3_f32 v238, v254, %[m60], %[p60]\n\t" \
    "v_mfma_f32_16x16x32_f16 a[144:147], v[36:39], v[148:151], a[144:147]\n\t" \
    "v_med3_f32 v239, v255, %[m60], %[p60]\n\t" \
    "v_mfma_f32_16x16x32_f16 a[112:115], v[32:35], v[148:151], a[112:115]\n\t" \
    "s_add_u32 s60, s50, 33554432\n\t" \
    "s_cmp_eq_u32 %[own], 0\n\t" \
    "v_mfma_f32_16x16x32_f16 a[80:83], v[28:31], v[148:151], a[80:83]\n\t" \
    "s_cselect_b32 s60, 1073741824, s60\n\t" \
    "v_mul_f32 v236, 0xbfb8aa3b, v236\n\t" \
    "v_mfma_f32_16x16x32_f16 a[48:51], v[24:27], v[148:151], a[48:51]\n\t" \
    "v_mul_f32 v237, 0xbfb8aa3b, v237\n\t" \
    "v_mfma_f32_16x16x32_f16 a[16:19], v[20:23], v[148:151], a[16:19]\n\t" \
    "ds_read_b128 v[148:151], %[vA0] offset:45056\n\t" \
    "s_waitcnt lgkmcnt(3)\n\t" \
    "v_mfma_f32_16x16x32_f16 a[20:23], v[20:23], v[152:155], a[20:23]\n\t" \
    "buffer_store_dwordx4 a[0:3], %[vA0], %[rWS], s60 offen offset:0 sc0 sc1\n\t" \
    "v_mfma_f32_16x16x32_f16 a[52:55], v[24:27], v[152:155], a[52:55]\n\t" \
    "v_mul_f32 v238, 0xbfb8aa3b, v238\n\t" \
    "v_mul_f32 v239, 0xbfb8aa3b, v239\n\t" \
    "v_mfma_f32_16x16x32_f16 a[84:87], v[28:31], v[152:155], a[84:87]\n\t" \
    "buffer_store_dwordx4 a[4:7], %[vA0], %[rWS], s60 offen offset:1024 sc0 sc1\n\t" \
    "v_mfma_f32_16x16x32_f16 a[116:119], v[32:35], v[152:155], a[116:119]\n\t" \
    "v_exp_f32 v236, v236\n\t" \
    "v_mfma_f32_16x16x32_f16 a[148:151], v[36:39], v[152:155], a[148:151]\n\t" \
    "v_exp_f32 v237, v237\n\t" \
    "v_mfma_f32_16x16x32_f16 a[180:183], v[40:43], v[152:155], a[180:183]\n\t" \
    "buffer_store_dwordx4 a[8:11], %[vA0], %[rWS], s60 offen offset:2048 sc0 sc1\n\t" \
    "v_mfma_f32_16x16x32_f16 a[212:215], v[44:47], v[152:155], a[212:215]\n\t" \
    "v_exp_f32 v238, v238\n\t" \
    "v_mfma_f32_16x16x32_f16 a[244:247], v[48:51], v[152:155], a[244:247]\n\t" \
    "ds_read_b128 v[152:155], %[vA0] offset:46080\n\t" \
    "s_waitcnt lgkmcnt(3)\n\t" \
    "v_mfma_f32_16x16x32_f16 a[248:251], v[48:51], v[156:159], a[248:251]\n\t" \
    "v_exp_f32 v239, v239\n\t" \
    "v_mfma_f32_16x16x32_f16 a[216:219], v[44:47], v[156:159], a[216:219]\n\t" \
    "buffer_store_dwordx4 a[12:15], %[vA0], %[rWS], s60 offen offset:3072 sc0 sc1\n\t" \
    "v_mfma_f32_16x16x32_f16 a[184:187], v[40:43], v[156:159], a[184:187]\n\t" \
    "s_add_u32 s60, s50, 33562624\n\t" \
    "s_cmp_eq_u32 %[own], 1\n\t" \
    "v_mfma_f32_16x16x32_f16 a[152:155], v[36:39], v[156:159], a[152:155]\n\t" \
    "s_cselect_b32 s60, 1073741824, s60\n\t" \
    "v_add_f32 v236, 1.0, v236\n\t" \
    "v_mfma_f32_16x16x32_f16 a[120:123], v[32:35], v[156:159], a[120:123]\n\t" \
    "v_add_f32 v237, 1.0, v237\n\t" \
    "v_mfma_f32_16x16x32_f16 a[88:91], v[28:31], v[156:159], a[88:91]\n\t" \
    "buffer_store_dwordx4 a[64:67], %[vA0], %[rWS], s60 offen offset:0 sc0 sc1\n\t" \
    "v_mfma_f32_16x16x32_f16 a[56:59], v[24:27], v[156:159], a[56:59]\n\t" \
    "v_add_f32 v238, 1.0, v238\n\t" \
    "v_add_f32 v239, 1.0, v239\n\t" \
    "v_mfma_f32_16x16x32_f16 a[24:27], v[20:23], v[156:159], a[24:27]\n\t" \
    "ds_read_b128 v[156:159], %[vA0] offset:47104\n\t" \
    "s_waitcnt lgkmcnt(3)\n\t" \
    "v_mfma_f32_16x16x32_f16 a[28:31], v[20:23], v[160:163], a[28:31]\n\t" \
    "buffer_store_dwordx4 a[68:71], %[vA0], %[rWS], s60 offen offset:1024 sc0 sc1\n\t" \
    "v_mfma_f32_16x16x32_f16 a[60:63], v[24:27], v[160:163], a[60:63]\n\t" \
    "v_rcp_f32 v236, v236\n\t" \
    "v_mfma_f32_16x16x32_f16 a[92:95], v[28:31], v[160:163], a[92:95]\n\t" \
    "v_rcp_f32 v237, v237\n\t" \
    "v_mfma_f32_16x16x32_f16 a[124:127], v[32:35], v[160:163], a[124:127]\n\t" \
    "buffer_store_dwordx4 a[72:75], %[vA0], %[rWS], s60 offen offset:2048 sc0 sc1\n\t" \
    "v_mfma_f32_16x16x32_f16 a[156:159], v[36:39], v[160:163], a[156:159]\n\t" \
    "v_rcp_f32 v238, v238\n\t" \
    "v_mfma_f32_16x16x32_f16 a[188:191], v[40:43], v[160:163], a[188:191]\n\t" \
    "v_rcp_f32 v239, v239\n\t" \
    "v_mfma_f32_16x16x32_f16 a[220:223], v[44:47], v[160:163], a[220:223]\n\t" \
    "buffer_store_dwordx4 a[76:79], %[vA0], %[rWS], s60 offen offset:3072 sc0 sc1\n\t" \
    "v_mfma_f32_16x16x32_f16 a[252:255], v[48:51], v[160:163], a[252:255]\n\t" \
    "ds_read_b128 v[160:163], %[vA0] offset:48128\n\t" \
    "s_waitcnt lgkmcnt(3)\n\t" \
    "v_mfma_f32_16x16x32_f16 a[240:243], v[80:83], v[148:151], a[240:243]\n\t" \
    "s_add_u32 s60, s50, 33570816\n\t" \
    "s_cmp_eq_u32 %[own], 2\n\t" \
    "v_mfma_f32_16x16x32_f16 a[208:211], v[76:79], v[148:151], a[208:211]\n\t" \
    "s_cselect_b32 s60, 1073741824, s60\n\t" \
    "v_mul_f32 v232, v252, v236\n\t" \
    "v_mfma_f32_16x16x32_f16 a[176:179], v[72:75], v[148:151], a[176:179]\n\t" \
    "v_mul_f32 v233, v253, v237\n\t" \
    "v_mfma_f32_16x16x32_f16 a[144:147], v[68:71], v[148:151], a[144:147]\n\t" \
    "buffer_store_dwordx4 a[128:131], %[vA0], %[rWS], s60 offen offset:0 sc0 sc1\n\t" \
    "v_mfma_f32_16x16x32_f16 a[112:115], v[64:67], v[148:151], a[112:115]\n\t" \
    "v_mul_f32 v234, v254, v238\n\t" \
    "v_mul_f32 v235, v255, v239\n\t" \
    "v_mfma_f32_16x16x32_f16 a[80:83], v[60:63], v[148:151], a[80:83]\n\t" \
    "buffer_store_dwordx4 a[132:135], %[vA0], %[rWS], s60 offen offset:1024 sc0 sc1\n\t" \
    "v_mfma_f32_16x16x32_f16 a[48:51], v[56:59], v[148:151], a[48:51]\n\t" \
    "buffer_store_dwordx4 a[136:139], %[vA0], %[rWS], s60 offen offset:2048 sc0 sc1\n\t" \
    "v_mfma_f32_16x16x32_f16 a[16:19], v[52:55], v[148:151], a[16:19]\n\t" \
    "ds_read_b128 v[148:151], %[vA0] offset:53248\n\t" \
    "s_waitcnt lgkmcnt(3)\n\t" \
    "v_mfma_f32_16x16x32_f16 a[20:23], v[52:55], v[152:155], a[20:23]\n\t" \
    "buffer_store_dwordx4 a[140:143], %[vA0], %[rWS], s60 offen offset:3072 sc0 sc1\n\t" \
    "v_mfma_f32_16x16x32_f16 a[52:55], v[56:59], v[152:155], a[52:55]\n\t" \
    "s_add_u32 s60, s50, 33579008\n\t" \
    "s_cmp_eq_u32 %[own], 3\n\t" \
    "v_mfma_f32_16x16x32_f16 a[84:87], v[60:63], v[152:155], a[84:87]\n\t" \
    "s_cselect_b32 s60, 1073741824, s60\n\t" \
    "v_mfma_f32_16x16x32_f16 a[116:119], v[64:67], v[152:155], a[116:119]\n\t" \
    "buffer_store_dwordx4 a[192:195], %[vA0], %[rWS], s60 offen offset:0 sc0 sc1\n\t" \
    "v_mfma_f32_16x16x32_f16 a[148:151], v[68:71], v[152:155], a[148:151]\n\t" \
    "buffer_store_dwordx4 a[196:199], %[vA0], %[rWS], s60 offen offset:1024 sc0 sc1\n\t" \
    "v_mfma_f32_16x16x32_f16 a[180:183], v[72:75], v[152:155], a[180:183]\n\t" \
    "buffer_store_dwordx4 a[200:203], %[vA0], %[rWS], s60 offen offset:2048 sc0 sc1\n\t" \
    "v_mfma_f32_16x16x32_f16 a[212:215], v[76:79], v[152:155], a[212:215]\n\t" \
    "buffer_store_dwordx4 a[204:207], %[vA0], %[rWS], s60 offen offset:3072 sc0 sc1\n\t" \
    "v_mfma_f32_16x16x32_f16 a[244:247], v[80:83], v[152:155], a[244:247]\n\t" \
    "ds_read_b128 v[152:155], %[vA0] offset:54272\n\t" \
    "s_waitcnt lgkmcnt(3)\n\t" \
    "v_mfma_f32_16x16x32_f16 a[248:251], v[80:83], v[156:159], a[248:251]\n\t" \
    "s_add_u32 s60, s50, 33558528\n\t" \
    "s_cmp_eq_u32 %[own], 0\n\t" \
    "v_mfma_f32_16x16x32_f16 a[216:219], v[76:79], v[156:159], a[216:219]\n\t" \
    "s_cselect_b32 s60, 1073741824, s60\n\t" \
    "v_mfma_f32_16x16x32_f16 a[184:187], v[72:75], v[156:159], a[184:187]\n\t" \
    "buffer_store_dwordx4 a[32:35], %[vA0], %[rWS], s60 offen offset:0 sc0 sc1\n\t" \
    "v_mfma_f32_16x16x32_f16 a[152:155], v[68:71], v[156:159], a[152:155]\n\t" \
    "buffer_store_dwordx4 a[36:39], %[vA0], %[rWS], s60 offen offset:1024 sc0 sc1\n\t" \
    "v_mfma_f32_16x16x32_f16 a[120:123], v[64:67], v[156:159], a[120:123]\n\t" \
    "buffer_store_dwordx4 a[40:43], %[vA0], %[rWS], s60 offen offset:2048 sc0 sc1\n\t" \
    "v_mfma_f32_16x16x32_f16 a[88:91], v[60:63], v[156:159], a[88:91]\n\t" \
    "buffer_store_dwordx4 a[44:47], %[vA0], %[rWS], s60 offen offset:3072 sc0 sc1\n\t" \
    "v_mfma_f32_16x16x32_f16 a[56:59], v[56:59], v[156:159], a[56:59]\n\t" \
    "s_add_u32 s60, s50, 33566720\n\t" \
    "s_cmp_eq_u32 %[own], 1\n\t" \
    "v_mfma_f32_16x16x32_f16 a[24:27], v[52:55], v[156:159], a[24:27]\n\t" \
    "ds_read_b128 v[156:159], %[vA0] offset:55296\n\t" \
    "s_waitcnt lgkmcnt(3)\n\t" \
    "v_mfma_f32_16x16x32_f16 a[28:31], v[52:55], v[160:163], a[28:31]\n\t" \
    "s_cselect_b32 s60, 1073741824, s60\n\t" \
    "v_mfma_f32_16x16x32_f16 a[60:63], v[56:59], v[160:163], a[60:63]\n\t" \
    "buffer_store_dwordx4 a[96:99], %[vA0], %[rWS], s60 offen offset:0 sc0 sc1\n\t" \
    "v_mfma_f32_16x16x32_f16 a[92:95], v[60:63], v[160:163], a[92:95]\n\t" \
    "buffer_store_dwordx4 a[100:103], %[vA0], %[rWS], s60 offen offset:1024 sc0 sc1\n\t" \
    "v_mfma_f32_16x16x32_f16 a[124:127], v[64:67], v[160:163], a[124:127]\n\t" \
    "buffer_store_dwordx4 a[104:107], %[vA0], %[rWS], s60 offen offset:2048 sc0 sc1\n\t" \
    "v_mfma_f32_16x16x32_f16 a[156:159], v[68:71], v[160:163], a[156:159]\n\t" \
    "buffer_store_dwordx4 a[108:111], %[vA0], %[rWS], s60 offen offset:3072 sc0 sc1\n\t" \
    "v_mfma_f32_16x16x32_f16 a[188:191], v[72:75], v[160:163], a[188:191]\n\t" \
    "s_add_u32 s60, s50, 33574912\n\t" \
    "s_cmp_eq_u32 %[own], 2\n\t" \
    "v_mfma_f32_16x16x32_f16 a[220:223], v[76:79], v[160:163], a[220:223]\n\t" \
    "s_cselect_b32 s60, 1073741824, s60\n\t" \
    "v_mfma_f32_16x16x32_f16 a[252:255], v[80:83], v[160:163], a[252:255]\n\t" \
    "ds_read_b128 v[160:163], %[vA0] offset:56320\n\t" \
    "s_waitcnt lgkmcnt(3)\n\t" \
    "v_mfma_f32_16x16x32_f16 a[240:243], v[112:115], v[148:151], a[240:243]\n\t" \
    "buffer_store_dwordx4 a[160:163], %[vA0], %[rWS], s60 offen offset:0 sc0 sc1\n\t" \
    "v_mfma_f32_16x16x32_f16 a[208:211], v[108:111], v[148:151], a[208:211]\n\t" \
    "buffer_store_dwordx4 a[164:167], %[vA0], %[rWS], s60 offen offset:1024 sc0 sc1\n\t" \
    "v_mfma_f32_16x16x32_f16 a[176:179], v[104:107], v[148:151], a[176:179]\n\t" \
    "buffer_store_dwordx4 a[168:171], %[vA0], %[rWS], s60 offen offset:2048 sc0 sc1\n\t" \
    "v_mfma_f32_16x16x32_f16 a[144:147], v[100:103], v[148:151], a[144:147]\n\t" \
    "buffer_store_dwordx4 a[172:175], %[vA0], %[rWS], s60 offen offset:3072 sc0 sc1\n\t" \
    "v_mfma_f32_16x16x32_f16 a[112:115], v[96:99], v[148:151], a[112:115]\n\t" \
    "s_add_u32 s60, s50, 33583104\n\t" \
    "s_cmp_eq_u32 %[own], 3\n\t" \
    "v_mfma_f32_16x16x32_f16 a[80:83], v[92:95], v[148:151], a[80:83]\n\t" \
    "s_cselect_b32 s60, 1073741824, s60\n\t" \
    "v_mfma_f32_16x16x32_f16 a[48:51], v[88:91], v[148:151], a[48:51]\n\t" \
    "buffer_store_dwordx4 a[224:227], %[vA0], %[rWS], s60 offen offset:0 sc0 sc1\n\t" \
    "v_mfma_f32_16x16x32_f16 a[16:19], v[84:87], v[148:151], a[16:19]\n\t" \
    "ds_read_b128 v[148:151], %[vA0] offset:61440\n\t" \
    "s_waitcnt lgkmcnt(3)\n\t" \
    "v_mfma_f32_16x16x32_f16 a[20:23], v[84:87], v[152:155], a[20:23]\n\t" \
    "buffer_store_dwordx4 a[228:231], %[vA0], %[rWS], s60 offen offset:1024 sc0 sc1\n\t" \
    "v_mfma_f32_16x16x32_f16 a[52:55], v[88:91], v[152:155], a[52:55]\n\t" \
    "buffer_store_dwordx4 a[232:235], %[vA0], %[rWS], s60 offen offset:2048 sc0 sc1\n\t" \
    "v_mfma_f32_16x16x32_f16 a[84:87], v[92:95], v[152:155], a[84:87]\n\t" \
    "buffer_store_dwordx4 a[236:239], %[vA0], %[rWS], s60 offen offset:3072 sc0 sc1\n\t" \
    "v_mfma_f32_16x16x32_f16 a[116:119], v[96:99], v[152:155], a[116:119]\n\t" \
    "v_mfma_f32_16x16x32_f16 a[148:151], v[100:103], v[152:155], a[148:151]\n\t" \
    "v_mfma_f32_16x16x32_f16 a[180:183], v[104:107], v[152:155], a[180:183]\n\t" \
    "v_mfma_f32_16x16x32_f16 a[212:215], v[108:111], v[152:155], a[212:215]\n\t" \
    "v_mfma_f32_16x16x32_f16 a[244:247], v[112:115], v[152:155], a[244:247]\n\t" \
    "ds_read_b128 v[152:155], %[vA0] offset:62464\n\t" \
    "s_waitcnt lgkmcnt(3)\n\t" \
    "v_mfma_f32_16x16x32_f16 a[248:251], v[112:115], v[156:159], a[248:251]\n\t" \
    "v_mfma_f32_16x16x32_f16 a[216:219], v[108:111], v[156:159], a[216:219]\n\t" \
    "v_mfma_f32_16x16x32_f16 a[184:187], v[104:107], v[156:159], a[184:187]\n\t" \
    "v_mfma_f32_16x16x32_f16 a[152:155], v[100:103], v[156:159], a[152:155]\n\t" \
    "v_mfma_f32_16x16x32_f16 a[120:123], v[96:99], v[156:159], a[120:123]\n\t" \
    "v_mfma_f32_16x16x32_f16 a[88:91], v[92:95], v[156:159], a[88:91]\n\t" \
    "v_mfma_f32_16x16x32_f16 a[56:59], v[88:91], v[156:159], a[56:59]\n\t" \
    "v_mfma_f32_16x16x32_f16 a[24:27], v[84:87], v[156:159], a[24:27]\n\t" \
    "ds_read_b128 v[156:159], %[vA0] offset:63488\n\t" \
    "s_waitcnt lgkmcnt(3)\n\t" \
    "v_mfma_f32_16x16x32_f16 a[28:31], v[84:87], v[160:163], a[28:31]\n\t" \
    "v_mfma_f32_16x16x32_f16 a[60:63], v[88:91], v[160:163], a[60:63]\n\t" \
    "v_mfma_f32_16x16x32_f16 a[92:95], v[92:95], v[160:163], a[92:95]\n\t" \
    "v_mfma_f32_16x16x32_f16 a[124:127], v[96:99], v[160:163], a[124:127]\n\t" \
    "v_mfma_f32_16x16x32_f16 a[156:159], v[100:103], v[160:163], a[156:159]\n\t" \
    "v_mfma_f32_16x16x32_f16 a[188:191], v[104:107], v[160:163], a[188:191]\n\t" \
    "v_mfma_f32_16x16x32_f16 a[220:223], v[108:111], v[160:163], a[220:223]\n\t" \
    "v_mfma_f32_16x16x32_f16 a[252:255], v[112:115], v[160:163], a[252:255]\n\t" \
    "ds_read_b128 v[160:163], %[vA0] offset:64512\n\t" \
    "s_waitcnt lgkmcnt(3)\n\t" \
    "v_mfma_f32_16x16x32_f16 a[240:243], v[144:147], v[148:151], a[240:243]\n\t" \
    "v_mfma_f32_16x16x32_f16 a[208:211], v[140:143], v[148:151], a[208:211]\n\t" \
    "v_mfma_f32_16x16x32_f16 a[176:179], v[136:139], v[148:151], a[176:179]\n\t" \
    "v_mfma_f32_16x16x32_f16 a[144:147], v[132:135], v[148:151], a[144:147]\n\t" \
    "v_mfma_f32_16x16x32_f16 a[112:115], v[128:131], v[148:151], a[112:115]\n\t" \
    "v_mfma_f32_16x16x32_f16 a[80:83], v[124:127], v[148:151], a[80:83]\n\t" \
    "v_mfma_f32_16x16x32_f16 a[48:51], v[120:123], v[148:151], a[48:51]\n\t" \
    "v_mfma_f32_16x16x32_f16 a[16:19], v[116:119], v[148:151], a[16:19]\n\t" \
    "s_waitcnt lgkmcnt(2)\n\t" \
    "v_mfma_f32_16x16x32_f16 a[20:23], v[116:119], v[152:155], a[20:23]\n\t" \
    "v_mfma_f32_16x16x32_f16 a[52:55], v[120:123], v[152:155], a[52:55]\n\t" \
    "v_mfma_f32_16x16x32_f16 a[84:87], v[124:127], v[152:155], a[84:87]\n\t" \
    "v_mfma_f32_16x16x32_f16 a[116:119], v[128:131], v[152:155], a[116:119]\n\t" \
    "v_mfma_f32_16x16x32_f16 a[148:151], v[132:135], v[152:155], a[148:151]\n\t" \
    "v_mfma_f32_16x16x32_f16 a[180:183], v[136:139], v[152:155], a[180:183]\n\t" \
    "v_mfma_f32_16x16x32_f16 a[212:215], v[140:143], v[152:155], a[212:215]\n\t" \
    "v_mfma_f32_16x16x32_f16 a[244:247], v[144:147], v[152:155], a[244:247]\n\t" \
    "s_waitcnt lgkmcnt(1)\n\t" \
    "v_mfma_f32_16x16x32_f16 a[248:251], v[144:147], v[156:159], a[248:251]\n\t" \
    "v_mfma_f32_16x16x32_f16 a[216:219], v[140:143], v[156:159], a[216:219]\n\t" \
    "v_mfma_f32_16x16x32_f16 a[184:187], v[136:139], v[156:159], a[184:187]\n\t" \
    "v_mfma_f32_16x16x32_f16 a[152:155], v[132:135], v[156:159], a[152:155]\n\t" \
    "v_mfma_f32_16x16x32_f16 a[120:123], v[128:131], v[156:159], a[120:123]\n\t" \
    "v_mfma_f32_16x16x32_f16 a[88:91], v[124:127], v[156:159], a[88:91]\n\t" \
    "v_mfma_f32_16x16x32_f16 a[56:59], v[120:123], v[156:159], a[56:59]\n\t" \
    "v_mfma_f32_16x16x32_f16 a[24:27], v[116:119], v[156:159], a[24:27]\n\t" \
    "s_waitcnt lgkmcnt(0)\n\t" \
    "v_mfma_f32_16x16x32_f16 a[28:31], v[116:119], v[160:163], a[28:31]\n\t" \
    "v_mfma_f32_16x16x32_f16 a[60:63], v[120:123], v[160:163], a[60:63]\n\t" \
    "v_mfma_f32_16x16x32_f16 a[92:95], v[124:127], v[160:163], a[92:95]\n\t" \
    "v_mfma_f32_16x16x32_f16 a[124:127], v[128:131], v[160:163], a[124:127]\n\t" \
    "v_mfma_f32_16x16x32_f16 a[156:159], v[132:135], v[160:163], a[156:159]\n\t" \
    "v_mfma_f32_16x16x32_f16 a[188:191], v[136:139], v[160:163], a[188:191]\n\t" \
    "v_mfma_f32_16x16x32_f16 a[220:223], v[140:143], v[160:163], a[220:223]\n\t" \
    "v_mfma_f32_16x16x32_f16 a[252:255], v[144:147], v[160:163], a[252:255]\n\t" \
    "s_nop 7\n\t" \
    "s_waitcnt lgkmcnt(0)\n\t" \
    "s_barrier\n\t" \
    "s_cmp_eq_u32 %[own], 0\n\t" \
    "s_cbranch_scc0 OWN0%=\n\t" \
    "ds_write_b128 %[vEnd], a[0:3] offset:32768\n\t" \
    "ds_write_b128 %[vEnd], a[4:7] offset:33792\n\t" \
    "ds_write_b128 %[vEnd], a[8:11] offset:34816\n\t" \
    "ds_write_b128 %[vEnd], a[12:15] offset:35840\n\t" \
    "ds_write_b128 %[vEnd], a[32:35] offset:36864\n\t" \
    "ds_write_b128 %[vEnd], a[36:39] offset:37888\n\t" \
    "ds_write_b128 %[vEnd], a[40:43] offset:38912\n\t" \
    "ds_write_b128 %[vEnd], a[44:47] offset:39936\n\t" \
    "OWN0%=:\n\t" \
    "s_cmp_eq_u32 %[own], 1\n\t" \
    "s_cbranch_scc0 OWN1%=\n\t" \
    "ds_write_b128 %[vEnd], a[64:67] offset:32768\n\t" \
    "ds_write_b128 %[vEnd], a[68:71] offset:33792\n\t" \
    "ds_write_b128 %[vEnd], a[72:75] offset:34816\n\t" \
    "ds_write_b128 %[vEnd], a[76:79] offset:35840\n\t" \
    "ds_write_b128 %[vEnd], a[96:99] offset:36864\n\t" \
    "ds_write_b128 %[vEnd], a[100:103] offset:37888\n\t" \
    "ds_write_b128 %[vEnd], a[104:107] offset:38912\n\t" \
    "ds_write_b128 %[vEnd], a[108:111] offset:39936\n\t" \
    "OWN1%=:\n\t" \
    "s_cmp_eq_u32 %[own], 2\n\t" \
    "s_cbranch_scc0 OWN2%=\n\t" \
    "ds_write_b128 %[vEnd], a[128:131] offset:32768\n\t" \
    "ds_write_b128 %[vEnd], a[132:135] offset:33792\n\t" \
    "ds_write_b128 %[vEnd], a[136:139] offset:34816\n\t" \
    "ds_write_b128 %[vEnd], a[140:143] offset:35840\n\t" \
    "ds_write_b128 %[vEnd], a[160:163] offset:36864\n\t" \
    "ds_write_b128 %[vEnd], a[164:167] offset:37888\n\t" \
    "ds_write_b128 %[vEnd], a[168:171] offset:38912\n\t" \
    "ds_write_b128 %[vEnd], a[172:175] offset:39936\n\t" \
    "OWN2%=:\n\t" \
    "s_cmp_eq_u32 %[own], 3\n\t" \
    "s_cbranch_scc0 OWN3%=\n\t" \
    "ds_write_b128 %[vEnd], a[192:195] offset:32768\n\t" \
    "ds_write_b128 %[vEnd], a[196:199] offset:33792\n\t" \
    "ds_write_b128 %[vEnd], a[200:203] offset:34816\n\t" \
    "ds_write_b128 %[vEnd], a[204:207] offset:35840\n\t" \
    "ds_write_b128 %[vEnd], a[224:227] offset:36864\n\t" \
    "ds_write_b128 %[vEnd], a[228:231] offset:37888\n\t" \
    "ds_write_b128 %[vEnd], a[232:235] offset:38912\n\t" \
    "ds_write_b128 %[vEnd], a[236:239] offset:39936\n\t" \
    "OWN3%=:\n\t" \
    "ds_write_b128 %[vEnd], a[16:19] offset:0\n\t" \
    "ds_write_b128 %[vEnd], a[20:23] offset:1024\n\t" \
    "ds_write_b128 %[vEnd], a[24:27] offset:2048\n\t" \
    "ds_write_b128 %[vEnd], a[28:31] offset:3072\n\t" \
    "ds_write_b128 %[vEnd], a[48:51] offset:4096\n\t" \
    "ds_write_b128 %[vEnd], a[52:55] offset:5120\n\t" \
    "ds_write_b128 %[vEnd], a[56:59] offset:6144\n\t" \
    "ds_write_b128 %[vEnd], a[60:63] offset:7168\n\t" \
    "ds_write_b128 %[vEnd], a[80:83] offset:8192\n\t" \
    "ds_write_b128 %[vEnd], a[84:87] offset:9216\n\t" \
    "ds_write_b128 %[vEnd], a[88:91] offset:10240\n\t" \
    "ds_write_b128 %[vEnd], a[92:95] offset:11264\n\t" \
    "ds_write_b128 %[vEnd], a[112:115] offset:12288\n\t" \
    "ds_write_b128 %[vEnd], a[116:119] offset:13312\n\t" \
    "ds_write_b128 %[vEnd], a[120:123] offset:14336\n\t" \
    "ds_write_b128 %[vEnd], a[124:127] offset:15360\n\t" \
    "ds_write_b128 %[vEnd], a[144:147] offset:16384\n\t" \
    "ds_write_b128 %[vEnd], a[148:151] offset:17408\n\t" \
    "ds_write_b128 %[vEnd], a[152:155] offset:18432\n\t" \
    "ds_write_b128 %[vEnd], a[156:159] offset:19456\n\t" \
    "ds_write_b128 %[vEnd], a[176:179] offset:20480\n\t" \
    "ds_write_b128 %[vEnd], a[180:183] offset:21504\n\t" \
    "ds_write_b128 %[vEnd], a[184:187] offset:22528\n\t" \
    "ds_write_b128 %[vEnd], a[188:191] offset:23552\n\t" \
    "ds_write_b128 %[vEnd], a[208:211] offset:24576\n\t" \
    "ds_write_b128 %[vEnd], a[212:215] offset:25600\n\t" \
    "ds_write_b128 %[vEnd], a[216:219] offset:26624\n\t" \
    "ds_write_b128 %[vEnd], a[220:223] offset:27648\n\t" \
    "ds_write_b128 %[vEnd], a[240:243] offset:28672\n\t" \
    "ds_write_b128 %[vEnd], a[244:247] offset:29696\n\t" \
    "ds_write_b128 %[vEnd], a[248:251] offset:30720\n\t" \
    "ds_write_b128 %[vEnd], a[252:255] offset:31744\n\t" \
    "s_add_u32 s49, s49, 1\n\t" \
    "v_mov_b32 v19, s49\n\t" \
    "s_waitcnt vmcnt(0)\n\t" \
    "buffer_store_dword v19, %[vFs], %[rWS], 0 offen sc0 sc1\n\t" \
    "s_waitcnt lgkmcnt(0)\n\t" \
    "s_barrier\n\t" \
    "buffer_store_dwordx4 v[232:235], %[vO], s[56:59], 0 offen sc0 sc1 nt\n\t" \
    "s_nop 1\n\t" \
    "s_mov_b32 %[ocall], s35\n\t" \
    "s_mov_b32 %[bad], s54\n\t" \
    "s_branch END%=\n\t" \
    "SLOW_PX%=:\n\t" \
    "s_memrealtime s[62:63]\n\t" \
    "SPIN_PX%=:\n\t" \
    "buffer_load_dword v19, off, %[rWS], s55 sc0 sc1\n\t" \
    "s_waitcnt vmcnt(0) lgkmcnt(0)\n\t" \
    "v_cmp_le_u32 s[76:77], s70, v19\n\t" \
    "s_nop 1\n\t" \
    "s_nop 0\n\t" \
    "s_cmp_eq_u64 s[76:77], exec\n\t" \
    "s_cbranch_scc1 BACK_PX%=\n\t" \
    "buffer_load_dword v19, off, %[rWS], %[sTmo] sc1\n\t" \
    "s_memrealtime s[64:65]\n\t" \
    "s_waitcnt vmcnt(0) lgkmcnt(0)\n\t" \
    "v_readfirstlane_b32 s65, v19\n\t" \
    "s_sub_u32 s64, s64, s62\n\t" \
    "s_nop 1\n\t" \
    "s_cmp_eq_u32 s65, s35\n\t" \
    "s_cbranch_scc1 BAD_PX%=\n\t" \
    "s_cmp_gt_u32 s64, %[ddl]\n\t" \
    "s_cbranch_scc1 BAD_PX%=\n\t" \
    "s_sleep 1\n\t" \
    "s_branch SPIN_PX%=\n\t" \
    "BAD_PX%=:\n\t" \
    "v_mov_b32 v19, s35\n\t" \
    "s_mov_b32 s54, 1\n\t" \
    "buffer_store_dword v19, off, %[rWS], %[sTmo] sc1\n\t" \
    "s_waitcnt vmcnt(0) lgkmcnt(0)\n\t" \
    "s_branch BACK_PX%=\n\t" \
    "SLOW_PQ0%=:\n\t" \
    "s_memrealtime s[62:63]\n\t" \
    "SPIN_PQ0%=:\n\t" \
    "buffer_load_dword v19, %[vXf], %[rWS], s34 offen sc0 sc1\n\t" \
    "s_waitcnt vmcnt(0) lgkmcnt(0)\n\t" \
    "v_cmp_le_u32 s[76:77], s70, v19\n\t" \
    "s_nop 1\n\t" \
    "s_nop 0\n\t" \
    "s_cmp_eq_u64 s[76:77], exec\n\t" \
    "s_cbranch_scc1 BACK_PQ0%=\n\t" \
    "buffer_load_dword v19, off, %[rWS], %[sTmo] sc1\n\t" \
    "s_memrealtime s[64:65]\n\t" \
    "s_waitcnt vmcnt(0) lgkmcnt(0)\n\t" \
    "v_readfirstlane_b32 s65, v19\n\t" \
    "s_sub_u32 s64, s64, s62\n\t" \
    "s_nop 1\n\t" \
    "s_cmp_eq_u32 s65, s35\n\t" \
    "s_cbranch_scc1 BAD_PQ0%=\n\t" \
    "s_cmp_gt_u32 s64, %[ddl]\n\t" \
    "s_cbranch_scc1 BAD_PQ0%=\n\t" \
    "s_sleep 1\n\t" \
    "s_branch SPIN_PQ0%=\n\t" \
    "BAD_PQ0%=:\n\t" \
    "v_mov_b32 v19, s35\n\t" \
    "s_mov_b32 s54, 1\n\t" \
    "buffer_store_dword v19, off, %[rWS], %[sTmo] sc1\n\t" \
    "s_waitcnt vmcnt(0) lgkmcnt(0)\n\t" \
    "s_branch BACK_PQ0%=\n\t" \
    "SLOW_PQ1%=:\n\t" \
    "s_memrealtime s[62:63]\n\t" \
    "SPIN_PQ1%=:\n\t" \
    "buffer_load_dword v19, %[vXf], %[rWS], s34 offen sc0 sc1\n\t" \
    "s_waitcnt vmcnt(0) lgkmcnt(0)\n\t" \
    "v_cmp_le_u32 s[76:77], s70, v19\n\t" \
    "s_nop 1\n\t" \
    "s_nop 0\n\t" \
    "s_cmp_eq_u64 s[76:77], exec\n\t" \
    "s_cbranch_scc1 BACK_PQ1%=\n\t" \
    "buffer_load_dword v19, off, %[rWS], %[sTmo] sc1\n\t" \
    "s_memrealtime s[64:65]\n\t" \
    "s_waitcnt vmcnt(0) lgkmcnt(0)\n\t" \
    "v_readfirstlane_b32 s65, v19\n\t" \
    "s_sub_u32 s64, s64, s62\n\t" \
    "s_nop 1\n\t" \
    "s_cmp_eq_u32 s65, s35\n\t" \
    "s_cbranch_scc1 BAD_PQ1%=\n\t" \
    "s_cmp_gt_u32 s64, %[ddl]\n\t" \
    "s_cbranch_scc1 BAD_PQ1%=\n\t" \
    "s_sleep 1\n\t" \
    "s_branch SPIN_PQ1%=\n\t" \
    "BAD_PQ1%=:\n\t" \
    "v_mov_b32 v19, s35\n\t" \
    "s_mov_b32 s54, 1\n\t" \
    "buffer_store_dword v19, off, %[rWS], %[sTmo] sc1\n\t" \
    "s_waitcnt vmcnt(0) lgkmcnt(0)\n\t" \
    "s_branch BACK_PQ1%=\n\t" \
    "FO1_0%=:\n\t" \
    "v_accvgpr_read_b32 v236, a16\n\t" \
    "v_accvgpr_read_b32 v237, a20\n\t" \
    "v_accvgpr_read_b32 v238, a24\n\t" \
    "v_accvgpr_read_b32 v239, a28\n\t" \
    "ds_write_b128 %[vStg], v[236:239] offset:16384\n\t" \
    "ds_write_b128 %[vStg], a[48:51] offset:0\n\t" \
    "v_accvgpr_read_b32 v240, a17\n\t" \
    "v_accvgpr_read_b32 v241, a21\n\t" \
    "v_accvgpr_read_b32 v242, a25\n\t" \
    "v_accvgpr_read_b32 v243, a29\n\t" \
    "ds_write_b128 %[vStg], v[240:243] offset:17408\n\t" \
    "ds_write_b128 %[vStg], a[52:55] offset:1024\n\t" \
    "v_accvgpr_read_b32 v244, a18\n\t" \
    "v_accvgpr_read_b32 v245, a22\n\t" \
    "v_accvgpr_read_b32 v246, a26\n\t" \
    "v_accvgpr_read_b32 v247, a30\n\t" \
    "ds_write_b128 %[vStg], v[244:247] offset:18432\n\t" \
    "ds_write_b128 %[vStg], a[56:59] offset:2048\n\t" \
    "v_accvgpr_read_b32 v248, a19\n\t" \
    "v_accvgpr_read_b32 v249, a23\n\t" \
    "v_accvgpr_read_b32 v250, a27\n\t" \
    "v_accvgpr_read_b32 v251, a31\n\t" \
    "ds_write_b128 %[vStg], v[248:251] offset:19456\n\t" \
    "ds_write_b128 %[vStg], a[60:63] offset:3072\n\t" \
    "s_branch FB1_0%=\n\t" \
    "FO1_2%=:\n\t" \
    "v_accvgpr_read_b32 v236, a80\n\t" \
    "v_accvgpr_read_b32 v237, a84\n\t" \
    "v_accvgpr_read_b32 v238, a88\n\t" \
    "v_accvgpr_read_b32 v239, a92\n\t" \
    "ds_write_b128 %[vStg], v[236:239] offset:16384\n\t" \
    "ds_write_b128 %[vStg], a[112:115] offset:4096\n\t" \
    "v_accvgpr_read_b32 v240, a81\n\t" \
    "v_accvgpr_read_b32 v241, a85\n\t" \
    "v_accvgpr_read_b32 v242, a89\n\t" \
    "v_accvgpr_read_b32 v243, a93\n\t" \
    "ds_write_b128 %[vStg], v[240:243] offset:17408\n\t" \
    "ds_write_b128 %[vStg], a[116:119] offset:5120\n\t" \
    "v_accvgpr_read_b32 v244, a82\n\t" \
    "v_accvgpr_read_b32 v245, a86\n\t" \
    "v_accvgpr_read_b32 v246, a90\n\t" \
    "v_accvgpr_read_b32 v247, a94\n\t" \
    "ds_write_b128 %[vStg], v[244:247] offset:18432\n\t" \
    "ds_write_b128 %[vStg], a[120:123] offset:6144\n\t" \
    "v_accvgpr_read_b32 v248, a83\n\t" \
    "v_accvgpr_read_b32 v249, a87\n\t" \
    "v_accvgpr_read_b32 v250, a91\n\t" \
    "v_accvgpr_read_b32 v251, a95\n\t" \
    "ds_write_b128 %[vStg], v[248:251] offset:19456\n\t" \
    "ds_write_b128 %[vStg], a[124:127] offset:7168\n\t" \
    "s_branch FB1_2%=\n\t" \
    "FO1_4%=:\n\t" \
    "v_accvgpr_read_b32 v236, a144\n\t" \
    "v_accvgpr_read_b32 v237, a148\n\t" \
    "v_accvgpr_read_b32 v238, a152\n\t" \
    "v_accvgpr_read_b32 v239, a156\n\t" \
    "ds_write_b128 %[vStg], v[236:239] offset:16384\n\t" \
    "ds_write_b128 %[vStg], a[176:179] offset:8192\n\t" \
    "v_accvgpr_read_b32 v240, a145\n\t" \
    "v_accvgpr_read_b32 v241, a149\n\t" \
    "v_accvgpr_read_b32 v242, a153\n\t" \
    "v_accvgpr_read_b32 v243, a157\n\t" \
    "ds_write_b128 %[vStg], v[240:243] offset:17408\n\t" \
    "ds_write_b128 %[vStg], a[180:183] offset:9216\n\t" \
    "v_accvgpr_read_b32 v244, a146\n\t" \
    "v_accvgpr_read_b32 v245, a150\n\t" \
    "v_accvgpr_read_b32 v246, a154\n\t" \
    "v_accvgpr_read_b32 v247, a158\n\t" \
    "ds_write_b128 %[vStg], v[244:247] offset:18432\n\t" \
    "ds_write_b128 %[vStg], a[184:187] offset:10240\n\t" \
    "v_accvgpr_read_b32 v248, a147\n\t" \
    "v_accvgpr_read_b32 v249, a151\n\t" \
    "v_accvgpr_read_b32 v250, a155\n\t" \
    "v_accvgpr_read_b32 v251, a159\n\t" \
    "ds_write_b128 %[vStg], v[248:251] offset:19456\n\t" \
    "ds_write_b128 %[vStg], a[188:191] offset:11264\n\t" \
    "s_branch FB1_4%=\n\t" \
    "FO1_6%=:\n\t" \
    "v_accvgpr_read_b32 v236, a208\n\t" \
    "v_accvgpr_read_b32 v237, a212\n\t" \
    "v_accvgpr_read_b32 v238, a216\n\t" \
    "v_accvgpr_read_b32 v239, a220\n\t" \
    "ds_write_b128 %[vStg], v[236:239] offset:16384\n\t" \
    "ds_write_b128 %[vStg], a[240:243] offset:12288\n\t" \
    "v_accvgpr_read_b32 v240, a209\n\t" \
    "v_accvgpr_read_b32 v241, a213\n\t" \
    "v_accvgpr_read_b32 v242, a217\n\t" \
    "v_accvgpr_read_b32 v243, a221\n\t" \
    "ds_write_b128 %[vStg], v[240:243] offset:17408\n\t" \
    "ds_write_b128 %[vStg], a[244:247] offset:13312\n\t" \
    "v_accvgpr_read_b32 v244, a210\n\t" \
    "v_accvgpr_read_b32 v245, a214\n\t" \
    "v_accvgpr_read_b32 v246, a218\n\t" \
    "v_accvgpr_read_b32 v247, a222\n\t" \
    "ds_write_b128 %[vStg], v[244:247] offset:18432\n\t" \
    "ds_write_b128 %[vStg], a[248:251] offset:14336\n\t" \
    "v_accvgpr_read_b32 v248, a211\n\t" \
    "v_accvgpr_read_b32 v249, a215\n\t" \
    "v_accvgpr_read_b32 v250, a219\n\t" \
    "v_accvgpr_read_b32 v251, a223\n\t" \
    "ds_write_b128 %[vStg], v[248:251] offset:19456\n\t" \
    "ds_write_b128 %[vStg], a[252:255] offset:15360\n\t" \
    "s_branch FB1_6%=\n\t" \
    "FO0_0%=:\n\t" \
    "v_accvgpr_read_b32 v236, a0\n\t" \
    "v_accvgpr_read_b32 v237, a4\n\t" \
    "v_accvgpr_read_b32 v238, a8\n\t" \
    "v_accvgpr_read_b32 v239, a12\n\t" \
    "ds_write_b128 %[vStg], v[236:239] offset:16384\n\t" \
    "ds_write_b128 %[vStg], a[32:35] offset:0\n\t" \
    "v_accvgpr_read_b32 v240, a1\n\t" \
    "v_accvgpr_read_b32 v241, a5\n\t" \
    "v_accvgpr_read_b32 v242, a9\n\t" \
    "v_accvgpr_read_b32 v243, a13\n\t" \
    "ds_write_b128 %[vStg], v[240:243] offset:17408\n\t" \
    "ds_write_b128 %[vStg], a[36:39] offset:1024\n\t" \
    "v_accvgpr_read_b32 v244, a2\n\t" \
    "v_accvgpr_read_b32 v245, a6\n\t" \
    "v_accvgpr_read_b32 v246, a10\n\t" \
    "v_accvgpr_read_b32 v247, a14\n\t" \
    "ds_write_b128 %[vStg], v[244:247] offset:18432\n\t" \
    "ds_write_b128 %[vStg], a[40:43] offset:2048\n\t" \
    "v_accvgpr_read_b32 v248, a3\n\t" \
    "v_accvgpr_read_b32 v249, a7\n\t" \
    "v_accvgpr_read_b32 v250, a11\n\t" \
    "v_accvgpr_read_b32 v251, a15\n\t" \
    "ds_write_b128 %[vStg], v[248:251] offset:19456\n\t" \
    "ds_write_b128 %[vStg], a[44:47] offset:3072\n\t" \
    "s_branch FB0_0%=\n\t" \
    "FO0_2%=:\n\t" \
    "v_accvgpr_read_b32 v236, a64\n\t" \
    "v_accvgpr_read_b32 v237, a68\n\t" \
    "v_accvgpr_read_b32 v238, a72\n\t" \
    "v_accvgpr_read_b32 v239, a76\n\t" \
    "ds_write_b128 %[vStg], v[236:239] offset:16384\n\t" \
    "ds_write_b128 %[vStg], a[96:99] offset:4096\n\t" \
    "v_accvgpr_read_b32 v240, a65\n\t" \
    "v_accvgpr_read_b32 v241, a69\n\t" \
    "v_accvgpr_read_b32 v242, a73\n\t" \
    "v_accvgpr_read_b32 v243, a77\n\t" \
    "ds_write_b128 %[vStg], v[240:243] offset:17408\n\t" \
    "ds_write_b128 %[vStg], a[100:103] offset:5120\n\t" \
    "v_accvgpr_read_b32 v244, a66\n\t" \
    "v_accvgpr_read_b32 v245, a70\n\t" \
    "v_accvgpr_read_b32 v246, a74\n\t" \
    "v_accvgpr_read_b32 v247, a78\n\t" \
    "ds_write_b128 %[vStg], v[244:247] offset:18432\n\t" \
    "ds_write_b128 %[vStg], a[104:107] offset:6144\n\t" \
    "v_accvgpr_read_b32 v248, a67\n\t" \
    "v_accvgpr_read_b32 v249, a71\n\t" \
    "v_accvgpr_read_b32 v250, a75\n\t" \
    "v_accvgpr_read_b32 v251, a79\n\t" \
    "ds_write_b128 %[vStg], v[248:251] offset:19456\n\t" \
    "ds_write_b128 %[vStg], a[108:111] offset:7168\n\t" \
    "s_branch FB0_2%=\n\t" \
    "FO0_4%=:\n\t" \
    "v_accvgpr_read_b32 v236, a128\n\t" \
    "v_accvgpr_read_b32 v237, a132\n\t" \
    "v_accvgpr_read_b32 v238, a136\n\t" \
    "v_accvgpr_read_b32 v239, a140\n\t" \
    "ds_write_b128 %[vStg], v[236:239] offset:16384\n\t" \
    "ds_write_b128 %[vStg], a[160:163] offset:8192\n\t" \
    "v_accvgpr_read_b32 v240, a129\n\t" \
    "v_accvgpr_read_b32 v241, a133\n\t" \
    "v_accvgpr_read_b32 v242, a137\n\t" \
    "v_accvgpr_read_b32 v243, a141\n\t" \
    "ds_write_b128 %[vStg], v[240:243] offset:17408\n\t" \
    "ds_write_b128 %[vStg], a[164:167] offset:9216\n\t" \
    "v_accvgpr_read_b32 v244, a130\n\t" \
    "v_accvgpr_read_b32 v245, a134\n\t" \
    "v_accvgpr_read_b32 v246, a138\n\t" \
    "v_accvgpr_read_b32 v247, a142\n\t" \
    "ds_write_b128 %[vStg], v[244:247] offset:18432\n\t" \
    "ds_write_b128 %[vStg], a[168:171] offset:10240\n\t" \
    "v_accvgpr_read_b32 v248, a131\n\t" \
    "v_accvgpr_read_b32 v249, a135\n\t" \
    "v_accvgpr_read_b32 v250, a139\n\t" \
    "v_accvgpr_read_b32 v251, a143\n\t" \
    "ds_write_b128 %[vStg], v[248:251] offset:19456\n\t" \
    "ds_write_b128 %[vStg], a[172:175] offset:11264\n\t" \
    "s_branch FB0_4%=\n\t" \
    "FO0_6%=:\n\t" \
    "v_accvgpr_read_b32 v236, a192\n\t" \
    "v_accvgpr_read_b32 v237, a196\n\t" \
    "v_accvgpr_read_b32 v238, a200\n\t" \
    "v_accvgpr_read_b32 v239, a204\n\t" \
    "ds_write_b128 %[vStg], v[236:239] offset:16384\n\t" \
    "ds_write_b128 %[vStg], a[224:227] offset:12288\n\t" \
    "v_accvgpr_read_b32 v240, a193\n\t" \
    "v_accvgpr_read_b32 v241, a197\n\t" \
    "v_accvgpr_read_b32 v242, a201\n\t" \
    "v_accvgpr_read_b32 v243, a205\n\t" \
    "ds_write_b128 %[vStg], v[240:243] offset:17408\n\t" \
    "ds_write_b128 %[vStg], a[228:231] offset:13312\n\t" \
    "v_accvgpr_read_b32 v244, a194\n\t" \
    "v_accvgpr_read_b32 v245, a198\n\t" \
    "v_accvgpr_read_b32 v246, a202\n\t" \
    "v_accvgpr_read_b32 v247, a206\n\t" \
    "ds_write_b128 %[vStg], v[244:247] offset:18432\n\t" \
    "ds_write_b128 %[vStg], a[232:235] offset:14336\n\t" \
    "v_accvgpr_read_b32 v248, a195\n\t" \
    "v_accvgpr_read_b32 v249, a199\n\t" \
    "v_accvgpr_read_b32 v250, a203\n\t" \
    "v_accvgpr_read_b32 v251, a207\n\t" \
    "ds_write_b128 %[vStg], v[248:251] offset:19456\n\t" \
    "ds_write_b128 %[vStg], a[236:239] offset:15360\n\t" \
    "s_branch FB0_6%=\n\t" \
    "SLOW_P0%=:\n\t" \
    "s_memrealtime s[62:63]\n\t" \
    "SPIN_P0%=:\n\t" \
    "buffer_load_dword v19, %[vFl], %[rWS], 0 offen sc0 sc1\n\t" \
    "s_waitcnt vmcnt(0) lgkmcnt(0)\n\t" \
    "v_cmp_le_u32 s[76:77], s49, v19\n\t" \
    "s_nop 1\n\t" \
    "s_or_b64 s[76:77], s[76:77], s[70:71]\n\t" \
    "s_cmp_eq_u64 s[76:77], exec\n\t" \
    "s_cbranch_scc1 BACK_P0%=\n\t" \
    "buffer_load_dword v19, off, %[rWS], %[sTmo] sc1\n\t" \
    "s_memrealtime s[64:65]\n\t" \
    "s_waitcnt vmcnt(0) lgkmcnt(0)\n\t" \
    "v_readfirstlane_b32 s65, v19\n\t" \
    "s_sub_u32 s64, s64, s62\n\t" \
    "s_nop 1\n\t" \
    "s_cmp_eq_u32 s65, s35\n\t" \
    "s_cbranch_scc1 BAD_P0%=\n\t" \
    "s_cmp_gt_u32 s64, %[ddl]\n\t" \
    "s_cbranch_scc1 BAD_P0%=\n\t" \
    "s_sleep 1\n\t" \
    "s_branch SPIN_P0%=\n\t" \
    "BAD_P0%=:\n\t" \
    "v_mov_b32 v19, s35\n\t" \
    "s_mov_b32 s54, 1\n\t" \
    "buffer_store_dword v19, off, %[rWS], %[sTmo] sc1\n\t" \
    "s_waitcnt vmcnt(0) lgkmcnt(0)\n\t" \
    "s_branch BACK_P0%=\n\t" \
    "SLOW_P4%=:\n\t" \
    "s_memrealtime s[62:63]\n\t" \
    "SPIN_P4%=:\n\t" \
    "buffer_load_dword v19, %[vFl], %[rWS], 0 offen sc0 sc1\n\t" \
    "s_waitcnt vmcnt(0) lgkmcnt(0)\n\t" \
    "v_cmp_le_u32 s[76:77], s49, v19\n\t" \
    "s_nop 1\n\t" \
    "s_or_b64 s[76:77], s[76:77], s[70:71]\n\t" \
    "s_cmp_eq_u64 s[76:77], exec\n\t" \
    "s_cbranch_scc1 BACK_P4%=\n\t" \
    "buffer_load_dword v19, off, %[rWS], %[sTmo] sc1\n\t" \
    "s_memrealtime s[64:65]\n\t" \
    "s_waitcnt vmcnt(0) lgkmcnt(0)\n\t" \
    "v_readfirstlane_b32 s65, v19\n\t" \
    "s_sub_u32 s64, s64, s62\n\t" \
    "s_nop 1\n\t" \
    "s_cmp_eq_u32 s65, s35\n\t" \
    "s_cbranch_scc1 BAD_P4%=\n\t" \
    "s_cmp_gt_u32 s64, %[ddl]\n\t" \
    "s_cbranch_scc1 BAD_P4%=\n\t" \
    "s_sleep 1\n\t" \
    "s_branch SPIN_P4%=\n\t" \
    "BAD_P4%=:\n\t" \
    "v_mov_b32 v19, s35\n\t" \
    "s_mov_b32 s54, 1\n\t" \
    "buffer_store_dword v19, off, %[rWS], %[sTmo] sc1\n\t" \
    "s_waitcnt vmcnt(0) lgkmcnt(0)\n\t" \
    "s_branch BACK_P4%=\n\t" \
    "END%=:\n\t" \
    ""
#define GEMM_CLOBBERS "v18", "v19", "v20", "v21", "v22", "v23", "v24", "v25", "v26", "v27", "v28", "v29", "v30", "v31", "v32", "v33", "v34", "v35", "v36", "v37", "v38", "v39", "v40", "v41", "v42", "v43", "v44", "v45", "v46", "v47", "v48", "v49", "v50", "v51", "v52", "v53", "v54", "v55", "v56", "v57", "v58", "v59", "v60", "v61", "v62", "v63", "v64", "v65", "v66", "v67", "v68", "v69", "v70", "v71", "v72", "v73", "v74", "v75", "v76", "v77", "v78", "v79", "v80", "v81", "v82", "v83", "v84", "v85", "v86", "v87", "v88", "v89", "v90", "v91", "v92", "v93", "v94", "v95", "v96", "v97", "v98", "v99", "v100", "v101", "v102", "v103", "v104", "v105", "v106", "v107", "v108", "v109", "v110", "v111", "v112", "v113", "v114", "v115", "v116", "v117", "v118", "v119", "v120", "v121", "v122", "v123", "v124", "v125", "v126", "v127", "v128", "v129", "v130", "v131", "v132", "v133", "v134", "v135", "v136", "v137", "v138", "v139", "v140", "v141", "v142", "v143", "v144", "v145", "v146", "v147", "v148", "v149", "v150", "v151", "v152", "v153", "v154", "v155", "v156", "v157", "v158", "v159", "v160", "v161", "v162", "v163", "v164", "v165", "v166", "v167", "v168", "v169", "v170", "v171", "v172", "v173", "v174", "v175", "v176", "v177", "v178", "v179", "v180", "v181", "v182", "v183", "v184", "v185", "v186", "v187", "v188", "v189", "v190", "v191", "v192", "v193", "v194", "v195", "v196", "v197", "v198", "v199", "v200", "v201", "v202", "v203", "v204", "v205", "v206", "v207", "v208", "v209", "v210", "v211", "v212", "v213", "v214", "v215", "v216", "v217", "v218", "v219", "v220", "v221", "v222", "v223", "v224", "v225", "v226", "v227", "v228", "v229", "v230", "v231", "v232", "v233", "v234", "v235", "v236", "v237", "v238", "v239", "v240", "v241", "v242", "v243", "v244", "v245", "v246", "v247", "v248", "v249", "v250", "v251", "v252", "v253", "v254", "v255", "a0", "a1", "a2", "a3", "a4", "a5", "a6", "a7", "a8", "a9", "a10", "a11", "a12", "a13", "a14", "a15", "a16", "a17", "a18", "a19", "a20", "a21", "a22", "a23", "a24", "a25", "a26", "a27", "a28", "a29", "a30", "a31", "a32", "a33", "a34", "a35", "a36", "a37", "a38", "a39", "a40", "a41", "a42", "a43", "a44", "a45", "a46", "a47", "a48", "a49", "a50", "a51", "a52", "a53", "a54", "a55", "a56", "a57", "a58", "a59", "a60", "a61", "a62", "a63", "a64", "a65", "a66", "a67", "a68", "a69", "a70", "a71", "a72", "a73", "a74", "a75", "a76", "a77", "a78", "a79", "a80", "a81", "a82", "a83", "a84", "a85", "a86", "a87", "a88", "a89", "a90", "a91", "a92", "a93", "a94", "a95", "a96", "a97", "a98", "a99", "a100", "a101", "a102", "a103", "a104", "a105", "a106", "a107", "a108", "a109", "a110", "a111", "a112", "a113", "a114", "a115", "a116", "a117", "a118", "a119", "a120", "a121", "a122", "a123", "a124", "a125", "a126", "a127", "a128", "a129", "a130", "a131", "a132", "a133", "a134", "a135", "a136", "a137", "a138", "a139", "a140", "a141", "a142", "a143", "a144", "a145", "a146", "a147", "a148", "a149", "a150", "a151", "a152", "a153", "a154", "a155", "a156", "a157", "a158", "a159", "a160", "a161", "a162", "a163", "a164", "a165", "a166", "a167", "a168", "a169", "a170", "a171", "a172", "a173", "a174", "a175", "a176", "a177", "a178", "a179", "a180", "a181", "a182", "a183", "a184", "a185", "a186", "a187", "a188", "a189", "a190", "a191", "a192", "a193", "a194", "a195", "a196", "a197", "a198", "a199", "a200", "a201", "a202", "a203", "a204", "a205", "a206", "a207", "a208", "a209", "a210", "a211", "a212", "a213", "a214", "a215", "a216", "a217", "a218", "a219", "a220", "a221", "a222", "a223", "a224", "a225", "a226", "a227", "a228", "a229", "a230", "a231", "a232", "a233", "a234", "a235", "a236", "a237", "a238", "a239", "a240", "a241", "a242", "a243", "a244", "a245", "a246", "a247", "a248", "a249", "a250", "a251", "a252", "a253", "a254", "a255", "s34", "s35", "s36", "s37", "s38", "s39", "s40", "s41", "s42", "s43", "s44", "s45", "s46", "s47", "s48", "s49", "s50", "s51", "s52", "s53", "s54", "s55", "s56", "s57", "s58", "s59", "s60", "s61", "s62", "s63", "s64", "s65", "s66", "s67", "s68", "s69", "s70", "s71", "s72", "s73", "s74", "s75", "s76", "s77", "s78", "s79", "s80", "s81", "vcc", "scc", "memory"
#define HIP_CHECK(expr) do { hipError_t _e = (expr); if (_e != hipSuccess) { \
    fprintf(stderr, "HIP error %s at %s:%d\n", hipGetErrorString(_e), __FILE__, __LINE__); \
    return; } } while (0)
typedef float    f32x4 __attribute__((ext_vector_type(4)));
typedef unsigned u32x4 __attribute__((ext_vector_type(4)));
#define LAS __attribute__((address_space(3)))
#define GAS __attribute__((address_space(1)))
#define DI __device__ __forceinline__
#define SB() __builtin_amdgcn_sched_barrier(0)
static_assert(GEN_RD == 0 && GEN_OWN == 1, "the tail below matches this schedule");

constexpr int NR = 8, ML = 1024, NN = 8192, KK = 8192, NL = NN / NR;
constexpr unsigned WS_XH = 0;
constexpr unsigned WS_XCH = 16u << 20;
constexpr unsigned XCH_B = 32u << 20;
constexpr size_t WS_FLAGS = (size_t)256 << 20;
constexpr unsigned WS_BYTES = (256u << 20) + (1u << 20);
constexpr unsigned OUT_BYTES = (unsigned)NN * NL * 4u;
constexpr int FL_TMO = 32, FL_DONE = 64  , FL_XCNT = 4096  ;
constexpr unsigned FL_XCH_B = 65536;
constexpr unsigned DEADLINE_TICKS = 50000000u;
constexpr int LDS_BYTES = 163840;
constexpr int AUX_SC = 17, AUX_OUT = 19;
struct Args { const float* x; const float* w; void* out[NR]; void* ws[NR]; int rank; int pad; };

DI float silu(float y) {
    const float c = __builtin_amdgcn_fmed3f(y, -60.0f, 60.0f);
    const float e = __builtin_amdgcn_exp2f(c * -1.4426950408889634f);
    return y * __builtin_amdgcn_rcpf(1.0f + e);
}
DI unsigned rfl(unsigned v) { return __builtin_amdgcn_readfirstlane(v); }
DI f32x4 ldx(__amdgpu_buffer_rsrc_t r, unsigned v, unsigned s) { return __builtin_bit_cast(f32x4, __builtin_amdgcn_raw_buffer_load_b128(r, v, s, AUX_SC)); }
DI void stx(f32x4 d, __amdgpu_buffer_rsrc_t r, unsigned v) { __builtin_amdgcn_raw_buffer_store_b128(__builtin_bit_cast(u32x4, d), r, v, 0, AUX_SC); }
DI void sto(f32x4 d, __amdgpu_buffer_rsrc_t r, unsigned v) { __builtin_amdgcn_raw_buffer_store_b128(__builtin_bit_cast(u32x4, d), r, v, 0, AUX_OUT); }
DI __amdgpu_buffer_rsrc_t mkrs(unsigned long long u, unsigned bytes) {
    const unsigned long long v = ((unsigned long long)rfl((unsigned)(u >> 32)) << 32) | rfl((unsigned)u);
    return __builtin_amdgcn_make_buffer_rsrc((void*)v, (short)0, (int)bytes, 0x00020000);
}
DI bool wait_ge(GAS unsigned* word, unsigned need, GAS unsigned* tmo, unsigned call) {
    const unsigned long long t0 = __builtin_amdgcn_s_memrealtime();
    while (!__all(__hip_atomic_load(word, __ATOMIC_RELAXED, __HIP_MEMORY_SCOPE_SYSTEM) >= need)) {
        if (rfl(__hip_atomic_load(tmo, __ATOMIC_RELAXED, __HIP_MEMORY_SCOPE_AGENT)) == call || __builtin_amdgcn_s_memrealtime() - t0 > DEADLINE_TICKS) {
            __hip_atomic_store(tmo, call, __ATOMIC_RELAXED, __HIP_MEMORY_SCOPE_AGENT);
            return true;
        }
        __builtin_amdgcn_s_sleep(1);
    }
    asm volatile("" ::: "memory");
    return false;
}
DI int rel_of(int i, int j) { const int r = i + (j == 15 ? 3 : j == 14 ? 0 : j >> 1); return i == 7 ? 7 : r >= 7 ? r - 7 : r; }
DI unsigned col_of(int j, int dest) { return (unsigned)((j + dest) & 15) * 64u; }

__global__ __launch_bounds__(256, 1) void fused_kernel(Args a) {
    extern __shared__ __attribute__((aligned(16))) char smem[];
    int lane = threadIdx.x & 63;
    const int wv = rfl(threadIdx.x >> 6);
    int g = lane >> 4, c = lane & 15;
    const int b = blockIdx.x & 255, cls = b & 7, idx = b >> 3, mt = idx & 1, j = idx >> 1, par = cls & 1, own = cls >> 1;
    const int e0 = rel_of(0, j), e1 = rel_of(1 + par, j), e2 = rel_of(3 + par, j), e3 = rel_of(5 + par, j);
    unsigned long long q0 = (unsigned long long)a.out[e0], q1 = (unsigned long long)a.out[e1], q2 = (unsigned long long)a.out[e2],
                       wsu = (unsigned long long)a.ws[7], xu = (unsigned long long)a.x, wu = (unsigned long long)a.w;
    int rank = a.rank;
    asm volatile("" : "+s"(q0), "+s"(q1), "+s"(q2), "+s"(wsu), "+s"(xu), "+s"(wu), "+s"(rank));
    GAS char* ws = (GAS char*)wsu;
    GAS unsigned* fl = (GAS unsigned*)(ws + WS_FLAGS);
    const __amdgpu_buffer_rsrc_t rWS = mkrs(wsu, WS_BYTES);
    const int d0 = (rank + 1 + e0) & 7, d1 = (rank + 1 + e1) & 7, d2 = (rank + 1 + e2) & 7, d3 = (rank + 1 + e3) & 7;
    unsigned bad, call;
    {
        const __amdgpu_buffer_rsrc_t rW = mkrs(wu, (unsigned)KK * NN * 4u), rX = mkrs(xu, (unsigned)ML * KK * 4u);
        const int mfg = mt * 32 + (j & ~1) * 2 + wv;
        const int h = wv & 1, lead = (mt + (j >> 1)) & 1;
        const int k0 = lead * 2 + (wv >> 1), k1 = (lead ^ 1) * 2 + (wv >> 1);
        unsigned sW[4];
#pragma unroll
        for (int q = 0; q < 4; ++q) {
            const int dd = h == 0 ? (q < 2 ? d1 : d3) : (q == 0 ? d0 : q == 3 ? rank : d2);
            sW[q] = rfl(((unsigned)dd * NL + col_of(j, dd)) * 4u);
        }
        unsigned tab0, tab1;
        {
            const int i = lane & 31, st = i & 7;
            const bool ok = i < 24;
            const int e = ok ? i >> 3 : 1;
            const int mf = 2 * own + (st >> 2);
            const int r = e == 0 ? 2 * par + ((st >> 1) & 1) : st & 3;
            const int p0 = e == 0 ? st & 1 : par;
            const unsigned src = WS_XCH + (unsigned)e * XCH_B + (unsigned)((idx * 8 + p0) * 4 + wv) * 32768u + mf * 4096u + r * 1024u;
            const int dest = e == 0 ? d0 : e == 1 ? d1 : d2;
            const unsigned long long o = (e == 0 ? q0 : e == 1 ? q1 : q2) + ((size_t)(rank * ML + mt * 512 + wv * 128 + mf * 16 + 4 * r) * NL + col_of(j, dest)) * 4;
            tab0 = lane < 32 ? src : (unsigned)o;
            tab1 = ((unsigned)(o >> 32) & 0xffffu) | (ok && (e != 0 || (st & 1)) ? 0x30000u : 0u);
        }
        const unsigned fb = (unsigned)WS_FLAGS;
        asm volatile("s_nop 4\n\t" GEMM_ASM
            : [bad] "=s"(bad), [ocall] "=s"(call)
            : [vA0] "v"(lane * 16), [vW] "v"((unsigned)(g * 8 * NN + 4 * c) * 4u),
              [vWr0] "v"(lane * 16 + (k0 * 8 + h * 4) * 1024), [vWr1] "v"(lane * 16 + (k1 * 8 + h * 4) * 1024),
              [vStg] "v"(65536 + wv * 24576 + lane * 16), [vEnd] "v"(wv * 40960 + lane * 16), [vO] "v"(g * 4096 + c * 16),
              [vFs] "v"(lane < 8 ? fb + FL_XCH_B + (unsigned)((idx * 8 + lane) * 4 + wv) * 128u + cls * 4u : 0x80000000u),
              [vFl] "v"(fb + FL_XCH_B + (unsigned)(b * 4 + wv) * 128u + (lane & 7) * 4u),
              [vXf] "v"(fb + (FL_XCNT + 32 * ((mt * 4 + wv) * 8 + (lane & 7))) * 4u), [vXs] "v"(g * 32768u + c * 16u),
              [vXw] "v"(65536 + wv * 24576 + (c >> 3) * 1024 + ((c & 7) >> 1) * 256 + g * 64 + (c & 1) * 8),
              [vTab0] "v"(tab0), [vTab1] "v"(tab1), [p60] "v"(60.0f), [m60] "s"(-60.0f),
              [rWS] "s"(rWS), [rW] "s"(rW), [rX] "s"(rX),
              [sJunk] "s"(rfl(fb + FL_XCH_B + (unsigned)(b * 4 + wv) * 128u + 64u)),
              [sXs0] "s"(rfl((unsigned)mfg * 524288u + cls * 4096u)),
              [sXd0] "s"(rfl(WS_XH + (unsigned)(((mfg >> 3) * 256 + cls * 32) * 8 + (mfg & 7)) * 1024u)),
              [sXc0] "s"(fb + (FL_XCNT + 32 * ((mt * 4 + (j >> 2)) * 8 + cls)) * 4u),
              [sXp0] "s"(rfl(fb + (FL_XCNT + 32 * ((mt * 4 + wv) * 8 + cls)) * 4u)),
              [ign] "s"(par ? 0x55555555u : 0xaaaaaaaau),
              [sA0] "s"(rfl(WS_XH + (unsigned)(mt * 4 + wv) * 2097152u)), [t0] "s"(cls * 8), [pop] "s"(j & 1),
              [sW0] "s"(sW[0]), [sW1] "s"(sW[1]), [sW2] "s"(sW[2]), [sW3] "s"(sW[3]),
              [sWk0] "s"(rfl((unsigned)k0 * 1048576u)), [sWk1] "s"(rfl((unsigned)k1 * 1048576u)),
              [sExp0] "s"(rfl(WS_XCH + (unsigned)(b * 4 + wv) * 32768u)), [own] "s"(own),
              [wsLo] "s"((unsigned)wsu), [wsHi] "s"((unsigned)(wsu >> 32)),
              [sTmo] "s"(fb + FL_TMO * 4u), [ddl] "i"(DEADLINE_TICKS)
            : GEMM_CLOBBERS);
    }
    asm volatile("v_mbcnt_lo_u32_b32 %0, -1, 0\n\tv_mbcnt_hi_u32_b32 %0, -1, %0" : "=v"(lane));
    g = lane >> 4; c = lane & 15;
    const unsigned vA = lane * 16, vO = g * 4096 + c * 16;
    unsigned long long pdone = 0;
    if (wv < 2) {
        const __amdgpu_buffer_rsrc_t rO = mkrs((unsigned long long)a.out[e3], OUT_BYTES);
        GAS unsigned* fsrc = (GAS unsigned*)(ws + WS_FLAGS + FL_XCH_B + (size_t)(b * 4 + wv + ((lane >> 2) & 2)) * 128 + (par + 2 * (lane & 3)) * 4);
        if (wait_ge(fsrc, call * 16 + 7, fl + FL_TMO, call)) bad = 1;
        f32x4 p[2][2][3][4];
#pragma unroll
        for (int h = 0; h < 2; ++h)
#pragma unroll
            for (int m = 0; m < 2; ++m)
#pragma unroll
                for (int k = 0; k < 3; ++k) {
                    const int pc = par + 2 * ((own + 1 + k) & 3);
                    const unsigned sb = WS_XCH + 3 * XCH_B + (unsigned)((idx * 8 + pc) * 4 + wv + 2 * h) * 32768u + (2 * own + m) * 4096u;
#pragma unroll
                    for (int t = 0; t < 4; ++t) p[h][m][k][t] = ldx(rWS, vA + t * 1024, sb);
                }
        SB();
        unsigned long long w0 = (unsigned long long)a.ws[e0], w1 = (unsigned long long)a.ws[e1], w2 = (unsigned long long)a.ws[e2], w3 = (unsigned long long)a.ws[e3];
        asm volatile("" : "+s"(w0), "+s"(w1), "+s"(w2), "+s"(w3));
        pdone = (lane == 0 ? w0 : lane == 1 ? w1 : lane == 2 ? w2 : w3) + WS_FLAGS + (FL_DONE + 32 * rank) * 4;
        asm volatile("" : "+v"(pdone));
        SB();
#pragma unroll
        for (int h = 0; h < 2; ++h) {
            LAS const f32x4* L = (LAS const f32x4*)(smem + (wv + 2 * h) * 40960 + lane * 16);
            const unsigned ro = vO + ((unsigned)(rank * ML + mt * 512 + (wv + 2 * h) * 128) * NL + col_of(j, d3)) * 4u;
#pragma unroll
            for (int m = 0; m < 2; ++m) {
                f32x4 y[4];
#pragma unroll
                for (int t = 0; t < 4; ++t) y[t] = (L[(32 + m * 4 + t) * 64] + p[h][m][0][t]) + (p[h][m][1][t] + p[h][m][2][t]);
#pragma unroll
                for (int r = 0; r < 4; ++r) {
                    const f32x4 o = {silu(y[0][r]), silu(y[1][r]), silu(y[2][r]), silu(y[3][r])};
                    SB();
                    if ((h | m | r) != 0) asm volatile("s_waitcnt vmcnt(0)" ::: "memory");
                    sto(o, rO, ro + ((2 * own + m) * 16 + 4 * r) * 4096u);
                    SB();
                }
            }
        }
    } else {
        const __amdgpu_buffer_rsrc_t rO = mkrs((unsigned long long)a.out[7], OUT_BYTES);
        __builtin_amdgcn_s_sleep(110);
#pragma unroll
        for (int h = 0; h < 2; ++h) {
            const int rg = wv - 2 + 2 * h;
            LAS const f32x4* L = (LAS const f32x4*)(smem + rg * 40960 + lane * 16);
            const unsigned sb = WS_XCH + 4 * XCH_B + (unsigned)(b * 4 + rg) * 32768u;
#pragma unroll
            for (int k = 0; k < 7; ++k) {
                const int mf = (cls + 1 + k) & 7;
#pragma unroll
                for (int t = 0; t < 4; ++t) stx(L[(mf * 4 + t) * 64], rWS, vA + t * 1024 + sb + mf * 4096u);
                SB();
                asm volatile("s_waitcnt vmcnt(6)" ::: "memory");
            }
        }
        asm volatile("s_waitcnt vmcnt(0)" ::: "memory");
        GAS unsigned* fdst = (GAS unsigned*)(ws + WS_FLAGS + FL_XCH_B + (size_t)((idx * 8 + (lane & 7)) * 4 + wv) * 128 + cls * 4);
        GAS unsigned* fsrc = (GAS unsigned*)(ws + WS_FLAGS + FL_XCH_B + (size_t)(b * 4 + wv) * 128 + (lane & 7) * 4);
        if (lane < 8) __hip_atomic_store(fdst, call * 16 + 8, __ATOMIC_RELAXED, __HIP_MEMORY_SCOPE_SYSTEM);
        if (wait_ge(fsrc, call * 16 + 8, fl + FL_TMO, call)) bad = 1;
        f32x4 p[2][7][4];
#pragma unroll
        for (int h = 0; h < 2; ++h)
#pragma unroll
            for (int k = 0; k < 7; ++k) {
                const int pc = (cls + 1 + k) & 7;
                const unsigned sb = WS_XCH + 4 * XCH_B + (unsigned)((idx * 8 + pc) * 4 + wv - 2 + 2 * h) * 32768u + cls * 4096u;
#pragma unroll
                for (int t = 0; t < 4; ++t) p[h][k][t] = ldx(rWS, vA + t * 1024, sb);
            }
        SB();
#pragma unroll
        for (int h = 0; h < 2; ++h) {
            const int rg = wv - 2 + 2 * h;
            LAS const f32x4* L = (LAS const f32x4*)(smem + rg * 40960 + lane * 16);
            const unsigned ro = vO + ((unsigned)(rank * ML + mt * 512 + rg * 128) * NL + col_of(j, rank)) * 4u;
            f32x4 y[4];
#pragma unroll
            for (int t = 0; t < 4; ++t) y[t] = ((L[(cls * 4 + t) * 64] + p[h][0][t]) + (p[h][1][t] + p[h][2][t])) + ((p[h][3][t] + p[h][4][t]) + (p[h][5][t] + p[h][6][t]));
#pragma unroll
            for (int r = 0; r < 4; ++r) sto((f32x4){silu(y[0][r]), silu(y[1][r]), silu(y[2][r]), silu(y[3][r])}, rO, ro + (cls * 16 + 4 * r) * 4096u);
        }
    }
    asm volatile("s_waitcnt vmcnt(0)" ::: "memory");
    __syncthreads();
    if (wv == 0) {
        if (lane < 4) __hip_atomic_fetch_add((GAS unsigned*)pdone, lane == 0 ? 1u : 2u, __ATOMIC_RELAXED, __HIP_MEMORY_SCOPE_SYSTEM);
        if (b == 0) {
            const int p = (rank + 1 + (lane & 7) % 7) & 7;
            if (wait_ge(fl + FL_DONE + 32 * p, call * 256u, fl + FL_TMO, call)) bad = 1;
        }
    }
    if (bad && lane == 0) ((GAS float*)a.out[7])[(size_t)(rank * ML + mt * 512 + cls * 16) * NL + col_of(j, rank)] = __builtin_nanf("");
}

extern "C" void kernel_launch(int rank, int nranks, ncclComm_t comm, hipStream_t stream, void* const* d_in, const int* in_sizes, int n_in,
                              void* d_out, int out_size, const int* in_dtypes, int out_dtype, void* const* d_out_all, void* const* d_ws_all, size_t ws_bytes) {
    if (nranks != NR || (unsigned)rank >= (unsigned)NR || n_in != 2 || in_sizes[0] != ML * KK || in_sizes[1] != KK * NN
        || out_size != NN * NL || ws_bytes < WS_BYTES) {
        fprintf(stderr, "kernel_launch: unexpected problem shape\n");
        return;
    }
    HIP_CHECK(hipFuncSetAttribute((const void*)fused_kernel, hipFuncAttributeMaxDynamicSharedMemorySize, LDS_BYTES));
    Args a;
    a.x = (const float*)d_in[0]; a.w = (const float*)d_in[1];
    for (int p = 0; p < NR; ++p) { a.out[p] = d_out_all[(rank + 1 + p) & 7]; a.ws[p] = d_ws_all[(rank + 1 + p) & 7]; }
    a.rank = rank; a.pad = 0;
    hipLaunchKernelGGL(fused_kernel, dim3(256), dim3(256), LDS_BYTES, stream, a);
    HIP_CHECK(hipGetLastError());
}
```
